```python
import numpy as np

kernel_name = "gpu_gemm_ar_m8192_k8192_n8192_bf16_gelu"


ABSMAX_RTOL = 0.008

COMPUTE_DTYPE = "bfloat16"
EPILOGUE = "gelu"
IN_QUANT = None
OUT_QUANT = None
IN_DTYPES = {"x": "bfloat16", "w_mat": "bfloat16"}
WORLD_SIZE = 8

def _to_bf16(x):
    u = np.ascontiguousarray(x, dtype=np.float32).view(np.uint32)
    return ((u + 0x7FFF + ((u >> 16) & 1)) & 0xFFFF0000).view(np.float32)

def setup_inputs():

    x = _to_bf16(np.random.randn(8192, 8192).astype(np.float32))
    w_mat = _to_bf16((np.random.randn(8192, 8192) / np.sqrt(8192)).astype(np.float32))
    return {"x": x, "w_mat": w_mat}

def _epilogue(y):
    c = 0.7978845608028654
    return (0.5 * y * (1.0 + np.tanh(c * (y + 0.044715 * y ** 3)))).astype(np.float32)

def reference(x, w_mat):
    y = (x @ w_mat).astype(np.float32)
    return _epilogue(y)
```

```cpp
#include <hip/hip_runtime.h>
#include <rccl/rccl.h>
#include <cstdio>
#include <cstdint>
#include <cstring>

namespace gar {

#define LAS __attribute__((address_space(3)))
typedef unsigned u32x4 __attribute__((ext_vector_type(4)));
typedef unsigned short u16x2 __attribute__((ext_vector_type(2)));
typedef short bf16x8 __attribute__((ext_vector_type(8)));
typedef float f32x4 __attribute__((ext_vector_type(4)));
typedef __amdgpu_buffer_rsrc_t rsrc_t;

constexpr int WORLD = 8;
constexpr int DIM = 8192;
constexpr int KSH = 1024;
constexpr int NGEMM = 128;
constexpr int GRID = 256;
#define PULL_WG_PER_LINK 6
#define PULL_WAVES 4
constexpr int PWL = PULL_WG_PER_LINK;
constexpr int PWAVES = PULL_WAVES;
constexpr int NPULL = 7 * PWL;
constexpr int NPACK = GRID - NGEMM - NPULL;
constexpr int NSEQ = 3584;
constexpr unsigned long long DEADLINE = 40000000ull;

constexpr unsigned OFF_TICKET = 0, OFF_TMO = 128;
constexpr unsigned OFF_CNT = 4096;
constexpr unsigned OFF_XFLAG = OFF_CNT + 256 * 128;
constexpr unsigned OFF_GRAN = 131072;
constexpr unsigned OFF_SLOT = 1u << 20;
constexpr unsigned OFF_AIMG = 64u << 20;
constexpr unsigned OFF_BIMG = 128u << 20;
constexpr unsigned OFF_XBUF = 160u << 20;
constexpr unsigned WS_USED = 192u << 20;
constexpr unsigned WS_NEED = WS_USED;
static_assert(OFF_XFLAG + 256 * 128 <= OFF_GRAN, "map");
static_assert(OFF_GRAN + WORLD * NSEQ * 16 <= OFF_SLOT, "map");
static_assert(OFF_SLOT + 6144u * 8192u <= OFF_AIMG, "map");
constexpr int LDS_BYTES = 131072 + 256;

struct Params {
    const void* x; const void* w; void* out;
    char* ws[WORLD];
    int rank; int pad;
};

constexpr int AUX_SC1 = 16, AUX_SYS = 17;

__device__ __forceinline__ rsrc_t mk_rsrc(const void* p, unsigned bytes) {
    return __builtin_amdgcn_make_buffer_rsrc(const_cast<void*>(p), 0, bytes, 0x00020000);
}
__device__ __forceinline__ unsigned rfl(unsigned v) { return __builtin_amdgcn_readfirstlane(v); }
__device__ __forceinline__ unsigned bfi(unsigned mask, unsigned a, unsigned b) { return (a & mask) | (b & ~mask); }
__device__ __forceinline__ unsigned perm(unsigned hi, unsigned lo, unsigned sel) { return __builtin_amdgcn_perm(hi, lo, sel); }
#define WAIT_VM0() asm volatile("s_waitcnt vmcnt(0)" ::: "memory")

#define STAMP(ws, idx) do { } while (0)

__device__ __forceinline__ bool give_up(unsigned long long t0, unsigned* tmo, unsigned call) {
    if (rfl(__hip_atomic_load(tmo, __ATOMIC_RELAXED, __HIP_MEMORY_SCOPE_AGENT)) == call) return true;
    if (__builtin_amdgcn_s_memrealtime() - t0 > DEADLINE) { __hip_atomic_store(tmo, call, __ATOMIC_RELAXED, __HIP_MEMORY_SCOPE_AGENT); return true; }
    return false;
}
__device__ __forceinline__ void wait_word(unsigned* p, unsigned target, unsigned* tmo, unsigned call) {
    if ((int)(rfl(__hip_atomic_load(p, __ATOMIC_RELAXED, __HIP_MEMORY_SCOPE_AGENT)) - target) >= 0) return;
    const unsigned long long t0 = __builtin_amdgcn_s_memrealtime();
    while ((int)(rfl(__hip_atomic_load(p, __ATOMIC_RELAXED, __HIP_MEMORY_SCOPE_AGENT)) - target) < 0) {
        if (give_up(t0, tmo, call)) break;
        __builtin_amdgcn_s_sleep(4);
    }
}

struct Hdr { unsigned h0, h1, h2; };

__device__ __forceinline__ Hdr encode_block(const u32x4 (&c)[8], u32x4 (&pl)[6], int lane) {
    unsigned H[16];
    u16x2 m1 = {0, 0}, m2 = {0, 0};
#pragma unroll
    for (int j = 0; j < 8; ++j) {
        H[2 * j] = perm(c[j].y, c[j].x, 0x07050301u); H[2 * j + 1] = perm(c[j].w, c[j].z, 0x07050301u);
        const unsigned l0 = perm(c[j].y, c[j].x, 0x06040200u), l1 = perm(c[j].w, c[j].z, 0x06040200u);
        if (j & 1) { pl[j >> 1].z = l0; pl[j >> 1].w = l1; } else { pl[j >> 1].x = l0; pl[j >> 1].y = l1; }
    }
#pragma unroll
    for (int g = 0; g < 16; ++g) {
        const unsigned t = H[g] & 0x7F7F7F7Fu;
        m1 = __builtin_elementwise_max(m1, __builtin_bit_cast(u16x2, t));
        m2 = __builtin_elementwise_max(m2, __builtin_bit_cast(u16x2, t << 8));
    }
    const u16x2 mm = __builtin_elementwise_max(m1, m2);
    unsigned mx = (mm.x > mm.y ? mm.x : mm.y) >> 8;
#pragma unroll
    for (int o = 32; o >= 1; o >>= 1) { const unsigned y = __shfl_xor(mx, o); mx = mx > y ? mx : y; }
    const unsigned base = rfl(mx), B = base * 0x01010101u;
    unsigned D[16], any = 0;
#pragma unroll
    for (int g = 0; g < 16; ++g) { D[g] = B - (H[g] & 0x7F7F7F7Fu); any |= D[g]; }
#pragma unroll
    for (int j = 0; j < 8; ++j) {
        const unsigned a = bfi(0x07070707u, D[2 * j], H[2 * j] >> 4), b = bfi(0x70707070u, D[2 * j + 1] << 4, H[2 * j + 1]);
        const unsigned wv = bfi(0x0F0F0F0Fu, a, b);
        if (j == 0) pl[4].x = wv; if (j == 1) pl[4].y = wv; if (j == 2) pl[4].z = wv; if (j == 3) pl[4].w = wv;
        if (j == 4) pl[5].x = wv; if (j == 5) pl[5].y = wv; if (j == 6) pl[5].z = wv; if (j == 7) pl[5].w = wv;
    }
    unsigned r0 = 0, r1 = 0, r2 = 0, n = 0;
    if (__ballot((any & 0x78787878u) != 0) != 0ull) {
#pragma unroll
        for (int g = 0; g < 16; ++g) {
            const unsigned f = D[g] & 0x78787878u;
            unsigned long long mask = __ballot(f != 0);
            while (mask != 0ull && n <= 3) {
                const int L = __builtin_ctzll(mask); mask &= mask - 1;
                const unsigned fl = __builtin_amdgcn_readlane(f, L);
                const unsigned lo = __builtin_amdgcn_readlane((g & 1) ? c[g >> 1].z : c[g >> 1].x, L);
                const unsigned hi = __builtin_amdgcn_readlane((g & 1) ? c[g >> 1].w : c[g >> 1].y, L);
#pragma unroll
                for (int b = 0; b < 4; ++b)
                    if ((fl >> (8 * b)) & 0xFFu) {
                        const unsigned val = (((b & 2) ? hi : lo) >> (16 * (b & 1))) & 0xFFFFu;
                        const unsigned rec = (unsigned)(L * 64 + g * 4 + b) | (val << 12);
                        if (n == 0) r0 = rec; else if (n == 1) r1 = rec; else if (n == 2) r2 = rec;
                        ++n;
                    }
            }
        }
    }
    Hdr h;
    const unsigned raw = n > 3 ? 1u : 0u;
    h.h0 = r0 | ((base & 15u) << 28);
    h.h1 = r1 | ((base >> 4) << 28) | (raw << 31);
    h.h2 = r2 | ((n & 3u) << 28);
    (void)lane;
    return h;
}

__device__ __forceinline__ void decode_block(const u32x4 (&pl)[6], unsigned base, u32x4 (&c)[8]) {
    const unsigned B = base * 0x01010101u;
#pragma unroll
    for (int j = 0; j < 8; ++j) {
        const unsigned wv = j == 0 ? pl[4].x : j == 1 ? pl[4].y : j == 2 ? pl[4].z : j == 3 ? pl[4].w : j == 4 ? pl[5].x : j == 5 ? pl[5].y : j == 6 ? pl[5].z : pl[5].w;
        const unsigned l0 = (j & 1) ? pl[j >> 1].z : pl[j >> 1].x, l1 = (j & 1) ? pl[j >> 1].w : pl[j >> 1].y;
        const unsigned ha = (B - (wv & 0x07070707u)) | ((wv << 4) & 0x80808080u);
        const unsigned hb = (B - ((wv >> 4) & 0x07070707u)) | (wv & 0x80808080u);
        c[j].x = perm(ha, l0, 0x05010400u); c[j].y = perm(ha, l0, 0x07030602u);
        c[j].z = perm(hb, l1, 0x05010400u); c[j].w = perm(hb, l1, 0x07030602u);
    }
}

__device__ __forceinline__ void store_granule(char* p, u32x4 g) {
    asm volatile("global_store_dwordx4 %0, %1, off sc0 sc1\n\ts_nop 1" :: "v"(p), "v"(g) : "memory");
}

__device__ __forceinline__ void publish_block(const Params& P, rsrc_t wsr, const u32x4 (&c)[8], unsigned slot, unsigned n, unsigned dmask, unsigned call, int lane) {
    u32x4 pl[6];
    const Hdr h = encode_block(c, pl, lane);
    const unsigned so = OFF_SLOT + slot * 8192u;
    if (h.h1 >> 31) {
#pragma unroll
        for (int j = 0; j < 8; ++j) __builtin_amdgcn_raw_buffer_store_b128(c[j], wsr, lane * 16 + j * 1024, so, AUX_SYS);
    } else {
#pragma unroll
        for (int j = 0; j < 6; ++j) __builtin_amdgcn_raw_buffer_store_b128(pl[j], wsr, lane * 16 + j * 1024, so, AUX_SYS);
    }
    WAIT_VM0();
    if (lane < WORLD && ((dmask >> lane) & 1u)) {
        char* p = P.ws[lane & 7] + OFF_GRAN + ((unsigned)P.rank * NSEQ + n) * 16u;
        store_granule(p, (u32x4){call, h.h0, h.h1, h.h2});
    }
}

__device__ __forceinline__ void packer_role(const Params& P, LAS unsigned char* lds, unsigned call, int wg) {
    const int lane = threadIdx.x & 63, wid = rfl(threadIdx.x >> 6);
    char* ws = P.ws[P.rank];
    const rsrc_t wsr = mk_rsrc(ws, WS_USED);
    const rsrc_t xr = mk_rsrc(P.x, DIM * KSH * 2), wr = mk_rsrc(P.w, DIM * KSH * 2);
    const int myi = P.rank >> 2, myj = P.rank & 3;
    LAS unsigned char* my = lds + wid * 8192;
    for (int p = wg * 8 + wid; p < 4096; p += NPACK * 8) {
        bool is_w; int kk, sel, r, bi;
        if (p < 3072) { kk = p / 192; const int rem = p - kk * 192; bi = rem & 31; r = 0; is_w = rem < 128; sel = is_w ? (rem >> 5) : ((rem - 128) >> 5); }
        else { const int q = p - 3072; kk = q >> 6; sel = (q >> 5) & 1; bi = q & 31; r = 1; is_w = false; }
        u32x4 c[8];
        if (!is_w) {
            const int rr = lane >> 3;
            const unsigned vo = (unsigned)rr * 2048u + (unsigned)(((lane & 7) ^ rr) * 16);
            const unsigned so = (unsigned)(sel * 4096 + r * 2048 + bi * 64) * 2048u + (unsigned)kk * 128u;
#pragma unroll
            for (int j = 0; j < 8; ++j) c[j] = __builtin_amdgcn_raw_buffer_load_b128(xr, vo + j * 8 * 2048, so, 0);
        } else {
            const int pn = bi >> 2, bj = (bi >> 1) & 1, q = bi & 1;
            const int gk = lane >> 3, cc = lane & 7, wcl = cc >> 2, cq = cc & 3;
            const unsigned vo = (unsigned)(gk * 8) * 16384u + (unsigned)((64 * (2 * q + wcl) + 32 * bj + 8 * cq) * 2);
            const unsigned so = (unsigned)(kk * 64) * 16384u + (unsigned)((sel * 2048 + pn * 256) * 2);
            u32x4 R[8];
#pragma unroll
            for (int jj = 0; jj < 8; ++jj) R[jj] = __builtin_amdgcn_raw_buffer_load_b128(wr, vo + jj * 16384, so, 0);
#pragma unroll
            for (int j = 0; j < 8; ++j) {
                const unsigned s = (j & 1) ? 0x07060302u : 0x05040100u;
                u32x4 t;
                t.x = perm(R[1][j >> 1], R[0][j >> 1], s); t.y = perm(R[3][j >> 1], R[2][j >> 1], s);
                t.z = perm(R[5][j >> 1], R[4][j >> 1], s); t.w = perm(R[7][j >> 1], R[6][j >> 1], s);
                const int s64 = 32 * wcl + 16 * (j >> 2) + 4 * cq + (j & 3);
                *(LAS u32x4*)(my + s64 * 128 + ((gk ^ (s64 & 7)) * 16)) = t;
            }
            asm volatile("s_waitcnt lgkmcnt(0)" ::: "memory");
#pragma unroll
            for (int j = 0; j < 8; ++j) c[j] = *(LAS u32x4*)(my + j * 1024 + lane * 16);
            asm volatile("s_waitcnt lgkmcnt(0)" ::: "memory");
        }
        const int g = P.rank * 16 + kk;
        const bool self = is_w ? (sel == myj) : (sel == myi);
        if (self) {
            const unsigned so = is_w ? OFF_BIMG + (unsigned)(g * 32 + bi) * 8192u : OFF_AIMG + (unsigned)((r * 128 + g) * 32 + bi) * 8192u;
#pragma unroll
            for (int j = 0; j < 8; ++j) __builtin_amdgcn_raw_buffer_store_b128(c[j], wsr, lane * 16 + j * 1024, so, AUX_SC1);
        }
        const unsigned slot = is_w ? 2048u + (unsigned)((sel * 16 + kk) * 32 + bi) : (unsigned)(((sel * 2 + r) * 16 + kk) * 32 + bi);
        const unsigned n = is_w ? (unsigned)(kk * 64 + bi) : (r == 0 ? (unsigned)(kk * 64 + 32 + bi) : (unsigned)(1024 + kk * 32 + bi));
        unsigned dmask = is_w ? (0x11u << sel) : (0x0Fu << (4 * sel));
        dmask &= ~(1u << P.rank);
        publish_block(P, wsr, c, slot, n, dmask, call, lane);
        if (self && lane == 0)
            __hip_atomic_fetch_add((unsigned*)(ws + OFF_CNT + (r * 128 + g) * 128), 1u, __ATOMIC_RELAXED, __HIP_MEMORY_SCOPE_AGENT);
    }
}

__device__ __forceinline__ u32x4 load_granule(rsrc_t wsr, unsigned off) { return __builtin_amdgcn_raw_buffer_load_b128(wsr, 0, off, AUX_SYS); }

__device__ __forceinline__ void puller_role(const Params& P, unsigned call, int wg) {
    const int lane = threadIdx.x & 63, wid = rfl(threadIdx.x >> 6);
    if (wid >= PWAVES) return;
    const int link = wg % 7, src = (P.rank + 1 + link) & 7;
    constexpr int NW = PWL * PWAVES;
    char* ws = P.ws[P.rank];
    unsigned* tmo = (unsigned*)(ws + OFF_TMO);
    const rsrc_t wsr = mk_rsrc(ws, WS_USED), rem = mk_rsrc(P.ws[src], WS_USED), outr = mk_rsrc(P.out, (unsigned)DIM * DIM * 2u);
    const int myi = P.rank >> 2, myj = P.rank & 3;
    const unsigned gbase = OFF_GRAN + (unsigned)src * NSEQ * 16u;
    unsigned pending = 0;
    int n = (wg / 7) * PWAVES + wid;
    u32x4 gr = load_granule(wsr, gbase + n * 16);
    for (; n < NSEQ; n += NW) {
        bool dead = false;
        if ((int)(rfl(gr.x) - call) < 0) {
            if (pending) { WAIT_VM0(); if (lane == 0) __hip_atomic_fetch_add((unsigned*)(ws + pending), 1u, __ATOMIC_RELAXED, __HIP_MEMORY_SCOPE_AGENT); pending = 0; }
            const unsigned long long t0 = __builtin_amdgcn_s_memrealtime();
            for (;;) {
                gr = load_granule(wsr, gbase + n * 16);
                if ((int)(rfl(gr.x) - call) >= 0) break;
                if (give_up(t0, tmo, call)) { dead = true; break; }
                __builtin_amdgcn_s_sleep(4);
                asm volatile("" ::: "memory");
            }
        }
        const unsigned h0 = rfl(gr.y), h1 = rfl(gr.z), h2 = rfl(gr.w);
        unsigned slot, dso, sj, sr, cnt; bool is_out = false;
        if (n < 1024) { const int kk = n >> 6, bi = n & 31, g = src * 16 + kk; cnt = OFF_CNT + g * 128;
            if ((n & 63) < 32) { slot = 2048u + (myj * 16 + kk) * 32 + bi; dso = OFF_BIMG + (unsigned)(g * 32 + bi) * 8192u; }
            else { slot = ((myi * 2) * 16 + kk) * 32 + bi; dso = OFF_AIMG + (unsigned)(g * 32 + bi) * 8192u; } }
        else if (n < 1536) { const int kk = (n - 1024) >> 5, bi = n & 31, g = src * 16 + kk; cnt = OFF_CNT + (128 + g) * 128;
            slot = ((myi * 2 + 1) * 16 + kk) * 32 + bi; dso = OFF_AIMG + (unsigned)((128 + g) * 32 + bi) * 8192u; }
        else { const int o = n - 1536, w = o & 7, h = (o >> 3) & 1, tile = (o >> 4) & 63, r = o >> 10; cnt = 0; is_out = true;
            slot = 4096u + o;
            const unsigned row0 = (src >> 2) * 4096 + r * 2048 + (tile >> 3) * 256 + h * 128 + (w >> 2) * 64, col0 = (src & 3) * 2048 + (tile & 7) * 256 + (w & 3) * 64;
            dso = (row0 * DIM + col0) * 2u; }
        sj = is_out ? 8u * DIM * 2u : 1024u; sr = is_out ? DIM * 2u : 128u;
        const unsigned so = OFF_SLOT + slot * 8192u;
        const bool raw = (h1 >> 31) != 0;
        u32x4 c[8], pl[6];
        if (!dead) {
            if (raw) {
#pragma unroll
                for (int j = 0; j < 8; ++j) c[j] = __builtin_amdgcn_raw_buffer_load_b128(rem, lane * 16 + j * 1024, so, AUX_SYS);
            } else {
#pragma unroll
                for (int j = 0; j < 6; ++j) pl[j] = __builtin_amdgcn_raw_buffer_load_b128(rem, lane * 16 + j * 1024, so, AUX_SYS);
            }
        }
        const int n2 = n + NW < NSEQ ? n + NW : n;
        gr = load_granule(wsr, gbase + n2 * 16);
        WAIT_VM0();
        if (pending) { if (lane == 0) __hip_atomic_fetch_add((unsigned*)(ws + pending), 1u, __ATOMIC_RELAXED, __HIP_MEMORY_SCOPE_AGENT); pending = 0; }
        if (is_out && rfl(__hip_atomic_load(tmo, __ATOMIC_RELAXED, __HIP_MEMORY_SCOPE_AGENT)) == call) dead = true;
        if (dead) {
#pragma unroll
            for (int j = 0; j < 8; ++j) c[j] = (u32x4){0x7FC07FC0u, 0x7FC07FC0u, 0x7FC07FC0u, 0x7FC07FC0u};
        } else if (!raw) decode_block(pl, (h0 >> 28) | (((h1 >> 28) & 7u) << 4), c);
        const rsrc_t dr = is_out ? outr : wsr;
        const unsigned vo = (unsigned)(lane >> 3) * sr + (unsigned)(lane & 7) * 16u;
#pragma unroll
        for (int j = 0; j < 8; ++j) __builtin_amdgcn_raw_buffer_store_b128(c[j], dr, vo, dso + j * sj, AUX_SC1);
        if (!dead && !raw) {
            const unsigned ne = (h2 >> 28) & 3u;
            for (unsigned e = 0; e < ne; ++e) {
                const unsigned rec = e == 0 ? h0 : e == 1 ? h1 : h2, pos = rec & 0xFFFu, v = pos & 63u;
                if ((unsigned)lane == (pos >> 6)) __builtin_amdgcn_raw_buffer_store_b16((unsigned short)((rec >> 12) & 0xFFFFu), dr, vo + (v & 7u) * 2u, dso + (v >> 3) * sj, AUX_SC1);
            }
        }
        pending = cnt;
    }
    WAIT_VM0();
    if (pending && lane == 0) __hip_atomic_fetch_add((unsigned*)(ws + pending), 1u, __ATOMIC_RELAXED, __HIP_MEMORY_SCOPE_AGENT);
}

__device__ __forceinline__ float gelu_f(float y) {
    const float c1 = -2.0f * 0.7978845608028654f * 1.4426950408889634f, c2 = c1 * 0.044715f;
    const float e = __builtin_amdgcn_exp2f(y * (c1 + c2 * y * y));
    return y * __builtin_amdgcn_rcpf(1.0f + e);
}
__device__ __forceinline__ unsigned cvt_pk_bf16(float lo, float hi) { unsigned r; asm("v_cvt_pk_bf16_f32 %0, %1, %2" : "=v"(r) : "v"(lo), "v"(hi)); return r; }

__device__ __forceinline__ void gemm_role(const Params& P, LAS unsigned char* lds, unsigned call, int wg) {
    const int tid = threadIdx.x, wid = rfl(tid >> 6), lane = tid & 63, wr = wid >> 2, wc = wid & 3, fr = lane & 15, fq = lane >> 4;
    char* ws = P.ws[P.rank];
    unsigned* tmo = (unsigned*)(ws + OFF_TMO);
    const rsrc_t wsr = mk_rsrc(ws, WS_USED);
    const int xcd = wg & 7, q = wg >> 3, h = xcd & 1, blk = xcd >> 1;
    const int tm = (blk >> 1) * 4 + (q >> 2), tn = (blk & 1) * 4 + (q & 3), tile = tm * 8 + tn;
    const unsigned voff = (unsigned)tid * 16u, ldsw = (unsigned)wid * 1024u;
    const int aoff0 = (wr * 64 + fr) * 128 + ((fq ^ (fr & 7)) << 4), aoff1 = aoff0 ^ 64;
    const int boff0 = (wc * 32 + fr) * 128 + ((fq ^ (fr & 7)) << 4), boff1 = boff0 ^ 64;
#define G_SA(b, hh) (((b) * 2 + (hh)) * 16384)
#define G_SB(b, hh) ((4 + (b) * 2 + (hh)) * 16384)
#define G_STAGE(bufoff, soff) do { _Pragma("unroll") for (int _i = 0; _i < 2; ++_i) \
        __builtin_amdgcn_raw_ptr_buffer_load_lds(wsr, (LAS void*)(lds + (bufoff) + ldsw + _i * 8192), 16, voff, (soff) + _i * 8192, 0, AUX_SC1); } while (0)
#define G_LDA(dst, b, hh) do { _Pragma("unroll") for (int m = 0; m < 4; ++m) { \
        dst[m][0] = *(const LAS bf16x8*)(lds + G_SA(b, hh) + aoff0 + m * 2048); dst[m][1] = *(const LAS bf16x8*)(lds + G_SA(b, hh) + aoff1 + m * 2048); } } while (0)
#define G_LDB(dst, b, hh) do { _Pragma("unroll") for (int n = 0; n < 2; ++n) { \
        dst[n][0] = *(const LAS bf16x8*)(lds + G_SB(b, hh) + boff0 + n * 2048); dst[n][1] = *(const LAS bf16x8*)(lds + G_SB(b, hh) + boff1 + n * 2048); } } while (0)
#define G_MMA(ai, bj, At, Bt) do { __builtin_amdgcn_s_setprio(1); _Pragma("unroll") for (int m = 0; m < 4; ++m) _Pragma("unroll") for (int n = 0; n < 2; ++n) _Pragma("unroll") for (int k = 0; k < 2; ++k) \
        acc[ai][bj][m][n] = __builtin_amdgcn_mfma_f32_16x16x32_bf16(Bt[n][k], At[m][k], acc[ai][bj][m][n], 0, 0, 0); __builtin_amdgcn_s_setprio(0); } while (0)
#define G_WAIT_V(n) asm volatile("s_waitcnt vmcnt(" #n ")" ::: "memory")
#define G_WAIT_L(n) asm volatile("s_waitcnt lgkmcnt(" #n ")" ::: "memory")
#define G_BAR __builtin_amdgcn_s_barrier()
#define G_SCHED __builtin_amdgcn_sched_barrier(0)
#define G_GIDX(t) ((unsigned)((4 * h + ((t) & 3)) * 16 + ((t) >> 2)))
    const int zero = 0;
    for (int r = 0; r < 2; ++r) {
        const unsigned target = call * (r == 0 ? 64u : 32u);
        const unsigned abase = OFF_AIMG + (unsigned)r * (128u * 262144u) + (unsigned)tm * 32768u, bbase = OFF_BIMG + (unsigned)tn * 32768u;
        char* cbase = ws + OFF_CNT + r * 128 * 128;
        f32x4 acc[2][2][4][2];
#pragma unroll
        for (int a = 0; a < 2; ++a)
#pragma unroll
            for (int b = 0; b < 2; ++b)
#pragma unroll
                for (int m = 0; m < 4; ++m)
#pragma unroll
                    for (int n = 0; n < 2; ++n) acc[a][b][m][n] = (f32x4){0.f, 0.f, 0.f, 0.f};
        bf16x8 At[4][2], B0[2][2], B1[2][2];
        STAMP(ws, r * 8 + 0);
        wait_word((unsigned*)(cbase + G_GIDX(0) * 128), target, tmo, call);
        wait_word((unsigned*)(cbase + G_GIDX(1) * 128), target, tmo, call);
        wait_word((unsigned*)(cbase + G_GIDX(2) * 128), target, tmo, call);
        STAMP(ws, r * 8 + 1);
        {
            const unsigned a0 = abase + G_GIDX(0) * 262144u, b0 = bbase + G_GIDX(0) * 262144u, a1 = abase + G_GIDX(1) * 262144u, b1 = bbase + G_GIDX(1) * 262144u;
            G_STAGE(G_SB(0, 0), b0); G_STAGE(G_SA(0, 0), a0); G_STAGE(G_SB(0, 1), b0 + 16384); G_STAGE(G_SA(0, 1), a0 + 16384);
            if (wr == 1) G_BAR;
            G_WAIT_V(4); G_BAR;
            G_STAGE(G_SB(1, 0), b1); G_STAGE(G_SA(1, 0), a1); G_STAGE(G_SB(1, 1), b1 + 16384);
            G_WAIT_V(6); G_BAR;
        }
        unsigned fA = target, fB;
        for (int t = 0; t < 64; t += 2) {
            const int t2 = t + 2 < 64 ? t + 2 : 63, t3 = t + 3 < 64 ? t + 3 : 63, t4 = t + 4 < 64 ? t + 4 : 63;
            const unsigned g1 = G_GIDX(t + 1), g2 = G_GIDX(t2), g3 = G_GIDX(t3), g4 = G_GIDX(t4);
            const unsigned a1 = abase + g1 * 262144u, a2 = abase + g2 * 262144u, b2 = bbase + g2 * 262144u, a3 = abase + g3 * 262144u, b3 = bbase + g3 * 262144u;
            asm volatile("global_load_dword %0, %1, %2 sc1" : "=v"(fB) : "v"(zero), "s"(cbase + g3 * 128));
            G_LDB(B0, 0, 0); G_SCHED; G_LDA(At, 0, 0); G_STAGE(G_SA(1, 1), a1 + 16384);
            G_WAIT_L(8); G_BAR; G_WAIT_L(0); G_MMA(0, 0, At, B0); G_BAR; G_SCHED;
            asm volatile("" : "+v"(fA));
            if ((int)(rfl(fA) - target) < 0) wait_word((unsigned*)(cbase + g2 * 128), target, tmo, call);
            G_LDB(B1, 0, 1); G_STAGE(G_SB(0, 0), b2);
            G_BAR; G_WAIT_L(0); G_MMA(0, 1, At, B1); G_BAR;
            G_LDA(At, 0, 1); G_STAGE(G_SA(0, 0), a2);
            G_BAR; G_WAIT_L(0); G_MMA(1, 0, At, B0); G_BAR; G_SCHED;
            G_STAGE(G_SB(0, 1), b2 + 16384);
            G_WAIT_V(6); G_BAR; G_MMA(1, 1, At, B1); G_BAR;
            asm volatile("global_load_dword %0, %1, %2 sc1" : "=v"(fA) : "v"(zero), "s"(cbase + g4 * 128));
            G_LDB(B0, 1, 0); G_SCHED; G_LDA(At, 1, 0); G_STAGE(G_SA(0, 1), a2 + 16384);
            G_WAIT_L(8); G_BAR; G_WAIT_L(0); G_MMA(0, 0, At, B0); G_BAR; G_SCHED;
            asm volatile("" : "+v"(fB));
            if ((int)(rfl(fB) - target) < 0) wait_word((unsigned*)(cbase + g3 * 128), target, tmo, call);
            G_LDB(B1, 1, 1); G_STAGE(G_SB(1, 0), b3);
            G_BAR; G_WAIT_L(0); G_MMA(0, 1, At, B1); G_BAR;
            G_LDA(At, 1, 1); G_STAGE(G_SA(1, 0), a3);
            G_BAR; G_WAIT_L(0); G_MMA(1, 0, At, B0); G_BAR; G_SCHED;
            G_STAGE(G_SB(1, 1), b3 + 16384);
            G_WAIT_V(6); G_BAR; G_MMA(1, 1, At, B1); G_BAR;
        }
        G_WAIT_V(0);
        asm volatile("" : "+v"(fA));
        if (wr == 0) G_BAR;
        G_BAR;
        STAMP(ws, r * 8 + 2);
        if (h) {
#pragma unroll
            for (int b = 0; b < 2; ++b)
#pragma unroll
                for (int m = 0; m < 4; ++m)
#pragma unroll
                    for (int n = 0; n < 2; ++n) { const f32x4 tq = acc[0][b][m][n]; acc[0][b][m][n] = acc[1][b][m][n]; acc[1][b][m][n] = tq; }
        }
        const unsigned xl = (unsigned)((r * 64 + tile) * 2);
#pragma unroll
        for (int b = 0; b < 2; ++b)
#pragma unroll
            for (int m = 0; m < 4; ++m)
#pragma unroll
                for (int n = 0; n < 2; ++n)
                    __builtin_amdgcn_raw_buffer_store_b128(__builtin_bit_cast(u32x4, acc[1][b][m][n]), wsr, lane * 16 + ((b * 4 + m) * 2 + n) * 1024, OFF_XBUF + ((xl + h) * 8 + wid) * 16384u, AUX_SC1);
        WAIT_VM0();
        if (lane == 0) __hip_atomic_store((unsigned*)(ws + OFF_XFLAG + (xl + h) * 128 + wid * 16), call, __ATOMIC_RELAXED, __HIP_MEMORY_SCOPE_AGENT);
        wait_word((unsigned*)(ws + OFF_XFLAG + (xl + (h ^ 1)) * 128 + wid * 16), call, tmo, call);
        const bool dead = rfl(__hip_atomic_load(tmo, __ATOMIC_RELAXED, __HIP_MEMORY_SCOPE_AGENT)) == call;
        STAMP(ws, r * 8 + 3);
        LAS unsigned char* my = lds + wid * 8192;
#pragma unroll
        for (int b = 0; b < 2; ++b)
#pragma unroll
            for (int m = 0; m < 4; ++m) {
                const f32x4 p0 = __builtin_bit_cast(f32x4, __builtin_amdgcn_raw_buffer_load_b128(wsr, lane * 16 + ((b * 4 + m) * 2 + 0) * 1024, OFF_XBUF + ((xl + (h ^ 1)) * 8 + wid) * 16384u, AUX_SC1));
                const f32x4 p1 = __builtin_bit_cast(f32x4, __builtin_amdgcn_raw_buffer_load_b128(wsr, lane * 16 + ((b * 4 + m) * 2 + 1) * 1024, OFF_XBUF + ((xl + (h ^ 1)) * 8 + wid) * 16384u, AUX_SC1));
                const f32x4 v0 = acc[0][b][m][0] + p0, v1 = acc[0][b][m][1] + p1;
                u32x4 o;
                o.x = cvt_pk_bf16(gelu_f(v0[0]), gelu_f(v0[1])); o.y = cvt_pk_bf16(gelu_f(v0[2]), gelu_f(v0[3]));
                o.z = cvt_pk_bf16(gelu_f(v1[0]), gelu_f(v1[1])); o.w = cvt_pk_bf16(gelu_f(v1[2]), gelu_f(v1[3]));
                const int row = m * 16 + fr, ch = b * 4 + fq;
                *(LAS u32x4*)(my + row * 128 + ((ch ^ (row & 7)) * 16)) = o;
            }
        asm volatile("s_waitcnt lgkmcnt(0)" ::: "memory");
        u32x4 c[8];
#pragma unroll
        for (int j = 0; j < 8; ++j) { const int row = j * 8 + (lane >> 3); c[j] = *(LAS u32x4*)(my + row * 128 + (((lane & 7) ^ (row & 7)) * 16)); }
        if (dead) {
#pragma unroll
            for (int j = 0; j < 8; ++j) c[j] = (u32x4){0x7FC07FC0u, 0x7FC07FC0u, 0x7FC07FC0u, 0x7FC07FC0u};
        }
        {
            const rsrc_t outr = mk_rsrc(P.out, (unsigned)DIM * DIM * 2u);
            const unsigned row0 = (P.rank >> 2) * 4096 + r * 2048 + tm * 256 + h * 128 + wr * 64, col0 = (P.rank & 3) * 2048 + tn * 256 + wc * 64;
            const unsigned dso = (row0 * DIM + col0) * 2u, vo = (unsigned)(lane >> 3) * (DIM * 2u) + (unsigned)(lane & 7) * 16u;
#pragma unroll
            for (int j = 0; j < 8; ++j) __builtin_amdgcn_raw_buffer_store_b128(c[j], outr, vo, dso + j * (8u * DIM * 2u), AUX_SC1);
            const unsigned o = (xl + h) * 8 + wid;
            publish_block(P, wsr, c, 4096u + o, 1536u + o, 0xFFu & ~(1u << P.rank), call, lane);
        }
        STAMP(ws, r * 8 + 4);
        asm volatile("s_waitcnt lgkmcnt(0)" ::: "memory");
        G_BAR;
    }
#undef G_SA
#undef G_SB
#undef G_STAGE
#undef G_LDA
#undef G_LDB
#undef G_MMA
#undef G_WAIT_V
#undef G_WAIT_L
#undef G_BAR
#undef G_SCHED
#undef G_GIDX
}

__global__ __launch_bounds__(512, 2) void gemm_allgather_kernel(Params P) {
    extern __shared__ __attribute__((aligned(16))) unsigned char shm[];
    LAS unsigned char* lds = (LAS unsigned char*)shm;
    char* ws = P.ws[P.rank];
    if (threadIdx.x == 0) {
        const unsigned t = __hip_atomic_fetch_add((unsigned*)(ws + OFF_TICKET), 1u, __ATOMIC_RELAXED, __HIP_MEMORY_SCOPE_AGENT);
        *(LAS unsigned*)(lds + 131072) = t / GRID + 1;
    }
    __syncthreads();
    const unsigned call = rfl(*(LAS unsigned*)(lds + 131072));
    const int b = blockIdx.x;
    STAMP(ws, 60);
    if (b < NGEMM) gemm_role(P, lds, call, b);
    else if (b < NGEMM + NPULL) puller_role(P, call, b - NGEMM);
    else packer_role(P, lds, call, b - NGEMM - NPULL);
    STAMP(ws, 61);
}

}

extern "C" void kernel_launch(int rank, int nranks, ncclComm_t comm,
                              hipStream_t stream,
                              void* const* d_in, const int* in_sizes, int n_in,
                              void* d_out, int out_size,
                              const int* in_dtypes, int out_dtype,
                              void* const* d_out_all,
                              void* const* d_ws_all, size_t ws_bytes) {
    (void)comm; (void)in_sizes; (void)in_dtypes; (void)out_dtype; (void)d_out_all;
    if (nranks != gar::WORLD || n_in < 2 || ws_bytes < gar::WS_NEED || out_size != gar::DIM * gar::DIM || (unsigned)rank >= (unsigned)gar::WORLD) {
        fprintf(stderr, "unexpected problem shape\n");
        return;
    }
    static bool attr = false;
    if (!attr) {
        hipError_t e = hipFuncSetAttribute((const void*)gar::gemm_allgather_kernel, hipFuncAttributeMaxDynamicSharedMemorySize, gar::LDS_BYTES);
        if (e != hipSuccess) { fprintf(stderr, "hipFuncSetAttribute: %s\n", hipGetErrorString(e)); return; }
        attr = true;
    }
    gar::Params P;
    memset(&P, 0, sizeof P);
    P.x = d_in[0]; P.w = d_in[1]; P.out = d_out; P.rank = rank;
    for (int p = 0; p < gar::WORLD; ++p) P.ws[p] = (char*)d_ws_all[p];
    hipLaunchKernelGGL(gar::gemm_allgather_kernel, dim3(gar::GRID), dim3(512), gar::LDS_BYTES, stream, P);
    hipError_t e = hipGetLastError();
    if (e != hipSuccess) fprintf(stderr, "launch: %s\n", hipGetErrorString(e));
}
```

```cpp
#include <hip/hip_runtime.h>
#include <rccl/rccl.h>
#include <cstdio>
#include <cstdint>
#include <cstring>

namespace gar {

#define LAS __attribute__((address_space(3)))
typedef unsigned u32x4 __attribute__((ext_vector_type(4)));
typedef unsigned short u16x2 __attribute__((ext_vector_type(2)));
typedef short bf16x8 __attribute__((ext_vector_type(8)));
typedef float f32x4 __attribute__((ext_vector_type(4)));
typedef __amdgpu_buffer_rsrc_t rsrc_t;

constexpr int WORLD = 8;
constexpr int DIM = 8192;
constexpr int KSH = 1024;
constexpr int NGEMM = 128;
constexpr int GRID = 256;
#define PULL_WG_PER_LINK 12
#define PULL_WAVES 1
constexpr int PWL = PULL_WG_PER_LINK;
constexpr int PWAVES = PULL_WAVES;
constexpr int NPULL = 7 * PWL;
constexpr int NPACK = GRID - NGEMM - NPULL;
constexpr int NSEQ = 3584;
constexpr unsigned long long DEADLINE = 40000000ull;

constexpr unsigned OFF_TICKET = 0, OFF_TMO = 128;
constexpr unsigned OFF_LINK = 1024;
constexpr unsigned OFF_CNT = 4096;
constexpr unsigned OFF_XFLAG = OFF_CNT + 256 * 128;
constexpr unsigned OFF_GRAN = 131072;
constexpr unsigned OFF_SLOT = 1u << 20;
constexpr unsigned OFF_AIMG = 64u << 20;
constexpr unsigned OFF_BIMG = 128u << 20;
constexpr unsigned OFF_XBUF = 160u << 20;
constexpr unsigned WS_USED = 192u << 20;
constexpr unsigned WS_NEED = WS_USED;
static_assert(OFF_XFLAG + 256 * 128 <= OFF_GRAN, "map");
static_assert(OFF_GRAN + WORLD * NSEQ * 16 <= OFF_SLOT, "map");
static_assert(OFF_SLOT + 6144u * 8192u <= OFF_AIMG, "map");
constexpr int LDS_BYTES = 131072 + 256;

struct Params {
    const void* x; const void* w; void* out;
    char* ws[WORLD];
    int rank; int pad;
};

constexpr int AUX_SC1 = 16, AUX_SYS = 17;

__device__ __forceinline__ rsrc_t mk_rsrc(const void* p, unsigned bytes) {
    return __builtin_amdgcn_make_buffer_rsrc(const_cast<void*>(p), 0, bytes, 0x00020000);
}
__device__ __forceinline__ unsigned rfl(unsigned v) { return __builtin_amdgcn_readfirstlane(v); }
__device__ __forceinline__ unsigned bfi(unsigned mask, unsigned a, unsigned b) { return (a & mask) | (b & ~mask); }
__device__ __forceinline__ unsigned perm(unsigned hi, unsigned lo, unsigned sel) { return __builtin_amdgcn_perm(hi, lo, sel); }
#define WAIT_VM0() asm volatile("s_waitcnt vmcnt(0)" ::: "memory")

#define STAMP(ws, idx) do { } while (0)

__device__ __forceinline__ bool give_up(unsigned long long t0, unsigned* tmo, unsigned call) {
    if (rfl(__hip_atomic_load(tmo, __ATOMIC_RELAXED, __HIP_MEMORY_SCOPE_AGENT)) == call) return true;
    if (__builtin_amdgcn_s_memrealtime() - t0 > DEADLINE) { __hip_atomic_store(tmo, call, __ATOMIC_RELAXED, __HIP_MEMORY_SCOPE_AGENT); return true; }
    return false;
}
__device__ __forceinline__ void wait_word(unsigned* p, unsigned target, unsigned* tmo, unsigned call) {
    if ((int)(rfl(__hip_atomic_load(p, __ATOMIC_RELAXED, __HIP_MEMORY_SCOPE_AGENT)) - target) >= 0) return;
    const unsigned long long t0 = __builtin_amdgcn_s_memrealtime();
    while ((int)(rfl(__hip_atomic_load(p, __ATOMIC_RELAXED, __HIP_MEMORY_SCOPE_AGENT)) - target) < 0) {
        if (give_up(t0, tmo, call)) break;
        __builtin_amdgcn_s_sleep(4);
    }
}

struct Hdr { unsigned h0, h1, h2; };

__device__ __forceinline__ Hdr encode_block(const u32x4 (&c)[8], u32x4 (&pl)[6], int lane) {
    unsigned H[16];
    u16x2 m1 = {0, 0}, m2 = {0, 0};
#pragma unroll
    for (int j = 0; j < 8; ++j) {
        H[2 * j] = perm(c[j].y, c[j].x, 0x07050301u); H[2 * j + 1] = perm(c[j].w, c[j].z, 0x07050301u);
        const unsigned l0 = perm(c[j].y, c[j].x, 0x06040200u), l1 = perm(c[j].w, c[j].z, 0x06040200u);
        if (j & 1) { pl[j >> 1].z = l0; pl[j >> 1].w = l1; } else { pl[j >> 1].x = l0; pl[j >> 1].y = l1; }
    }
#pragma unroll
    for (int g = 0; g < 16; ++g) {
        const unsigned t = H[g] & 0x7F7F7F7Fu;
        m1 = __builtin_elementwise_max(m1, __builtin_bit_cast(u16x2, t));
        m2 = __builtin_elementwise_max(m2, __builtin_bit_cast(u16x2, t << 8));
    }
    const u16x2 mm = __builtin_elementwise_max(m1, m2);
    unsigned mx = (mm.x > mm.y ? mm.x : mm.y) >> 8;
#pragma unroll
    for (int o = 32; o >= 1; o >>= 1) { const unsigned y = __shfl_xor(mx, o); mx = mx > y ? mx : y; }
    const unsigned base = rfl(mx), B = base * 0x01010101u;
    unsigned D[16], any = 0;
#pragma unroll
    for (int g = 0; g < 16; ++g) { D[g] = B - (H[g] & 0x7F7F7F7Fu); any |= D[g]; }
#pragma unroll
    for (int j = 0; j < 8; ++j) {
        const unsigned a = bfi(0x07070707u, D[2 * j], H[2 * j] >> 4), b = bfi(0x70707070u, D[2 * j + 1] << 4, H[2 * j + 1]);
        const unsigned wv = bfi(0x0F0F0F0Fu, a, b);
        if (j == 0) pl[4].x = wv; if (j == 1) pl[4].y = wv; if (j == 2) pl[4].z = wv; if (j == 3) pl[4].w = wv;
        if (j == 4) pl[5].x = wv; if (j == 5) pl[5].y = wv; if (j == 6) pl[5].z = wv; if (j == 7) pl[5].w = wv;
    }
    unsigned r0 = 0, r1 = 0, r2 = 0, n = 0;
    if (__ballot((any & 0x78787878u) != 0) != 0ull) {
#pragma unroll
        for (int g = 0; g < 16; ++g) {
            const unsigned f = D[g] & 0x78787878u;
            unsigned long long mask = __ballot(f != 0);
            while (mask != 0ull && n <= 3) {
                const int L = __builtin_ctzll(mask); mask &= mask - 1;
                const unsigned fl = __builtin_amdgcn_readlane(f, L);
                const unsigned lo = __builtin_amdgcn_readlane((g & 1) ? c[g >> 1].z : c[g >> 1].x, L);
                const unsigned hi = __builtin_amdgcn_readlane((g & 1) ? c[g >> 1].w : c[g >> 1].y, L);
#pragma unroll
                for (int b = 0; b < 4; ++b)
                    if ((fl >> (8 * b)) & 0xFFu) {
                        const unsigned val = (((b & 2) ? hi : lo) >> (16 * (b & 1))) & 0xFFFFu;
                        const unsigned rec = (unsigned)(L * 64 + g * 4 + b) | (val << 12);
                        if (n == 0) r0 = rec; else if (n == 1) r1 = rec; else if (n == 2) r2 = rec;
                        ++n;
                    }
            }
        }
    }
    Hdr h;
    const unsigned raw = n > 3 ? 1u : 0u;
    h.h0 = r0 | ((base & 15u) << 28);
    h.h1 = r1 | ((base >> 4) << 28) | (raw << 31);
    h.h2 = r2 | ((n & 3u) << 28);
    (void)lane;
    return h;
}

__device__ __forceinline__ void decode_block(const u32x4 (&pl)[6], unsigned base, u32x4 (&c)[8]) {
    const unsigned B = base * 0x01010101u;
#pragma unroll
    for (int j = 0; j < 8; ++j) {
        const unsigned wv = j == 0 ? pl[4].x : j == 1 ? pl[4].y : j == 2 ? pl[4].z : j == 3 ? pl[4].w : j == 4 ? pl[5].x : j == 5 ? pl[5].y : j == 6 ? pl[5].z : pl[5].w;
        const unsigned l0 = (j & 1) ? pl[j >> 1].z : pl[j >> 1].x, l1 = (j & 1) ? pl[j >> 1].w : pl[j >> 1].y;
        const unsigned ha = (B - (wv & 0x07070707u)) | ((wv << 4) & 0x80808080u);
        const unsigned hb = (B - ((wv >> 4) & 0x07070707u)) | (wv & 0x80808080u);
        c[j].x = perm(ha, l0, 0x05010400u); c[j].y = perm(ha, l0, 0x07030602u);
        c[j].z = perm(hb, l1, 0x05010400u); c[j].w = perm(hb, l1, 0x07030602u);
    }
}

__device__ __forceinline__ void store_granule(char* p, u32x4 g) {
    asm volatile("global_store_dwordx4 %0, %1, off sc0 sc1\n\ts_nop 1" :: "v"(p), "v"(g) : "memory");
}

__device__ __forceinline__ void publish_block(const Params& P, rsrc_t wsr, const u32x4 (&c)[8], unsigned slot, unsigned n, unsigned dmask, unsigned call, int lane) {
    u32x4 pl[6];
    const Hdr h = encode_block(c, pl, lane);
    const unsigned so = OFF_SLOT + slot * 8192u;
    if (h.h1 >> 31) {
#pragma unroll
        for (int j = 0; j < 8; ++j) __builtin_amdgcn_raw_buffer_store_b128(c[j], wsr, lane * 16 + j * 1024, so, AUX_SYS);
    } else {
#pragma unroll
        for (int j = 0; j < 6; ++j) __builtin_amdgcn_raw_buffer_store_b128(pl[j], wsr, lane * 16 + j * 1024, so, AUX_SYS);
    }
    WAIT_VM0();
    if (lane < WORLD && ((dmask >> lane) & 1u)) {
        char* p = P.ws[lane & 7] + OFF_GRAN + ((unsigned)P.rank * NSEQ + n) * 16u;
        store_granule(p, (u32x4){call, h.h0, h.h1, h.h2});
    }
}

__device__ __forceinline__ void packer_role(const Params& P, LAS unsigned char* lds, unsigned call, int wg) {
    const int lane = threadIdx.x & 63, wid = rfl(threadIdx.x >> 6);
    char* ws = P.ws[P.rank];
    const rsrc_t wsr = mk_rsrc(ws, WS_USED);
    const rsrc_t xr = mk_rsrc(P.x, DIM * KSH * 2), wr = mk_rsrc(P.w, DIM * KSH * 2);
    const int myi = P.rank >> 2, myj = P.rank & 3;
    LAS unsigned char* my = lds + wid * 8192;
    for (int p = wg * 8 + wid; p < 4096; p += NPACK * 8) {
        bool is_w; int kk, sel, r, bi;
        if (p < 3072) { kk = p / 192; const int rem = p - kk * 192; bi = rem & 31; r = 0; is_w = rem < 128; sel = is_w ? (rem >> 5) : ((rem - 128) >> 5); }
        else { const int q = p - 3072; kk = q >> 6; sel = (q >> 5) & 1; bi = q & 31; r = 1; is_w = false; }
        u32x4 c[8];
        if (!is_w) {
            const int rr = lane >> 3;
            const unsigned vo = (unsigned)rr * 2048u + (unsigned)(((lane & 7) ^ rr) * 16);
            const unsigned so = (unsigned)(sel * 4096 + r * 2048 + bi * 64) * 2048u + (unsigned)kk * 128u;
#pragma unroll
            for (int j = 0; j < 8; ++j) c[j] = __builtin_amdgcn_raw_buffer_load_b128(xr, vo + j * 8 * 2048, so, 0);
        } else {
            const int pn = bi >> 2, bj = (bi >> 1) & 1, q = bi & 1;
            const int gk = lane >> 3, cc = lane & 7, wcl = cc >> 2, cq = cc & 3;
            const unsigned vo = (unsigned)(gk * 8) * 16384u + (unsigned)((64 * (2 * q + wcl) + 32 * bj + 8 * cq) * 2);
            const unsigned so = (unsigned)(kk * 64) * 16384u + (unsigned)((sel * 2048 + pn * 256) * 2);
            u32x4 R[8];
#pragma unroll
            for (int jj = 0; jj < 8; ++jj) R[jj] = __builtin_amdgcn_raw_buffer_load_b128(wr, vo + jj * 16384, so, 0);
#pragma unroll
            for (int j = 0; j < 8; ++j) {
                const unsigned s = (j & 1) ? 0x07060302u : 0x05040100u;
                u32x4 t;
                t.x = perm(R[1][j >> 1], R[0][j >> 1], s); t.y = perm(R[3][j >> 1], R[2][j >> 1], s);
                t.z = perm(R[5][j >> 1], R[4][j >> 1], s); t.w = perm(R[7][j >> 1], R[6][j >> 1], s);
                const int s64 = 32 * wcl + 16 * (j >> 2) + 4 * cq + (j & 3);
                *(LAS u32x4*)(my + s64 * 128 + ((gk ^ (s64 & 7)) * 16)) = t;
            }
            asm volatile("s_waitcnt lgkmcnt(0)" ::: "memory");
#pragma unroll
            for (int j = 0; j < 8; ++j) c[j] = *(LAS u32x4*)(my + j * 1024 + lane * 16);
            asm volatile("s_waitcnt lgkmcnt(0)" ::: "memory");
        }
        const int g = P.rank * 16 + kk;
        const bool self = is_w ? (sel == myj) : (sel == myi);
        if (self) {
            const unsigned so = is_w ? OFF_BIMG + (unsigned)(g * 32 + bi) * 8192u : OFF_AIMG + (unsigned)((r * 128 + g) * 32 + bi) * 8192u;
#pragma unroll
            for (int j = 0; j < 8; ++j) __builtin_amdgcn_raw_buffer_store_b128(c[j], wsr, lane * 16 + j * 1024, so, AUX_SC1);
        }
        const unsigned slot = is_w ? 2048u + (unsigned)((sel * 16 + kk) * 32 + bi) : (unsigned)(((sel * 2 + r) * 16 + kk) * 32 + bi);
        const unsigned n = is_w ? (unsigned)(kk * 64 + bi) : (r == 0 ? (unsigned)(kk * 64 + 32 + bi) : (unsigned)(1024 + kk * 32 + bi));
        unsigned dmask = is_w ? (0x11u << sel) : (0x0Fu << (4 * sel));
        dmask &= ~(1u << P.rank);
        publish_block(P, wsr, c, slot, n, dmask, call, lane);
        if (self && lane == 0)
            __hip_atomic_fetch_add((unsigned*)(ws + OFF_CNT + (r * 128 + g) * 128), 1u, __ATOMIC_RELAXED, __HIP_MEMORY_SCOPE_AGENT);
    }
}

__device__ __forceinline__ u32x4 load_granule(rsrc_t wsr, unsigned off) { return __builtin_amdgcn_raw_buffer_load_b128(wsr, 0, off, AUX_SYS); }

struct Blk { unsigned slot, dso, cnt; bool is_out; };
__device__ __forceinline__ Blk locate(unsigned n, int src, int myi, int myj) {
    Blk b; b.is_out = false;
    if (n < 1024) { const int kk = n >> 6, bi = n & 31, g = src * 16 + kk; b.cnt = OFF_CNT + g * 128;
        if ((n & 63) < 32) { b.slot = 2048u + (myj * 16 + kk) * 32 + bi; b.dso = OFF_BIMG + (unsigned)(g * 32 + bi) * 8192u; }
        else { b.slot = ((myi * 2) * 16 + kk) * 32 + bi; b.dso = OFF_AIMG + (unsigned)(g * 32 + bi) * 8192u; } }
    else if (n < 1536) { const int kk = (n - 1024) >> 5, bi = n & 31, g = src * 16 + kk; b.cnt = OFF_CNT + (128 + g) * 128;
        b.slot = ((myi * 2 + 1) * 16 + kk) * 32 + bi; b.dso = OFF_AIMG + (unsigned)((128 + g) * 32 + bi) * 8192u; }
    else { const int o = n - 1536, w = o & 7, h = (o >> 3) & 1, tile = (o >> 4) & 63, r = o >> 10; b.cnt = 0; b.is_out = true;
        b.slot = 4096u + o;
        const unsigned row0 = (src >> 2) * 4096 + r * 2048 + (tile >> 3) * 256 + h * 128 + (w >> 2) * 64, col0 = (src & 3) * 2048 + (tile & 7) * 256 + (w & 3) * 64;
        b.dso = (row0 * DIM + col0) * 2u; }
    return b;
}
__device__ __forceinline__ void fetch_payload(rsrc_t rem, LAS unsigned char* dst, unsigned so, bool raw, int lane) {
#pragma unroll
    for (int j = 0; j < 6; ++j) __builtin_amdgcn_raw_ptr_buffer_load_lds(rem, (LAS void*)(dst + j * 1024), 16, lane * 16, so + j * 1024, 0, AUX_SYS);
    if (raw) {
#pragma unroll
        for (int j = 6; j < 8; ++j) __builtin_amdgcn_raw_ptr_buffer_load_lds(rem, (LAS void*)(dst + j * 1024), 16, lane * 16, so + j * 1024, 0, AUX_SYS);
    }
}
__device__ __forceinline__ void signal_cnt(char* ws, unsigned off, int lane) {
    if (off && lane == 0) __hip_atomic_fetch_add((unsigned*)(ws + off), 1u, __ATOMIC_RELAXED, __HIP_MEMORY_SCOPE_AGENT);
}

__device__ __forceinline__ void puller_role(const Params& P, LAS unsigned char* lds, unsigned call, int wg) {
    const int lane = threadIdx.x & 63, wid = rfl(threadIdx.x >> 6);
    if (wid >= PWAVES) return;
    const int link = wg % 7, src = (P.rank + 1 + link) & 7;
    constexpr unsigned NW = PWL * PWAVES;
    char* ws = P.ws[P.rank];
    unsigned* tmo = (unsigned*)(ws + OFF_TMO);
    const rsrc_t wsr = mk_rsrc(ws, WS_USED), rem = mk_rsrc(P.ws[src], WS_USED), outr = mk_rsrc(P.out, (unsigned)DIM * DIM * 2u);
    const int myi = P.rank >> 2, myj = P.rank & 3;
    const unsigned gbase = OFF_GRAN + (unsigned)src * NSEQ * 16u;
    char* const gptr = ws + gbase;
    LAS unsigned char* ring = lds + wid * 16384;
    const unsigned tko = OFF_LINK + (unsigned)src * 128u, tbase = (call - 1u) * (NSEQ + 3u * NW);
    unsigned n, n1, n2;
    {
        unsigned t0 = 0;
        if (lane == 0) t0 = __hip_atomic_fetch_add((unsigned*)(ws + tko), 3u, __ATOMIC_RELAXED, __HIP_MEMORY_SCOPE_AGENT);
        n = rfl(t0) - tbase; n1 = n + 1; n2 = n + 2;
    }
    unsigned h0 = 0, h1 = 0, h2 = 0;
    unsigned g1x, g1y, g1z, g1w;
    bool dead_cur = false;
    unsigned pend = 0;
    int s = 0;
    const int zero = 0, one = 1;
    if (n < NSEQ) {
        u32x4 gr = load_granule(wsr, gbase + n * 16u);
        if ((int)(rfl(gr.x) - call) < 0) {
            const unsigned long long t0 = __builtin_amdgcn_s_memrealtime();
            for (;;) {
                gr = load_granule(wsr, gbase + n * 16u);
                if ((int)(rfl(gr.x) - call) >= 0) break;
                if (give_up(t0, tmo, call)) { dead_cur = true; break; }
                __builtin_amdgcn_s_sleep(2);
                asm volatile("" ::: "memory");
            }
        }
        h0 = rfl(gr.y); h1 = rfl(gr.z); h2 = rfl(gr.w);
        if (!dead_cur) fetch_payload(rem, ring, OFF_SLOT + locate(n, src, myi, myj).slot * 8192u, (h1 >> 31) != 0, lane);
    }
    {
        const u32x4 gr = load_granule(wsr, gbase + (n1 < NSEQ ? n1 : 0u) * 16u);
        g1x = rfl(gr.x); g1y = rfl(gr.y); g1z = rfl(gr.z); g1w = rfl(gr.w);
    }
    WAIT_VM0();
    while (n < NSEQ) {
        const bool has1 = n1 < NSEQ;
        const bool ready1 = has1 && (int)(g1x - call) >= 0;
        if (ready1) fetch_payload(rem, ring + (s ^ 1) * 8192, OFF_SLOT + locate(n1, src, myi, myj).slot * 8192u, (g1z >> 31) != 0, lane);
        u32x4 g2; unsigned tk = 0, tmv;
        asm volatile("global_load_dwordx4 %0, %1, %2 sc0 sc1" : "=v"(g2) : "v"(zero), "s"(gptr + (n2 < NSEQ ? n2 : 0u) * 16u) : "memory");
        asm volatile("global_load_dword %0, %1, %2 sc1" : "=v"(tmv) : "v"(zero), "s"(tmo) : "memory");
        if (lane == 0) asm volatile("global_atomic_add %0, %1, %2, %3 sc0" : "=v"(tk) : "v"(zero), "v"(one), "s"(ws + tko) : "memory");
        const Blk b = locate(n, src, myi, myj);
        const bool raw = (h1 >> 31) != 0;
        u32x4 c[8];
        LAS unsigned char* cur = ring + s * 8192 + lane * 16;
        if (dead_cur) {
#pragma unroll
            for (int j = 0; j < 8; ++j) c[j] = (u32x4){0x7FC07FC0u, 0x7FC07FC0u, 0x7FC07FC0u, 0x7FC07FC0u};
        } else if (raw) {
#pragma unroll
            for (int j = 0; j < 8; ++j) c[j] = *(LAS u32x4*)(cur + j * 1024);
        } else {
            u32x4 pl[6];
#pragma unroll
            for (int j = 0; j < 6; ++j) pl[j] = *(LAS u32x4*)(cur + j * 1024);
            decode_block(pl, (h0 >> 28) | (((h1 >> 28) & 7u) << 4), c);
        }
        const rsrc_t dr = b.is_out ? outr : wsr;
        const unsigned sj = b.is_out ? 8u * DIM * 2u : 1024u, sr = b.is_out ? DIM * 2u : 128u;
        const unsigned vo = (unsigned)(lane >> 3) * sr + (unsigned)(lane & 7) * 16u;
#pragma unroll
        for (int j = 0; j < 8; ++j) __builtin_amdgcn_raw_buffer_store_b128(c[j], dr, vo, b.dso + j * sj, AUX_SC1);
        if (!dead_cur && !raw) {
            const unsigned ne = (h2 >> 28) & 3u;
            for (unsigned e = 0; e < ne; ++e) {
                const unsigned rec = e == 0 ? h0 : e == 1 ? h1 : h2, pos = rec & 0xFFFu, v = pos & 63u;
                if ((unsigned)lane == (pos >> 6)) __builtin_amdgcn_raw_buffer_store_b16((unsigned short)((rec >> 12) & 0xFFFFu), dr, vo + (v & 7u) * 2u, b.dso + (v >> 3) * sj, AUX_SC1);
            }
        }
        asm volatile("s_waitcnt vmcnt(8)" : "+v"(g2), "+v"(tk), "+v"(tmv) :: "memory");
        signal_cnt(ws, pend, lane);
        pend = b.cnt;
        bool dead1 = false;
        if (has1 && !ready1) {
            WAIT_VM0();
            signal_cnt(ws, pend, lane); pend = 0;
            const unsigned long long t0 = __builtin_amdgcn_s_memrealtime();
            for (;;) {
                const u32x4 gr = load_granule(wsr, gbase + n1 * 16u);
                g1x = rfl(gr.x); g1y = rfl(gr.y); g1z = rfl(gr.z); g1w = rfl(gr.w);
                if ((int)(g1x - call) >= 0) break;
                if (give_up(t0, tmo, call)) { dead1 = true; break; }
                __builtin_amdgcn_s_sleep(2);
                asm volatile("" ::: "memory");
            }
            if (!dead1) fetch_payload(rem, ring + (s ^ 1) * 8192, OFF_SLOT + locate(n1, src, myi, myj).slot * 8192u, (g1z >> 31) != 0, lane);
            WAIT_VM0();
        }
        dead_cur = dead1 || (n1 >= 1536 && rfl(tmv) == call);
        n = n1; h0 = g1y; h1 = g1z; h2 = g1w;
        n1 = n2; g1x = rfl(g2.x); g1y = rfl(g2.y); g1z = rfl(g2.z); g1w = rfl(g2.w);
        n2 = rfl(tk) - tbase;
        s ^= 1;
    }
    WAIT_VM0();
    signal_cnt(ws, pend, lane);
}

__device__ __forceinline__ float gelu_f(float y) {
    const float c1 = -2.0f * 0.7978845608028654f * 1.4426950408889634f, c2 = c1 * 0.044715f;
    const float e = __builtin_amdgcn_exp2f(y * (c1 + c2 * y * y));
    return y * __builtin_amdgcn_rcpf(1.0f + e);
}
__device__ __forceinline__ unsigned cvt_pk_bf16(float lo, float hi) { unsigned r; asm("v_cvt_pk_bf16_f32 %0, %1, %2" : "=v"(r) : "v"(lo), "v"(hi)); return r; }

__device__ __forceinline__ void gemm_role(const Params& P, LAS unsigned char* lds, unsigned call, int wg) {
    const int tid = threadIdx.x, wid = rfl(tid >> 6), lane = tid & 63, wr = wid >> 2, wc = wid & 3, fr = lane & 15, fq = lane >> 4;
    char* ws = P.ws[P.rank];
    unsigned* tmo = (unsigned*)(ws + OFF_TMO);
    const rsrc_t wsr = mk_rsrc(ws, WS_USED);
    const int xcd = wg & 7, q = wg >> 3, h = xcd & 1, blk = xcd >> 1;
    const int tm = (blk >> 1) * 4 + (q >> 2), tn = (blk & 1) * 4 + (q & 3), tile = tm * 8 + tn;
    const unsigned voff = (unsigned)tid * 16u, ldsw = (unsigned)wid * 1024u;
    const int aoff0 = (wr * 64 + fr) * 128 + ((fq ^ (fr & 7)) << 4), aoff1 = aoff0 ^ 64;
    const int boff0 = (wc * 32 + fr) * 128 + ((fq ^ (fr & 7)) << 4), boff1 = boff0 ^ 64;
#define G_SA(b, hh) (((b) * 2 + (hh)) * 16384)
#define G_SB(b, hh) ((4 + (b) * 2 + (hh)) * 16384)
#define G_STAGE(bufoff, soff) do { _Pragma("unroll") for (int _i = 0; _i < 2; ++_i) \
        __builtin_amdgcn_raw_ptr_buffer_load_lds(wsr, (LAS void*)(lds + (bufoff) + ldsw + _i * 8192), 16, voff, (soff) + _i * 8192, 0, AUX_SC1); } while (0)
#define G_LDA(dst, b, hh) do { _Pragma("unroll") for (int m = 0; m < 4; ++m) { \
        dst[m][0] = *(const LAS bf16x8*)(lds + G_SA(b, hh) + aoff0 + m * 2048); dst[m][1] = *(const LAS bf16x8*)(lds + G_SA(b, hh) + aoff1 + m * 2048); } } while (0)
#define G_LDB(dst, b, hh) do { _Pragma("unroll") for (int n = 0; n < 2; ++n) { \
        dst[n][0] = *(const LAS bf16x8*)(lds + G_SB(b, hh) + boff0 + n * 2048); dst[n][1] = *(const LAS bf16x8*)(lds + G_SB(b, hh) + boff1 + n * 2048); } } while (0)
#define G_MMA(ai, bj, At, Bt) do { __builtin_amdgcn_s_setprio(1); _Pragma("unroll") for (int m = 0; m < 4; ++m) _Pragma("unroll") for (int n = 0; n < 2; ++n) _Pragma("unroll") for (int k = 0; k < 2; ++k) \
        acc[ai][bj][m][n] = __builtin_amdgcn_mfma_f32_16x16x32_bf16(Bt[n][k], At[m][k], acc[ai][bj][m][n], 0, 0, 0); __builtin_amdgcn_s_setprio(0); } while (0)
#define G_WAIT_V(n) asm volatile("s_waitcnt vmcnt(" #n ")" ::: "memory")
#define G_WAIT_L(n) asm volatile("s_waitcnt lgkmcnt(" #n ")" ::: "memory")
#define G_BAR __builtin_amdgcn_s_barrier()
#define G_SCHED __builtin_amdgcn_sched_barrier(0)
#define G_GIDX(t) ((unsigned)((4 * h + ((t) & 3)) * 16 + ((t) >> 2)))
    const int zero = 0;
    for (int r = 0; r < 2; ++r) {
        const unsigned target = call * (r == 0 ? 64u : 32u);
        const unsigned abase = OFF_AIMG + (unsigned)r * (128u * 262144u) + (unsigned)tm * 32768u, bbase = OFF_BIMG + (unsigned)tn * 32768u;
        char* cbase = ws + OFF_CNT + r * 128 * 128;
        f32x4 acc[2][2][4][2];
#pragma unroll
        for (int a = 0; a < 2; ++a)
#pragma unroll
            for (int b = 0; b < 2; ++b)
#pragma unroll
                for (int m = 0; m < 4; ++m)
#pragma unroll
                    for (int n = 0; n < 2; ++n) acc[a][b][m][n] = (f32x4){0.f, 0.f, 0.f, 0.f};
        bf16x8 At[4][2], B0[2][2], B1[2][2];
        STAMP(ws, r * 8 + 0);
        wait_word((unsigned*)(cbase + G_GIDX(0) * 128), target, tmo, call);
        wait_word((unsigned*)(cbase + G_GIDX(1) * 128), target, tmo, call);
        wait_word((unsigned*)(cbase + G_GIDX(2) * 128), target, tmo, call);
        STAMP(ws, r * 8 + 1);
        {
            const unsigned a0 = abase + G_GIDX(0) * 262144u, b0 = bbase + G_GIDX(0) * 262144u, a1 = abase + G_GIDX(1) * 262144u, b1 = bbase + G_GIDX(1) * 262144u;
            G_STAGE(G_SB(0, 0), b0); G_STAGE(G_SA(0, 0), a0); G_STAGE(G_SB(0, 1), b0 + 16384); G_STAGE(G_SA(0, 1), a0 + 16384);
            if (wr == 1) G_BAR;
            G_WAIT_V(4); G_BAR;
            G_STAGE(G_SB(1, 0), b1); G_STAGE(G_SA(1, 0), a1); G_STAGE(G_SB(1, 1), b1 + 16384);
            G_WAIT_V(6); G_BAR;
        }
        unsigned fA = target, fB;
        for (int t = 0; t < 64; t += 2) {
            const int t2 = t + 2 < 64 ? t + 2 : 63, t3 = t + 3 < 64 ? t + 3 : 63, t4 = t + 4 < 64 ? t + 4 : 63;
            const unsigned g1 = G_GIDX(t + 1), g2 = G_GIDX(t2), g3 = G_GIDX(t3), g4 = G_GIDX(t4);
            const unsigned a1 = abase + g1 * 262144u, a2 = abase + g2 * 262144u, b2 = bbase + g2 * 262144u, a3 = abase + g3 * 262144u, b3 = bbase + g3 * 262144u;
            asm volatile("global_load_dword %0, %1, %2 sc1" : "=v"(fB) : "v"(zero), "s"(cbase + g3 * 128));
            G_LDB(B0, 0, 0); G_SCHED; G_LDA(At, 0, 0); G_STAGE(G_SA(1, 1), a1 + 16384);
            G_WAIT_L(8); G_BAR; G_WAIT_L(0); G_MMA(0, 0, At, B0); G_BAR; G_SCHED;
            asm volatile("" : "+v"(fA));
            if ((int)(rfl(fA) - target) < 0) wait_word((unsigned*)(cbase + g2 * 128), target, tmo, call);
            G_LDB(B1, 0, 1); G_STAGE(G_SB(0, 0), b2);
            G_BAR; G_WAIT_L(0); G_MMA(0, 1, At, B1); G_BAR;
            G_LDA(At, 0, 1); G_STAGE(G_SA(0, 0), a2);
            G_BAR; G_WAIT_L(0); G_MMA(1, 0, At, B0); G_BAR; G_SCHED;
            G_STAGE(G_SB(0, 1), b2 + 16384);
            G_WAIT_V(6); G_BAR; G_MMA(1, 1, At, B1); G_BAR;
            asm volatile("global_load_dword %0, %1, %2 sc1" : "=v"(fA) : "v"(zero), "s"(cbase + g4 * 128));
            G_LDB(B0, 1, 0); G_SCHED; G_LDA(At, 1, 0); G_STAGE(G_SA(0, 1), a2 + 16384);
            G_WAIT_L(8); G_BAR; G_WAIT_L(0); G_MMA(0, 0, At, B0); G_BAR; G_SCHED;
            asm volatile("" : "+v"(fB));
            if ((int)(rfl(fB) - target) < 0) wait_word((unsigned*)(cbase + g3 * 128), target, tmo, call);
            G_LDB(B1, 1, 1); G_STAGE(G_SB(1, 0), b3);
            G_BAR; G_WAIT_L(0); G_MMA(0, 1, At, B1); G_BAR;
            G_LDA(At, 1, 1); G_STAGE(G_SA(1, 0), a3);
            G_BAR; G_WAIT_L(0); G_MMA(1, 0, At, B0); G_BAR; G_SCHED;
            G_STAGE(G_SB(1, 1), b3 + 16384);
            G_WAIT_V(6); G_BAR; G_MMA(1, 1, At, B1); G_BAR;
        }
        G_WAIT_V(0);
        asm volatile("" : "+v"(fA));
        if (wr == 0) G_BAR;
        G_BAR;
        STAMP(ws, r * 8 + 2);
        if (h) {
#pragma unroll
            for (int b = 0; b < 2; ++b)
#pragma unroll
                for (int m = 0; m < 4; ++m)
#pragma unroll
                    for (int n = 0; n < 2; ++n) { const f32x4 tq = acc[0][b][m][n]; acc[0][b][m][n] = acc[1][b][m][n]; acc[1][b][m][n] = tq; }
        }
        const unsigned xl = (unsigned)((r * 64 + tile) * 2);
#pragma unroll
        for (int b = 0; b < 2; ++b)
#pragma unroll
            for (int m = 0; m < 4; ++m)
#pragma unroll
                for (int n = 0; n < 2; ++n)
                    __builtin_amdgcn_raw_buffer_store_b128(__builtin_bit_cast(u32x4, acc[1][b][m][n]), wsr, lane * 16 + ((b * 4 + m) * 2 + n) * 1024, OFF_XBUF + ((xl + h) * 8 + wid) * 16384u, AUX_SC1);
        WAIT_VM0();
        if (lane == 0) __hip_atomic_store((unsigned*)(ws + OFF_XFLAG + (xl + h) * 128 + wid * 16), call, __ATOMIC_RELAXED, __HIP_MEMORY_SCOPE_AGENT);
        wait_word((unsigned*)(ws + OFF_XFLAG + (xl + (h ^ 1)) * 128 + wid * 16), call, tmo, call);
        const bool dead = rfl(__hip_atomic_load(tmo, __ATOMIC_RELAXED, __HIP_MEMORY_SCOPE_AGENT)) == call;
        STAMP(ws, r * 8 + 3);
        LAS unsigned char* my = lds + wid * 8192;
#pragma unroll
        for (int b = 0; b < 2; ++b)
#pragma unroll
            for (int m = 0; m < 4; ++m) {
                const f32x4 p0 = __builtin_bit_cast(f32x4, __builtin_amdgcn_raw_buffer_load_b128(wsr, lane * 16 + ((b * 4 + m) * 2 + 0) * 1024, OFF_XBUF + ((xl + (h ^ 1)) * 8 + wid) * 16384u, AUX_SC1));
                const f32x4 p1 = __builtin_bit_cast(f32x4, __builtin_amdgcn_raw_buffer_load_b128(wsr, lane * 16 + ((b * 4 + m) * 2 + 1) * 1024, OFF_XBUF + ((xl + (h ^ 1)) * 8 + wid) * 16384u, AUX_SC1));
                const f32x4 v0 = acc[0][b][m][0] + p0, v1 = acc[0][b][m][1] + p1;
                u32x4 o;
                o.x = cvt_pk_bf16(gelu_f(v0[0]), gelu_f(v0[1])); o.y = cvt_pk_bf16(gelu_f(v0[2]), gelu_f(v0[3]));
                o.z = cvt_pk_bf16(gelu_f(v1[0]), gelu_f(v1[1])); o.w = cvt_pk_bf16(gelu_f(v1[2]), gelu_f(v1[3]));
                const int row = m * 16 + fr, ch = b * 4 + fq;
                *(LAS u32x4*)(my + row * 128 + ((ch ^ (row & 7)) * 16)) = o;
            }
        asm volatile("s_waitcnt lgkmcnt(0)" ::: "memory");
        u32x4 c[8];
#pragma unroll
        for (int j = 0; j < 8; ++j) { const int row = j * 8 + (lane >> 3); c[j] = *(LAS u32x4*)(my + row * 128 + (((lane & 7) ^ (row & 7)) * 16)); }
        if (dead) {
#pragma unroll
            for (int j = 0; j < 8; ++j) c[j] = (u32x4){0x7FC07FC0u, 0x7FC07FC0u, 0x7FC07FC0u, 0x7FC07FC0u};
        }
        {
            const rsrc_t outr = mk_rsrc(P.out, (unsigned)DIM * DIM * 2u);
            const unsigned row0 = (P.rank >> 2) * 4096 + r * 2048 + tm * 256 + h * 128 + wr * 64, col0 = (P.rank & 3) * 2048 + tn * 256 + wc * 64;
            const unsigned dso = (row0 * DIM + col0) * 2u, vo = (unsigned)(lane >> 3) * (DIM * 2u) + (unsigned)(lane & 7) * 16u;
#pragma unroll
            for (int j = 0; j < 8; ++j) __builtin_amdgcn_raw_buffer_store_b128(c[j], outr, vo, dso + j * (8u * DIM * 2u), AUX_SC1);
            const unsigned o = (xl + h) * 8 + wid;
            publish_block(P, wsr, c, 4096u + o, 1536u + o, 0xFFu & ~(1u << P.rank), call, lane);
        }
        STAMP(ws, r * 8 + 4);
        asm volatile("s_waitcnt lgkmcnt(0)" ::: "memory");
        G_BAR;
    }
#undef G_SA
#undef G_SB
#undef G_STAGE
#undef G_LDA
#undef G_LDB
#undef G_MMA
#undef G_WAIT_V
#undef G_WAIT_L
#undef G_BAR
#undef G_SCHED
#undef G_GIDX
}

__global__ __launch_bounds__(512, 2) void gemm_allgather_kernel(Params P) {
    extern __shared__ __attribute__((aligned(16))) unsigned char shm[];
    LAS unsigned char* lds = (LAS unsigned char*)shm;
    char* ws = P.ws[P.rank];
    if (threadIdx.x == 0) {
        const unsigned t = __hip_atomic_fetch_add((unsigned*)(ws + OFF_TICKET), 1u, __ATOMIC_RELAXED, __HIP_MEMORY_SCOPE_AGENT);
        *(LAS unsigned*)(lds + 131072) = t / GRID + 1;
    }
    __syncthreads();
    const unsigned call = rfl(*(LAS unsigned*)(lds + 131072));
    const int b = blockIdx.x;
    STAMP(ws, 60);
    if (b < NGEMM) gemm_role(P, lds, call, b);
    else if (b < NGEMM + NPULL) puller_role(P, lds, call, b - NGEMM);
    else packer_role(P, lds, call, b - NGEMM - NPULL);
    STAMP(ws, 61);
}

}

extern "C" void kernel_launch(int rank, int nranks, ncclComm_t comm,
                              hipStream_t stream,
                              void* const* d_in, const int* in_sizes, int n_in,
                              void* d_out, int out_size,
                              const int* in_dtypes, int out_dtype,
                              void* const* d_out_all,
                              void* const* d_ws_all, size_t ws_bytes) {
    (void)comm; (void)in_sizes; (void)in_dtypes; (void)out_dtype; (void)d_out_all;
    if (nranks != gar::WORLD || n_in < 2 || ws_bytes < gar::WS_NEED || out_size != gar::DIM * gar::DIM || (unsigned)rank >= (unsigned)gar::WORLD) {
        fprintf(stderr, "unexpected problem shape\n");
        return;
    }
    static bool attr = false;
    if (!attr) {
        hipError_t e = hipFuncSetAttribute((const void*)gar::gemm_allgather_kernel, hipFuncAttributeMaxDynamicSharedMemorySize, gar::LDS_BYTES);
        if (e != hipSuccess) { fprintf(stderr, "hipFuncSetAttribute: %s\n", hipGetErrorString(e)); return; }
        attr = true;
    }
    gar::Params P;
    memset(&P, 0, sizeof P);
    P.x = d_in[0]; P.w = d_in[1]; P.out = d_out; P.rank = rank;
    for (int p = 0; p < gar::WORLD; ++p) P.ws[p] = (char*)d_ws_all[p];
    hipLaunchKernelGGL(gar::gemm_allgather_kernel, dim3(gar::GRID), dim3(512), gar::LDS_BYTES, stream, P);
    hipError_t e = hipGetLastError();
    if (e != hipSuccess) fprintf(stderr, "launch: %s\n", hipGetErrorString(e));
}
```

```cpp
#include <hip/hip_runtime.h>
#include <rccl/rccl.h>
#include <cstdio>
#include <cstdint>
#include <cstring>

namespace gar {

#define LAS __attribute__((address_space(3)))
typedef unsigned u32x4 __attribute__((ext_vector_type(4)));
typedef unsigned short u16x2 __attribute__((ext_vector_type(2)));
typedef short bf16x8 __attribute__((ext_vector_type(8)));
typedef float f32x4 __attribute__((ext_vector_type(4)));
typedef __amdgpu_buffer_rsrc_t rsrc_t;

constexpr int WORLD = 8;
constexpr int DIM = 8192;
constexpr int KSH = 1024;
constexpr int NGEMM = 128;
constexpr int GRID = 256;
#define PULL_WG_PER_LINK 12
#define PULL_DECODERS 4
constexpr int PWL = PULL_WG_PER_LINK;
constexpr int NDEC = PULL_DECODERS;
constexpr int PWAVES = 1;
constexpr int NPULL = 7 * PWL;
constexpr int NPACK = GRID - NGEMM - NPULL;
constexpr int NSEQ = 3584;
constexpr unsigned long long DEADLINE = 40000000ull;

constexpr unsigned OFF_TICKET = 0, OFF_TMO = 128;
constexpr unsigned OFF_LINK = 1024;
constexpr unsigned OFF_CNT = 4096;
constexpr unsigned OFF_XFLAG = OFF_CNT + 256 * 128;
constexpr unsigned OFF_GRAN = 131072;
constexpr unsigned OFF_SLOT = 1u << 20;
constexpr unsigned OFF_AIMG = 64u << 20;
constexpr unsigned OFF_BIMG = 128u << 20;
constexpr unsigned OFF_XBUF = 160u << 20;
constexpr unsigned WS_USED = 192u << 20;
constexpr unsigned WS_NEED = WS_USED;
static_assert(OFF_XFLAG + 256 * 128 <= OFF_GRAN, "map");
static_assert(OFF_GRAN + WORLD * NSEQ * 16 <= OFF_SLOT, "map");
static_assert(OFF_SLOT + 6144u * 8192u <= OFF_AIMG, "map");
constexpr int LDS_BYTES = 131072 + 256;

struct Params {
    const void* x; const void* w; void* out;
    char* ws[WORLD];
    int rank; int pad;
};

constexpr int AUX_SC1 = 16, AUX_SYS = 17;

__device__ __forceinline__ rsrc_t mk_rsrc(const void* p, unsigned bytes) {
    return __builtin_amdgcn_make_buffer_rsrc(const_cast<void*>(p), 0, bytes, 0x00020000);
}
__device__ __forceinline__ unsigned rfl(unsigned v) { return __builtin_amdgcn_readfirstlane(v); }
__device__ __forceinline__ unsigned bfi(unsigned mask, unsigned a, unsigned b) { return (a & mask) | (b & ~mask); }
__device__ __forceinline__ unsigned perm(unsigned hi, unsigned lo, unsigned sel) { return __builtin_amdgcn_perm(hi, lo, sel); }
#define WAIT_VM0() asm volatile("s_waitcnt vmcnt(0)" ::: "memory")

#define STAMP(ws, idx) do { } while (0)

__device__ __forceinline__ bool give_up(unsigned long long t0, unsigned* tmo, unsigned call) {
    if (rfl(__hip_atomic_load(tmo, __ATOMIC_RELAXED, __HIP_MEMORY_SCOPE_AGENT)) == call) return true;
    if (__builtin_amdgcn_s_memrealtime() - t0 > DEADLINE) { __hip_atomic_store(tmo, call, __ATOMIC_RELAXED, __HIP_MEMORY_SCOPE_AGENT); return true; }
    return false;
}
__device__ __forceinline__ void wait_word(unsigned* p, unsigned target, unsigned* tmo, unsigned call) {
    if ((int)(rfl(__hip_atomic_load(p, __ATOMIC_RELAXED, __HIP_MEMORY_SCOPE_AGENT)) - target) >= 0) return;
    const unsigned long long t0 = __builtin_amdgcn_s_memrealtime();
    while ((int)(rfl(__hip_atomic_load(p, __ATOMIC_RELAXED, __HIP_MEMORY_SCOPE_AGENT)) - target) < 0) {
        if (give_up(t0, tmo, call)) break;
        __builtin_amdgcn_s_sleep(4);
    }
}

struct Hdr { unsigned h0, h1, h2; };
constexpr unsigned AUX_LEN = 32, AUX_STR = 96, AUX_MAX = 2048, ENC_HIST = AUX_MAX + 16;
constexpr unsigned LANE_BITS_MAX = 256;

__device__ __forceinline__ unsigned bytemask(unsigned x, int sh) { return perm(0u, 0u, ((x >> sh) & 0x01010101u) | 0x0C0C0C0Cu); }
template <int CTRL, int RM = 0xf> __device__ __forceinline__ unsigned dpp0(unsigned v) { return (unsigned)__builtin_amdgcn_update_dpp(0, (int)v, CTRL, RM, 0xf, RM == 0xf); }
__device__ __forceinline__ unsigned wave_scan_incl(unsigned v) {
    v += dpp0<0x111>(v); v += dpp0<0x112>(v); v += dpp0<0x114>(v); v += dpp0<0x118>(v);
    v += dpp0<0x142, 0xa>(v); v += dpp0<0x143, 0xc>(v);
    return v;
}
__device__ __forceinline__ unsigned lut2x16(const unsigned (&T)[8], unsigned idx, unsigned sg) {
    const unsigned sel = idx & 0x07070707u, m8 = bytemask(idx, 3), ms = bytemask(sg, 7);
    const unsigned p = bfi(m8, perm(T[3], T[2], sel), perm(T[1], T[0], sel)), n = bfi(m8, perm(T[7], T[6], sel), perm(T[5], T[4], sel));
    return bfi(ms, n, p);
}
#define LDS_FENCE() asm volatile("" ::: "memory")

__device__ __forceinline__ Hdr encode_block(const u32x4 (&c)[8], u32x4 (&R4)[4], u32x4& a0, u32x4& a1, LAS unsigned char* scr, int lane) {
    unsigned E[16], R[16];
    u16x2 m1 = {0, 0}, m2 = {0, 0};
#pragma unroll
    for (int g = 0; g < 16; ++g) {
        const unsigned a = (g & 1) ? c[g >> 1].z : c[g >> 1].x, b = (g & 1) ? c[g >> 1].w : c[g >> 1].y;
        const unsigned H = perm(b, a, 0x07050301u), L = perm(b, a, 0x06040200u);
        E[g] = bfi(0xFEFEFEFEu, H << 1, L >> 7);
        R[g] = bfi(0x80808080u, H, L);
        m1 = __builtin_elementwise_max(m1, __builtin_bit_cast(u16x2, E[g]));
        m2 = __builtin_elementwise_max(m2, __builtin_bit_cast(u16x2, E[g] << 8));
    }
#pragma unroll
    for (int p = 0; p < 4; ++p) R4[p] = (u32x4){R[4 * p], R[4 * p + 1], R[4 * p + 2], R[4 * p + 3]};
    const u32x4 z4 = {0u, 0u, 0u, 0u};
    *(LAS u32x4*)(scr + lane * 16) = z4; *(LAS u32x4*)(scr + 1024 + lane * 16) = z4;
    if (lane < 9) *(LAS u32x4*)(scr + AUX_MAX + lane * 16) = z4;
    const u16x2 mm = __builtin_elementwise_max(m1, m2);
    unsigned mx = (mm.x > mm.y ? mm.x : mm.y) >> 8;
#pragma unroll
    for (int o = 32; o >= 1; o >>= 1) { const unsigned y = __shfl_xor(mx, o); mx = mx > y ? mx : y; }
    const unsigned base = rfl(mx), B = base * 0x01010101u;
    unsigned any = 0;
#pragma unroll
    for (int g = 0; g < 16; ++g) { E[g] = B - E[g]; any |= E[g]; }
    LDS_FENCE();
#pragma unroll
    for (int gi = 0; gi < 4; ++gi) {
        const int g = gi * 5;
        const unsigned k4 = ((E[g] & 0x0F0F0F0Fu) | ((R[g] >> 3) & 0x10101010u)) << 2;
#pragma unroll
        for (int b = 0; b < 4; ++b)
            __hip_atomic_fetch_add((LAS unsigned*)(scr + ENC_HIST + ((k4 >> (8 * b)) & 0x7Cu)), 1u, __ATOMIC_RELAXED, __HIP_MEMORY_SCOPE_WORKGROUP);
    }
    LDS_FENCE();
    const unsigned key = (*(LAS unsigned*)(scr + ENC_HIST + (lane & 31) * 4) << 4) | (15u - (lane & 15));
    unsigned rank = 0;
    rank += dpp0<0x121>(key) > key; rank += dpp0<0x122>(key) > key; rank += dpp0<0x123>(key) > key; rank += dpp0<0x124>(key) > key; rank += dpp0<0x125>(key) > key;
    rank += dpp0<0x126>(key) > key; rank += dpp0<0x127>(key) > key; rank += dpp0<0x128>(key) > key; rank += dpp0<0x129>(key) > key; rank += dpp0<0x12A>(key) > key;
    rank += dpp0<0x12B>(key) > key; rank += dpp0<0x12C>(key) > key; rank += dpp0<0x12D>(key) > key; rank += dpp0<0x12E>(key) > key; rank += dpp0<0x12F>(key) > key;
    unsigned fv = (rank + 1u) << (8 * (lane & 3));
    fv |= dpp0<0xB1>(fv); fv |= dpp0<0x4E>(fv);
    unsigned F[8];
#pragma unroll
    for (int q = 0; q < 8; ++q) F[q] = __builtin_amdgcn_readlane(fv, q * 4);
    if (lane < 32) *(LAS unsigned char*)(scr + (lane & 16) + rank) = (unsigned char)(base - (lane & 15));
    unsigned W[16], O[16], T = 0, bad = 0;
#pragma unroll
    for (int g = 0; g < 16; ++g) {
        unsigned ln = lut2x16(F, E[g] & 0x0F0F0F0Fu, R[g]);
        const bool over = __builtin_amdgcn_sad_u8(ln, 0u, 0u) > 31u;
        if (__ballot(over) != 0ull) {
            any |= 0x80u;
            const unsigned a = (ln & 0xFFu) > ((ln >> 8) & 0xFFu) ? (ln & 0xFFu) : ((ln >> 8) & 0xFFu), b = ((ln >> 16) & 0xFFu) > (ln >> 24) ? ((ln >> 16) & 0xFFu) : (ln >> 24);
            const unsigned eq = over ? (((ln ^ ((a > b ? a : b) * 0x01010101u)) + 0x7F7F7F7Fu) & 0x80808080u) ^ 0x80808080u : 0u;
            E[g] |= eq;
            ln = bfi(bytemask(eq, 7), 0x01010101u, ln);
        }
        unsigned pos = (ln & 0xFFu) - 1u, w = 1u << (pos & 31u);
        pos += (ln >> 8) & 0xFFu; w |= 1u << (pos & 31u);
        pos += (ln >> 16) & 0xFFu; w |= 1u << (pos & 31u);
        pos += ln >> 24; w |= 1u << (pos & 31u);
        W[g] = w; O[g] = T; T += pos + 1u; bad = bad > pos ? bad : pos;
    }
    bad = (bad > 30u) | (T > LANE_BITS_MAX);
    const unsigned incl = wave_scan_incl(T), total = __builtin_amdgcn_readlane(incl, 63), p0 = AUX_STR * 8u + incl - T;
    *(LAS unsigned char*)(scr + AUX_LEN + lane) = (unsigned char)(T - 64u);
    unsigned raw = (__ballot(bad != 0) != 0ull) || total > (AUX_MAX - AUX_STR) * 8u - 32u;
    if (!raw) {
#pragma unroll
        for (int g = 0; g < 16; ++g) {
            const unsigned p = p0 + O[g], s = p & 31u;
            LAS unsigned* q = (LAS unsigned*)(scr + ((p >> 5) << 2));
            __hip_atomic_fetch_or(q, W[g] << s, __ATOMIC_RELAXED, __HIP_MEMORY_SCOPE_WORKGROUP);
            __hip_atomic_fetch_or(q + 1, (W[g] >> 1) >> (31u - s), __ATOMIC_RELAXED, __HIP_MEMORY_SCOPE_WORKGROUP);
        }
    }
    unsigned r0 = 0, r1 = 0, r2 = 0, n = 0;
    if (__ballot((any & 0xF0F0F0F0u) != 0) != 0ull) {
#pragma unroll
        for (int g = 0; g < 16; ++g) {
            const unsigned f = E[g] & 0xF0F0F0F0u;
            unsigned long long mask = __ballot(f != 0);
            while (mask != 0ull && n <= 3) {
                const int L = __builtin_ctzll(mask); mask &= mask - 1;
                const unsigned fl = __builtin_amdgcn_readlane(f, L);
                const unsigned lo = __builtin_amdgcn_readlane((g & 1) ? c[g >> 1].z : c[g >> 1].x, L);
                const unsigned hi = __builtin_amdgcn_readlane((g & 1) ? c[g >> 1].w : c[g >> 1].y, L);
#pragma unroll
                for (int b = 0; b < 4; ++b) {
                    const bool e = ((fl >> (8 * b)) & 0xFFu) != 0;
                    const unsigned val = (((b & 2) ? hi : lo) >> (16 * (b & 1))) & 0xFFFFu;
                    const unsigned rec = (unsigned)(L * 64 + g * 4 + b) | (val << 12);
                    r0 = (e && n == 0) ? rec : r0; r1 = (e && n == 1) ? rec : r1; r2 = (e && n == 2) ? rec : r2;
                    n += e;
                }
            }
        }
    }
    raw |= n > 3;
    LDS_FENCE();
    a0 = *(LAS u32x4*)(scr + lane * 16); a1 = *(LAS u32x4*)(scr + 1024 + lane * 16);
    const unsigned n16 = (AUX_STR + ((total + 7u) >> 3) + 15u) >> 4;
    Hdr h;
    h.h0 = r0 | ((n16 & 15u) << 28);
    h.h1 = r1 | ((n16 >> 4) << 28);
    h.h2 = r2 | ((n & 3u) << 28) | (raw << 31);
    return h;
}

__device__ __forceinline__ void decode_block(LAS const unsigned char* src, u32x4 (&c)[8], int lane) {
    LAS const unsigned char* aux = src + 4096;
    const u32x4 t0 = *(LAS const u32x4*)(aux), t1 = *(LAS const u32x4*)(aux + 16);
    const unsigned T = *(LAS const unsigned char*)(aux + AUX_LEN + lane) + 64u;
    u32x4 R4[4];
#pragma unroll
    for (int p = 0; p < 4; ++p) R4[p] = *(LAS const u32x4*)(src + p * 1024 + lane * 16);
    const unsigned p0 = AUX_STR * 8u + wave_scan_incl(T) - T, s = p0 & 31u;
    LAS const unsigned* q = (LAS const unsigned*)(aux + ((p0 >> 5) << 2));
    unsigned S[9];
#pragma unroll
    for (int i = 0; i < 9; ++i) S[i] = q[i];
#pragma unroll
    for (int i = 0; i < 8; ++i) S[i] = __builtin_amdgcn_alignbit(S[i + 1], S[i], s);
    const unsigned I[8] = {rfl(t0.x), rfl(t0.y), rfl(t0.z), rfl(t0.w), rfl(t1.x), rfl(t1.y), rfl(t1.z), rfl(t1.w)};
#pragma unroll
    for (int g = 0; g < 16; ++g) {
        const unsigned w0 = S[0], q0 = __builtin_ctz(w0), w1 = w0 & (w0 - 1u), q1 = __builtin_ctz(w1);
        const unsigned w2 = w1 & (w1 - 1u), q2 = __builtin_ctz(w2), w3 = w2 & (w2 - 1u), q3 = __builtin_ctz(w3);
        const unsigned Q = q0 | (q1 << 8) | (q2 << 16) | (q3 << 24);
        const unsigned rk = Q - (Q << 8) - 0x01010100u;
        if (g < 15) {
            const unsigned sh = q3 + 1u;
#pragma unroll
            for (int i = 0; i < 7; ++i) if (32 * i < 256 - 4 * (g + 1)) S[i] = (32 * (i + 1) < 256 - 4 * g) ? __builtin_amdgcn_alignbit(S[i + 1], S[i], sh) : S[i] >> sh;
            if (32 * 7 < 256 - 4 * (g + 1)) S[7] >>= sh;
        }
        const unsigned Rg = (g & 3) == 0 ? R4[g >> 2].x : (g & 3) == 1 ? R4[g >> 2].y : (g & 3) == 2 ? R4[g >> 2].z : R4[g >> 2].w;
        const unsigned E = lut2x16(I, rk, Rg);
        const unsigned H = bfi(0x80808080u, Rg, E >> 1), L = bfi(0x80808080u, E << 7, Rg);
        const unsigned x = perm(H, L, 0x05010400u), y = perm(H, L, 0x07030602u);
        if (g & 1) { c[g >> 1].z = x; c[g >> 1].w = y; } else { c[g >> 1].x = x; c[g >> 1].y = y; }
    }
}

__device__ __forceinline__ void store_granule(char* p, u32x4 g) {
    asm volatile("global_store_dwordx4 %0, %1, off sc0 sc1\n\ts_nop 1" :: "v"(p), "v"(g) : "memory");
}

__device__ __forceinline__ void publish_block(const Params& P, rsrc_t wsr, const u32x4 (&c)[8], unsigned slot, unsigned n, unsigned dmask, unsigned call, LAS unsigned char* scr, int lane) {
    u32x4 R4[4], a0, a1;
    const Hdr h = encode_block(c, R4, a0, a1, scr, lane);
    const unsigned so = OFF_SLOT + slot * 8192u;
    if (h.h2 >> 31) {
#pragma unroll
        for (int j = 0; j < 8; ++j) __builtin_amdgcn_raw_buffer_store_b128(c[j], wsr, lane * 16 + j * 1024, so, AUX_SYS);
    } else {
        const unsigned n16 = (h.h0 >> 28) | ((h.h1 >> 28) << 4);
#pragma unroll
        for (int j = 0; j < 4; ++j) __builtin_amdgcn_raw_buffer_store_b128(R4[j], wsr, lane * 16 + j * 1024, so, AUX_SYS);
        if ((unsigned)lane < n16) __builtin_amdgcn_raw_buffer_store_b128(a0, wsr, lane * 16 + 4096, so, AUX_SYS);
        if ((unsigned)lane + 64u < n16) __builtin_amdgcn_raw_buffer_store_b128(a1, wsr, lane * 16 + 5120, so, AUX_SYS);
    }
    WAIT_VM0();
    if (lane < WORLD && ((dmask >> lane) & 1u)) {
        char* p = P.ws[lane & 7] + OFF_GRAN + ((unsigned)P.rank * NSEQ + n) * 16u;
        store_granule(p, (u32x4){call, h.h0, h.h1, h.h2});
    }
}

__device__ __forceinline__ void packer_role(const Params& P, LAS unsigned char* lds, unsigned call, int wg) {
    const int lane = threadIdx.x & 63, wid = rfl(threadIdx.x >> 6);
    char* ws = P.ws[P.rank];
    const rsrc_t wsr = mk_rsrc(ws, WS_USED);
    const rsrc_t xr = mk_rsrc(P.x, DIM * KSH * 2), wr = mk_rsrc(P.w, DIM * KSH * 2);
    const int myi = P.rank >> 2, myj = P.rank & 3;
    LAS unsigned char* my = lds + wid * 8192;
    for (int p = wg * 8 + wid; p < 4096; p += NPACK * 8) {
        bool is_w; int kk, sel, r, bi;
        if (p < 3072) { kk = p / 192; const int rem = p - kk * 192; bi = rem & 31; r = 0; is_w = rem < 128; sel = is_w ? (rem >> 5) : ((rem - 128) >> 5); }
        else { const int q = p - 3072; kk = q >> 6; sel = (q >> 5) & 1; bi = q & 31; r = 1; is_w = false; }
        u32x4 c[8];
        if (!is_w) {
            const int rr = lane >> 3;
            const unsigned vo = (unsigned)rr * 2048u + (unsigned)(((lane & 7) ^ rr) * 16);
            const unsigned so = (unsigned)(sel * 4096 + r * 2048 + bi * 64) * 2048u + (unsigned)kk * 128u;
#pragma unroll
            for (int j = 0; j < 8; ++j) c[j] = __builtin_amdgcn_raw_buffer_load_b128(xr, vo + j * 8 * 2048, so, 0);
        } else {
            const int pn = bi >> 2, bj = (bi >> 1) & 1, q = bi & 1;
            const int gk = lane >> 3, cc = lane & 7, wcl = cc >> 2, cq = cc & 3;
            const unsigned vo = (unsigned)(gk * 8) * 16384u + (unsigned)((64 * (2 * q + wcl) + 32 * bj + 8 * cq) * 2);
            const unsigned so = (unsigned)(kk * 64) * 16384u + (unsigned)((sel * 2048 + pn * 256) * 2);
            u32x4 R[8];
#pragma unroll
            for (int jj = 0; jj < 8; ++jj) R[jj] = __builtin_amdgcn_raw_buffer_load_b128(wr, vo + jj * 16384, so, 0);
#pragma unroll
            for (int j = 0; j < 8; ++j) {
                const unsigned s = (j & 1) ? 0x07060302u : 0x05040100u;
                u32x4 t;
                t.x = perm(R[1][j >> 1], R[0][j >> 1], s); t.y = perm(R[3][j >> 1], R[2][j >> 1], s);
                t.z = perm(R[5][j >> 1], R[4][j >> 1], s); t.w = perm(R[7][j >> 1], R[6][j >> 1], s);
                const int s64 = 32 * wcl + 16 * (j >> 2) + 4 * cq + (j & 3);
                *(LAS u32x4*)(my + s64 * 128 + ((gk ^ (s64 & 7)) * 16)) = t;
            }
            asm volatile("s_waitcnt lgkmcnt(0)" ::: "memory");
#pragma unroll
            for (int j = 0; j < 8; ++j) c[j] = *(LAS u32x4*)(my + j * 1024 + lane * 16);
            asm volatile("s_waitcnt lgkmcnt(0)" ::: "memory");
        }
        const int g = P.rank * 16 + kk;
        const bool self = is_w ? (sel == myj) : (sel == myi);
        if (self) {
            const unsigned so = is_w ? OFF_BIMG + (unsigned)(g * 32 + bi) * 8192u : OFF_AIMG + (unsigned)((r * 128 + g) * 32 + bi) * 8192u;
#pragma unroll
            for (int j = 0; j < 8; ++j) __builtin_amdgcn_raw_buffer_store_b128(c[j], wsr, lane * 16 + j * 1024, so, AUX_SC1);
        }
        const unsigned slot = is_w ? 2048u + (unsigned)((sel * 16 + kk) * 32 + bi) : (unsigned)(((sel * 2 + r) * 16 + kk) * 32 + bi);
        const unsigned n = is_w ? (unsigned)(kk * 64 + bi) : (r == 0 ? (unsigned)(kk * 64 + 32 + bi) : (unsigned)(1024 + kk * 32 + bi));
        unsigned dmask = is_w ? (0x11u << sel) : (0x0Fu << (4 * sel));
        dmask &= ~(1u << P.rank);
        publish_block(P, wsr, c, slot, n, dmask, call, my, lane);
        if (self && lane == 0)
            __hip_atomic_fetch_add((unsigned*)(ws + OFF_CNT + (r * 128 + g) * 128), 1u, __ATOMIC_RELAXED, __HIP_MEMORY_SCOPE_AGENT);
    }
}

__device__ __forceinline__ u32x4 load_granule(rsrc_t wsr, unsigned off) { return __builtin_amdgcn_raw_buffer_load_b128(wsr, 0, off, AUX_SYS); }

struct Blk { unsigned slot, dso, cnt; bool is_out; };
__device__ __forceinline__ Blk locate(unsigned n, int src, int myi, int myj) {
    Blk b; b.is_out = false;
    if (n < 1024) { const int kk = n >> 6, bi = n & 31, g = src * 16 + kk; b.cnt = OFF_CNT + g * 128;
        if ((n & 63) < 32) { b.slot = 2048u + (myj * 16 + kk) * 32 + bi; b.dso = OFF_BIMG + (unsigned)(g * 32 + bi) * 8192u; }
        else { b.slot = ((myi * 2) * 16 + kk) * 32 + bi; b.dso = OFF_AIMG + (unsigned)(g * 32 + bi) * 8192u; } }
    else if (n < 1536) { const int kk = (n - 1024) >> 5, bi = n & 31, g = src * 16 + kk; b.cnt = OFF_CNT + (128 + g) * 128;
        b.slot = ((myi * 2 + 1) * 16 + kk) * 32 + bi; b.dso = OFF_AIMG + (unsigned)((128 + g) * 32 + bi) * 8192u; }
    else { const int o = n - 1536, w = o & 7, h = (o >> 3) & 1, tile = (o >> 4) & 63, r = o >> 10; b.cnt = 0; b.is_out = true;
        b.slot = 4096u + o;
        const unsigned row0 = (src >> 2) * 4096 + r * 2048 + (tile >> 3) * 256 + h * 128 + (w >> 2) * 64, col0 = (src & 3) * 2048 + (tile & 7) * 256 + (w & 3) * 64;
        b.dso = (row0 * DIM + col0) * 2u; }
    return b;
}
__device__ __forceinline__ void signal_cnt(char* ws, unsigned off, int lane) {
    if (off && lane == 0) __hip_atomic_fetch_add((unsigned*)(ws + off), 1u, __ATOMIC_RELAXED, __HIP_MEMORY_SCOPE_AGENT);
}
constexpr unsigned NSLOT = 15, CTL = NSLOT * 8192;
constexpr unsigned M_DEAD = 1u << 30, N_STOP = 0xFFFFFFFFu;
__device__ __forceinline__ unsigned lds_ld(LAS unsigned char* p) { return *(volatile LAS unsigned*)p; }
__device__ __forceinline__ void lds_st(LAS unsigned char* p, unsigned v) { *(volatile LAS unsigned*)p = v; }
__device__ __forceinline__ void ring_publish(LAS unsigned char* lds, unsigned i, unsigned n, unsigned h0, unsigned h1, unsigned h2, int lane) {
    const unsigned sl = i % NSLOT;
    if (lane == 0) { *(volatile LAS u32x4*)(lds + CTL + sl * 16) = (u32x4){n, h0, h1, h2}; lds_st(lds + CTL + 256 + sl * 4, i + 1u); }
}
__device__ __forceinline__ void dma_op(rsrc_t r, LAS unsigned char* dst, int j, unsigned so, int lane) {
    __builtin_amdgcn_raw_ptr_buffer_load_lds(r, (LAS void*)(dst + j * 1024), 16, lane * 16 + j * 1024, so, 0, AUX_SYS);
}
#define VM_WAIT_(n) asm volatile("s_waitcnt vmcnt(" #n ")" ::: "memory")
#define VM_WAIT(n) VM_WAIT_(n)
#define WIN 5

__device__ __forceinline__ void loader_wave(const Params& P, LAS unsigned char* lds, unsigned call, int src, int lane) {
    char* ws = P.ws[P.rank];
    unsigned* tmo = (unsigned*)(ws + OFF_TMO);
    const rsrc_t wsr = mk_rsrc(ws, WS_USED);
    char* const remp = P.ws[src];
    const int myi = P.rank >> 2, myj = P.rank & 3;
    const unsigned gbase = OFF_GRAN + (unsigned)src * NSEQ * 16u;
    char* const gptr = ws + gbase;
    const unsigned tko = OFF_LINK + (unsigned)src * 128u, tbase = (call - 1u) * (NSEQ + 3u * PWL);
    __builtin_amdgcn_s_setprio(3);
    unsigned n, n1, n2;
    {
        unsigned t0 = 0;
        if (lane == 0) t0 = __hip_atomic_fetch_add((unsigned*)(ws + tko), 3u, __ATOMIC_RELAXED, __HIP_MEMORY_SCOPE_AGENT);
        n = rfl(t0) - tbase; n1 = n + 1; n2 = n + 2;
    }
    unsigned gx, gy, gz, gw, hx, hy, hz, hw;
    { const u32x4 a = load_granule(wsr, gbase + (n < NSEQ ? n : 0u) * 16u), b = load_granule(wsr, gbase + (n1 < NSEQ ? n1 : 0u) * 16u);
      gx = rfl(a.x); gy = rfl(a.y); gz = rfl(a.z); gw = rfl(a.w); hx = rfl(b.x); hy = rfl(b.y); hz = rfl(b.z); hw = rfl(b.w); }
    const int zero = 0, one = 1;
    unsigned i = 0;
    bool pend = false; unsigned pn = 0, p0 = 0, p1 = 0, p2 = 0;
    while (n < NSEQ) {
        const unsigned sl = i % NSLOT;
        bool dead = false;
        const bool need_gran = (int)(gx - call) < 0;
        const bool need_slot = i >= NSLOT && (int)(lds_ld(lds + CTL + 320 + sl * 4) - (i - NSLOT + 1u)) < 0;
        if (need_gran || need_slot) {
            if (pend) { VM_WAIT(0); ring_publish(lds, i - 1u, pn, p0, p1, p2, lane); pend = false; }
            const unsigned long long t0 = __builtin_amdgcn_s_memrealtime();
            while ((int)(gx - call) < 0) {
                const u32x4 a = load_granule(wsr, gbase + n * 16u);
                gx = rfl(a.x); gy = rfl(a.y); gz = rfl(a.z); gw = rfl(a.w);
                if ((int)(gx - call) >= 0) break;
                if (give_up(t0, tmo, call)) { dead = true; break; }
                __builtin_amdgcn_s_sleep(2);
                asm volatile("" ::: "memory");
            }
            while (i >= NSLOT && (int)(lds_ld(lds + CTL + 320 + sl * 4) - (i - NSLOT + 1u)) < 0) __builtin_amdgcn_s_sleep(1);
        }
        const bool raw = (gw >> 31) != 0;
        const unsigned so = OFF_SLOT + locate(n, src, myi, myj).slot * 8192u;
        const unsigned bytes = dead ? 0u : raw ? 8192u : 4096u + (((gy >> 28) | ((gz >> 28) << 4)) << 4);
        const rsrc_t rem = mk_rsrc(remp, so + bytes);
        LAS unsigned char* dst = lds + sl * 8192;
        u32x4 g2; unsigned tk = 0;
        asm volatile("global_load_dwordx4 %0, %1, %2 sc0 sc1" : "=v"(g2) : "v"(zero), "s"(gptr + (n2 < NSEQ ? n2 : 0u) * 16u) : "memory");
        if (lane == 0) asm volatile("global_atomic_add %0, %1, %2, %3 sc0" : "=v"(tk) : "v"(zero), "v"(one), "s"(ws + tko) : "memory");
        VM_WAIT(WIN); dma_op(rem, dst, 0, so, lane);
        VM_WAIT(WIN); dma_op(rem, dst, 1, so, lane);
        VM_WAIT(WIN); dma_op(rem, dst, 2, so, lane);
        VM_WAIT(WIN); if (pend) ring_publish(lds, i - 1u, pn, p0, p1, p2, lane); dma_op(rem, dst, 3, so, lane);
        VM_WAIT(WIN); dma_op(rem, dst, 4, so, lane);
        asm volatile("s_waitcnt vmcnt(5)" : "+v"(g2), "+v"(tk) :: "memory"); dma_op(rem, dst, 5, so, lane);
        if (raw) { dma_op(rem, dst, 6, so, lane); dma_op(rem, dst, 7, so, lane); VM_WAIT(0); }
        pend = true; pn = n; p0 = gy; p1 = gz; p2 = gw | (dead ? M_DEAD : 0u);
        n = n1; gx = hx; gy = hy; gz = hz; gw = hw;
        n1 = n2; hx = rfl(g2.x); hy = rfl(g2.y); hz = rfl(g2.z); hw = rfl(g2.w);
        n2 = rfl(tk) - tbase;
        ++i;
    }
    VM_WAIT(0);
    if (pend) ring_publish(lds, i - 1u, pn, p0, p1, p2, lane);
    for (int d = 0; d < NDEC; ++d, ++i) {
        const unsigned sl = i % NSLOT;
        while (i >= NSLOT && (int)(lds_ld(lds + CTL + 320 + sl * 4) - (i - NSLOT + 1u)) < 0) __builtin_amdgcn_s_sleep(1);
        ring_publish(lds, i, N_STOP, 0u, 0u, 0u, lane);
    }
}

__device__ __forceinline__ void decoder_wave(const Params& P, LAS unsigned char* lds, unsigned call, int src, int d, int lane) {
    char* ws = P.ws[P.rank];
    unsigned* tmo = (unsigned*)(ws + OFF_TMO);
    const rsrc_t wsr = mk_rsrc(ws, WS_USED), outr = mk_rsrc(P.out, (unsigned)DIM * DIM * 2u);
    const int myi = P.rank >> 2, myj = P.rank & 3;
    for (unsigned i = d;; i += NDEC) {
        const unsigned sl = i % NSLOT;
        while (lds_ld(lds + CTL + 256 + sl * 4) != i + 1u) __builtin_amdgcn_s_sleep(1);
        asm volatile("" ::: "memory");
        const u32x4 meta = *(volatile LAS u32x4*)(lds + CTL + sl * 16);
        const unsigned n = rfl(meta.x), h0 = rfl(meta.y), h1 = rfl(meta.z), h2 = rfl(meta.w);
        if (n == N_STOP) break;
        const Blk b = locate(n, src, myi, myj);
        const bool raw = (h2 >> 31) != 0;
        bool dead = (h2 & M_DEAD) != 0;
        if (b.is_out && rfl(__hip_atomic_load(tmo, __ATOMIC_RELAXED, __HIP_MEMORY_SCOPE_AGENT)) == call) dead = true;
        u32x4 c[8];
        LAS unsigned char* cur = lds + sl * 8192;
        if (dead) {
#pragma unroll
            for (int j = 0; j < 8; ++j) c[j] = (u32x4){0x7FC07FC0u, 0x7FC07FC0u, 0x7FC07FC0u, 0x7FC07FC0u};
        } else if (raw) {
#pragma unroll
            for (int j = 0; j < 8; ++j) c[j] = *(LAS u32x4*)(cur + j * 1024 + lane * 16);
        } else decode_block(cur, c, lane);
#pragma unroll
        for (int j = 0; j < 8; ++j) asm volatile("" : "+v"(c[j]));
        if (lane == 0) lds_st(lds + CTL + 320 + sl * 4, i + 1u);
        const rsrc_t dr = b.is_out ? outr : wsr;
        const unsigned sj = b.is_out ? 8u * DIM * 2u : 1024u, sr = b.is_out ? DIM * 2u : 128u;
        const unsigned vo = (unsigned)(lane >> 3) * sr + (unsigned)(lane & 7) * 16u;
#pragma unroll
        for (int j = 0; j < 8; ++j) __builtin_amdgcn_raw_buffer_store_b128(c[j], dr, vo, b.dso + j * sj, AUX_SC1);
        if (!dead && !raw) {
            const unsigned ne = (h2 >> 28) & 3u;
            for (unsigned e = 0; e < ne; ++e) {
                const unsigned rec = e == 0 ? h0 : e == 1 ? h1 : h2, pos = rec & 0xFFFu, v = pos & 63u;
                if ((unsigned)lane == (pos >> 6)) __builtin_amdgcn_raw_buffer_store_b16((unsigned short)((rec >> 12) & 0xFFFFu), dr, vo + (v & 7u) * 2u, b.dso + (v >> 3) * sj, AUX_SC1);
            }
        }
        if (b.cnt) { WAIT_VM0(); signal_cnt(ws, b.cnt, lane); }
    }
}

__device__ __forceinline__ void puller_role(const Params& P, LAS unsigned char* lds, unsigned call, int wg) {
    const int lane = threadIdx.x & 63, wid = rfl(threadIdx.x >> 6);
    const int link = wg % 7, src = (P.rank + 1 + link) & 7;
    if (threadIdx.x < 32) *(volatile LAS unsigned*)(lds + CTL + 256 + threadIdx.x * 4) = 0u;
    __syncthreads();
    if (wid == 0) loader_wave(P, lds, call, src, lane);
    else if (wid <= NDEC) decoder_wave(P, lds, call, src, wid - 1, lane);
}

__device__ __forceinline__ float gelu_f(float y) {
    const float c1 = -2.0f * 0.7978845608028654f * 1.4426950408889634f, c2 = c1 * 0.044715f;
    const float e = __builtin_amdgcn_exp2f(y * (c1 + c2 * y * y));
    return y * __builtin_amdgcn_rcpf(1.0f + e);
}
__device__ __forceinline__ unsigned cvt_pk_bf16(float lo, float hi) { unsigned r; asm("v_cvt_pk_bf16_f32 %0, %1, %2" : "=v"(r) : "v"(lo), "v"(hi)); return r; }

__device__ __forceinline__ void gemm_role(const Params& P, LAS unsigned char* lds, unsigned call, int wg) {
    const int tid = threadIdx.x, wid = rfl(tid >> 6), lane = tid & 63, wr = wid >> 2, wc = wid & 3, fr = lane & 15, fq = lane >> 4;
    char* ws = P.ws[P.rank];
    unsigned* tmo = (unsigned*)(ws + OFF_TMO);
    const rsrc_t wsr = mk_rsrc(ws, WS_USED);
    const int xcd = wg & 7, q = wg >> 3, h = xcd & 1, blk = xcd >> 1;
    const int tm = (blk >> 1) * 4 + (q >> 2), tn = (blk & 1) * 4 + (q & 3), tile = tm * 8 + tn;
    const unsigned voff = (unsigned)tid * 16u, ldsw = (unsigned)wid * 1024u;
    const int aoff0 = (wr * 64 + fr) * 128 + ((fq ^ (fr & 7)) << 4), aoff1 = aoff0 ^ 64;
    const int boff0 = (wc * 32 + fr) * 128 + ((fq ^ (fr & 7)) << 4), boff1 = boff0 ^ 64;
#define G_SA(b, hh) (((b) * 2 + (hh)) * 16384)
#define G_SB(b, hh) ((4 + (b) * 2 + (hh)) * 16384)
#define G_STAGE(bufoff, soff) do { _Pragma("unroll") for (int _i = 0; _i < 2; ++_i) \
        __builtin_amdgcn_raw_ptr_buffer_load_lds(wsr, (LAS void*)(lds + (bufoff) + ldsw + _i * 8192), 16, voff, (soff) + _i * 8192, 0, AUX_SC1); } while (0)
#define G_LDA(dst, b, hh) do { _Pragma("unroll") for (int m = 0; m < 4; ++m) { \
        dst[m][0] = *(const LAS bf16x8*)(lds + G_SA(b, hh) + aoff0 + m * 2048); dst[m][1] = *(const LAS bf16x8*)(lds + G_SA(b, hh) + aoff1 + m * 2048); } } while (0)
#define G_LDB(dst, b, hh) do { _Pragma("unroll") for (int n = 0; n < 2; ++n) { \
        dst[n][0] = *(const LAS bf16x8*)(lds + G_SB(b, hh) + boff0 + n * 2048); dst[n][1] = *(const LAS bf16x8*)(lds + G_SB(b, hh) + boff1 + n * 2048); } } while (0)
#define G_MMA(ai, bj, At, Bt) do { __builtin_amdgcn_s_setprio(1); _Pragma("unroll") for (int m = 0; m < 4; ++m) _Pragma("unroll") for (int n = 0; n < 2; ++n) _Pragma("unroll") for (int k = 0; k < 2; ++k) \
        acc[ai][bj][m][n] = __builtin_amdgcn_mfma_f32_16x16x32_bf16(Bt[n][k], At[m][k], acc[ai][bj][m][n], 0, 0, 0); __builtin_amdgcn_s_setprio(0); } while (0)
#define G_WAIT_V(n) asm volatile("s_waitcnt vmcnt(" #n ")" ::: "memory")
#define G_WAIT_L(n) asm volatile("s_waitcnt lgkmcnt(" #n ")" ::: "memory")
#define G_BAR __builtin_amdgcn_s_barrier()
#define G_SCHED __builtin_amdgcn_sched_barrier(0)
#define G_GIDX(t) ((unsigned)((4 * h + ((t) & 3)) * 16 + ((t) >> 2)))
    const int zero = 0;
    for (int r = 0; r < 2; ++r) {
        const unsigned target = call * (r == 0 ? 64u : 32u);
        const unsigned abase = OFF_AIMG + (unsigned)r * (128u * 262144u) + (unsigned)tm * 32768u, bbase = OFF_BIMG + (unsigned)tn * 32768u;
        char* cbase = ws + OFF_CNT + r * 128 * 128;
        f32x4 acc[2][2][4][2];
#pragma unroll
        for (int a = 0; a < 2; ++a)
#pragma unroll
            for (int b = 0; b < 2; ++b)
#pragma unroll
                for (int m = 0; m < 4; ++m)
#pragma unroll
                    for (int n = 0; n < 2; ++n) acc[a][b][m][n] = (f32x4){0.f, 0.f, 0.f, 0.f};
        bf16x8 At[4][2], B0[2][2], B1[2][2];
        STAMP(ws, r * 8 + 0);
        wait_word((unsigned*)(cbase + G_GIDX(0) * 128), target, tmo, call);
        wait_word((unsigned*)(cbase + G_GIDX(1) * 128), target, tmo, call);
        wait_word((unsigned*)(cbase + G_GIDX(2) * 128), target, tmo, call);
        STAMP(ws, r * 8 + 1);
        {
            const unsigned a0 = abase + G_GIDX(0) * 262144u, b0 = bbase + G_GIDX(0) * 262144u, a1 = abase + G_GIDX(1) * 262144u, b1 = bbase + G_GIDX(1) * 262144u;
            G_STAGE(G_SB(0, 0), b0); G_STAGE(G_SA(0, 0), a0); G_STAGE(G_SB(0, 1), b0 + 16384); G_STAGE(G_SA(0, 1), a0 + 16384);
            if (wr == 1) G_BAR;
            G_WAIT_V(4); G_BAR;
            G_STAGE(G_SB(1, 0), b1); G_STAGE(G_SA(1, 0), a1); G_STAGE(G_SB(1, 1), b1 + 16384);
            G_WAIT_V(6); G_BAR;
        }
        unsigned fA = target, fB;
        for (int t = 0; t < 64; t += 2) {
            const int t2 = t + 2 < 64 ? t + 2 : 63, t3 = t + 3 < 64 ? t + 3 : 63, t4 = t + 4 < 64 ? t + 4 : 63;
            const unsigned g1 = G_GIDX(t + 1), g2 = G_GIDX(t2), g3 = G_GIDX(t3), g4 = G_GIDX(t4);
            const unsigned a1 = abase + g1 * 262144u, a2 = abase + g2 * 262144u, b2 = bbase + g2 * 262144u, a3 = abase + g3 * 262144u, b3 = bbase + g3 * 262144u;
            asm volatile("global_load_dword %0, %1, %2 sc1" : "=v"(fB) : "v"(zero), "s"(cbase + g3 * 128));
            G_LDB(B0, 0, 0); G_SCHED; G_LDA(At, 0, 0); G_STAGE(G_SA(1, 1), a1 + 16384);
            G_WAIT_L(8); G_BAR; G_WAIT_L(0); G_MMA(0, 0, At, B0); G_BAR; G_SCHED;
            asm volatile("" : "+v"(fA));
            if ((int)(rfl(fA) - target) < 0) wait_word((unsigned*)(cbase + g2 * 128), target, tmo, call);
            G_LDB(B1, 0, 1); G_STAGE(G_SB(0, 0), b2);
            G_BAR; G_WAIT_L(0); G_MMA(0, 1, At, B1); G_BAR;
            G_LDA(At, 0, 1); G_STAGE(G_SA(0, 0), a2);
            G_BAR; G_WAIT_L(0); G_MMA(1, 0, At, B0); G_BAR; G_SCHED;
            G_STAGE(G_SB(0, 1), b2 + 16384);
            G_WAIT_V(6); G_BAR; G_MMA(1, 1, At, B1); G_BAR;
            asm volatile("global_load_dword %0, %1, %2 sc1" : "=v"(fA) : "v"(zero), "s"(cbase + g4 * 128));
            G_LDB(B0, 1, 0); G_SCHED; G_LDA(At, 1, 0); G_STAGE(G_SA(0, 1), a2 + 16384);
            G_WAIT_L(8); G_BAR; G_WAIT_L(0); G_MMA(0, 0, At, B0); G_BAR; G_SCHED;
            asm volatile("" : "+v"(fB));
            if ((int)(rfl(fB) - target) < 0) wait_word((unsigned*)(cbase + g3 * 128), target, tmo, call);
            G_LDB(B1, 1, 1); G_STAGE(G_SB(1, 0), b3);
            G_BAR; G_WAIT_L(0); G_MMA(0, 1, At, B1); G_BAR;
            G_LDA(At, 1, 1); G_STAGE(G_SA(1, 0), a3);
            G_BAR; G_WAIT_L(0); G_MMA(1, 0, At, B0); G_BAR; G_SCHED;
            G_STAGE(G_SB(1, 1), b3 + 16384);
            G_WAIT_V(6); G_BAR; G_MMA(1, 1, At, B1); G_BAR;
        }
        G_WAIT_V(0);
        asm volatile("" : "+v"(fA));
        if (wr == 0) G_BAR;
        G_BAR;
        STAMP(ws, r * 8 + 2);
        if (h) {
#pragma unroll
            for (int b = 0; b < 2; ++b)
#pragma unroll
                for (int m = 0; m < 4; ++m)
#pragma unroll
                    for (int n = 0; n < 2; ++n) { const f32x4 tq = acc[0][b][m][n]; acc[0][b][m][n] = acc[1][b][m][n]; acc[1][b][m][n] = tq; }
        }
        const unsigned xl = (unsigned)((r * 64 + tile) * 2);
#pragma unroll
        for (int b = 0; b < 2; ++b)
#pragma unroll
            for (int m = 0; m < 4; ++m)
#pragma unroll
                for (int n = 0; n < 2; ++n)
                    __builtin_amdgcn_raw_buffer_store_b128(__builtin_bit_cast(u32x4, acc[1][b][m][n]), wsr, lane * 16 + ((b * 4 + m) * 2 + n) * 1024, OFF_XBUF + ((xl + h) * 8 + wid) * 16384u, AUX_SC1);
        WAIT_VM0();
        if (lane == 0) __hip_atomic_store((unsigned*)(ws + OFF_XFLAG + (xl + h) * 128 + wid * 16), call, __ATOMIC_RELAXED, __HIP_MEMORY_SCOPE_AGENT);
        wait_word((unsigned*)(ws + OFF_XFLAG + (xl + (h ^ 1)) * 128 + wid * 16), call, tmo, call);
        const bool dead = rfl(__hip_atomic_load(tmo, __ATOMIC_RELAXED, __HIP_MEMORY_SCOPE_AGENT)) == call;
        STAMP(ws, r * 8 + 3);
        LAS unsigned char* my = lds + wid * 8192;
#pragma unroll
        for (int b = 0; b < 2; ++b)
#pragma unroll
            for (int m = 0; m < 4; ++m) {
                const f32x4 p0 = __builtin_bit_cast(f32x4, __builtin_amdgcn_raw_buffer_load_b128(wsr, lane * 16 + ((b * 4 + m) * 2 + 0) * 1024, OFF_XBUF + ((xl + (h ^ 1)) * 8 + wid) * 16384u, AUX_SC1));
                const f32x4 p1 = __builtin_bit_cast(f32x4, __builtin_amdgcn_raw_buffer_load_b128(wsr, lane * 16 + ((b * 4 + m) * 2 + 1) * 1024, OFF_XBUF + ((xl + (h ^ 1)) * 8 + wid) * 16384u, AUX_SC1));
                const f32x4 v0 = acc[0][b][m][0] + p0, v1 = acc[0][b][m][1] + p1;
                u32x4 o;
                o.x = cvt_pk_bf16(gelu_f(v0[0]), gelu_f(v0[1])); o.y = cvt_pk_bf16(gelu_f(v0[2]), gelu_f(v0[3]));
                o.z = cvt_pk_bf16(gelu_f(v1[0]), gelu_f(v1[1])); o.w = cvt_pk_bf16(gelu_f(v1[2]), gelu_f(v1[3]));
                const int row = m * 16 + fr, ch = b * 4 + fq;
                *(LAS u32x4*)(my + row * 128 + ((ch ^ (row & 7)) * 16)) = o;
            }
        asm volatile("s_waitcnt lgkmcnt(0)" ::: "memory");
        u32x4 c[8];
#pragma unroll
        for (int j = 0; j < 8; ++j) { const int row = j * 8 + (lane >> 3); c[j] = *(LAS u32x4*)(my + row * 128 + (((lane & 7) ^ (row & 7)) * 16)); }
        if (dead) {
#pragma unroll
            for (int j = 0; j < 8; ++j) c[j] = (u32x4){0x7FC07FC0u, 0x7FC07FC0u, 0x7FC07FC0u, 0x7FC07FC0u};
        }
        {
            const rsrc_t outr = mk_rsrc(P.out, (unsigned)DIM * DIM * 2u);
            const unsigned row0 = (P.rank >> 2) * 4096 + r * 2048 + tm * 256 + h * 128 + wr * 64, col0 = (P.rank & 3) * 2048 + tn * 256 + wc * 64;
            const unsigned dso = (row0 * DIM + col0) * 2u, vo = (unsigned)(lane >> 3) * (DIM * 2u) + (unsigned)(lane & 7) * 16u;
#pragma unroll
            for (int j = 0; j < 8; ++j) __builtin_amdgcn_raw_buffer_store_b128(c[j], outr, vo, dso + j * (8u * DIM * 2u), AUX_SC1);
            const unsigned o = (xl + h) * 8 + wid;
            publish_block(P, wsr, c, 4096u + o, 1536u + o, 0xFFu & ~(1u << P.rank), call, my, lane);
        }
        STAMP(ws, r * 8 + 4);
        asm volatile("s_waitcnt lgkmcnt(0)" ::: "memory");
        G_BAR;
    }
#undef G_SA
#undef G_SB
#undef G_STAGE
#undef G_LDA
#undef G_LDB
#undef G_MMA
#undef G_WAIT_V
#undef G_WAIT_L
#undef G_BAR
#undef G_SCHED
#undef G_GIDX
}

__global__ __launch_bounds__(512, 2) void gemm_allgather_kernel(Params P) {
    extern __shared__ __attribute__((aligned(16))) unsigned char shm[];
    LAS unsigned char* lds = (LAS unsigned char*)shm;
    char* ws = P.ws[P.rank];
    if (threadIdx.x == 0) {
        const unsigned t = __hip_atomic_fetch_add((unsigned*)(ws + OFF_TICKET), 1u, __ATOMIC_RELAXED, __HIP_MEMORY_SCOPE_AGENT);
        *(LAS unsigned*)(lds + 131072) = t / GRID + 1;
    }
    __syncthreads();
    const unsigned call = rfl(*(LAS unsigned*)(lds + 131072));
    const int b = blockIdx.x;
    STAMP(ws, 60);
    if (b < NGEMM) gemm_role(P, lds, call, b);
    else if (b < NGEMM + NPULL) puller_role(P, lds, call, b - NGEMM);
    else packer_role(P, lds, call, b - NGEMM - NPULL);
    STAMP(ws, 61);
}

}

extern "C" void kernel_launch(int rank, int nranks, ncclComm_t comm,
                              hipStream_t stream,
                              void* const* d_in, const int* in_sizes, int n_in,
                              void* d_out, int out_size,
                              const int* in_dtypes, int out_dtype,
                              void* const* d_out_all,
                              void* const* d_ws_all, size_t ws_bytes) {
    (void)comm; (void)in_sizes; (void)in_dtypes; (void)out_dtype; (void)d_out_all;
    if (nranks != gar::WORLD || n_in < 2 || ws_bytes < gar::WS_NEED || out_size != gar::DIM * gar::DIM || (unsigned)rank >= (unsigned)gar::WORLD) {
        fprintf(stderr, "unexpected problem shape\n");
        return;
    }
    static bool attr = false;
    if (!attr) {
        hipError_t e = hipFuncSetAttribute((const void*)gar::gemm_allgather_kernel, hipFuncAttributeMaxDynamicSharedMemorySize, gar::LDS_BYTES);
        if (e != hipSuccess) { fprintf(stderr, "hipFuncSetAttribute: %s\n", hipGetErrorString(e)); return; }
        attr = true;
    }
    gar::Params P;
    memset(&P, 0, sizeof P);
    P.x = d_in[0]; P.w = d_in[1]; P.out = d_out; P.rank = rank;
    for (int p = 0; p < gar::WORLD; ++p) P.ws[p] = (char*)d_ws_all[p];
    hipLaunchKernelGGL(gar::gemm_allgather_kernel, dim3(gar::GRID), dim3(512), gar::LDS_BYTES, stream, P);
    hipError_t e = hipGetLastError();
    if (e != hipSuccess) fprintf(stderr, "launch: %s\n", hipGetErrorString(e));
}
```

```cpp
#include <hip/hip_runtime.h>
#include <rccl/rccl.h>
#include <cstdio>
#include <cstdint>
#include <cstring>

namespace gar {

#define LAS __attribute__((address_space(3)))
typedef unsigned u32x4 __attribute__((ext_vector_type(4)));
typedef unsigned short u16x2 __attribute__((ext_vector_type(2)));
typedef short bf16x8 __attribute__((ext_vector_type(8)));
typedef float f32x4 __attribute__((ext_vector_type(4)));
typedef __amdgpu_buffer_rsrc_t rsrc_t;

constexpr int WORLD = 8;
constexpr int DIM = 8192;
constexpr int KSH = 1024;
constexpr int NGEMM = 128;
constexpr int GRID = 256;
#define PULL_WG_PER_LINK 12
#define PULL_DECODERS 4
constexpr int PWL = PULL_WG_PER_LINK;
constexpr int NDEC = PULL_DECODERS;
constexpr int PWAVES = 1;
constexpr int NPULL = 7 * PWL;
constexpr int NPACK = GRID - NGEMM - NPULL;
#define NRAW 24
constexpr int NFIRST = (NPACK + NGEMM) * 4;
static_assert(NPACK * 4 == 176, "roles");
constexpr int NSEQ = 3584;
constexpr unsigned long long DEADLINE = 40000000ull;

constexpr unsigned OFF_TMO = 128;
constexpr unsigned OFF_LINK = 1024;
constexpr unsigned OFF_CNT = 4096;
constexpr unsigned OFF_XFLAG = OFF_CNT + 256 * 128;
constexpr unsigned OFF_GRAN = 131072;
constexpr unsigned OFF_CALL = 640u << 10;
constexpr unsigned OFF_SLOT = 1u << 20;
constexpr unsigned OFF_AIMG = 64u << 20;
constexpr unsigned OFF_BIMG = 128u << 20;
constexpr unsigned OFF_XBUF = 160u << 20;
constexpr unsigned WS_USED = 192u << 20;
constexpr unsigned WS_NEED = WS_USED;
static_assert(OFF_XFLAG + 256 * 128 <= OFF_GRAN, "map");
static_assert(OFF_GRAN + WORLD * NSEQ * 16 <= OFF_CALL && OFF_CALL + 2048 * 64 <= OFF_SLOT, "map");
static_assert(OFF_SLOT + 6144u * 8192u <= OFF_AIMG, "map");
constexpr int LDS_BYTES = 131072;

struct Params {
    const void* x; const void* w; void* out;
    char* ws[WORLD];
    int rank; int pad;
};

constexpr int AUX_SC1 = 16, AUX_SYS = 17;

__device__ __forceinline__ rsrc_t mk_rsrc(const void* p, unsigned bytes) {
    return __builtin_amdgcn_make_buffer_rsrc(const_cast<void*>(p), 0, bytes, 0x00020000);
}
__device__ __forceinline__ unsigned rfl(unsigned v) { return __builtin_amdgcn_readfirstlane(v); }
__device__ __forceinline__ unsigned bfi(unsigned mask, unsigned a, unsigned b) { return (a & mask) | (b & ~mask); }
__device__ __forceinline__ unsigned perm(unsigned hi, unsigned lo, unsigned sel) { return __builtin_amdgcn_perm(hi, lo, sel); }
#define WAIT_VM0() asm volatile("s_waitcnt vmcnt(0)" ::: "memory")

#define STAMP(ws, idx) do { } while (0)

__device__ __forceinline__ unsigned call_issue(char* ws) {
    unsigned t = 0;
    const int zero = 0, one = 1;
    if ((threadIdx.x & 63) == 0) asm volatile("global_atomic_add %0, %1, %2, %3 sc0" : "=v"(t) : "v"(zero), "v"(one), "s"(ws + OFF_CALL + (blockIdx.x * 8 + rfl(threadIdx.x >> 6)) * 64) : "memory");
    return t;
}
__device__ __forceinline__ unsigned call_get(unsigned& t) {
    asm volatile("s_waitcnt vmcnt(0)" : "+v"(t) :: "memory");
    return rfl(t) + 1u;
}

__device__ __forceinline__ bool give_up(unsigned long long t0, unsigned* tmo, unsigned call) {
    if (rfl(__hip_atomic_load(tmo, __ATOMIC_RELAXED, __HIP_MEMORY_SCOPE_AGENT)) == call) return true;
    if (__builtin_amdgcn_s_memrealtime() - t0 > DEADLINE) { __hip_atomic_store(tmo, call, __ATOMIC_RELAXED, __HIP_MEMORY_SCOPE_AGENT); return true; }
    return false;
}
__device__ __forceinline__ void wait_word(unsigned* p, unsigned target, unsigned* tmo, unsigned call) {
    if ((int)(rfl(__hip_atomic_load(p, __ATOMIC_RELAXED, __HIP_MEMORY_SCOPE_SYSTEM)) - target) >= 0) return;
    const unsigned long long t0 = __builtin_amdgcn_s_memrealtime();
    while ((int)(rfl(__hip_atomic_load(p, __ATOMIC_RELAXED, __HIP_MEMORY_SCOPE_SYSTEM)) - target) < 0) {
        if (give_up(t0, tmo, call)) break;
        __builtin_amdgcn_s_sleep(4);
    }
}

struct Hdr { unsigned h0, h1, h2; };
constexpr unsigned AUX_LEN = 32, AUX_STR = 96, AUX_MAX = 2048, ENC_HIST = AUX_MAX + 16;
constexpr unsigned LANE_BITS_MAX = 256;

__device__ __forceinline__ unsigned bytemask(unsigned x, int sh) { return perm(0u, 0u, ((x >> sh) & 0x01010101u) | 0x0C0C0C0Cu); }
template <int CTRL, int RM = 0xf> __device__ __forceinline__ unsigned dpp0(unsigned v) { return (unsigned)__builtin_amdgcn_update_dpp(0, (int)v, CTRL, RM, 0xf, RM == 0xf); }
__device__ __forceinline__ unsigned wave_scan_incl(unsigned v) {
    v += dpp0<0x111>(v); v += dpp0<0x112>(v); v += dpp0<0x114>(v); v += dpp0<0x118>(v);
    v += dpp0<0x142, 0xa>(v); v += dpp0<0x143, 0xc>(v);
    return v;
}
__device__ __forceinline__ unsigned lut2x16(const unsigned (&T)[8], unsigned idx, unsigned sg) {
    const unsigned sel = idx & 0x07070707u, m8 = bytemask(idx, 3), ms = bytemask(sg, 7);
    const unsigned p = bfi(m8, perm(T[3], T[2], sel), perm(T[1], T[0], sel)), n = bfi(m8, perm(T[7], T[6], sel), perm(T[5], T[4], sel));
    return bfi(ms, n, p);
}
#define LDS_FENCE() asm volatile("" ::: "memory")

__device__ __forceinline__ Hdr encode_block(const u32x4 (&c)[8], u32x4 (&R4)[4], u32x4& a0, u32x4& a1, LAS unsigned char* scr, int lane) {
    unsigned E[16], R[16];
    u16x2 m1 = {0, 0}, m2 = {0, 0};
#pragma unroll
    for (int g = 0; g < 16; ++g) {
        const unsigned a = (g & 1) ? c[g >> 1].z : c[g >> 1].x, b = (g & 1) ? c[g >> 1].w : c[g >> 1].y;
        const unsigned H = perm(b, a, 0x07050301u), L = perm(b, a, 0x06040200u);
        E[g] = bfi(0xFEFEFEFEu, H << 1, L >> 7);
        R[g] = bfi(0x80808080u, H, L);
        m1 = __builtin_elementwise_max(m1, __builtin_bit_cast(u16x2, E[g]));
        m2 = __builtin_elementwise_max(m2, __builtin_bit_cast(u16x2, E[g] << 8));
    }
#pragma unroll
    for (int p = 0; p < 4; ++p) R4[p] = (u32x4){R[4 * p], R[4 * p + 1], R[4 * p + 2], R[4 * p + 3]};
    const u32x4 z4 = {0u, 0u, 0u, 0u};
    *(LAS u32x4*)(scr + lane * 16) = z4; *(LAS u32x4*)(scr + 1024 + lane * 16) = z4;
    if (lane < 9) *(LAS u32x4*)(scr + AUX_MAX + lane * 16) = z4;
    const u16x2 mm = __builtin_elementwise_max(m1, m2);
    unsigned mx = (mm.x > mm.y ? mm.x : mm.y) >> 8;
#pragma unroll
    for (int o = 32; o >= 1; o >>= 1) { const unsigned y = __shfl_xor(mx, o); mx = mx > y ? mx : y; }
    const unsigned base = rfl(mx), B = base * 0x01010101u;
    unsigned any = 0;
#pragma unroll
    for (int g = 0; g < 16; ++g) { E[g] = B - E[g]; any |= E[g]; }
    LDS_FENCE();
#pragma unroll
    for (int gi = 0; gi < 4; ++gi) {
        const int g = gi * 5;
        const unsigned k4 = ((E[g] & 0x0F0F0F0Fu) | ((R[g] >> 3) & 0x10101010u)) << 2;
#pragma unroll
        for (int b = 0; b < 4; ++b)
            __hip_atomic_fetch_add((LAS unsigned*)(scr + ENC_HIST + ((k4 >> (8 * b)) & 0x7Cu)), 1u, __ATOMIC_RELAXED, __HIP_MEMORY_SCOPE_WORKGROUP);
    }
    LDS_FENCE();
    const unsigned key = (*(LAS unsigned*)(scr + ENC_HIST + (lane & 31) * 4) << 4) | (15u - (lane & 15));
    unsigned rank = 0;
    rank += dpp0<0x121>(key) > key; rank += dpp0<0x122>(key) > key; rank += dpp0<0x123>(key) > key; rank += dpp0<0x124>(key) > key; rank += dpp0<0x125>(key) > key;
    rank += dpp0<0x126>(key) > key; rank += dpp0<0x127>(key) > key; rank += dpp0<0x128>(key) > key; rank += dpp0<0x129>(key) > key; rank += dpp0<0x12A>(key) > key;
    rank += dpp0<0x12B>(key) > key; rank += dpp0<0x12C>(key) > key; rank += dpp0<0x12D>(key) > key; rank += dpp0<0x12E>(key) > key; rank += dpp0<0x12F>(key) > key;
    unsigned fv = (rank + 1u) << (8 * (lane & 3));
    fv |= dpp0<0xB1>(fv); fv |= dpp0<0x4E>(fv);
    unsigned F[8];
#pragma unroll
    for (int q = 0; q < 8; ++q) F[q] = __builtin_amdgcn_readlane(fv, q * 4);
    if (lane < 32) *(LAS unsigned char*)(scr + (lane & 16) + rank) = (unsigned char)(base - (lane & 15));
    unsigned W[16], O[16], T = 0, bad = 0;
#pragma unroll
    for (int g = 0; g < 16; ++g) {
        unsigned ln = lut2x16(F, E[g] & 0x0F0F0F0Fu, R[g]);
        const bool over = __builtin_amdgcn_sad_u8(ln, 0u, 0u) > 31u;
        if (__ballot(over) != 0ull) {
            any |= 0x80u;
            const unsigned a = (ln & 0xFFu) > ((ln >> 8) & 0xFFu) ? (ln & 0xFFu) : ((ln >> 8) & 0xFFu), b = ((ln >> 16) & 0xFFu) > (ln >> 24) ? ((ln >> 16) & 0xFFu) : (ln >> 24);
            const unsigned eq = over ? (((ln ^ ((a > b ? a : b) * 0x01010101u)) + 0x7F7F7F7Fu) & 0x80808080u) ^ 0x80808080u : 0u;
            E[g] |= eq;
            ln = bfi(bytemask(eq, 7), 0x01010101u, ln);
        }
        unsigned pos = (ln & 0xFFu) - 1u, w = 1u << (pos & 31u);
        pos += (ln >> 8) & 0xFFu; w |= 1u << (pos & 31u);
        pos += (ln >> 16) & 0xFFu; w |= 1u << (pos & 31u);
        pos += ln >> 24; w |= 1u << (pos & 31u);
        W[g] = w; O[g] = T; T += pos + 1u; bad = bad > pos ? bad : pos;
    }
    bad = (bad > 30u) | (T > LANE_BITS_MAX);
    const unsigned incl = wave_scan_incl(T), total = __builtin_amdgcn_readlane(incl, 63), p0 = AUX_STR * 8u + incl - T;
    *(LAS unsigned char*)(scr + AUX_LEN + lane) = (unsigned char)(T - 64u);
    unsigned raw = (__ballot(bad != 0) != 0ull) || total > (AUX_MAX - AUX_STR) * 8u - 32u;
    if (!raw) {
#pragma unroll
        for (int g = 0; g < 16; ++g) {
            const unsigned p = p0 + O[g], s = p & 31u;
            LAS unsigned* q = (LAS unsigned*)(scr + ((p >> 5) << 2));
            __hip_atomic_fetch_or(q, W[g] << s, __ATOMIC_RELAXED, __HIP_MEMORY_SCOPE_WORKGROUP);
            __hip_atomic_fetch_or(q + 1, (W[g] >> 1) >> (31u - s), __ATOMIC_RELAXED, __HIP_MEMORY_SCOPE_WORKGROUP);
        }
    }
    unsigned r0 = 0, r1 = 0, r2 = 0, n = 0;
    if (__ballot((any & 0xF0F0F0F0u) != 0) != 0ull) {
#pragma unroll
        for (int g = 0; g < 16; ++g) {
            const unsigned f = E[g] & 0xF0F0F0F0u;
            unsigned long long mask = __ballot(f != 0);
            while (mask != 0ull && n <= 3) {
                const int L = __builtin_ctzll(mask); mask &= mask - 1;
                const unsigned fl = __builtin_amdgcn_readlane(f, L);
                const unsigned lo = __builtin_amdgcn_readlane((g & 1) ? c[g >> 1].z : c[g >> 1].x, L);
                const unsigned hi = __builtin_amdgcn_readlane((g & 1) ? c[g >> 1].w : c[g >> 1].y, L);
#pragma unroll
                for (int b = 0; b < 4; ++b) {
                    const bool e = ((fl >> (8 * b)) & 0xFFu) != 0;
                    const unsigned val = (((b & 2) ? hi : lo) >> (16 * (b & 1))) & 0xFFFFu;
                    const unsigned rec = (unsigned)(L * 64 + g * 4 + b) | (val << 12);
                    r0 = (e && n == 0) ? rec : r0; r1 = (e && n == 1) ? rec : r1; r2 = (e && n == 2) ? rec : r2;
                    n += e;
                }
            }
        }
    }
    raw |= n > 3;
    LDS_FENCE();
    a0 = *(LAS u32x4*)(scr + lane * 16); a1 = *(LAS u32x4*)(scr + 1024 + lane * 16);
    const unsigned n16 = (AUX_STR + ((total + 7u) >> 3) + 15u) >> 4;
    Hdr h;
    h.h0 = r0 | ((n16 & 15u) << 28);
    h.h1 = r1 | ((n16 >> 4) << 28);
    h.h2 = r2 | ((n & 3u) << 28) | (raw << 31);
    return h;
}

__device__ __forceinline__ void decode_block(LAS const unsigned char* src, u32x4 (&c)[8], int lane) {
    LAS const unsigned char* aux = src + 4096;
    const u32x4 t0 = *(LAS const u32x4*)(aux), t1 = *(LAS const u32x4*)(aux + 16);
    const unsigned T = *(LAS const unsigned char*)(aux + AUX_LEN + lane) + 64u;
    u32x4 R4[4];
#pragma unroll
    for (int p = 0; p < 4; ++p) R4[p] = *(LAS const u32x4*)(src + p * 1024 + lane * 16);
    const unsigned p0 = AUX_STR * 8u + wave_scan_incl(T) - T, s = p0 & 31u;
    LAS const unsigned* q = (LAS const unsigned*)(aux + ((p0 >> 5) << 2));
    unsigned S[9];
#pragma unroll
    for (int i = 0; i < 9; ++i) S[i] = q[i];
#pragma unroll
    for (int i = 0; i < 8; ++i) S[i] = __builtin_amdgcn_alignbit(S[i + 1], S[i], s);
    const unsigned I[8] = {rfl(t0.x), rfl(t0.y), rfl(t0.z), rfl(t0.w), rfl(t1.x), rfl(t1.y), rfl(t1.z), rfl(t1.w)};
#pragma unroll
    for (int g = 0; g < 16; ++g) {
        const unsigned w0 = S[0], q0 = __builtin_ctz(w0), w1 = w0 & (w0 - 1u), q1 = __builtin_ctz(w1);
        const unsigned w2 = w1 & (w1 - 1u), q2 = __builtin_ctz(w2), w3 = w2 & (w2 - 1u), q3 = __builtin_ctz(w3);
        const unsigned Q = q0 | (q1 << 8) | (q2 << 16) | (q3 << 24);
        const unsigned rk = Q - (Q << 8) - 0x01010100u;
        if (g < 15) {
            const unsigned sh = q3 + 1u;
#pragma unroll
            for (int i = 0; i < 7; ++i) if (32 * i < 256 - 4 * (g + 1)) S[i] = (32 * (i + 1) < 256 - 4 * g) ? __builtin_amdgcn_alignbit(S[i + 1], S[i], sh) : S[i] >> sh;
            if (32 * 7 < 256 - 4 * (g + 1)) S[7] >>= sh;
        }
        const unsigned Rg = (g & 3) == 0 ? R4[g >> 2].x : (g & 3) == 1 ? R4[g >> 2].y : (g & 3) == 2 ? R4[g >> 2].z : R4[g >> 2].w;
        const unsigned E = lut2x16(I, rk, Rg);
        const unsigned H = bfi(0x80808080u, Rg, E >> 1), L = bfi(0x80808080u, E << 7, Rg);
        const unsigned x = perm(H, L, 0x05010400u), y = perm(H, L, 0x07030602u);
        if (g & 1) { c[g >> 1].z = x; c[g >> 1].w = y; } else { c[g >> 1].x = x; c[g >> 1].y = y; }
    }
}

__device__ __forceinline__ void store_granule(char* p, u32x4 g) {
    asm volatile("global_store_dwordx4 %0, %1, off sc0 sc1\n\ts_nop 1" :: "v"(p), "v"(g) : "memory");
}

__device__ __forceinline__ void publish_block(const Params& P, rsrc_t wsr, const u32x4 (&c)[8], unsigned slot, unsigned n, unsigned dmask, unsigned call, LAS unsigned char* scr, int lane) {
    u32x4 R4[4], a0, a1;
    Hdr h = {0u, 0u, 1u << 31};
    if (n >= NRAW) h = encode_block(c, R4, a0, a1, scr, lane);
    const unsigned so = OFF_SLOT + slot * 8192u;
    if (h.h2 >> 31) {
#pragma unroll
        for (int j = 0; j < 8; ++j) __builtin_amdgcn_raw_buffer_store_b128(c[j], wsr, lane * 16 + j * 1024, so, AUX_SYS);
    } else {
        const unsigned n16 = (h.h0 >> 28) | ((h.h1 >> 28) << 4);
#pragma unroll
        for (int j = 0; j < 4; ++j) __builtin_amdgcn_raw_buffer_store_b128(R4[j], wsr, lane * 16 + j * 1024, so, AUX_SYS);
        if ((unsigned)lane < n16) __builtin_amdgcn_raw_buffer_store_b128(a0, wsr, lane * 16 + 4096, so, AUX_SYS);
        if ((unsigned)lane + 64u < n16) __builtin_amdgcn_raw_buffer_store_b128(a1, wsr, lane * 16 + 5120, so, AUX_SYS);
    }
    WAIT_VM0();
    if (lane < WORLD && ((dmask >> lane) & 1u)) {
        char* p = P.ws[lane & 7] + OFF_GRAN + ((unsigned)P.rank * NSEQ + n) * 16u;
        store_granule(p, (u32x4){call, h.h0, h.h1, h.h2});
    }
}

__device__ __forceinline__ void packer_role(const Params& P, LAS unsigned char* lds, unsigned& ticket, int first, int next, int step) {
    const int lane = threadIdx.x & 63, wid = rfl(threadIdx.x >> 6);
    char* ws = P.ws[P.rank];
    const rsrc_t wsr = mk_rsrc(ws, WS_USED);
    const rsrc_t xr = mk_rsrc(P.x, DIM * KSH * 2), wr = mk_rsrc(P.w, DIM * KSH * 2);
    const int myi = P.rank >> 2, myj = P.rank & 3;
    LAS unsigned char* my = lds + wid * 8192;
    for (int p = first; p < 4096; p = p < NFIRST ? next : p + step) {
        bool is_w; int kk, sel, r, bi;
        if (p < 3072) {
            int rem;
            if (p < 64) { kk = 0; rem = 128 + p; } else if (p < 176) { kk = 1; rem = p - 64; } else if (p < 304) { kk = 0; rem = p - 176; } else if (p < 384) { kk = 1; rem = p - 192; }
            else { kk = p / 192; rem = p - kk * 192; }
            bi = rem & 31; r = 0; is_w = rem < 128; sel = is_w ? (rem >> 5) : ((rem - 128) >> 5);
        }
        else { const int q = p - 3072; kk = q >> 6; sel = (q >> 5) & 1; bi = q & 31; r = 1; is_w = false; }
        u32x4 c[8];
        if (!is_w) {
            const int rr = lane >> 3;
            const unsigned vo = (unsigned)rr * 2048u + (unsigned)(((lane & 7) ^ rr) * 16);
            const unsigned so = (unsigned)(sel * 4096 + r * 2048 + bi * 64) * 2048u + (unsigned)kk * 128u;
#pragma unroll
            for (int j = 0; j < 8; ++j) c[j] = __builtin_amdgcn_raw_buffer_load_b128(xr, vo + j * 8 * 2048, so, 0);
        } else {
            const int pn = bi >> 2, bj = (bi >> 1) & 1, q = bi & 1;
            const int gk = lane >> 3, cc = lane & 7, wcl = cc >> 2, cq = cc & 3;
            const unsigned vo = (unsigned)(gk * 8) * 16384u + (unsigned)((64 * (2 * q + wcl) + 32 * bj + 8 * cq) * 2);
            const unsigned so = (unsigned)(kk * 64) * 16384u + (unsigned)((sel * 2048 + pn * 256) * 2);
            u32x4 R[8];
#pragma unroll
            for (int jj = 0; jj < 8; ++jj) R[jj] = __builtin_amdgcn_raw_buffer_load_b128(wr, vo + jj * 16384, so, 0);
#pragma unroll
            for (int j = 0; j < 8; ++j) {
                const unsigned s = (j & 1) ? 0x07060302u : 0x05040100u;
                u32x4 t;
                t.x = perm(R[1][j >> 1], R[0][j >> 1], s); t.y = perm(R[3][j >> 1], R[2][j >> 1], s);
                t.z = perm(R[5][j >> 1], R[4][j >> 1], s); t.w = perm(R[7][j >> 1], R[6][j >> 1], s);
                const int s64 = 32 * wcl + 16 * (j >> 2) + 4 * cq + (j & 3);
                *(LAS u32x4*)(my + s64 * 128 + ((gk ^ (s64 & 7)) * 16)) = t;
            }
            asm volatile("s_waitcnt lgkmcnt(0)" ::: "memory");
#pragma unroll
            for (int j = 0; j < 8; ++j) c[j] = *(LAS u32x4*)(my + j * 1024 + lane * 16);
            asm volatile("s_waitcnt lgkmcnt(0)" ::: "memory");
        }
        const unsigned call = call_get(ticket);
        const int g = P.rank * 16 + kk;
        const bool self = is_w ? (sel == myj) : (sel == myi);
        if (self) {
            const unsigned so = is_w ? OFF_BIMG + (unsigned)(g * 32 + bi) * 8192u : OFF_AIMG + (unsigned)((r * 128 + g) * 32 + bi) * 8192u;
#pragma unroll
            for (int j = 0; j < 8; ++j) __builtin_amdgcn_raw_buffer_store_b128(c[j], wsr, lane * 16 + j * 1024, so, AUX_SC1);
        }
        const unsigned slot = is_w ? 2048u + (unsigned)((sel * 16 + kk) * 32 + bi) : (unsigned)(((sel * 2 + r) * 16 + kk) * 32 + bi);
        const unsigned n = is_w ? (unsigned)(kk * 64 + 32 + bi) : (r == 0 ? (unsigned)(kk * 64 + bi) : (unsigned)(1024 + kk * 32 + bi));
        unsigned dmask = is_w ? (0x11u << sel) : (0x0Fu << (4 * sel));
        dmask &= ~(1u << P.rank);
        publish_block(P, wsr, c, slot, n, dmask, call, my, lane);
        if (self && lane == 0)
            __hip_atomic_fetch_add((unsigned*)(ws + OFF_CNT + (r * 128 + g) * 128), 1u, __ATOMIC_RELAXED, __HIP_MEMORY_SCOPE_SYSTEM);
    }
}

__device__ __forceinline__ u32x4 load_granule(rsrc_t wsr, unsigned off) { return __builtin_amdgcn_raw_buffer_load_b128(wsr, 0, off, AUX_SYS); }

struct Blk { unsigned slot, dso, cnt; bool is_out; };
__device__ __forceinline__ Blk locate(unsigned n, int src, int myi, int myj) {
    Blk b; b.is_out = false;
    if (n < 1024) { const int kk = n >> 6, bi = n & 31, g = src * 16 + kk; b.cnt = OFF_CNT + g * 128;
        if ((n & 63) >= 32) { b.slot = 2048u + (myj * 16 + kk) * 32 + bi; b.dso = OFF_BIMG + (unsigned)(g * 32 + bi) * 8192u; }
        else { b.slot = ((myi * 2) * 16 + kk) * 32 + bi; b.dso = OFF_AIMG + (unsigned)(g * 32 + bi) * 8192u; } }
    else if (n < 1536) { const int kk = (n - 1024) >> 5, bi = n & 31, g = src * 16 + kk; b.cnt = OFF_CNT + (128 + g) * 128;
        b.slot = ((myi * 2 + 1) * 16 + kk) * 32 + bi; b.dso = OFF_AIMG + (unsigned)((128 + g) * 32 + bi) * 8192u; }
    else { const int o = n - 1536, w = o & 7, h = (o >> 3) & 1, tile = (o >> 4) & 63, r = o >> 10; b.cnt = 0; b.is_out = true;
        b.slot = 4096u + o;
        const unsigned row0 = (src >> 2) * 4096 + r * 2048 + (tile >> 3) * 256 + h * 128 + (w >> 2) * 64, col0 = (src & 3) * 2048 + (tile & 7) * 256 + (w & 3) * 64;
        b.dso = (row0 * DIM + col0) * 2u; }
    return b;
}
__device__ __forceinline__ void signal_cnt(char* ws, unsigned off, int lane) {
    if (off && lane == 0) __hip_atomic_fetch_add((unsigned*)(ws + off), 1u, __ATOMIC_RELAXED, __HIP_MEMORY_SCOPE_SYSTEM);
}
constexpr unsigned NSLOT = 15, CTL = NSLOT * 8192;
constexpr unsigned M_DEAD = 1u << 30, N_STOP = 0xFFFFFFFFu;
__device__ __forceinline__ unsigned lds_ld(LAS unsigned char* p) { return *(volatile LAS unsigned*)p; }
__device__ __forceinline__ void lds_st(LAS unsigned char* p, unsigned v) { *(volatile LAS unsigned*)p = v; }
__device__ __forceinline__ void ring_publish(LAS unsigned char* lds, unsigned i, unsigned n, unsigned h0, unsigned h1, unsigned h2, int lane) {
    const unsigned sl = i % NSLOT;
    if (lane == 0) { *(volatile LAS u32x4*)(lds + CTL + sl * 16) = (u32x4){n, h0, h1, h2}; lds_st(lds + CTL + 256 + sl * 4, i + 1u); }
}
__device__ __forceinline__ void dma_op(rsrc_t r, LAS unsigned char* dst, int j, unsigned so, int lane) {
    __builtin_amdgcn_raw_ptr_buffer_load_lds(r, (LAS void*)(dst + j * 1024), 16, lane * 16 + j * 1024, so, 0, AUX_SYS);
}
#define VM_WAIT_(n) asm volatile("s_waitcnt vmcnt(" #n ")" ::: "memory")
#define VM_WAIT(n) VM_WAIT_(n)
#define WIN 5

__device__ __forceinline__ void loader_wave(const Params& P, LAS unsigned char* lds, unsigned call, int src, int lane) {
    char* ws = P.ws[P.rank];
    unsigned* tmo = (unsigned*)(ws + OFF_TMO);
    const rsrc_t wsr = mk_rsrc(ws, WS_USED);
    char* const remp = P.ws[src];
    const int myi = P.rank >> 2, myj = P.rank & 3;
    const unsigned gbase = OFF_GRAN + (unsigned)src * NSEQ * 16u;
    char* const gptr = ws + gbase;
    const unsigned tko = OFF_LINK + (unsigned)src * 128u, tbase = (call - 1u) * (NSEQ + 3u * PWL);
    __builtin_amdgcn_s_setprio(3);
    unsigned n, n1, n2;
    {
        unsigned t0 = 0;
        if (lane == 0) t0 = __hip_atomic_fetch_add((unsigned*)(ws + tko), 3u, __ATOMIC_RELAXED, __HIP_MEMORY_SCOPE_AGENT);
        n = rfl(t0) - tbase; n1 = n + 1; n2 = n + 2;
    }
    unsigned gx, gy, gz, gw, hx, hy, hz, hw;
    { const u32x4 a = load_granule(wsr, gbase + (n < NSEQ ? n : 0u) * 16u), b = load_granule(wsr, gbase + (n1 < NSEQ ? n1 : 0u) * 16u);
      gx = rfl(a.x); gy = rfl(a.y); gz = rfl(a.z); gw = rfl(a.w); hx = rfl(b.x); hy = rfl(b.y); hz = rfl(b.z); hw = rfl(b.w); }
    const int zero = 0, one = 1;
    unsigned i = 0;
    bool pend = false; unsigned pn = 0, p0 = 0, p1 = 0, p2 = 0;
    while (n < NSEQ) {
        const unsigned sl = i % NSLOT;
        bool dead = false;
        const bool need_gran = (int)(gx - call) < 0;
        const bool need_slot = i >= NSLOT && (int)(lds_ld(lds + CTL + 320 + sl * 4) - (i - NSLOT + 1u)) < 0;
        if (need_gran || need_slot) {
            if (pend) { VM_WAIT(0); ring_publish(lds, i - 1u, pn, p0, p1, p2, lane); pend = false; }
            const unsigned long long t0 = __builtin_amdgcn_s_memrealtime();
            while ((int)(gx - call) < 0) {
                const u32x4 a = load_granule(wsr, gbase + n * 16u);
                gx = rfl(a.x); gy = rfl(a.y); gz = rfl(a.z); gw = rfl(a.w);
                if ((int)(gx - call) >= 0) break;
                if (give_up(t0, tmo, call)) { dead = true; break; }
                __builtin_amdgcn_s_sleep(2);
                asm volatile("" ::: "memory");
            }
            while (i >= NSLOT && (int)(lds_ld(lds + CTL + 320 + sl * 4) - (i - NSLOT + 1u)) < 0) __builtin_amdgcn_s_sleep(1);
        }
        const bool raw = (gw >> 31) != 0;
        const unsigned so = OFF_SLOT + locate(n, src, myi, myj).slot * 8192u;
        const unsigned bytes = dead ? 0u : raw ? 8192u : 4096u + (((gy >> 28) | ((gz >> 28) << 4)) << 4);
        const rsrc_t rem = mk_rsrc(remp, so + bytes);
        LAS unsigned char* dst = lds + sl * 8192;
        u32x4 g2; unsigned tk = 0;
        asm volatile("global_load_dwordx4 %0, %1, %2 sc0 sc1" : "=v"(g2) : "v"(zero), "s"(gptr + (n2 < NSEQ ? n2 : 0u) * 16u) : "memory");
        if (lane == 0) asm volatile("global_atomic_add %0, %1, %2, %3 sc0" : "=v"(tk) : "v"(zero), "v"(one), "s"(ws + tko) : "memory");
        VM_WAIT(WIN); dma_op(rem, dst, 0, so, lane);
        VM_WAIT(WIN); dma_op(rem, dst, 1, so, lane);
        VM_WAIT(WIN); dma_op(rem, dst, 2, so, lane);
        VM_WAIT(WIN); if (pend) ring_publish(lds, i - 1u, pn, p0, p1, p2, lane); dma_op(rem, dst, 3, so, lane);
        VM_WAIT(WIN); dma_op(rem, dst, 4, so, lane);
        asm volatile("s_waitcnt vmcnt(5)" : "+v"(g2), "+v"(tk) :: "memory"); dma_op(rem, dst, 5, so, lane);
        if (raw) { dma_op(rem, dst, 6, so, lane); dma_op(rem, dst, 7, so, lane); VM_WAIT(0); }
        pend = true; pn = n; p0 = gy; p1 = gz; p2 = gw | (dead ? M_DEAD : 0u);
        n = n1; gx = hx; gy = hy; gz = hz; gw = hw;
        n1 = n2; hx = rfl(g2.x); hy = rfl(g2.y); hz = rfl(g2.z); hw = rfl(g2.w);
        n2 = rfl(tk) - tbase;
        ++i;
    }
    VM_WAIT(0);
    if (pend) ring_publish(lds, i - 1u, pn, p0, p1, p2, lane);
    for (int d = 0; d < NDEC; ++d, ++i) {
        const unsigned sl = i % NSLOT;
        while (i >= NSLOT && (int)(lds_ld(lds + CTL + 320 + sl * 4) - (i - NSLOT + 1u)) < 0) __builtin_amdgcn_s_sleep(1);
        ring_publish(lds, i, N_STOP, 0u, 0u, 0u, lane);
    }
}

__device__ __forceinline__ void decoder_wave(const Params& P, LAS unsigned char* lds, unsigned call, int src, int d, int lane) {
    char* ws = P.ws[P.rank];
    unsigned* tmo = (unsigned*)(ws + OFF_TMO);
    const rsrc_t wsr = mk_rsrc(ws, WS_USED), outr = mk_rsrc(P.out, (unsigned)DIM * DIM * 2u);
    const int myi = P.rank >> 2, myj = P.rank & 3;
    for (unsigned i = d;; i += NDEC) {
        const unsigned sl = i % NSLOT;
        while (lds_ld(lds + CTL + 256 + sl * 4) != i + 1u) __builtin_amdgcn_s_sleep(1);
        asm volatile("" ::: "memory");
        const u32x4 meta = *(volatile LAS u32x4*)(lds + CTL + sl * 16);
        const unsigned n = rfl(meta.x), h0 = rfl(meta.y), h1 = rfl(meta.z), h2 = rfl(meta.w);
        if (n == N_STOP) break;
        const Blk b = locate(n, src, myi, myj);
        const bool raw = (h2 >> 31) != 0;
        bool dead = (h2 & M_DEAD) != 0;
        if (b.is_out && rfl(__hip_atomic_load(tmo, __ATOMIC_RELAXED, __HIP_MEMORY_SCOPE_AGENT)) == call) dead = true;
        u32x4 c[8];
        LAS unsigned char* cur = lds + sl * 8192;
        if (dead) {
#pragma unroll
            for (int j = 0; j < 8; ++j) c[j] = (u32x4){0x7FC07FC0u, 0x7FC07FC0u, 0x7FC07FC0u, 0x7FC07FC0u};
        } else if (raw) {
#pragma unroll
            for (int j = 0; j < 8; ++j) c[j] = *(LAS u32x4*)(cur + j * 1024 + lane * 16);
        } else decode_block(cur, c, lane);
#pragma unroll
        for (int j = 0; j < 8; ++j) asm volatile("" : "+v"(c[j]));
        if (lane == 0) lds_st(lds + CTL + 320 + sl * 4, i + 1u);
        const rsrc_t dr = b.is_out ? outr : wsr;
        const unsigned sj = b.is_out ? 8u * DIM * 2u : 1024u, sr = b.is_out ? DIM * 2u : 128u;
        const unsigned vo = (unsigned)(lane >> 3) * sr + (unsigned)(lane & 7) * 16u;
#pragma unroll
        for (int j = 0; j < 8; ++j) __builtin_amdgcn_raw_buffer_store_b128(c[j], dr, vo, b.dso + j * sj, AUX_SC1);
        if (!dead && !raw) {
            const unsigned ne = (h2 >> 28) & 3u;
            for (unsigned e = 0; e < ne; ++e) {
                const unsigned rec = e == 0 ? h0 : e == 1 ? h1 : h2, pos = rec & 0xFFFu, v = pos & 63u;
                if ((unsigned)lane == (pos >> 6)) __builtin_amdgcn_raw_buffer_store_b16((unsigned short)((rec >> 12) & 0xFFFFu), dr, vo + (v & 7u) * 2u, b.dso + (v >> 3) * sj, AUX_SC1);
            }
        }
        if (b.cnt) { WAIT_VM0(); signal_cnt(ws, b.cnt, lane); }
    }
}

__device__ __forceinline__ void puller_role(const Params& P, LAS unsigned char* lds, unsigned call, int wg) {
    const int lane = threadIdx.x & 63, wid = rfl(threadIdx.x >> 6);
    const int link = wg % 7, src = rfl((P.rank + 1 + link) & 7);
    if (threadIdx.x < 32) *(volatile LAS unsigned*)(lds + CTL + 256 + threadIdx.x * 4) = 0u;
    __syncthreads();
    if (wid == 0) loader_wave(P, lds, call, src, lane);
    else if (wid <= NDEC) decoder_wave(P, lds, call, src, wid - 1, lane);
}

__device__ __forceinline__ float gelu_f(float y) {
    const float c1 = -2.0f * 0.7978845608028654f * 1.4426950408889634f, c2 = c1 * 0.044715f;
    const float e = __builtin_amdgcn_exp2f(y * (c1 + c2 * y * y));
    return y * __builtin_amdgcn_rcpf(1.0f + e);
}
__device__ __forceinline__ unsigned cvt_pk_bf16(float lo, float hi) { unsigned r; asm("v_cvt_pk_bf16_f32 %0, %1, %2" : "=v"(r) : "v"(lo), "v"(hi)); return r; }

__device__ __forceinline__ void gemm_role(const Params& P, LAS unsigned char* lds, unsigned call, int wg) {
    const int tid = threadIdx.x, wid = rfl(tid >> 6), lane = tid & 63, wr = wid >> 2, wc = wid & 3, fr = lane & 15, fq = lane >> 4;
    char* ws = P.ws[P.rank];
    unsigned* tmo = (unsigned*)(ws + OFF_TMO);
    const rsrc_t wsr = mk_rsrc(ws, WS_USED);
    const int xcd = wg & 7, q = (wg - NPACK) >> 3, h = xcd & 1, blk = xcd >> 1;
    const int tm = (blk >> 1) * 4 + (q >> 2), tn = (blk & 1) * 4 + (q & 3), tile = tm * 8 + tn;
    const unsigned voff = (unsigned)tid * 16u, ldsw = (unsigned)wid * 1024u;
    const int aoff0 = (wr * 64 + fr) * 128 + ((fq ^ (fr & 7)) << 4), aoff1 = aoff0 ^ 64;
    const int boff0 = (wc * 32 + fr) * 128 + ((fq ^ (fr & 7)) << 4), boff1 = boff0 ^ 64;
#define G_SA(b, hh) (((b) * 2 + (hh)) * 16384)
#define G_SB(b, hh) ((4 + (b) * 2 + (hh)) * 16384)
#define G_STAGE(bufoff, soff) do { _Pragma("unroll") for (int _i = 0; _i < 2; ++_i) \
        __builtin_amdgcn_raw_ptr_buffer_load_lds(wsr, (LAS void*)(lds + (bufoff) + ldsw + _i * 8192), 16, voff, (soff) + _i * 8192, 0, AUX_SYS); } while (0)
#define G_LDA(dst, b, hh) do { _Pragma("unroll") for (int m = 0; m < 4; ++m) { \
        dst[m][0] = *(const LAS bf16x8*)(lds + G_SA(b, hh) + aoff0 + m * 2048); dst[m][1] = *(const LAS bf16x8*)(lds + G_SA(b, hh) + aoff1 + m * 2048); } } while (0)
#define G_LDB(dst, b, hh) do { _Pragma("unroll") for (int n = 0; n < 2; ++n) { \
        dst[n][0] = *(const LAS bf16x8*)(lds + G_SB(b, hh) + boff0 + n * 2048); dst[n][1] = *(const LAS bf16x8*)(lds + G_SB(b, hh) + boff1 + n * 2048); } } while (0)
#define G_MMA(ai, bj, At, Bt) do { __builtin_amdgcn_s_setprio(1); _Pragma("unroll") for (int m = 0; m < 4; ++m) _Pragma("unroll") for (int n = 0; n < 2; ++n) _Pragma("unroll") for (int k = 0; k < 2; ++k) \
        acc[ai][bj][m][n] = __builtin_amdgcn_mfma_f32_16x16x32_bf16(Bt[n][k], At[m][k], acc[ai][bj][m][n], 0, 0, 0); __builtin_amdgcn_s_setprio(0); } while (0)
#define G_WAIT_V(n) asm volatile("s_waitcnt vmcnt(" #n ")" ::: "memory")
#define G_WAIT_L(n) asm volatile("s_waitcnt lgkmcnt(" #n ")" ::: "memory")
#define G_BAR __builtin_amdgcn_s_barrier()
#define G_SCHED __builtin_amdgcn_sched_barrier(0)
#define G_GIDX(t) ((unsigned)((4 * h + ((t) & 3)) * 16 + ((t) >> 2)))
    const int zero = 0;
    for (int r = 0; r < 2; ++r) {
        const unsigned target = call * (r == 0 ? 64u : 32u);
        const unsigned abase = OFF_AIMG + (unsigned)r * (128u * 262144u) + (unsigned)tm * 32768u, bbase = OFF_BIMG + (unsigned)tn * 32768u;
        char* cbase = ws + OFF_CNT + r * 128 * 128;
        f32x4 acc[2][2][4][2];
#pragma unroll
        for (int a = 0; a < 2; ++a)
#pragma unroll
            for (int b = 0; b < 2; ++b)
#pragma unroll
                for (int m = 0; m < 4; ++m)
#pragma unroll
                    for (int n = 0; n < 2; ++n) acc[a][b][m][n] = (f32x4){0.f, 0.f, 0.f, 0.f};
        bf16x8 At[4][2], B0[2][2], B1[2][2];
        STAMP(ws, r * 8 + 0);
        wait_word((unsigned*)(cbase + G_GIDX(0) * 128), target, tmo, call);
        wait_word((unsigned*)(cbase + G_GIDX(1) * 128), target, tmo, call);
        wait_word((unsigned*)(cbase + G_GIDX(2) * 128), target, tmo, call);
        STAMP(ws, r * 8 + 1);
        {
            const unsigned a0 = abase + G_GIDX(0) * 262144u, b0 = bbase + G_GIDX(0) * 262144u, a1 = abase + G_GIDX(1) * 262144u, b1 = bbase + G_GIDX(1) * 262144u;
            G_STAGE(G_SB(0, 0), b0); G_STAGE(G_SA(0, 0), a0); G_STAGE(G_SB(0, 1), b0 + 16384); G_STAGE(G_SA(0, 1), a0 + 16384);
            if (wr == 1) G_BAR;
            G_WAIT_V(4); G_BAR;
            G_STAGE(G_SB(1, 0), b1); G_STAGE(G_SA(1, 0), a1); G_STAGE(G_SB(1, 1), b1 + 16384);
            G_WAIT_V(6); G_BAR;
        }
        unsigned fA = target, fB;
        for (int t = 0; t < 64; t += 2) {
            const int t2 = t + 2 < 64 ? t + 2 : 63, t3 = t + 3 < 64 ? t + 3 : 63, t4 = t + 4 < 64 ? t + 4 : 63;
            const unsigned g1 = G_GIDX(t + 1), g2 = G_GIDX(t2), g3 = G_GIDX(t3), g4 = G_GIDX(t4);
            const unsigned a1 = abase + g1 * 262144u, a2 = abase + g2 * 262144u, b2 = bbase + g2 * 262144u, a3 = abase + g3 * 262144u, b3 = bbase + g3 * 262144u;
            asm volatile("global_load_dword %0, %1, %2 sc0 sc1" : "=v"(fB) : "v"(zero), "s"(cbase + g3 * 128));
            G_LDB(B0, 0, 0); G_SCHED; G_LDA(At, 0, 0); G_STAGE(G_SA(1, 1), a1 + 16384);
            G_WAIT_L(8); G_BAR; G_WAIT_L(0); G_MMA(0, 0, At, B0); G_BAR; G_SCHED;
            asm volatile("" : "+v"(fA));
            if ((int)(rfl(fA) - target) < 0) wait_word((unsigned*)(cbase + g2 * 128), target, tmo, call);
            G_LDB(B1, 0, 1); G_STAGE(G_SB(0, 0), b2);
            G_BAR; G_WAIT_L(0); G_MMA(0, 1, At, B1); G_BAR;
            G_LDA(At, 0, 1); G_STAGE(G_SA(0, 0), a2);
            G_BAR; G_WAIT_L(0); G_MMA(1, 0, At, B0); G_BAR; G_SCHED;
            G_STAGE(G_SB(0, 1), b2 + 16384);
            G_WAIT_V(6); G_BAR; G_MMA(1, 1, At, B1); G_BAR;
            asm volatile("global_load_dword %0, %1, %2 sc0 sc1" : "=v"(fA) : "v"(zero), "s"(cbase + g4 * 128));
            G_LDB(B0, 1, 0); G_SCHED; G_LDA(At, 1, 0); G_STAGE(G_SA(0, 1), a2 + 16384);
            G_WAIT_L(8); G_BAR; G_WAIT_L(0); G_MMA(0, 0, At, B0); G_BAR; G_SCHED;
            asm volatile("" : "+v"(fB));
            if ((int)(rfl(fB) - target) < 0) wait_word((unsigned*)(cbase + g3 * 128), target, tmo, call);
            G_LDB(B1, 1, 1); G_STAGE(G_SB(1, 0), b3);
            G_BAR; G_WAIT_L(0); G_MMA(0, 1, At, B1); G_BAR;
            G_LDA(At, 1, 1); G_STAGE(G_SA(1, 0), a3);
            G_BAR; G_WAIT_L(0); G_MMA(1, 0, At, B0); G_BAR; G_SCHED;
            G_STAGE(G_SB(1, 1), b3 + 16384);
            G_WAIT_V(6); G_BAR; G_MMA(1, 1, At, B1); G_BAR;
        }
        G_WAIT_V(0);
        asm volatile("" : "+v"(fA));
        if (wr == 0) G_BAR;
        G_BAR;
        STAMP(ws, r * 8 + 2);
        if (h) {
#pragma unroll
            for (int b = 0; b < 2; ++b)
#pragma unroll
                for (int m = 0; m < 4; ++m)
#pragma unroll
                    for (int n = 0; n < 2; ++n) { const f32x4 tq = acc[0][b][m][n]; acc[0][b][m][n] = acc[1][b][m][n]; acc[1][b][m][n] = tq; }
        }
        const unsigned xl = (unsigned)((r * 64 + tile) * 2);
#pragma unroll
        for (int b = 0; b < 2; ++b)
#pragma unroll
            for (int m = 0; m < 4; ++m)
#pragma unroll
                for (int n = 0; n < 2; ++n)
                    __builtin_amdgcn_raw_buffer_store_b128(__builtin_bit_cast(u32x4, acc[1][b][m][n]), wsr, lane * 16 + ((b * 4 + m) * 2 + n) * 1024, OFF_XBUF + ((xl + h) * 8 + wid) * 16384u, AUX_SC1);
        WAIT_VM0();
        if (lane == 0) __hip_atomic_store((unsigned*)(ws + OFF_XFLAG + (xl + h) * 128 + wid * 16), call, __ATOMIC_RELAXED, __HIP_MEMORY_SCOPE_AGENT);
        wait_word((unsigned*)(ws + OFF_XFLAG + (xl + (h ^ 1)) * 128 + wid * 16), call, tmo, call);
        const bool dead = rfl(__hip_atomic_load(tmo, __ATOMIC_RELAXED, __HIP_MEMORY_SCOPE_AGENT)) == call;
        STAMP(ws, r * 8 + 3);
        LAS unsigned char* my = lds + wid * 8192;
#pragma unroll
        for (int b = 0; b < 2; ++b)
#pragma unroll
            for (int m = 0; m < 4; ++m) {
                const f32x4 p0 = __builtin_bit_cast(f32x4, __builtin_amdgcn_raw_buffer_load_b128(wsr, lane * 16 + ((b * 4 + m) * 2 + 0) * 1024, OFF_XBUF + ((xl + (h ^ 1)) * 8 + wid) * 16384u, AUX_SC1));
                const f32x4 p1 = __builtin_bit_cast(f32x4, __builtin_amdgcn_raw_buffer_load_b128(wsr, lane * 16 + ((b * 4 + m) * 2 + 1) * 1024, OFF_XBUF + ((xl + (h ^ 1)) * 8 + wid) * 16384u, AUX_SC1));
                const f32x4 v0 = acc[0][b][m][0] + p0, v1 = acc[0][b][m][1] + p1;
                u32x4 o;
                o.x = cvt_pk_bf16(gelu_f(v0[0]), gelu_f(v0[1])); o.y = cvt_pk_bf16(gelu_f(v0[2]), gelu_f(v0[3]));
                o.z = cvt_pk_bf16(gelu_f(v1[0]), gelu_f(v1[1])); o.w = cvt_pk_bf16(gelu_f(v1[2]), gelu_f(v1[3]));
                const int row = m * 16 + fr, ch = b * 4 + fq;
                *(LAS u32x4*)(my + row * 128 + ((ch ^ (row & 7)) * 16)) = o;
            }
        asm volatile("s_waitcnt lgkmcnt(0)" ::: "memory");
        u32x4 c[8];
#pragma unroll
        for (int j = 0; j < 8; ++j) { const int row = j * 8 + (lane >> 3); c[j] = *(LAS u32x4*)(my + row * 128 + (((lane & 7) ^ (row & 7)) * 16)); }
        if (dead) {
#pragma unroll
            for (int j = 0; j < 8; ++j) c[j] = (u32x4){0x7FC07FC0u, 0x7FC07FC0u, 0x7FC07FC0u, 0x7FC07FC0u};
        }
        {
            const rsrc_t outr = mk_rsrc(P.out, (unsigned)DIM * DIM * 2u);
            const unsigned row0 = (P.rank >> 2) * 4096 + r * 2048 + tm * 256 + h * 128 + wr * 64, col0 = (P.rank & 3) * 2048 + tn * 256 + wc * 64;
            const unsigned dso = (row0 * DIM + col0) * 2u, vo = (unsigned)(lane >> 3) * (DIM * 2u) + (unsigned)(lane & 7) * 16u;
#pragma unroll
            for (int j = 0; j < 8; ++j) __builtin_amdgcn_raw_buffer_store_b128(c[j], outr, vo, dso + j * (8u * DIM * 2u), AUX_SC1);
            const unsigned o = (xl + h) * 8 + wid;
            publish_block(P, wsr, c, 4096u + o, 1536u + o, 0xFFu & ~(1u << P.rank), call, my, lane);
        }
        STAMP(ws, r * 8 + 4);
        asm volatile("s_waitcnt lgkmcnt(0)" ::: "memory");
        G_BAR;
    }
#undef G_SA
#undef G_SB
#undef G_STAGE
#undef G_LDA
#undef G_LDB
#undef G_MMA
#undef G_WAIT_V
#undef G_WAIT_L
#undef G_BAR
#undef G_SCHED
#undef G_GIDX
}

__global__ __launch_bounds__(512, 2) void gemm_allgather_kernel(Params P) {
    extern __shared__ __attribute__((aligned(16))) unsigned char shm[];
    LAS unsigned char* lds = (LAS unsigned char*)shm;
    char* ws = P.ws[P.rank];
    unsigned ticket = call_issue(ws);
    const int b = blockIdx.x;
    STAMP(ws, 60);
    unsigned call;
    if (b >= NPACK + NGEMM) { call = call_get(ticket); puller_role(P, lds, rfl(call), rfl(b - NPACK - NGEMM)); }
    else {
        const int wid = rfl(threadIdx.x >> 6);
        const bool gemm = b >= NPACK;
        const int next = gemm ? 4096 : NFIRST + b * 8 + wid;
        packer_role(P, lds, ticket, wid < 4 ? b * 4 + wid : next, next, NPACK * 8);
        call = call_get(ticket);
        if (gemm) { __syncthreads(); gemm_role(P, lds, rfl(call), rfl(b)); }
    }
    STAMP(ws, 61);
    STAMP(ws, 58 + (call & 1));
}

}

extern "C" void kernel_launch(int rank, int nranks, ncclComm_t comm,
                              hipStream_t stream,
                              void* const* d_in, const int* in_sizes, int n_in,
                              void* d_out, int out_size,
                              const int* in_dtypes, int out_dtype,
                              void* const* d_out_all,
                              void* const* d_ws_all, size_t ws_bytes) {
    (void)comm; (void)in_sizes; (void)in_dtypes; (void)out_dtype; (void)d_out_all;
    if (nranks != gar::WORLD || n_in < 2 || ws_bytes < gar::WS_NEED || out_size != gar::DIM * gar::DIM || (unsigned)rank >= (unsigned)gar::WORLD) {
        fprintf(stderr, "unexpected problem shape\n");
        return;
    }
    static bool attr = false;
    if (!attr) {
        hipError_t e = hipFuncSetAttribute((const void*)gar::gemm_allgather_kernel, hipFuncAttributeMaxDynamicSharedMemorySize, gar::LDS_BYTES);
        if (e != hipSuccess) { fprintf(stderr, "hipFuncSetAttribute: %s\n", hipGetErrorString(e)); return; }
        attr = true;
    }
    gar::Params P;
    memset(&P, 0, sizeof P);
    P.x = d_in[0]; P.w = d_in[1]; P.out = d_out; P.rank = rank;
    for (int p = 0; p < gar::WORLD; ++p) P.ws[p] = (char*)d_ws_all[p];
    hipLaunchKernelGGL(gar::gemm_allgather_kernel, dim3(gar::GRID), dim3(512), gar::LDS_BYTES, stream, P);
    hipError_t e = hipGetLastError();
    if (e != hipSuccess) fprintf(stderr, "launch: %s\n", hipGetErrorString(e));
}
```

```cpp
#include <hip/hip_runtime.h>
#include <rccl/rccl.h>
#include <cstdio>
#include <cstdint>
#include <cstring>

namespace gar {

#define LAS __attribute__((address_space(3)))
typedef unsigned u32x4 __attribute__((ext_vector_type(4)));
typedef unsigned short u16x2 __attribute__((ext_vector_type(2)));
typedef short bf16x8 __attribute__((ext_vector_type(8)));
typedef float f32x4 __attribute__((ext_vector_type(4)));
typedef __amdgpu_buffer_rsrc_t rsrc_t;

constexpr int WORLD = 8;
constexpr int DIM = 8192;
constexpr int KSH = 1024;
constexpr int NGEMM = 128;
constexpr int GRID = 256;
#define PULL_WG_PER_LINK 12
#define PULL_DECODERS 7
constexpr int PWL = PULL_WG_PER_LINK;
constexpr int NDEC = PULL_DECODERS;
constexpr int PWAVES = 1;
constexpr int NPULL = 7 * PWL;
constexpr int NPACK = GRID - NGEMM - NPULL;
#define NRAW 24
constexpr int NFIRST = (NPACK + NGEMM) * 4;
#define GEMM_PACK_ROUNDS 1
constexpr int GROUNDS = GEMM_PACK_ROUNDS;
static_assert(NFIRST + GROUNDS * NGEMM * 8 <= 4096, "roles");
constexpr int NSEQ = 3584;
constexpr unsigned long long DEADLINE = 40000000ull;

constexpr unsigned OFF_TMO = 128;
constexpr unsigned OFF_LINK = 1024;
constexpr unsigned OFF_CNT = 4096;
constexpr unsigned OFF_XFLAG = OFF_CNT + 256 * 128;
constexpr unsigned OFF_GRAN = 131072;
constexpr unsigned OFF_CALL = 640u << 10;
constexpr unsigned OFF_SLOT = 1u << 20;
constexpr unsigned OFF_AIMG = 64u << 20;
constexpr unsigned OFF_BIMG = 128u << 20;
constexpr unsigned OFF_XBUF = 160u << 20;
constexpr unsigned WS_USED = 192u << 20;
constexpr unsigned WS_NEED = WS_USED;
static_assert(OFF_XFLAG + 256 * 128 <= OFF_GRAN, "map");
static_assert(OFF_GRAN + WORLD * NSEQ * 8 <= OFF_CALL && OFF_CALL + 2048 * 64 <= OFF_SLOT, "map");
static_assert(OFF_SLOT + 6144u * 8192u <= OFF_AIMG, "map");
constexpr int LDS_BYTES = 131072;

struct Params {
    const void* x; const void* w; void* out;
    char* ws[WORLD];
    int rank; int pad;
};

constexpr int AUX_SC1 = 16, AUX_SYS = 17;

__device__ __forceinline__ rsrc_t mk_rsrc(const void* p, unsigned bytes) {
    return __builtin_amdgcn_make_buffer_rsrc(const_cast<void*>(p), 0, bytes, 0x00020000);
}
__device__ __forceinline__ unsigned rfl(unsigned v) { return __builtin_amdgcn_readfirstlane(v); }
__device__ __forceinline__ unsigned bfi(unsigned mask, unsigned a, unsigned b) { return (a & mask) | (b & ~mask); }
__device__ __forceinline__ unsigned perm(unsigned hi, unsigned lo, unsigned sel) { return __builtin_amdgcn_perm(hi, lo, sel); }
#define WAIT_VM0() asm volatile("s_waitcnt vmcnt(0)" ::: "memory")

#define STAMP(ws, idx) do { } while (0)

__device__ __forceinline__ unsigned call_issue(char* ws) {
    unsigned t = 0;
    const int zero = 0, one = 1;
    if ((threadIdx.x & 63) == 0) asm volatile("global_atomic_add %0, %1, %2, %3 sc0" : "=v"(t) : "v"(zero), "v"(one), "s"(ws + OFF_CALL + (blockIdx.x * 8 + rfl(threadIdx.x >> 6)) * 64) : "memory");
    return t;
}
__device__ __forceinline__ unsigned call_get(unsigned& t) {
    asm volatile("s_waitcnt vmcnt(0)" : "+v"(t) :: "memory");
    return rfl(t) + 1u;
}

__device__ __forceinline__ bool give_up(unsigned long long t0, unsigned* tmo, unsigned call) {
    if (rfl(__hip_atomic_load(tmo, __ATOMIC_RELAXED, __HIP_MEMORY_SCOPE_AGENT)) == call) return true;
    if (__builtin_amdgcn_s_memrealtime() - t0 > DEADLINE) { __hip_atomic_store(tmo, call, __ATOMIC_RELAXED, __HIP_MEMORY_SCOPE_AGENT); return true; }
    return false;
}
__device__ __forceinline__ void wait_word(unsigned* p, unsigned target, unsigned* tmo, unsigned call) {
    if ((int)(rfl(__hip_atomic_load(p, __ATOMIC_RELAXED, __HIP_MEMORY_SCOPE_SYSTEM)) - target) >= 0) return;
    const unsigned long long t0 = __builtin_amdgcn_s_memrealtime();
    while ((int)(rfl(__hip_atomic_load(p, __ATOMIC_RELAXED, __HIP_MEMORY_SCOPE_SYSTEM)) - target) < 0) {
        if (give_up(t0, tmo, call)) break;
        __builtin_amdgcn_s_sleep(4);
    }
}

constexpr unsigned MAX_EXC = 31;
constexpr unsigned RAWB = 3584, AUX_U = 20, AUX_MAX = 2560, U_MAX = 10240, LANE_U_MAX = 224;
constexpr unsigned ENC_HIST = AUX_MAX + 16, ENC_SORT = ENC_HIST + 256, ENC_END = ENC_SORT + 128;
constexpr unsigned DEC_POS = 6144;

__device__ __forceinline__ unsigned bfi_c(unsigned mask, unsigned a, unsigned b) { unsigned r; asm("v_bfi_b32 %0, %1, %2, %3" : "=v"(r) : "s"(mask), "v"(a), "v"(b)); return r; }
__device__ __forceinline__ unsigned opaque(unsigned v) { asm("" : "+v"(v)); return v; }
__device__ __forceinline__ unsigned bytemask(unsigned x, int sh, unsigned c0c) {
    unsigned t; asm("v_and_or_b32 %0, %1, %2, %3" : "=v"(t) : "v"(x >> sh), "s"(0x01010101u), "v"(c0c));
    return perm(0u, 0u, t);
}
template <int CTRL, int RM = 0xf> __device__ __forceinline__ unsigned dpp0(unsigned v) { return (unsigned)__builtin_amdgcn_update_dpp(0, (int)v, CTRL, RM, 0xf, RM == 0xf); }
__device__ __forceinline__ unsigned wave_scan_incl(unsigned v) {
    v += dpp0<0x111>(v); v += dpp0<0x112>(v); v += dpp0<0x114>(v); v += dpp0<0x118>(v);
    v += dpp0<0x142, 0xa>(v); v += dpp0<0x143, 0xc>(v);
    return v;
}
__device__ __forceinline__ unsigned umin(unsigned a, unsigned b) { return a < b ? a : b; }
__device__ __forceinline__ unsigned umax(unsigned a, unsigned b) { return a > b ? a : b; }
__device__ __forceinline__ unsigned sad4(unsigned x) { return __builtin_amdgcn_sad_u8(x, 0u, 0u); }
__device__ __forceinline__ unsigned lut2x16(const unsigned (&T)[8], unsigned idx, unsigned ms, unsigned c0c) {
    const unsigned sel = idx & 0x07070707u, m8 = bytemask(idx, 3, c0c);
    const unsigned p = bfi(m8, perm(T[3], T[2], sel), perm(T[1], T[0], sel)), n = bfi(m8, perm(T[7], T[6], sel), perm(T[5], T[4], sel));
    return bfi(ms, n, p);
}
#define LDS_FENCE() asm volatile("" ::: "memory")
__device__ __forceinline__ void lds_or2(LAS unsigned char* scr, unsigned p, unsigned w) {
    const unsigned s = p & 31u;
    LAS unsigned* q = (LAS unsigned*)(scr + ((p >> 5) << 2));
    __hip_atomic_fetch_or(q, w << s, __ATOMIC_RELAXED, __HIP_MEMORY_SCOPE_WORKGROUP);
    __hip_atomic_fetch_or(q + 1, (w >> 1) >> (31u - s), __ATOMIC_RELAXED, __HIP_MEMORY_SCOPE_WORKGROUP);
}

__device__ __forceinline__ unsigned encode_block(const u32x4 (&c)[8], unsigned (&P)[14], u32x4 (&a)[3], LAS unsigned char* scr, int lane) {
    asm volatile("" : "+v"(lane));
    const unsigned c0c = opaque(0x0C0C0C0Cu);
    unsigned E[16], R[16];
    u16x2 p1 = {0, 0}, p2 = {0, 0}, n1 = {0, 0}, n2 = {0, 0};
#pragma unroll
    for (int g = 0; g < 16; ++g) {
        const unsigned x = (g & 1) ? c[g >> 1].z : c[g >> 1].x, y = (g & 1) ? c[g >> 1].w : c[g >> 1].y;
        const unsigned H = perm(y, x, 0x07050301u), L = perm(y, x, 0x06040200u);
        E[g] = bfi_c(0xFEFEFEFEu, H << 1, L >> 7);
        R[g] = bfi_c(0x80808080u, L << 1, bfi_c(0x40404040u, H >> 1, L));
        const unsigned en = E[g] & bytemask(H, 7, c0c), ep = E[g] ^ en;
        p1 = __builtin_elementwise_max(p1, __builtin_bit_cast(u16x2, ep)); p2 = __builtin_elementwise_max(p2, __builtin_bit_cast(u16x2, ep << 8));
        n1 = __builtin_elementwise_max(n1, __builtin_bit_cast(u16x2, en)); n2 = __builtin_elementwise_max(n2, __builtin_bit_cast(u16x2, en << 8));
        __builtin_amdgcn_sched_barrier(0);
    }
    const u32x4 z4 = {0u, 0u, 0u, 0u};
    *(LAS u32x4*)(scr + lane * 16) = z4; *(LAS u32x4*)(scr + 1024 + lane * 16) = z4;
    if (lane < 49) *(LAS u32x4*)(scr + 2048 + lane * 16) = z4;
    unsigned baseP, baseN;
    {
        const u16x2 mp = __builtin_elementwise_max(p1, p2), mn = __builtin_elementwise_max(n1, n2);
        u16x2 m = {(unsigned short)((mp.x > mp.y ? mp.x : mp.y) >> 8), (unsigned short)((mn.x > mn.y ? mn.x : mn.y) >> 8)};
        m = __builtin_elementwise_max(m, __builtin_bit_cast(u16x2, dpp0<0x128>(__builtin_bit_cast(unsigned, m)))); m = __builtin_elementwise_max(m, __builtin_bit_cast(u16x2, dpp0<0x124>(__builtin_bit_cast(unsigned, m))));
        m = __builtin_elementwise_max(m, __builtin_bit_cast(u16x2, dpp0<0x122>(__builtin_bit_cast(unsigned, m)))); m = __builtin_elementwise_max(m, __builtin_bit_cast(u16x2, dpp0<0x121>(__builtin_bit_cast(unsigned, m))));
        const unsigned mu = __builtin_bit_cast(unsigned, m);
        const u16x2 r01 = __builtin_elementwise_max(__builtin_bit_cast(u16x2, (unsigned)__builtin_amdgcn_readlane(mu, 0)), __builtin_bit_cast(u16x2, (unsigned)__builtin_amdgcn_readlane(mu, 16)));
        const u16x2 r23 = __builtin_elementwise_max(__builtin_bit_cast(u16x2, (unsigned)__builtin_amdgcn_readlane(mu, 32)), __builtin_bit_cast(u16x2, (unsigned)__builtin_amdgcn_readlane(mu, 48)));
        const unsigned mm = rfl(__builtin_bit_cast(unsigned, __builtin_elementwise_max(r01, r23)));
        baseP = mm & 0xFFu; baseN = mm >> 16;
    }
    const unsigned BP = baseP * 0x01010101u, BN = baseN * 0x01010101u;
    unsigned any = 0, D[16];
#pragma unroll
    for (int g = 0; g < 16; ++g) {
        E[g] = bfi(bytemask(R[g], 6, c0c), BN, BP) - E[g]; any |= E[g];
        D[g] = bfi_c(0x1E1E1E1Eu, E[g] << 1, (R[g] >> 7) & 0x01010101u);
        R[g] &= 0x7F7F7F7Fu;
    }
    LDS_FENCE();
#pragma unroll
    for (int gi = 0; gi < 4; ++gi) {
        const int g = gi * 5;
        const unsigned k4 = ((D[g] & 0x0F0F0F0Fu) | ((R[g] >> 2) & 0x10101010u) | ((D[g] << 1) & 0x20202020u)) << 2;
#pragma unroll
        for (int b = 0; b < 4; ++b)
            __hip_atomic_fetch_add((LAS unsigned*)(scr + ENC_HIST + ((k4 >> (8 * b)) & 0xFFu)), 1u, __ATOMIC_RELAXED, __HIP_MEMORY_SCOPE_WORKGROUP);
    }
    LDS_FENCE();
    const unsigned cnt = *(LAS unsigned*)(scr + ENC_HIST + lane * 4);
    const unsigned key = (cnt << 4) | (15u - (lane & 15));
    unsigned rank = 0;
    rank += dpp0<0x121>(key) > key; rank += dpp0<0x122>(key) > key; rank += dpp0<0x123>(key) > key; rank += dpp0<0x124>(key) > key; rank += dpp0<0x125>(key) > key;
    rank += dpp0<0x126>(key) > key; rank += dpp0<0x127>(key) > key; rank += dpp0<0x128>(key) > key; rank += dpp0<0x129>(key) > key; rank += dpp0<0x12A>(key) > key;
    rank += dpp0<0x12B>(key) > key; rank += dpp0<0x12C>(key) > key; rank += dpp0<0x12D>(key) > key; rank += dpp0<0x12E>(key) > key; rank += dpp0<0x12F>(key) > key;
    unsigned tail = cnt; tail += dpp0<0x128>(tail); tail += dpp0<0x124>(tail); tail += dpp0<0x122>(tail); tail += dpp0<0x121>(tail);
    const int sg = (lane >> 4) & 1;
    const unsigned ct = sg ? __builtin_amdgcn_readlane(tail, 48) : __builtin_amdgcn_readlane(tail, 32);
    if (lane < 32) *(LAS unsigned*)(scr + ENC_SORT + ((lane & 16) + rank) * 4) = cnt;
    LDS_FENCE();
    unsigned best;
    {
        const unsigned cb = lane & 15, s0 = cb / 3u, s1 = cb - 3u * s0, A = 1u << s0, AB = A + (1u << s1);
        unsigned cost = ct * (16u - AB);
#pragma unroll
        for (int q = 0; q < 4; ++q) {
            const u32x4 h = *(LAS u32x4*)(scr + ENC_SORT + sg * 64 + q * 16);
#pragma unroll
            for (int j = 0; j < 4; ++j) {
                const unsigned r = q * 4 + j, len = r < A ? 1u + s0 : r < AB ? 2u + s1 : 4u + ((r - AB) >> 1);
                cost += 2u * len * h[j];
            }
        }
        best = cb < 9 ? (cost << 4) | cb : 0xFFFFFFFFu;
        best = umin(best, dpp0<0x128>(best)); best = umin(best, dpp0<0x124>(best)); best = umin(best, dpp0<0x122>(best)); best = umin(best, dpp0<0x121>(best));
    }
    const unsigned cbP = __builtin_amdgcn_readlane(best, 0) & 15u, cbN = __builtin_amdgcn_readlane(best, 16) & 15u;
    const unsigned s0P = cbP / 3u, s1P = cbP - 3u * s0P, s0N = cbN / 3u, s1N = cbN - 3u * s0N;
    const unsigned ABP = (1u << s0P) + (1u << s1P), ABN = (1u << s0N) + (1u << s1N);
    unsigned F[8];
    {
        const unsigned s0 = sg ? s0N : s0P, s1 = sg ? s1N : s1P, A = 1u << s0, AB = sg ? ABN : ABP;
        const unsigned ent = rank < A ? (rank << 4) | (s0 << 6) : rank < AB ? 1u | ((rank - A) << 4) | (s1 << 6) : (2u + ((rank - AB) >> 1)) | (((rank - AB) & 1u) << 4) | (1u << 6);
        unsigned fv = ent << (8 * (lane & 3));
        fv |= dpp0<0xB1>(fv); fv |= dpp0<0x4E>(fv);
#pragma unroll
        for (int q = 0; q < 8; ++q) F[q] = __builtin_amdgcn_readlane(fv, q * 4);
        if (lane < 32)
            __hip_atomic_fetch_or((LAS unsigned*)(scr + (sg * 2 + (rank >> 3)) * 4), (unsigned)(lane & 15) << (8 * (rank & 3) + 4 * ((rank >> 2) & 1)), __ATOMIC_RELAXED, __HIP_MEMORY_SCOPE_WORKGROUP);
        if (lane == 0) *(LAS unsigned*)(scr + 16) = baseP | (baseN << 8) | (s0P << 16) | (s1P << 18) | (s0N << 20) | (s1N << 22);
    }
    const unsigned ABP4 = ABP * 0x01010101u, ABN4 = ABN * 0x01010101u, S0P4 = s0P * 0x01010101u, S0N4 = s0N * 0x01010101u;
    unsigned KK[16], XW[4], OX[4], TU = 0, TX = 0, bad = 0, xl = 0, xw = 0;
#pragma unroll
    for (int g = 0; g < 16; ++g) {
        const unsigned ms = bytemask(R[g], 6, c0c), m16 = bytemask(D[g], 4, c0c), ent = lut2x16(F, D[g], ms, c0c);
        const unsigned q = ((D[g] | 0x80808080u) - bfi(ms, ABN4, ABP4)) & 0x7F7F7F7Fu;
        unsigned K = bfi(m16, ((q >> 1) & 0x3F3F3F3Fu) + 0x02020202u, ent & 0x0F0F0F0Fu);
        unsigned SUF = bfi(m16, q & 0x01010101u, (ent >> 4) & 0x03030303u);
        unsigned S = bfi(m16, 0x01010101u, (ent >> 6) & 0x03030303u);
        unsigned gl = sad4(K) + 4u;
        if (__ballot(gl > 31u) != 0ull) {
            any |= 0x80u;
            const unsigned km = umax(umax(K & 0xFFu, (K >> 8) & 0xFFu), umax((K >> 16) & 0xFFu, K >> 24));
            const unsigned eq = gl > 31u ? (((K ^ (km * 0x01010101u)) + 0x7F7F7F7Fu) & 0x80808080u) ^ 0x80808080u : 0u, em = bytemask(eq, 7, c0c);
            E[g] |= eq;
            K &= ~em; SUF &= ~em; S = bfi(em, bfi(ms, S0N4, S0P4), S);
            gl = sad4(K) + 4u;
        }
        KK[g] = K; TU += gl; bad = umax(bad, gl);
        xw |= (SUF & 0xFFu) << xl; xl += S & 0xFFu;
        xw |= ((SUF >> 8) & 0xFFu) << xl; xl += (S >> 8) & 0xFFu;
        xw |= ((SUF >> 16) & 0xFFu) << xl; xl += (S >> 16) & 0xFFu;
        xw |= (SUF >> 24) << (xl & 31u); xl += S >> 24;
        if ((g & 3) == 3) { XW[g >> 2] = xw; OX[g >> 2] = TX; TX += xl; xw = 0; xl = 0; }
        __builtin_amdgcn_sched_barrier(0);
    }
    bad = (bad > 31u) | (TU > LANE_U_MAX);
    const unsigned tt = TU | (TX << 16), incl = wave_scan_incl(tt), tot = __builtin_amdgcn_readlane(incl, 63), totU = tot & 0xFFFFu, totX = tot >> 16;
    const unsigned xstart = (AUX_U * 8u + totU + 31u) & ~31u, endbit = xstart + totX;
    unsigned raw = (__ballot(bad != 0) != 0ull) || totU > U_MAX || endbit > (AUX_MAX - 8u) * 8u;
    if (!raw) {
        const unsigned px = xstart + ((incl - tt) >> 16);
        unsigned p = AUX_U * 8u + ((incl - tt) & 0xFFFFu);
#pragma unroll
        for (int g = 0; g < 16; ++g) {
            const unsigned ln = KK[g] + 0x01010101u;
            unsigned pos = (ln & 0xFFu) - 1u, w = 1u << pos;
            pos += (ln >> 8) & 0xFFu; w |= 1u << pos;
            pos += (ln >> 16) & 0xFFu; w |= 1u << pos;
            pos += ln >> 24; w |= 1u << pos;
            lds_or2(scr, p, w); p += pos + 1u;
            __builtin_amdgcn_sched_barrier(0);
        }
#pragma unroll
        for (int q = 0; q < 4; ++q) lds_or2(scr, px + OX[q], XW[q]);
    }
#pragma unroll
    for (int i = 0; i < 7; ++i) { P[i] = R[i] | ((R[14] << (7 - i)) & 0x80808080u); P[7 + i] = R[7 + i] | ((R[15] << (7 - i)) & 0x80808080u); }
    unsigned n = 0;
    const unsigned xo = (endbit + 31u) >> 5;
    if (!raw && __ballot((any & 0xF0F0F0F0u) != 0) != 0ull) {
#pragma unroll
        for (int g = 0; g < 16; ++g) {
            const unsigned f = E[g] & 0xF0F0F0F0u;
            unsigned long long mask = __ballot(f != 0);
            while (mask != 0ull && n <= MAX_EXC) {
                const int L = __builtin_ctzll(mask); mask &= mask - 1;
                const unsigned fl = __builtin_amdgcn_readlane(f, L);
                const unsigned lo = __builtin_amdgcn_readlane((g & 1) ? c[g >> 1].z : c[g >> 1].x, L);
                const unsigned hi = __builtin_amdgcn_readlane((g & 1) ? c[g >> 1].w : c[g >> 1].y, L);
                const bool e = lane < 4 && ((fl >> (8 * (lane & 3))) & 0xFFu) != 0;
                const unsigned long long em = __ballot(e);
                const unsigned slot = xo + n + __builtin_popcountll(em & ((1ull << lane) - 1ull));
                const unsigned val = (((lane & 2) ? hi : lo) >> (16 * (lane & 1))) & 0xFFFFu;
                if (e && slot * 4u < AUX_MAX) *(LAS unsigned*)(scr + slot * 4u) = (unsigned)(L * 64 + g * 4 + (lane & 3)) | (val << 12);
                n += __builtin_popcountll(em);
            }
        }
    }
    raw |= (n > MAX_EXC) | ((xo + n) * 4u > AUX_MAX);
    LDS_FENCE();
    a[0] = *(LAS u32x4*)(scr + lane * 16); a[1] = *(LAS u32x4*)(scr + 1024 + lane * 16); a[2] = *(LAS u32x4*)(scr + 2048 + (lane & 31) * 16);
    return ((xo + n + 3u) >> 2) | (n << 8) | (xo << 16) | (raw << 31);
}

__device__ __forceinline__ void decode_block(LAS unsigned char* src, u32x4 (&c)[8], int lane) {
    LAS unsigned char* aux = src + RAWB;
    const unsigned c0c = opaque(0x0C0C0C0Cu);
    const u32x4 nt = *(LAS const u32x4*)(aux);
    const unsigned prm_v = *(LAS const unsigned*)(aux + 16);
    unsigned u[5];
#pragma unroll
    for (int i = 0; i < 5; ++i) u[i] = *(LAS const unsigned*)(aux + AUX_U + lane * 20 + i * 4);
    u32x4 P4[3];
#pragma unroll
    for (int p = 0; p < 3; ++p) P4[p] = *(LAS const u32x4*)(src + p * 1024 + lane * 16);
    const unsigned long long ph = *(LAS const unsigned long long*)(src + 3072 + lane * 8);
    const unsigned c0 = __builtin_popcount(u[0]), c1 = c0 + __builtin_popcount(u[1]), c2 = c1 + __builtin_popcount(u[2]), c3 = c2 + __builtin_popcount(u[3]), cn = c3 + __builtin_popcount(u[4]);
    const unsigned C = wave_scan_incl(cn) - cn;
    if (lane == 0) *(LAS unsigned*)(src + DEC_POS) = AUX_U * 8u;
    const int rounds = __ballot(cn > 128u) != 0ull ? 3 : 2;
    for (int j = 0; j < rounds; ++j) {
        const unsigned t = (C >> 6) + 1u + j, kth = 64u * t - C;
        const unsigned i = (kth > c0) + (kth > c1) + (kth > c2) + (kth > c3);
        unsigned rem = kth - (i == 0 ? 0u : i == 1 ? c0 : i == 2 ? c1 : i == 3 ? c2 : c3), w = i == 0 ? u[0] : i == 1 ? u[1] : i == 2 ? u[2] : i == 3 ? u[3] : u[4], pos = 0;
#pragma unroll
        for (int st = 16; st >= 1; st >>= 1) {
            const unsigned lo = __builtin_popcount(w & ((1u << st) - 1u));
            const bool up = rem > lo;
            rem -= up ? lo : 0u; w = up ? w >> st : w; pos += up ? st : 0;
        }
        if (kth <= cn && t < 64u) *(LAS unsigned*)(src + DEC_POS + t * 4) = AUX_U * 8u + lane * 160u + i * 32u + pos + 1u;
    }
    LDS_FENCE();
    const unsigned p0 = *(LAS const unsigned*)(src + DEC_POS + lane * 4);
    unsigned S[8];
    {
        LAS const unsigned* q = (LAS const unsigned*)(aux + ((p0 >> 5) << 2));
#pragma unroll
        for (int i = 0; i < 8; ++i) S[i] = q[i];
#pragma unroll
        for (int i = 0; i < 7; ++i) S[i] = __builtin_amdgcn_alignbit(S[i + 1], S[i], p0 & 31u);
    }
    unsigned R[16];
    {
        const unsigned P[14] = {P4[0].x, P4[0].y, P4[0].z, P4[0].w, P4[1].x, P4[1].y, P4[1].z, P4[1].w, P4[2].x, P4[2].y, P4[2].z, P4[2].w, (unsigned)ph, (unsigned)(ph >> 32)};
        unsigned x = 0, y = 0;
#pragma unroll
        for (int i = 0; i < 7; ++i) { x |= (P[i] >> (7 - i)) & (0x01010101u << i); y |= (P[7 + i] >> (7 - i)) & (0x01010101u << i); }
#pragma unroll
        for (int i = 0; i < 14; ++i) R[i] = P[i] & 0x7F7F7F7Fu;
        R[14] = x; R[15] = y;
    }
    const unsigned n0 = rfl(nt.x), n1 = rfl(nt.y), n2 = rfl(nt.z), n3 = rfl(nt.w), prm = rfl(prm_v);
    const unsigned I[8] = {n0 & 0x0F0F0F0Fu, (n0 >> 4) & 0x0F0F0F0Fu, n1 & 0x0F0F0F0Fu, (n1 >> 4) & 0x0F0F0F0Fu, n2 & 0x0F0F0F0Fu, (n2 >> 4) & 0x0F0F0F0Fu, n3 & 0x0F0F0F0Fu, (n3 >> 4) & 0x0F0F0F0Fu};
    const unsigned s0P = (prm >> 16) & 3u, s1P = (prm >> 18) & 3u, s0N = (prm >> 20) & 3u, s1N = (prm >> 22) & 3u;
    const unsigned STP = (s0P << 8) | (s1P << 16) | (1u << 24), STN = (s0N << 8) | (s1N << 16) | (1u << 24);
    const unsigned OTP = (4u << 8) | (((1u << s0P) + 2u) << 16) | (((1u << s0P) + (1u << s1P)) << 24);
    const unsigned OTN = (4u << 8) | (((1u << s0N) + 2u) << 16) | (((1u << s0N) + (1u << s1N)) << 24);
    const unsigned BP = (prm & 0xFFu) * 0x01010101u, BN = ((prm >> 8) & 0xFFu) * 0x01010101u;
    unsigned RB[16], SS[16], TU = 0, TX = 0;
#pragma unroll
    for (int g = 0; g < 16; ++g) {
        const unsigned w0 = S[0], q0 = __builtin_ctz(w0), w1 = w0 & (w0 - 1u), q1 = __builtin_ctz(w1), w2 = w1 & (w1 - 1u), q2 = __builtin_ctz(w2), w3 = w2 & (w2 - 1u), q3 = __builtin_ctz(w3);
        const unsigned Q = q0 | (q1 << 8) | (q2 << 16) | (q3 << 24);
        const unsigned LN = Q - opaque((Q << 8) - 1u), sh = q3 + 1u;
        TU += sh;
        if (g < 15) {
#pragma unroll
            for (int i = 0; i < 7; ++i) if (32 * i < (int)LANE_U_MAX - 4 * (g + 1)) S[i] = (i < 6 && 32 * (i + 1) < (int)LANE_U_MAX - 4 * g) ? __builtin_amdgcn_alignbit(S[i + 1], S[i], sh) : S[i] >> sh;
        }
        const unsigned sel = bfi_c(0x04040404u, R[g] >> 4, bfi(bytemask(LN + 0x7D7D7D7Du, 7, c0c), 0x03030303u, LN));
        SS[g] = perm(STN, STP, sel);
        RB[g] = perm(OTN, OTP, sel) + (LN << 1);
        TX = __builtin_amdgcn_sad_u8(SS[g], 0u, TX);
    }
    const unsigned xstart = (__builtin_amdgcn_readlane(p0 + TU, 63) + 31u) & ~31u, px = xstart + wave_scan_incl(TX) - TX;
    unsigned X[5];
    {
        LAS const unsigned* q = (LAS const unsigned*)(aux + ((px >> 5) << 2));
#pragma unroll
        for (int i = 0; i < 5; ++i) X[i] = q[i];
#pragma unroll
        for (int i = 0; i < 4; ++i) X[i] = __builtin_amdgcn_alignbit(X[i + 1], X[i], px & 31u);
    }
    unsigned xoff = 0;
#pragma unroll
    for (int g = 0; g < 16; ++g) {
        const unsigned s = SS[g], M = perm(0u, 0x00030100u, s);
        const unsigned y = (g & 1) ? X[0] >> xoff : X[0], Y = perm(y, y, 0x01000100u);
        const unsigned t1 = opaque(s << 8), t2 = opaque((t1 << 8) + t1), Cs = (t2 << 16) + t2;
        const u16x2 z02 = __builtin_bit_cast(u16x2, Y) >> __builtin_bit_cast(u16x2, Cs & 0x00FF00FFu);
        const u16x2 z13 = __builtin_bit_cast(u16x2, Y) >> __builtin_bit_cast(u16x2, perm(0u, Cs, 0x0C030C01u));
        const unsigned suf = perm(__builtin_bit_cast(unsigned, z13), __builtin_bit_cast(unsigned, z02), 0x06020400u) & M;
        if (g & 1) {
            if (g < 15) {
                const unsigned t = __builtin_amdgcn_sad_u8(s, 0u, xoff);
#pragma unroll
                for (int i = 0; i < 4; ++i) if (32 * i < 8 * (15 - g)) X[i] = (i < 3 && 32 * (i + 1) < 8 * (17 - g)) ? __builtin_amdgcn_alignbit(X[i + 1], X[i], t) : X[i] >> t;
            }
        } else xoff = sad4(s);
        const unsigned rank = RB[g] + suf - 0x06060606u, ms = bytemask(R[g], 6, c0c);
        const unsigned D = bfi(bytemask(rank, 4, c0c), rank, lut2x16(I, rank, ms, c0c));
        const unsigned E = bfi(ms, BN, BP) - ((D >> 1) & 0x0F0F0F0Fu);
        const unsigned H = bfi_c(0x80808080u, R[g] << 1, E >> 1), L = bfi_c(0x3F3F3F3Fu, R[g], bfi_c(0x40404040u, D << 6, E << 7));
        const unsigned x = perm(H, L, 0x05010400u), yv = perm(H, L, 0x07030602u);
        if (g & 1) { c[g >> 1].z = x; c[g >> 1].w = yv; } else { c[g >> 1].x = x; c[g >> 1].y = yv; }
    }
}

typedef unsigned u32x2 __attribute__((ext_vector_type(2)));
__device__ __forceinline__ void store_granule(char* p, u32x2 g) {
    asm volatile("global_store_dwordx2 %0, %1, off sc0 sc1\n\ts_nop 1" :: "v"(p), "v"(g) : "memory");
}

__device__ __forceinline__ void publish_block(const Params& P, rsrc_t wsr, const u32x4 (&c)[8], unsigned slot, unsigned n, unsigned dmask, unsigned call, LAS unsigned char* scr, int lane) {
    unsigned R[14], h = 1u << 31; u32x4 a[3];
    if (n >= NRAW) h = encode_block(c, R, a, scr, lane);
    const unsigned so = OFF_SLOT + slot * 8192u;
    if (h >> 31) {
#pragma unroll
        for (int j = 0; j < 8; ++j) __builtin_amdgcn_raw_buffer_store_b128(c[j], wsr, lane * 16 + j * 1024, so, AUX_SYS);
    } else {
        const unsigned n16 = h & 255u;
#pragma unroll
        for (int j = 0; j < 3; ++j) __builtin_amdgcn_raw_buffer_store_b128((u32x4){R[4 * j], R[4 * j + 1], R[4 * j + 2], R[4 * j + 3]}, wsr, lane * 16 + j * 1024, so, AUX_SYS);
        __builtin_amdgcn_raw_buffer_store_b64((u32x2){R[12], R[13]}, wsr, lane * 8 + 3072, so, AUX_SYS);
        if ((unsigned)lane < n16) __builtin_amdgcn_raw_buffer_store_b128(a[0], wsr, lane * 16 + RAWB, so, AUX_SYS);
        if ((unsigned)lane + 64u < n16) __builtin_amdgcn_raw_buffer_store_b128(a[1], wsr, lane * 16 + RAWB + 1024, so, AUX_SYS);
        if ((unsigned)lane + 128u < n16) __builtin_amdgcn_raw_buffer_store_b128(a[2], wsr, lane * 16 + RAWB + 2048, so, AUX_SYS);
    }
    WAIT_VM0();
    if (lane < WORLD && ((dmask >> lane) & 1u)) {
        char* p = P.ws[lane & 7] + OFF_GRAN + ((unsigned)P.rank * NSEQ + n) * 8u;
        store_granule(p, (u32x2){call, h});
    }
}

__device__ __forceinline__ void packer_role(const Params& P, LAS unsigned char* lds, unsigned& ticket, int first, int next, int step, int end) {
    const int lane = threadIdx.x & 63, wid = rfl(threadIdx.x >> 6);
    char* ws = P.ws[P.rank];
    const rsrc_t wsr = mk_rsrc(ws, WS_USED);
    const rsrc_t xr = mk_rsrc(P.x, DIM * KSH * 2), wr = mk_rsrc(P.w, DIM * KSH * 2);
    const int myi = P.rank >> 2, myj = P.rank & 3;
    LAS unsigned char* my = lds + wid * 8192;
    for (int p = first; p < end; p = p < NFIRST ? next : p + step) {
        bool is_w; int kk, sel, r, bi;
        if (p < 3072) {
            kk = p / 192; const int rem = p - kk * 192;
            bi = rem & 31; r = 0; is_w = rem >= 64; sel = is_w ? ((rem - 64) >> 5) : (rem >> 5);
        }
        else { const int q = p - 3072; kk = q >> 6; sel = (q >> 5) & 1; bi = q & 31; r = 1; is_w = false; }
        u32x4 c[8];
        if (!is_w) {
            const int rr = lane >> 3;
            const unsigned vo = (unsigned)rr * 2048u + (unsigned)(((lane & 7) ^ rr) * 16);
            const unsigned so = (unsigned)(sel * 4096 + r * 2048 + bi * 64) * 2048u + (unsigned)kk * 128u;
#pragma unroll
            for (int j = 0; j < 8; ++j) c[j] = __builtin_amdgcn_raw_buffer_load_b128(xr, vo + j * 8 * 2048, so, 0);
        } else {
            const int pn = bi >> 2, bj = (bi >> 1) & 1, q = bi & 1;
            const int gk = lane >> 3, cc = lane & 7, wcl = cc >> 2, cq = cc & 3;
            const unsigned vo = (unsigned)(gk * 8) * 16384u + (unsigned)((64 * (2 * q + wcl) + 32 * bj + 8 * cq) * 2);
            const unsigned so = (unsigned)(kk * 64) * 16384u + (unsigned)((sel * 2048 + pn * 256) * 2);
            u32x4 R[8];
#pragma unroll
            for (int jj = 0; jj < 8; ++jj) R[jj] = __builtin_amdgcn_raw_buffer_load_b128(wr, vo + jj * 16384, so, 0);
#pragma unroll
            for (int j = 0; j < 8; ++j) {
                const unsigned s = (j & 1) ? 0x07060302u : 0x05040100u;
                u32x4 t;
                t.x = perm(R[1][j >> 1], R[0][j >> 1], s); t.y = perm(R[3][j >> 1], R[2][j >> 1], s);
                t.z = perm(R[5][j >> 1], R[4][j >> 1], s); t.w = perm(R[7][j >> 1], R[6][j >> 1], s);
                const int s64 = 32 * wcl + 16 * (j >> 2) + 4 * cq + (j & 3);
                *(LAS u32x4*)(my + s64 * 128 + ((gk ^ (s64 & 7)) * 16)) = t;
            }
            asm volatile("s_waitcnt lgkmcnt(0)" ::: "memory");
#pragma unroll
            for (int j = 0; j < 8; ++j) c[j] = *(LAS u32x4*)(my + j * 1024 + lane * 16);
            asm volatile("s_waitcnt lgkmcnt(0)" ::: "memory");
        }
        const unsigned call = call_get(ticket);
        const int g = P.rank * 16 + kk;
        const bool self = is_w ? (sel == myj) : (sel == myi);
        if (self) {
            const unsigned so = is_w ? OFF_BIMG + (unsigned)(g * 32 + bi) * 8192u : OFF_AIMG + (unsigned)((r * 128 + g) * 32 + bi) * 8192u;
#pragma unroll
            for (int j = 0; j < 8; ++j) __builtin_amdgcn_raw_buffer_store_b128(c[j], wsr, lane * 16 + j * 1024, so, AUX_SC1);
        }
        const unsigned slot = is_w ? 2048u + (unsigned)((sel * 16 + kk) * 32 + bi) : (unsigned)(((sel * 2 + r) * 16 + kk) * 32 + bi);
        const unsigned n = is_w ? (unsigned)(kk * 64 + 32 + bi) : (r == 0 ? (unsigned)(kk * 64 + bi) : (unsigned)(1024 + kk * 32 + bi));
        unsigned dmask = is_w ? (0x11u << sel) : (0x0Fu << (4 * sel));
        dmask &= ~(1u << P.rank);
        publish_block(P, wsr, c, slot, n, dmask, call, my, lane);
        if (self && lane == 0)
            __hip_atomic_fetch_add((unsigned*)(ws + OFF_CNT + (r * 128 + g) * 128), 1u, __ATOMIC_RELAXED, __HIP_MEMORY_SCOPE_SYSTEM);
    }
}

__device__ __forceinline__ u32x2 load_granule(rsrc_t wsr, unsigned off) { return __builtin_amdgcn_raw_buffer_load_b64(wsr, 0, off, AUX_SYS); }

struct Blk { unsigned slot, dso, cnt; bool is_out; };
__device__ __forceinline__ Blk locate(unsigned n, int src, int myi, int myj) {
    Blk b; b.is_out = false;
    if (n < 1024) { const int kk = n >> 6, bi = n & 31, g = src * 16 + kk; b.cnt = OFF_CNT + g * 128;
        if ((n & 63) >= 32) { b.slot = 2048u + (myj * 16 + kk) * 32 + bi; b.dso = OFF_BIMG + (unsigned)(g * 32 + bi) * 8192u; }
        else { b.slot = ((myi * 2) * 16 + kk) * 32 + bi; b.dso = OFF_AIMG + (unsigned)(g * 32 + bi) * 8192u; } }
    else if (n < 1536) { const int kk = (n - 1024) >> 5, bi = n & 31, g = src * 16 + kk; b.cnt = OFF_CNT + (128 + g) * 128;
        b.slot = ((myi * 2 + 1) * 16 + kk) * 32 + bi; b.dso = OFF_AIMG + (unsigned)((128 + g) * 32 + bi) * 8192u; }
    else { const int o = n - 1536, w = o & 7, h = (o >> 3) & 1, tile = (o >> 4) & 63, r = o >> 10; b.cnt = 0; b.is_out = true;
        b.slot = 4096u + o;
        const unsigned row0 = (src >> 2) * 4096 + r * 2048 + (tile >> 3) * 256 + h * 128 + (w >> 2) * 64, col0 = (src & 3) * 2048 + (tile & 7) * 256 + (w & 3) * 64;
        b.dso = (row0 * DIM + col0) * 2u; }
    return b;
}
__device__ __forceinline__ void signal_cnt(char* ws, unsigned off, int lane) {
    if (off && lane == 0) __hip_atomic_fetch_add((unsigned*)(ws + off), 1u, __ATOMIC_RELAXED, __HIP_MEMORY_SCOPE_SYSTEM);
}
constexpr unsigned NSLOT = 15, CTL = NSLOT * 8192;
constexpr unsigned M_DEAD = 1u << 30, N_STOP = 0xFFFFFFFFu;
__device__ __forceinline__ unsigned lds_ld(LAS unsigned char* p) { return *(volatile LAS unsigned*)p; }
__device__ __forceinline__ void lds_st(LAS unsigned char* p, unsigned v) { *(volatile LAS unsigned*)p = v; }
__device__ __forceinline__ void ring_publish(LAS unsigned char* lds, unsigned i, unsigned n, unsigned h, int lane) {
    const unsigned sl = i % NSLOT;
    if (lane == 0) { *(volatile LAS u32x2*)(lds + CTL + sl * 8) = (u32x2){n, h}; lds_st(lds + CTL + 256 + sl * 4, i + 1u); }
}
__device__ __forceinline__ void dma_op(rsrc_t r, LAS unsigned char* dst, int j, unsigned so, int lane) {
    __builtin_amdgcn_raw_ptr_buffer_load_lds(r, (LAS void*)(dst + j * 1024), 16, lane * 16 + j * 1024, so, 0, AUX_SYS);
}
#define VM_WAIT_(n) asm volatile("s_waitcnt vmcnt(" #n ")" ::: "memory")
#define VM_WAIT(n) VM_WAIT_(n)
#define WIN 5

__device__ __forceinline__ void loader_wave(const Params& P, LAS unsigned char* lds, unsigned call, int src, int lane) {
    char* ws = P.ws[P.rank];
    unsigned* tmo = (unsigned*)(ws + OFF_TMO);
    const rsrc_t wsr = mk_rsrc(ws, WS_USED);
    char* const remp = P.ws[src];
    const int myi = P.rank >> 2, myj = P.rank & 3;
    const unsigned gbase = OFF_GRAN + (unsigned)src * NSEQ * 8u;
    char* const gptr = ws + gbase;
    const unsigned tko = OFF_LINK + (unsigned)src * 128u, tbase = (call - 1u) * (NSEQ + 3u * PWL);
    __builtin_amdgcn_s_setprio(3);
    unsigned n, n1, n2;
    {
        unsigned t0 = 0;
        if (lane == 0) t0 = __hip_atomic_fetch_add((unsigned*)(ws + tko), 3u, __ATOMIC_RELAXED, __HIP_MEMORY_SCOPE_AGENT);
        n = rfl(t0) - tbase; n1 = n + 1; n2 = n + 2;
    }
    unsigned gx, gy, hx, hy;
    { const u32x2 a = load_granule(wsr, gbase + (n < NSEQ ? n : 0u) * 8u), b = load_granule(wsr, gbase + (n1 < NSEQ ? n1 : 0u) * 8u);
      gx = rfl(a.x); gy = rfl(a.y); hx = rfl(b.x); hy = rfl(b.y); }
    const int zero = 0, one = 1;
    unsigned i = 0;
    bool pend = false; unsigned pn = 0, ph = 0;
    while (n < NSEQ) {
        const unsigned sl = i % NSLOT;
        bool dead = false;
        const bool need_gran = (int)(gx - call) < 0;
        const bool need_slot = i >= NSLOT && (int)(lds_ld(lds + CTL + 320 + sl * 4) - (i - NSLOT + 1u)) < 0;
        if (need_gran || need_slot) {
            if (pend) { VM_WAIT(0); ring_publish(lds, i - 1u, pn, ph, lane); pend = false; }
            const unsigned long long t0 = __builtin_amdgcn_s_memrealtime();
            while ((int)(gx - call) < 0) {
                const u32x2 a = load_granule(wsr, gbase + n * 8u);
                gx = rfl(a.x); gy = rfl(a.y);
                if ((int)(gx - call) >= 0) break;
                if (give_up(t0, tmo, call)) { dead = true; break; }
                __builtin_amdgcn_s_sleep(2);
                asm volatile("" ::: "memory");
            }
            while (i >= NSLOT && (int)(lds_ld(lds + CTL + 320 + sl * 4) - (i - NSLOT + 1u)) < 0) __builtin_amdgcn_s_sleep(1);
        }
        const bool raw = (gy >> 31) != 0;
        const unsigned so = OFF_SLOT + locate(n, src, myi, myj).slot * 8192u;
        const unsigned bytes = dead ? 0u : raw ? 8192u : RAWB + ((gy & 255u) << 4);
        const rsrc_t rem = mk_rsrc(remp, so + bytes);
        LAS unsigned char* dst = lds + sl * 8192;
        u32x2 g2; unsigned tk = 0;
        asm volatile("global_load_dwordx2 %0, %1, %2 sc0 sc1" : "=v"(g2) : "v"(zero), "s"(gptr + (n2 < NSEQ ? n2 : 0u) * 8u) : "memory");
        if (lane == 0) asm volatile("global_atomic_add %0, %1, %2, %3 sc0" : "=v"(tk) : "v"(zero), "v"(one), "s"(ws + tko) : "memory");
        VM_WAIT(WIN); dma_op(rem, dst, 0, so, lane);
        VM_WAIT(WIN); dma_op(rem, dst, 1, so, lane);
        VM_WAIT(WIN); dma_op(rem, dst, 2, so, lane);
        VM_WAIT(WIN); if (pend) ring_publish(lds, i - 1u, pn, ph, lane); dma_op(rem, dst, 3, so, lane);
        VM_WAIT(WIN); dma_op(rem, dst, 4, so, lane);
        asm volatile("s_waitcnt vmcnt(5)" : "+v"(g2), "+v"(tk) :: "memory"); dma_op(rem, dst, 5, so, lane);
        if (raw) { dma_op(rem, dst, 6, so, lane); dma_op(rem, dst, 7, so, lane); VM_WAIT(0); }
        pend = true; pn = n; ph = gy | (dead ? M_DEAD : 0u);
        n = n1; gx = hx; gy = hy;
        n1 = n2; hx = rfl(g2.x); hy = rfl(g2.y);
        n2 = rfl(tk) - tbase;
        ++i;
    }
    VM_WAIT(0);
    if (pend) ring_publish(lds, i - 1u, pn, ph, lane);
    for (int d = 0; d < NDEC; ++d, ++i) {
        const unsigned sl = i % NSLOT;
        while (i >= NSLOT && (int)(lds_ld(lds + CTL + 320 + sl * 4) - (i - NSLOT + 1u)) < 0) __builtin_amdgcn_s_sleep(1);
        ring_publish(lds, i, N_STOP, 0u, lane);
    }
}

__device__ __forceinline__ void decoder_wave(const Params& P, LAS unsigned char* lds, unsigned call, int src, int d, int lane) {
    char* ws = P.ws[P.rank];
    unsigned* tmo = (unsigned*)(ws + OFF_TMO);
    const rsrc_t wsr = mk_rsrc(ws, WS_USED), outr = mk_rsrc(P.out, (unsigned)DIM * DIM * 2u);
    const int myi = P.rank >> 2, myj = P.rank & 3;
    for (unsigned i = d;; i += NDEC) {
        const unsigned sl = i % NSLOT;
        while (lds_ld(lds + CTL + 256 + sl * 4) != i + 1u) __builtin_amdgcn_s_sleep(1);
        asm volatile("" ::: "memory");
        const u32x2 meta = *(volatile LAS u32x2*)(lds + CTL + sl * 8);
        const unsigned n = rfl(meta.x), h = rfl(meta.y);
        if (n == N_STOP) break;
        const Blk b = locate(n, src, myi, myj);
        const bool raw = (h >> 31) != 0;
        bool dead = (h & M_DEAD) != 0;
        if (b.is_out && rfl(__hip_atomic_load(tmo, __ATOMIC_RELAXED, __HIP_MEMORY_SCOPE_AGENT)) == call) dead = true;
        u32x4 c[8];
        unsigned recs = 0;
        const unsigned ne = (dead || raw) ? 0u : (h >> 8) & 63u;
        LAS unsigned char* cur = lds + sl * 8192;
        if (dead) {
#pragma unroll
            for (int j = 0; j < 8; ++j) c[j] = (u32x4){0x7FC07FC0u, 0x7FC07FC0u, 0x7FC07FC0u, 0x7FC07FC0u};
        } else if (raw) {
#pragma unroll
            for (int j = 0; j < 8; ++j) c[j] = *(LAS u32x4*)(cur + j * 1024 + lane * 16);
        } else {
            recs = *(LAS unsigned*)(cur + RAWB + (((h >> 16) & 0x3FFu) + (lane & 31)) * 4u);
            decode_block(cur, c, lane);
        }
#pragma unroll
        for (int j = 0; j < 8; ++j) asm volatile("" : "+v"(c[j]));
        asm volatile("" : "+v"(recs));
        if (lane == 0) lds_st(lds + CTL + 320 + sl * 4, i + 1u);
        const rsrc_t dr = b.is_out ? outr : wsr;
        const unsigned sj = b.is_out ? 8u * DIM * 2u : 1024u, sr = b.is_out ? DIM * 2u : 128u;
        const unsigned vo = (unsigned)(lane >> 3) * sr + (unsigned)(lane & 7) * 16u;
#pragma unroll
        for (int j = 0; j < 8; ++j) __builtin_amdgcn_raw_buffer_store_b128(c[j], dr, vo, b.dso + j * sj, AUX_SC1);
        for (unsigned e = 0; e < ne; ++e) {
            const unsigned rec = __builtin_amdgcn_readlane(recs, e), pos = rec & 0xFFFu, v = pos & 63u;
            if ((unsigned)lane == (pos >> 6)) __builtin_amdgcn_raw_buffer_store_b16((unsigned short)((rec >> 12) & 0xFFFFu), dr, vo + (v & 7u) * 2u, b.dso + (v >> 3) * sj, AUX_SC1);
        }
        if (b.cnt) { WAIT_VM0(); signal_cnt(ws, b.cnt, lane); }
    }
}

__device__ __forceinline__ void puller_role(const Params& P, LAS unsigned char* lds, unsigned call, int wg) {
    const int lane = threadIdx.x & 63, wid = rfl(threadIdx.x >> 6);
    const int link = wg % 7, src = rfl((P.rank + 1 + link) & 7);
    if (threadIdx.x < 32) *(volatile LAS unsigned*)(lds + CTL + 256 + threadIdx.x * 4) = 0u;
    __syncthreads();
    if (wid == 0) loader_wave(P, lds, call, src, lane);
    else if (wid <= NDEC) decoder_wave(P, lds, call, src, wid - 1, lane);
}

__device__ __forceinline__ float gelu_f(float y) {
    const float c1 = -2.0f * 0.7978845608028654f * 1.4426950408889634f, c2 = c1 * 0.044715f;
    const float e = __builtin_amdgcn_exp2f(y * (c1 + c2 * y * y));
    return y * __builtin_amdgcn_rcpf(1.0f + e);
}
__device__ __forceinline__ unsigned cvt_pk_bf16(float lo, float hi) { unsigned r; asm("v_cvt_pk_bf16_f32 %0, %1, %2" : "=v"(r) : "v"(lo), "v"(hi)); return r; }

__device__ __forceinline__ void gemm_role(const Params& P, LAS unsigned char* lds, unsigned call, int wg) {
    const int tid = threadIdx.x, wid = rfl(tid >> 6), lane = tid & 63, wr = wid >> 2, wc = wid & 3, fr = lane & 15, fq = lane >> 4;
    char* ws = P.ws[P.rank];
    unsigned* tmo = (unsigned*)(ws + OFF_TMO);
    const rsrc_t wsr = mk_rsrc(ws, WS_USED);
    const int xcd = wg & 7, q = (wg - NPACK) >> 3, h = xcd & 1, blk = xcd >> 1;
    const int tm = (blk >> 1) * 4 + (q >> 2), tn = (blk & 1) * 4 + (q & 3), tile = tm * 8 + tn;
    const unsigned voff = (unsigned)tid * 16u, ldsw = (unsigned)wid * 1024u;
    const int aoff0 = (wr * 64 + fr) * 128 + ((fq ^ (fr & 7)) << 4), aoff1 = aoff0 ^ 64;
    const int boff0 = (wc * 32 + fr) * 128 + ((fq ^ (fr & 7)) << 4), boff1 = boff0 ^ 64;
#define G_SA(b, hh) (((b) * 2 + (hh)) * 16384)
#define G_SB(b, hh) ((4 + (b) * 2 + (hh)) * 16384)
#define G_STAGE(bufoff, soff) do { _Pragma("unroll") for (int _i = 0; _i < 2; ++_i) \
        __builtin_amdgcn_raw_ptr_buffer_load_lds(wsr, (LAS void*)(lds + (bufoff) + ldsw + _i * 8192), 16, voff, (soff) + _i * 8192, 0, AUX_SYS); } while (0)
#define G_LDA(dst, b, hh) do { _Pragma("unroll") for (int m = 0; m < 4; ++m) { \
        dst[m][0] = *(const LAS bf16x8*)(lds + G_SA(b, hh) + aoff0 + m * 2048); dst[m][1] = *(const LAS bf16x8*)(lds + G_SA(b, hh) + aoff1 + m * 2048); } } while (0)
#define G_LDB(dst, b, hh) do { _Pragma("unroll") for (int n = 0; n < 2; ++n) { \
        dst[n][0] = *(const LAS bf16x8*)(lds + G_SB(b, hh) + boff0 + n * 2048); dst[n][1] = *(const LAS bf16x8*)(lds + G_SB(b, hh) + boff1 + n * 2048); } } while (0)
#define G_MMA(ai, bj, At, Bt) do { __builtin_amdgcn_s_setprio(1); _Pragma("unroll") for (int m = 0; m < 4; ++m) _Pragma("unroll") for (int n = 0; n < 2; ++n) _Pragma("unroll") for (int k = 0; k < 2; ++k) \
        acc[ai][bj][m][n] = __builtin_amdgcn_mfma_f32_16x16x32_bf16(Bt[n][k], At[m][k], acc[ai][bj][m][n], 0, 0, 0); __builtin_amdgcn_s_setprio(0); } while (0)
#define G_WAIT_V(n) asm volatile("s_waitcnt vmcnt(" #n ")" ::: "memory")
#define G_WAIT_L(n) asm volatile("s_waitcnt lgkmcnt(" #n ")" ::: "memory")
#define G_BAR __builtin_amdgcn_s_barrier()
#define G_SCHED __builtin_amdgcn_sched_barrier(0)
#define G_GIDX(t) ((unsigned)((4 * h + ((t) & 3)) * 16 + ((t) >> 2)))
    const int zero = 0;
    for (int r = 0; r < 2; ++r) {
        const unsigned target = call * (r == 0 ? 64u : 32u);
        const unsigned abase = OFF_AIMG + (unsigned)r * (128u * 262144u) + (unsigned)tm * 32768u, bbase = OFF_BIMG + (unsigned)tn * 32768u;
        char* cbase = ws + OFF_CNT + r * 128 * 128;
        f32x4 acc[2][2][4][2];
#pragma unroll
        for (int a = 0; a < 2; ++a)
#pragma unroll
            for (int b = 0; b < 2; ++b)
#pragma unroll
                for (int m = 0; m < 4; ++m)
#pragma unroll
                    for (int n = 0; n < 2; ++n) acc[a][b][m][n] = (f32x4){0.f, 0.f, 0.f, 0.f};
        bf16x8 At[4][2], B0[2][2], B1[2][2];
        STAMP(ws, r * 8 + 0);
        wait_word((unsigned*)(cbase + G_GIDX(0) * 128), target, tmo, call);
        wait_word((unsigned*)(cbase + G_GIDX(1) * 128), target, tmo, call);
        wait_word((unsigned*)(cbase + G_GIDX(2) * 128), target, tmo, call);
        STAMP(ws, r * 8 + 1);
        {
            const unsigned a0 = abase + G_GIDX(0) * 262144u, b0 = bbase + G_GIDX(0) * 262144u, a1 = abase + G_GIDX(1) * 262144u, b1 = bbase + G_GIDX(1) * 262144u;
            G_STAGE(G_SB(0, 0), b0); G_STAGE(G_SA(0, 0), a0); G_STAGE(G_SB(0, 1), b0 + 16384); G_STAGE(G_SA(0, 1), a0 + 16384);
            if (wr == 1) G_BAR;
            G_WAIT_V(4); G_BAR;
            G_STAGE(G_SB(1, 0), b1); G_STAGE(G_SA(1, 0), a1); G_STAGE(G_SB(1, 1), b1 + 16384);
            G_WAIT_V(6); G_BAR;
        }
        unsigned fA = target, fB;
        for (int t = 0; t < 64; t += 2) {
            const int t2 = t + 2 < 64 ? t + 2 : 63, t3 = t + 3 < 64 ? t + 3 : 63, t4 = t + 4 < 64 ? t + 4 : 63;
            const unsigned g1 = G_GIDX(t + 1), g2 = G_GIDX(t2), g3 = G_GIDX(t3), g4 = G_GIDX(t4);
            const unsigned a1 = abase + g1 * 262144u, a2 = abase + g2 * 262144u, b2 = bbase + g2 * 262144u, a3 = abase + g3 * 262144u, b3 = bbase + g3 * 262144u;
            asm volatile("global_load_dword %0, %1, %2 sc0 sc1" : "=v"(fB) : "v"(zero), "s"(cbase + g3 * 128));
            G_LDB(B0, 0, 0); G_SCHED; G_LDA(At, 0, 0); G_STAGE(G_SA(1, 1), a1 + 16384);
            G_WAIT_L(8); G_BAR; G_WAIT_L(0); G_MMA(0, 0, At, B0); G_BAR; G_SCHED;
            asm volatile("" : "+v"(fA));
            if ((int)(rfl(fA) - target) < 0) wait_word((unsigned*)(cbase + g2 * 128), target, tmo, call);
            G_LDB(B1, 0, 1); G_STAGE(G_SB(0, 0), b2);
            G_BAR; G_WAIT_L(0); G_MMA(0, 1, At, B1); G_BAR;
            G_LDA(At, 0, 1); G_STAGE(G_SA(0, 0), a2);
            G_BAR; G_WAIT_L(0); G_MMA(1, 0, At, B0); G_BAR; G_SCHED;
            G_STAGE(G_SB(0, 1), b2 + 16384);
            G_WAIT_V(6); G_BAR; G_MMA(1, 1, At, B1); G_BAR;
            asm volatile("global_load_dword %0, %1, %2 sc0 sc1" : "=v"(fA) : "v"(zero), "s"(cbase + g4 * 128));
            G_LDB(B0, 1, 0); G_SCHED; G_LDA(At, 1, 0); G_STAGE(G_SA(0, 1), a2 + 16384);
            G_WAIT_L(8); G_BAR; G_WAIT_L(0); G_MMA(0, 0, At, B0); G_BAR; G_SCHED;
            asm volatile("" : "+v"(fB));
            if ((int)(rfl(fB) - target) < 0) wait_word((unsigned*)(cbase + g3 * 128), target, tmo, call);
            G_LDB(B1, 1, 1); G_STAGE(G_SB(1, 0), b3);
            G_BAR; G_WAIT_L(0); G_MMA(0, 1, At, B1); G_BAR;
            G_LDA(At, 1, 1); G_STAGE(G_SA(1, 0), a3);
            G_BAR; G_WAIT_L(0); G_MMA(1, 0, At, B0); G_BAR; G_SCHED;
            G_STAGE(G_SB(1, 1), b3 + 16384);
            G_WAIT_V(6); G_BAR; G_MMA(1, 1, At, B1); G_BAR;
        }
        G_WAIT_V(0);
        asm volatile("" : "+v"(fA));
        if (wr == 0) G_BAR;
        G_BAR;
        STAMP(ws, r * 8 + 2);
        int lane = threadIdx.x & 63;
        asm volatile("" : "+v"(lane));
        const int fr = lane & 15, fq = lane >> 4;
        if (h) {
#pragma unroll
            for (int b = 0; b < 2; ++b)
#pragma unroll
                for (int m = 0; m < 4; ++m)
#pragma unroll
                    for (int n = 0; n < 2; ++n) { const f32x4 tq = acc[0][b][m][n]; acc[0][b][m][n] = acc[1][b][m][n]; acc[1][b][m][n] = tq; }
        }
        const unsigned xl = (unsigned)((r * 64 + tile) * 2);
#pragma unroll
        for (int b = 0; b < 2; ++b)
#pragma unroll
            for (int m = 0; m < 4; ++m)
#pragma unroll
                for (int n = 0; n < 2; ++n)
                    __builtin_amdgcn_raw_buffer_store_b128(__builtin_bit_cast(u32x4, acc[1][b][m][n]), wsr, lane * 16 + ((b * 4 + m) * 2 + n) * 1024, OFF_XBUF + ((xl + h) * 8 + wid) * 16384u, AUX_SC1);
        WAIT_VM0();
        if (lane == 0) __hip_atomic_store((unsigned*)(ws + OFF_XFLAG + (xl + h) * 128 + wid * 16), call, __ATOMIC_RELAXED, __HIP_MEMORY_SCOPE_AGENT);
        wait_word((unsigned*)(ws + OFF_XFLAG + (xl + (h ^ 1)) * 128 + wid * 16), call, tmo, call);
        const bool dead = rfl(__hip_atomic_load(tmo, __ATOMIC_RELAXED, __HIP_MEMORY_SCOPE_AGENT)) == call;
        STAMP(ws, r * 8 + 3);
        LAS unsigned char* my = lds + wid * 8192;
#pragma unroll
        for (int b = 0; b < 2; ++b)
#pragma unroll
            for (int m = 0; m < 4; ++m) {
                const f32x4 p0 = __builtin_bit_cast(f32x4, __builtin_amdgcn_raw_buffer_load_b128(wsr, lane * 16 + ((b * 4 + m) * 2 + 0) * 1024, OFF_XBUF + ((xl + (h ^ 1)) * 8 + wid) * 16384u, AUX_SC1));
                const f32x4 p1 = __builtin_bit_cast(f32x4, __builtin_amdgcn_raw_buffer_load_b128(wsr, lane * 16 + ((b * 4 + m) * 2 + 1) * 1024, OFF_XBUF + ((xl + (h ^ 1)) * 8 + wid) * 16384u, AUX_SC1));
                const f32x4 v0 = acc[0][b][m][0] + p0, v1 = acc[0][b][m][1] + p1;
                u32x4 o;
                o.x = cvt_pk_bf16(gelu_f(v0[0]), gelu_f(v0[1])); o.y = cvt_pk_bf16(gelu_f(v0[2]), gelu_f(v0[3]));
                o.z = cvt_pk_bf16(gelu_f(v1[0]), gelu_f(v1[1])); o.w = cvt_pk_bf16(gelu_f(v1[2]), gelu_f(v1[3]));
                const int row = m * 16 + fr, ch = b * 4 + fq;
                *(LAS u32x4*)(my + row * 128 + ((ch ^ (row & 7)) * 16)) = o;
            }
        asm volatile("s_waitcnt lgkmcnt(0)" ::: "memory");
        u32x4 c[8];
#pragma unroll
        for (int j = 0; j < 8; ++j) { const int row = j * 8 + (lane >> 3); c[j] = *(LAS u32x4*)(my + row * 128 + (((lane & 7) ^ (row & 7)) * 16)); }
        if (dead) {
#pragma unroll
            for (int j = 0; j < 8; ++j) c[j] = (u32x4){0x7FC07FC0u, 0x7FC07FC0u, 0x7FC07FC0u, 0x7FC07FC0u};
        }
        {
            const rsrc_t outr = mk_rsrc(P.out, (unsigned)DIM * DIM * 2u);
            const unsigned row0 = (P.rank >> 2) * 4096 + r * 2048 + tm * 256 + h * 128 + wr * 64, col0 = (P.rank & 3) * 2048 + tn * 256 + wc * 64;
            const unsigned dso = (row0 * DIM + col0) * 2u, vo = (unsigned)(lane >> 3) * (DIM * 2u) + (unsigned)(lane & 7) * 16u;
#pragma unroll
            for (int j = 0; j < 8; ++j) __builtin_amdgcn_raw_buffer_store_b128(c[j], outr, vo, dso + j * (8u * DIM * 2u), AUX_SC1);
            const unsigned o = (xl + h) * 8 + wid;
            publish_block(P, wsr, c, 4096u + o, 1536u + o, 0xFFu & ~(1u << P.rank), call, my, lane);
        }
        STAMP(ws, r * 8 + 4);
        asm volatile("s_waitcnt lgkmcnt(0)" ::: "memory");
        G_BAR;
    }
#undef G_SA
#undef G_SB
#undef G_STAGE
#undef G_LDA
#undef G_LDB
#undef G_MMA
#undef G_WAIT_V
#undef G_WAIT_L
#undef G_BAR
#undef G_SCHED
#undef G_GIDX
}

__global__ __launch_bounds__(512, 2) void gemm_allgather_kernel(Params P) {
    extern __shared__ __attribute__((aligned(16))) unsigned char shm[];
    LAS unsigned char* lds = (LAS unsigned char*)shm;
    char* ws = P.ws[P.rank];
    unsigned ticket = call_issue(ws);
    const int b = blockIdx.x;
    STAMP(ws, 60);
    unsigned call;
    if (b >= NPACK + NGEMM) { call = call_get(ticket); puller_role(P, lds, rfl(call), rfl(b - NPACK - NGEMM)); }
    else {
        const int wid = rfl(threadIdx.x >> 6);
        const bool gemm = b >= NPACK;
        const int next = gemm ? NFIRST + (b - NPACK) * 8 + wid : NFIRST + GROUNDS * NGEMM * 8 + b * 8 + wid;
        packer_role(P, lds, ticket, wid < 4 ? b * 4 + wid : next, next, gemm ? NGEMM * 8 : NPACK * 8, gemm ? NFIRST + GROUNDS * NGEMM * 8 : 4096);
        call = call_get(ticket);
        if (gemm) { __syncthreads(); gemm_role(P, lds, rfl(call), rfl(b)); }
    }
    STAMP(ws, 61);
    STAMP(ws, 58 + (call & 1));
}

}

extern "C" void kernel_launch(int rank, int nranks, ncclComm_t comm,
                              hipStream_t stream,
                              void* const* d_in, const int* in_sizes, int n_in,
                              void* d_out, int out_size,
                              const int* in_dtypes, int out_dtype,
                              void* const* d_out_all,
                              void* const* d_ws_all, size_t ws_bytes) {
    (void)comm; (void)in_sizes; (void)in_dtypes; (void)out_dtype; (void)d_out_all;
    if (nranks != gar::WORLD || n_in < 2 || ws_bytes < gar::WS_NEED || out_size != gar::DIM * gar::DIM || (unsigned)rank >= (unsigned)gar::WORLD) {
        fprintf(stderr, "unexpected problem shape\n");
        return;
    }
    static bool attr = false;
    if (!attr) {
        hipError_t e = hipFuncSetAttribute((const void*)gar::gemm_allgather_kernel, hipFuncAttributeMaxDynamicSharedMemorySize, gar::LDS_BYTES);
        if (e != hipSuccess) { fprintf(stderr, "hipFuncSetAttribute: %s\n", hipGetErrorString(e)); return; }
        attr = true;
    }
    gar::Params P;
    memset(&P, 0, sizeof P);
    P.x = d_in[0]; P.w = d_in[1]; P.out = d_out; P.rank = rank;
    for (int p = 0; p < gar::WORLD; ++p) P.ws[p] = (char*)d_ws_all[p];
    hipLaunchKernelGGL(gar::gemm_allgather_kernel, dim3(gar::GRID), dim3(512), gar::LDS_BYTES, stream, P);
    hipError_t e = hipGetLastError();
    if (e != hipSuccess) fprintf(stderr, "launch: %s\n", hipGetErrorString(e));
}
```

```cpp
#include <hip/hip_runtime.h>
#include <rccl/rccl.h>
#include <cstdio>
#include <cstdint>
#include <cstring>

namespace gar {

#define LAS __attribute__((address_space(3)))
typedef unsigned u32x4 __attribute__((ext_vector_type(4)));
typedef unsigned short u16x2 __attribute__((ext_vector_type(2)));
typedef short bf16x8 __attribute__((ext_vector_type(8)));
typedef float f32x4 __attribute__((ext_vector_type(4)));
typedef __amdgpu_buffer_rsrc_t rsrc_t;

constexpr int WORLD = 8;
constexpr int DIM = 8192;
constexpr int KSH = 1024;
constexpr int NGEMM = 128;
constexpr int GRID = 256;
#define PULL_WG_PER_LINK 12
#define PULL_DECODERS 7
constexpr int PWL = PULL_WG_PER_LINK;
constexpr int NDEC = PULL_DECODERS;
constexpr int PWAVES = 1;
constexpr int NPULL = 7 * PWL;
constexpr int NPACK = GRID - NGEMM - NPULL;
#define EPI_PACE 0
#define NRAW 40
constexpr int NFIRST = (NPACK + NGEMM) * 4;
#define GEMM_PACK_ROUNDS 1
constexpr int GROUNDS = GEMM_PACK_ROUNDS;
static_assert(NFIRST + GROUNDS * NGEMM * 8 <= 4096 && NFIRST % 8 == 0, "roles");
constexpr int NSEQ = 3584;
constexpr unsigned long long DEADLINE = 40000000ull;

constexpr unsigned OFF_TMO = 128;
constexpr unsigned OFF_LINK = 1024;
constexpr unsigned OFF_CNT = 4096;
constexpr unsigned OFF_XFLAG = OFF_CNT + 256 * 128;
constexpr unsigned OFF_GRAN = 131072;
constexpr unsigned OFF_CALL = 640u << 10;
constexpr unsigned OFF_SLOT = 1u << 20;
constexpr unsigned OFF_AIMG = 64u << 20;
constexpr unsigned OFF_BIMG = 128u << 20;
constexpr unsigned OFF_XBUF = 160u << 20;
constexpr unsigned WS_USED = 192u << 20;
constexpr unsigned WS_NEED = WS_USED;
static_assert(OFF_XFLAG + 256 * 128 <= OFF_GRAN, "map");
static_assert(OFF_GRAN + WORLD * NSEQ * 8 <= OFF_CALL && OFF_CALL + 2048 * 64 <= OFF_SLOT, "map");
static_assert(OFF_SLOT + 6144u * 8192u <= OFF_AIMG, "map");
constexpr int LDS_BYTES = 131072 + 256;

struct Params {
    const void* x; const void* w; void* out;
    char* ws[WORLD];
    int rank; int pad;
};

constexpr int AUX_SC1 = 16, AUX_SYS = 17;

__device__ __forceinline__ rsrc_t mk_rsrc(const void* p, unsigned bytes) {
    return __builtin_amdgcn_make_buffer_rsrc(const_cast<void*>(p), 0, bytes, 0x00020000);
}
__device__ __forceinline__ unsigned rfl(unsigned v) { return __builtin_amdgcn_readfirstlane(v); }
__device__ __forceinline__ unsigned bfi(unsigned mask, unsigned a, unsigned b) { return (a & mask) | (b & ~mask); }
__device__ __forceinline__ unsigned perm(unsigned hi, unsigned lo, unsigned sel) { return __builtin_amdgcn_perm(hi, lo, sel); }
#define WAIT_VM0() asm volatile("s_waitcnt vmcnt(0)" ::: "memory")

#define STAMP(ws, idx) do { } while (0)

__device__ __forceinline__ unsigned call_issue(char* ws) {
    unsigned t = 0;
    const int zero = 0, one = 1;
    if ((threadIdx.x & 63) == 0) asm volatile("global_atomic_add %0, %1, %2, %3 sc0" : "=v"(t) : "v"(zero), "v"(one), "s"(ws + OFF_CALL + (blockIdx.x * 8 + rfl(threadIdx.x >> 6)) * 64) : "memory");
    return t;
}
__device__ __forceinline__ unsigned call_get(unsigned& t) {
    asm volatile("s_waitcnt vmcnt(0)" : "+v"(t) :: "memory");
    return rfl(t) + 1u;
}

__device__ __forceinline__ bool give_up(unsigned long long t0, unsigned* tmo, unsigned call) {
    if (rfl(__hip_atomic_load(tmo, __ATOMIC_RELAXED, __HIP_MEMORY_SCOPE_AGENT)) == call) return true;
    if (__builtin_amdgcn_s_memrealtime() - t0 > DEADLINE) { __hip_atomic_store(tmo, call, __ATOMIC_RELAXED, __HIP_MEMORY_SCOPE_AGENT); return true; }
    return false;
}
__device__ __forceinline__ void wait_word(unsigned* p, unsigned target, unsigned* tmo, unsigned call) {
    if ((int)(rfl(__hip_atomic_load(p, __ATOMIC_RELAXED, __HIP_MEMORY_SCOPE_SYSTEM)) - target) >= 0) return;
    const unsigned long long t0 = __builtin_amdgcn_s_memrealtime();
    while ((int)(rfl(__hip_atomic_load(p, __ATOMIC_RELAXED, __HIP_MEMORY_SCOPE_SYSTEM)) - target) < 0) {
        if (give_up(t0, tmo, call)) break;
        __builtin_amdgcn_s_sleep(4);
    }
}

constexpr unsigned MAX_EXC = 31;
constexpr unsigned RAWB = 3584, AUX_U = 20, AUX_MAX = 2560, U_MAX = 10240, LANE_U_MAX = 224;
constexpr unsigned ENC_HIST = AUX_MAX + 16, ENC_SORT = ENC_HIST + 256, ENC_END = ENC_SORT + 128;
constexpr unsigned DEC_POS = 6144;

__device__ __forceinline__ unsigned bfi_c(unsigned mask, unsigned a, unsigned b) { unsigned r; asm("v_bfi_b32 %0, %1, %2, %3" : "=v"(r) : "s"(mask), "v"(a), "v"(b)); return r; }
__device__ __forceinline__ unsigned opaque(unsigned v) { asm("" : "+v"(v)); return v; }
__device__ __forceinline__ unsigned bytemask(unsigned x, int sh, unsigned c0c) {
    unsigned t; asm("v_and_or_b32 %0, %1, %2, %3" : "=v"(t) : "v"(x >> sh), "s"(0x01010101u), "v"(c0c));
    return perm(0u, 0u, t);
}
template <int CTRL, int RM = 0xf> __device__ __forceinline__ unsigned dpp0(unsigned v) { return (unsigned)__builtin_amdgcn_update_dpp(0, (int)v, CTRL, RM, 0xf, RM == 0xf); }
__device__ __forceinline__ unsigned wave_scan_incl(unsigned v) {
    v += dpp0<0x111>(v); v += dpp0<0x112>(v); v += dpp0<0x114>(v); v += dpp0<0x118>(v);
    v += dpp0<0x142, 0xa>(v); v += dpp0<0x143, 0xc>(v);
    return v;
}
__device__ __forceinline__ unsigned umin(unsigned a, unsigned b) { return a < b ? a : b; }
__device__ __forceinline__ unsigned umax(unsigned a, unsigned b) { return a > b ? a : b; }
__device__ __forceinline__ unsigned sad4(unsigned x) { return __builtin_amdgcn_sad_u8(x, 0u, 0u); }
__device__ __forceinline__ unsigned lut2x16(const unsigned (&T)[8], unsigned idx, unsigned ms, unsigned c0c) {
    const unsigned sel = idx & 0x07070707u, m8 = bytemask(idx, 3, c0c);
    const unsigned p = bfi(m8, perm(T[3], T[2], sel), perm(T[1], T[0], sel)), n = bfi(m8, perm(T[7], T[6], sel), perm(T[5], T[4], sel));
    return bfi(ms, n, p);
}
#define LDS_FENCE() asm volatile("" ::: "memory")
__device__ __forceinline__ void lds_or2(LAS unsigned char* scr, unsigned p, unsigned w) {
    const unsigned s = p & 31u;
    LAS unsigned* q = (LAS unsigned*)(scr + ((p >> 5) << 2));
    __hip_atomic_fetch_or(q, w << s, __ATOMIC_RELAXED, __HIP_MEMORY_SCOPE_WORKGROUP);
    __hip_atomic_fetch_or(q + 1, (w >> 1) >> (31u - s), __ATOMIC_RELAXED, __HIP_MEMORY_SCOPE_WORKGROUP);
}

__device__ __forceinline__ unsigned encode_block(const u32x4 (&c)[8], unsigned (&P)[14], u32x4 (&a)[3], LAS unsigned char* scr, int lane) {
    asm volatile("" : "+v"(lane));
    const unsigned c0c = opaque(0x0C0C0C0Cu);
    unsigned E[16], R[16];
    u16x2 p1 = {0, 0}, p2 = {0, 0}, n1 = {0, 0}, n2 = {0, 0};
#pragma unroll
    for (int g = 0; g < 16; ++g) {
        const unsigned x = (g & 1) ? c[g >> 1].z : c[g >> 1].x, y = (g & 1) ? c[g >> 1].w : c[g >> 1].y;
        const unsigned H = perm(y, x, 0x07050301u), L = perm(y, x, 0x06040200u);
        E[g] = bfi_c(0xFEFEFEFEu, H << 1, L >> 7);
        R[g] = bfi_c(0x80808080u, L << 1, bfi_c(0x40404040u, H >> 1, L));
        const unsigned en = E[g] & bytemask(H, 7, c0c), ep = E[g] ^ en;
        p1 = __builtin_elementwise_max(p1, __builtin_bit_cast(u16x2, ep)); p2 = __builtin_elementwise_max(p2, __builtin_bit_cast(u16x2, ep << 8));
        n1 = __builtin_elementwise_max(n1, __builtin_bit_cast(u16x2, en)); n2 = __builtin_elementwise_max(n2, __builtin_bit_cast(u16x2, en << 8));
        __builtin_amdgcn_sched_barrier(0);
    }
    const u32x4 z4 = {0u, 0u, 0u, 0u};
    *(LAS u32x4*)(scr + lane * 16) = z4; *(LAS u32x4*)(scr + 1024 + lane * 16) = z4;
    if (lane < 49) *(LAS u32x4*)(scr + 2048 + lane * 16) = z4;
    unsigned baseP, baseN;
    {
        const u16x2 mp = __builtin_elementwise_max(p1, p2), mn = __builtin_elementwise_max(n1, n2);
        u16x2 m = {(unsigned short)((mp.x > mp.y ? mp.x : mp.y) >> 8), (unsigned short)((mn.x > mn.y ? mn.x : mn.y) >> 8)};
        m = __builtin_elementwise_max(m, __builtin_bit_cast(u16x2, dpp0<0x128>(__builtin_bit_cast(unsigned, m)))); m = __builtin_elementwise_max(m, __builtin_bit_cast(u16x2, dpp0<0x124>(__builtin_bit_cast(unsigned, m))));
        m = __builtin_elementwise_max(m, __builtin_bit_cast(u16x2, dpp0<0x122>(__builtin_bit_cast(unsigned, m)))); m = __builtin_elementwise_max(m, __builtin_bit_cast(u16x2, dpp0<0x121>(__builtin_bit_cast(unsigned, m))));
        const unsigned mu = __builtin_bit_cast(unsigned, m);
        const u16x2 r01 = __builtin_elementwise_max(__builtin_bit_cast(u16x2, (unsigned)__builtin_amdgcn_readlane(mu, 0)), __builtin_bit_cast(u16x2, (unsigned)__builtin_amdgcn_readlane(mu, 16)));
        const u16x2 r23 = __builtin_elementwise_max(__builtin_bit_cast(u16x2, (unsigned)__builtin_amdgcn_readlane(mu, 32)), __builtin_bit_cast(u16x2, (unsigned)__builtin_amdgcn_readlane(mu, 48)));
        const unsigned mm = rfl(__builtin_bit_cast(unsigned, __builtin_elementwise_max(r01, r23)));
        baseP = mm & 0xFFu; baseN = mm >> 16;
    }
    const unsigned BP = baseP * 0x01010101u, BN = baseN * 0x01010101u;
    unsigned any = 0, D[16];
#pragma unroll
    for (int g = 0; g < 16; ++g) {
        E[g] = bfi(bytemask(R[g], 6, c0c), BN, BP) - E[g]; any |= E[g];
        D[g] = bfi_c(0x1E1E1E1Eu, E[g] << 1, (R[g] >> 7) & 0x01010101u);
        R[g] &= 0x7F7F7F7Fu;
    }
    LDS_FENCE();
#pragma unroll
    for (int gi = 0; gi < 4; ++gi) {
        const int g = gi * 5;
        const unsigned k4 = ((D[g] & 0x0F0F0F0Fu) | ((R[g] >> 2) & 0x10101010u) | ((D[g] << 1) & 0x20202020u)) << 2;
#pragma unroll
        for (int b = 0; b < 4; ++b)
            __hip_atomic_fetch_add((LAS unsigned*)(scr + ENC_HIST + ((k4 >> (8 * b)) & 0xFFu)), 1u, __ATOMIC_RELAXED, __HIP_MEMORY_SCOPE_WORKGROUP);
    }
    LDS_FENCE();
    const unsigned cnt = *(LAS unsigned*)(scr + ENC_HIST + lane * 4);
    const unsigned key = (cnt << 4) | (15u - (lane & 15));
    unsigned rank = 0;
    rank += dpp0<0x121>(key) > key; rank += dpp0<0x122>(key) > key; rank += dpp0<0x123>(key) > key; rank += dpp0<0x124>(key) > key; rank += dpp0<0x125>(key) > key;
    rank += dpp0<0x126>(key) > key; rank += dpp0<0x127>(key) > key; rank += dpp0<0x128>(key) > key; rank += dpp0<0x129>(key) > key; rank += dpp0<0x12A>(key) > key;
    rank += dpp0<0x12B>(key) > key; rank += dpp0<0x12C>(key) > key; rank += dpp0<0x12D>(key) > key; rank += dpp0<0x12E>(key) > key; rank += dpp0<0x12F>(key) > key;
    unsigned tail = cnt; tail += dpp0<0x128>(tail); tail += dpp0<0x124>(tail); tail += dpp0<0x122>(tail); tail += dpp0<0x121>(tail);
    const int sg = (lane >> 4) & 1;
    const unsigned ct = sg ? __builtin_amdgcn_readlane(tail, 48) : __builtin_amdgcn_readlane(tail, 32);
    if (lane < 32) *(LAS unsigned*)(scr + ENC_SORT + ((lane & 16) + rank) * 4) = cnt;
    LDS_FENCE();
    unsigned best;
    {
        const unsigned cb = lane & 15, s0 = cb / 3u, s1 = cb - 3u * s0, A = 1u << s0, AB = A + (1u << s1);
        unsigned cost = ct * (16u - AB);
#pragma unroll
        for (int q = 0; q < 4; ++q) {
            const u32x4 h = *(LAS u32x4*)(scr + ENC_SORT + sg * 64 + q * 16);
#pragma unroll
            for (int j = 0; j < 4; ++j) {
                const unsigned r = q * 4 + j, len = r < A ? 1u + s0 : r < AB ? 2u + s1 : 4u + ((r - AB) >> 1);
                cost += 2u * len * h[j];
            }
        }
        best = cb < 9 ? (cost << 4) | cb : 0xFFFFFFFFu;
        best = umin(best, dpp0<0x128>(best)); best = umin(best, dpp0<0x124>(best)); best = umin(best, dpp0<0x122>(best)); best = umin(best, dpp0<0x121>(best));
    }
    const unsigned cbP = __builtin_amdgcn_readlane(best, 0) & 15u, cbN = __builtin_amdgcn_readlane(best, 16) & 15u;
    const unsigned s0P = cbP / 3u, s1P = cbP - 3u * s0P, s0N = cbN / 3u, s1N = cbN - 3u * s0N;
    const unsigned ABP = (1u << s0P) + (1u << s1P), ABN = (1u << s0N) + (1u << s1N);
    unsigned F[8];
    {
        const unsigned s0 = sg ? s0N : s0P, s1 = sg ? s1N : s1P, A = 1u << s0, AB = sg ? ABN : ABP;
        const unsigned ent = rank < A ? (rank << 4) | (s0 << 6) : rank < AB ? 1u | ((rank - A) << 4) | (s1 << 6) : (2u + ((rank - AB) >> 1)) | (((rank - AB) & 1u) << 4) | (1u << 6);
        unsigned fv = ent << (8 * (lane & 3));
        fv |= dpp0<0xB1>(fv); fv |= dpp0<0x4E>(fv);
#pragma unroll
        for (int q = 0; q < 8; ++q) F[q] = __builtin_amdgcn_readlane(fv, q * 4);
        if (lane < 32)
            __hip_atomic_fetch_or((LAS unsigned*)(scr + (sg * 2 + (rank >> 3)) * 4), (unsigned)(lane & 15) << (8 * (rank & 3) + 4 * ((rank >> 2) & 1)), __ATOMIC_RELAXED, __HIP_MEMORY_SCOPE_WORKGROUP);
        if (lane == 0) *(LAS unsigned*)(scr + 16) = baseP | (baseN << 8) | (s0P << 16) | (s1P << 18) | (s0N << 20) | (s1N << 22);
    }
    const unsigned ABP4 = ABP * 0x01010101u, ABN4 = ABN * 0x01010101u, S0P4 = s0P * 0x01010101u, S0N4 = s0N * 0x01010101u;
    unsigned KK[16], XW[4], OX[4], TU = 0, TX = 0, bad = 0, xl = 0, xw = 0;
#pragma unroll
    for (int g = 0; g < 16; ++g) {
        const unsigned ms = bytemask(R[g], 6, c0c), m16 = bytemask(D[g], 4, c0c), ent = lut2x16(F, D[g], ms, c0c);
        const unsigned q = ((D[g] | 0x80808080u) - bfi(ms, ABN4, ABP4)) & 0x7F7F7F7Fu;
        unsigned K = bfi(m16, ((q >> 1) & 0x3F3F3F3Fu) + 0x02020202u, ent & 0x0F0F0F0Fu);
        unsigned SUF = bfi(m16, q & 0x01010101u, (ent >> 4) & 0x03030303u);
        unsigned S = bfi(m16, 0x01010101u, (ent >> 6) & 0x03030303u);
        unsigned gl = sad4(K) + 4u;
        if (__ballot(gl > 31u) != 0ull) {
            any |= 0x80u;
            const unsigned km = umax(umax(K & 0xFFu, (K >> 8) & 0xFFu), umax((K >> 16) & 0xFFu, K >> 24));
            const unsigned eq = gl > 31u ? (((K ^ (km * 0x01010101u)) + 0x7F7F7F7Fu) & 0x80808080u) ^ 0x80808080u : 0u, em = bytemask(eq, 7, c0c);
            E[g] |= eq;
            K &= ~em; SUF &= ~em; S = bfi(em, bfi(ms, S0N4, S0P4), S);
            gl = sad4(K) + 4u;
        }
        KK[g] = K; TU += gl; bad = umax(bad, gl);
        xw |= (SUF & 0xFFu) << xl; xl += S & 0xFFu;
        xw |= ((SUF >> 8) & 0xFFu) << xl; xl += (S >> 8) & 0xFFu;
        xw |= ((SUF >> 16) & 0xFFu) << xl; xl += (S >> 16) & 0xFFu;
        xw |= (SUF >> 24) << (xl & 31u); xl += S >> 24;
        if ((g & 3) == 3) { XW[g >> 2] = xw; OX[g >> 2] = TX; TX += xl; xw = 0; xl = 0; }
        __builtin_amdgcn_sched_barrier(0);
    }
    bad = (bad > 31u) | (TU > LANE_U_MAX);
    const unsigned tt = TU | (TX << 16), incl = wave_scan_incl(tt), tot = __builtin_amdgcn_readlane(incl, 63), totU = tot & 0xFFFFu, totX = tot >> 16;
    const unsigned xstart = (AUX_U * 8u + totU + 31u) & ~31u, endbit = xstart + totX;
    unsigned raw = (__ballot(bad != 0) != 0ull) || totU > U_MAX || endbit > (AUX_MAX - 8u) * 8u;
    if (!raw) {
        const unsigned px = xstart + ((incl - tt) >> 16);
        unsigned p = AUX_U * 8u + ((incl - tt) & 0xFFFFu);
#pragma unroll
        for (int g = 0; g < 16; ++g) {
            const unsigned ln = KK[g] + 0x01010101u;
            unsigned pos = (ln & 0xFFu) - 1u, w = 1u << pos;
            pos += (ln >> 8) & 0xFFu; w |= 1u << pos;
            pos += (ln >> 16) & 0xFFu; w |= 1u << pos;
            pos += ln >> 24; w |= 1u << pos;
            lds_or2(scr, p, w); p += pos + 1u;
            __builtin_amdgcn_sched_barrier(0);
        }
#pragma unroll
        for (int q = 0; q < 4; ++q) lds_or2(scr, px + OX[q], XW[q]);
    }
#pragma unroll
    for (int i = 0; i < 7; ++i) { P[i] = R[i] | ((R[14] << (7 - i)) & 0x80808080u); P[7 + i] = R[7 + i] | ((R[15] << (7 - i)) & 0x80808080u); }
    unsigned n = 0;
    const unsigned xo = (endbit + 31u) >> 5;
    if (!raw && __ballot((any & 0xF0F0F0F0u) != 0) != 0ull) {
#pragma unroll
        for (int g = 0; g < 16; ++g) {
            const unsigned f = E[g] & 0xF0F0F0F0u;
            unsigned long long mask = __ballot(f != 0);
            while (mask != 0ull && n <= MAX_EXC) {
                const int L = __builtin_ctzll(mask); mask &= mask - 1;
                const unsigned fl = __builtin_amdgcn_readlane(f, L);
                const unsigned lo = __builtin_amdgcn_readlane((g & 1) ? c[g >> 1].z : c[g >> 1].x, L);
                const unsigned hi = __builtin_amdgcn_readlane((g & 1) ? c[g >> 1].w : c[g >> 1].y, L);
                const bool e = lane < 4 && ((fl >> (8 * (lane & 3))) & 0xFFu) != 0;
                const unsigned long long em = __ballot(e);
                const unsigned slot = xo + n + __builtin_popcountll(em & ((1ull << lane) - 1ull));
                const unsigned val = (((lane & 2) ? hi : lo) >> (16 * (lane & 1))) & 0xFFFFu;
                if (e && slot * 4u < AUX_MAX) *(LAS unsigned*)(scr + slot * 4u) = (unsigned)(L * 64 + g * 4 + (lane & 3)) | (val << 12);
                n += __builtin_popcountll(em);
            }
        }
    }
    raw |= (n > MAX_EXC) | ((xo + n) * 4u > AUX_MAX);
    LDS_FENCE();
    a[0] = *(LAS u32x4*)(scr + lane * 16); a[1] = *(LAS u32x4*)(scr + 1024 + lane * 16); a[2] = *(LAS u32x4*)(scr + 2048 + (lane & 31) * 16);
    return ((xo + n + 3u) >> 2) | (n << 8) | (xo << 16) | (raw << 31);
}

__device__ __forceinline__ void decode_block(LAS unsigned char* src, u32x4 (&c)[8], int lane) {
    LAS unsigned char* aux = src + RAWB;
    const unsigned c0c = opaque(0x0C0C0C0Cu);
    const u32x4 nt = *(LAS const u32x4*)(aux);
    const unsigned prm_v = *(LAS const unsigned*)(aux + 16);
    unsigned u[5];
#pragma unroll
    for (int i = 0; i < 5; ++i) u[i] = *(LAS const unsigned*)(aux + AUX_U + lane * 20 + i * 4);
    u32x4 P4[3];
#pragma unroll
    for (int p = 0; p < 3; ++p) P4[p] = *(LAS const u32x4*)(src + p * 1024 + lane * 16);
    const unsigned long long ph = *(LAS const unsigned long long*)(src + 3072 + lane * 8);
    const unsigned c0 = __builtin_popcount(u[0]), c1 = c0 + __builtin_popcount(u[1]), c2 = c1 + __builtin_popcount(u[2]), c3 = c2 + __builtin_popcount(u[3]), cn = c3 + __builtin_popcount(u[4]);
    const unsigned C = wave_scan_incl(cn) - cn;
    if (lane == 0) *(LAS unsigned*)(src + DEC_POS) = AUX_U * 8u;
    const int rounds = __ballot(cn > 128u) != 0ull ? 3 : 2;
    for (int j = 0; j < rounds; ++j) {
        const unsigned t = (C >> 6) + 1u + j, kth = 64u * t - C;
        const unsigned i = (kth > c0) + (kth > c1) + (kth > c2) + (kth > c3);
        unsigned rem = kth - (i == 0 ? 0u : i == 1 ? c0 : i == 2 ? c1 : i == 3 ? c2 : c3), w = i == 0 ? u[0] : i == 1 ? u[1] : i == 2 ? u[2] : i == 3 ? u[3] : u[4], pos = 0;
#pragma unroll
        for (int st = 16; st >= 1; st >>= 1) {
            const unsigned lo = __builtin_popcount(w & ((1u << st) - 1u));
            const bool up = rem > lo;
            rem -= up ? lo : 0u; w = up ? w >> st : w; pos += up ? st : 0;
        }
        if (kth <= cn && t < 64u) *(LAS unsigned*)(src + DEC_POS + t * 4) = AUX_U * 8u + lane * 160u + i * 32u + pos + 1u;
    }
    LDS_FENCE();
    const unsigned p0 = *(LAS const unsigned*)(src + DEC_POS + lane * 4);
    unsigned S[8];
    {
        LAS const unsigned* q = (LAS const unsigned*)(aux + ((p0 >> 5) << 2));
#pragma unroll
        for (int i = 0; i < 8; ++i) S[i] = q[i];
#pragma unroll
        for (int i = 0; i < 7; ++i) S[i] = __builtin_amdgcn_alignbit(S[i + 1], S[i], p0 & 31u);
    }
    unsigned R[16];
    {
        const unsigned P[14] = {P4[0].x, P4[0].y, P4[0].z, P4[0].w, P4[1].x, P4[1].y, P4[1].z, P4[1].w, P4[2].x, P4[2].y, P4[2].z, P4[2].w, (unsigned)ph, (unsigned)(ph >> 32)};
        unsigned x = 0, y = 0;
#pragma unroll
        for (int i = 0; i < 7; ++i) { x |= (P[i] >> (7 - i)) & (0x01010101u << i); y |= (P[7 + i] >> (7 - i)) & (0x01010101u << i); }
#pragma unroll
        for (int i = 0; i < 14; ++i) R[i] = P[i] & 0x7F7F7F7Fu;
        R[14] = x; R[15] = y;
    }
    const unsigned n0 = rfl(nt.x), n1 = rfl(nt.y), n2 = rfl(nt.z), n3 = rfl(nt.w), prm = rfl(prm_v);
    const unsigned I[8] = {n0 & 0x0F0F0F0Fu, (n0 >> 4) & 0x0F0F0F0Fu, n1 & 0x0F0F0F0Fu, (n1 >> 4) & 0x0F0F0F0Fu, n2 & 0x0F0F0F0Fu, (n2 >> 4) & 0x0F0F0F0Fu, n3 & 0x0F0F0F0Fu, (n3 >> 4) & 0x0F0F0F0Fu};
    const unsigned s0P = (prm >> 16) & 3u, s1P = (prm >> 18) & 3u, s0N = (prm >> 20) & 3u, s1N = (prm >> 22) & 3u;
    const unsigned STP = (s0P << 8) | (s1P << 16) | (1u << 24), STN = (s0N << 8) | (s1N << 16) | (1u << 24);
    const unsigned OTP = (4u << 8) | (((1u << s0P) + 2u) << 16) | (((1u << s0P) + (1u << s1P)) << 24);
    const unsigned OTN = (4u << 8) | (((1u << s0N) + 2u) << 16) | (((1u << s0N) + (1u << s1N)) << 24);
    const unsigned BP = (prm & 0xFFu) * 0x01010101u, BN = ((prm >> 8) & 0xFFu) * 0x01010101u;
    unsigned RB[16], SS[16], TU = 0, TX = 0;
#pragma unroll
    for (int g = 0; g < 16; ++g) {
        const unsigned w0 = S[0], q0 = __builtin_ctz(w0), w1 = w0 & (w0 - 1u), q1 = __builtin_ctz(w1), w2 = w1 & (w1 - 1u), q2 = __builtin_ctz(w2), w3 = w2 & (w2 - 1u), q3 = __builtin_ctz(w3);
        const unsigned Q = q0 | (q1 << 8) | (q2 << 16) | (q3 << 24);
        const unsigned LN = Q - opaque((Q << 8) - 1u), sh = q3 + 1u;
        TU += sh;
        if (g < 15) {
#pragma unroll
            for (int i = 0; i < 7; ++i) if (32 * i < (int)LANE_U_MAX - 4 * (g + 1)) S[i] = (i < 6 && 32 * (i + 1) < (int)LANE_U_MAX - 4 * g) ? __builtin_amdgcn_alignbit(S[i + 1], S[i], sh) : S[i] >> sh;
        }
        const unsigned sel = bfi_c(0x04040404u, R[g] >> 4, bfi(bytemask(LN + 0x7D7D7D7Du, 7, c0c), 0x03030303u, LN));
        SS[g] = perm(STN, STP, sel);
        RB[g] = perm(OTN, OTP, sel) + (LN << 1);
        TX = __builtin_amdgcn_sad_u8(SS[g], 0u, TX);
    }
    const unsigned xstart = (__builtin_amdgcn_readlane(p0 + TU, 63) + 31u) & ~31u, px = xstart + wave_scan_incl(TX) - TX;
    unsigned X[5];
    {
        LAS const unsigned* q = (LAS const unsigned*)(aux + ((px >> 5) << 2));
#pragma unroll
        for (int i = 0; i < 5; ++i) X[i] = q[i];
#pragma unroll
        for (int i = 0; i < 4; ++i) X[i] = __builtin_amdgcn_alignbit(X[i + 1], X[i], px & 31u);
    }
    unsigned xoff = 0;
#pragma unroll
    for (int g = 0; g < 16; ++g) {
        const unsigned s = SS[g], M = perm(0u, 0x00030100u, s);
        const unsigned y = (g & 1) ? X[0] >> xoff : X[0], Y = perm(y, y, 0x01000100u);
        const unsigned t1 = opaque(s << 8), t2 = opaque((t1 << 8) + t1), Cs = (t2 << 16) + t2;
        const u16x2 z02 = __builtin_bit_cast(u16x2, Y) >> __builtin_bit_cast(u16x2, Cs & 0x00FF00FFu);
        const u16x2 z13 = __builtin_bit_cast(u16x2, Y) >> __builtin_bit_cast(u16x2, perm(0u, Cs, 0x0C030C01u));
        const unsigned suf = perm(__builtin_bit_cast(unsigned, z13), __builtin_bit_cast(unsigned, z02), 0x06020400u) & M;
        if (g & 1) {
            if (g < 15) {
                const unsigned t = __builtin_amdgcn_sad_u8(s, 0u, xoff);
#pragma unroll
                for (int i = 0; i < 4; ++i) if (32 * i < 8 * (15 - g)) X[i] = (i < 3 && 32 * (i + 1) < 8 * (17 - g)) ? __builtin_amdgcn_alignbit(X[i + 1], X[i], t) : X[i] >> t;
            }
        } else xoff = sad4(s);
        const unsigned rank = RB[g] + suf - 0x06060606u, ms = bytemask(R[g], 6, c0c);
        const unsigned D = bfi(bytemask(rank, 4, c0c), rank, lut2x16(I, rank, ms, c0c));
        const unsigned E = bfi(ms, BN, BP) - ((D >> 1) & 0x0F0F0F0Fu);
        const unsigned H = bfi_c(0x80808080u, R[g] << 1, E >> 1), L = bfi_c(0x3F3F3F3Fu, R[g], bfi_c(0x40404040u, D << 6, E << 7));
        const unsigned x = perm(H, L, 0x05010400u), yv = perm(H, L, 0x07030602u);
        if (g & 1) { c[g >> 1].z = x; c[g >> 1].w = yv; } else { c[g >> 1].x = x; c[g >> 1].y = yv; }
    }
}

typedef unsigned u32x2 __attribute__((ext_vector_type(2)));
__device__ __forceinline__ void store_granule(char* p, u32x2 g) {
    asm volatile("global_store_dwordx2 %0, %1, off sc0 sc1\n\ts_nop 1" :: "v"(p), "v"(g) : "memory");
}

constexpr unsigned GCTL = 131072;
__device__ __forceinline__ void publish_block(const Params& P, rsrc_t wsr, const u32x4 (&c)[8], unsigned slot, unsigned n, unsigned dmask, unsigned call, LAS unsigned char* lds, LAS unsigned char* scr,
                                              unsigned j, int wid, int lane) {
    unsigned R[14], h = 1u << 31; u32x4 a[3];
    if (n >= NRAW) h = encode_block(c, R, a, scr, lane);
    const unsigned so = OFF_SLOT + slot * 8192u;
    if (h >> 31) {
#pragma unroll
        for (int q = 0; q < 8; ++q) __builtin_amdgcn_raw_buffer_store_b128(c[q], wsr, lane * 16 + q * 1024, so, AUX_SYS);
    } else {
        const unsigned n16 = h & 255u;
#pragma unroll
        for (int q = 0; q < 3; ++q) __builtin_amdgcn_raw_buffer_store_b128((u32x4){R[4 * q], R[4 * q + 1], R[4 * q + 2], R[4 * q + 3]}, wsr, lane * 16 + q * 1024, so, AUX_SYS);
        __builtin_amdgcn_raw_buffer_store_b64((u32x2){R[12], R[13]}, wsr, lane * 8 + 3072, so, AUX_SYS);
        if ((unsigned)lane < n16) __builtin_amdgcn_raw_buffer_store_b128(a[0], wsr, lane * 16 + RAWB, so, AUX_SYS);
        if ((unsigned)lane + 64u < n16) __builtin_amdgcn_raw_buffer_store_b128(a[1], wsr, lane * 16 + RAWB + 1024, so, AUX_SYS);
        if ((unsigned)lane + 128u < n16) __builtin_amdgcn_raw_buffer_store_b128(a[2], wsr, lane * 16 + RAWB + 2048, so, AUX_SYS);
    }
    WAIT_VM0();
    const unsigned u = j & 3u, gsize = j == 0 ? 4u : 8u;
    volatile LAS unsigned* ctl = (volatile LAS unsigned*)(lds + GCTL);
    while (j >= 4u && (int)(ctl[36 + u] - (j - 3u)) < 0) __builtin_amdgcn_s_sleep(1);
    unsigned arrived = 0;
    if (lane == 0) {
        ctl[u * 8 + wid] = h;
        arrived = __hip_atomic_fetch_add((LAS unsigned*)(lds + GCTL) + 32 + u, 1u, __ATOMIC_RELAXED, __HIP_MEMORY_SCOPE_WORKGROUP);
    }
    if (rfl(arrived) == gsize - 1u) {
        const unsigned d = lane >> 3, w = lane & 7, hw = ctl[u * 8 + w];
        if (w < gsize && ((dmask >> d) & 1u)) store_granule(P.ws[d] + OFF_GRAN + ((unsigned)P.rank * NSEQ + n - wid + w) * 8u, (u32x2){call, hw});
        if (lane == 0) { ctl[32 + u] = 0u; ctl[36 + u] = j + 1u; }
    }
}

__device__ __forceinline__ void packer_role(const Params& P, LAS unsigned char* lds, unsigned& ticket, int first, int next, int step, int end) {
    const int lane = threadIdx.x & 63, wid = rfl(threadIdx.x >> 6);
    char* ws = P.ws[P.rank];
    const rsrc_t wsr = mk_rsrc(ws, WS_USED);
    const rsrc_t xr = mk_rsrc(P.x, DIM * KSH * 2), wr = mk_rsrc(P.w, DIM * KSH * 2);
    const int myi = P.rank >> 2, myj = P.rank & 3;
    LAS unsigned char* my = lds + wid * 8192;
    unsigned j = first < NFIRST ? 0u : 1u;
    for (int p = first; p < end; p = p < NFIRST ? next : p + step, ++j) {
        bool is_w; int kk, sel, r, bi;
        if (p < 3072) {
            kk = p / 192; const int rem = p - kk * 192;
            bi = rem & 31; r = 0; is_w = rem >= 64; sel = is_w ? ((rem - 64) >> 5) : (rem >> 5);
        }
        else { const int q = p - 3072; kk = q >> 6; sel = (q >> 5) & 1; bi = q & 31; r = 1; is_w = false; }
        u32x4 c[8];
        if (!is_w) {
            const int rr = lane >> 3;
            const unsigned vo = (unsigned)rr * 2048u + (unsigned)(((lane & 7) ^ rr) * 16);
            const unsigned so = (unsigned)(sel * 4096 + r * 2048 + bi * 64) * 2048u + (unsigned)kk * 128u;
#pragma unroll
            for (int j = 0; j < 8; ++j) c[j] = __builtin_amdgcn_raw_buffer_load_b128(xr, vo + j * 8 * 2048, so, 0);
        } else {
            const int pn = bi >> 2, bj = (bi >> 1) & 1, q = bi & 1;
            const int gk = lane >> 3, cc = lane & 7, wcl = cc >> 2, cq = cc & 3;
            const unsigned vo = (unsigned)(gk * 8) * 16384u + (unsigned)((64 * (2 * q + wcl) + 32 * bj + 8 * cq) * 2);
            const unsigned so = (unsigned)(kk * 64) * 16384u + (unsigned)((sel * 2048 + pn * 256) * 2);
            u32x4 R[8];
#pragma unroll
            for (int jj = 0; jj < 8; ++jj) R[jj] = __builtin_amdgcn_raw_buffer_load_b128(wr, vo + jj * 16384, so, 0);
#pragma unroll
            for (int j = 0; j < 8; ++j) {
                const unsigned s = (j & 1) ? 0x07060302u : 0x05040100u;
                u32x4 t;
                t.x = perm(R[1][j >> 1], R[0][j >> 1], s); t.y = perm(R[3][j >> 1], R[2][j >> 1], s);
                t.z = perm(R[5][j >> 1], R[4][j >> 1], s); t.w = perm(R[7][j >> 1], R[6][j >> 1], s);
                const int s64 = 32 * wcl + 16 * (j >> 2) + 4 * cq + (j & 3);
                *(LAS u32x4*)(my + s64 * 128 + ((gk ^ (s64 & 7)) * 16)) = t;
            }
            asm volatile("s_waitcnt lgkmcnt(0)" ::: "memory");
#pragma unroll
            for (int j = 0; j < 8; ++j) c[j] = *(LAS u32x4*)(my + j * 1024 + lane * 16);
            asm volatile("s_waitcnt lgkmcnt(0)" ::: "memory");
        }
        const unsigned call = call_get(ticket);
        const int g = P.rank * 16 + kk;
        const bool self = is_w ? (sel == myj) : (sel == myi);
        if (self) {
            const unsigned so = is_w ? OFF_BIMG + (unsigned)(g * 32 + bi) * 8192u : OFF_AIMG + (unsigned)((r * 128 + g) * 32 + bi) * 8192u;
#pragma unroll
            for (int j = 0; j < 8; ++j) __builtin_amdgcn_raw_buffer_store_b128(c[j], wsr, lane * 16 + j * 1024, so, AUX_SC1);
        }
        const unsigned slot = is_w ? 2048u + (unsigned)((sel * 16 + kk) * 32 + bi) : (unsigned)(((sel * 2 + r) * 16 + kk) * 32 + bi);
        const unsigned n = is_w ? (unsigned)(kk * 64 + 32 + bi) : (r == 0 ? (unsigned)(kk * 64 + bi) : (unsigned)(1024 + kk * 32 + bi));
        unsigned dmask = is_w ? (0x11u << sel) : (0x0Fu << (4 * sel));
        dmask &= ~(1u << P.rank);
        publish_block(P, wsr, c, slot, n, dmask, call, lds, my, j, wid, lane);
        if (self && lane == 0)
            __hip_atomic_fetch_add((unsigned*)(ws + OFF_CNT + (r * 128 + g) * 128), 1u, __ATOMIC_RELAXED, __HIP_MEMORY_SCOPE_SYSTEM);
    }
}

__device__ __forceinline__ u32x2 load_granule(rsrc_t wsr, unsigned off) { return __builtin_amdgcn_raw_buffer_load_b64(wsr, 0, off, AUX_SYS); }

struct Blk { unsigned slot, dso, cnt; bool is_out; };
__device__ __forceinline__ Blk locate(unsigned n, int src, int myi, int myj) {
    Blk b; b.is_out = false;
    if (n < 1024) { const int kk = n >> 6, bi = n & 31, g = src * 16 + kk; b.cnt = OFF_CNT + g * 128;
        if ((n & 63) >= 32) { b.slot = 2048u + (myj * 16 + kk) * 32 + bi; b.dso = OFF_BIMG + (unsigned)(g * 32 + bi) * 8192u; }
        else { b.slot = ((myi * 2) * 16 + kk) * 32 + bi; b.dso = OFF_AIMG + (unsigned)(g * 32 + bi) * 8192u; } }
    else if (n < 1536) { const int kk = (n - 1024) >> 5, bi = n & 31, g = src * 16 + kk; b.cnt = OFF_CNT + (128 + g) * 128;
        b.slot = ((myi * 2 + 1) * 16 + kk) * 32 + bi; b.dso = OFF_AIMG + (unsigned)((128 + g) * 32 + bi) * 8192u; }
    else { const int o = n - 1536, w = o & 7, h = (o >> 3) & 1, tile = (o >> 4) & 63, r = o >> 10; b.cnt = 0; b.is_out = true;
        b.slot = 4096u + o;
        const unsigned row0 = (src >> 2) * 4096 + r * 2048 + (tile >> 3) * 256 + h * 128 + (w >> 2) * 64, col0 = (src & 3) * 2048 + (tile & 7) * 256 + (w & 3) * 64;
        b.dso = (row0 * DIM + col0) * 2u; }
    return b;
}
__device__ __forceinline__ void signal_cnt(char* ws, unsigned off, int lane) {
    if (off && lane == 0) __hip_atomic_fetch_add((unsigned*)(ws + off), 1u, __ATOMIC_RELAXED, __HIP_MEMORY_SCOPE_SYSTEM);
}
constexpr unsigned NSLOT = 15, CTL = NSLOT * 8192;
constexpr unsigned M_DEAD = 1u << 30, N_STOP = 0xFFFFFFFFu;
__device__ __forceinline__ unsigned lds_ld(LAS unsigned char* p) { return *(volatile LAS unsigned*)p; }
__device__ __forceinline__ void lds_st(LAS unsigned char* p, unsigned v) { *(volatile LAS unsigned*)p = v; }
__device__ __forceinline__ void ring_publish(LAS unsigned char* lds, unsigned i, unsigned n, unsigned h, int lane) {
    const unsigned sl = i % NSLOT;
    if (lane == 0) { *(volatile LAS u32x2*)(lds + CTL + sl * 8) = (u32x2){n, h}; lds_st(lds + CTL + 256 + sl * 4, i + 1u); }
}
__device__ __forceinline__ void dma_op(rsrc_t r, LAS unsigned char* dst, int j, unsigned so, int lane) {
    __builtin_amdgcn_raw_ptr_buffer_load_lds(r, (LAS void*)(dst + j * 1024), 16, lane * 16 + j * 1024, so, 0, AUX_SYS);
}
#define VM_WAIT_(n) asm volatile("s_waitcnt vmcnt(" #n ")" ::: "memory")
#define VM_WAIT(n) VM_WAIT_(n)
#define WIN 5

__device__ __forceinline__ void loader_wave(const Params& P, LAS unsigned char* lds, unsigned call, int src, int lane) {
    char* ws = P.ws[P.rank];
    unsigned* tmo = (unsigned*)(ws + OFF_TMO);
    const rsrc_t wsr = mk_rsrc(ws, WS_USED);
    char* const remp = P.ws[src];
    const int myi = P.rank >> 2, myj = P.rank & 3;
    const unsigned gbase = OFF_GRAN + (unsigned)src * NSEQ * 8u;
    char* const gptr = ws + gbase;
    const unsigned tko = OFF_LINK + (unsigned)src * 128u, tbase = (call - 1u) * (NSEQ + 3u * PWL);
    __builtin_amdgcn_s_setprio(3);
    unsigned n, n1, n2;
    {
        unsigned t0 = 0;
        if (lane == 0) t0 = __hip_atomic_fetch_add((unsigned*)(ws + tko), 3u, __ATOMIC_RELAXED, __HIP_MEMORY_SCOPE_AGENT);
        n = rfl(t0) - tbase; n1 = n + 1; n2 = n + 2;
    }
    unsigned gx, gy, hx, hy;
    { const u32x2 a = load_granule(wsr, gbase + (n < NSEQ ? n : 0u) * 8u), b = load_granule(wsr, gbase + (n1 < NSEQ ? n1 : 0u) * 8u);
      gx = rfl(a.x); gy = rfl(a.y); hx = rfl(b.x); hy = rfl(b.y); }
    const int zero = 0, one = 1;
    unsigned i = 0;
    bool pend = false; unsigned pn = 0, ph = 0;
    while (n < NSEQ) {
        const unsigned sl = i % NSLOT;
        bool dead = false;
        const bool need_gran = (int)(gx - call) < 0;
        const bool need_slot = i >= NSLOT && (int)(lds_ld(lds + CTL + 320 + sl * 4) - (i - NSLOT + 1u)) < 0;
        if (need_gran || need_slot) {
            if (pend) { VM_WAIT(0); ring_publish(lds, i - 1u, pn, ph, lane); pend = false; }
            const unsigned long long t0 = __builtin_amdgcn_s_memrealtime();
            while ((int)(gx - call) < 0) {
                const u32x2 a = load_granule(wsr, gbase + n * 8u);
                gx = rfl(a.x); gy = rfl(a.y);
                if ((int)(gx - call) >= 0) break;
                if (give_up(t0, tmo, call)) { dead = true; break; }
                __builtin_amdgcn_s_sleep(2);
                asm volatile("" ::: "memory");
            }
            while (i >= NSLOT && (int)(lds_ld(lds + CTL + 320 + sl * 4) - (i - NSLOT + 1u)) < 0) __builtin_amdgcn_s_sleep(1);
        }
        const bool raw = (gy >> 31) != 0;
        const unsigned so = OFF_SLOT + locate(n, src, myi, myj).slot * 8192u;
        const unsigned bytes = dead ? 0u : raw ? 8192u : RAWB + ((gy & 255u) << 4);
        const rsrc_t rem = mk_rsrc(remp, so + bytes);
        LAS unsigned char* dst = lds + sl * 8192;
        u32x2 g2; unsigned tk = 0;
        asm volatile("global_load_dwordx2 %0, %1, %2 sc0 sc1" : "=v"(g2) : "v"(zero), "s"(gptr + (n2 < NSEQ ? n2 : 0u) * 8u) : "memory");
        if (lane == 0) asm volatile("global_atomic_add %0, %1, %2, %3 sc0" : "=v"(tk) : "v"(zero), "v"(one), "s"(ws + tko) : "memory");
        VM_WAIT(WIN); dma_op(rem, dst, 0, so, lane);
        VM_WAIT(WIN); dma_op(rem, dst, 1, so, lane);
        VM_WAIT(WIN); dma_op(rem, dst, 2, so, lane);
        VM_WAIT(WIN); if (pend) ring_publish(lds, i - 1u, pn, ph, lane); dma_op(rem, dst, 3, so, lane);
        VM_WAIT(WIN); dma_op(rem, dst, 4, so, lane);
        asm volatile("s_waitcnt vmcnt(5)" : "+v"(g2), "+v"(tk) :: "memory"); dma_op(rem, dst, 5, so, lane);
        if (raw) { dma_op(rem, dst, 6, so, lane); dma_op(rem, dst, 7, so, lane); VM_WAIT(0); }
        pend = true; pn = n; ph = gy | (dead ? M_DEAD : 0u);
        n = n1; gx = hx; gy = hy;
        n1 = n2; hx = rfl(g2.x); hy = rfl(g2.y);
        n2 = rfl(tk) - tbase;
        ++i;
    }
    VM_WAIT(0);
    if (pend) ring_publish(lds, i - 1u, pn, ph, lane);
    for (int d = 0; d < NDEC; ++d, ++i) {
        const unsigned sl = i % NSLOT;
        while (i >= NSLOT && (int)(lds_ld(lds + CTL + 320 + sl * 4) - (i - NSLOT + 1u)) < 0) __builtin_amdgcn_s_sleep(1);
        ring_publish(lds, i, N_STOP, 0u, lane);
    }
}

__device__ __forceinline__ void decoder_wave(const Params& P, LAS unsigned char* lds, unsigned call, int src, int d, int lane) {
    char* ws = P.ws[P.rank];
    unsigned* tmo = (unsigned*)(ws + OFF_TMO);
    const rsrc_t wsr = mk_rsrc(ws, WS_USED), outr = mk_rsrc(P.out, (unsigned)DIM * DIM * 2u);
    const int myi = P.rank >> 2, myj = P.rank & 3;
    for (unsigned i = d;; i += NDEC) {
        const unsigned sl = i % NSLOT;
        while (lds_ld(lds + CTL + 256 + sl * 4) != i + 1u) __builtin_amdgcn_s_sleep(1);
        asm volatile("" ::: "memory");
        const u32x2 meta = *(volatile LAS u32x2*)(lds + CTL + sl * 8);
        const unsigned n = rfl(meta.x), h = rfl(meta.y);
        if (n == N_STOP) break;
        const Blk b = locate(n, src, myi, myj);
        const bool raw = (h >> 31) != 0;
        bool dead = (h & M_DEAD) != 0;
        if (b.is_out && rfl(__hip_atomic_load(tmo, __ATOMIC_RELAXED, __HIP_MEMORY_SCOPE_AGENT)) == call) dead = true;
        u32x4 c[8];
        unsigned recs = 0;
        const unsigned ne = (dead || raw) ? 0u : (h >> 8) & 63u;
        LAS unsigned char* cur = lds + sl * 8192;
        if (dead) {
#pragma unroll
            for (int j = 0; j < 8; ++j) c[j] = (u32x4){0x7FC07FC0u, 0x7FC07FC0u, 0x7FC07FC0u, 0x7FC07FC0u};
        } else if (raw) {
#pragma unroll
            for (int j = 0; j < 8; ++j) c[j] = *(LAS u32x4*)(cur + j * 1024 + lane * 16);
        } else {
            recs = *(LAS unsigned*)(cur + RAWB + (((h >> 16) & 0x3FFu) + (lane & 31)) * 4u);
            decode_block(cur, c, lane);
        }
#pragma unroll
        for (int j = 0; j < 8; ++j) asm volatile("" : "+v"(c[j]));
        asm volatile("" : "+v"(recs));
        if (lane == 0) lds_st(lds + CTL + 320 + sl * 4, i + 1u);
        const rsrc_t dr = b.is_out ? outr : wsr;
        const unsigned sj = b.is_out ? 8u * DIM * 2u : 1024u, sr = b.is_out ? DIM * 2u : 128u;
        const unsigned vo = (unsigned)(lane >> 3) * sr + (unsigned)(lane & 7) * 16u;
#pragma unroll
        for (int j = 0; j < 8; ++j) __builtin_amdgcn_raw_buffer_store_b128(c[j], dr, vo, b.dso + j * sj, AUX_SC1);
        for (unsigned e = 0; e < ne; ++e) {
            const unsigned rec = __builtin_amdgcn_readlane(recs, e), pos = rec & 0xFFFu, v = pos & 63u;
            if ((unsigned)lane == (pos >> 6)) __builtin_amdgcn_raw_buffer_store_b16((unsigned short)((rec >> 12) & 0xFFFFu), dr, vo + (v & 7u) * 2u, b.dso + (v >> 3) * sj, AUX_SC1);
        }
        if (b.cnt) { WAIT_VM0(); signal_cnt(ws, b.cnt, lane); }
    }
}

__device__ __forceinline__ void puller_role(const Params& P, LAS unsigned char* lds, unsigned call, int wg) {
    const int lane = threadIdx.x & 63, wid = rfl(threadIdx.x >> 6);
    const int link = wg % 7, src = rfl((P.rank + 1 + link) & 7);
    if (threadIdx.x < 32) *(volatile LAS unsigned*)(lds + CTL + 256 + threadIdx.x * 4) = 0u;
    __syncthreads();
    if (wid == 0) loader_wave(P, lds, call, src, lane);
    else if (wid <= NDEC) decoder_wave(P, lds, call, src, wid - 1, lane);
}

__device__ __forceinline__ float gelu_f(float y) {
    const float c1 = -2.0f * 0.7978845608028654f * 1.4426950408889634f, c2 = c1 * 0.044715f;
    const float e = __builtin_amdgcn_exp2f(y * (c1 + c2 * y * y));
    return y * __builtin_amdgcn_rcpf(1.0f + e);
}
__device__ __forceinline__ unsigned cvt_pk_bf16(float lo, float hi) { unsigned r; asm("v_cvt_pk_bf16_f32 %0, %1, %2" : "=v"(r) : "v"(lo), "v"(hi)); return r; }

__device__ __forceinline__ void gemm_role(const Params& P, LAS unsigned char* lds, unsigned call, int wg) {
    const int tid = threadIdx.x, wid = rfl(tid >> 6), lane = tid & 63, wr = wid >> 2, wc = wid & 3, fr = lane & 15, fq = lane >> 4;
    char* ws = P.ws[P.rank];
    unsigned* tmo = (unsigned*)(ws + OFF_TMO);
    const rsrc_t wsr = mk_rsrc(ws, WS_USED);
    const int xcd = wg & 7, q = (wg - NPACK) >> 3, h = xcd & 1, blk = xcd >> 1;
    const int tm = (blk >> 1) * 4 + (q >> 2), tn = (blk & 1) * 4 + (q & 3), tile = tm * 8 + tn;
    const unsigned voff = (unsigned)tid * 16u, ldsw = (unsigned)wid * 1024u;
    const int aoff0 = (wr * 64 + fr) * 128 + ((fq ^ (fr & 7)) << 4), aoff1 = aoff0 ^ 64;
    const int boff0 = (wc * 32 + fr) * 128 + ((fq ^ (fr & 7)) << 4), boff1 = boff0 ^ 64;
#define G_SA(b, hh) (((b) * 2 + (hh)) * 16384)
#define G_SB(b, hh) ((4 + (b) * 2 + (hh)) * 16384)
#define G_STAGE(bufoff, soff) do { _Pragma("unroll") for (int _i = 0; _i < 2; ++_i) \
        __builtin_amdgcn_raw_ptr_buffer_load_lds(wsr, (LAS void*)(lds + (bufoff) + ldsw + _i * 8192), 16, voff, (soff) + _i * 8192, 0, AUX_SYS); } while (0)
#define G_LDA(dst, b, hh) do { _Pragma("unroll") for (int m = 0; m < 4; ++m) { \
        dst[m][0] = *(const LAS bf16x8*)(lds + G_SA(b, hh) + aoff0 + m * 2048); dst[m][1] = *(const LAS bf16x8*)(lds + G_SA(b, hh) + aoff1 + m * 2048); } } while (0)
#define G_LDB(dst, b, hh) do { _Pragma("unroll") for (int n = 0; n < 2; ++n) { \
        dst[n][0] = *(const LAS bf16x8*)(lds + G_SB(b, hh) + boff0 + n * 2048); dst[n][1] = *(const LAS bf16x8*)(lds + G_SB(b, hh) + boff1 + n * 2048); } } while (0)
#define G_MMA(ai, bj, At, Bt) do { __builtin_amdgcn_s_setprio(1); _Pragma("unroll") for (int m = 0; m < 4; ++m) _Pragma("unroll") for (int n = 0; n < 2; ++n) _Pragma("unroll") for (int k = 0; k < 2; ++k) \
        acc[ai][bj][m][n] = __builtin_amdgcn_mfma_f32_16x16x32_bf16(Bt[n][k], At[m][k], acc[ai][bj][m][n], 0, 0, 0); __builtin_amdgcn_s_setprio(0); } while (0)
#define G_WAIT_V(n) asm volatile("s_waitcnt vmcnt(" #n ")" ::: "memory")
#define G_WAIT_L(n) asm volatile("s_waitcnt lgkmcnt(" #n ")" ::: "memory")
#define G_BAR __builtin_amdgcn_s_barrier()
#define G_SCHED __builtin_amdgcn_sched_barrier(0)
#define G_GIDX(t) ((unsigned)((4 * h + ((t) & 3)) * 16 + ((t) >> 2)))
    const int zero = 0;
    for (int r = 0; r < 2; ++r) {
        const unsigned target = call * (r == 0 ? 64u : 32u);
        const unsigned abase = OFF_AIMG + (unsigned)r * (128u * 262144u) + (unsigned)tm * 32768u, bbase = OFF_BIMG + (unsigned)tn * 32768u;
        char* cbase = ws + OFF_CNT + r * 128 * 128;
        f32x4 acc[2][2][4][2];
#pragma unroll
        for (int a = 0; a < 2; ++a)
#pragma unroll
            for (int b = 0; b < 2; ++b)
#pragma unroll
                for (int m = 0; m < 4; ++m)
#pragma unroll
                    for (int n = 0; n < 2; ++n) acc[a][b][m][n] = (f32x4){0.f, 0.f, 0.f, 0.f};
        bf16x8 At[4][2], B0[2][2], B1[2][2];
        STAMP(ws, r * 8 + 0);
        wait_word((unsigned*)(cbase + G_GIDX(0) * 128), target, tmo, call);
        wait_word((unsigned*)(cbase + G_GIDX(1) * 128), target, tmo, call);
        wait_word((unsigned*)(cbase + G_GIDX(2) * 128), target, tmo, call);
        STAMP(ws, r * 8 + 1);
        {
            const unsigned a0 = abase + G_GIDX(0) * 262144u, b0 = bbase + G_GIDX(0) * 262144u, a1 = abase + G_GIDX(1) * 262144u, b1 = bbase + G_GIDX(1) * 262144u;
            G_STAGE(G_SB(0, 0), b0); G_STAGE(G_SA(0, 0), a0); G_STAGE(G_SB(0, 1), b0 + 16384); G_STAGE(G_SA(0, 1), a0 + 16384);
            if (wr == 1) G_BAR;
            G_WAIT_V(4); G_BAR;
            G_STAGE(G_SB(1, 0), b1); G_STAGE(G_SA(1, 0), a1); G_STAGE(G_SB(1, 1), b1 + 16384);
            G_WAIT_V(6); G_BAR;
        }
        unsigned fA = target, fB;
        for (int t = 0; t < 64; t += 2) {
            const int t2 = t + 2 < 64 ? t + 2 : 63, t3 = t + 3 < 64 ? t + 3 : 63, t4 = t + 4 < 64 ? t + 4 : 63;
            const unsigned g1 = G_GIDX(t + 1), g2 = G_GIDX(t2), g3 = G_GIDX(t3), g4 = G_GIDX(t4);
            const unsigned a1 = abase + g1 * 262144u, a2 = abase + g2 * 262144u, b2 = bbase + g2 * 262144u, a3 = abase + g3 * 262144u, b3 = bbase + g3 * 262144u;
            asm volatile("global_load_dword %0, %1, %2 sc0 sc1" : "=v"(fB) : "v"(zero), "s"(cbase + g3 * 128));
            G_LDB(B0, 0, 0); G_SCHED; G_LDA(At, 0, 0); G_STAGE(G_SA(1, 1), a1 + 16384);
            G_WAIT_L(8); G_BAR; G_WAIT_L(0); G_MMA(0, 0, At, B0); G_BAR; G_SCHED;
            asm volatile("" : "+v"(fA));
            if ((int)(rfl(fA) - target) < 0) wait_word((unsigned*)(cbase + g2 * 128), target, tmo, call);
            G_LDB(B1, 0, 1); G_STAGE(G_SB(0, 0), b2);
            G_BAR; G_WAIT_L(0); G_MMA(0, 1, At, B1); G_BAR;
            G_LDA(At, 0, 1); G_STAGE(G_SA(0, 0), a2);
            G_BAR; G_WAIT_L(0); G_MMA(1, 0, At, B0); G_BAR; G_SCHED;
            G_STAGE(G_SB(0, 1), b2 + 16384);
            G_WAIT_V(6); G_BAR; G_MMA(1, 1, At, B1); G_BAR;
            asm volatile("global_load_dword %0, %1, %2 sc0 sc1" : "=v"(fA) : "v"(zero), "s"(cbase + g4 * 128));
            G_LDB(B0, 1, 0); G_SCHED; G_LDA(At, 1, 0); G_STAGE(G_SA(0, 1), a2 + 16384);
            G_WAIT_L(8); G_BAR; G_WAIT_L(0); G_MMA(0, 0, At, B0); G_BAR; G_SCHED;
            asm volatile("" : "+v"(fB));
            if ((int)(rfl(fB) - target) < 0) wait_word((unsigned*)(cbase + g3 * 128), target, tmo, call);
            G_LDB(B1, 1, 1); G_STAGE(G_SB(1, 0), b3);
            G_BAR; G_WAIT_L(0); G_MMA(0, 1, At, B1); G_BAR;
            G_LDA(At, 1, 1); G_STAGE(G_SA(1, 0), a3);
            G_BAR; G_WAIT_L(0); G_MMA(1, 0, At, B0); G_BAR; G_SCHED;
            G_STAGE(G_SB(1, 1), b3 + 16384);
            G_WAIT_V(6); G_BAR; G_MMA(1, 1, At, B1); G_BAR;
        }
        G_WAIT_V(0);
        asm volatile("" : "+v"(fA));
        if (wr == 0) G_BAR;
        G_BAR;
        STAMP(ws, r * 8 + 2);
        int lane = threadIdx.x & 63;
        asm volatile("" : "+v"(lane));
        const int fr = lane & 15, fq = lane >> 4;
        if (h) {
#pragma unroll
            for (int b = 0; b < 2; ++b)
#pragma unroll
                for (int m = 0; m < 4; ++m)
#pragma unroll
                    for (int n = 0; n < 2; ++n) { const f32x4 tq = acc[0][b][m][n]; acc[0][b][m][n] = acc[1][b][m][n]; acc[1][b][m][n] = tq; }
        }
        const unsigned xl = (unsigned)((r * 64 + tile) * 2);
#pragma unroll
        for (int b = 0; b < 2; ++b)
#pragma unroll
            for (int m = 0; m < 4; ++m)
#pragma unroll
                for (int n = 0; n < 2; ++n) {
                    __builtin_amdgcn_raw_buffer_store_b128(__builtin_bit_cast(u32x4, acc[1][b][m][n]), wsr, lane * 16 + ((b * 4 + m) * 2 + n) * 1024, OFF_XBUF + ((xl + h) * 8 + wid) * 16384u, AUX_SC1);
                    if (EPI_PACE) __builtin_amdgcn_s_sleep(EPI_PACE);
                }
        WAIT_VM0();
        if (lane == 0) __hip_atomic_store((unsigned*)(ws + OFF_XFLAG + (xl + h) * 128 + wid * 16), call, __ATOMIC_RELAXED, __HIP_MEMORY_SCOPE_AGENT);
        wait_word((unsigned*)(ws + OFF_XFLAG + (xl + (h ^ 1)) * 128 + wid * 16), call, tmo, call);
        const bool dead = rfl(__hip_atomic_load(tmo, __ATOMIC_RELAXED, __HIP_MEMORY_SCOPE_AGENT)) == call;
        STAMP(ws, r * 8 + 3);
        LAS unsigned char* my = lds + wid * 8192;
#pragma unroll
        for (int b = 0; b < 2; ++b)
#pragma unroll
            for (int m = 0; m < 4; ++m) {
                const f32x4 p0 = __builtin_bit_cast(f32x4, __builtin_amdgcn_raw_buffer_load_b128(wsr, lane * 16 + ((b * 4 + m) * 2 + 0) * 1024, OFF_XBUF + ((xl + (h ^ 1)) * 8 + wid) * 16384u, AUX_SC1));
                const f32x4 p1 = __builtin_bit_cast(f32x4, __builtin_amdgcn_raw_buffer_load_b128(wsr, lane * 16 + ((b * 4 + m) * 2 + 1) * 1024, OFF_XBUF + ((xl + (h ^ 1)) * 8 + wid) * 16384u, AUX_SC1));
                const f32x4 v0 = acc[0][b][m][0] + p0, v1 = acc[0][b][m][1] + p1;
                if (EPI_PACE) { asm volatile("" :: "v"(v0), "v"(v1)); __builtin_amdgcn_s_sleep(EPI_PACE); __builtin_amdgcn_sched_barrier(0); }
                u32x4 o;
                o.x = cvt_pk_bf16(gelu_f(v0[0]), gelu_f(v0[1])); o.y = cvt_pk_bf16(gelu_f(v0[2]), gelu_f(v0[3]));
                o.z = cvt_pk_bf16(gelu_f(v1[0]), gelu_f(v1[1])); o.w = cvt_pk_bf16(gelu_f(v1[2]), gelu_f(v1[3]));
                const int row = m * 16 + fr, ch = b * 4 + fq;
                *(LAS u32x4*)(my + row * 128 + ((ch ^ (row & 7)) * 16)) = o;
            }
        asm volatile("s_waitcnt lgkmcnt(0)" ::: "memory");
        u32x4 c[8];
#pragma unroll
        for (int j = 0; j < 8; ++j) { const int row = j * 8 + (lane >> 3); c[j] = *(LAS u32x4*)(my + row * 128 + (((lane & 7) ^ (row & 7)) * 16)); }
        if (dead) {
#pragma unroll
            for (int j = 0; j < 8; ++j) c[j] = (u32x4){0x7FC07FC0u, 0x7FC07FC0u, 0x7FC07FC0u, 0x7FC07FC0u};
        }
        {
            const rsrc_t outr = mk_rsrc(P.out, (unsigned)DIM * DIM * 2u);
            const unsigned row0 = (P.rank >> 2) * 4096 + r * 2048 + tm * 256 + h * 128 + wr * 64, col0 = (P.rank & 3) * 2048 + tn * 256 + wc * 64;
            const unsigned dso = (row0 * DIM + col0) * 2u, vo = (unsigned)(lane >> 3) * (DIM * 2u) + (unsigned)(lane & 7) * 16u;
#pragma unroll
            for (int j = 0; j < 8; ++j) __builtin_amdgcn_raw_buffer_store_b128(c[j], outr, vo, dso + j * (8u * DIM * 2u), AUX_SC1);
            const unsigned o = (xl + h) * 8 + wid;
            publish_block(P, wsr, c, 4096u + o, 1536u + o, 0xFFu & ~(1u << P.rank), call, lds, my, 1u + GROUNDS + r, wid, lane);
        }
        STAMP(ws, r * 8 + 4);
        asm volatile("s_waitcnt lgkmcnt(0)" ::: "memory");
        G_BAR;
    }
#undef G_SA
#undef G_SB
#undef G_STAGE
#undef G_LDA
#undef G_LDB
#undef G_MMA
#undef G_WAIT_V
#undef G_WAIT_L
#undef G_BAR
#undef G_SCHED
#undef G_GIDX
}

__global__ __launch_bounds__(512, 2) void gemm_allgather_kernel(Params P) {
    extern __shared__ __attribute__((aligned(16))) unsigned char shm[];
    LAS unsigned char* lds = (LAS unsigned char*)shm;
    char* ws = P.ws[P.rank];
    unsigned ticket = call_issue(ws);
    const int b = blockIdx.x;
    STAMP(ws, 60);
    unsigned call;
    if (b >= NPACK + NGEMM) { call = call_get(ticket); puller_role(P, lds, rfl(call), rfl(b - NPACK - NGEMM)); }
    else {
        if (threadIdx.x < 8) *(volatile LAS unsigned*)(lds + GCTL + 128 + threadIdx.x * 4) = 0u;
        __syncthreads();
        const int wid = rfl(threadIdx.x >> 6);
        const bool gemm = b >= NPACK;
        const int next = gemm ? NFIRST + (b - NPACK) * 8 + wid : NFIRST + GROUNDS * NGEMM * 8 + b * 8 + wid;
        packer_role(P, lds, ticket, wid < 4 ? b * 4 + wid : next, next, gemm ? NGEMM * 8 : NPACK * 8, gemm ? NFIRST + GROUNDS * NGEMM * 8 : 4096);
        call = call_get(ticket);
        if (gemm) { __syncthreads(); gemm_role(P, lds, rfl(call), rfl(b)); }
    }
    STAMP(ws, 61);
    STAMP(ws, 58 + (call & 1));
}

}

extern "C" void kernel_launch(int rank, int nranks, ncclComm_t comm,
                              hipStream_t stream,
                              void* const* d_in, const int* in_sizes, int n_in,
                              void* d_out, int out_size,
                              const int* in_dtypes, int out_dtype,
                              void* const* d_out_all,
                              void* const* d_ws_all, size_t ws_bytes) {
    (void)comm; (void)in_sizes; (void)in_dtypes; (void)out_dtype; (void)d_out_all;
    if (nranks != gar::WORLD || n_in < 2 || ws_bytes < gar::WS_NEED || out_size != gar::DIM * gar::DIM || (unsigned)rank >= (unsigned)gar::WORLD) {
        fprintf(stderr, "unexpected problem shape\n");
        return;
    }
    static bool attr = false;
    if (!attr) {
        hipError_t e = hipFuncSetAttribute((const void*)gar::gemm_allgather_kernel, hipFuncAttributeMaxDynamicSharedMemorySize, gar::LDS_BYTES);
        if (e != hipSuccess) { fprintf(stderr, "hipFuncSetAttribute: %s\n", hipGetErrorString(e)); return; }
        attr = true;
    }
    gar::Params P;
    memset(&P, 0, sizeof P);
    P.x = d_in[0]; P.w = d_in[1]; P.out = d_out; P.rank = rank;
    for (int p = 0; p < gar::WORLD; ++p) P.ws[p] = (char*)d_ws_all[p];
    hipLaunchKernelGGL(gar::gemm_allgather_kernel, dim3(gar::GRID), dim3(512), gar::LDS_BYTES, stream, P);
    hipError_t e = hipGetLastError();
    if (e != hipSuccess) fprintf(stderr, "launch: %s\n", hipGetErrorString(e));
}
```

```cpp
#include <hip/hip_runtime.h>
#include <rccl/rccl.h>
#include <cstdio>
#include <cstdint>
#include <cstring>

namespace gar {

#define LAS __attribute__((address_space(3)))
typedef unsigned u32x4 __attribute__((ext_vector_type(4)));
typedef unsigned short u16x2 __attribute__((ext_vector_type(2)));
typedef short bf16x8 __attribute__((ext_vector_type(8)));
typedef float f32x4 __attribute__((ext_vector_type(4)));
typedef __amdgpu_buffer_rsrc_t rsrc_t;

constexpr int WORLD = 8;
constexpr int DIM = 8192;
constexpr int KSH = 1024;
constexpr int NGEMM = 128;
constexpr int GRID = 256;
#define PULL_WG_PER_LINK 12
#define PULL_DECODERS 7
constexpr int PWL = PULL_WG_PER_LINK;
constexpr int NDEC = PULL_DECODERS;
constexpr int PWAVES = 1;
constexpr int NPULL = 7 * PWL;
constexpr int NPACK = GRID - NGEMM - NPULL;
#define NRAWEND 24
#define WARM_TLB 1
#define EPI_PACE 0
#define NRAW 40
constexpr int NFIRST = (NPACK + NGEMM) * 4;
#define GEMM_PACK_ROUNDS 1
constexpr int GROUNDS = GEMM_PACK_ROUNDS;
static_assert(NFIRST + GROUNDS * NGEMM * 8 <= 4096 && NFIRST % 8 == 0, "roles");
constexpr int NSEQ = 3584;
constexpr unsigned long long DEADLINE = 40000000ull;

constexpr unsigned OFF_TMO = 128;
constexpr unsigned OFF_LINK = 1024;
constexpr unsigned OFF_CNT = 4096;
constexpr unsigned OFF_XFLAG = OFF_CNT + 256 * 128;
constexpr unsigned OFF_GRAN = 131072;
constexpr unsigned OFF_CALL = 640u << 10;
constexpr unsigned OFF_SLOT = 1u << 20;
constexpr unsigned OFF_AIMG = 64u << 20;
constexpr unsigned OFF_BIMG = 128u << 20;
constexpr unsigned OFF_XBUF = 160u << 20;
constexpr unsigned WS_USED = 192u << 20;
constexpr unsigned WS_NEED = WS_USED;
static_assert(OFF_XFLAG + 256 * 128 <= OFF_GRAN, "map");
static_assert(OFF_GRAN + WORLD * NSEQ * 8 <= OFF_CALL && OFF_CALL + 2048 * 64 <= OFF_SLOT, "map");
static_assert(OFF_SLOT + 6144u * 8192u <= OFF_AIMG, "map");
constexpr int LDS_BYTES = 131072 + 256;

struct Params {
    const void* x; const void* w; void* out;
    char* ws[WORLD];
    int rank; int pad;
};

constexpr int AUX_SC1 = 16, AUX_SYS = 17;

__device__ __forceinline__ rsrc_t mk_rsrc(const void* p, unsigned bytes) {
    return __builtin_amdgcn_make_buffer_rsrc(const_cast<void*>(p), 0, bytes, 0x00020000);
}
__device__ __forceinline__ unsigned rfl(unsigned v) { return __builtin_amdgcn_readfirstlane(v); }
__device__ __forceinline__ unsigned bfi(unsigned mask, unsigned a, unsigned b) { return (a & mask) | (b & ~mask); }
__device__ __forceinline__ unsigned perm(unsigned hi, unsigned lo, unsigned sel) { return __builtin_amdgcn_perm(hi, lo, sel); }
#define WAIT_VM0() asm volatile("s_waitcnt vmcnt(0)" ::: "memory")

#define STAMP(ws, idx) do { } while (0)

__device__ __forceinline__ unsigned call_issue(char* ws) {
    unsigned t = 0;
    const int zero = 0, one = 1;
    if ((threadIdx.x & 63) == 0) asm volatile("global_atomic_add %0, %1, %2, %3 sc0" : "=v"(t) : "v"(zero), "v"(one), "s"(ws + OFF_CALL + (blockIdx.x * 8 + rfl(threadIdx.x >> 6)) * 64) : "memory");
    return t;
}
__device__ __forceinline__ unsigned call_get(unsigned& t) {
    asm volatile("s_waitcnt vmcnt(0)" : "+v"(t) :: "memory");
    return rfl(t) + 1u;
}

__device__ __forceinline__ bool give_up(unsigned long long t0, unsigned* tmo, unsigned call) {
    if (rfl(__hip_atomic_load(tmo, __ATOMIC_RELAXED, __HIP_MEMORY_SCOPE_AGENT)) == call) return true;
    if (__builtin_amdgcn_s_memrealtime() - t0 > DEADLINE) { __hip_atomic_store(tmo, call, __ATOMIC_RELAXED, __HIP_MEMORY_SCOPE_AGENT); return true; }
    return false;
}
__device__ __forceinline__ void wait_word(unsigned* p, unsigned target, unsigned* tmo, unsigned call) {
    if ((int)(rfl(__hip_atomic_load(p, __ATOMIC_RELAXED, __HIP_MEMORY_SCOPE_SYSTEM)) - target) >= 0) return;
    const unsigned long long t0 = __builtin_amdgcn_s_memrealtime();
    while ((int)(rfl(__hip_atomic_load(p, __ATOMIC_RELAXED, __HIP_MEMORY_SCOPE_SYSTEM)) - target) < 0) {
        if (give_up(t0, tmo, call)) break;
        __builtin_amdgcn_s_sleep(4);
    }
}

constexpr unsigned MAX_EXC = 31;
constexpr unsigned RAWB = 3584, AUX_U = 20, AUX_MAX = 2560, U_MAX = 10240, LANE_U_MAX = 224;
constexpr unsigned ENC_HIST = AUX_MAX + 16, ENC_SORT = ENC_HIST + 256, ENC_END = ENC_SORT + 128;
constexpr unsigned DEC_POS = 6144;

__device__ __forceinline__ unsigned bfi_c(unsigned mask, unsigned a, unsigned b) { unsigned r; asm("v_bfi_b32 %0, %1, %2, %3" : "=v"(r) : "s"(mask), "v"(a), "v"(b)); return r; }
__device__ __forceinline__ unsigned opaque(unsigned v) { asm("" : "+v"(v)); return v; }
__device__ __forceinline__ unsigned bytemask(unsigned x, int sh, unsigned c0c) {
    unsigned t; asm("v_and_or_b32 %0, %1, %2, %3" : "=v"(t) : "v"(x >> sh), "s"(0x01010101u), "v"(c0c));
    return perm(0u, 0u, t);
}
template <int CTRL, int RM = 0xf> __device__ __forceinline__ unsigned dpp0(unsigned v) { return (unsigned)__builtin_amdgcn_update_dpp(0, (int)v, CTRL, RM, 0xf, RM == 0xf); }
__device__ __forceinline__ unsigned wave_scan_incl(unsigned v) {
    v += dpp0<0x111>(v); v += dpp0<0x112>(v); v += dpp0<0x114>(v); v += dpp0<0x118>(v);
    v += dpp0<0x142, 0xa>(v); v += dpp0<0x143, 0xc>(v);
    return v;
}
__device__ __forceinline__ unsigned umin(unsigned a, unsigned b) { return a < b ? a : b; }
__device__ __forceinline__ unsigned umax(unsigned a, unsigned b) { return a > b ? a : b; }
__device__ __forceinline__ unsigned sad4(unsigned x) { return __builtin_amdgcn_sad_u8(x, 0u, 0u); }
__device__ __forceinline__ unsigned lut2x16(const unsigned (&T)[8], unsigned idx, unsigned ms, unsigned c0c) {
    const unsigned sel = idx & 0x07070707u, m8 = bytemask(idx, 3, c0c);
    const unsigned p = bfi(m8, perm(T[3], T[2], sel), perm(T[1], T[0], sel)), n = bfi(m8, perm(T[7], T[6], sel), perm(T[5], T[4], sel));
    return bfi(ms, n, p);
}
#define LDS_FENCE() asm volatile("" ::: "memory")
__device__ __forceinline__ void lds_or2(LAS unsigned char* scr, unsigned p, unsigned w) {
    const unsigned s = p & 31u;
    LAS unsigned* q = (LAS unsigned*)(scr + ((p >> 5) << 2));
    __hip_atomic_fetch_or(q, w << s, __ATOMIC_RELAXED, __HIP_MEMORY_SCOPE_WORKGROUP);
    __hip_atomic_fetch_or(q + 1, (w >> 1) >> (31u - s), __ATOMIC_RELAXED, __HIP_MEMORY_SCOPE_WORKGROUP);
}

__device__ __forceinline__ unsigned encode_block(const u32x4 (&c)[8], unsigned (&P)[14], u32x4 (&a)[3], LAS unsigned char* scr, int lane) {
    asm volatile("" : "+v"(lane));
    const unsigned c0c = opaque(0x0C0C0C0Cu);
    unsigned E[16], R[16];
    u16x2 p1 = {0, 0}, p2 = {0, 0}, n1 = {0, 0}, n2 = {0, 0};
#pragma unroll
    for (int g = 0; g < 16; ++g) {
        const unsigned x = (g & 1) ? c[g >> 1].z : c[g >> 1].x, y = (g & 1) ? c[g >> 1].w : c[g >> 1].y;
        const unsigned H = perm(y, x, 0x07050301u), L = perm(y, x, 0x06040200u);
        E[g] = bfi_c(0xFEFEFEFEu, H << 1, L >> 7);
        R[g] = bfi_c(0x80808080u, L << 1, bfi_c(0x40404040u, H >> 1, L));
        const unsigned en = E[g] & bytemask(H, 7, c0c), ep = E[g] ^ en;
        p1 = __builtin_elementwise_max(p1, __builtin_bit_cast(u16x2, ep)); p2 = __builtin_elementwise_max(p2, __builtin_bit_cast(u16x2, ep << 8));
        n1 = __builtin_elementwise_max(n1, __builtin_bit_cast(u16x2, en)); n2 = __builtin_elementwise_max(n2, __builtin_bit_cast(u16x2, en << 8));
        __builtin_amdgcn_sched_barrier(0);
    }
    const u32x4 z4 = {0u, 0u, 0u, 0u};
    *(LAS u32x4*)(scr + lane * 16) = z4; *(LAS u32x4*)(scr + 1024 + lane * 16) = z4;
    if (lane < 49) *(LAS u32x4*)(scr + 2048 + lane * 16) = z4;
    unsigned baseP, baseN;
    {
        const u16x2 mp = __builtin_elementwise_max(p1, p2), mn = __builtin_elementwise_max(n1, n2);
        u16x2 m = {(unsigned short)((mp.x > mp.y ? mp.x : mp.y) >> 8), (unsigned short)((mn.x > mn.y ? mn.x : mn.y) >> 8)};
        m = __builtin_elementwise_max(m, __builtin_bit_cast(u16x2, dpp0<0x128>(__builtin_bit_cast(unsigned, m)))); m = __builtin_elementwise_max(m, __builtin_bit_cast(u16x2, dpp0<0x124>(__builtin_bit_cast(unsigned, m))));
        m = __builtin_elementwise_max(m, __builtin_bit_cast(u16x2, dpp0<0x122>(__builtin_bit_cast(unsigned, m)))); m = __builtin_elementwise_max(m, __builtin_bit_cast(u16x2, dpp0<0x121>(__builtin_bit_cast(unsigned, m))));
        const unsigned mu = __builtin_bit_cast(unsigned, m);
        const u16x2 r01 = __builtin_elementwise_max(__builtin_bit_cast(u16x2, (unsigned)__builtin_amdgcn_readlane(mu, 0)), __builtin_bit_cast(u16x2, (unsigned)__builtin_amdgcn_readlane(mu, 16)));
        const u16x2 r23 = __builtin_elementwise_max(__builtin_bit_cast(u16x2, (unsigned)__builtin_amdgcn_readlane(mu, 32)), __builtin_bit_cast(u16x2, (unsigned)__builtin_amdgcn_readlane(mu, 48)));
        const unsigned mm = rfl(__builtin_bit_cast(unsigned, __builtin_elementwise_max(r01, r23)));
        baseP = mm & 0xFFu; baseN = mm >> 16;
    }
    const unsigned BP = baseP * 0x01010101u, BN = baseN * 0x01010101u;
    unsigned any = 0, D[16];
#pragma unroll
    for (int g = 0; g < 16; ++g) {
        E[g] = bfi(bytemask(R[g], 6, c0c), BN, BP) - E[g]; any |= E[g];
        D[g] = bfi_c(0x1E1E1E1Eu, E[g] << 1, (R[g] >> 7) & 0x01010101u);
        R[g] &= 0x7F7F7F7Fu;
    }
    LDS_FENCE();
#pragma unroll
    for (int gi = 0; gi < 4; ++gi) {
        const int g = gi * 5;
        const unsigned k4 = ((D[g] & 0x0F0F0F0Fu) | ((R[g] >> 2) & 0x10101010u) | ((D[g] << 1) & 0x20202020u)) << 2;
#pragma unroll
        for (int b = 0; b < 4; ++b)
            __hip_atomic_fetch_add((LAS unsigned*)(scr + ENC_HIST + ((k4 >> (8 * b)) & 0xFFu)), 1u, __ATOMIC_RELAXED, __HIP_MEMORY_SCOPE_WORKGROUP);
    }
    LDS_FENCE();
    const unsigned cnt = *(LAS unsigned*)(scr + ENC_HIST + lane * 4);
    const unsigned key = (cnt << 4) | (15u - (lane & 15));
    unsigned rank = 0;
    rank += dpp0<0x121>(key) > key; rank += dpp0<0x122>(key) > key; rank += dpp0<0x123>(key) > key; rank += dpp0<0x124>(key) > key; rank += dpp0<0x125>(key) > key;
    rank += dpp0<0x126>(key) > key; rank += dpp0<0x127>(key) > key; rank += dpp0<0x128>(key) > key; rank += dpp0<0x129>(key) > key; rank += dpp0<0x12A>(key) > key;
    rank += dpp0<0x12B>(key) > key; rank += dpp0<0x12C>(key) > key; rank += dpp0<0x12D>(key) > key; rank += dpp0<0x12E>(key) > key; rank += dpp0<0x12F>(key) > key;
    unsigned tail = cnt; tail += dpp0<0x128>(tail); tail += dpp0<0x124>(tail); tail += dpp0<0x122>(tail); tail += dpp0<0x121>(tail);
    const int sg = (lane >> 4) & 1;
    const unsigned ct = sg ? __builtin_amdgcn_readlane(tail, 48) : __builtin_amdgcn_readlane(tail, 32);
    if (lane < 32) *(LAS unsigned*)(scr + ENC_SORT + ((lane & 16) + rank) * 4) = cnt;
    LDS_FENCE();
    unsigned best;
    {
        const unsigned cb = lane & 15, s0 = cb / 3u, s1 = cb - 3u * s0, A = 1u << s0, AB = A + (1u << s1);
        unsigned cost = ct * (16u - AB);
#pragma unroll
        for (int q = 0; q < 4; ++q) {
            const u32x4 h = *(LAS u32x4*)(scr + ENC_SORT + sg * 64 + q * 16);
#pragma unroll
            for (int j = 0; j < 4; ++j) {
                const unsigned r = q * 4 + j, len = r < A ? 1u + s0 : r < AB ? 2u + s1 : 4u + ((r - AB) >> 1);
                cost += 2u * len * h[j];
            }
        }
        best = cb < 9 ? (cost << 4) | cb : 0xFFFFFFFFu;
        best = umin(best, dpp0<0x128>(best)); best = umin(best, dpp0<0x124>(best)); best = umin(best, dpp0<0x122>(best)); best = umin(best, dpp0<0x121>(best));
    }
    const unsigned cbP = __builtin_amdgcn_readlane(best, 0) & 15u, cbN = __builtin_amdgcn_readlane(best, 16) & 15u;
    const unsigned s0P = cbP / 3u, s1P = cbP - 3u * s0P, s0N = cbN / 3u, s1N = cbN - 3u * s0N;
    const unsigned ABP = (1u << s0P) + (1u << s1P), ABN = (1u << s0N) + (1u << s1N);
    unsigned F[8];
    {
        const unsigned s0 = sg ? s0N : s0P, s1 = sg ? s1N : s1P, A = 1u << s0, AB = sg ? ABN : ABP;
        const unsigned ent = rank < A ? (rank << 4) | (s0 << 6) : rank < AB ? 1u | ((rank - A) << 4) | (s1 << 6) : (2u + ((rank - AB) >> 1)) | (((rank - AB) & 1u) << 4) | (1u << 6);
        unsigned fv = ent << (8 * (lane & 3));
        fv |= dpp0<0xB1>(fv); fv |= dpp0<0x4E>(fv);
#pragma unroll
        for (int q = 0; q < 8; ++q) F[q] = __builtin_amdgcn_readlane(fv, q * 4);
        if (lane < 32)
            __hip_atomic_fetch_or((LAS unsigned*)(scr + (sg * 2 + (rank >> 3)) * 4), (unsigned)(lane & 15) << (8 * (rank & 3) + 4 * ((rank >> 2) & 1)), __ATOMIC_RELAXED, __HIP_MEMORY_SCOPE_WORKGROUP);
        if (lane == 0) *(LAS unsigned*)(scr + 16) = baseP | (baseN << 8) | (s0P << 16) | (s1P << 18) | (s0N << 20) | (s1N << 22);
    }
    const unsigned ABP4 = ABP * 0x01010101u, ABN4 = ABN * 0x01010101u, S0P4 = s0P * 0x01010101u, S0N4 = s0N * 0x01010101u;
    unsigned KK[16], XW[4], OX[4], TU = 0, TX = 0, bad = 0, xl = 0, xw = 0;
#pragma unroll
    for (int g = 0; g < 16; ++g) {
        const unsigned ms = bytemask(R[g], 6, c0c), m16 = bytemask(D[g], 4, c0c), ent = lut2x16(F, D[g], ms, c0c);
        const unsigned q = ((D[g] | 0x80808080u) - bfi(ms, ABN4, ABP4)) & 0x7F7F7F7Fu;
        unsigned K = bfi(m16, ((q >> 1) & 0x3F3F3F3Fu) + 0x02020202u, ent & 0x0F0F0F0Fu);
        unsigned SUF = bfi(m16, q & 0x01010101u, (ent >> 4) & 0x03030303u);
        unsigned S = bfi(m16, 0x01010101u, (ent >> 6) & 0x03030303u);
        unsigned gl = sad4(K) + 4u;
        if (__ballot(gl > 31u) != 0ull) {
            any |= 0x80u;
            const unsigned km = umax(umax(K & 0xFFu, (K >> 8) & 0xFFu), umax((K >> 16) & 0xFFu, K >> 24));
            const unsigned eq = gl > 31u ? (((K ^ (km * 0x01010101u)) + 0x7F7F7F7Fu) & 0x80808080u) ^ 0x80808080u : 0u, em = bytemask(eq, 7, c0c);
            E[g] |= eq;
            K &= ~em; SUF &= ~em; S = bfi(em, bfi(ms, S0N4, S0P4), S);
            gl = sad4(K) + 4u;
        }
        KK[g] = K; TU += gl; bad = umax(bad, gl);
        xw |= (SUF & 0xFFu) << xl; xl += S & 0xFFu;
        xw |= ((SUF >> 8) & 0xFFu) << xl; xl += (S >> 8) & 0xFFu;
        xw |= ((SUF >> 16) & 0xFFu) << xl; xl += (S >> 16) & 0xFFu;
        xw |= (SUF >> 24) << (xl & 31u); xl += S >> 24;
        if ((g & 3) == 3) { XW[g >> 2] = xw; OX[g >> 2] = TX; TX += xl; xw = 0; xl = 0; }
        __builtin_amdgcn_sched_barrier(0);
    }
    bad = (bad > 31u) | (TU > LANE_U_MAX);
    const unsigned tt = TU | (TX << 16), incl = wave_scan_incl(tt), tot = __builtin_amdgcn_readlane(incl, 63), totU = tot & 0xFFFFu, totX = tot >> 16;
    const unsigned xstart = (AUX_U * 8u + totU + 31u) & ~31u, endbit = xstart + totX;
    unsigned raw = (__ballot(bad != 0) != 0ull) || totU > U_MAX || endbit > (AUX_MAX - 8u) * 8u;
    if (!raw) {
        const unsigned px = xstart + ((incl - tt) >> 16);
        unsigned p = AUX_U * 8u + ((incl - tt) & 0xFFFFu);
#pragma unroll
        for (int g = 0; g < 16; ++g) {
            const unsigned ln = KK[g] + 0x01010101u;
            unsigned pos = (ln & 0xFFu) - 1u, w = 1u << pos;
            pos += (ln >> 8) & 0xFFu; w |= 1u << pos;
            pos += (ln >> 16) & 0xFFu; w |= 1u << pos;
            pos += ln >> 24; w |= 1u << pos;
            lds_or2(scr, p, w); p += pos + 1u;
            __builtin_amdgcn_sched_barrier(0);
        }
#pragma unroll
        for (int q = 0; q < 4; ++q) lds_or2(scr, px + OX[q], XW[q]);
    }
#pragma unroll
    for (int i = 0; i < 7; ++i) { P[i] = R[i] | ((R[14] << (7 - i)) & 0x80808080u); P[7 + i] = R[7 + i] | ((R[15] << (7 - i)) & 0x80808080u); }
    unsigned n = 0;
    const unsigned xo = (endbit + 31u) >> 5;
    if (!raw && __ballot((any & 0xF0F0F0F0u) != 0) != 0ull) {
#pragma unroll
        for (int g = 0; g < 16; ++g) {
            const unsigned f = E[g] & 0xF0F0F0F0u;
            unsigned long long mask = __ballot(f != 0);
            while (mask != 0ull && n <= MAX_EXC) {
                const int L = __builtin_ctzll(mask); mask &= mask - 1;
                const unsigned fl = __builtin_amdgcn_readlane(f, L);
                const unsigned lo = __builtin_amdgcn_readlane((g & 1) ? c[g >> 1].z : c[g >> 1].x, L);
                const unsigned hi = __builtin_amdgcn_readlane((g & 1) ? c[g >> 1].w : c[g >> 1].y, L);
                const bool e = lane < 4 && ((fl >> (8 * (lane & 3))) & 0xFFu) != 0;
                const unsigned long long em = __ballot(e);
                const unsigned slot = xo + n + __builtin_popcountll(em & ((1ull << lane) - 1ull));
                const unsigned val = (((lane & 2) ? hi : lo) >> (16 * (lane & 1))) & 0xFFFFu;
                if (e && slot * 4u < AUX_MAX) *(LAS unsigned*)(scr + slot * 4u) = (unsigned)(L * 64 + g * 4 + (lane & 3)) | (val << 12);
                n += __builtin_popcountll(em);
            }
        }
    }
    raw |= (n > MAX_EXC) | ((xo + n) * 4u > AUX_MAX);
    LDS_FENCE();
    a[0] = *(LAS u32x4*)(scr + lane * 16); a[1] = *(LAS u32x4*)(scr + 1024 + lane * 16); a[2] = *(LAS u32x4*)(scr + 2048 + (lane & 31) * 16);
    return ((xo + n + 3u) >> 2) | (n << 8) | (xo << 16) | (raw << 31);
}

__device__ __forceinline__ void decode_block(LAS unsigned char* src, u32x4 (&c)[8], int lane) {
    LAS unsigned char* aux = src + RAWB;
    const unsigned c0c = opaque(0x0C0C0C0Cu);
    const u32x4 nt = *(LAS const u32x4*)(aux);
    const unsigned prm_v = *(LAS const unsigned*)(aux + 16);
    unsigned u[5];
#pragma unroll
    for (int i = 0; i < 5; ++i) u[i] = *(LAS const unsigned*)(aux + AUX_U + lane * 20 + i * 4);
    u32x4 P4[3];
#pragma unroll
    for (int p = 0; p < 3; ++p) P4[p] = *(LAS const u32x4*)(src + p * 1024 + lane * 16);
    const unsigned long long ph = *(LAS const unsigned long long*)(src + 3072 + lane * 8);
    const unsigned c0 = __builtin_popcount(u[0]), c1 = c0 + __builtin_popcount(u[1]), c2 = c1 + __builtin_popcount(u[2]), c3 = c2 + __builtin_popcount(u[3]), cn = c3 + __builtin_popcount(u[4]);
    const unsigned C = wave_scan_incl(cn) - cn;
    if (lane == 0) *(LAS unsigned*)(src + DEC_POS) = AUX_U * 8u;
    const int rounds = __ballot(cn > 128u) != 0ull ? 3 : 2;
    for (int j = 0; j < rounds; ++j) {
        const unsigned t = (C >> 6) + 1u + j, kth = 64u * t - C;
        const unsigned i = (kth > c0) + (kth > c1) + (kth > c2) + (kth > c3);
        unsigned rem = kth - (i == 0 ? 0u : i == 1 ? c0 : i == 2 ? c1 : i == 3 ? c2 : c3), w = i == 0 ? u[0] : i == 1 ? u[1] : i == 2 ? u[2] : i == 3 ? u[3] : u[4], pos = 0;
#pragma unroll
        for (int st = 16; st >= 1; st >>= 1) {
            const unsigned lo = __builtin_popcount(w & ((1u << st) - 1u));
            const bool up = rem > lo;
            rem -= up ? lo : 0u; w = up ? w >> st : w; pos += up ? st : 0;
        }
        if (kth <= cn && t < 64u) *(LAS unsigned*)(src + DEC_POS + t * 4) = AUX_U * 8u + lane * 160u + i * 32u + pos + 1u;
    }
    LDS_FENCE();
    const unsigned p0 = *(LAS const unsigned*)(src + DEC_POS + lane * 4);
    unsigned S[8];
    {
        LAS const unsigned* q = (LAS const unsigned*)(aux + ((p0 >> 5) << 2));
#pragma unroll
        for (int i = 0; i < 8; ++i) S[i] = q[i];
#pragma unroll
        for (int i = 0; i < 7; ++i) S[i] = __builtin_amdgcn_alignbit(S[i + 1], S[i], p0 & 31u);
    }
    unsigned R[16];
    {
        const unsigned P[14] = {P4[0].x, P4[0].y, P4[0].z, P4[0].w, P4[1].x, P4[1].y, P4[1].z, P4[1].w, P4[2].x, P4[2].y, P4[2].z, P4[2].w, (unsigned)ph, (unsigned)(ph >> 32)};
        unsigned x = 0, y = 0;
#pragma unroll
        for (int i = 0; i < 7; ++i) { x |= (P[i] >> (7 - i)) & (0x01010101u << i); y |= (P[7 + i] >> (7 - i)) & (0x01010101u << i); }
#pragma unroll
        for (int i = 0; i < 14; ++i) R[i] = P[i] & 0x7F7F7F7Fu;
        R[14] = x; R[15] = y;
    }
    const unsigned n0 = rfl(nt.x), n1 = rfl(nt.y), n2 = rfl(nt.z), n3 = rfl(nt.w), prm = rfl(prm_v);
    const unsigned I[8] = {n0 & 0x0F0F0F0Fu, (n0 >> 4) & 0x0F0F0F0Fu, n1 & 0x0F0F0F0Fu, (n1 >> 4) & 0x0F0F0F0Fu, n2 & 0x0F0F0F0Fu, (n2 >> 4) & 0x0F0F0F0Fu, n3 & 0x0F0F0F0Fu, (n3 >> 4) & 0x0F0F0F0Fu};
    const unsigned s0P = (prm >> 16) & 3u, s1P = (prm >> 18) & 3u, s0N = (prm >> 20) & 3u, s1N = (prm >> 22) & 3u;
    const unsigned STP = (s0P << 8) | (s1P << 16) | (1u << 24), STN = (s0N << 8) | (s1N << 16) | (1u << 24);
    const unsigned OTP = (4u << 8) | (((1u << s0P) + 2u) << 16) | (((1u << s0P) + (1u << s1P)) << 24);
    const unsigned OTN = (4u << 8) | (((1u << s0N) + 2u) << 16) | (((1u << s0N) + (1u << s1N)) << 24);
    const unsigned BP = (prm & 0xFFu) * 0x01010101u, BN = ((prm >> 8) & 0xFFu) * 0x01010101u;
    unsigned RB[16], SS[16], TU = 0, TX = 0;
#pragma unroll
    for (int g = 0; g < 16; ++g) {
        const unsigned w0 = S[0], q0 = __builtin_ctz(w0), w1 = w0 & (w0 - 1u), q1 = __builtin_ctz(w1), w2 = w1 & (w1 - 1u), q2 = __builtin_ctz(w2), w3 = w2 & (w2 - 1u), q3 = __builtin_ctz(w3);
        const unsigned Q = q0 | (q1 << 8) | (q2 << 16) | (q3 << 24);
        const unsigned LN = Q - opaque((Q << 8) - 1u), sh = q3 + 1u;
        TU += sh;
        if (g < 15) {
#pragma unroll
            for (int i = 0; i < 7; ++i) if (32 * i < (int)LANE_U_MAX - 4 * (g + 1)) S[i] = (i < 6 && 32 * (i + 1) < (int)LANE_U_MAX - 4 * g) ? __builtin_amdgcn_alignbit(S[i + 1], S[i], sh) : S[i] >> sh;
        }
        const unsigned sel = bfi_c(0x04040404u, R[g] >> 4, bfi(bytemask(LN + 0x7D7D7D7Du, 7, c0c), 0x03030303u, LN));
        SS[g] = perm(STN, STP, sel);
        RB[g] = perm(OTN, OTP, sel) + (LN << 1);
        TX = __builtin_amdgcn_sad_u8(SS[g], 0u, TX);
    }
    const unsigned xstart = (__builtin_amdgcn_readlane(p0 + TU, 63) + 31u) & ~31u, px = xstart + wave_scan_incl(TX) - TX;
    unsigned X[5];
    {
        LAS const unsigned* q = (LAS const unsigned*)(aux + ((px >> 5) << 2));
#pragma unroll
        for (int i = 0; i < 5; ++i) X[i] = q[i];
#pragma unroll
        for (int i = 0; i < 4; ++i) X[i] = __builtin_amdgcn_alignbit(X[i + 1], X[i], px & 31u);
    }
    unsigned xoff = 0;
#pragma unroll
    for (int g = 0; g < 16; ++g) {
        const unsigned s = SS[g], M = perm(0u, 0x00030100u, s);
        const unsigned y = (g & 1) ? X[0] >> xoff : X[0], Y = perm(y, y, 0x01000100u);
        const unsigned t1 = opaque(s << 8), t2 = opaque((t1 << 8) + t1), Cs = (t2 << 16) + t2;
        const u16x2 z02 = __builtin_bit_cast(u16x2, Y) >> __builtin_bit_cast(u16x2, Cs & 0x00FF00FFu);
        const u16x2 z13 = __builtin_bit_cast(u16x2, Y) >> __builtin_bit_cast(u16x2, perm(0u, Cs, 0x0C030C01u));
        const unsigned suf = perm(__builtin_bit_cast(unsigned, z13), __builtin_bit_cast(unsigned, z02), 0x06020400u) & M;
        if (g & 1) {
            if (g < 15) {
                const unsigned t = __builtin_amdgcn_sad_u8(s, 0u, xoff);
#pragma unroll
                for (int i = 0; i < 4; ++i) if (32 * i < 8 * (15 - g)) X[i] = (i < 3 && 32 * (i + 1) < 8 * (17 - g)) ? __builtin_amdgcn_alignbit(X[i + 1], X[i], t) : X[i] >> t;
            }
        } else xoff = sad4(s);
        const unsigned rank = RB[g] + suf - 0x06060606u, ms = bytemask(R[g], 6, c0c);
        const unsigned D = bfi(bytemask(rank, 4, c0c), rank, lut2x16(I, rank, ms, c0c));
        const unsigned E = bfi(ms, BN, BP) - ((D >> 1) & 0x0F0F0F0Fu);
        const unsigned H = bfi_c(0x80808080u, R[g] << 1, E >> 1), L = bfi_c(0x3F3F3F3Fu, R[g], bfi_c(0x40404040u, D << 6, E << 7));
        const unsigned x = perm(H, L, 0x05010400u), yv = perm(H, L, 0x07030602u);
        if (g & 1) { c[g >> 1].z = x; c[g >> 1].w = yv; } else { c[g >> 1].x = x; c[g >> 1].y = yv; }
    }
}

typedef unsigned u32x2 __attribute__((ext_vector_type(2)));
__device__ __forceinline__ void store_granule(char* p, u32x2 g) {
    asm volatile("global_store_dwordx2 %0, %1, off sc0 sc1\n\ts_nop 1" :: "v"(p), "v"(g) : "memory");
}

constexpr unsigned GCTL = 131072;
__device__ __forceinline__ void publish_block(const Params& P, rsrc_t wsr, const u32x4 (&c)[8], unsigned slot, unsigned n, unsigned dmask, unsigned call, LAS unsigned char* lds, LAS unsigned char* scr,
                                              unsigned j, int wid, int lane) {
    unsigned R[14], h = 1u << 31; u32x4 a[3];
    if (n >= NRAW && n < NSEQ - NRAWEND) h = encode_block(c, R, a, scr, lane);
    const unsigned so = OFF_SLOT + slot * 8192u;
    if (h >> 31) {
#pragma unroll
        for (int q = 0; q < 8; ++q) __builtin_amdgcn_raw_buffer_store_b128(c[q], wsr, lane * 16 + q * 1024, so, AUX_SYS);
    } else {
        const unsigned n16 = h & 255u;
#pragma unroll
        for (int q = 0; q < 3; ++q) __builtin_amdgcn_raw_buffer_store_b128((u32x4){R[4 * q], R[4 * q + 1], R[4 * q + 2], R[4 * q + 3]}, wsr, lane * 16 + q * 1024, so, AUX_SYS);
        __builtin_amdgcn_raw_buffer_store_b64((u32x2){R[12], R[13]}, wsr, lane * 8 + 3072, so, AUX_SYS);
        if ((unsigned)lane < n16) __builtin_amdgcn_raw_buffer_store_b128(a[0], wsr, lane * 16 + RAWB, so, AUX_SYS);
        if ((unsigned)lane + 64u < n16) __builtin_amdgcn_raw_buffer_store_b128(a[1], wsr, lane * 16 + RAWB + 1024, so, AUX_SYS);
        if ((unsigned)lane + 128u < n16) __builtin_amdgcn_raw_buffer_store_b128(a[2], wsr, lane * 16 + RAWB + 2048, so, AUX_SYS);
    }
    WAIT_VM0();
    const unsigned u = j & 3u, gsize = j == 0 ? 4u : 8u;
    volatile LAS unsigned* ctl = (volatile LAS unsigned*)(lds + GCTL);
    while (j >= 4u && (int)(ctl[36 + u] - (j - 3u)) < 0) __builtin_amdgcn_s_sleep(1);
    unsigned arrived = 0;
    if (lane == 0) {
        ctl[u * 8 + wid] = h;
        arrived = __hip_atomic_fetch_add((LAS unsigned*)(lds + GCTL) + 32 + u, 1u, __ATOMIC_RELAXED, __HIP_MEMORY_SCOPE_WORKGROUP);
    }
    if (rfl(arrived) == gsize - 1u) {
        const unsigned d = lane >> 3, w = lane & 7, hw = ctl[u * 8 + w];
        if (w < gsize && ((dmask >> d) & 1u)) store_granule(P.ws[d] + OFF_GRAN + ((unsigned)P.rank * NSEQ + n - wid + w) * 8u, (u32x2){call, hw});
        if (lane == 0) { ctl[32 + u] = 0u; ctl[36 + u] = j + 1u; }
    }
}

__device__ __forceinline__ void packer_role(const Params& P, LAS unsigned char* lds, unsigned& ticket, int first, int next, int step, int end) {
    const int lane = threadIdx.x & 63, wid = rfl(threadIdx.x >> 6);
    char* ws = P.ws[P.rank];
    const rsrc_t wsr = mk_rsrc(ws, WS_USED);
    const rsrc_t xr = mk_rsrc(P.x, DIM * KSH * 2), wr = mk_rsrc(P.w, DIM * KSH * 2);
    const int myi = P.rank >> 2, myj = P.rank & 3;
    LAS unsigned char* my = lds + wid * 8192;
    unsigned j = first < NFIRST ? 0u : 1u;
    for (int p = first; p < end; p = p < NFIRST ? next : p + step, ++j) {
        bool is_w; int kk, sel, r, bi;
        if (p < 3072) {
            kk = p / 192; const int rem = p - kk * 192;
            bi = rem & 31; r = 0; is_w = rem >= 64; sel = is_w ? ((rem - 64) >> 5) : (rem >> 5);
        }
        else { const int q = p - 3072; kk = q >> 6; sel = (q >> 5) & 1; bi = q & 31; r = 1; is_w = false; }
        u32x4 c[8];
        if (!is_w) {
            const int rr = lane >> 3;
            const unsigned vo = (unsigned)rr * 2048u + (unsigned)(((lane & 7) ^ rr) * 16);
            const unsigned so = (unsigned)(sel * 4096 + r * 2048 + bi * 64) * 2048u + (unsigned)kk * 128u;
#pragma unroll
            for (int j = 0; j < 8; ++j) c[j] = __builtin_amdgcn_raw_buffer_load_b128(xr, vo + j * 8 * 2048, so, 0);
        } else {
            const int pn = bi >> 2, bj = (bi >> 1) & 1, q = bi & 1;
            const int gk = lane >> 3, cc = lane & 7, wcl = cc >> 2, cq = cc & 3;
            const unsigned vo = (unsigned)(gk * 8) * 16384u + (unsigned)((64 * (2 * q + wcl) + 32 * bj + 8 * cq) * 2);
            const unsigned so = (unsigned)(kk * 64) * 16384u + (unsigned)((sel * 2048 + pn * 256) * 2);
            u32x4 R[8];
#pragma unroll
            for (int jj = 0; jj < 8; ++jj) R[jj] = __builtin_amdgcn_raw_buffer_load_b128(wr, vo + jj * 16384, so, 0);
#pragma unroll
            for (int j = 0; j < 8; ++j) {
                const unsigned s = (j & 1) ? 0x07060302u : 0x05040100u;
                u32x4 t;
                t.x = perm(R[1][j >> 1], R[0][j >> 1], s); t.y = perm(R[3][j >> 1], R[2][j >> 1], s);
                t.z = perm(R[5][j >> 1], R[4][j >> 1], s); t.w = perm(R[7][j >> 1], R[6][j >> 1], s);
                const int s64 = 32 * wcl + 16 * (j >> 2) + 4 * cq + (j & 3);
                *(LAS u32x4*)(my + s64 * 128 + ((gk ^ (s64 & 7)) * 16)) = t;
            }
            asm volatile("s_waitcnt lgkmcnt(0)" ::: "memory");
#pragma unroll
            for (int j = 0; j < 8; ++j) c[j] = *(LAS u32x4*)(my + j * 1024 + lane * 16);
            asm volatile("s_waitcnt lgkmcnt(0)" ::: "memory");
        }
        const unsigned call = call_get(ticket);
        const int g = P.rank * 16 + kk;
        const bool self = is_w ? (sel == myj) : (sel == myi);
        if (self) {
            const unsigned so = is_w ? OFF_BIMG + (unsigned)(g * 32 + bi) * 8192u : OFF_AIMG + (unsigned)((r * 128 + g) * 32 + bi) * 8192u;
#pragma unroll
            for (int j = 0; j < 8; ++j) __builtin_amdgcn_raw_buffer_store_b128(c[j], wsr, lane * 16 + j * 1024, so, AUX_SC1);
        }
        const unsigned slot = is_w ? 2048u + (unsigned)((sel * 16 + kk) * 32 + bi) : (unsigned)(((sel * 2 + r) * 16 + kk) * 32 + bi);
        const unsigned n = is_w ? (unsigned)(kk * 64 + 32 + bi) : (r == 0 ? (unsigned)(kk * 64 + bi) : (unsigned)(1024 + kk * 32 + bi));
        unsigned dmask = is_w ? (0x11u << sel) : (0x0Fu << (4 * sel));
        dmask &= ~(1u << P.rank);
        publish_block(P, wsr, c, slot, n, dmask, call, lds, my, j, wid, lane);
        if (self && lane == 0)
            __hip_atomic_fetch_add((unsigned*)(ws + OFF_CNT + (r * 128 + g) * 128), 1u, __ATOMIC_RELAXED, __HIP_MEMORY_SCOPE_SYSTEM);
    }
}

__device__ __forceinline__ u32x2 load_granule(rsrc_t wsr, unsigned off) { return __builtin_amdgcn_raw_buffer_load_b64(wsr, 0, off, AUX_SYS); }

struct Blk { unsigned slot, dso, cnt; bool is_out; };
__device__ __forceinline__ Blk locate(unsigned n, int src, int myi, int myj) {
    Blk b; b.is_out = false;
    if (n < 1024) { const int kk = n >> 6, bi = n & 31, g = src * 16 + kk; b.cnt = OFF_CNT + g * 128;
        if ((n & 63) >= 32) { b.slot = 2048u + (myj * 16 + kk) * 32 + bi; b.dso = OFF_BIMG + (unsigned)(g * 32 + bi) * 8192u; }
        else { b.slot = ((myi * 2) * 16 + kk) * 32 + bi; b.dso = OFF_AIMG + (unsigned)(g * 32 + bi) * 8192u; } }
    else if (n < 1536) { const int kk = (n - 1024) >> 5, bi = n & 31, g = src * 16 + kk; b.cnt = OFF_CNT + (128 + g) * 128;
        b.slot = ((myi * 2 + 1) * 16 + kk) * 32 + bi; b.dso = OFF_AIMG + (unsigned)((128 + g) * 32 + bi) * 8192u; }
    else { const int o = n - 1536, w = o & 7, h = (o >> 3) & 1, tile = (o >> 4) & 63, r = o >> 10; b.cnt = 0; b.is_out = true;
        b.slot = 4096u + o;
        const unsigned row0 = (src >> 2) * 4096 + r * 2048 + (tile >> 3) * 256 + h * 128 + (w >> 2) * 64, col0 = (src & 3) * 2048 + (tile & 7) * 256 + (w & 3) * 64;
        b.dso = (row0 * DIM + col0) * 2u; }
    return b;
}
__device__ __forceinline__ void signal_cnt(char* ws, unsigned off, int lane) {
    if (off && lane == 0) __hip_atomic_fetch_add((unsigned*)(ws + off), 1u, __ATOMIC_RELAXED, __HIP_MEMORY_SCOPE_SYSTEM);
}
constexpr unsigned NSLOT = 15, CTL = NSLOT * 8192;
constexpr unsigned M_DEAD = 1u << 30, N_STOP = 0xFFFFFFFFu;
__device__ __forceinline__ unsigned lds_ld(LAS unsigned char* p) { return *(volatile LAS unsigned*)p; }
__device__ __forceinline__ void lds_st(LAS unsigned char* p, unsigned v) { *(volatile LAS unsigned*)p = v; }
__device__ __forceinline__ void ring_publish(LAS unsigned char* lds, unsigned i, unsigned n, unsigned h, int lane) {
    const unsigned sl = i % NSLOT;
    if (lane == 0) { *(volatile LAS u32x2*)(lds + CTL + sl * 8) = (u32x2){n, h}; lds_st(lds + CTL + 256 + sl * 4, i + 1u); }
}
__device__ __forceinline__ void dma_op(rsrc_t r, LAS unsigned char* dst, int j, unsigned so, int lane) {
    __builtin_amdgcn_raw_ptr_buffer_load_lds(r, (LAS void*)(dst + j * 1024), 16, lane * 16 + j * 1024, so, 0, AUX_SYS);
}
#define VM_WAIT_(n) asm volatile("s_waitcnt vmcnt(" #n ")" ::: "memory")
#define VM_WAIT(n) VM_WAIT_(n)
#define WIN 5

__device__ __forceinline__ void loader_wave(const Params& P, LAS unsigned char* lds, unsigned call, int src, int lane) {
    char* ws = P.ws[P.rank];
    unsigned* tmo = (unsigned*)(ws + OFF_TMO);
    const rsrc_t wsr = mk_rsrc(ws, WS_USED);
    char* const remp = P.ws[src];
    const int myi = P.rank >> 2, myj = P.rank & 3;
    const unsigned gbase = OFF_GRAN + (unsigned)src * NSEQ * 8u;
    char* const gptr = ws + gbase;
    const unsigned tko = OFF_LINK + (unsigned)src * 128u, tbase = (call - 1u) * (NSEQ + 3u * PWL);
    __builtin_amdgcn_s_setprio(3);
    unsigned n, n1, n2;
    {
        unsigned t0 = 0;
        if (lane == 0) t0 = __hip_atomic_fetch_add((unsigned*)(ws + tko), 3u, __ATOMIC_RELAXED, __HIP_MEMORY_SCOPE_AGENT);
        n = rfl(t0) - tbase; n1 = n + 1; n2 = n + 2;
    }
    {
        const rsrc_t rw = mk_rsrc(remp, OFF_AIMG);
        const unsigned v = __builtin_amdgcn_raw_buffer_load_b32(rw, lane < 25 ? OFF_SLOT + (unsigned)lane * (2u << 20) + 8188u - (lane == 24 ? 8192u : 0u) : 0xFFFFFFF0u, 0, AUX_SYS);
        asm volatile("" :: "v"(v));
    }
    unsigned gx, gy, hx, hy;
    { const u32x2 a = load_granule(wsr, gbase + (n < NSEQ ? n : 0u) * 8u), b = load_granule(wsr, gbase + (n1 < NSEQ ? n1 : 0u) * 8u);
      gx = rfl(a.x); gy = rfl(a.y); hx = rfl(b.x); hy = rfl(b.y); }
    const int zero = 0, one = 1;
    unsigned i = 0;
    bool pend = false; unsigned pn = 0, ph = 0;
    while (n < NSEQ) {
        const unsigned sl = i % NSLOT;
        bool dead = false;
        const bool need_gran = (int)(gx - call) < 0;
        const bool need_slot = i >= NSLOT && (int)(lds_ld(lds + CTL + 320 + sl * 4) - (i - NSLOT + 1u)) < 0;
        if (need_gran || need_slot) {
            if (pend) { VM_WAIT(0); ring_publish(lds, i - 1u, pn, ph, lane); pend = false; }
            const unsigned long long t0 = __builtin_amdgcn_s_memrealtime();
            while ((int)(gx - call) < 0) {
                const u32x2 a = load_granule(wsr, gbase + n * 8u);
                gx = rfl(a.x); gy = rfl(a.y);
                if ((int)(gx - call) >= 0) break;
                if (give_up(t0, tmo, call)) { dead = true; break; }
                __builtin_amdgcn_s_sleep(2);
                asm volatile("" ::: "memory");
            }
            while (i >= NSLOT && (int)(lds_ld(lds + CTL + 320 + sl * 4) - (i - NSLOT + 1u)) < 0) __builtin_amdgcn_s_sleep(1);
        }
        const bool raw = (gy >> 31) != 0;
        const unsigned so = OFF_SLOT + locate(n, src, myi, myj).slot * 8192u;
                const unsigned bytes = dead ? 0u : raw ? 8192u : RAWB + ((gy & 255u) << 4);
        const rsrc_t rem = mk_rsrc(remp, so + bytes);
        LAS unsigned char* dst = lds + sl * 8192;
        u32x2 g2; unsigned tk = 0;
        asm volatile("global_load_dwordx2 %0, %1, %2 sc0 sc1" : "=v"(g2) : "v"(zero), "s"(gptr + (n2 < NSEQ ? n2 : 0u) * 8u) : "memory");
        if (lane == 0) asm volatile("global_atomic_add %0, %1, %2, %3 sc0" : "=v"(tk) : "v"(zero), "v"(one), "s"(ws + tko) : "memory");
        VM_WAIT(WIN); dma_op(rem, dst, 0, so, lane);
        VM_WAIT(WIN); dma_op(rem, dst, 1, so, lane);
        VM_WAIT(WIN); dma_op(rem, dst, 2, so, lane);
        VM_WAIT(WIN); if (pend) ring_publish(lds, i - 1u, pn, ph, lane); dma_op(rem, dst, 3, so, lane);
        VM_WAIT(WIN); dma_op(rem, dst, 4, so, lane);
        asm volatile("s_waitcnt vmcnt(5)" : "+v"(g2), "+v"(tk) :: "memory"); dma_op(rem, dst, 5, so, lane);
        if (raw) { dma_op(rem, dst, 6, so, lane); dma_op(rem, dst, 7, so, lane); VM_WAIT(0); }
        pend = true; pn = n; ph = gy | (dead ? M_DEAD : 0u);
        n = n1; gx = hx; gy = hy;
        n1 = n2; hx = rfl(g2.x); hy = rfl(g2.y);
        n2 = rfl(tk) - tbase;
        ++i;
    }
    VM_WAIT(0);
    if (pend) ring_publish(lds, i - 1u, pn, ph, lane);
    for (int d = 0; d < NDEC; ++d, ++i) {
        const unsigned sl = i % NSLOT;
        while (i >= NSLOT && (int)(lds_ld(lds + CTL + 320 + sl * 4) - (i - NSLOT + 1u)) < 0) __builtin_amdgcn_s_sleep(1);
        ring_publish(lds, i, N_STOP, 0u, lane);
    }
}

__device__ __forceinline__ void decoder_wave(const Params& P, LAS unsigned char* lds, unsigned call, int src, int d, int lane) {
    char* ws = P.ws[P.rank];
    unsigned* tmo = (unsigned*)(ws + OFF_TMO);
    const rsrc_t wsr = mk_rsrc(ws, WS_USED), outr = mk_rsrc(P.out, (unsigned)DIM * DIM * 2u);
    const int myi = P.rank >> 2, myj = P.rank & 3;
    for (unsigned i = d;; i += NDEC) {
        const unsigned sl = i % NSLOT;
        while (lds_ld(lds + CTL + 256 + sl * 4) != i + 1u) __builtin_amdgcn_s_sleep(1);
        asm volatile("" ::: "memory");
        const u32x2 meta = *(volatile LAS u32x2*)(lds + CTL + sl * 8);
        const unsigned n = rfl(meta.x), h = rfl(meta.y);
        if (n == N_STOP) break;
        const Blk b = locate(n, src, myi, myj);
        const bool raw = (h >> 31) != 0;
        bool dead = (h & M_DEAD) != 0;
        if (b.is_out && rfl(__hip_atomic_load(tmo, __ATOMIC_RELAXED, __HIP_MEMORY_SCOPE_AGENT)) == call) dead = true;
        u32x4 c[8];
        unsigned recs = 0;
        const unsigned ne = (dead || raw) ? 0u : (h >> 8) & 63u;
        LAS unsigned char* cur = lds + sl * 8192;
        if (dead) {
#pragma unroll
            for (int j = 0; j < 8; ++j) c[j] = (u32x4){0x7FC07FC0u, 0x7FC07FC0u, 0x7FC07FC0u, 0x7FC07FC0u};
        } else if (raw) {
#pragma unroll
            for (int j = 0; j < 8; ++j) c[j] = *(LAS u32x4*)(cur + j * 1024 + lane * 16);
        } else {
            recs = *(LAS unsigned*)(cur + RAWB + (((h >> 16) & 0x3FFu) + (lane & 31)) * 4u);
            decode_block(cur, c, lane);
        }
#pragma unroll
        for (int j = 0; j < 8; ++j) asm volatile("" : "+v"(c[j]));
        asm volatile("" : "+v"(recs));
        if (lane == 0) lds_st(lds + CTL + 320 + sl * 4, i + 1u);
        const rsrc_t dr = b.is_out ? outr : wsr;
        const unsigned sj = b.is_out ? 8u * DIM * 2u : 1024u, sr = b.is_out ? DIM * 2u : 128u;
        const unsigned vo = (unsigned)(lane >> 3) * sr + (unsigned)(lane & 7) * 16u;
#pragma unroll
        for (int j = 0; j < 8; ++j) __builtin_amdgcn_raw_buffer_store_b128(c[j], dr, vo, b.dso + j * sj, AUX_SC1);
        for (unsigned e = 0; e < ne; ++e) {
            const unsigned rec = __builtin_amdgcn_readlane(recs, e), pos = rec & 0xFFFu, v = pos & 63u;
            if ((unsigned)lane == (pos >> 6)) __builtin_amdgcn_raw_buffer_store_b16((unsigned short)((rec >> 12) & 0xFFFFu), dr, vo + (v & 7u) * 2u, b.dso + (v >> 3) * sj, AUX_SC1);
        }
        if (b.cnt) { WAIT_VM0(); signal_cnt(ws, b.cnt, lane); }
    }
}

__device__ __forceinline__ void puller_role(const Params& P, LAS unsigned char* lds, unsigned call, int wg) {
    const int lane = threadIdx.x & 63, wid = rfl(threadIdx.x >> 6);
    const int link = wg % 7, src = rfl((P.rank + 1 + link) & 7);
    if (threadIdx.x < 32) *(volatile LAS unsigned*)(lds + CTL + 256 + threadIdx.x * 4) = 0u;
    __syncthreads();
    if (wid == 0) loader_wave(P, lds, call, src, lane);
    else if (wid <= NDEC) decoder_wave(P, lds, call, src, wid - 1, lane);
}

__device__ __forceinline__ float gelu_f(float y) {
    const float c1 = -2.0f * 0.7978845608028654f * 1.4426950408889634f, c2 = c1 * 0.044715f;
    const float e = __builtin_amdgcn_exp2f(y * (c1 + c2 * y * y));
    return y * __builtin_amdgcn_rcpf(1.0f + e);
}
__device__ __forceinline__ unsigned cvt_pk_bf16(float lo, float hi) { unsigned r; asm("v_cvt_pk_bf16_f32 %0, %1, %2" : "=v"(r) : "v"(lo), "v"(hi)); return r; }

__device__ __forceinline__ void gemm_role(const Params& P, LAS unsigned char* lds, unsigned call, int wg) {
    const int tid = threadIdx.x, wid = rfl(tid >> 6), lane = tid & 63, wr = wid >> 2, wc = wid & 3, fr = lane & 15, fq = lane >> 4;
    char* ws = P.ws[P.rank];
    unsigned* tmo = (unsigned*)(ws + OFF_TMO);
    const rsrc_t wsr = mk_rsrc(ws, WS_USED);
    const int xcd = wg & 7, q = (wg - NPACK) >> 3, h = xcd & 1, blk = xcd >> 1;
    const int tm = (blk >> 1) * 4 + (q >> 2), tn = (blk & 1) * 4 + (q & 3), tile = tm * 8 + tn;
    const unsigned voff = (unsigned)tid * 16u, ldsw = (unsigned)wid * 1024u;
    const int aoff0 = (wr * 64 + fr) * 128 + ((fq ^ (fr & 7)) << 4), aoff1 = aoff0 ^ 64;
    const int boff0 = (wc * 32 + fr) * 128 + ((fq ^ (fr & 7)) << 4), boff1 = boff0 ^ 64;
#define G_SA(b, hh) (((b) * 2 + (hh)) * 16384)
#define G_SB(b, hh) ((4 + (b) * 2 + (hh)) * 16384)
#define G_STAGE(bufoff, soff) do { _Pragma("unroll") for (int _i = 0; _i < 2; ++_i) \
        __builtin_amdgcn_raw_ptr_buffer_load_lds(wsr, (LAS void*)(lds + (bufoff) + ldsw + _i * 8192), 16, voff, (soff) + _i * 8192, 0, AUX_SYS); } while (0)
#define G_LDA(dst, b, hh) do { _Pragma("unroll") for (int m = 0; m < 4; ++m) { \
        dst[m][0] = *(const LAS bf16x8*)(lds + G_SA(b, hh) + aoff0 + m * 2048); dst[m][1] = *(const LAS bf16x8*)(lds + G_SA(b, hh) + aoff1 + m * 2048); } } while (0)
#define G_LDB(dst, b, hh) do { _Pragma("unroll") for (int n = 0; n < 2; ++n) { \
        dst[n][0] = *(const LAS bf16x8*)(lds + G_SB(b, hh) + boff0 + n * 2048); dst[n][1] = *(const LAS bf16x8*)(lds + G_SB(b, hh) + boff1 + n * 2048); } } while (0)
#define G_MMA(ai, bj, At, Bt) do { __builtin_amdgcn_s_setprio(1); _Pragma("unroll") for (int m = 0; m < 4; ++m) _Pragma("unroll") for (int n = 0; n < 2; ++n) _Pragma("unroll") for (int k = 0; k < 2; ++k) \
        acc[ai][bj][m][n] = __builtin_amdgcn_mfma_f32_16x16x32_bf16(Bt[n][k], At[m][k], acc[ai][bj][m][n], 0, 0, 0); __builtin_amdgcn_s_setprio(0); } while (0)
#define G_WAIT_V(n) asm volatile("s_waitcnt vmcnt(" #n ")" ::: "memory")
#define G_WAIT_L(n) asm volatile("s_waitcnt lgkmcnt(" #n ")" ::: "memory")
#define G_BAR __builtin_amdgcn_s_barrier()
#define G_SCHED __builtin_amdgcn_sched_barrier(0)
#define G_GIDX(t) ((unsigned)((4 * h + ((t) & 3)) * 16 + ((t) >> 2)))
    const int zero = 0;
    for (int r = 0; r < 2; ++r) {
        const unsigned target = call * (r == 0 ? 64u : 32u);
        const unsigned abase = OFF_AIMG + (unsigned)r * (128u * 262144u) + (unsigned)tm * 32768u, bbase = OFF_BIMG + (unsigned)tn * 32768u;
        char* cbase = ws + OFF_CNT + r * 128 * 128;
        f32x4 acc[2][2][4][2];
#pragma unroll
        for (int a = 0; a < 2; ++a)
#pragma unroll
            for (int b = 0; b < 2; ++b)
#pragma unroll
                for (int m = 0; m < 4; ++m)
#pragma unroll
                    for (int n = 0; n < 2; ++n) acc[a][b][m][n] = (f32x4){0.f, 0.f, 0.f, 0.f};
        bf16x8 At[4][2], B0[2][2], B1[2][2];
        STAMP(ws, r * 8 + 0);
        wait_word((unsigned*)(cbase + G_GIDX(0) * 128), target, tmo, call);
        wait_word((unsigned*)(cbase + G_GIDX(1) * 128), target, tmo, call);
        wait_word((unsigned*)(cbase + G_GIDX(2) * 128), target, tmo, call);
        STAMP(ws, r * 8 + 1);
        {
            const unsigned a0 = abase + G_GIDX(0) * 262144u, b0 = bbase + G_GIDX(0) * 262144u, a1 = abase + G_GIDX(1) * 262144u, b1 = bbase + G_GIDX(1) * 262144u;
            G_STAGE(G_SB(0, 0), b0); G_STAGE(G_SA(0, 0), a0); G_STAGE(G_SB(0, 1), b0 + 16384); G_STAGE(G_SA(0, 1), a0 + 16384);
            if (wr == 1) G_BAR;
            G_WAIT_V(4); G_BAR;
            G_STAGE(G_SB(1, 0), b1); G_STAGE(G_SA(1, 0), a1); G_STAGE(G_SB(1, 1), b1 + 16384);
            G_WAIT_V(6); G_BAR;
        }
        unsigned fA = target, fB;
        for (int t = 0; t < 64; t += 2) {
            const int t2 = t + 2 < 64 ? t + 2 : 63, t3 = t + 3 < 64 ? t + 3 : 63, t4 = t + 4 < 64 ? t + 4 : 63;
            const unsigned g1 = G_GIDX(t + 1), g2 = G_GIDX(t2), g3 = G_GIDX(t3), g4 = G_GIDX(t4);
            const unsigned a1 = abase + g1 * 262144u, a2 = abase + g2 * 262144u, b2 = bbase + g2 * 262144u, a3 = abase + g3 * 262144u, b3 = bbase + g3 * 262144u;
            asm volatile("global_load_dword %0, %1, %2 sc0 sc1" : "=v"(fB) : "v"(zero), "s"(cbase + g3 * 128));
            G_LDB(B0, 0, 0); G_SCHED; G_LDA(At, 0, 0); G_STAGE(G_SA(1, 1), a1 + 16384);
            G_WAIT_L(8); G_BAR; G_WAIT_L(0); G_MMA(0, 0, At, B0); G_BAR; G_SCHED;
            asm volatile("" : "+v"(fA));
            if ((int)(rfl(fA) - target) < 0) wait_word((unsigned*)(cbase + g2 * 128), target, tmo, call);
            G_LDB(B1, 0, 1); G_STAGE(G_SB(0, 0), b2);
            G_BAR; G_WAIT_L(0); G_MMA(0, 1, At, B1); G_BAR;
            G_LDA(At, 0, 1); G_STAGE(G_SA(0, 0), a2);
            G_BAR; G_WAIT_L(0); G_MMA(1, 0, At, B0); G_BAR; G_SCHED;
            G_STAGE(G_SB(0, 1), b2 + 16384);
            G_WAIT_V(6); G_BAR; G_MMA(1, 1, At, B1); G_BAR;
            asm volatile("global_load_dword %0, %1, %2 sc0 sc1" : "=v"(fA) : "v"(zero), "s"(cbase + g4 * 128));
            G_LDB(B0, 1, 0); G_SCHED; G_LDA(At, 1, 0); G_STAGE(G_SA(0, 1), a2 + 16384);
            G_WAIT_L(8); G_BAR; G_WAIT_L(0); G_MMA(0, 0, At, B0); G_BAR; G_SCHED;
            asm volatile("" : "+v"(fB));
            if ((int)(rfl(fB) - target) < 0) wait_word((unsigned*)(cbase + g3 * 128), target, tmo, call);
            G_LDB(B1, 1, 1); G_STAGE(G_SB(1, 0), b3);
            G_BAR; G_WAIT_L(0); G_MMA(0, 1, At, B1); G_BAR;
            G_LDA(At, 1, 1); G_STAGE(G_SA(1, 0), a3);
            G_BAR; G_WAIT_L(0); G_MMA(1, 0, At, B0); G_BAR; G_SCHED;
            G_STAGE(G_SB(1, 1), b3 + 16384);
            G_WAIT_V(6); G_BAR; G_MMA(1, 1, At, B1); G_BAR;
        }
        G_WAIT_V(0);
        asm volatile("" : "+v"(fA));
        if (wr == 0) G_BAR;
        G_BAR;
        STAMP(ws, r * 8 + 2);
        int lane = threadIdx.x & 63;
        asm volatile("" : "+v"(lane));
        const int fr = lane & 15, fq = lane >> 4;
        if (h) {
#pragma unroll
            for (int b = 0; b < 2; ++b)
#pragma unroll
                for (int m = 0; m < 4; ++m)
#pragma unroll
                    for (int n = 0; n < 2; ++n) { const f32x4 tq = acc[0][b][m][n]; acc[0][b][m][n] = acc[1][b][m][n]; acc[1][b][m][n] = tq; }
        }
        const unsigned xl = (unsigned)((r * 64 + tile) * 2);
#pragma unroll
        for (int b = 0; b < 2; ++b)
#pragma unroll
            for (int m = 0; m < 4; ++m)
#pragma unroll
                for (int n = 0; n < 2; ++n) {
                    __builtin_amdgcn_raw_buffer_store_b128(__builtin_bit_cast(u32x4, acc[1][b][m][n]), wsr, lane * 16 + ((b * 4 + m) * 2 + n) * 1024, OFF_XBUF + ((xl + h) * 8 + wid) * 16384u, AUX_SC1);
                    if (EPI_PACE) __builtin_amdgcn_s_sleep(EPI_PACE);
                }
        WAIT_VM0();
        if (lane == 0) __hip_atomic_store((unsigned*)(ws + OFF_XFLAG + (xl + h) * 128 + wid * 16), call, __ATOMIC_RELAXED, __HIP_MEMORY_SCOPE_AGENT);
        wait_word((unsigned*)(ws + OFF_XFLAG + (xl + (h ^ 1)) * 128 + wid * 16), call, tmo, call);
        const bool dead = rfl(__hip_atomic_load(tmo, __ATOMIC_RELAXED, __HIP_MEMORY_SCOPE_AGENT)) == call;
        STAMP(ws, r * 8 + 3);
        LAS unsigned char* my = lds + wid * 8192;
#pragma unroll
        for (int b = 0; b < 2; ++b)
#pragma unroll
            for (int m = 0; m < 4; ++m) {
                const f32x4 p0 = __builtin_bit_cast(f32x4, __builtin_amdgcn_raw_buffer_load_b128(wsr, lane * 16 + ((b * 4 + m) * 2 + 0) * 1024, OFF_XBUF + ((xl + (h ^ 1)) * 8 + wid) * 16384u, AUX_SC1));
                const f32x4 p1 = __builtin_bit_cast(f32x4, __builtin_amdgcn_raw_buffer_load_b128(wsr, lane * 16 + ((b * 4 + m) * 2 + 1) * 1024, OFF_XBUF + ((xl + (h ^ 1)) * 8 + wid) * 16384u, AUX_SC1));
                const f32x4 v0 = acc[0][b][m][0] + p0, v1 = acc[0][b][m][1] + p1;
                if (EPI_PACE) { asm volatile("" :: "v"(v0), "v"(v1)); __builtin_amdgcn_s_sleep(EPI_PACE); __builtin_amdgcn_sched_barrier(0); }
                u32x4 o;
                o.x = cvt_pk_bf16(gelu_f(v0[0]), gelu_f(v0[1])); o.y = cvt_pk_bf16(gelu_f(v0[2]), gelu_f(v0[3]));
                o.z = cvt_pk_bf16(gelu_f(v1[0]), gelu_f(v1[1])); o.w = cvt_pk_bf16(gelu_f(v1[2]), gelu_f(v1[3]));
                const int row = m * 16 + fr, ch = b * 4 + fq;
                *(LAS u32x4*)(my + row * 128 + ((ch ^ (row & 7)) * 16)) = o;
            }
        asm volatile("s_waitcnt lgkmcnt(0)" ::: "memory");
        u32x4 c[8];
#pragma unroll
        for (int j = 0; j < 8; ++j) { const int row = j * 8 + (lane >> 3); c[j] = *(LAS u32x4*)(my + row * 128 + (((lane & 7) ^ (row & 7)) * 16)); }
        if (dead) {
#pragma unroll
            for (int j = 0; j < 8; ++j) c[j] = (u32x4){0x7FC07FC0u, 0x7FC07FC0u, 0x7FC07FC0u, 0x7FC07FC0u};
        }
        {
            const rsrc_t outr = mk_rsrc(P.out, (unsigned)DIM * DIM * 2u);
            const unsigned row0 = (P.rank >> 2) * 4096 + r * 2048 + tm * 256 + h * 128 + wr * 64, col0 = (P.rank & 3) * 2048 + tn * 256 + wc * 64;
            const unsigned dso = (row0 * DIM + col0) * 2u, vo = (unsigned)(lane >> 3) * (DIM * 2u) + (unsigned)(lane & 7) * 16u;
#pragma unroll
            for (int j = 0; j < 8; ++j) __builtin_amdgcn_raw_buffer_store_b128(c[j], outr, vo, dso + j * (8u * DIM * 2u), AUX_SC1);
            const unsigned o = (xl + h) * 8 + wid;
            publish_block(P, wsr, c, 4096u + o, 1536u + o, 0xFFu & ~(1u << P.rank), call, lds, my, 1u + GROUNDS + r, wid, lane);
        }
        STAMP(ws, r * 8 + 4);
        asm volatile("s_waitcnt lgkmcnt(0)" ::: "memory");
        G_BAR;
    }
#undef G_SA
#undef G_SB
#undef G_STAGE
#undef G_LDA
#undef G_LDB
#undef G_MMA
#undef G_WAIT_V
#undef G_WAIT_L
#undef G_BAR
#undef G_SCHED
#undef G_GIDX
}

__global__ __launch_bounds__(512, 2) void gemm_allgather_kernel(Params P) {
    extern __shared__ __attribute__((aligned(16))) unsigned char shm[];
    LAS unsigned char* lds = (LAS unsigned char*)shm;
    char* ws = P.ws[P.rank];
    unsigned ticket = call_issue(ws);
    const int b = blockIdx.x;
    STAMP(ws, 60);
    unsigned call;
    if (b >= NPACK + NGEMM) { call = call_get(ticket); puller_role(P, lds, rfl(call), rfl(b - NPACK - NGEMM)); }
    else {
        if (threadIdx.x < 8) *(volatile LAS unsigned*)(lds + GCTL + 128 + threadIdx.x * 4) = 0u;
        __syncthreads();
        const int wid = rfl(threadIdx.x >> 6);
        const bool gemm = b >= NPACK;
        const int next = gemm ? NFIRST + (b - NPACK) * 8 + wid : NFIRST + GROUNDS * NGEMM * 8 + b * 8 + wid;
        packer_role(P, lds, ticket, wid < 4 ? b * 4 + wid : next, next, gemm ? NGEMM * 8 : NPACK * 8, gemm ? NFIRST + GROUNDS * NGEMM * 8 : 4096);
        call = call_get(ticket);
        if (gemm) { __syncthreads(); gemm_role(P, lds, rfl(call), rfl(b)); }
    }
    STAMP(ws, 61);
    STAMP(ws, 58 + (call & 1));
}

}

extern "C" void kernel_launch(int rank, int nranks, ncclComm_t comm,
                              hipStream_t stream,
                              void* const* d_in, const int* in_sizes, int n_in,
                              void* d_out, int out_size,
                              const int* in_dtypes, int out_dtype,
                              void* const* d_out_all,
                              void* const* d_ws_all, size_t ws_bytes) {
    (void)comm; (void)in_sizes; (void)in_dtypes; (void)out_dtype; (void)d_out_all;
    if (nranks != gar::WORLD || n_in < 2 || ws_bytes < gar::WS_NEED || out_size != gar::DIM * gar::DIM || (unsigned)rank >= (unsigned)gar::WORLD) {
        fprintf(stderr, "unexpected problem shape\n");
        return;
    }
    static bool attr = false;
    if (!attr) {
        hipError_t e = hipFuncSetAttribute((const void*)gar::gemm_allgather_kernel, hipFuncAttributeMaxDynamicSharedMemorySize, gar::LDS_BYTES);
        if (e != hipSuccess) { fprintf(stderr, "hipFuncSetAttribute: %s\n", hipGetErrorString(e)); return; }
        attr = true;
    }
    gar::Params P;
    memset(&P, 0, sizeof P);
    P.x = d_in[0]; P.w = d_in[1]; P.out = d_out; P.rank = rank;
    for (int p = 0; p < gar::WORLD; ++p) P.ws[p] = (char*)d_ws_all[p];
    hipLaunchKernelGGL(gar::gemm_allgather_kernel, dim3(gar::GRID), dim3(512), gar::LDS_BYTES, stream, P);
    hipError_t e = hipGetLastError();
    if (e != hipSuccess) fprintf(stderr, "launch: %s\n", hipGetErrorString(e));
}
```

```cpp
#include <hip/hip_runtime.h>
#include <rccl/rccl.h>
#include <cstdio>
#include <cstdint>
#include <cstring>

namespace gar {

#define LAS __attribute__((address_space(3)))
typedef unsigned u32x4 __attribute__((ext_vector_type(4)));
typedef unsigned short u16x2 __attribute__((ext_vector_type(2)));
typedef short bf16x8 __attribute__((ext_vector_type(8)));
typedef float f32x4 __attribute__((ext_vector_type(4)));
typedef __amdgpu_buffer_rsrc_t rsrc_t;

constexpr int WORLD = 8;
constexpr int DIM = 8192;
constexpr int KSH = 1024;
constexpr int NGEMM = 128;
constexpr int GRID = 256;
#define PULL_WG_PER_LINK 12
#define PULL_DECODERS 7
constexpr int PWL = PULL_WG_PER_LINK;
constexpr int NDEC = PULL_DECODERS;
constexpr int PWAVES = 1;
constexpr int NPULL = 7 * PWL;
constexpr int NPACK = GRID - NGEMM - NPULL;
#define NRAWEND 24
#define WARM_TLB 1
#define EPI_PACE 0
#define NRAW 48
constexpr int NFIRST = (NPACK + NGEMM) * 4;
#define GEMM_PACK_ROUNDS 1
constexpr int GROUNDS = GEMM_PACK_ROUNDS;
static_assert(NFIRST + GROUNDS * NGEMM * 8 <= 4096 && NFIRST % 8 == 0, "roles");
constexpr int NSEQ = 3584;
constexpr unsigned long long DEADLINE = 40000000ull;

constexpr unsigned OFF_TMO = 128;
constexpr unsigned OFF_LINK = 1024;
constexpr unsigned OFF_CNT = 4096;
constexpr unsigned OFF_XFLAG = OFF_CNT + 256 * 128;
constexpr unsigned OFF_OCNT = OFF_XFLAG + 256 * 128;
constexpr unsigned OFF_GRAN = 131072;
constexpr unsigned OFF_CALL = 640u << 10;
constexpr unsigned OFF_SLOT = 1u << 20;
constexpr unsigned OFF_AIMG = 64u << 20;
constexpr unsigned OFF_BIMG = 128u << 20;
constexpr unsigned OFF_XBUF = 160u << 20;
constexpr unsigned WS_USED = 192u << 20;
constexpr unsigned WS_NEED = WS_USED;
static_assert(OFF_OCNT + 128 * 128 <= OFF_GRAN, "map");
static_assert(OFF_GRAN + WORLD * NSEQ * 8 <= OFF_CALL && OFF_CALL + 2048 * 64 <= OFF_SLOT, "map");
static_assert(OFF_SLOT + 6144u * 8192u <= OFF_AIMG, "map");
constexpr int LDS_BYTES = 131072 + 256;

struct Params {
    const void* x; const void* w; void* out;
    char* ws[WORLD];
    int rank; int pad;
};

constexpr int AUX_SC1 = 16, AUX_SYS = 17;

__device__ __forceinline__ rsrc_t mk_rsrc(const void* p, unsigned bytes) {
    return __builtin_amdgcn_make_buffer_rsrc(const_cast<void*>(p), 0, bytes, 0x00020000);
}
__device__ __forceinline__ unsigned rfl(unsigned v) { return __builtin_amdgcn_readfirstlane(v); }
__device__ __forceinline__ unsigned bfi(unsigned mask, unsigned a, unsigned b) { return (a & mask) | (b & ~mask); }
__device__ __forceinline__ unsigned perm(unsigned hi, unsigned lo, unsigned sel) { return __builtin_amdgcn_perm(hi, lo, sel); }
#define WAIT_VM0() asm volatile("s_waitcnt vmcnt(0)" ::: "memory")

#define STAMP(ws, idx) do { } while (0)

__device__ __forceinline__ unsigned call_draw(char* ws) {
    unsigned t;
    const int zero = 0, one = 1;
    unsigned long long keep;
    asm volatile("s_mov_b64 %1, exec\n\ts_mov_b64 exec, 1\n\tglobal_atomic_add %0, %2, %3, %4 sc0\n\ts_mov_b64 exec, %1\n\ts_waitcnt vmcnt(0)"
                 : "=&v"(t), "=&s"(keep) : "v"(zero), "v"(one), "s"(ws + OFF_CALL + (blockIdx.x * 8 + rfl(threadIdx.x >> 6)) * 64) : "memory");
    return rfl(t) + 1u;
}

__device__ __forceinline__ bool give_up(unsigned long long t0, unsigned* tmo, unsigned call) {
    if (rfl(__hip_atomic_load(tmo, __ATOMIC_RELAXED, __HIP_MEMORY_SCOPE_AGENT)) == call) return true;
    if (__builtin_amdgcn_s_memrealtime() - t0 > DEADLINE) { __hip_atomic_store(tmo, call, __ATOMIC_RELAXED, __HIP_MEMORY_SCOPE_AGENT); return true; }
    return false;
}
__device__ __forceinline__ void wait_word(unsigned* p, unsigned target, unsigned* tmo, unsigned call) {
    if ((int)(rfl(__hip_atomic_load(p, __ATOMIC_RELAXED, __HIP_MEMORY_SCOPE_SYSTEM)) - target) >= 0) return;
    const unsigned long long t0 = __builtin_amdgcn_s_memrealtime();
    while ((int)(rfl(__hip_atomic_load(p, __ATOMIC_RELAXED, __HIP_MEMORY_SCOPE_SYSTEM)) - target) < 0) {
        if (give_up(t0, tmo, call)) break;
        __builtin_amdgcn_s_sleep(4);
    }
}

constexpr unsigned MAX_EXC = 31;
constexpr unsigned RAWB = 3584, AUX_U = 20, AUX_MAX = 2560, U_MAX = 10240, LANE_U_MAX = 224;
constexpr unsigned ENC_HIST = AUX_MAX + 16, ENC_SORT = ENC_HIST + 256, ENC_END = ENC_SORT + 128;
constexpr unsigned DEC_POS = 6144;

__device__ __forceinline__ unsigned bfi_c(unsigned mask, unsigned a, unsigned b) { unsigned r; asm("v_bfi_b32 %0, %1, %2, %3" : "=v"(r) : "s"(mask), "v"(a), "v"(b)); return r; }
__device__ __forceinline__ unsigned opaque(unsigned v) { asm("" : "+v"(v)); return v; }
__device__ __forceinline__ unsigned bytemask(unsigned x, int sh, unsigned c0c) {
    unsigned t; asm("v_and_or_b32 %0, %1, %2, %3" : "=v"(t) : "v"(x >> sh), "s"(0x01010101u), "v"(c0c));
    return perm(0u, 0u, t);
}
template <int CTRL, int RM = 0xf> __device__ __forceinline__ unsigned dpp0(unsigned v) { return (unsigned)__builtin_amdgcn_update_dpp(0, (int)v, CTRL, RM, 0xf, RM == 0xf); }
__device__ __forceinline__ unsigned wave_scan_incl(unsigned v) {
    v += dpp0<0x111>(v); v += dpp0<0x112>(v); v += dpp0<0x114>(v); v += dpp0<0x118>(v);
    v += dpp0<0x142, 0xa>(v); v += dpp0<0x143, 0xc>(v);
    return v;
}
__device__ __forceinline__ unsigned umin(unsigned a, unsigned b) { return a < b ? a : b; }
__device__ __forceinline__ unsigned umax(unsigned a, unsigned b) { return a > b ? a : b; }
__device__ __forceinline__ unsigned sad4(unsigned x) { return __builtin_amdgcn_sad_u8(x, 0u, 0u); }
__device__ __forceinline__ unsigned lut2x16(const unsigned (&T)[8], unsigned idx, unsigned ms, unsigned c0c) {
    const unsigned sel = idx & 0x07070707u, m8 = bytemask(idx, 3, c0c);
    const unsigned p = bfi(m8, perm(T[3], T[2], sel), perm(T[1], T[0], sel)), n = bfi(m8, perm(T[7], T[6], sel), perm(T[5], T[4], sel));
    return bfi(ms, n, p);
}
#define LDS_FENCE() asm volatile("" ::: "memory")
__device__ __forceinline__ void lds_or2(LAS unsigned char* scr, unsigned p, unsigned w) {
    const unsigned s = p & 31u;
    LAS unsigned* q = (LAS unsigned*)(scr + ((p >> 5) << 2));
    __hip_atomic_fetch_or(q, w << s, __ATOMIC_RELAXED, __HIP_MEMORY_SCOPE_WORKGROUP);
    __hip_atomic_fetch_or(q + 1, (w >> 1) >> (31u - s), __ATOMIC_RELAXED, __HIP_MEMORY_SCOPE_WORKGROUP);
}

__device__ __forceinline__ unsigned encode_block(const u32x4 (&c)[8], unsigned (&P)[14], u32x4 (&a)[3], LAS unsigned char* scr, int lane) {
    asm volatile("" : "+v"(lane));
    const unsigned c0c = opaque(0x0C0C0C0Cu);
    unsigned E[16], R[16];
    u16x2 p1 = {0, 0}, p2 = {0, 0}, n1 = {0, 0}, n2 = {0, 0};
#pragma unroll
    for (int g = 0; g < 16; ++g) {
        const unsigned x = (g & 1) ? c[g >> 1].z : c[g >> 1].x, y = (g & 1) ? c[g >> 1].w : c[g >> 1].y;
        const unsigned H = perm(y, x, 0x07050301u), L = perm(y, x, 0x06040200u);
        E[g] = bfi_c(0xFEFEFEFEu, H << 1, L >> 7);
        R[g] = bfi_c(0x80808080u, L << 1, bfi_c(0x40404040u, H >> 1, L));
        const unsigned en = E[g] & bytemask(H, 7, c0c), ep = E[g] ^ en;
        p1 = __builtin_elementwise_max(p1, __builtin_bit_cast(u16x2, ep)); p2 = __builtin_elementwise_max(p2, __builtin_bit_cast(u16x2, ep << 8));
        n1 = __builtin_elementwise_max(n1, __builtin_bit_cast(u16x2, en)); n2 = __builtin_elementwise_max(n2, __builtin_bit_cast(u16x2, en << 8));
        __builtin_amdgcn_sched_barrier(0);
    }
    const u32x4 z4 = {0u, 0u, 0u, 0u};
    *(LAS u32x4*)(scr + lane * 16) = z4; *(LAS u32x4*)(scr + 1024 + lane * 16) = z4;
    if (lane < 49) *(LAS u32x4*)(scr + 2048 + lane * 16) = z4;
    unsigned baseP, baseN;
    {
        const u16x2 mp = __builtin_elementwise_max(p1, p2), mn = __builtin_elementwise_max(n1, n2);
        u16x2 m = {(unsigned short)((mp.x > mp.y ? mp.x : mp.y) >> 8), (unsigned short)((mn.x > mn.y ? mn.x : mn.y) >> 8)};
        m = __builtin_elementwise_max(m, __builtin_bit_cast(u16x2, dpp0<0x128>(__builtin_bit_cast(unsigned, m)))); m = __builtin_elementwise_max(m, __builtin_bit_cast(u16x2, dpp0<0x124>(__builtin_bit_cast(unsigned, m))));
        m = __builtin_elementwise_max(m, __builtin_bit_cast(u16x2, dpp0<0x122>(__builtin_bit_cast(unsigned, m)))); m = __builtin_elementwise_max(m, __builtin_bit_cast(u16x2, dpp0<0x121>(__builtin_bit_cast(unsigned, m))));
        const unsigned mu = __builtin_bit_cast(unsigned, m);
        const u16x2 r01 = __builtin_elementwise_max(__builtin_bit_cast(u16x2, (unsigned)__builtin_amdgcn_readlane(mu, 0)), __builtin_bit_cast(u16x2, (unsigned)__builtin_amdgcn_readlane(mu, 16)));
        const u16x2 r23 = __builtin_elementwise_max(__builtin_bit_cast(u16x2, (unsigned)__builtin_amdgcn_readlane(mu, 32)), __builtin_bit_cast(u16x2, (unsigned)__builtin_amdgcn_readlane(mu, 48)));
        const unsigned mm = rfl(__builtin_bit_cast(unsigned, __builtin_elementwise_max(r01, r23)));
        baseP = mm & 0xFFu; baseN = mm >> 16;
    }
    const unsigned BP = baseP * 0x01010101u, BN = baseN * 0x01010101u;
    unsigned any = 0, D[16];
#pragma unroll
    for (int g = 0; g < 16; ++g) {
        E[g] = bfi(bytemask(R[g], 6, c0c), BN, BP) - E[g]; any |= E[g];
        D[g] = bfi_c(0x1E1E1E1Eu, E[g] << 1, (R[g] >> 7) & 0x01010101u);
        R[g] &= 0x7F7F7F7Fu;
    }
    LDS_FENCE();
#pragma unroll
    for (int gi = 0; gi < 4; ++gi) {
        const int g = gi * 5;
        const unsigned k4 = ((D[g] & 0x0F0F0F0Fu) | ((R[g] >> 2) & 0x10101010u) | ((D[g] << 1) & 0x20202020u)) << 2;
#pragma unroll
        for (int b = 0; b < 4; ++b)
            __hip_atomic_fetch_add((LAS unsigned*)(scr + ENC_HIST + ((k4 >> (8 * b)) & 0xFFu)), 1u, __ATOMIC_RELAXED, __HIP_MEMORY_SCOPE_WORKGROUP);
    }
    LDS_FENCE();
    const unsigned cnt = *(LAS unsigned*)(scr + ENC_HIST + lane * 4);
    const unsigned key = (cnt << 4) | (15u - (lane & 15));
    unsigned rank = 0;
    rank += dpp0<0x121>(key) > key; rank += dpp0<0x122>(key) > key; rank += dpp0<0x123>(key) > key; rank += dpp0<0x124>(key) > key; rank += dpp0<0x125>(key) > key;
    rank += dpp0<0x126>(key) > key; rank += dpp0<0x127>(key) > key; rank += dpp0<0x128>(key) > key; rank += dpp0<0x129>(key) > key; rank += dpp0<0x12A>(key) > key;
    rank += dpp0<0x12B>(key) > key; rank += dpp0<0x12C>(key) > key; rank += dpp0<0x12D>(key) > key; rank += dpp0<0x12E>(key) > key; rank += dpp0<0x12F>(key) > key;
    unsigned tail = cnt; tail += dpp0<0x128>(tail); tail += dpp0<0x124>(tail); tail += dpp0<0x122>(tail); tail += dpp0<0x121>(tail);
    const int sg = (lane >> 4) & 1;
    const unsigned ct = sg ? __builtin_amdgcn_readlane(tail, 48) : __builtin_amdgcn_readlane(tail, 32);
    if (lane < 32) *(LAS unsigned*)(scr + ENC_SORT + ((lane & 16) + rank) * 4) = cnt;
    LDS_FENCE();
    unsigned best;
    {
        const unsigned cb = lane & 15, s0 = cb / 3u, s1 = cb - 3u * s0, A = 1u << s0, AB = A + (1u << s1);
        unsigned cost = ct * (16u - AB);
#pragma unroll
        for (int q = 0; q < 4; ++q) {
            const u32x4 h = *(LAS u32x4*)(scr + ENC_SORT + sg * 64 + q * 16);
#pragma unroll
            for (int j = 0; j < 4; ++j) {
                const unsigned r = q * 4 + j, len = r < A ? 1u + s0 : r < AB ? 2u + s1 : 4u + ((r - AB) >> 1);
                cost += 2u * len * h[j];
            }
        }
        best = cb < 9 ? (cost << 4) | cb : 0xFFFFFFFFu;
        best = umin(best, dpp0<0x128>(best)); best = umin(best, dpp0<0x124>(best)); best = umin(best, dpp0<0x122>(best)); best = umin(best, dpp0<0x121>(best));
    }
    const unsigned cbP = __builtin_amdgcn_readlane(best, 0) & 15u, cbN = __builtin_amdgcn_readlane(best, 16) & 15u;
    const unsigned s0P = cbP / 3u, s1P = cbP - 3u * s0P, s0N = cbN / 3u, s1N = cbN - 3u * s0N;
    const unsigned ABP = (1u << s0P) + (1u << s1P), ABN = (1u << s0N) + (1u << s1N);
    unsigned F[8];
    {
        const unsigned s0 = sg ? s0N : s0P, s1 = sg ? s1N : s1P, A = 1u << s0, AB = sg ? ABN : ABP;
        const unsigned ent = rank < A ? (rank << 4) | (s0 << 6) : rank < AB ? 1u | ((rank - A) << 4) | (s1 << 6) : (2u + ((rank - AB) >> 1)) | (((rank - AB) & 1u) << 4) | (1u << 6);
        unsigned fv = ent << (8 * (lane & 3));
        fv |= dpp0<0xB1>(fv); fv |= dpp0<0x4E>(fv);
#pragma unroll
        for (int q = 0; q < 8; ++q) F[q] = __builtin_amdgcn_readlane(fv, q * 4);
        if (lane < 32)
            __hip_atomic_fetch_or((LAS unsigned*)(scr + (sg * 2 + (rank >> 3)) * 4), (unsigned)(lane & 15) << (8 * (rank & 3) + 4 * ((rank >> 2) & 1)), __ATOMIC_RELAXED, __HIP_MEMORY_SCOPE_WORKGROUP);
        if (lane == 0) *(LAS unsigned*)(scr + 16) = baseP | (baseN << 8) | (s0P << 16) | (s1P << 18) | (s0N << 20) | (s1N << 22);
    }
    const unsigned ABP4 = ABP * 0x01010101u, ABN4 = ABN * 0x01010101u, S0P4 = s0P * 0x01010101u, S0N4 = s0N * 0x01010101u;
    unsigned KK[16], XW[4], OX[4], TU = 0, TX = 0, bad = 0, xl = 0, xw = 0;
#pragma unroll
    for (int g = 0; g < 16; ++g) {
        const unsigned ms = bytemask(R[g], 6, c0c), m16 = bytemask(D[g], 4, c0c), ent = lut2x16(F, D[g], ms, c0c);
        const unsigned q = ((D[g] | 0x80808080u) - bfi(ms, ABN4, ABP4)) & 0x7F7F7F7Fu;
        unsigned K = bfi(m16, ((q >> 1) & 0x3F3F3F3Fu) + 0x02020202u, ent & 0x0F0F0F0Fu);
        unsigned SUF = bfi(m16, q & 0x01010101u, (ent >> 4) & 0x03030303u);
        unsigned S = bfi(m16, 0x01010101u, (ent >> 6) & 0x03030303u);
        unsigned gl = sad4(K) + 4u;
        if (__ballot(gl > 31u) != 0ull) {
            any |= 0x80u;
            const unsigned km = umax(umax(K & 0xFFu, (K >> 8) & 0xFFu), umax((K >> 16) & 0xFFu, K >> 24));
            const unsigned eq = gl > 31u ? (((K ^ (km * 0x01010101u)) + 0x7F7F7F7Fu) & 0x80808080u) ^ 0x80808080u : 0u, em = bytemask(eq, 7, c0c);
            E[g] |= eq;
            K &= ~em; SUF &= ~em; S = bfi(em, bfi(ms, S0N4, S0P4), S);
            gl = sad4(K) + 4u;
        }
        KK[g] = K; TU += gl; bad = umax(bad, gl);
        xw |= (SUF & 0xFFu) << xl; xl += S & 0xFFu;
        xw |= ((SUF >> 8) & 0xFFu) << xl; xl += (S >> 8) & 0xFFu;
        xw |= ((SUF >> 16) & 0xFFu) << xl; xl += (S >> 16) & 0xFFu;
        xw |= (SUF >> 24) << (xl & 31u); xl += S >> 24;
        if ((g & 3) == 3) { XW[g >> 2] = xw; OX[g >> 2] = TX; TX += xl; xw = 0; xl = 0; }
        __builtin_amdgcn_sched_barrier(0);
    }
    bad = (bad > 31u) | (TU > LANE_U_MAX);
    const unsigned tt = TU | (TX << 16), incl = wave_scan_incl(tt), tot = __builtin_amdgcn_readlane(incl, 63), totU = tot & 0xFFFFu, totX = tot >> 16;
    const unsigned xstart = (AUX_U * 8u + totU + 31u) & ~31u, endbit = xstart + totX;
    unsigned raw = (__ballot(bad != 0) != 0ull) || totU > U_MAX || endbit > (AUX_MAX - 8u) * 8u;
    if (!raw) {
        const unsigned px = xstart + ((incl - tt) >> 16);
        unsigned p = AUX_U * 8u + ((incl - tt) & 0xFFFFu);
#pragma unroll
        for (int g = 0; g < 16; ++g) {
            const unsigned ln = KK[g] + 0x01010101u;
            unsigned pos = (ln & 0xFFu) - 1u, w = 1u << pos;
            pos += (ln >> 8) & 0xFFu; w |= 1u << pos;
            pos += (ln >> 16) & 0xFFu; w |= 1u << pos;
            pos += ln >> 24; w |= 1u << pos;
            lds_or2(scr, p, w); p += pos + 1u;
            __builtin_amdgcn_sched_barrier(0);
        }
#pragma unroll
        for (int q = 0; q < 4; ++q) lds_or2(scr, px + OX[q], XW[q]);
    }
#pragma unroll
    for (int i = 0; i < 7; ++i) { P[i] = R[i] | ((R[14] << (7 - i)) & 0x80808080u); P[7 + i] = R[7 + i] | ((R[15] << (7 - i)) & 0x80808080u); }
    unsigned n = 0;
    const unsigned xo = (endbit + 31u) >> 5;
    if (!raw && __ballot((any & 0xF0F0F0F0u) != 0) != 0ull) {
#pragma unroll
        for (int g = 0; g < 16; ++g) {
            const unsigned f = E[g] & 0xF0F0F0F0u;
            unsigned long long mask = __ballot(f != 0);
            while (mask != 0ull && n <= MAX_EXC) {
                const int L = __builtin_ctzll(mask); mask &= mask - 1;
                const unsigned fl = __builtin_amdgcn_readlane(f, L);
                const unsigned lo = __builtin_amdgcn_readlane((g & 1) ? c[g >> 1].z : c[g >> 1].x, L);
                const unsigned hi = __builtin_amdgcn_readlane((g & 1) ? c[g >> 1].w : c[g >> 1].y, L);
                const bool e = lane < 4 && ((fl >> (8 * (lane & 3))) & 0xFFu) != 0;
                const unsigned long long em = __ballot(e);
                const unsigned slot = xo + n + __builtin_popcountll(em & ((1ull << lane) - 1ull));
                const unsigned val = (((lane & 2) ? hi : lo) >> (16 * (lane & 1))) & 0xFFFFu;
                if (e && slot * 4u < AUX_MAX) *(LAS unsigned*)(scr + slot * 4u) = (unsigned)(L * 64 + g * 4 + (lane & 3)) | (val << 12);
                n += __builtin_popcountll(em);
            }
        }
    }
    raw |= (n > MAX_EXC) | ((xo + n) * 4u > AUX_MAX);
    LDS_FENCE();
    a[0] = *(LAS u32x4*)(scr + lane * 16); a[1] = *(LAS u32x4*)(scr + 1024 + lane * 16); a[2] = *(LAS u32x4*)(scr + 2048 + (lane & 31) * 16);
    return ((xo + n + 3u) >> 2) | (n << 8) | (xo << 16) | (raw << 31);
}

__device__ __forceinline__ void decode_block(LAS unsigned char* src, u32x4 (&c)[8], int lane) {
    LAS unsigned char* aux = src + RAWB;
    const unsigned c0c = opaque(0x0C0C0C0Cu);
    const u32x4 nt = *(LAS const u32x4*)(aux);
    const unsigned prm_v = *(LAS const unsigned*)(aux + 16);
    unsigned u[5];
#pragma unroll
    for (int i = 0; i < 5; ++i) u[i] = *(LAS const unsigned*)(aux + AUX_U + lane * 20 + i * 4);
    u32x4 P4[3];
#pragma unroll
    for (int p = 0; p < 3; ++p) P4[p] = *(LAS const u32x4*)(src + p * 1024 + lane * 16);
    const unsigned long long ph = *(LAS const unsigned long long*)(src + 3072 + lane * 8);
    const unsigned c0 = __builtin_popcount(u[0]), c1 = c0 + __builtin_popcount(u[1]), c2 = c1 + __builtin_popcount(u[2]), c3 = c2 + __builtin_popcount(u[3]), cn = c3 + __builtin_popcount(u[4]);
    const unsigned C = wave_scan_incl(cn) - cn;
    if (lane == 0) *(LAS unsigned*)(src + DEC_POS) = AUX_U * 8u;
    const int rounds = __ballot(cn > 128u) != 0ull ? 3 : 2;
    for (int j = 0; j < rounds; ++j) {
        const unsigned t = (C >> 6) + 1u + j, kth = 64u * t - C;
        const unsigned i = (kth > c0) + (kth > c1) + (kth > c2) + (kth > c3);
        unsigned rem = kth - (i == 0 ? 0u : i == 1 ? c0 : i == 2 ? c1 : i == 3 ? c2 : c3), w = i == 0 ? u[0] : i == 1 ? u[1] : i == 2 ? u[2] : i == 3 ? u[3] : u[4], pos = 0;
#pragma unroll
        for (int st = 16; st >= 1; st >>= 1) {
            const unsigned lo = __builtin_popcount(w & ((1u << st) - 1u));
            const bool up = rem > lo;
            rem -= up ? lo : 0u; w = up ? w >> st : w; pos += up ? st : 0;
        }
        if (kth <= cn && t < 64u) *(LAS unsigned*)(src + DEC_POS + t * 4) = AUX_U * 8u + lane * 160u + i * 32u + pos + 1u;
    }
    LDS_FENCE();
    const unsigned p0 = *(LAS const unsigned*)(src + DEC_POS + lane * 4);
    unsigned S[8];
    {
        LAS const unsigned* q = (LAS const unsigned*)(aux + ((p0 >> 5) << 2));
#pragma unroll
        for (int i = 0; i < 8; ++i) S[i] = q[i];
#pragma unroll
        for (int i = 0; i < 7; ++i) S[i] = __builtin_amdgcn_alignbit(S[i + 1], S[i], p0 & 31u);
    }
    unsigned R[16];
    {
        const unsigned P[14] = {P4[0].x, P4[0].y, P4[0].z, P4[0].w, P4[1].x, P4[1].y, P4[1].z, P4[1].w, P4[2].x, P4[2].y, P4[2].z, P4[2].w, (unsigned)ph, (unsigned)(ph >> 32)};
        unsigned x = 0, y = 0;
#pragma unroll
        for (int i = 0; i < 7; ++i) { x |= (P[i] >> (7 - i)) & (0x01010101u << i); y |= (P[7 + i] >> (7 - i)) & (0x01010101u << i); }
#pragma unroll
        for (int i = 0; i < 14; ++i) R[i] = P[i] & 0x7F7F7F7Fu;
        R[14] = x; R[15] = y;
    }
    const unsigned n0 = rfl(nt.x), n1 = rfl(nt.y), n2 = rfl(nt.z), n3 = rfl(nt.w), prm = rfl(prm_v);
    const unsigned I[8] = {n0 & 0x0F0F0F0Fu, (n0 >> 4) & 0x0F0F0F0Fu, n1 & 0x0F0F0F0Fu, (n1 >> 4) & 0x0F0F0F0Fu, n2 & 0x0F0F0F0Fu, (n2 >> 4) & 0x0F0F0F0Fu, n3 & 0x0F0F0F0Fu, (n3 >> 4) & 0x0F0F0F0Fu};
    const unsigned s0P = (prm >> 16) & 3u, s1P = (prm >> 18) & 3u, s0N = (prm >> 20) & 3u, s1N = (prm >> 22) & 3u;
    const unsigned STP = (s0P << 8) | (s1P << 16) | (1u << 24), STN = (s0N << 8) | (s1N << 16) | (1u << 24);
    const unsigned OTP = (4u << 8) | (((1u << s0P) + 2u) << 16) | (((1u << s0P) + (1u << s1P)) << 24);
    const unsigned OTN = (4u << 8) | (((1u << s0N) + 2u) << 16) | (((1u << s0N) + (1u << s1N)) << 24);
    const unsigned BP = (prm & 0xFFu) * 0x01010101u, BN = ((prm >> 8) & 0xFFu) * 0x01010101u;
    unsigned RB[16], SS[16], TU = 0, TX = 0;
#pragma unroll
    for (int g = 0; g < 16; ++g) {
        const unsigned w0 = S[0], q0 = __builtin_ctz(w0), w1 = w0 & (w0 - 1u), q1 = __builtin_ctz(w1), w2 = w1 & (w1 - 1u), q2 = __builtin_ctz(w2), w3 = w2 & (w2 - 1u), q3 = __builtin_ctz(w3);
        const unsigned Q = q0 | (q1 << 8) | (q2 << 16) | (q3 << 24);
        const unsigned LN = Q - opaque((Q << 8) - 1u), sh = q3 + 1u;
        TU += sh;
        if (g < 15) {
#pragma unroll
            for (int i = 0; i < 7; ++i) if (32 * i < (int)LANE_U_MAX - 4 * (g + 1)) S[i] = (i < 6 && 32 * (i + 1) < (int)LANE_U_MAX - 4 * g) ? __builtin_amdgcn_alignbit(S[i + 1], S[i], sh) : S[i] >> sh;
        }
        const unsigned sel = bfi_c(0x04040404u, R[g] >> 4, bfi(bytemask(LN + 0x7D7D7D7Du, 7, c0c), 0x03030303u, LN));
        SS[g] = perm(STN, STP, sel);
        RB[g] = perm(OTN, OTP, sel) + (LN << 1);
        TX = __builtin_amdgcn_sad_u8(SS[g], 0u, TX);
    }
    const unsigned xstart = (__builtin_amdgcn_readlane(p0 + TU, 63) + 31u) & ~31u, px = xstart + wave_scan_incl(TX) - TX;
    unsigned X[5];
    {
        LAS const unsigned* q = (LAS const unsigned*)(aux + ((px >> 5) << 2));
#pragma unroll
        for (int i = 0; i < 5; ++i) X[i] = q[i];
#pragma unroll
        for (int i = 0; i < 4; ++i) X[i] = __builtin_amdgcn_alignbit(X[i + 1], X[i], px & 31u);
    }
    unsigned xoff = 0;
#pragma unroll
    for (int g = 0; g < 16; ++g) {
        const unsigned s = SS[g], M = perm(0u, 0x00030100u, s);
        const unsigned y = (g & 1) ? X[0] >> xoff : X[0], Y = perm(y, y, 0x01000100u);
        const unsigned t1 = opaque(s << 8), t2 = opaque((t1 << 8) + t1), Cs = (t2 << 16) + t2;
        const u16x2 z02 = __builtin_bit_cast(u16x2, Y) >> __builtin_bit_cast(u16x2, Cs & 0x00FF00FFu);
        const u16x2 z13 = __builtin_bit_cast(u16x2, Y) >> __builtin_bit_cast(u16x2, perm(0u, Cs, 0x0C030C01u));
        const unsigned suf = perm(__builtin_bit_cast(unsigned, z13), __builtin_bit_cast(unsigned, z02), 0x06020400u) & M;
        if (g & 1) {
            if (g < 15) {
                const unsigned t = __builtin_amdgcn_sad_u8(s, 0u, xoff);
#pragma unroll
                for (int i = 0; i < 4; ++i) if (32 * i < 8 * (15 - g)) X[i] = (i < 3 && 32 * (i + 1) < 8 * (17 - g)) ? __builtin_amdgcn_alignbit(X[i + 1], X[i], t) : X[i] >> t;
            }
        } else xoff = sad4(s);
        const unsigned rank = RB[g] + suf - 0x06060606u, ms = bytemask(R[g], 6, c0c);
        const unsigned D = bfi(bytemask(rank, 4, c0c), rank, lut2x16(I, rank, ms, c0c));
        const unsigned E = bfi(ms, BN, BP) - ((D >> 1) & 0x0F0F0F0Fu);
        const unsigned H = bfi_c(0x80808080u, R[g] << 1, E >> 1), L = bfi_c(0x3F3F3F3Fu, R[g], bfi_c(0x40404040u, D << 6, E << 7));
        const unsigned x = perm(H, L, 0x05010400u), yv = perm(H, L, 0x07030602u);
        if (g & 1) { c[g >> 1].z = x; c[g >> 1].w = yv; } else { c[g >> 1].x = x; c[g >> 1].y = yv; }
    }
}

typedef unsigned u32x2 __attribute__((ext_vector_type(2)));
__device__ __forceinline__ void store_granule(char* p, u32x2 g) {
    asm volatile("global_store_dwordx2 %0, %1, off sc0 sc1\n\ts_nop 1" :: "v"(p), "v"(g) : "memory");
}

constexpr unsigned GCTL = 131072;
__device__ __forceinline__ void publish_block(const Params& P, rsrc_t wsr, const u32x4 (&c)[8], unsigned slot, unsigned n, unsigned dmask, unsigned call, LAS unsigned char* lds, LAS unsigned char* scr,
                                              unsigned j, int wid, int lane) {
    unsigned R[14], h = 1u << 31; u32x4 a[3];
    if (n >= NRAW && n < NSEQ - NRAWEND) h = encode_block(c, R, a, scr, lane);
    const unsigned so = OFF_SLOT + slot * 8192u;
    if (h >> 31) {
#pragma unroll
        for (int q = 0; q < 8; ++q) __builtin_amdgcn_raw_buffer_store_b128(c[q], wsr, lane * 16 + q * 1024, so, AUX_SYS);
    } else {
        const unsigned n16 = h & 255u;
#pragma unroll
        for (int q = 0; q < 3; ++q) __builtin_amdgcn_raw_buffer_store_b128((u32x4){R[4 * q], R[4 * q + 1], R[4 * q + 2], R[4 * q + 3]}, wsr, lane * 16 + q * 1024, so, AUX_SYS);
        __builtin_amdgcn_raw_buffer_store_b64((u32x2){R[12], R[13]}, wsr, lane * 8 + 3072, so, AUX_SYS);
        if ((unsigned)lane < n16) __builtin_amdgcn_raw_buffer_store_b128(a[0], wsr, lane * 16 + RAWB, so, AUX_SYS);
        if ((unsigned)lane + 64u < n16) __builtin_amdgcn_raw_buffer_store_b128(a[1], wsr, lane * 16 + RAWB + 1024, so, AUX_SYS);
        if ((unsigned)lane + 128u < n16) __builtin_amdgcn_raw_buffer_store_b128(a[2], wsr, lane * 16 + RAWB + 2048, so, AUX_SYS);
    }
    WAIT_VM0();
    const unsigned u = j & 3u, gsize = j == 0 ? 4u : 8u;
    volatile LAS unsigned* ctl = (volatile LAS unsigned*)(lds + GCTL);
    while (j >= 4u && (int)(ctl[36 + u] - (j - 3u)) < 0) __builtin_amdgcn_s_sleep(1);
    unsigned arrived = 0;
    if (lane == 0) {
        ctl[u * 8 + wid] = h;
        arrived = __hip_atomic_fetch_add((LAS unsigned*)(lds + GCTL) + 32 + u, 1u, __ATOMIC_RELAXED, __HIP_MEMORY_SCOPE_WORKGROUP);
    }
    if (rfl(arrived) == gsize - 1u) {
        const unsigned d = lane >> 3, w = lane & 7, hw = ctl[u * 8 + w];
        if (w < gsize && ((dmask >> d) & 1u)) store_granule(P.ws[d] + OFF_GRAN + ((unsigned)P.rank * NSEQ + n - wid + w) * 8u, (u32x2){call, hw});
        if (lane == 0) { ctl[32 + u] = 0u; ctl[36 + u] = j + 1u; }
    }
}

__device__ __forceinline__ void signal_cnt(char* ws, unsigned off, int lane) {
    if (off && lane == 0) __hip_atomic_fetch_add((unsigned*)(ws + off), 1u, __ATOMIC_RELAXED, __HIP_MEMORY_SCOPE_SYSTEM);
}
__device__ __forceinline__ unsigned packer_role(const Params& P, LAS unsigned char* lds, int first, int next, int step, int end) {
    unsigned call = 0;
    const int lane = threadIdx.x & 63, wid = rfl(threadIdx.x >> 6);
    char* ws = P.ws[P.rank];
    const rsrc_t wsr = mk_rsrc(ws, WS_USED);
    const rsrc_t xr = mk_rsrc(P.x, DIM * KSH * 2), wr = mk_rsrc(P.w, DIM * KSH * 2);
    const int myi = P.rank >> 2, myj = P.rank & 3;
    LAS unsigned char* my = lds + wid * 8192;
    unsigned j = first < NFIRST ? 0u : 1u;
    for (int p = first; p < end; p = p < NFIRST ? next : p + step, ++j) {
        u32x4 c[8];
        unsigned slot, n, dmask, selfcnt = 0;
        if (p >= 4096) {
            const unsigned o = p - 4096, w = o & 7, h = (o >> 3) & 1, tile = (o >> 4) & 63;
            if (call == 0) call = call_draw(ws);
            unsigned* tmo = (unsigned*)(ws + OFF_TMO);
            wait_word((unsigned*)(ws + OFF_OCNT + (o >> 3) * 128), call * 8u, tmo, call);
            const unsigned row0 = (P.rank >> 2) * 4096 + (tile >> 3) * 256 + h * 128 + (w >> 2) * 64, col0 = (P.rank & 3) * 2048 + (tile & 7) * 256 + (w & 3) * 64;
            const rsrc_t outr = mk_rsrc(P.out, (unsigned)DIM * DIM * 2u);
#pragma unroll
            for (int q = 0; q < 8; ++q) c[q] = __builtin_amdgcn_raw_buffer_load_b128(outr, (unsigned)(lane >> 3) * (DIM * 2u) + (unsigned)(lane & 7) * 16u, (row0 * DIM + col0) * 2u + q * (8u * DIM * 2u), AUX_SYS);
            if (rfl(__hip_atomic_load(tmo, __ATOMIC_RELAXED, __HIP_MEMORY_SCOPE_AGENT)) == call) {
#pragma unroll
                for (int q = 0; q < 8; ++q) c[q] = (u32x4){0x7FC07FC0u, 0x7FC07FC0u, 0x7FC07FC0u, 0x7FC07FC0u};
            }
            slot = 4096u + o; n = 1536u + o; dmask = 0xFFu;
        } else {
            bool is_w; int kk, sel, r, bi;
            if (p < 3072) {
                kk = p / 192; const int rem = p - kk * 192;
                bi = rem & 31; r = 0; is_w = rem >= 64; sel = is_w ? ((rem - 64) >> 5) : (rem >> 5);
            } else { const int q = p - 3072; kk = q >> 6; sel = (q >> 5) & 1; bi = q & 31; r = 1; is_w = false; }
            if (!is_w) {
                const int rr = lane >> 3;
                const unsigned vo = (unsigned)rr * 2048u + (unsigned)(((lane & 7) ^ rr) * 16);
                const unsigned so = (unsigned)(sel * 4096 + r * 2048 + bi * 64) * 2048u + (unsigned)kk * 128u;
#pragma unroll
                for (int q = 0; q < 8; ++q) c[q] = __builtin_amdgcn_raw_buffer_load_b128(xr, vo + q * 8 * 2048, so, 0);
            } else {
                const int pn = bi >> 2, bj = (bi >> 1) & 1, q = bi & 1;
                const int gk = lane >> 3, cc = lane & 7, wcl = cc >> 2, cq = cc & 3;
                const unsigned vo = (unsigned)(gk * 8) * 16384u + (unsigned)((64 * (2 * q + wcl) + 32 * bj + 8 * cq) * 2);
                const unsigned so = (unsigned)(kk * 64) * 16384u + (unsigned)((sel * 2048 + pn * 256) * 2);
                u32x4 R[8];
#pragma unroll
                for (int jj = 0; jj < 8; ++jj) R[jj] = __builtin_amdgcn_raw_buffer_load_b128(wr, vo + jj * 16384, so, 0);
#pragma unroll
                for (int t = 0; t < 8; ++t) {
                    const unsigned s = (t & 1) ? 0x07060302u : 0x05040100u;
                    u32x4 v;
                    v.x = perm(R[1][t >> 1], R[0][t >> 1], s); v.y = perm(R[3][t >> 1], R[2][t >> 1], s);
                    v.z = perm(R[5][t >> 1], R[4][t >> 1], s); v.w = perm(R[7][t >> 1], R[6][t >> 1], s);
                    const int s64 = 32 * wcl + 16 * (t >> 2) + 4 * cq + (t & 3);
                    *(LAS u32x4*)(my + s64 * 128 + ((gk ^ (s64 & 7)) * 16)) = v;
                }
                asm volatile("s_waitcnt lgkmcnt(0)" ::: "memory");
#pragma unroll
                for (int q2 = 0; q2 < 8; ++q2) c[q2] = *(LAS u32x4*)(my + q2 * 1024 + lane * 16);
                asm volatile("s_waitcnt lgkmcnt(0)" ::: "memory");
            }
            if (call == 0) call = call_draw(ws);
            const int g = P.rank * 16 + kk;
            if (is_w ? (sel == myj) : (sel == myi)) {
                const unsigned so = is_w ? OFF_BIMG + (unsigned)(g * 32 + bi) * 8192u : OFF_AIMG + (unsigned)((r * 128 + g) * 32 + bi) * 8192u;
#pragma unroll
                for (int q = 0; q < 8; ++q) __builtin_amdgcn_raw_buffer_store_b128(c[q], wsr, lane * 16 + q * 1024, so, AUX_SC1);
                selfcnt = OFF_CNT + (r * 128 + g) * 128;
            }
            slot = is_w ? 2048u + (unsigned)((sel * 16 + kk) * 32 + bi) : (unsigned)(((sel * 2 + r) * 16 + kk) * 32 + bi);
            n = is_w ? (unsigned)(kk * 64 + 32 + bi) : (r == 0 ? (unsigned)(kk * 64 + bi) : (unsigned)(1024 + kk * 32 + bi));
            dmask = is_w ? (0x11u << sel) : (0x0Fu << (4 * sel));
        }
        publish_block(P, wsr, c, slot, n, dmask & ~(1u << P.rank), call, lds, my, j, wid, lane);
        signal_cnt(ws, selfcnt, lane);
    }
    if (call == 0) call = call_draw(ws);
    return call;
}

__device__ __forceinline__ u32x2 load_granule(rsrc_t wsr, unsigned off) { return __builtin_amdgcn_raw_buffer_load_b64(wsr, 0, off, AUX_SYS); }

struct Blk { unsigned slot, dso, cnt; bool is_out; };
__device__ __forceinline__ Blk locate(unsigned n, int src, int myi, int myj) {
    Blk b; b.is_out = false;
    if (n < 1024) { const int kk = n >> 6, bi = n & 31, g = src * 16 + kk; b.cnt = OFF_CNT + g * 128;
        if ((n & 63) >= 32) { b.slot = 2048u + (myj * 16 + kk) * 32 + bi; b.dso = OFF_BIMG + (unsigned)(g * 32 + bi) * 8192u; }
        else { b.slot = ((myi * 2) * 16 + kk) * 32 + bi; b.dso = OFF_AIMG + (unsigned)(g * 32 + bi) * 8192u; } }
    else if (n < 1536) { const int kk = (n - 1024) >> 5, bi = n & 31, g = src * 16 + kk; b.cnt = OFF_CNT + (128 + g) * 128;
        b.slot = ((myi * 2 + 1) * 16 + kk) * 32 + bi; b.dso = OFF_AIMG + (unsigned)((128 + g) * 32 + bi) * 8192u; }
    else { const int o = n - 1536, w = o & 7, h = (o >> 3) & 1, tile = (o >> 4) & 63, r = o >> 10; b.cnt = 0; b.is_out = true;
        b.slot = 4096u + o;
        const unsigned row0 = (src >> 2) * 4096 + r * 2048 + (tile >> 3) * 256 + h * 128 + (w >> 2) * 64, col0 = (src & 3) * 2048 + (tile & 7) * 256 + (w & 3) * 64;
        b.dso = (row0 * DIM + col0) * 2u; }
    return b;
}
constexpr unsigned NSLOT = 15, CTL = NSLOT * 8192;
constexpr unsigned M_DEAD = 1u << 30, N_STOP = 0xFFFFFFFFu;
__device__ __forceinline__ unsigned lds_ld(LAS unsigned char* p) { return *(volatile LAS unsigned*)p; }
__device__ __forceinline__ void lds_st(LAS unsigned char* p, unsigned v) { *(volatile LAS unsigned*)p = v; }
__device__ __forceinline__ void ring_publish(LAS unsigned char* lds, unsigned i, unsigned n, unsigned h, int lane) {
    const unsigned sl = i % NSLOT;
    if (lane == 0) { *(volatile LAS u32x2*)(lds + CTL + sl * 8) = (u32x2){n, h}; lds_st(lds + CTL + 256 + sl * 4, i + 1u); }
}
__device__ __forceinline__ void dma_op(rsrc_t r, LAS unsigned char* dst, int j, unsigned so, int lane) {
    __builtin_amdgcn_raw_ptr_buffer_load_lds(r, (LAS void*)(dst + j * 1024), 16, lane * 16 + j * 1024, so, 0, AUX_SYS);
}
#define VM_WAIT_(n) asm volatile("s_waitcnt vmcnt(" #n ")" ::: "memory")
#define VM_WAIT(n) VM_WAIT_(n)
#define WIN 5

__device__ __forceinline__ void loader_wave(const Params& P, LAS unsigned char* lds, unsigned call, int src, int lane) {
    char* ws = P.ws[P.rank];
    unsigned* tmo = (unsigned*)(ws + OFF_TMO);
    const rsrc_t wsr = mk_rsrc(ws, WS_USED);
    char* const remp = P.ws[src];
    const int myi = P.rank >> 2, myj = P.rank & 3;
    const unsigned gbase = OFF_GRAN + (unsigned)src * NSEQ * 8u;
    char* const gptr = ws + gbase;
    const unsigned tko = OFF_LINK + (unsigned)src * 128u, tbase = (call - 1u) * (NSEQ + 3u * PWL);
    __builtin_amdgcn_s_setprio(3);
    unsigned n, n1, n2;
    {
        unsigned t0 = 0;
        if (lane == 0) t0 = __hip_atomic_fetch_add((unsigned*)(ws + tko), 3u, __ATOMIC_RELAXED, __HIP_MEMORY_SCOPE_AGENT);
        n = rfl(t0) - tbase; n1 = n + 1; n2 = n + 2;
    }
    {
        const rsrc_t rw = mk_rsrc(remp, OFF_AIMG);
        const unsigned v = __builtin_amdgcn_raw_buffer_load_b32(rw, lane < 25 ? OFF_SLOT + (unsigned)lane * (2u << 20) + 8188u - (lane == 24 ? 8192u : 0u) : 0xFFFFFFF0u, 0, AUX_SYS);
        asm volatile("" :: "v"(v));
    }
    unsigned gx, gy, hx, hy;
    { const u32x2 a = load_granule(wsr, gbase + (n < NSEQ ? n : 0u) * 8u), b = load_granule(wsr, gbase + (n1 < NSEQ ? n1 : 0u) * 8u);
      gx = rfl(a.x); gy = rfl(a.y); hx = rfl(b.x); hy = rfl(b.y); }
    const int zero = 0, one = 1;
    unsigned i = 0;
    bool pend = false; unsigned pn = 0, ph = 0;
    while (n < NSEQ) {
        const unsigned sl = i % NSLOT;
        bool dead = false;
        const bool need_gran = (int)(gx - call) < 0;
        const bool need_slot = i >= NSLOT && (int)(lds_ld(lds + CTL + 320 + sl * 4) - (i - NSLOT + 1u)) < 0;
        if (need_gran || need_slot) {
            if (pend) { VM_WAIT(0); ring_publish(lds, i - 1u, pn, ph, lane); pend = false; }
            const unsigned long long t0 = __builtin_amdgcn_s_memrealtime();
            while ((int)(gx - call) < 0) {
                const u32x2 a = load_granule(wsr, gbase + n * 8u);
                gx = rfl(a.x); gy = rfl(a.y);
                if ((int)(gx - call) >= 0) break;
                if (give_up(t0, tmo, call)) { dead = true; break; }
                __builtin_amdgcn_s_sleep(2);
                asm volatile("" ::: "memory");
            }
            while (i >= NSLOT && (int)(lds_ld(lds + CTL + 320 + sl * 4) - (i - NSLOT + 1u)) < 0) __builtin_amdgcn_s_sleep(1);
        }
        const bool raw = (gy >> 31) != 0;
        const unsigned so = OFF_SLOT + locate(n, src, myi, myj).slot * 8192u;
                const unsigned bytes = dead ? 0u : raw ? 8192u : RAWB + ((gy & 255u) << 4);
        const rsrc_t rem = mk_rsrc(remp, so + bytes);
        LAS unsigned char* dst = lds + sl * 8192;
        u32x2 g2; unsigned tk = 0;
        asm volatile("global_load_dwordx2 %0, %1, %2 sc0 sc1" : "=v"(g2) : "v"(zero), "s"(gptr + (n2 < NSEQ ? n2 : 0u) * 8u) : "memory");
        if (lane == 0) asm volatile("global_atomic_add %0, %1, %2, %3 sc0" : "=v"(tk) : "v"(zero), "v"(one), "s"(ws + tko) : "memory");
        VM_WAIT(WIN); dma_op(rem, dst, 0, so, lane);
        VM_WAIT(WIN); dma_op(rem, dst, 1, so, lane);
        VM_WAIT(WIN); dma_op(rem, dst, 2, so, lane);
        VM_WAIT(WIN); if (pend) ring_publish(lds, i - 1u, pn, ph, lane); dma_op(rem, dst, 3, so, lane);
        VM_WAIT(WIN); dma_op(rem, dst, 4, so, lane);
        asm volatile("s_waitcnt vmcnt(5)" : "+v"(g2), "+v"(tk) :: "memory"); dma_op(rem, dst, 5, so, lane);
        if (raw) { dma_op(rem, dst, 6, so, lane); dma_op(rem, dst, 7, so, lane); VM_WAIT(0); }
        pend = true; pn = n; ph = gy | (dead ? M_DEAD : 0u);
        n = n1; gx = hx; gy = hy;
        n1 = n2; hx = rfl(g2.x); hy = rfl(g2.y);
        n2 = rfl(tk) - tbase;
        ++i;
    }
    VM_WAIT(0);
    if (pend) ring_publish(lds, i - 1u, pn, ph, lane);
    for (int d = 0; d < NDEC; ++d, ++i) {
        const unsigned sl = i % NSLOT;
        while (i >= NSLOT && (int)(lds_ld(lds + CTL + 320 + sl * 4) - (i - NSLOT + 1u)) < 0) __builtin_amdgcn_s_sleep(1);
        ring_publish(lds, i, N_STOP, 0u, lane);
    }
}

__device__ __forceinline__ void decoder_wave(const Params& P, LAS unsigned char* lds, unsigned call, int src, int d, int lane) {
    char* ws = P.ws[P.rank];
    unsigned* tmo = (unsigned*)(ws + OFF_TMO);
    const rsrc_t wsr = mk_rsrc(ws, WS_USED), outr = mk_rsrc(P.out, (unsigned)DIM * DIM * 2u);
    const int myi = P.rank >> 2, myj = P.rank & 3;
    for (unsigned i = d;; i += NDEC) {
        const unsigned sl = i % NSLOT;
        while (lds_ld(lds + CTL + 256 + sl * 4) != i + 1u) __builtin_amdgcn_s_sleep(1);
        asm volatile("" ::: "memory");
        const u32x2 meta = *(volatile LAS u32x2*)(lds + CTL + sl * 8);
        const unsigned n = rfl(meta.x), h = rfl(meta.y);
        if (n == N_STOP) break;
        const Blk b = locate(n, src, myi, myj);
        const bool raw = (h >> 31) != 0;
        bool dead = (h & M_DEAD) != 0;
        if (b.is_out && rfl(__hip_atomic_load(tmo, __ATOMIC_RELAXED, __HIP_MEMORY_SCOPE_AGENT)) == call) dead = true;
        u32x4 c[8];
        unsigned recs = 0;
        const unsigned ne = (dead || raw) ? 0u : (h >> 8) & 63u;
        LAS unsigned char* cur = lds + sl * 8192;
        if (dead) {
#pragma unroll
            for (int j = 0; j < 8; ++j) c[j] = (u32x4){0x7FC07FC0u, 0x7FC07FC0u, 0x7FC07FC0u, 0x7FC07FC0u};
        } else if (raw) {
#pragma unroll
            for (int j = 0; j < 8; ++j) c[j] = *(LAS u32x4*)(cur + j * 1024 + lane * 16);
        } else {
            recs = *(LAS unsigned*)(cur + RAWB + (((h >> 16) & 0x3FFu) + (lane & 31)) * 4u);
            decode_block(cur, c, lane);
        }
#pragma unroll
        for (int j = 0; j < 8; ++j) asm volatile("" : "+v"(c[j]));
        asm volatile("" : "+v"(recs));
        if (lane == 0) lds_st(lds + CTL + 320 + sl * 4, i + 1u);
        const rsrc_t dr = b.is_out ? outr : wsr;
        const unsigned sj = b.is_out ? 8u * DIM * 2u : 1024u, sr = b.is_out ? DIM * 2u : 128u;
        const unsigned vo = (unsigned)(lane >> 3) * sr + (unsigned)(lane & 7) * 16u;
#pragma unroll
        for (int j = 0; j < 8; ++j) __builtin_amdgcn_raw_buffer_store_b128(c[j], dr, vo, b.dso + j * sj, AUX_SC1);
        for (unsigned e = 0; e < ne; ++e) {
            const unsigned rec = __builtin_amdgcn_readlane(recs, e), pos = rec & 0xFFFu, v = pos & 63u;
            if ((unsigned)lane == (pos >> 6)) __builtin_amdgcn_raw_buffer_store_b16((unsigned short)((rec >> 12) & 0xFFFFu), dr, vo + (v & 7u) * 2u, b.dso + (v >> 3) * sj, AUX_SC1);
        }
        if (b.cnt) { WAIT_VM0(); signal_cnt(ws, b.cnt, lane); }
    }
}

__device__ __forceinline__ void puller_role(const Params& P, LAS unsigned char* lds, unsigned call, int wg) {
    const int lane = threadIdx.x & 63, wid = rfl(threadIdx.x >> 6);
    const int link = wg % 7, src = rfl((P.rank + 1 + link) & 7);
    if (threadIdx.x < 32) *(volatile LAS unsigned*)(lds + CTL + 256 + threadIdx.x * 4) = 0u;
    __syncthreads();
    if (wid == 0) loader_wave(P, lds, call, src, lane);
    else if (wid <= NDEC) decoder_wave(P, lds, call, src, wid - 1, lane);
}

__device__ __forceinline__ float gelu_f(float y) {
    const float c1 = -2.0f * 0.7978845608028654f * 1.4426950408889634f, c2 = c1 * 0.044715f;
    const float e = __builtin_amdgcn_exp2f(y * (c1 + c2 * y * y));
    return y * __builtin_amdgcn_rcpf(1.0f + e);
}
__device__ __forceinline__ unsigned cvt_pk_bf16(float lo, float hi) { unsigned r; asm("v_cvt_pk_bf16_f32 %0, %1, %2" : "=v"(r) : "v"(lo), "v"(hi)); return r; }

__device__ __forceinline__ void gemm_role(const Params& P, LAS unsigned char* lds, unsigned call, int wg) {
    const int tid = threadIdx.x, wid = rfl(tid >> 6), lane = tid & 63, wr = wid >> 2, wc = wid & 3, fr = lane & 15, fq = lane >> 4;
    char* ws = P.ws[P.rank];
    unsigned* tmo = (unsigned*)(ws + OFF_TMO);
    const rsrc_t wsr = mk_rsrc(ws, WS_USED);
    const int xcd = wg & 7, q = (wg - NPACK) >> 3, h = xcd & 1, blk = xcd >> 1;
    const int tm = (blk >> 1) * 4 + (q >> 2), tn = (blk & 1) * 4 + (q & 3), tile = tm * 8 + tn;
    const unsigned voff = (unsigned)tid * 16u, ldsw = (unsigned)wid * 1024u;
    const int aoff0 = (wr * 64 + fr) * 128 + ((fq ^ (fr & 7)) << 4), aoff1 = aoff0 ^ 64;
    const int boff0 = (wc * 32 + fr) * 128 + ((fq ^ (fr & 7)) << 4), boff1 = boff0 ^ 64;
#define G_SA(b, hh) (((b) * 2 + (hh)) * 16384)
#define G_SB(b, hh) ((4 + (b) * 2 + (hh)) * 16384)
#define G_STAGE(bufoff, soff) do { _Pragma("unroll") for (int _i = 0; _i < 2; ++_i) \
        __builtin_amdgcn_raw_ptr_buffer_load_lds(wsr, (LAS void*)(lds + (bufoff) + ldsw + _i * 8192), 16, voff, (soff) + _i * 8192, 0, AUX_SYS); } while (0)
#define G_LDA(dst, b, hh) do { _Pragma("unroll") for (int m = 0; m < 4; ++m) { \
        dst[m][0] = *(const LAS bf16x8*)(lds + G_SA(b, hh) + aoff0 + m * 2048); dst[m][1] = *(const LAS bf16x8*)(lds + G_SA(b, hh) + aoff1 + m * 2048); } } while (0)
#define G_LDB(dst, b, hh) do { _Pragma("unroll") for (int n = 0; n < 2; ++n) { \
        dst[n][0] = *(const LAS bf16x8*)(lds + G_SB(b, hh) + boff0 + n * 2048); dst[n][1] = *(const LAS bf16x8*)(lds + G_SB(b, hh) + boff1 + n * 2048); } } while (0)
#define G_MMA(ai, bj, At, Bt) do { __builtin_amdgcn_s_setprio(1); _Pragma("unroll") for (int m = 0; m < 4; ++m) _Pragma("unroll") for (int n = 0; n < 2; ++n) _Pragma("unroll") for (int k = 0; k < 2; ++k) \
        acc[ai][bj][m][n] = __builtin_amdgcn_mfma_f32_16x16x32_bf16(Bt[n][k], At[m][k], acc[ai][bj][m][n], 0, 0, 0); __builtin_amdgcn_s_setprio(0); } while (0)
#define G_WAIT_V(n) asm volatile("s_waitcnt vmcnt(" #n ")" ::: "memory")
#define G_WAIT_L(n) asm volatile("s_waitcnt lgkmcnt(" #n ")" ::: "memory")
#define G_BAR __builtin_amdgcn_s_barrier()
#define G_SCHED __builtin_amdgcn_sched_barrier(0)
#define G_GIDX(t) ((unsigned)((4 * h + ((t) & 3)) * 16 + ((t) >> 2)))
    const int zero = 0;
    for (int r = 0; r < 2; ++r) {
        const unsigned target = call * (r == 0 ? 64u : 32u);
        const unsigned abase = OFF_AIMG + (unsigned)r * (128u * 262144u) + (unsigned)tm * 32768u, bbase = OFF_BIMG + (unsigned)tn * 32768u;
        char* cbase = ws + OFF_CNT + r * 128 * 128;
        f32x4 acc[2][2][4][2];
#pragma unroll
        for (int a = 0; a < 2; ++a)
#pragma unroll
            for (int b = 0; b < 2; ++b)
#pragma unroll
                for (int m = 0; m < 4; ++m)
#pragma unroll
                    for (int n = 0; n < 2; ++n) acc[a][b][m][n] = (f32x4){0.f, 0.f, 0.f, 0.f};
        bf16x8 At[4][2], B0[2][2], B1[2][2];
        STAMP(ws, r * 8 + 0);
        wait_word((unsigned*)(cbase + G_GIDX(0) * 128), target, tmo, call);
        wait_word((unsigned*)(cbase + G_GIDX(1) * 128), target, tmo, call);
        wait_word((unsigned*)(cbase + G_GIDX(2) * 128), target, tmo, call);
        STAMP(ws, r * 8 + 1);
        {
            const unsigned a0 = abase + G_GIDX(0) * 262144u, b0 = bbase + G_GIDX(0) * 262144u, a1 = abase + G_GIDX(1) * 262144u, b1 = bbase + G_GIDX(1) * 262144u;
            G_STAGE(G_SB(0, 0), b0); G_STAGE(G_SA(0, 0), a0); G_STAGE(G_SB(0, 1), b0 + 16384); G_STAGE(G_SA(0, 1), a0 + 16384);
            if (wr == 1) G_BAR;
            G_WAIT_V(4); G_BAR;
            G_STAGE(G_SB(1, 0), b1); G_STAGE(G_SA(1, 0), a1); G_STAGE(G_SB(1, 1), b1 + 16384);
            G_WAIT_V(6); G_BAR;
        }
        unsigned fA = target, fB;
        for (int t = 0; t < 64; t += 2) {
            const int t2 = t + 2 < 64 ? t + 2 : 63, t3 = t + 3 < 64 ? t + 3 : 63, t4 = t + 4 < 64 ? t + 4 : 63;
            const unsigned g1 = G_GIDX(t + 1), g2 = G_GIDX(t2), g3 = G_GIDX(t3), g4 = G_GIDX(t4);
            const unsigned a1 = abase + g1 * 262144u, a2 = abase + g2 * 262144u, b2 = bbase + g2 * 262144u, a3 = abase + g3 * 262144u, b3 = bbase + g3 * 262144u;
            asm volatile("global_load_dword %0, %1, %2 sc0 sc1" : "=v"(fB) : "v"(zero), "s"(cbase + g3 * 128));
            G_LDB(B0, 0, 0); G_SCHED; G_LDA(At, 0, 0); G_STAGE(G_SA(1, 1), a1 + 16384);
            G_WAIT_L(8); G_BAR; G_WAIT_L(0); G_MMA(0, 0, At, B0); G_BAR; G_SCHED;
            asm volatile("" : "+v"(fA));
            if ((int)(rfl(fA) - target) < 0) wait_word((unsigned*)(cbase + g2 * 128), target, tmo, call);
            G_LDB(B1, 0, 1); G_STAGE(G_SB(0, 0), b2);
            G_BAR; G_WAIT_L(0); G_MMA(0, 1, At, B1); G_BAR;
            G_LDA(At, 0, 1); G_STAGE(G_SA(0, 0), a2);
            G_BAR; G_WAIT_L(0); G_MMA(1, 0, At, B0); G_BAR; G_SCHED;
            G_STAGE(G_SB(0, 1), b2 + 16384);
            G_WAIT_V(6); G_BAR; G_MMA(1, 1, At, B1); G_BAR;
            asm volatile("global_load_dword %0, %1, %2 sc0 sc1" : "=v"(fA) : "v"(zero), "s"(cbase + g4 * 128));
            G_LDB(B0, 1, 0); G_SCHED; G_LDA(At, 1, 0); G_STAGE(G_SA(0, 1), a2 + 16384);
            G_WAIT_L(8); G_BAR; G_WAIT_L(0); G_MMA(0, 0, At, B0); G_BAR; G_SCHED;
            asm volatile("" : "+v"(fB));
            if ((int)(rfl(fB) - target) < 0) wait_word((unsigned*)(cbase + g3 * 128), target, tmo, call);
            G_LDB(B1, 1, 1); G_STAGE(G_SB(1, 0), b3);
            G_BAR; G_WAIT_L(0); G_MMA(0, 1, At, B1); G_BAR;
            G_LDA(At, 1, 1); G_STAGE(G_SA(1, 0), a3);
            G_BAR; G_WAIT_L(0); G_MMA(1, 0, At, B0); G_BAR; G_SCHED;
            G_STAGE(G_SB(1, 1), b3 + 16384);
            G_WAIT_V(6); G_BAR; G_MMA(1, 1, At, B1); G_BAR;
        }
        G_WAIT_V(0);
        asm volatile("" : "+v"(fA));
        if (wr == 0) G_BAR;
        G_BAR;
        STAMP(ws, r * 8 + 2);
        int lane = threadIdx.x & 63;
        asm volatile("" : "+v"(lane));
        const int fr = lane & 15, fq = lane >> 4;
        if (h) {
#pragma unroll
            for (int b = 0; b < 2; ++b)
#pragma unroll
                for (int m = 0; m < 4; ++m)
#pragma unroll
                    for (int n = 0; n < 2; ++n) { const f32x4 tq = acc[0][b][m][n]; acc[0][b][m][n] = acc[1][b][m][n]; acc[1][b][m][n] = tq; }
        }
        const unsigned xl = (unsigned)((r * 64 + tile) * 2);
#pragma unroll
        for (int b = 0; b < 2; ++b)
#pragma unroll
            for (int m = 0; m < 4; ++m)
#pragma unroll
                for (int n = 0; n < 2; ++n) {
                    __builtin_amdgcn_raw_buffer_store_b128(__builtin_bit_cast(u32x4, acc[1][b][m][n]), wsr, lane * 16 + ((b * 4 + m) * 2 + n) * 1024, OFF_XBUF + ((xl + h) * 8 + wid) * 16384u, AUX_SC1);
                    if (EPI_PACE) __builtin_amdgcn_s_sleep(EPI_PACE);
                }
        WAIT_VM0();
        if (lane == 0) __hip_atomic_store((unsigned*)(ws + OFF_XFLAG + (xl + h) * 128 + wid * 16), call, __ATOMIC_RELAXED, __HIP_MEMORY_SCOPE_AGENT);
        wait_word((unsigned*)(ws + OFF_XFLAG + (xl + (h ^ 1)) * 128 + wid * 16), call, tmo, call);
        const bool dead = rfl(__hip_atomic_load(tmo, __ATOMIC_RELAXED, __HIP_MEMORY_SCOPE_AGENT)) == call;
        STAMP(ws, r * 8 + 3);
        LAS unsigned char* my = lds + wid * 8192;
#pragma unroll
        for (int b = 0; b < 2; ++b)
#pragma unroll
            for (int m = 0; m < 4; ++m) {
                const f32x4 p0 = __builtin_bit_cast(f32x4, __builtin_amdgcn_raw_buffer_load_b128(wsr, lane * 16 + ((b * 4 + m) * 2 + 0) * 1024, OFF_XBUF + ((xl + (h ^ 1)) * 8 + wid) * 16384u, AUX_SC1));
                const f32x4 p1 = __builtin_bit_cast(f32x4, __builtin_amdgcn_raw_buffer_load_b128(wsr, lane * 16 + ((b * 4 + m) * 2 + 1) * 1024, OFF_XBUF + ((xl + (h ^ 1)) * 8 + wid) * 16384u, AUX_SC1));
                const f32x4 v0 = acc[0][b][m][0] + p0, v1 = acc[0][b][m][1] + p1;
                if (EPI_PACE) { asm volatile("" :: "v"(v0), "v"(v1)); __builtin_amdgcn_s_sleep(EPI_PACE); __builtin_amdgcn_sched_barrier(0); }
                u32x4 o;
                o.x = cvt_pk_bf16(gelu_f(v0[0]), gelu_f(v0[1])); o.y = cvt_pk_bf16(gelu_f(v0[2]), gelu_f(v0[3]));
                o.z = cvt_pk_bf16(gelu_f(v1[0]), gelu_f(v1[1])); o.w = cvt_pk_bf16(gelu_f(v1[2]), gelu_f(v1[3]));
                const int row = m * 16 + fr, ch = b * 4 + fq;
                *(LAS u32x4*)(my + row * 128 + ((ch ^ (row & 7)) * 16)) = o;
            }
        asm volatile("s_waitcnt lgkmcnt(0)" ::: "memory");
        u32x4 c[8];
#pragma unroll
        for (int j = 0; j < 8; ++j) { const int row = j * 8 + (lane >> 3); c[j] = *(LAS u32x4*)(my + row * 128 + (((lane & 7) ^ (row & 7)) * 16)); }
        if (dead) {
#pragma unroll
            for (int j = 0; j < 8; ++j) c[j] = (u32x4){0x7FC07FC0u, 0x7FC07FC0u, 0x7FC07FC0u, 0x7FC07FC0u};
        }
        {
            const rsrc_t outr = mk_rsrc(P.out, (unsigned)DIM * DIM * 2u);
            const unsigned row0 = (P.rank >> 2) * 4096 + r * 2048 + tm * 256 + h * 128 + wr * 64, col0 = (P.rank & 3) * 2048 + tn * 256 + wc * 64;
            const unsigned dso = (row0 * DIM + col0) * 2u, vo = (unsigned)(lane >> 3) * (DIM * 2u) + (unsigned)(lane & 7) * 16u;
#pragma unroll
            for (int j = 0; j < 8; ++j) __builtin_amdgcn_raw_buffer_store_b128(c[j], outr, vo, dso + j * (8u * DIM * 2u), AUX_SC1);
            const unsigned o = (xl + h) * 8 + wid;
            if (r == 0) { WAIT_VM0(); signal_cnt(ws, OFF_OCNT + (o >> 3) * 128, lane); }
            else publish_block(P, wsr, c, 4096u + o, 1536u + o, 0xFFu & ~(1u << P.rank), call, lds, my, 1u + GROUNDS, wid, lane);
        }
        STAMP(ws, r * 8 + 4);
        asm volatile("s_waitcnt lgkmcnt(0)" ::: "memory");
        G_BAR;
    }
#undef G_SA
#undef G_SB
#undef G_STAGE
#undef G_LDA
#undef G_LDB
#undef G_MMA
#undef G_WAIT_V
#undef G_WAIT_L
#undef G_BAR
#undef G_SCHED
#undef G_GIDX
}

__global__ __launch_bounds__(512, 2) void gemm_allgather_kernel(Params P) {
    extern __shared__ __attribute__((aligned(16))) unsigned char shm[];
    LAS unsigned char* lds = (LAS unsigned char*)shm;
    char* ws = P.ws[P.rank];
    const int b = rfl(blockIdx.x);
    STAMP(ws, 60);
    unsigned call;
    if (b >= NPACK + NGEMM) { call = call_draw(ws); puller_role(P, lds, rfl(call), rfl(b - NPACK - NGEMM)); }
    else {
        if (threadIdx.x < 8) *(volatile LAS unsigned*)(lds + GCTL + 128 + threadIdx.x * 4) = 0u;
        __syncthreads();
        const int wid = rfl(threadIdx.x >> 6);
        const bool gemm = b >= NPACK;
        const int next = gemm ? NFIRST + (b - NPACK) * 8 + wid : NFIRST + GROUNDS * NGEMM * 8 + b * 8 + wid;
        call = packer_role(P, lds, wid < 4 ? b * 4 + wid : next, next, gemm ? NGEMM * 8 : NPACK * 8, gemm ? NFIRST + GROUNDS * NGEMM * 8 : 4096 + 1024);
        if (gemm) { __syncthreads(); gemm_role(P, lds, rfl(call), rfl(b)); }
    }
    STAMP(ws, 61);
    STAMP(ws, 58 + (call & 1));
}

}

extern "C" void kernel_launch(int rank, int nranks, ncclComm_t comm,
                              hipStream_t stream,
                              void* const* d_in, const int* in_sizes, int n_in,
                              void* d_out, int out_size,
                              const int* in_dtypes, int out_dtype,
                              void* const* d_out_all,
                              void* const* d_ws_all, size_t ws_bytes) {
    (void)comm; (void)in_sizes; (void)in_dtypes; (void)out_dtype; (void)d_out_all;
    if (nranks != gar::WORLD || n_in < 2 || ws_bytes < gar::WS_NEED || out_size != gar::DIM * gar::DIM || (unsigned)rank >= (unsigned)gar::WORLD) {
        fprintf(stderr, "unexpected problem shape\n");
        return;
    }
    static bool attr = false;
    if (!attr) {
        hipError_t e = hipFuncSetAttribute((const void*)gar::gemm_allgather_kernel, hipFuncAttributeMaxDynamicSharedMemorySize, gar::LDS_BYTES);
        if (e != hipSuccess) { fprintf(stderr, "hipFuncSetAttribute: %s\n", hipGetErrorString(e)); return; }
        attr = true;
    }
    gar::Params P;
    memset(&P, 0, sizeof P);
    P.x = d_in[0]; P.w = d_in[1]; P.out = d_out; P.rank = rank;
    for (int p = 0; p < gar::WORLD; ++p) P.ws[p] = (char*)d_ws_all[p];
    hipLaunchKernelGGL(gar::gemm_allgather_kernel, dim3(gar::GRID), dim3(512), gar::LDS_BYTES, stream, P);
    hipError_t e = hipGetLastError();
    if (e != hipSuccess) fprintf(stderr, "launch: %s\n", hipGetErrorString(e));
}
```

```cpp
#include <hip/hip_runtime.h>
#include <rccl/rccl.h>
#include <cstdio>
#include <cstdint>
#include <cstring>

namespace gar {

#define LAS __attribute__((address_space(3)))
typedef unsigned u32x4 __attribute__((ext_vector_type(4)));
typedef unsigned short u16x2 __attribute__((ext_vector_type(2)));
typedef short bf16x8 __attribute__((ext_vector_type(8)));
typedef float f32x4 __attribute__((ext_vector_type(4)));
typedef __amdgpu_buffer_rsrc_t rsrc_t;

constexpr int WORLD = 8;
constexpr int DIM = 8192;
constexpr int KSH = 1024;
constexpr int NGEMM = 128;
constexpr int GRID = 256;
#define PULL_WG_PER_LINK 12
#define PULL_DECODERS 7
constexpr int PWL = PULL_WG_PER_LINK;
constexpr int NDEC = PULL_DECODERS;
constexpr int NPULL = 7 * PWL;
constexpr int NPACK = GRID - NGEMM - NPULL;
#define NRAWEND 24
#define WARM_TLB 1
#define NRAW 48
constexpr int NFIRST = (NPACK + NGEMM) * 4;
#define GEMM_PACK_ROUNDS 1
constexpr int GROUNDS = GEMM_PACK_ROUNDS;
static_assert(NFIRST + GROUNDS * NGEMM * 8 <= 4096 && NFIRST % 8 == 0, "roles");
constexpr int NSEQ = 3584;
constexpr unsigned long long DEADLINE = 40000000ull;

constexpr unsigned OFF_TMO = 128;
constexpr unsigned OFF_LINK = 1024;
constexpr unsigned OFF_CNT = 4096;
constexpr unsigned OFF_XFLAG = OFF_CNT + 256 * 128;
constexpr unsigned OFF_OCNT = OFF_XFLAG + 256 * 128;
constexpr unsigned OFF_GRAN = 131072;
constexpr unsigned OFF_CALL = 640u << 10;
constexpr unsigned OFF_SLOT = 1u << 20;
constexpr unsigned OFF_AIMG = 64u << 20;
constexpr unsigned OFF_BIMG = 128u << 20;
constexpr unsigned OFF_XBUF = 160u << 20;
constexpr unsigned WS_USED = 192u << 20;
constexpr unsigned WS_NEED = WS_USED;
static_assert(OFF_OCNT + 128 * 128 <= OFF_GRAN, "map");
static_assert(OFF_GRAN + WORLD * NSEQ * 8 <= OFF_CALL && OFF_CALL + 2048 * 64 <= OFF_SLOT, "map");
static_assert(OFF_SLOT + 6144u * 8192u <= OFF_AIMG, "map");
constexpr int LDS_BYTES = 131072 + 256;

struct Params {
    const void* x; const void* w; void* out;
    char* ws[WORLD];
    int rank; int pad;
};

constexpr int AUX_SC1 = 16, AUX_SYS = 17;

__device__ __forceinline__ rsrc_t mk_rsrc(const void* p, unsigned bytes) {
    return __builtin_amdgcn_make_buffer_rsrc(const_cast<void*>(p), 0, bytes, 0x00020000);
}
__device__ __forceinline__ unsigned rfl(unsigned v) { return __builtin_amdgcn_readfirstlane(v); }
__device__ __forceinline__ unsigned bfi(unsigned mask, unsigned a, unsigned b) { return (a & mask) | (b & ~mask); }
__device__ __forceinline__ unsigned perm(unsigned hi, unsigned lo, unsigned sel) { return __builtin_amdgcn_perm(hi, lo, sel); }
#define WAIT_VM0() asm volatile("s_waitcnt vmcnt(0)" ::: "memory")

#define STAMP(ws, idx) do { } while (0)

__device__ __forceinline__ unsigned call_draw(char* ws) {
    unsigned t;
    const int zero = 0, one = 1;
    unsigned long long keep;
    asm volatile("s_mov_b64 %1, exec\n\ts_mov_b64 exec, 1\n\tglobal_atomic_add %0, %2, %3, %4 sc0\n\ts_mov_b64 exec, %1\n\ts_waitcnt vmcnt(0)"
                 : "=&v"(t), "=&s"(keep) : "v"(zero), "v"(one), "s"(ws + OFF_CALL + (blockIdx.x * 8 + rfl(threadIdx.x >> 6)) * 64) : "memory");
    return rfl(t) + 1u;
}

__device__ __forceinline__ bool give_up(unsigned long long t0, unsigned* tmo, unsigned call) {
    if (rfl(__hip_atomic_load(tmo, __ATOMIC_RELAXED, __HIP_MEMORY_SCOPE_AGENT)) == call) return true;
    if (__builtin_amdgcn_s_memrealtime() - t0 > DEADLINE) { __hip_atomic_store(tmo, call, __ATOMIC_RELAXED, __HIP_MEMORY_SCOPE_AGENT); return true; }
    return false;
}
__device__ __forceinline__ void wait_word(unsigned* p, unsigned target, unsigned* tmo, unsigned call) {
    if ((int)(rfl(__hip_atomic_load(p, __ATOMIC_RELAXED, __HIP_MEMORY_SCOPE_SYSTEM)) - target) >= 0) return;
    const unsigned long long t0 = __builtin_amdgcn_s_memrealtime();
    while ((int)(rfl(__hip_atomic_load(p, __ATOMIC_RELAXED, __HIP_MEMORY_SCOPE_SYSTEM)) - target) < 0) {
        if (give_up(t0, tmo, call)) break;
        __builtin_amdgcn_s_sleep(4);
    }
}

constexpr unsigned MAX_EXC = 31;
constexpr unsigned RAWB = 3584, AUX_U = 20, AUX_MAX = 2560, U_MAX = 10240, LANE_U_MAX = 224;
constexpr unsigned ENC_HIST = AUX_MAX + 16, ENC_SORT = ENC_HIST + 256, ENC_END = ENC_SORT + 128;
constexpr unsigned DEC_POS = 6144;

__device__ __forceinline__ unsigned bfi_c(unsigned mask, unsigned a, unsigned b) { unsigned r; asm("v_bfi_b32 %0, %1, %2, %3" : "=v"(r) : "s"(mask), "v"(a), "v"(b)); return r; }
__device__ __forceinline__ unsigned opaque(unsigned v) { asm("" : "+v"(v)); return v; }
__device__ __forceinline__ unsigned bytemask(unsigned x, int sh, unsigned c0c) {
    unsigned t; asm("v_and_or_b32 %0, %1, %2, %3" : "=v"(t) : "v"(x >> sh), "s"(0x01010101u), "v"(c0c));
    return perm(0u, 0u, t);
}
template <int CTRL, int RM = 0xf> __device__ __forceinline__ unsigned dpp0(unsigned v) { return (unsigned)__builtin_amdgcn_update_dpp(0, (int)v, CTRL, RM, 0xf, RM == 0xf); }
__device__ __forceinline__ unsigned wave_scan_incl(unsigned v) {
    v += dpp0<0x111>(v); v += dpp0<0x112>(v); v += dpp0<0x114>(v); v += dpp0<0x118>(v);
    v += dpp0<0x142, 0xa>(v); v += dpp0<0x143, 0xc>(v);
    return v;
}
__device__ __forceinline__ unsigned umin(unsigned a, unsigned b) { return a < b ? a : b; }
__device__ __forceinline__ unsigned umax(unsigned a, unsigned b) { return a > b ? a : b; }
__device__ __forceinline__ unsigned sad4(unsigned x) { return __builtin_amdgcn_sad_u8(x, 0u, 0u); }
__device__ __forceinline__ unsigned lut2x16(const unsigned (&T)[8], unsigned idx, unsigned ms, unsigned c0c) {
    const unsigned sel = idx & 0x07070707u, m8 = bytemask(idx, 3, c0c);
    const unsigned p = bfi(m8, perm(T[3], T[2], sel), perm(T[1], T[0], sel)), n = bfi(m8, perm(T[7], T[6], sel), perm(T[5], T[4], sel));
    return bfi(ms, n, p);
}
#define LDS_FENCE() asm volatile("" ::: "memory")
__device__ __forceinline__ void lds_or2(LAS unsigned char* scr, unsigned p, unsigned w) {
    const unsigned s = p & 31u;
    LAS unsigned* q = (LAS unsigned*)(scr + ((p >> 5) << 2));
    __hip_atomic_fetch_or(q, w << s, __ATOMIC_RELAXED, __HIP_MEMORY_SCOPE_WORKGROUP);
    __hip_atomic_fetch_or(q + 1, (w >> 1) >> (31u - s), __ATOMIC_RELAXED, __HIP_MEMORY_SCOPE_WORKGROUP);
}

__device__ __forceinline__ unsigned encode_block(const u32x4 (&c)[8], unsigned (&P)[14], u32x4 (&a)[3], LAS unsigned char* scr, int lane) {
    asm volatile("" : "+v"(lane));
    const unsigned c0c = opaque(0x0C0C0C0Cu);
    unsigned E[16], R[16];
    u16x2 p1 = {0, 0}, p2 = {0, 0}, n1 = {0, 0}, n2 = {0, 0};
#pragma unroll
    for (int g = 0; g < 16; ++g) {
        const unsigned x = (g & 1) ? c[g >> 1].z : c[g >> 1].x, y = (g & 1) ? c[g >> 1].w : c[g >> 1].y;
        const unsigned H = perm(y, x, 0x07050301u), L = perm(y, x, 0x06040200u);
        E[g] = bfi_c(0xFEFEFEFEu, H << 1, L >> 7);
        R[g] = bfi_c(0x80808080u, L << 1, bfi_c(0x40404040u, H >> 1, L));
        const unsigned en = E[g] & bytemask(H, 7, c0c), ep = E[g] ^ en;
        p1 = __builtin_elementwise_max(p1, __builtin_bit_cast(u16x2, ep)); p2 = __builtin_elementwise_max(p2, __builtin_bit_cast(u16x2, ep << 8));
        n1 = __builtin_elementwise_max(n1, __builtin_bit_cast(u16x2, en)); n2 = __builtin_elementwise_max(n2, __builtin_bit_cast(u16x2, en << 8));
        __builtin_amdgcn_sched_barrier(0);
    }
    const u32x4 z4 = {0u, 0u, 0u, 0u};
    *(LAS u32x4*)(scr + lane * 16) = z4; *(LAS u32x4*)(scr + 1024 + lane * 16) = z4;
    if (lane < 49) *(LAS u32x4*)(scr + 2048 + lane * 16) = z4;
    unsigned baseP, baseN;
    {
        const u16x2 mp = __builtin_elementwise_max(p1, p2), mn = __builtin_elementwise_max(n1, n2);
        u16x2 m = {(unsigned short)((mp.x > mp.y ? mp.x : mp.y) >> 8), (unsigned short)((mn.x > mn.y ? mn.x : mn.y) >> 8)};
        m = __builtin_elementwise_max(m, __builtin_bit_cast(u16x2, dpp0<0x128>(__builtin_bit_cast(unsigned, m)))); m = __builtin_elementwise_max(m, __builtin_bit_cast(u16x2, dpp0<0x124>(__builtin_bit_cast(unsigned, m))));
        m = __builtin_elementwise_max(m, __builtin_bit_cast(u16x2, dpp0<0x122>(__builtin_bit_cast(unsigned, m)))); m = __builtin_elementwise_max(m, __builtin_bit_cast(u16x2, dpp0<0x121>(__builtin_bit_cast(unsigned, m))));
        const unsigned mu = __builtin_bit_cast(unsigned, m);
        const u16x2 r01 = __builtin_elementwise_max(__builtin_bit_cast(u16x2, (unsigned)__builtin_amdgcn_readlane(mu, 0)), __builtin_bit_cast(u16x2, (unsigned)__builtin_amdgcn_readlane(mu, 16)));
        const u16x2 r23 = __builtin_elementwise_max(__builtin_bit_cast(u16x2, (unsigned)__builtin_amdgcn_readlane(mu, 32)), __builtin_bit_cast(u16x2, (unsigned)__builtin_amdgcn_readlane(mu, 48)));
        const unsigned mm = rfl(__builtin_bit_cast(unsigned, __builtin_elementwise_max(r01, r23)));
        baseP = mm & 0xFFu; baseN = mm >> 16;
    }
    const unsigned BP = baseP * 0x01010101u, BN = baseN * 0x01010101u;
    unsigned any = 0, D[16];
#pragma unroll
    for (int g = 0; g < 16; ++g) {
        E[g] = bfi(bytemask(R[g], 6, c0c), BN, BP) - E[g]; any |= E[g];
        D[g] = bfi_c(0x1E1E1E1Eu, E[g] << 1, (R[g] >> 7) & 0x01010101u);
        R[g] &= 0x7F7F7F7Fu;
    }
    LDS_FENCE();
#pragma unroll
    for (int gi = 0; gi < 4; ++gi) {
        const int g = gi * 5;
        const unsigned k4 = ((D[g] & 0x0F0F0F0Fu) | ((R[g] >> 2) & 0x10101010u) | ((D[g] << 1) & 0x20202020u)) << 2;
#pragma unroll
        for (int b = 0; b < 4; ++b)
            __hip_atomic_fetch_add((LAS unsigned*)(scr + ENC_HIST + ((k4 >> (8 * b)) & 0xFFu)), 1u, __ATOMIC_RELAXED, __HIP_MEMORY_SCOPE_WORKGROUP);
    }
    LDS_FENCE();
    const unsigned cnt = *(LAS unsigned*)(scr + ENC_HIST + lane * 4);
    const unsigned key = (cnt << 4) | (15u - (lane & 15));
    unsigned rank = 0;
    rank += dpp0<0x121>(key) > key; rank += dpp0<0x122>(key) > key; rank += dpp0<0x123>(key) > key; rank += dpp0<0x124>(key) > key; rank += dpp0<0x125>(key) > key;
    rank += dpp0<0x126>(key) > key; rank += dpp0<0x127>(key) > key; rank += dpp0<0x128>(key) > key; rank += dpp0<0x129>(key) > key; rank += dpp0<0x12A>(key) > key;
    rank += dpp0<0x12B>(key) > key; rank += dpp0<0x12C>(key) > key; rank += dpp0<0x12D>(key) > key; rank += dpp0<0x12E>(key) > key; rank += dpp0<0x12F>(key) > key;
    unsigned tail = cnt; tail += dpp0<0x128>(tail); tail += dpp0<0x124>(tail); tail += dpp0<0x122>(tail); tail += dpp0<0x121>(tail);
    const int sg = (lane >> 4) & 1;
    const unsigned ct = sg ? __builtin_amdgcn_readlane(tail, 48) : __builtin_amdgcn_readlane(tail, 32);
    if (lane < 32) *(LAS unsigned*)(scr + ENC_SORT + ((lane & 16) + rank) * 4) = cnt;
    LDS_FENCE();
    unsigned best;
    {
        const unsigned cb = lane & 15, s0 = cb / 3u, s1 = cb - 3u * s0, A = 1u << s0, AB = A + (1u << s1);
        unsigned cost = ct * (16u - AB);
#pragma unroll
        for (int q = 0; q < 4; ++q) {
            const u32x4 h = *(LAS u32x4*)(scr + ENC_SORT + sg * 64 + q * 16);
#pragma unroll
            for (int j = 0; j < 4; ++j) {
                const unsigned r = q * 4 + j, len = r < A ? 1u + s0 : r < AB ? 2u + s1 : 4u + ((r - AB) >> 1);
                cost += 2u * len * h[j];
            }
        }
        best = cb < 9 ? (cost << 4) | cb : 0xFFFFFFFFu;
        best = umin(best, dpp0<0x128>(best)); best = umin(best, dpp0<0x124>(best)); best = umin(best, dpp0<0x122>(best)); best = umin(best, dpp0<0x121>(best));
    }
    const unsigned cbP = __builtin_amdgcn_readlane(best, 0) & 15u, cbN = __builtin_amdgcn_readlane(best, 16) & 15u;
    const unsigned s0P = cbP / 3u, s1P = cbP - 3u * s0P, s0N = cbN / 3u, s1N = cbN - 3u * s0N;
    const unsigned ABP = (1u << s0P) + (1u << s1P), ABN = (1u << s0N) + (1u << s1N);
    unsigned F[8];
    {
        const unsigned s0 = sg ? s0N : s0P, s1 = sg ? s1N : s1P, A = 1u << s0, AB = sg ? ABN : ABP;
        const unsigned ent = rank < A ? (rank << 4) | (s0 << 6) : rank < AB ? 1u | ((rank - A) << 4) | (s1 << 6) : (2u + ((rank - AB) >> 1)) | (((rank - AB) & 1u) << 4) | (1u << 6);
        unsigned fv = ent << (8 * (lane & 3));
        fv |= dpp0<0xB1>(fv); fv |= dpp0<0x4E>(fv);
#pragma unroll
        for (int q = 0; q < 8; ++q) F[q] = __builtin_amdgcn_readlane(fv, q * 4);
        if (lane < 32)
            __hip_atomic_fetch_or((LAS unsigned*)(scr + (sg * 2 + (rank >> 3)) * 4), (unsigned)(lane & 15) << (8 * (rank & 3) + 4 * ((rank >> 2) & 1)), __ATOMIC_RELAXED, __HIP_MEMORY_SCOPE_WORKGROUP);
        if (lane == 0) *(LAS unsigned*)(scr + 16) = baseP | (baseN << 8) | (s0P << 16) | (s1P << 18) | (s0N << 20) | (s1N << 22);
    }
    const unsigned ABP4 = ABP * 0x01010101u, ABN4 = ABN * 0x01010101u, S0P4 = s0P * 0x01010101u, S0N4 = s0N * 0x01010101u;
    unsigned KK[16], XW[4], OX[4], TU = 0, TX = 0, bad = 0, xl = 0, xw = 0;
#pragma unroll
    for (int g = 0; g < 16; ++g) {
        const unsigned ms = bytemask(R[g], 6, c0c), m16 = bytemask(D[g], 4, c0c), ent = lut2x16(F, D[g], ms, c0c);
        const unsigned q = ((D[g] | 0x80808080u) - bfi(ms, ABN4, ABP4)) & 0x7F7F7F7Fu;
        unsigned K = bfi(m16, ((q >> 1) & 0x3F3F3F3Fu) + 0x02020202u, ent & 0x0F0F0F0Fu);
        unsigned SUF = bfi(m16, q & 0x01010101u, (ent >> 4) & 0x03030303u);
        unsigned S = bfi(m16, 0x01010101u, (ent >> 6) & 0x03030303u);
        unsigned gl = sad4(K) + 4u;
        if (__ballot(gl > 31u) != 0ull) {
            any |= 0x80u;
            const unsigned km = umax(umax(K & 0xFFu, (K >> 8) & 0xFFu), umax((K >> 16) & 0xFFu, K >> 24));
            const unsigned eq = gl > 31u ? (((K ^ (km * 0x01010101u)) + 0x7F7F7F7Fu) & 0x80808080u) ^ 0x80808080u : 0u, em = bytemask(eq, 7, c0c);
            E[g] |= eq;
            K &= ~em; SUF &= ~em; S = bfi(em, bfi(ms, S0N4, S0P4), S);
            gl = sad4(K) + 4u;
        }
        KK[g] = K; TU += gl; bad = umax(bad, gl);
        xw |= (SUF & 0xFFu) << xl; xl += S & 0xFFu;
        xw |= ((SUF >> 8) & 0xFFu) << xl; xl += (S >> 8) & 0xFFu;
        xw |= ((SUF >> 16) & 0xFFu) << xl; xl += (S >> 16) & 0xFFu;
        xw |= (SUF >> 24) << (xl & 31u); xl += S >> 24;
        if ((g & 3) == 3) { XW[g >> 2] = xw; OX[g >> 2] = TX; TX += xl; xw = 0; xl = 0; }
        __builtin_amdgcn_sched_barrier(0);
    }
    bad = (bad > 31u) | (TU > LANE_U_MAX);
    const unsigned tt = TU | (TX << 16), incl = wave_scan_incl(tt), tot = __builtin_amdgcn_readlane(incl, 63), totU = tot & 0xFFFFu, totX = tot >> 16;
    const unsigned xstart = (AUX_U * 8u + totU + 31u) & ~31u, endbit = xstart + totX;
    unsigned raw = (__ballot(bad != 0) != 0ull) || totU > U_MAX || endbit > (AUX_MAX - 8u) * 8u;
    if (!raw) {
        const unsigned px = xstart + ((incl - tt) >> 16);
        unsigned p = AUX_U * 8u + ((incl - tt) & 0xFFFFu);
#pragma unroll
        for (int g = 0; g < 16; ++g) {
            const unsigned ln = KK[g] + 0x01010101u;
            unsigned pos = (ln & 0xFFu) - 1u, w = 1u << pos;
            pos += (ln >> 8) & 0xFFu; w |= 1u << pos;
            pos += (ln >> 16) & 0xFFu; w |= 1u << pos;
            pos += ln >> 24; w |= 1u << pos;
            lds_or2(scr, p, w); p += pos + 1u;
            __builtin_amdgcn_sched_barrier(0);
        }
#pragma unroll
        for (int q = 0; q < 4; ++q) lds_or2(scr, px + OX[q], XW[q]);
    }
#pragma unroll
    for (int i = 0; i < 7; ++i) { P[i] = R[i] | ((R[14] << (7 - i)) & 0x80808080u); P[7 + i] = R[7 + i] | ((R[15] << (7 - i)) & 0x80808080u); }
    unsigned n = 0;
    const unsigned xo = (endbit + 31u) >> 5;
    if (!raw && __ballot((any & 0xF0F0F0F0u) != 0) != 0ull) {
#pragma unroll
        for (int g = 0; g < 16; ++g) {
            const unsigned f = E[g] & 0xF0F0F0F0u;
            unsigned long long mask = __ballot(f != 0);
            while (mask != 0ull && n <= MAX_EXC) {
                const int L = __builtin_ctzll(mask); mask &= mask - 1;
                const unsigned fl = __builtin_amdgcn_readlane(f, L);
                const unsigned lo = __builtin_amdgcn_readlane((g & 1) ? c[g >> 1].z : c[g >> 1].x, L);
                const unsigned hi = __builtin_amdgcn_readlane((g & 1) ? c[g >> 1].w : c[g >> 1].y, L);
                const bool e = lane < 4 && ((fl >> (8 * (lane & 3))) & 0xFFu) != 0;
                const unsigned long long em = __ballot(e);
                const unsigned slot = xo + n + __builtin_popcountll(em & ((1ull << lane) - 1ull));
                const unsigned val = (((lane & 2) ? hi : lo) >> (16 * (lane & 1))) & 0xFFFFu;
                if (e && slot * 4u < AUX_MAX) *(LAS unsigned*)(scr + slot * 4u) = (unsigned)(L * 64 + g * 4 + (lane & 3)) | (val << 12);
                n += __builtin_popcountll(em);
            }
        }
    }
    raw |= (n > MAX_EXC) | ((xo + n) * 4u > AUX_MAX);
    LDS_FENCE();
    a[0] = *(LAS u32x4*)(scr + lane * 16); a[1] = *(LAS u32x4*)(scr + 1024 + lane * 16); a[2] = *(LAS u32x4*)(scr + 2048 + (lane & 31) * 16);
    return ((xo + n + 3u) >> 2) | (n << 8) | (xo << 16) | (raw << 31);
}

__device__ __forceinline__ void decode_block(LAS unsigned char* src, u32x4 (&c)[8], int lane) {
    LAS unsigned char* aux = src + RAWB;
    const unsigned c0c = opaque(0x0C0C0C0Cu);
    const u32x4 nt = *(LAS const u32x4*)(aux);
    const unsigned prm_v = *(LAS const unsigned*)(aux + 16);
    unsigned u[5];
#pragma unroll
    for (int i = 0; i < 5; ++i) u[i] = *(LAS const unsigned*)(aux + AUX_U + lane * 20 + i * 4);
    u32x4 P4[3];
#pragma unroll
    for (int p = 0; p < 3; ++p) P4[p] = *(LAS const u32x4*)(src + p * 1024 + lane * 16);
    const unsigned long long ph = *(LAS const unsigned long long*)(src + 3072 + lane * 8);
    const unsigned c0 = __builtin_popcount(u[0]), c1 = c0 + __builtin_popcount(u[1]), c2 = c1 + __builtin_popcount(u[2]), c3 = c2 + __builtin_popcount(u[3]), cn = c3 + __builtin_popcount(u[4]);
    const unsigned C = wave_scan_incl(cn) - cn;
    if (lane == 0) *(LAS unsigned*)(src + DEC_POS) = AUX_U * 8u;
    const int rounds = __ballot(cn > 128u) != 0ull ? 3 : 2;
    for (int j = 0; j < rounds; ++j) {
        const unsigned t = (C >> 6) + 1u + j, kth = 64u * t - C;
        const unsigned i = (kth > c0) + (kth > c1) + (kth > c2) + (kth > c3);
        unsigned rem = kth - (i == 0 ? 0u : i == 1 ? c0 : i == 2 ? c1 : i == 3 ? c2 : c3), w = i == 0 ? u[0] : i == 1 ? u[1] : i == 2 ? u[2] : i == 3 ? u[3] : u[4], pos = 0;
#pragma unroll
        for (int st = 16; st >= 1; st >>= 1) {
            const unsigned lo = __builtin_popcount(w & ((1u << st) - 1u));
            const bool up = rem > lo;
            rem -= up ? lo : 0u; w = up ? w >> st : w; pos += up ? st : 0;
        }
        if (kth <= cn && t < 64u) *(LAS unsigned*)(src + DEC_POS + t * 4) = AUX_U * 8u + lane * 160u + i * 32u + pos + 1u;
    }
    LDS_FENCE();
    const unsigned p0 = *(LAS const unsigned*)(src + DEC_POS + lane * 4);
    unsigned S[8];
    {
        LAS const unsigned* q = (LAS const unsigned*)(aux + ((p0 >> 5) << 2));
#pragma unroll
        for (int i = 0; i < 8; ++i) S[i] = q[i];
#pragma unroll
        for (int i = 0; i < 7; ++i) S[i] = __builtin_amdgcn_alignbit(S[i + 1], S[i], p0 & 31u);
    }
    unsigned R[16];
    {
        const unsigned P[14] = {P4[0].x, P4[0].y, P4[0].z, P4[0].w, P4[1].x, P4[1].y, P4[1].z, P4[1].w, P4[2].x, P4[2].y, P4[2].z, P4[2].w, (unsigned)ph, (unsigned)(ph >> 32)};
        unsigned x = 0, y = 0;
#pragma unroll
        for (int i = 0; i < 7; ++i) { x |= (P[i] >> (7 - i)) & (0x01010101u << i); y |= (P[7 + i] >> (7 - i)) & (0x01010101u << i); }
#pragma unroll
        for (int i = 0; i < 14; ++i) R[i] = P[i] & 0x7F7F7F7Fu;
        R[14] = x; R[15] = y;
    }
    const unsigned n0 = rfl(nt.x), n1 = rfl(nt.y), n2 = rfl(nt.z), n3 = rfl(nt.w), prm = rfl(prm_v);
    const unsigned I[8] = {n0 & 0x0F0F0F0Fu, (n0 >> 4) & 0x0F0F0F0Fu, n1 & 0x0F0F0F0Fu, (n1 >> 4) & 0x0F0F0F0Fu, n2 & 0x0F0F0F0Fu, (n2 >> 4) & 0x0F0F0F0Fu, n3 & 0x0F0F0F0Fu, (n3 >> 4) & 0x0F0F0F0Fu};
    const unsigned s0P = (prm >> 16) & 3u, s1P = (prm >> 18) & 3u, s0N = (prm >> 20) & 3u, s1N = (prm >> 22) & 3u;
    const unsigned STP = (s0P << 8) | (s1P << 16) | (1u << 24), STN = (s0N << 8) | (s1N << 16) | (1u << 24);
    const unsigned OTP = (4u << 8) | (((1u << s0P) + 2u) << 16) | (((1u << s0P) + (1u << s1P)) << 24);
    const unsigned OTN = (4u << 8) | (((1u << s0N) + 2u) << 16) | (((1u << s0N) + (1u << s1N)) << 24);
    const unsigned BP = (prm & 0xFFu) * 0x01010101u, BN = ((prm >> 8) & 0xFFu) * 0x01010101u;
    unsigned RB[16], SS[16], TU = 0, TX = 0;
#pragma unroll
    for (int g = 0; g < 16; ++g) {
        const unsigned w0 = S[0], q0 = __builtin_ctz(w0), w1 = w0 & (w0 - 1u), q1 = __builtin_ctz(w1), w2 = w1 & (w1 - 1u), q2 = __builtin_ctz(w2), w3 = w2 & (w2 - 1u), q3 = __builtin_ctz(w3);
        const unsigned Q = q0 | (q1 << 8) | (q2 << 16) | (q3 << 24);
        const unsigned LN = Q - opaque((Q << 8) - 1u), sh = q3 + 1u;
        TU += sh;
        if (g < 15) {
#pragma unroll
            for (int i = 0; i < 7; ++i) if (32 * i < (int)LANE_U_MAX - 4 * (g + 1)) S[i] = (i < 6 && 32 * (i + 1) < (int)LANE_U_MAX - 4 * g) ? __builtin_amdgcn_alignbit(S[i + 1], S[i], sh) : S[i] >> sh;
        }
        const unsigned sel = bfi_c(0x04040404u, R[g] >> 4, bfi(bytemask(LN + 0x7D7D7D7Du, 7, c0c), 0x03030303u, LN));
        SS[g] = perm(STN, STP, sel);
        RB[g] = perm(OTN, OTP, sel) + (LN << 1);
        TX = __builtin_amdgcn_sad_u8(SS[g], 0u, TX);
    }
    const unsigned xstart = (__builtin_amdgcn_readlane(p0 + TU, 63) + 31u) & ~31u, px = xstart + wave_scan_incl(TX) - TX;
    unsigned X[5];
    {
        LAS const unsigned* q = (LAS const unsigned*)(aux + ((px >> 5) << 2));
#pragma unroll
        for (int i = 0; i < 5; ++i) X[i] = q[i];
#pragma unroll
        for (int i = 0; i < 4; ++i) X[i] = __builtin_amdgcn_alignbit(X[i + 1], X[i], px & 31u);
    }
    unsigned xoff = 0;
#pragma unroll
    for (int g = 0; g < 16; ++g) {
        const unsigned s = SS[g], M = perm(0u, 0x00030100u, s);
        const unsigned y = (g & 1) ? X[0] >> xoff : X[0], Y = perm(y, y, 0x01000100u);
        const unsigned t1 = opaque(s << 8), t2 = opaque((t1 << 8) + t1), Cs = (t2 << 16) + t2;
        const u16x2 z02 = __builtin_bit_cast(u16x2, Y) >> __builtin_bit_cast(u16x2, Cs & 0x00FF00FFu);
        const u16x2 z13 = __builtin_bit_cast(u16x2, Y) >> __builtin_bit_cast(u16x2, perm(0u, Cs, 0x0C030C01u));
        const unsigned suf = perm(__builtin_bit_cast(unsigned, z13), __builtin_bit_cast(unsigned, z02), 0x06020400u) & M;
        if (g & 1) {
            if (g < 15) {
                const unsigned t = __builtin_amdgcn_sad_u8(s, 0u, xoff);
#pragma unroll
                for (int i = 0; i < 4; ++i) if (32 * i < 8 * (15 - g)) X[i] = (i < 3 && 32 * (i + 1) < 8 * (17 - g)) ? __builtin_amdgcn_alignbit(X[i + 1], X[i], t) : X[i] >> t;
            }
        } else xoff = sad4(s);
        const unsigned rank = RB[g] + suf - 0x06060606u, ms = bytemask(R[g], 6, c0c);
        const unsigned D = bfi(bytemask(rank, 4, c0c), rank, lut2x16(I, rank, ms, c0c));
        const unsigned E = bfi(ms, BN, BP) - ((D >> 1) & 0x0F0F0F0Fu);
        const unsigned H = bfi_c(0x80808080u, R[g] << 1, E >> 1), L = bfi_c(0x3F3F3F3Fu, R[g], bfi_c(0x40404040u, D << 6, E << 7));
        const unsigned x = perm(H, L, 0x05010400u), yv = perm(H, L, 0x07030602u);
        if (g & 1) { c[g >> 1].z = x; c[g >> 1].w = yv; } else { c[g >> 1].x = x; c[g >> 1].y = yv; }
    }
}

typedef unsigned u32x2 __attribute__((ext_vector_type(2)));
__device__ __forceinline__ void store_granule(char* p, u32x2 g) {
    asm volatile("global_store_dwordx2 %0, %1, off sc0 sc1\n\ts_nop 1" :: "v"(p), "v"(g) : "memory");
}

constexpr unsigned GCTL = 131072;
__device__ __forceinline__ void publish_block(const Params& P, rsrc_t wsr, const u32x4 (&c)[8], unsigned slot, unsigned n, unsigned dmask, unsigned call, LAS unsigned char* lds, LAS unsigned char* scr,
                                              unsigned j, int wid, int lane) {
    unsigned R[14], h = 1u << 31; u32x4 a[3];
    if (n >= NRAW && n < NSEQ - NRAWEND) h = encode_block(c, R, a, scr, lane);
    const unsigned so = OFF_SLOT + slot * 8192u;
    if (h >> 31) {
#pragma unroll
        for (int q = 0; q < 8; ++q) __builtin_amdgcn_raw_buffer_store_b128(c[q], wsr, lane * 16 + q * 1024, so, AUX_SYS);
    } else {
        const unsigned n16 = h & 255u;
#pragma unroll
        for (int q = 0; q < 3; ++q) __builtin_amdgcn_raw_buffer_store_b128((u32x4){R[4 * q], R[4 * q + 1], R[4 * q + 2], R[4 * q + 3]}, wsr, lane * 16 + q * 1024, so, AUX_SYS);
        __builtin_amdgcn_raw_buffer_store_b64((u32x2){R[12], R[13]}, wsr, lane * 8 + 3072, so, AUX_SYS);
        if ((unsigned)lane < n16) __builtin_amdgcn_raw_buffer_store_b128(a[0], wsr, lane * 16 + RAWB, so, AUX_SYS);
        if ((unsigned)lane + 64u < n16) __builtin_amdgcn_raw_buffer_store_b128(a[1], wsr, lane * 16 + RAWB + 1024, so, AUX_SYS);
        if ((unsigned)lane + 128u < n16) __builtin_amdgcn_raw_buffer_store_b128(a[2], wsr, lane * 16 + RAWB + 2048, so, AUX_SYS);
    }
    WAIT_VM0();
    const unsigned u = j & 3u, gsize = j == 0 ? 4u : 8u;
    volatile LAS unsigned* ctl = (volatile LAS unsigned*)(lds + GCTL);
    while (j >= 4u && (int)(ctl[36 + u] - (j - 3u)) < 0) __builtin_amdgcn_s_sleep(1);
    unsigned arrived = 0;
    if (lane == 0) {
        ctl[u * 8 + wid] = h;
        arrived = __hip_atomic_fetch_add((LAS unsigned*)(lds + GCTL) + 32 + u, 1u, __ATOMIC_RELAXED, __HIP_MEMORY_SCOPE_WORKGROUP);
    }
    if (rfl(arrived) == gsize - 1u) {
        const unsigned d = lane >> 3, w = lane & 7, hw = ctl[u * 8 + w];
        if (w < gsize && ((dmask >> d) & 1u)) store_granule(P.ws[d] + OFF_GRAN + ((unsigned)P.rank * NSEQ + n - wid + w) * 8u, (u32x2){call, hw});
        if (lane == 0) { ctl[32 + u] = 0u; ctl[36 + u] = j + 1u; }
    }
}

__device__ __forceinline__ void signal_cnt(char* ws, unsigned off, int lane) {
    if (off && lane == 0) __hip_atomic_fetch_add((unsigned*)(ws + off), 1u, __ATOMIC_RELAXED, __HIP_MEMORY_SCOPE_SYSTEM);
}
__device__ __forceinline__ unsigned packer_role(const Params& P, LAS unsigned char* lds, int first, int next, int step, int end) {
    unsigned call = 0;
    const int lane = threadIdx.x & 63, wid = rfl(threadIdx.x >> 6);
    char* ws = P.ws[P.rank];
    const rsrc_t wsr = mk_rsrc(ws, WS_USED);
    const rsrc_t xr = mk_rsrc(P.x, DIM * KSH * 2), wr = mk_rsrc(P.w, DIM * KSH * 2);
    const int myi = P.rank >> 2, myj = P.rank & 3;
    LAS unsigned char* my = lds + wid * 8192;
    unsigned j = first < NFIRST ? 0u : 1u;
    for (int p = first; p < end; p = p < NFIRST ? next : p + step, ++j) {
        u32x4 c[8];
        unsigned slot, n, dmask, selfcnt = 0;
        if (p >= 4096) {
            const unsigned o = p - 4096, w = o & 7, h = (o >> 3) & 1, tile = (o >> 4) & 63;
            if (call == 0) call = call_draw(ws);
            unsigned* tmo = (unsigned*)(ws + OFF_TMO);
            wait_word((unsigned*)(ws + OFF_OCNT + (o >> 3) * 128), call * 8u, tmo, call);
            const unsigned row0 = (P.rank >> 2) * 4096 + (tile >> 3) * 256 + h * 128 + (w >> 2) * 64, col0 = (P.rank & 3) * 2048 + (tile & 7) * 256 + (w & 3) * 64;
            const rsrc_t outr = mk_rsrc(P.out, (unsigned)DIM * DIM * 2u);
#pragma unroll
            for (int q = 0; q < 8; ++q) c[q] = __builtin_amdgcn_raw_buffer_load_b128(outr, (unsigned)(lane >> 3) * (DIM * 2u) + (unsigned)(lane & 7) * 16u, (row0 * DIM + col0) * 2u + q * (8u * DIM * 2u), AUX_SYS);
            if (rfl(__hip_atomic_load(tmo, __ATOMIC_RELAXED, __HIP_MEMORY_SCOPE_AGENT)) == call) {
#pragma unroll
                for (int q = 0; q < 8; ++q) c[q] = (u32x4){0x7FC07FC0u, 0x7FC07FC0u, 0x7FC07FC0u, 0x7FC07FC0u};
            }
            slot = 4096u + o; n = 1536u + o; dmask = 0xFFu;
        } else {
            bool is_w; int kk, sel, r, bi;
            if (p < 3072) {
                kk = p / 192; const int rem = p - kk * 192;
                bi = rem & 31; r = 0; is_w = rem >= 64; sel = is_w ? ((rem - 64) >> 5) : (rem >> 5);
            } else { const int q = p - 3072; kk = q >> 6; sel = (q >> 5) & 1; bi = q & 31; r = 1; is_w = false; }
            if (!is_w) {
                const int rr = lane >> 3;
                const unsigned vo = (unsigned)rr * 2048u + (unsigned)(((lane & 7) ^ rr) * 16);
                const unsigned so = (unsigned)(sel * 4096 + r * 2048 + bi * 64) * 2048u + (unsigned)kk * 128u;
#pragma unroll
                for (int q = 0; q < 8; ++q) c[q] = __builtin_amdgcn_raw_buffer_load_b128(xr, vo + q * 8 * 2048, so, 0);
            } else {
                const int pn = bi >> 2, bj = (bi >> 1) & 1, q = bi & 1;
                const int gk = lane >> 3, cc = lane & 7, wcl = cc >> 2, cq = cc & 3;
                const unsigned vo = (unsigned)(gk * 8) * 16384u + (unsigned)((64 * (2 * q + wcl) + 32 * bj + 8 * cq) * 2);
                const unsigned so = (unsigned)(kk * 64) * 16384u + (unsigned)((sel * 2048 + pn * 256) * 2);
                u32x4 R[8];
#pragma unroll
                for (int jj = 0; jj < 8; ++jj) R[jj] = __builtin_amdgcn_raw_buffer_load_b128(wr, vo + jj * 16384, so, 0);
#pragma unroll
                for (int t = 0; t < 8; ++t) {
                    const unsigned s = (t & 1) ? 0x07060302u : 0x05040100u;
                    u32x4 v;
                    v.x = perm(R[1][t >> 1], R[0][t >> 1], s); v.y = perm(R[3][t >> 1], R[2][t >> 1], s);
                    v.z = perm(R[5][t >> 1], R[4][t >> 1], s); v.w = perm(R[7][t >> 1], R[6][t >> 1], s);
                    const int s64 = 32 * wcl + 16 * (t >> 2) + 4 * cq + (t & 3);
                    *(LAS u32x4*)(my + s64 * 128 + ((gk ^ (s64 & 7)) * 16)) = v;
                }
                asm volatile("s_waitcnt lgkmcnt(0)" ::: "memory");
#pragma unroll
                for (int q2 = 0; q2 < 8; ++q2) c[q2] = *(LAS u32x4*)(my + q2 * 1024 + lane * 16);
                asm volatile("s_waitcnt lgkmcnt(0)" ::: "memory");
            }
            if (call == 0) call = call_draw(ws);
            const int g = P.rank * 16 + kk;
            if (is_w ? (sel == myj) : (sel == myi)) {
                const unsigned so = is_w ? OFF_BIMG + (unsigned)(g * 32 + bi) * 8192u : OFF_AIMG + (unsigned)((r * 128 + g) * 32 + bi) * 8192u;
#pragma unroll
                for (int q = 0; q < 8; ++q) __builtin_amdgcn_raw_buffer_store_b128(c[q], wsr, lane * 16 + q * 1024, so, AUX_SC1);
                selfcnt = OFF_CNT + (r * 128 + g) * 128;
            }
            slot = is_w ? 2048u + (unsigned)((sel * 16 + kk) * 32 + bi) : (unsigned)(((sel * 2 + r) * 16 + kk) * 32 + bi);
            n = is_w ? (unsigned)(kk * 64 + 32 + bi) : (r == 0 ? (unsigned)(kk * 64 + bi) : (unsigned)(1024 + kk * 32 + bi));
            dmask = is_w ? (0x11u << sel) : (0x0Fu << (4 * sel));
        }
        publish_block(P, wsr, c, slot, n, dmask & ~(1u << P.rank), call, lds, my, j, wid, lane);
        signal_cnt(ws, selfcnt, lane);
    }
    if (call == 0) call = call_draw(ws);
    return call;
}

__device__ __forceinline__ u32x2 load_granule(rsrc_t wsr, unsigned off) { return __builtin_amdgcn_raw_buffer_load_b64(wsr, 0, off, AUX_SYS); }

struct Blk { unsigned slot, dso, cnt; bool is_out; };
__device__ __forceinline__ Blk locate(unsigned n, int src, int myi, int myj) {
    Blk b; b.is_out = false;
    if (n < 1024) { const int kk = n >> 6, bi = n & 31, g = src * 16 + kk; b.cnt = OFF_CNT + g * 128;
        if ((n & 63) >= 32) { b.slot = 2048u + (myj * 16 + kk) * 32 + bi; b.dso = OFF_BIMG + (unsigned)(g * 32 + bi) * 8192u; }
        else { b.slot = ((myi * 2) * 16 + kk) * 32 + bi; b.dso = OFF_AIMG + (unsigned)(g * 32 + bi) * 8192u; } }
    else if (n < 1536) { const int kk = (n - 1024) >> 5, bi = n & 31, g = src * 16 + kk; b.cnt = OFF_CNT + (128 + g) * 128;
        b.slot = ((myi * 2 + 1) * 16 + kk) * 32 + bi; b.dso = OFF_AIMG + (unsigned)((128 + g) * 32 + bi) * 8192u; }
    else { const int o = n - 1536, w = o & 7, h = (o >> 3) & 1, tile = (o >> 4) & 63, r = o >> 10; b.cnt = 0; b.is_out = true;
        b.slot = 4096u + o;
        const unsigned row0 = (src >> 2) * 4096 + r * 2048 + (tile >> 3) * 256 + h * 128 + (w >> 2) * 64, col0 = (src & 3) * 2048 + (tile & 7) * 256 + (w & 3) * 64;
        b.dso = (row0 * DIM + col0) * 2u; }
    return b;
}
constexpr unsigned NSLOT = 15, CTL = NSLOT * 8192;
constexpr unsigned M_DEAD = 1u << 30, N_STOP = 0xFFFFFFFFu;
__device__ __forceinline__ unsigned lds_ld(LAS unsigned char* p) { return *(volatile LAS unsigned*)p; }
__device__ __forceinline__ void lds_st(LAS unsigned char* p, unsigned v) { *(volatile LAS unsigned*)p = v; }
__device__ __forceinline__ void ring_publish(LAS unsigned char* lds, unsigned i, unsigned n, unsigned h, int lane) {
    const unsigned sl = i % NSLOT;
    if (lane == 0) { *(volatile LAS u32x2*)(lds + CTL + sl * 8) = (u32x2){n, h}; lds_st(lds + CTL + 256 + sl * 4, i + 1u); }
}
__device__ __forceinline__ void dma_op(rsrc_t r, LAS unsigned char* dst, int j, unsigned so, int lane) {
    __builtin_amdgcn_raw_ptr_buffer_load_lds(r, (LAS void*)(dst + j * 1024), 16, lane * 16 + j * 1024, so, 0, AUX_SYS);
}
#define VM_WAIT_(n) asm volatile("s_waitcnt vmcnt(" #n ")" ::: "memory")
#define VM_WAIT(n) VM_WAIT_(n)
#define WIN 5

__device__ __forceinline__ void loader_wave(const Params& P, LAS unsigned char* lds, unsigned call, int src, int lane) {
    char* ws = P.ws[P.rank];
    unsigned* tmo = (unsigned*)(ws + OFF_TMO);
    const rsrc_t wsr = mk_rsrc(ws, WS_USED);
    char* const remp = P.ws[src];
    const int myi = P.rank >> 2, myj = P.rank & 3;
    const unsigned gbase = OFF_GRAN + (unsigned)src * NSEQ * 8u;
    char* const gptr = ws + gbase;
    const unsigned tko = OFF_LINK + (unsigned)src * 128u, tbase = (call - 1u) * (NSEQ + 3u * PWL);
    __builtin_amdgcn_s_setprio(3);
    unsigned n, n1, n2;
    {
        unsigned t0 = 0;
        if (lane == 0) t0 = __hip_atomic_fetch_add((unsigned*)(ws + tko), 3u, __ATOMIC_RELAXED, __HIP_MEMORY_SCOPE_AGENT);
        n = rfl(t0) - tbase; n1 = n + 1; n2 = n + 2;
    }
    {
        const rsrc_t rw = mk_rsrc(remp, OFF_AIMG);
        const unsigned v = __builtin_amdgcn_raw_buffer_load_b32(rw, lane < 25 ? OFF_SLOT + (unsigned)lane * (2u << 20) + 8188u - (lane == 24 ? 8192u : 0u) : 0xFFFFFFF0u, 0, AUX_SYS);
        asm volatile("" :: "v"(v));
    }
    unsigned gx, gy, hx, hy;
    { const u32x2 a = load_granule(wsr, gbase + (n < NSEQ ? n : 0u) * 8u), b = load_granule(wsr, gbase + (n1 < NSEQ ? n1 : 0u) * 8u);
      gx = rfl(a.x); gy = rfl(a.y); hx = rfl(b.x); hy = rfl(b.y); }
    const int zero = 0, one = 1;
    unsigned i = 0;
    bool pend = false; unsigned pn = 0, ph = 0;
    while (n < NSEQ) {
        const unsigned sl = i % NSLOT;
        bool dead = false;
        const bool need_gran = (int)(gx - call) < 0;
        const bool need_slot = i >= NSLOT && (int)(lds_ld(lds + CTL + 320 + sl * 4) - (i - NSLOT + 1u)) < 0;
        if (need_gran || need_slot) {
            if (pend) { VM_WAIT(0); ring_publish(lds, i - 1u, pn, ph, lane); pend = false; }
            const unsigned long long t0 = __builtin_amdgcn_s_memrealtime();
            while ((int)(gx - call) < 0) {
                const u32x2 a = load_granule(wsr, gbase + n * 8u);
                gx = rfl(a.x); gy = rfl(a.y);
                if ((int)(gx - call) >= 0) break;
                if (give_up(t0, tmo, call)) { dead = true; break; }
                __builtin_amdgcn_s_sleep(2);
                asm volatile("" ::: "memory");
            }
            while (i >= NSLOT && (int)(lds_ld(lds + CTL + 320 + sl * 4) - (i - NSLOT + 1u)) < 0) __builtin_amdgcn_s_sleep(1);
        }
        const bool raw = (gy >> 31) != 0;
        const unsigned so = OFF_SLOT + locate(n, src, myi, myj).slot * 8192u;
                const unsigned bytes = dead ? 0u : raw ? 8192u : RAWB + ((gy & 255u) << 4);
        const rsrc_t rem = mk_rsrc(remp, so + bytes);
        LAS unsigned char* dst = lds + sl * 8192;
        u32x2 g2; unsigned tk = 0;
        asm volatile("global_load_dwordx2 %0, %1, %2 sc0 sc1" : "=v"(g2) : "v"(zero), "s"(gptr + (n2 < NSEQ ? n2 : 0u) * 8u) : "memory");
        if (lane == 0) asm volatile("global_atomic_add %0, %1, %2, %3 sc0" : "=v"(tk) : "v"(zero), "v"(one), "s"(ws + tko) : "memory");
        VM_WAIT(WIN); dma_op(rem, dst, 0, so, lane);
        VM_WAIT(WIN); dma_op(rem, dst, 1, so, lane);
        VM_WAIT(WIN); dma_op(rem, dst, 2, so, lane);
        VM_WAIT(WIN); if (pend) ring_publish(lds, i - 1u, pn, ph, lane); dma_op(rem, dst, 3, so, lane);
        VM_WAIT(WIN); dma_op(rem, dst, 4, so, lane);
        asm volatile("s_waitcnt vmcnt(5)" : "+v"(g2), "+v"(tk) :: "memory"); dma_op(rem, dst, 5, so, lane);
        if (raw) { dma_op(rem, dst, 6, so, lane); dma_op(rem, dst, 7, so, lane); VM_WAIT(0); }
        pend = true; pn = n; ph = gy | (dead ? M_DEAD : 0u);
        n = n1; gx = hx; gy = hy;
        n1 = n2; hx = rfl(g2.x); hy = rfl(g2.y);
        n2 = rfl(tk) - tbase;
        ++i;
    }
    VM_WAIT(0);
    if (pend) ring_publish(lds, i - 1u, pn, ph, lane);
    for (int d = 0; d < NDEC; ++d, ++i) {
        const unsigned sl = i % NSLOT;
        while (i >= NSLOT && (int)(lds_ld(lds + CTL + 320 + sl * 4) - (i - NSLOT + 1u)) < 0) __builtin_amdgcn_s_sleep(1);
        ring_publish(lds, i, N_STOP, 0u, lane);
    }
}

__device__ __forceinline__ void decoder_wave(const Params& P, LAS unsigned char* lds, unsigned call, int src, int d, int lane) {
    char* ws = P.ws[P.rank];
    unsigned* tmo = (unsigned*)(ws + OFF_TMO);
    const rsrc_t wsr = mk_rsrc(ws, WS_USED), outr = mk_rsrc(P.out, (unsigned)DIM * DIM * 2u);
    const int myi = P.rank >> 2, myj = P.rank & 3;
    for (unsigned i = d;; i += NDEC) {
        const unsigned sl = i % NSLOT;
        while (lds_ld(lds + CTL + 256 + sl * 4) != i + 1u) __builtin_amdgcn_s_sleep(1);
        asm volatile("" ::: "memory");
        const u32x2 meta = *(volatile LAS u32x2*)(lds + CTL + sl * 8);
        const unsigned n = rfl(meta.x), h = rfl(meta.y);
        if (n == N_STOP) break;
        const Blk b = locate(n, src, myi, myj);
        const bool raw = (h >> 31) != 0;
        bool dead = (h & M_DEAD) != 0;
        if (b.is_out && rfl(__hip_atomic_load(tmo, __ATOMIC_RELAXED, __HIP_MEMORY_SCOPE_AGENT)) == call) dead = true;
        u32x4 c[8];
        unsigned recs = 0;
        const unsigned ne = (dead || raw) ? 0u : (h >> 8) & 63u;
        LAS unsigned char* cur = lds + sl * 8192;
        if (dead) {
#pragma unroll
            for (int j = 0; j < 8; ++j) c[j] = (u32x4){0x7FC07FC0u, 0x7FC07FC0u, 0x7FC07FC0u, 0x7FC07FC0u};
        } else if (raw) {
#pragma unroll
            for (int j = 0; j < 8; ++j) c[j] = *(LAS u32x4*)(cur + j * 1024 + lane * 16);
        } else {
            recs = *(LAS unsigned*)(cur + RAWB + (((h >> 16) & 0x3FFu) + (lane & 31)) * 4u);
            decode_block(cur, c, lane);
        }
#pragma unroll
        for (int j = 0; j < 8; ++j) asm volatile("" : "+v"(c[j]));
        asm volatile("" : "+v"(recs));
        if (lane == 0) lds_st(lds + CTL + 320 + sl * 4, i + 1u);
        const rsrc_t dr = b.is_out ? outr : wsr;
        const unsigned sj = b.is_out ? 8u * DIM * 2u : 1024u, sr = b.is_out ? DIM * 2u : 128u;
        const unsigned vo = (unsigned)(lane >> 3) * sr + (unsigned)(lane & 7) * 16u;
#pragma unroll
        for (int j = 0; j < 8; ++j) __builtin_amdgcn_raw_buffer_store_b128(c[j], dr, vo, b.dso + j * sj, AUX_SC1);
        for (unsigned e = 0; e < ne; ++e) {
            const unsigned rec = __builtin_amdgcn_readlane(recs, e), pos = rec & 0xFFFu, v = pos & 63u;
            if ((unsigned)lane == (pos >> 6)) __builtin_amdgcn_raw_buffer_store_b16((unsigned short)((rec >> 12) & 0xFFFFu), dr, vo + (v & 7u) * 2u, b.dso + (v >> 3) * sj, AUX_SC1);
        }
        if (b.cnt) { WAIT_VM0(); signal_cnt(ws, b.cnt, lane); }
    }
}

__device__ __forceinline__ void puller_role(const Params& P, LAS unsigned char* lds, unsigned call, int wg) {
    const int lane = threadIdx.x & 63, wid = rfl(threadIdx.x >> 6);
    const int link = wg % 7, src = rfl((P.rank + 1 + link) & 7);
    if (threadIdx.x < 32) *(volatile LAS unsigned*)(lds + CTL + 256 + threadIdx.x * 4) = 0u;
    __syncthreads();
    if (wid == 0) loader_wave(P, lds, call, src, lane);
    else if (wid <= NDEC) decoder_wave(P, lds, call, src, wid - 1, lane);
}

__device__ __forceinline__ float gelu_f(float y) {
    const float c1 = -2.0f * 0.7978845608028654f * 1.4426950408889634f, c2 = c1 * 0.044715f;
    const float e = __builtin_amdgcn_exp2f(y * (c1 + c2 * y * y));
    return y * __builtin_amdgcn_rcpf(1.0f + e);
}
__device__ __forceinline__ unsigned cvt_pk_bf16(float lo, float hi) { unsigned r; asm("v_cvt_pk_bf16_f32 %0, %1, %2" : "=v"(r) : "v"(lo), "v"(hi)); return r; }

__device__ __forceinline__ void gemm_role(const Params& P, LAS unsigned char* lds, unsigned call, int wg) {
    const int tid = threadIdx.x, wid = rfl(tid >> 6), lane = tid & 63, wr = wid >> 2, wc = wid & 3, fr = lane & 15, fq = lane >> 4;
    char* ws = P.ws[P.rank];
    unsigned* tmo = (unsigned*)(ws + OFF_TMO);
    const rsrc_t wsr = mk_rsrc(ws, WS_USED);
    const int xcd = wg & 7, q = (wg - NPACK) >> 3, h = xcd & 1, blk = xcd >> 1;
    const int tm = (blk >> 1) * 4 + (q >> 2), tn = (blk & 1) * 4 + (q & 3), tile = tm * 8 + tn;
    const unsigned voff = (unsigned)tid * 16u, ldsw = (unsigned)wid * 1024u;
    const int aoff0 = (wr * 64 + fr) * 128 + ((fq ^ (fr & 7)) << 4), aoff1 = aoff0 ^ 64;
    const int boff0 = (wc * 32 + fr) * 128 + ((fq ^ (fr & 7)) << 4), boff1 = boff0 ^ 64;
#define G_SA(b, hh) (((b) * 2 + (hh)) * 16384)
#define G_SB(b, hh) ((4 + (b) * 2 + (hh)) * 16384)
#define G_STAGE(bufoff, soff) do { _Pragma("unroll") for (int _i = 0; _i < 2; ++_i) \
        __builtin_amdgcn_raw_ptr_buffer_load_lds(wsr, (LAS void*)(lds + (bufoff) + ldsw + _i * 8192), 16, voff, (soff) + _i * 8192, 0, AUX_SYS); } while (0)
#define G_LDA(dst, b, hh) do { _Pragma("unroll") for (int m = 0; m < 4; ++m) { \
        dst[m][0] = *(const LAS bf16x8*)(lds + G_SA(b, hh) + aoff0 + m * 2048); dst[m][1] = *(const LAS bf16x8*)(lds + G_SA(b, hh) + aoff1 + m * 2048); } } while (0)
#define G_LDB(dst, b, hh) do { _Pragma("unroll") for (int n = 0; n < 2; ++n) { \
        dst[n][0] = *(const LAS bf16x8*)(lds + G_SB(b, hh) + boff0 + n * 2048); dst[n][1] = *(const LAS bf16x8*)(lds + G_SB(b, hh) + boff1 + n * 2048); } } while (0)
#define G_MMA(ai, bj, At, Bt) do { __builtin_amdgcn_s_setprio(1); _Pragma("unroll") for (int m = 0; m < 4; ++m) _Pragma("unroll") for (int n = 0; n < 2; ++n) _Pragma("unroll") for (int k = 0; k < 2; ++k) \
        acc[ai][bj][m][n] = __builtin_amdgcn_mfma_f32_16x16x32_bf16(Bt[n][k], At[m][k], acc[ai][bj][m][n], 0, 0, 0); __builtin_amdgcn_s_setprio(0); } while (0)
#define G_WAIT_V(n) asm volatile("s_waitcnt vmcnt(" #n ")" ::: "memory")
#define G_WAIT_L(n) asm volatile("s_waitcnt lgkmcnt(" #n ")" ::: "memory")
#define G_BAR __builtin_amdgcn_s_barrier()
#define G_SCHED __builtin_amdgcn_sched_barrier(0)
#define G_GIDX(t) ((unsigned)((4 * h + ((t) & 3)) * 16 + ((t) >> 2)))
    const int zero = 0;
    for (int r = 0; r < 2; ++r) {
        const unsigned target = call * (r == 0 ? 64u : 32u);
        const unsigned abase = OFF_AIMG + (unsigned)r * (128u * 262144u) + (unsigned)tm * 32768u, bbase = OFF_BIMG + (unsigned)tn * 32768u;
        char* cbase = ws + OFF_CNT + r * 128 * 128;
        f32x4 acc[2][2][4][2];
#pragma unroll
        for (int a = 0; a < 2; ++a)
#pragma unroll
            for (int b = 0; b < 2; ++b)
#pragma unroll
                for (int m = 0; m < 4; ++m)
#pragma unroll
                    for (int n = 0; n < 2; ++n) acc[a][b][m][n] = (f32x4){0.f, 0.f, 0.f, 0.f};
        bf16x8 At[4][2], B0[2][2], B1[2][2];
        STAMP(ws, r * 8 + 0);
        wait_word((unsigned*)(cbase + G_GIDX(0) * 128), target, tmo, call);
        wait_word((unsigned*)(cbase + G_GIDX(1) * 128), target, tmo, call);
        wait_word((unsigned*)(cbase + G_GIDX(2) * 128), target, tmo, call);
        STAMP(ws, r * 8 + 1);
        {
            const unsigned a0 = abase + G_GIDX(0) * 262144u, b0 = bbase + G_GIDX(0) * 262144u, a1 = abase + G_GIDX(1) * 262144u, b1 = bbase + G_GIDX(1) * 262144u;
            G_STAGE(G_SB(0, 0), b0); G_STAGE(G_SA(0, 0), a0); G_STAGE(G_SB(0, 1), b0 + 16384); G_STAGE(G_SA(0, 1), a0 + 16384);
            if (wr == 1) G_BAR;
            G_WAIT_V(4); G_BAR;
            G_STAGE(G_SB(1, 0), b1); G_STAGE(G_SA(1, 0), a1); G_STAGE(G_SB(1, 1), b1 + 16384);
            G_WAIT_V(6); G_BAR;
        }
        unsigned fA = target, fB;
        for (int t = 0; t < 64; t += 2) {
            const int t2 = t + 2 < 64 ? t + 2 : 63, t3 = t + 3 < 64 ? t + 3 : 63, t4 = t + 4 < 64 ? t + 4 : 63;
            const unsigned g1 = G_GIDX(t + 1), g2 = G_GIDX(t2), g3 = G_GIDX(t3), g4 = G_GIDX(t4);
            const unsigned a1 = abase + g1 * 262144u, a2 = abase + g2 * 262144u, b2 = bbase + g2 * 262144u, a3 = abase + g3 * 262144u, b3 = bbase + g3 * 262144u;
            asm volatile("global_load_dword %0, %1, %2 sc0 sc1" : "=v"(fB) : "v"(zero), "s"(cbase + g3 * 128));
            G_LDB(B0, 0, 0); G_SCHED; G_LDA(At, 0, 0); G_STAGE(G_SA(1, 1), a1 + 16384);
            G_WAIT_L(8); G_BAR; G_WAIT_L(0); G_MMA(0, 0, At, B0); G_BAR; G_SCHED;
            asm volatile("" : "+v"(fA));
            if ((int)(rfl(fA) - target) < 0) wait_word((unsigned*)(cbase + g2 * 128), target, tmo, call);
            G_LDB(B1, 0, 1); G_STAGE(G_SB(0, 0), b2);
            G_BAR; G_WAIT_L(0); G_MMA(0, 1, At, B1); G_BAR;
            G_LDA(At, 0, 1); G_STAGE(G_SA(0, 0), a2);
            G_BAR; G_WAIT_L(0); G_MMA(1, 0, At, B0); G_BAR; G_SCHED;
            G_STAGE(G_SB(0, 1), b2 + 16384);
            G_WAIT_V(6); G_BAR; G_MMA(1, 1, At, B1); G_BAR;
            asm volatile("global_load_dword %0, %1, %2 sc0 sc1" : "=v"(fA) : "v"(zero), "s"(cbase + g4 * 128));
            G_LDB(B0, 1, 0); G_SCHED; G_LDA(At, 1, 0); G_STAGE(G_SA(0, 1), a2 + 16384);
            G_WAIT_L(8); G_BAR; G_WAIT_L(0); G_MMA(0, 0, At, B0); G_BAR; G_SCHED;
            asm volatile("" : "+v"(fB));
            if ((int)(rfl(fB) - target) < 0) wait_word((unsigned*)(cbase + g3 * 128), target, tmo, call);
            G_LDB(B1, 1, 1); G_STAGE(G_SB(1, 0), b3);
            G_BAR; G_WAIT_L(0); G_MMA(0, 1, At, B1); G_BAR;
            G_LDA(At, 1, 1); G_STAGE(G_SA(1, 0), a3);
            G_BAR; G_WAIT_L(0); G_MMA(1, 0, At, B0); G_BAR; G_SCHED;
            G_STAGE(G_SB(1, 1), b3 + 16384);
            G_WAIT_V(6); G_BAR; G_MMA(1, 1, At, B1); G_BAR;
        }
        G_WAIT_V(0);
        asm volatile("" : "+v"(fA));
        if (wr == 0) G_BAR;
        G_BAR;
        STAMP(ws, r * 8 + 2);
        int lane = threadIdx.x & 63;
        asm volatile("" : "+v"(lane));
        const int fr = lane & 15, fq = lane >> 4;
        if (h) {
#pragma unroll
            for (int b = 0; b < 2; ++b)
#pragma unroll
                for (int m = 0; m < 4; ++m)
#pragma unroll
                    for (int n = 0; n < 2; ++n) { const f32x4 tq = acc[0][b][m][n]; acc[0][b][m][n] = acc[1][b][m][n]; acc[1][b][m][n] = tq; }
        }
        const unsigned xl = (unsigned)((r * 64 + tile) * 2);
#pragma unroll
        for (int b = 0; b < 2; ++b)
#pragma unroll
            for (int m = 0; m < 4; ++m)
#pragma unroll
                for (int n = 0; n < 2; ++n)
                    __builtin_amdgcn_raw_buffer_store_b128(__builtin_bit_cast(u32x4, acc[1][b][m][n]), wsr, lane * 16 + ((b * 4 + m) * 2 + n) * 1024, OFF_XBUF + ((xl + h) * 8 + wid) * 16384u, AUX_SC1);
        WAIT_VM0();
        if (lane == 0) __hip_atomic_store((unsigned*)(ws + OFF_XFLAG + (xl + h) * 128 + wid * 16), call, __ATOMIC_RELAXED, __HIP_MEMORY_SCOPE_AGENT);
        wait_word((unsigned*)(ws + OFF_XFLAG + (xl + (h ^ 1)) * 128 + wid * 16), call, tmo, call);
        const bool dead = rfl(__hip_atomic_load(tmo, __ATOMIC_RELAXED, __HIP_MEMORY_SCOPE_AGENT)) == call;
        STAMP(ws, r * 8 + 3);
        LAS unsigned char* my = lds + wid * 8192;
#pragma unroll
        for (int b = 0; b < 2; ++b)
#pragma unroll
            for (int m = 0; m < 4; ++m) {
                const f32x4 p0 = __builtin_bit_cast(f32x4, __builtin_amdgcn_raw_buffer_load_b128(wsr, lane * 16 + ((b * 4 + m) * 2 + 0) * 1024, OFF_XBUF + ((xl + (h ^ 1)) * 8 + wid) * 16384u, AUX_SC1));
                const f32x4 p1 = __builtin_bit_cast(f32x4, __builtin_amdgcn_raw_buffer_load_b128(wsr, lane * 16 + ((b * 4 + m) * 2 + 1) * 1024, OFF_XBUF + ((xl + (h ^ 1)) * 8 + wid) * 16384u, AUX_SC1));
                const f32x4 v0 = acc[0][b][m][0] + p0, v1 = acc[0][b][m][1] + p1;
                u32x4 o;
                o.x = cvt_pk_bf16(gelu_f(v0[0]), gelu_f(v0[1])); o.y = cvt_pk_bf16(gelu_f(v0[2]), gelu_f(v0[3]));
                o.z = cvt_pk_bf16(gelu_f(v1[0]), gelu_f(v1[1])); o.w = cvt_pk_bf16(gelu_f(v1[2]), gelu_f(v1[3]));
                const int row = m * 16 + fr, ch = b * 4 + fq;
                *(LAS u32x4*)(my + row * 128 + ((ch ^ (row & 7)) * 16)) = o;
            }
        asm volatile("s_waitcnt lgkmcnt(0)" ::: "memory");
        u32x4 c[8];
#pragma unroll
        for (int j = 0; j < 8; ++j) { const int row = j * 8 + (lane >> 3); c[j] = *(LAS u32x4*)(my + row * 128 + (((lane & 7) ^ (row & 7)) * 16)); }
        if (dead) {
#pragma unroll
            for (int j = 0; j < 8; ++j) c[j] = (u32x4){0x7FC07FC0u, 0x7FC07FC0u, 0x7FC07FC0u, 0x7FC07FC0u};
        }
        {
            const rsrc_t outr = mk_rsrc(P.out, (unsigned)DIM * DIM * 2u);
            const unsigned row0 = (P.rank >> 2) * 4096 + r * 2048 + tm * 256 + h * 128 + wr * 64, col0 = (P.rank & 3) * 2048 + tn * 256 + wc * 64;
            const unsigned dso = (row0 * DIM + col0) * 2u, vo = (unsigned)(lane >> 3) * (DIM * 2u) + (unsigned)(lane & 7) * 16u;
#pragma unroll
            for (int j = 0; j < 8; ++j) __builtin_amdgcn_raw_buffer_store_b128(c[j], outr, vo, dso + j * (8u * DIM * 2u), AUX_SC1);
            const unsigned o = (xl + h) * 8 + wid;
            if (r == 0) { WAIT_VM0(); signal_cnt(ws, OFF_OCNT + (o >> 3) * 128, lane); }
            else publish_block(P, wsr, c, 4096u + o, 1536u + o, 0xFFu & ~(1u << P.rank), call, lds, my, 1u + GROUNDS, wid, lane);
        }
        STAMP(ws, r * 8 + 4);
        asm volatile("s_waitcnt lgkmcnt(0)" ::: "memory");
        G_BAR;
    }
#undef G_SA
#undef G_SB
#undef G_STAGE
#undef G_LDA
#undef G_LDB
#undef G_MMA
#undef G_WAIT_V
#undef G_WAIT_L
#undef G_BAR
#undef G_SCHED
#undef G_GIDX
}

__global__ __launch_bounds__(512, 2) void gemm_allgather_kernel(Params P) {
    extern __shared__ __attribute__((aligned(16))) unsigned char shm[];
    LAS unsigned char* lds = (LAS unsigned char*)shm;
    char* ws = P.ws[P.rank];
    const int b = rfl(blockIdx.x);
    STAMP(ws, 60);
    unsigned call;
    if (b >= NPACK + NGEMM) { call = call_draw(ws); puller_role(P, lds, rfl(call), rfl(b - NPACK - NGEMM)); }
    else {
        if (threadIdx.x < 8) *(volatile LAS unsigned*)(lds + GCTL + 128 + threadIdx.x * 4) = 0u;
        __syncthreads();
        const int wid = rfl(threadIdx.x >> 6);
        const bool gemm = b >= NPACK;
        const int next = gemm ? NFIRST + (b - NPACK) * 8 + wid : NFIRST + GROUNDS * NGEMM * 8 + b * 8 + wid;
        call = packer_role(P, lds, wid < 4 ? b * 4 + wid : next, next, gemm ? NGEMM * 8 : NPACK * 8, gemm ? NFIRST + GROUNDS * NGEMM * 8 : 4096 + 1024);
        if (gemm) { __syncthreads(); gemm_role(P, lds, rfl(call), rfl(b)); }
    }
    STAMP(ws, 61);
    STAMP(ws, 58 + (call & 1));
}

}

extern "C" void kernel_launch(int rank, int nranks, ncclComm_t comm,
                              hipStream_t stream,
                              void* const* d_in, const int* in_sizes, int n_in,
                              void* d_out, int out_size,
                              const int* in_dtypes, int out_dtype,
                              void* const* d_out_all,
                              void* const* d_ws_all, size_t ws_bytes) {
    (void)comm; (void)in_sizes; (void)in_dtypes; (void)out_dtype; (void)d_out_all;
    if (nranks != gar::WORLD || n_in < 2 || ws_bytes < gar::WS_NEED || out_size != gar::DIM * gar::DIM || (unsigned)rank >= (unsigned)gar::WORLD) {
        fprintf(stderr, "unexpected problem shape\n");
        return;
    }
    static bool attr = false;
    if (!attr) {
        hipError_t e = hipFuncSetAttribute((const void*)gar::gemm_allgather_kernel, hipFuncAttributeMaxDynamicSharedMemorySize, gar::LDS_BYTES);
        if (e != hipSuccess) { fprintf(stderr, "hipFuncSetAttribute: %s\n", hipGetErrorString(e)); return; }
        attr = true;
    }
    gar::Params P;
    memset(&P, 0, sizeof P);
    P.x = d_in[0]; P.w = d_in[1]; P.out = d_out; P.rank = rank;
    for (int p = 0; p < gar::WORLD; ++p) P.ws[p] = (char*)d_ws_all[p];
    hipLaunchKernelGGL(gar::gemm_allgather_kernel, dim3(gar::GRID), dim3(512), gar::LDS_BYTES, stream, P);
    hipError_t e = hipGetLastError();
    if (e != hipSuccess) fprintf(stderr, "launch: %s\n", hipGetErrorString(e));
}
```

```cpp
#include <hip/hip_runtime.h>
#include <rccl/rccl.h>
#include <cstdio>
#include <cstdint>
#include <cstring>

namespace gar {

#define LAS __attribute__((address_space(3)))
typedef unsigned u32x4 __attribute__((ext_vector_type(4)));
typedef unsigned short u16x2 __attribute__((ext_vector_type(2)));
typedef short bf16x8 __attribute__((ext_vector_type(8)));
typedef float f32x4 __attribute__((ext_vector_type(4)));
typedef __amdgpu_buffer_rsrc_t rsrc_t;

constexpr int WORLD = 8;
constexpr int DIM = 8192;
constexpr int KSH = 1024;
constexpr int NGEMM = 128;
constexpr int GRID = 256;
#define PULL_WG_PER_LINK 12
#define PULL_DECODERS 7
constexpr int PWL = PULL_WG_PER_LINK;
constexpr int NDEC = PULL_DECODERS;
constexpr int NPULL = 7 * PWL;
constexpr int NPACK = GRID - NGEMM - NPULL;
#define NRAWEND 24
#define WARM_TLB 1
#define OUT_AUX 18
#define IMG_AUX 18
#define STG_AUX AUX_SYS
#define XIN_AUX 0
#define JIT_LEAD 30
#define PAY_AUX AUX_SYS
#define NRAW 48
constexpr int NFIRST = (NPACK + NGEMM) * 4;
#define GEMM_PACK_ROUNDS 1
constexpr int GROUNDS = GEMM_PACK_ROUNDS;
static_assert(NFIRST + GROUNDS * NGEMM * 8 <= 4096 && NFIRST % 8 == 0, "roles");
constexpr int NSEQ = 3584;
constexpr unsigned long long DEADLINE = 40000000ull;

constexpr unsigned OFF_TMO = 128;
constexpr unsigned OFF_LINK = 1024;
constexpr unsigned OFF_CNT = 4096;
constexpr unsigned OFF_XFLAG = OFF_CNT + 256 * 128;
constexpr unsigned OFF_OCNT = OFF_XFLAG + 256 * 128;
constexpr unsigned OFF_GRAN = 131072;
constexpr unsigned OFF_CALL = 640u << 10;
constexpr unsigned OFF_SLOT = 1u << 20;
constexpr unsigned OFF_AIMG = 64u << 20;
constexpr unsigned OFF_BIMG = 128u << 20;
constexpr unsigned OFF_XBUF = 160u << 20;
constexpr unsigned WS_USED = 192u << 20;
constexpr unsigned WS_NEED = WS_USED;
static_assert(OFF_OCNT + 128 * 128 <= OFF_GRAN, "map");
static_assert(OFF_GRAN + WORLD * NSEQ * 8 <= OFF_CALL && OFF_CALL + 2048 * 64 <= OFF_SLOT, "map");
static_assert(OFF_SLOT + 6144u * 8192u <= OFF_AIMG, "map");
constexpr int LDS_BYTES = 131072 + 256;

struct Params {
    const void* x; const void* w; void* out;
    char* ws[WORLD];
    int rank; int pad;
};

constexpr int AUX_SC1 = 16, AUX_SYS = 17;

__device__ __forceinline__ rsrc_t mk_rsrc(const void* p, unsigned bytes) {
    return __builtin_amdgcn_make_buffer_rsrc(const_cast<void*>(p), 0, bytes, 0x00020000);
}
__device__ __forceinline__ unsigned rfl(unsigned v) { return __builtin_amdgcn_readfirstlane(v); }
__device__ __forceinline__ unsigned bfi(unsigned mask, unsigned a, unsigned b) { return (a & mask) | (b & ~mask); }
__device__ __forceinline__ unsigned perm(unsigned hi, unsigned lo, unsigned sel) { return __builtin_amdgcn_perm(hi, lo, sel); }
#define WAIT_VM0() asm volatile("s_waitcnt vmcnt(0)" ::: "memory")

#define STAMP(ws, idx) do { } while (0)

__device__ __forceinline__ unsigned call_draw(char* ws) {
    unsigned t;
    const int zero = 0, one = 1;
    unsigned long long keep;
    asm volatile("s_mov_b64 %1, exec\n\ts_mov_b64 exec, 1\n\tglobal_atomic_add %0, %2, %3, %4 sc0\n\ts_mov_b64 exec, %1\n\ts_waitcnt vmcnt(0)"
                 : "=&v"(t), "=&s"(keep) : "v"(zero), "v"(one), "s"(ws + OFF_CALL + (blockIdx.x * 8 + rfl(threadIdx.x >> 6)) * 64) : "memory");
    return rfl(t) + 1u;
}

__device__ __forceinline__ bool give_up(unsigned long long t0, unsigned* tmo, unsigned call) {
    if (rfl(__hip_atomic_load(tmo, __ATOMIC_RELAXED, __HIP_MEMORY_SCOPE_AGENT)) == call) return true;
    if (__builtin_amdgcn_s_memrealtime() - t0 > DEADLINE) { __hip_atomic_store(tmo, call, __ATOMIC_RELAXED, __HIP_MEMORY_SCOPE_AGENT); return true; }
    return false;
}
__device__ __forceinline__ void wait_word(unsigned* p, unsigned target, unsigned* tmo, unsigned call) {
    if ((int)(rfl(__hip_atomic_load(p, __ATOMIC_RELAXED, __HIP_MEMORY_SCOPE_SYSTEM)) - target) >= 0) return;
    const unsigned long long t0 = __builtin_amdgcn_s_memrealtime();
    while ((int)(rfl(__hip_atomic_load(p, __ATOMIC_RELAXED, __HIP_MEMORY_SCOPE_SYSTEM)) - target) < 0) {
        if (give_up(t0, tmo, call)) break;
        __builtin_amdgcn_s_sleep(4);
    }
}

constexpr unsigned MAX_EXC = 31;
constexpr unsigned RAWB = 3584, AUX_U = 20, AUX_MAX = 2560, U_MAX = 10240, LANE_U_MAX = 224;
constexpr unsigned ENC_HIST = AUX_MAX + 16, ENC_SORT = ENC_HIST + 256, ENC_END = ENC_SORT + 128;
constexpr unsigned DEC_POS = 6144;

__device__ __forceinline__ unsigned bfi_c(unsigned mask, unsigned a, unsigned b) { unsigned r; asm("v_bfi_b32 %0, %1, %2, %3" : "=v"(r) : "s"(mask), "v"(a), "v"(b)); return r; }
__device__ __forceinline__ unsigned opaque(unsigned v) { asm("" : "+v"(v)); return v; }
__device__ __forceinline__ unsigned bytemask(unsigned x, int sh, unsigned c0c) {
    unsigned t; asm("v_and_or_b32 %0, %1, %2, %3" : "=v"(t) : "v"(x >> sh), "s"(0x01010101u), "v"(c0c));
    return perm(0u, 0u, t);
}
template <int CTRL, int RM = 0xf> __device__ __forceinline__ unsigned dpp0(unsigned v) { return (unsigned)__builtin_amdgcn_update_dpp(0, (int)v, CTRL, RM, 0xf, RM == 0xf); }
__device__ __forceinline__ unsigned wave_scan_incl(unsigned v) {
    v += dpp0<0x111>(v); v += dpp0<0x112>(v); v += dpp0<0x114>(v); v += dpp0<0x118>(v);
    v += dpp0<0x142, 0xa>(v); v += dpp0<0x143, 0xc>(v);
    return v;
}
__device__ __forceinline__ unsigned umin(unsigned a, unsigned b) { return a < b ? a : b; }
__device__ __forceinline__ unsigned umax(unsigned a, unsigned b) { return a > b ? a : b; }
__device__ __forceinline__ unsigned sad4(unsigned x) { return __builtin_amdgcn_sad_u8(x, 0u, 0u); }
__device__ __forceinline__ unsigned lut2x16(const unsigned (&T)[8], unsigned idx, unsigned ms, unsigned c0c) {
    const unsigned sel = idx & 0x07070707u, m8 = bytemask(idx, 3, c0c);
    const unsigned p = bfi(m8, perm(T[3], T[2], sel), perm(T[1], T[0], sel)), n = bfi(m8, perm(T[7], T[6], sel), perm(T[5], T[4], sel));
    return bfi(ms, n, p);
}
#define LDS_FENCE() asm volatile("" ::: "memory")
__device__ __forceinline__ void lds_or2(LAS unsigned char* scr, unsigned p, unsigned w) {
    const unsigned s = p & 31u;
    LAS unsigned* q = (LAS unsigned*)(scr + ((p >> 5) << 2));
    __hip_atomic_fetch_or(q, w << s, __ATOMIC_RELAXED, __HIP_MEMORY_SCOPE_WORKGROUP);
    __hip_atomic_fetch_or(q + 1, (w >> 1) >> (31u - s), __ATOMIC_RELAXED, __HIP_MEMORY_SCOPE_WORKGROUP);
}

__device__ __forceinline__ unsigned encode_block(const u32x4 (&c)[8], unsigned (&P)[14], u32x4 (&a)[3], LAS unsigned char* scr, int lane) {
    asm volatile("" : "+v"(lane));
    const unsigned c0c = opaque(0x0C0C0C0Cu);
    unsigned E[16], R[16];
    u16x2 p1 = {0, 0}, p2 = {0, 0}, n1 = {0, 0}, n2 = {0, 0};
#pragma unroll
    for (int g = 0; g < 16; ++g) {
        const unsigned x = (g & 1) ? c[g >> 1].z : c[g >> 1].x, y = (g & 1) ? c[g >> 1].w : c[g >> 1].y;
        const unsigned H = perm(y, x, 0x07050301u), L = perm(y, x, 0x06040200u);
        E[g] = bfi_c(0xFEFEFEFEu, H << 1, L >> 7);
        R[g] = bfi_c(0x80808080u, L << 1, bfi_c(0x40404040u, H >> 1, L));
        const unsigned en = E[g] & bytemask(H, 7, c0c), ep = E[g] ^ en;
        p1 = __builtin_elementwise_max(p1, __builtin_bit_cast(u16x2, ep)); p2 = __builtin_elementwise_max(p2, __builtin_bit_cast(u16x2, ep << 8));
        n1 = __builtin_elementwise_max(n1, __builtin_bit_cast(u16x2, en)); n2 = __builtin_elementwise_max(n2, __builtin_bit_cast(u16x2, en << 8));
        __builtin_amdgcn_sched_barrier(0);
    }
    const u32x4 z4 = {0u, 0u, 0u, 0u};
    *(LAS u32x4*)(scr + lane * 16) = z4; *(LAS u32x4*)(scr + 1024 + lane * 16) = z4;
    if (lane < 49) *(LAS u32x4*)(scr + 2048 + lane * 16) = z4;
    unsigned baseP, baseN;
    {
        const u16x2 mp = __builtin_elementwise_max(p1, p2), mn = __builtin_elementwise_max(n1, n2);
        u16x2 m = {(unsigned short)((mp.x > mp.y ? mp.x : mp.y) >> 8), (unsigned short)((mn.x > mn.y ? mn.x : mn.y) >> 8)};
        m = __builtin_elementwise_max(m, __builtin_bit_cast(u16x2, dpp0<0x128>(__builtin_bit_cast(unsigned, m)))); m = __builtin_elementwise_max(m, __builtin_bit_cast(u16x2, dpp0<0x124>(__builtin_bit_cast(unsigned, m))));
        m = __builtin_elementwise_max(m, __builtin_bit_cast(u16x2, dpp0<0x122>(__builtin_bit_cast(unsigned, m)))); m = __builtin_elementwise_max(m, __builtin_bit_cast(u16x2, dpp0<0x121>(__builtin_bit_cast(unsigned, m))));
        const unsigned mu = __builtin_bit_cast(unsigned, m);
        const u16x2 r01 = __builtin_elementwise_max(__builtin_bit_cast(u16x2, (unsigned)__builtin_amdgcn_readlane(mu, 0)), __builtin_bit_cast(u16x2, (unsigned)__builtin_amdgcn_readlane(mu, 16)));
        const u16x2 r23 = __builtin_elementwise_max(__builtin_bit_cast(u16x2, (unsigned)__builtin_amdgcn_readlane(mu, 32)), __builtin_bit_cast(u16x2, (unsigned)__builtin_amdgcn_readlane(mu, 48)));
        const unsigned mm = rfl(__builtin_bit_cast(unsigned, __builtin_elementwise_max(r01, r23)));
        baseP = mm & 0xFFu; baseN = mm >> 16;
    }
    const unsigned BP = baseP * 0x01010101u, BN = baseN * 0x01010101u;
    unsigned any = 0, D[16];
#pragma unroll
    for (int g = 0; g < 16; ++g) {
        E[g] = bfi(bytemask(R[g], 6, c0c), BN, BP) - E[g]; any |= E[g];
        D[g] = bfi_c(0x1E1E1E1Eu, E[g] << 1, (R[g] >> 7) & 0x01010101u);
        R[g] &= 0x7F7F7F7Fu;
    }
    LDS_FENCE();
#pragma unroll
    for (int gi = 0; gi < 4; ++gi) {
        const int g = gi * 5;
        const unsigned k4 = ((D[g] & 0x0F0F0F0Fu) | ((R[g] >> 2) & 0x10101010u) | ((D[g] << 1) & 0x20202020u)) << 2;
#pragma unroll
        for (int b = 0; b < 4; ++b)
            __hip_atomic_fetch_add((LAS unsigned*)(scr + ENC_HIST + ((k4 >> (8 * b)) & 0xFFu)), 1u, __ATOMIC_RELAXED, __HIP_MEMORY_SCOPE_WORKGROUP);
    }
    LDS_FENCE();
    const unsigned cnt = *(LAS unsigned*)(scr + ENC_HIST + lane * 4);
    const unsigned key = (cnt << 4) | (15u - (lane & 15));
    unsigned rank = 0;
    rank += dpp0<0x121>(key) > key; rank += dpp0<0x122>(key) > key; rank += dpp0<0x123>(key) > key; rank += dpp0<0x124>(key) > key; rank += dpp0<0x125>(key) > key;
    rank += dpp0<0x126>(key) > key; rank += dpp0<0x127>(key) > key; rank += dpp0<0x128>(key) > key; rank += dpp0<0x129>(key) > key; rank += dpp0<0x12A>(key) > key;
    rank += dpp0<0x12B>(key) > key; rank += dpp0<0x12C>(key) > key; rank += dpp0<0x12D>(key) > key; rank += dpp0<0x12E>(key) > key; rank += dpp0<0x12F>(key) > key;
    unsigned tail = cnt; tail += dpp0<0x128>(tail); tail += dpp0<0x124>(tail); tail += dpp0<0x122>(tail); tail += dpp0<0x121>(tail);
    const int sg = (lane >> 4) & 1;
    const unsigned ct = sg ? __builtin_amdgcn_readlane(tail, 48) : __builtin_amdgcn_readlane(tail, 32);
    if (lane < 32) *(LAS unsigned*)(scr + ENC_SORT + ((lane & 16) + rank) * 4) = cnt;
    LDS_FENCE();
    unsigned best;
    {
        const unsigned cb = lane & 15, s0 = cb / 3u, s1 = cb - 3u * s0, A = 1u << s0, AB = A + (1u << s1);
        unsigned cost = ct * (16u - AB);
#pragma unroll
        for (int q = 0; q < 4; ++q) {
            const u32x4 h = *(LAS u32x4*)(scr + ENC_SORT + sg * 64 + q * 16);
#pragma unroll
            for (int j = 0; j < 4; ++j) {
                const unsigned r = q * 4 + j, len = r < A ? 1u + s0 : r < AB ? 2u + s1 : 4u + ((r - AB) >> 1);
                cost += 2u * len * h[j];
            }
        }
        best = cb < 9 ? (cost << 4) | cb : 0xFFFFFFFFu;
        best = umin(best, dpp0<0x128>(best)); best = umin(best, dpp0<0x124>(best)); best = umin(best, dpp0<0x122>(best)); best = umin(best, dpp0<0x121>(best));
    }
    const unsigned cbP = __builtin_amdgcn_readlane(best, 0) & 15u, cbN = __builtin_amdgcn_readlane(best, 16) & 15u;
    const unsigned s0P = cbP / 3u, s1P = cbP - 3u * s0P, s0N = cbN / 3u, s1N = cbN - 3u * s0N;
    const unsigned ABP = (1u << s0P) + (1u << s1P), ABN = (1u << s0N) + (1u << s1N);
    unsigned F[8];
    {
        const unsigned s0 = sg ? s0N : s0P, s1 = sg ? s1N : s1P, A = 1u << s0, AB = sg ? ABN : ABP;
        const unsigned ent = rank < A ? (rank << 4) | (s0 << 6) : rank < AB ? 1u | ((rank - A) << 4) | (s1 << 6) : (2u + ((rank - AB) >> 1)) | (((rank - AB) & 1u) << 4) | (1u << 6);
        unsigned fv = ent << (8 * (lane & 3));
        fv |= dpp0<0xB1>(fv); fv |= dpp0<0x4E>(fv);
#pragma unroll
        for (int q = 0; q < 8; ++q) F[q] = __builtin_amdgcn_readlane(fv, q * 4);
        if (lane < 32)
            __hip_atomic_fetch_or((LAS unsigned*)(scr + (sg * 2 + (rank >> 3)) * 4), (unsigned)(lane & 15) << (8 * (rank & 3) + 4 * ((rank >> 2) & 1)), __ATOMIC_RELAXED, __HIP_MEMORY_SCOPE_WORKGROUP);
        if (lane == 0) *(LAS unsigned*)(scr + 16) = baseP | (baseN << 8) | (s0P << 16) | (s1P << 18) | (s0N << 20) | (s1N << 22);
    }
    const unsigned ABP4 = ABP * 0x01010101u, ABN4 = ABN * 0x01010101u, S0P4 = s0P * 0x01010101u, S0N4 = s0N * 0x01010101u;
    unsigned KK[16], XW[4], OX[4], TU = 0, TX = 0, bad = 0, xl = 0, xw = 0;
#pragma unroll
    for (int g = 0; g < 16; ++g) {
        const unsigned ms = bytemask(R[g], 6, c0c), m16 = bytemask(D[g], 4, c0c), ent = lut2x16(F, D[g], ms, c0c);
        const unsigned q = ((D[g] | 0x80808080u) - bfi(ms, ABN4, ABP4)) & 0x7F7F7F7Fu;
        unsigned K = bfi(m16, ((q >> 1) & 0x3F3F3F3Fu) + 0x02020202u, ent & 0x0F0F0F0Fu);
        unsigned SUF = bfi(m16, q & 0x01010101u, (ent >> 4) & 0x03030303u);
        unsigned S = bfi(m16, 0x01010101u, (ent >> 6) & 0x03030303u);
        unsigned gl = sad4(K) + 4u;
        if (__ballot(gl > 31u) != 0ull) {
            any |= 0x80u;
            const unsigned km = umax(umax(K & 0xFFu, (K >> 8) & 0xFFu), umax((K >> 16) & 0xFFu, K >> 24));
            const unsigned eq = gl > 31u ? (((K ^ (km * 0x01010101u)) + 0x7F7F7F7Fu) & 0x80808080u) ^ 0x80808080u : 0u, em = bytemask(eq, 7, c0c);
            E[g] |= eq;
            K &= ~em; SUF &= ~em; S = bfi(em, bfi(ms, S0N4, S0P4), S);
            gl = sad4(K) + 4u;
        }
        KK[g] = K; TU += gl; bad = umax(bad, gl);
        xw |= (SUF & 0xFFu) << xl; xl += S & 0xFFu;
        xw |= ((SUF >> 8) & 0xFFu) << xl; xl += (S >> 8) & 0xFFu;
        xw |= ((SUF >> 16) & 0xFFu) << xl; xl += (S >> 16) & 0xFFu;
        xw |= (SUF >> 24) << (xl & 31u); xl += S >> 24;
        if ((g & 3) == 3) { XW[g >> 2] = xw; OX[g >> 2] = TX; TX += xl; xw = 0; xl = 0; }
        __builtin_amdgcn_sched_barrier(0);
    }
    bad = (bad > 31u) | (TU > LANE_U_MAX);
    const unsigned tt = TU | (TX << 16), incl = wave_scan_incl(tt), tot = __builtin_amdgcn_readlane(incl, 63), totU = tot & 0xFFFFu, totX = tot >> 16;
    const unsigned xstart = (AUX_U * 8u + totU + 31u) & ~31u, endbit = xstart + totX;
    unsigned raw = (__ballot(bad != 0) != 0ull) || totU > U_MAX || endbit > (AUX_MAX - 8u) * 8u;
    if (!raw) {
        const unsigned px = xstart + ((incl - tt) >> 16);
        unsigned p = AUX_U * 8u + ((incl - tt) & 0xFFFFu);
#pragma unroll
        for (int g = 0; g < 16; ++g) {
            const unsigned ln = KK[g] + 0x01010101u;
            unsigned pos = (ln & 0xFFu) - 1u, w = 1u << pos;
            pos += (ln >> 8) & 0xFFu; w |= 1u << pos;
            pos += (ln >> 16) & 0xFFu; w |= 1u << pos;
            pos += ln >> 24; w |= 1u << pos;
            lds_or2(scr, p, w); p += pos + 1u;
            __builtin_amdgcn_sched_barrier(0);
        }
#pragma unroll
        for (int q = 0; q < 4; ++q) lds_or2(scr, px + OX[q], XW[q]);
    }
#pragma unroll
    for (int i = 0; i < 7; ++i) { P[i] = R[i] | ((R[14] << (7 - i)) & 0x80808080u); P[7 + i] = R[7 + i] | ((R[15] << (7 - i)) & 0x80808080u); }
    unsigned n = 0;
    const unsigned xo = (endbit + 31u) >> 5;
    if (!raw && __ballot((any & 0xF0F0F0F0u) != 0) != 0ull) {
#pragma unroll
        for (int g = 0; g < 16; ++g) {
            const unsigned f = E[g] & 0xF0F0F0F0u;
            unsigned long long mask = __ballot(f != 0);
            while (mask != 0ull && n <= MAX_EXC) {
                const int L = __builtin_ctzll(mask); mask &= mask - 1;
                const unsigned fl = __builtin_amdgcn_readlane(f, L);
                const unsigned lo = __builtin_amdgcn_readlane((g & 1) ? c[g >> 1].z : c[g >> 1].x, L);
                const unsigned hi = __builtin_amdgcn_readlane((g & 1) ? c[g >> 1].w : c[g >> 1].y, L);
                const bool e = lane < 4 && ((fl >> (8 * (lane & 3))) & 0xFFu) != 0;
                const unsigned long long em = __ballot(e);
                const unsigned slot = xo + n + __builtin_popcountll(em & ((1ull << lane) - 1ull));
                const unsigned val = (((lane & 2) ? hi : lo) >> (16 * (lane & 1))) & 0xFFFFu;
                if (e && slot * 4u < AUX_MAX) *(LAS unsigned*)(scr + slot * 4u) = (unsigned)(L * 64 + g * 4 + (lane & 3)) | (val << 12);
                n += __builtin_popcountll(em);
            }
        }
    }
    raw |= (n > MAX_EXC) | ((xo + n) * 4u > AUX_MAX);
    LDS_FENCE();
    a[0] = *(LAS u32x4*)(scr + lane * 16); a[1] = *(LAS u32x4*)(scr + 1024 + lane * 16); a[2] = *(LAS u32x4*)(scr + 2048 + (lane & 31) * 16);
    return ((xo + n + 3u) >> 2) | (n << 8) | (xo << 16) | (raw << 31);
}

__device__ __forceinline__ void decode_block(LAS unsigned char* src, u32x4 (&c)[8], int lane) {
    LAS unsigned char* aux = src + RAWB;
    const unsigned c0c = opaque(0x0C0C0C0Cu);
    const u32x4 nt = *(LAS const u32x4*)(aux);
    const unsigned prm_v = *(LAS const unsigned*)(aux + 16);
    unsigned u[5];
#pragma unroll
    for (int i = 0; i < 5; ++i) u[i] = *(LAS const unsigned*)(aux + AUX_U + lane * 20 + i * 4);
    u32x4 P4[3];
#pragma unroll
    for (int p = 0; p < 3; ++p) P4[p] = *(LAS const u32x4*)(src + p * 1024 + lane * 16);
    const unsigned long long ph = *(LAS const unsigned long long*)(src + 3072 + lane * 8);
    const unsigned c0 = __builtin_popcount(u[0]), c1 = c0 + __builtin_popcount(u[1]), c2 = c1 + __builtin_popcount(u[2]), c3 = c2 + __builtin_popcount(u[3]), cn = c3 + __builtin_popcount(u[4]);
    const unsigned C = wave_scan_incl(cn) - cn;
    if (lane == 0) *(LAS unsigned*)(src + DEC_POS) = AUX_U * 8u;
    const int rounds = __ballot(cn > 128u) != 0ull ? 3 : 2;
    for (int j = 0; j < rounds; ++j) {
        const unsigned t = (C >> 6) + 1u + j, kth = 64u * t - C;
        const unsigned i = (kth > c0) + (kth > c1) + (kth > c2) + (kth > c3);
        unsigned rem = kth - (i == 0 ? 0u : i == 1 ? c0 : i == 2 ? c1 : i == 3 ? c2 : c3), w = i == 0 ? u[0] : i == 1 ? u[1] : i == 2 ? u[2] : i == 3 ? u[3] : u[4], pos = 0;
#pragma unroll
        for (int st = 16; st >= 1; st >>= 1) {
            const unsigned lo = __builtin_popcount(w & ((1u << st) - 1u));
            const bool up = rem > lo;
            rem -= up ? lo : 0u; w = up ? w >> st : w; pos += up ? st : 0;
        }
        if (kth <= cn && t < 64u) *(LAS unsigned*)(src + DEC_POS + t * 4) = AUX_U * 8u + lane * 160u + i * 32u + pos + 1u;
    }
    LDS_FENCE();
    const unsigned p0 = *(LAS const unsigned*)(src + DEC_POS + lane * 4);
    unsigned S[8];
    {
        LAS const unsigned* q = (LAS const unsigned*)(aux + ((p0 >> 5) << 2));
#pragma unroll
        for (int i = 0; i < 8; ++i) S[i] = q[i];
#pragma unroll
        for (int i = 0; i < 7; ++i) S[i] = __builtin_amdgcn_alignbit(S[i + 1], S[i], p0 & 31u);
    }
    unsigned R[16];
    {
        const unsigned P[14] = {P4[0].x, P4[0].y, P4[0].z, P4[0].w, P4[1].x, P4[1].y, P4[1].z, P4[1].w, P4[2].x, P4[2].y, P4[2].z, P4[2].w, (unsigned)ph, (unsigned)(ph >> 32)};
        unsigned x = 0, y = 0;
#pragma unroll
        for (int i = 0; i < 7; ++i) { x |= (P[i] >> (7 - i)) & (0x01010101u << i); y |= (P[7 + i] >> (7 - i)) & (0x01010101u << i); }
#pragma unroll
        for (int i = 0; i < 14; ++i) R[i] = P[i] & 0x7F7F7F7Fu;
        R[14] = x; R[15] = y;
    }
    const unsigned n0 = rfl(nt.x), n1 = rfl(nt.y), n2 = rfl(nt.z), n3 = rfl(nt.w), prm = rfl(prm_v);
    const unsigned I[8] = {n0 & 0x0F0F0F0Fu, (n0 >> 4) & 0x0F0F0F0Fu, n1 & 0x0F0F0F0Fu, (n1 >> 4) & 0x0F0F0F0Fu, n2 & 0x0F0F0F0Fu, (n2 >> 4) & 0x0F0F0F0Fu, n3 & 0x0F0F0F0Fu, (n3 >> 4) & 0x0F0F0F0Fu};
    const unsigned s0P = (prm >> 16) & 3u, s1P = (prm >> 18) & 3u, s0N = (prm >> 20) & 3u, s1N = (prm >> 22) & 3u;
    const unsigned STP = (s0P << 8) | (s1P << 16) | (1u << 24), STN = (s0N << 8) | (s1N << 16) | (1u << 24);
    const unsigned OTP = (4u << 8) | (((1u << s0P) + 2u) << 16) | (((1u << s0P) + (1u << s1P)) << 24);
    const unsigned OTN = (4u << 8) | (((1u << s0N) + 2u) << 16) | (((1u << s0N) + (1u << s1N)) << 24);
    const unsigned BP = (prm & 0xFFu) * 0x01010101u, BN = ((prm >> 8) & 0xFFu) * 0x01010101u;
    unsigned RB[16], SS[16], TU = 0, TX = 0;
#pragma unroll
    for (int g = 0; g < 16; ++g) {
        const unsigned w0 = S[0], q0 = __builtin_ctz(w0), w1 = w0 & (w0 - 1u), q1 = __builtin_ctz(w1), w2 = w1 & (w1 - 1u), q2 = __builtin_ctz(w2), w3 = w2 & (w2 - 1u), q3 = __builtin_ctz(w3);
        const unsigned Q = q0 | (q1 << 8) | (q2 << 16) | (q3 << 24);
        const unsigned LN = Q - opaque((Q << 8) - 1u), sh = q3 + 1u;
        TU += sh;
        if (g < 15) {
#pragma unroll
            for (int i = 0; i < 7; ++i) if (32 * i < (int)LANE_U_MAX - 4 * (g + 1)) S[i] = (i < 6 && 32 * (i + 1) < (int)LANE_U_MAX - 4 * g) ? __builtin_amdgcn_alignbit(S[i + 1], S[i], sh) : S[i] >> sh;
        }
        const unsigned sel = bfi_c(0x04040404u, R[g] >> 4, bfi(bytemask(LN + 0x7D7D7D7Du, 7, c0c), 0x03030303u, LN));
        SS[g] = perm(STN, STP, sel);
        RB[g] = perm(OTN, OTP, sel) + (LN << 1);
        TX = __builtin_amdgcn_sad_u8(SS[g], 0u, TX);
    }
    const unsigned xstart = (__builtin_amdgcn_readlane(p0 + TU, 63) + 31u) & ~31u, px = xstart + wave_scan_incl(TX) - TX;
    unsigned X[5];
    {
        LAS const unsigned* q = (LAS const unsigned*)(aux + ((px >> 5) << 2));
#pragma unroll
        for (int i = 0; i < 5; ++i) X[i] = q[i];
#pragma unroll
        for (int i = 0; i < 4; ++i) X[i] = __builtin_amdgcn_alignbit(X[i + 1], X[i], px & 31u);
    }
    unsigned xoff = 0;
#pragma unroll
    for (int g = 0; g < 16; ++g) {
        const unsigned s = SS[g], M = perm(0u, 0x00030100u, s);
        const unsigned y = (g & 1) ? X[0] >> xoff : X[0], Y = perm(y, y, 0x01000100u);
        const unsigned t1 = opaque(s << 8), t2 = opaque((t1 << 8) + t1), Cs = (t2 << 16) + t2;
        const u16x2 z02 = __builtin_bit_cast(u16x2, Y) >> __builtin_bit_cast(u16x2, Cs & 0x00FF00FFu);
        const u16x2 z13 = __builtin_bit_cast(u16x2, Y) >> __builtin_bit_cast(u16x2, perm(0u, Cs, 0x0C030C01u));
        const unsigned suf = perm(__builtin_bit_cast(unsigned, z13), __builtin_bit_cast(unsigned, z02), 0x06020400u) & M;
        if (g & 1) {
            if (g < 15) {
                const unsigned t = __builtin_amdgcn_sad_u8(s, 0u, xoff);
#pragma unroll
                for (int i = 0; i < 4; ++i) if (32 * i < 8 * (15 - g)) X[i] = (i < 3 && 32 * (i + 1) < 8 * (17 - g)) ? __builtin_amdgcn_alignbit(X[i + 1], X[i], t) : X[i] >> t;
            }
        } else xoff = sad4(s);
        const unsigned rank = RB[g] + suf - 0x06060606u, ms = bytemask(R[g], 6, c0c);
        const unsigned D = bfi(bytemask(rank, 4, c0c), rank, lut2x16(I, rank, ms, c0c));
        const unsigned E = bfi(ms, BN, BP) - ((D >> 1) & 0x0F0F0F0Fu);
        const unsigned H = bfi_c(0x80808080u, R[g] << 1, E >> 1), L = bfi_c(0x3F3F3F3Fu, R[g], bfi_c(0x40404040u, D << 6, E << 7));
        const unsigned x = perm(H, L, 0x05010400u), yv = perm(H, L, 0x07030602u);
        if (g & 1) { c[g >> 1].z = x; c[g >> 1].w = yv; } else { c[g >> 1].x = x; c[g >> 1].y = yv; }
    }
}

typedef unsigned u32x2 __attribute__((ext_vector_type(2)));
__device__ __forceinline__ void store_granule(char* p, u32x2 g) {
    asm volatile("global_store_dwordx2 %0, %1, off sc0 sc1\n\ts_nop 1" :: "v"(p), "v"(g) : "memory");
}

constexpr unsigned GCTL = 131072;
__device__ __forceinline__ void publish_block(const Params& P, rsrc_t wsr, const u32x4 (&c)[8], unsigned slot, unsigned n, unsigned dmask, unsigned call, LAS unsigned char* lds, LAS unsigned char* scr,
                                              unsigned j, int wid, int lane) {
    unsigned R[14], h = 1u << 31; u32x4 a[3];
    if (n >= NRAW && n < NSEQ - NRAWEND) h = encode_block(c, R, a, scr, lane);
    const unsigned so = OFF_SLOT + slot * 8192u;
    if (h >> 31) {
#pragma unroll
        for (int q = 0; q < 8; ++q) __builtin_amdgcn_raw_buffer_store_b128(c[q], wsr, lane * 16 + q * 1024, so, PAY_AUX);
    } else {
        const unsigned n16 = h & 255u;
#pragma unroll
        for (int q = 0; q < 3; ++q) __builtin_amdgcn_raw_buffer_store_b128((u32x4){R[4 * q], R[4 * q + 1], R[4 * q + 2], R[4 * q + 3]}, wsr, lane * 16 + q * 1024, so, PAY_AUX);
        __builtin_amdgcn_raw_buffer_store_b64((u32x2){R[12], R[13]}, wsr, lane * 8 + 3072, so, PAY_AUX);
        if ((unsigned)lane < n16) __builtin_amdgcn_raw_buffer_store_b128(a[0], wsr, lane * 16 + RAWB, so, PAY_AUX);
        if ((unsigned)lane + 64u < n16) __builtin_amdgcn_raw_buffer_store_b128(a[1], wsr, lane * 16 + RAWB + 1024, so, PAY_AUX);
        if ((unsigned)lane + 128u < n16) __builtin_amdgcn_raw_buffer_store_b128(a[2], wsr, lane * 16 + RAWB + 2048, so, PAY_AUX);
    }
    WAIT_VM0();
    const unsigned u = j & 3u, gsize = j == 0 ? 4u : 8u;
    volatile LAS unsigned* ctl = (volatile LAS unsigned*)(lds + GCTL);
    while (j >= 4u && (int)(ctl[36 + u] - (j - 3u)) < 0) __builtin_amdgcn_s_sleep(1);
    unsigned arrived = 0;
    if (lane == 0) {
        ctl[u * 8 + wid] = h;
        arrived = __hip_atomic_fetch_add((LAS unsigned*)(lds + GCTL) + 32 + u, 1u, __ATOMIC_RELAXED, __HIP_MEMORY_SCOPE_WORKGROUP);
    }
    if (rfl(arrived) == gsize - 1u) {
        const unsigned d = lane >> 3, w = lane & 7, hw = ctl[u * 8 + w];
        if (w < gsize && ((dmask >> d) & 1u)) store_granule(P.ws[d] + OFF_GRAN + ((unsigned)P.rank * NSEQ + n - wid + w) * 8u, (u32x2){call, hw});
        if (lane == 0) { ctl[32 + u] = 0u; ctl[36 + u] = j + 1u; }
    }
}

__device__ __forceinline__ void signal_cnt(char* ws, unsigned off, int lane) {
    if (off && lane == 0) __hip_atomic_fetch_add((unsigned*)(ws + off), 1u, __ATOMIC_RELAXED, __HIP_MEMORY_SCOPE_SYSTEM);
}
__device__ __forceinline__ unsigned packer_role(const Params& P, LAS unsigned char* lds, int first, int next, int step, int end) {
    unsigned call = 0;
    const int lane = threadIdx.x & 63, wid = rfl(threadIdx.x >> 6);
    char* ws = P.ws[P.rank];
    const rsrc_t wsr = mk_rsrc(ws, WS_USED);
    const rsrc_t xr = mk_rsrc(P.x, DIM * KSH * 2), wr = mk_rsrc(P.w, DIM * KSH * 2);
    const int myi = P.rank >> 2, myj = P.rank & 3;
    LAS unsigned char* my = lds + wid * 8192;
    unsigned j = first < NFIRST ? 0u : 1u;
    const unsigned long long tstart = __builtin_amdgcn_s_memrealtime();
    for (int p = first; p < end; p = p < NFIRST ? next : p + step, ++j) {
        if (p < 4096) {
            const int due = (p < 3072 ? (p / 192) * 580 : 9280 + ((p - 3072) >> 6) * 290) - JIT_LEAD * 100;
            while ((int)(__builtin_amdgcn_s_memrealtime() - tstart) < due) __builtin_amdgcn_s_sleep(16);
        }
        u32x4 c[8];
        unsigned slot, n, dmask, selfcnt = 0;
        if (p >= 4096) {
            const unsigned o = p - 4096, w = o & 7, h = (o >> 3) & 1, tile = (o >> 4) & 63;
            if (call == 0) call = call_draw(ws);
            unsigned* tmo = (unsigned*)(ws + OFF_TMO);
            wait_word((unsigned*)(ws + OFF_OCNT + (o >> 3) * 128), call * 8u, tmo, call);
            const unsigned row0 = (P.rank >> 2) * 4096 + (tile >> 3) * 256 + h * 128 + (w >> 2) * 64, col0 = (P.rank & 3) * 2048 + (tile & 7) * 256 + (w & 3) * 64;
            const rsrc_t outr = mk_rsrc(P.out, (unsigned)DIM * DIM * 2u);
#pragma unroll
            for (int q = 0; q < 8; ++q) c[q] = __builtin_amdgcn_raw_buffer_load_b128(outr, (unsigned)(lane >> 3) * (DIM * 2u) + (unsigned)(lane & 7) * 16u, (row0 * DIM + col0) * 2u + q * (8u * DIM * 2u), AUX_SYS);
            if (rfl(__hip_atomic_load(tmo, __ATOMIC_RELAXED, __HIP_MEMORY_SCOPE_AGENT)) == call) {
#pragma unroll
                for (int q = 0; q < 8; ++q) c[q] = (u32x4){0x7FC07FC0u, 0x7FC07FC0u, 0x7FC07FC0u, 0x7FC07FC0u};
            }
            slot = 4096u + o; n = 1536u + o; dmask = 0xFFu;
        } else {
            bool is_w; int kk, sel, r, bi;
            if (p < 3072) {
                kk = p / 192; const int rem = p - kk * 192;
                bi = rem & 31; r = 0; is_w = rem >= 64; sel = is_w ? ((rem - 64) >> 5) : (rem >> 5);
            } else { const int q = p - 3072; kk = q >> 6; sel = (q >> 5) & 1; bi = q & 31; r = 1; is_w = false; }
            if (!is_w) {
                const int rr = lane >> 3;
                const unsigned vo = (unsigned)rr * 2048u + (unsigned)(((lane & 7) ^ rr) * 16);
                const unsigned so = (unsigned)(sel * 4096 + r * 2048 + bi * 64) * 2048u + (unsigned)kk * 128u;
#pragma unroll
                for (int q = 0; q < 8; ++q) c[q] = __builtin_amdgcn_raw_buffer_load_b128(xr, vo + q * 8 * 2048, so, XIN_AUX);
            } else {
                const int pn = bi >> 2, bj = (bi >> 1) & 1, q = bi & 1;
                const int gk = lane >> 3, cc = lane & 7, wcl = cc >> 2, cq = cc & 3;
                const unsigned vo = (unsigned)(gk * 8) * 16384u + (unsigned)((64 * (2 * q + wcl) + 32 * bj + 8 * cq) * 2);
                const unsigned so = (unsigned)(kk * 64) * 16384u + (unsigned)((sel * 2048 + pn * 256) * 2);
                u32x4 R[8];
#pragma unroll
                for (int jj = 0; jj < 8; ++jj) R[jj] = __builtin_amdgcn_raw_buffer_load_b128(wr, vo + jj * 16384, so, XIN_AUX);
#pragma unroll
                for (int t = 0; t < 8; ++t) {
                    const unsigned s = (t & 1) ? 0x07060302u : 0x05040100u;
                    u32x4 v;
                    v.x = perm(R[1][t >> 1], R[0][t >> 1], s); v.y = perm(R[3][t >> 1], R[2][t >> 1], s);
                    v.z = perm(R[5][t >> 1], R[4][t >> 1], s); v.w = perm(R[7][t >> 1], R[6][t >> 1], s);
                    const int s64 = 32 * wcl + 16 * (t >> 2) + 4 * cq + (t & 3);
                    *(LAS u32x4*)(my + s64 * 128 + ((gk ^ (s64 & 7)) * 16)) = v;
                }
                asm volatile("s_waitcnt lgkmcnt(0)" ::: "memory");
#pragma unroll
                for (int q2 = 0; q2 < 8; ++q2) c[q2] = *(LAS u32x4*)(my + q2 * 1024 + lane * 16);
                asm volatile("s_waitcnt lgkmcnt(0)" ::: "memory");
            }
            if (call == 0) call = call_draw(ws);
            const int g = P.rank * 16 + kk;
            if (is_w ? (sel == myj) : (sel == myi)) {
                const unsigned so = is_w ? OFF_BIMG + (unsigned)(g * 32 + bi) * 8192u : OFF_AIMG + (unsigned)((r * 128 + g) * 32 + bi) * 8192u;
#pragma unroll
                for (int q = 0; q < 8; ++q) __builtin_amdgcn_raw_buffer_store_b128(c[q], wsr, lane * 16 + q * 1024, so, AUX_SC1);
                selfcnt = OFF_CNT + (r * 128 + g) * 128;
            }
            slot = is_w ? 2048u + (unsigned)((sel * 16 + kk) * 32 + bi) : (unsigned)(((sel * 2 + r) * 16 + kk) * 32 + bi);
            n = is_w ? (unsigned)(kk * 64 + 32 + bi) : (r == 0 ? (unsigned)(kk * 64 + bi) : (unsigned)(1024 + kk * 32 + bi));
            dmask = is_w ? (0x11u << sel) : (0x0Fu << (4 * sel));
        }
        publish_block(P, wsr, c, slot, n, dmask & ~(1u << P.rank), call, lds, my, j, wid, lane);
        signal_cnt(ws, selfcnt, lane);
    }
    if (call == 0) call = call_draw(ws);
    return call;
}

__device__ __forceinline__ u32x2 load_granule(rsrc_t wsr, unsigned off) { return __builtin_amdgcn_raw_buffer_load_b64(wsr, 0, off, AUX_SYS); }

struct Blk { unsigned slot, dso, cnt; bool is_out; };
__device__ __forceinline__ Blk locate(unsigned n, int src, int myi, int myj) {
    Blk b; b.is_out = false;
    if (n < 1024) { const int kk = n >> 6, bi = n & 31, g = src * 16 + kk; b.cnt = OFF_CNT + g * 128;
        if ((n & 63) >= 32) { b.slot = 2048u + (myj * 16 + kk) * 32 + bi; b.dso = OFF_BIMG + (unsigned)(g * 32 + bi) * 8192u; }
        else { b.slot = ((myi * 2) * 16 + kk) * 32 + bi; b.dso = OFF_AIMG + (unsigned)(g * 32 + bi) * 8192u; } }
    else if (n < 1536) { const int kk = (n - 1024) >> 5, bi = n & 31, g = src * 16 + kk; b.cnt = OFF_CNT + (128 + g) * 128;
        b.slot = ((myi * 2 + 1) * 16 + kk) * 32 + bi; b.dso = OFF_AIMG + (unsigned)((128 + g) * 32 + bi) * 8192u; }
    else { const int o = n - 1536, w = o & 7, h = (o >> 3) & 1, tile = (o >> 4) & 63, r = o >> 10; b.cnt = 0; b.is_out = true;
        b.slot = 4096u + o;
        const unsigned row0 = (src >> 2) * 4096 + r * 2048 + (tile >> 3) * 256 + h * 128 + (w >> 2) * 64, col0 = (src & 3) * 2048 + (tile & 7) * 256 + (w & 3) * 64;
        b.dso = (row0 * DIM + col0) * 2u; }
    return b;
}
constexpr unsigned NSLOT = 15, CTL = NSLOT * 8192;
constexpr unsigned M_DEAD = 1u << 30, N_STOP = 0xFFFFFFFFu;
__device__ __forceinline__ unsigned lds_ld(LAS unsigned char* p) { return *(volatile LAS unsigned*)p; }
__device__ __forceinline__ void lds_st(LAS unsigned char* p, unsigned v) { *(volatile LAS unsigned*)p = v; }
__device__ __forceinline__ void ring_publish(LAS unsigned char* lds, unsigned i, unsigned n, unsigned h, int lane) {
    const unsigned sl = i % NSLOT;
    if (lane == 0) { *(volatile LAS u32x2*)(lds + CTL + sl * 8) = (u32x2){n, h}; lds_st(lds + CTL + 256 + sl * 4, i + 1u); }
}
__device__ __forceinline__ void dma_op(rsrc_t r, LAS unsigned char* dst, int j, unsigned so, int lane) {
    __builtin_amdgcn_raw_ptr_buffer_load_lds(r, (LAS void*)(dst + j * 1024), 16, lane * 16 + j * 1024, so, 0, AUX_SYS);
}
#define VM_WAIT_(n) asm volatile("s_waitcnt vmcnt(" #n ")" ::: "memory")
#define VM_WAIT(n) VM_WAIT_(n)
#define WIN 5

__device__ __forceinline__ void loader_wave(const Params& P, LAS unsigned char* lds, unsigned call, int src, int lane) {
    char* ws = P.ws[P.rank];
    unsigned* tmo = (unsigned*)(ws + OFF_TMO);
    const rsrc_t wsr = mk_rsrc(ws, WS_USED);
    char* const remp = P.ws[src];
    const int myi = P.rank >> 2, myj = P.rank & 3;
    const unsigned gbase = OFF_GRAN + (unsigned)src * NSEQ * 8u;
    char* const gptr = ws + gbase;
    const unsigned tko = OFF_LINK + (unsigned)src * 128u, tbase = (call - 1u) * (NSEQ + 3u * PWL);
    __builtin_amdgcn_s_setprio(3);
    unsigned n, n1, n2;
    {
        unsigned t0 = 0;
        if (lane == 0) t0 = __hip_atomic_fetch_add((unsigned*)(ws + tko), 3u, __ATOMIC_RELAXED, __HIP_MEMORY_SCOPE_AGENT);
        n = rfl(t0) - tbase; n1 = n + 1; n2 = n + 2;
    }
    {
        const rsrc_t rw = mk_rsrc(remp, OFF_AIMG);
        const unsigned v = __builtin_amdgcn_raw_buffer_load_b32(rw, lane < 25 ? OFF_SLOT + (unsigned)lane * (2u << 20) + 8188u - (lane == 24 ? 8192u : 0u) : 0xFFFFFFF0u, 0, AUX_SYS);
        asm volatile("" :: "v"(v));
    }
    unsigned gx, gy, hx, hy;
    { const u32x2 a = load_granule(wsr, gbase + (n < NSEQ ? n : 0u) * 8u), b = load_granule(wsr, gbase + (n1 < NSEQ ? n1 : 0u) * 8u);
      gx = rfl(a.x); gy = rfl(a.y); hx = rfl(b.x); hy = rfl(b.y); }
    const int zero = 0, one = 1;
    unsigned i = 0;
    bool pend = false; unsigned pn = 0, ph = 0;
    while (n < NSEQ) {
        const unsigned sl = i % NSLOT;
        bool dead = false;
        const bool need_gran = (int)(gx - call) < 0;
        const bool need_slot = i >= NSLOT && (int)(lds_ld(lds + CTL + 320 + sl * 4) - (i - NSLOT + 1u)) < 0;
        if (need_gran || need_slot) {
            if (pend) { VM_WAIT(0); ring_publish(lds, i - 1u, pn, ph, lane); pend = false; }
            const unsigned long long t0 = __builtin_amdgcn_s_memrealtime();
            while ((int)(gx - call) < 0) {
                const u32x2 a = load_granule(wsr, gbase + n * 8u);
                gx = rfl(a.x); gy = rfl(a.y);
                if ((int)(gx - call) >= 0) break;
                if (give_up(t0, tmo, call)) { dead = true; break; }
                __builtin_amdgcn_s_sleep(2);
                asm volatile("" ::: "memory");
            }
            while (i >= NSLOT && (int)(lds_ld(lds + CTL + 320 + sl * 4) - (i - NSLOT + 1u)) < 0) __builtin_amdgcn_s_sleep(1);
        }
        const bool raw = (gy >> 31) != 0;
        const unsigned so = OFF_SLOT + locate(n, src, myi, myj).slot * 8192u;
                const unsigned bytes = dead ? 0u : raw ? 8192u : RAWB + ((gy & 255u) << 4);
        const rsrc_t rem = mk_rsrc(remp, so + bytes);
        LAS unsigned char* dst = lds + sl * 8192;
        u32x2 g2; unsigned tk = 0;
        asm volatile("global_load_dwordx2 %0, %1, %2 sc0 sc1" : "=v"(g2) : "v"(zero), "s"(gptr + (n2 < NSEQ ? n2 : 0u) * 8u) : "memory");
        if (lane == 0) asm volatile("global_atomic_add %0, %1, %2, %3 sc0" : "=v"(tk) : "v"(zero), "v"(one), "s"(ws + tko) : "memory");
        VM_WAIT(WIN); dma_op(rem, dst, 0, so, lane);
        VM_WAIT(WIN); dma_op(rem, dst, 1, so, lane);
        VM_WAIT(WIN); dma_op(rem, dst, 2, so, lane);
        VM_WAIT(WIN); if (pend) ring_publish(lds, i - 1u, pn, ph, lane); dma_op(rem, dst, 3, so, lane);
        VM_WAIT(WIN); dma_op(rem, dst, 4, so, lane);
        asm volatile("s_waitcnt vmcnt(5)" : "+v"(g2), "+v"(tk) :: "memory"); dma_op(rem, dst, 5, so, lane);
        if (raw) { dma_op(rem, dst, 6, so, lane); dma_op(rem, dst, 7, so, lane); VM_WAIT(0); }
        pend = true; pn = n; ph = gy | (dead ? M_DEAD : 0u);
        n = n1; gx = hx; gy = hy;
        n1 = n2; hx = rfl(g2.x); hy = rfl(g2.y);
        n2 = rfl(tk) - tbase;
        ++i;
    }
    VM_WAIT(0);
    if (pend) ring_publish(lds, i - 1u, pn, ph, lane);
    for (int d = 0; d < NDEC; ++d, ++i) {
        const unsigned sl = i % NSLOT;
        while (i >= NSLOT && (int)(lds_ld(lds + CTL + 320 + sl * 4) - (i - NSLOT + 1u)) < 0) __builtin_amdgcn_s_sleep(1);
        ring_publish(lds, i, N_STOP, 0u, lane);
    }
}

__device__ __forceinline__ void decoder_wave(const Params& P, LAS unsigned char* lds, unsigned call, int src, int d, int lane) {
    char* ws = P.ws[P.rank];
    unsigned* tmo = (unsigned*)(ws + OFF_TMO);
    const rsrc_t wsr = mk_rsrc(ws, WS_USED), outr = mk_rsrc(P.out, (unsigned)DIM * DIM * 2u);
    const int myi = P.rank >> 2, myj = P.rank & 3;
    for (unsigned i = d;; i += NDEC) {
        const unsigned sl = i % NSLOT;
        while (lds_ld(lds + CTL + 256 + sl * 4) != i + 1u) __builtin_amdgcn_s_sleep(1);
        asm volatile("" ::: "memory");
        const u32x2 meta = *(volatile LAS u32x2*)(lds + CTL + sl * 8);
        const unsigned n = rfl(meta.x), h = rfl(meta.y);
        if (n == N_STOP) break;
        const Blk b = locate(n, src, myi, myj);
        const bool raw = (h >> 31) != 0;
        bool dead = (h & M_DEAD) != 0;
        if (b.is_out && rfl(__hip_atomic_load(tmo, __ATOMIC_RELAXED, __HIP_MEMORY_SCOPE_AGENT)) == call) dead = true;
        u32x4 c[8];
        unsigned recs = 0;
        const unsigned ne = (dead || raw) ? 0u : (h >> 8) & 63u;
        LAS unsigned char* cur = lds + sl * 8192;
        if (dead) {
#pragma unroll
            for (int j = 0; j < 8; ++j) c[j] = (u32x4){0x7FC07FC0u, 0x7FC07FC0u, 0x7FC07FC0u, 0x7FC07FC0u};
        } else if (raw) {
#pragma unroll
            for (int j = 0; j < 8; ++j) c[j] = *(LAS u32x4*)(cur + j * 1024 + lane * 16);
        } else {
            recs = *(LAS unsigned*)(cur + RAWB + (((h >> 16) & 0x3FFu) + (lane & 31)) * 4u);
            decode_block(cur, c, lane);
        }
#pragma unroll
        for (int j = 0; j < 8; ++j) asm volatile("" : "+v"(c[j]));
        asm volatile("" : "+v"(recs));
        if (lane == 0) lds_st(lds + CTL + 320 + sl * 4, i + 1u);
        const rsrc_t dr = b.is_out ? outr : wsr;
        const unsigned sj = b.is_out ? 8u * DIM * 2u : 1024u, sr = b.is_out ? DIM * 2u : 128u;
        const unsigned vo = (unsigned)(lane >> 3) * sr + (unsigned)(lane & 7) * 16u;
        if (b.is_out) {
#pragma unroll
            for (int j = 0; j < 8; ++j) __builtin_amdgcn_raw_buffer_store_b128(c[j], outr, vo, b.dso + j * sj, OUT_AUX);
        } else {
#pragma unroll
            for (int j = 0; j < 8; ++j) __builtin_amdgcn_raw_buffer_store_b128(c[j], wsr, vo, b.dso + j * sj, IMG_AUX);
        }
        for (unsigned e = 0; e < ne; ++e) {
            const unsigned rec = __builtin_amdgcn_readlane(recs, e), pos = rec & 0xFFFu, v = pos & 63u;
            if ((unsigned)lane == (pos >> 6)) __builtin_amdgcn_raw_buffer_store_b16((unsigned short)((rec >> 12) & 0xFFFFu), dr, vo + (v & 7u) * 2u, b.dso + (v >> 3) * sj, AUX_SC1);
        }
        if (b.cnt) { WAIT_VM0(); signal_cnt(ws, b.cnt, lane); }
    }
}

__device__ __forceinline__ void puller_role(const Params& P, LAS unsigned char* lds, unsigned call, int wg) {
    const int lane = threadIdx.x & 63, wid = rfl(threadIdx.x >> 6);
    const int link = wg % 7, src = rfl((P.rank + 1 + link) & 7);
    if (threadIdx.x < 32) *(volatile LAS unsigned*)(lds + CTL + 256 + threadIdx.x * 4) = 0u;
    __syncthreads();
    if (wid == 0) loader_wave(P, lds, call, src, lane);
    else if (wid <= NDEC) decoder_wave(P, lds, call, src, wid - 1, lane);
}

__device__ __forceinline__ float gelu_f(float y) {
    const float c1 = -2.0f * 0.7978845608028654f * 1.4426950408889634f, c2 = c1 * 0.044715f;
    const float e = __builtin_amdgcn_exp2f(y * (c1 + c2 * y * y));
    return y * __builtin_amdgcn_rcpf(1.0f + e);
}
__device__ __forceinline__ unsigned cvt_pk_bf16(float lo, float hi) { unsigned r; asm("v_cvt_pk_bf16_f32 %0, %1, %2" : "=v"(r) : "v"(lo), "v"(hi)); return r; }

__device__ __forceinline__ void gemm_role(const Params& P, LAS unsigned char* lds, unsigned call, int wg) {
    const int tid = threadIdx.x, wid = rfl(tid >> 6), lane = tid & 63, wr = wid >> 2, wc = wid & 3, fr = lane & 15, fq = lane >> 4;
    char* ws = P.ws[P.rank];
    unsigned* tmo = (unsigned*)(ws + OFF_TMO);
    const rsrc_t wsr = mk_rsrc(ws, WS_USED);
    const int xcd = wg & 7, q = (wg - NPACK) >> 3, h = xcd & 1, blk = xcd >> 1;
    const int tm = (blk >> 1) * 4 + (q >> 2), tn = (blk & 1) * 4 + (q & 3), tile = tm * 8 + tn;
    const unsigned voff = (unsigned)tid * 16u, ldsw = (unsigned)wid * 1024u;
    const int aoff0 = (wr * 64 + fr) * 128 + ((fq ^ (fr & 7)) << 4), aoff1 = aoff0 ^ 64;
    const int boff0 = (wc * 32 + fr) * 128 + ((fq ^ (fr & 7)) << 4), boff1 = boff0 ^ 64;
#define G_SA(b, hh) (((b) * 2 + (hh)) * 16384)
#define G_SB(b, hh) ((4 + (b) * 2 + (hh)) * 16384)
#define G_STAGE(bufoff, soff) do { _Pragma("unroll") for (int _i = 0; _i < 2; ++_i) \
        __builtin_amdgcn_raw_ptr_buffer_load_lds(wsr, (LAS void*)(lds + (bufoff) + ldsw + _i * 8192), 16, voff, (soff) + _i * 8192, 0, STG_AUX); } while (0)
#define G_LDA(dst, b, hh) do { _Pragma("unroll") for (int m = 0; m < 4; ++m) { \
        dst[m][0] = *(const LAS bf16x8*)(lds + G_SA(b, hh) + aoff0 + m * 2048); dst[m][1] = *(const LAS bf16x8*)(lds + G_SA(b, hh) + aoff1 + m * 2048); } } while (0)
#define G_LDB(dst, b, hh) do { _Pragma("unroll") for (int n = 0; n < 2; ++n) { \
        dst[n][0] = *(const LAS bf16x8*)(lds + G_SB(b, hh) + boff0 + n * 2048); dst[n][1] = *(const LAS bf16x8*)(lds + G_SB(b, hh) + boff1 + n * 2048); } } while (0)
#define G_MMA(ai, bj, At, Bt) do { __builtin_amdgcn_s_setprio(1); _Pragma("unroll") for (int m = 0; m < 4; ++m) _Pragma("unroll") for (int n = 0; n < 2; ++n) _Pragma("unroll") for (int k = 0; k < 2; ++k) \
        acc[ai][bj][m][n] = __builtin_amdgcn_mfma_f32_16x16x32_bf16(Bt[n][k], At[m][k], acc[ai][bj][m][n], 0, 0, 0); __builtin_amdgcn_s_setprio(0); } while (0)
#define G_WAIT_V(n) asm volatile("s_waitcnt vmcnt(" #n ")" ::: "memory")
#define G_WAIT_L(n) asm volatile("s_waitcnt lgkmcnt(" #n ")" ::: "memory")
#define G_BAR __builtin_amdgcn_s_barrier()
#define G_SCHED __builtin_amdgcn_sched_barrier(0)
#define G_GIDX(t) ((unsigned)((4 * h + ((t) & 3)) * 16 + ((t) >> 2)))
    const int zero = 0;
    for (int r = 0; r < 2; ++r) {
        const unsigned target = call * (r == 0 ? 64u : 32u);
        const unsigned abase = OFF_AIMG + (unsigned)r * (128u * 262144u) + (unsigned)tm * 32768u, bbase = OFF_BIMG + (unsigned)tn * 32768u;
        char* cbase = ws + OFF_CNT + r * 128 * 128;
        f32x4 acc[2][2][4][2];
#pragma unroll
        for (int a = 0; a < 2; ++a)
#pragma unroll
            for (int b = 0; b < 2; ++b)
#pragma unroll
                for (int m = 0; m < 4; ++m)
#pragma unroll
                    for (int n = 0; n < 2; ++n) acc[a][b][m][n] = (f32x4){0.f, 0.f, 0.f, 0.f};
        bf16x8 At[4][2], B0[2][2], B1[2][2];
        STAMP(ws, r * 8 + 0);
        wait_word((unsigned*)(cbase + G_GIDX(0) * 128), target, tmo, call);
        wait_word((unsigned*)(cbase + G_GIDX(1) * 128), target, tmo, call);
        wait_word((unsigned*)(cbase + G_GIDX(2) * 128), target, tmo, call);
        STAMP(ws, r * 8 + 1);
        {
            const unsigned a0 = abase + G_GIDX(0) * 262144u, b0 = bbase + G_GIDX(0) * 262144u, a1 = abase + G_GIDX(1) * 262144u, b1 = bbase + G_GIDX(1) * 262144u;
            G_STAGE(G_SB(0, 0), b0); G_STAGE(G_SA(0, 0), a0); G_STAGE(G_SB(0, 1), b0 + 16384); G_STAGE(G_SA(0, 1), a0 + 16384);
            if (wr == 1) G_BAR;
            G_WAIT_V(4); G_BAR;
            G_STAGE(G_SB(1, 0), b1); G_STAGE(G_SA(1, 0), a1); G_STAGE(G_SB(1, 1), b1 + 16384);
            G_WAIT_V(6); G_BAR;
        }
        unsigned fA = target, fB;
        for (int t = 0; t < 64; t += 2) {
            const int t2 = t + 2 < 64 ? t + 2 : 63, t3 = t + 3 < 64 ? t + 3 : 63, t4 = t + 4 < 64 ? t + 4 : 63;
            const unsigned g1 = G_GIDX(t + 1), g2 = G_GIDX(t2), g3 = G_GIDX(t3), g4 = G_GIDX(t4);
            const unsigned a1 = abase + g1 * 262144u, a2 = abase + g2 * 262144u, b2 = bbase + g2 * 262144u, a3 = abase + g3 * 262144u, b3 = bbase + g3 * 262144u;
            asm volatile("global_load_dword %0, %1, %2 sc0 sc1" : "=v"(fB) : "v"(zero), "s"(cbase + g3 * 128));
            G_LDB(B0, 0, 0); G_SCHED; G_LDA(At, 0, 0); G_STAGE(G_SA(1, 1), a1 + 16384);
            G_WAIT_L(8); G_BAR; G_WAIT_L(0); G_MMA(0, 0, At, B0); G_BAR; G_SCHED;
            asm volatile("" : "+v"(fA));
            if ((int)(rfl(fA) - target) < 0) wait_word((unsigned*)(cbase + g2 * 128), target, tmo, call);
            G_LDB(B1, 0, 1); G_STAGE(G_SB(0, 0), b2);
            G_BAR; G_WAIT_L(0); G_MMA(0, 1, At, B1); G_BAR;
            G_LDA(At, 0, 1); G_STAGE(G_SA(0, 0), a2);
            G_BAR; G_WAIT_L(0); G_MMA(1, 0, At, B0); G_BAR; G_SCHED;
            G_STAGE(G_SB(0, 1), b2 + 16384);
            G_WAIT_V(6); G_BAR; G_MMA(1, 1, At, B1); G_BAR;
            asm volatile("global_load_dword %0, %1, %2 sc0 sc1" : "=v"(fA) : "v"(zero), "s"(cbase + g4 * 128));
            G_LDB(B0, 1, 0); G_SCHED; G_LDA(At, 1, 0); G_STAGE(G_SA(0, 1), a2 + 16384);
            G_WAIT_L(8); G_BAR; G_WAIT_L(0); G_MMA(0, 0, At, B0); G_BAR; G_SCHED;
            asm volatile("" : "+v"(fB));
            if ((int)(rfl(fB) - target) < 0) wait_word((unsigned*)(cbase + g3 * 128), target, tmo, call);
            G_LDB(B1, 1, 1); G_STAGE(G_SB(1, 0), b3);
            G_BAR; G_WAIT_L(0); G_MMA(0, 1, At, B1); G_BAR;
            G_LDA(At, 1, 1); G_STAGE(G_SA(1, 0), a3);
            G_BAR; G_WAIT_L(0); G_MMA(1, 0, At, B0); G_BAR; G_SCHED;
            G_STAGE(G_SB(1, 1), b3 + 16384);
            G_WAIT_V(6); G_BAR; G_MMA(1, 1, At, B1); G_BAR;
        }
        G_WAIT_V(0);
        asm volatile("" : "+v"(fA));
        if (wr == 0) G_BAR;
        G_BAR;
        STAMP(ws, r * 8 + 2);
        int lane = threadIdx.x & 63;
        asm volatile("" : "+v"(lane));
        const int fr = lane & 15, fq = lane >> 4;
        if (h) {
#pragma unroll
            for (int b = 0; b < 2; ++b)
#pragma unroll
                for (int m = 0; m < 4; ++m)
#pragma unroll
                    for (int n = 0; n < 2; ++n) { const f32x4 tq = acc[0][b][m][n]; acc[0][b][m][n] = acc[1][b][m][n]; acc[1][b][m][n] = tq; }
        }
        const unsigned xl = (unsigned)((r * 64 + tile) * 2);
#pragma unroll
        for (int b = 0; b < 2; ++b)
#pragma unroll
            for (int m = 0; m < 4; ++m)
#pragma unroll
                for (int n = 0; n < 2; ++n)
                    __builtin_amdgcn_raw_buffer_store_b128(__builtin_bit_cast(u32x4, acc[1][b][m][n]), wsr, lane * 16 + ((b * 4 + m) * 2 + n) * 1024, OFF_XBUF + ((xl + h) * 8 + wid) * 16384u, AUX_SC1);
        WAIT_VM0();
        if (lane == 0) __hip_atomic_store((unsigned*)(ws + OFF_XFLAG + (xl + h) * 128 + wid * 16), call, __ATOMIC_RELAXED, __HIP_MEMORY_SCOPE_AGENT);
        wait_word((unsigned*)(ws + OFF_XFLAG + (xl + (h ^ 1)) * 128 + wid * 16), call, tmo, call);
        const bool dead = rfl(__hip_atomic_load(tmo, __ATOMIC_RELAXED, __HIP_MEMORY_SCOPE_AGENT)) == call;
        STAMP(ws, r * 8 + 3);
        LAS unsigned char* my = lds + wid * 8192;
#pragma unroll
        for (int b = 0; b < 2; ++b)
#pragma unroll
            for (int m = 0; m < 4; ++m) {
                const f32x4 p0 = __builtin_bit_cast(f32x4, __builtin_amdgcn_raw_buffer_load_b128(wsr, lane * 16 + ((b * 4 + m) * 2 + 0) * 1024, OFF_XBUF + ((xl + (h ^ 1)) * 8 + wid) * 16384u, AUX_SC1));
                const f32x4 p1 = __builtin_bit_cast(f32x4, __builtin_amdgcn_raw_buffer_load_b128(wsr, lane * 16 + ((b * 4 + m) * 2 + 1) * 1024, OFF_XBUF + ((xl + (h ^ 1)) * 8 + wid) * 16384u, AUX_SC1));
                const f32x4 v0 = acc[0][b][m][0] + p0, v1 = acc[0][b][m][1] + p1;
                u32x4 o;
                o.x = cvt_pk_bf16(gelu_f(v0[0]), gelu_f(v0[1])); o.y = cvt_pk_bf16(gelu_f(v0[2]), gelu_f(v0[3]));
                o.z = cvt_pk_bf16(gelu_f(v1[0]), gelu_f(v1[1])); o.w = cvt_pk_bf16(gelu_f(v1[2]), gelu_f(v1[3]));
                const int row = m * 16 + fr, ch = b * 4 + fq;
                *(LAS u32x4*)(my + row * 128 + ((ch ^ (row & 7)) * 16)) = o;
            }
        asm volatile("s_waitcnt lgkmcnt(0)" ::: "memory");
        u32x4 c[8];
#pragma unroll
        for (int j = 0; j < 8; ++j) { const int row = j * 8 + (lane >> 3); c[j] = *(LAS u32x4*)(my + row * 128 + (((lane & 7) ^ (row & 7)) * 16)); }
        if (dead) {
#pragma unroll
            for (int j = 0; j < 8; ++j) c[j] = (u32x4){0x7FC07FC0u, 0x7FC07FC0u, 0x7FC07FC0u, 0x7FC07FC0u};
        }
        {
            const rsrc_t outr = mk_rsrc(P.out, (unsigned)DIM * DIM * 2u);
            const unsigned row0 = (P.rank >> 2) * 4096 + r * 2048 + tm * 256 + h * 128 + wr * 64, col0 = (P.rank & 3) * 2048 + tn * 256 + wc * 64;
            const unsigned dso = (row0 * DIM + col0) * 2u, vo = (unsigned)(lane >> 3) * (DIM * 2u) + (unsigned)(lane & 7) * 16u;
            if (r == 0) {
#pragma unroll
                for (int j = 0; j < 8; ++j) __builtin_amdgcn_raw_buffer_store_b128(c[j], outr, vo, dso + j * (8u * DIM * 2u), AUX_SC1);
            } else {
#pragma unroll
                for (int j = 0; j < 8; ++j) __builtin_amdgcn_raw_buffer_store_b128(c[j], outr, vo, dso + j * (8u * DIM * 2u), OUT_AUX);
            }
            const unsigned o = (xl + h) * 8 + wid;
            if (r == 0) { WAIT_VM0(); signal_cnt(ws, OFF_OCNT + (o >> 3) * 128, lane); }
            else publish_block(P, wsr, c, 4096u + o, 1536u + o, 0xFFu & ~(1u << P.rank), call, lds, my, 1u + GROUNDS, wid, lane);
        }
        STAMP(ws, r * 8 + 4);
        asm volatile("s_waitcnt lgkmcnt(0)" ::: "memory");
        G_BAR;
    }
#undef G_SA
#undef G_SB
#undef G_STAGE
#undef G_LDA
#undef G_LDB
#undef G_MMA
#undef G_WAIT_V
#undef G_WAIT_L
#undef G_BAR
#undef G_SCHED
#undef G_GIDX
}

__global__ __launch_bounds__(512, 2) void gemm_allgather_kernel(Params P) {
    extern __shared__ __attribute__((aligned(16))) unsigned char shm[];
    LAS unsigned char* lds = (LAS unsigned char*)shm;
    char* ws = P.ws[P.rank];
    const int b = rfl(blockIdx.x);
    STAMP(ws, 60);
    unsigned call;
    if (b >= NPACK + NGEMM) { call = call_draw(ws); puller_role(P, lds, rfl(call), rfl(b - NPACK - NGEMM)); }
    else {
        if (threadIdx.x < 8) *(volatile LAS unsigned*)(lds + GCTL + 128 + threadIdx.x * 4) = 0u;
        __syncthreads();
        const int wid = rfl(threadIdx.x >> 6);
        const bool gemm = b >= NPACK;
        const int next = gemm ? NFIRST + (b - NPACK) * 8 + wid : NFIRST + GROUNDS * NGEMM * 8 + b * 8 + wid;
        call = packer_role(P, lds, wid < 4 ? b * 4 + wid : next, next, gemm ? NGEMM * 8 : NPACK * 8, gemm ? NFIRST + GROUNDS * NGEMM * 8 : 4096 + 1024);
        if (gemm) { __syncthreads(); gemm_role(P, lds, rfl(call), rfl(b)); }
    }
    STAMP(ws, 61);
    STAMP(ws, 58 + (call & 1));
}

}

extern "C" void kernel_launch(int rank, int nranks, ncclComm_t comm,
                              hipStream_t stream,
                              void* const* d_in, const int* in_sizes, int n_in,
                              void* d_out, int out_size,
                              const int* in_dtypes, int out_dtype,
                              void* const* d_out_all,
                              void* const* d_ws_all, size_t ws_bytes) {
    (void)comm; (void)in_sizes; (void)in_dtypes; (void)out_dtype; (void)d_out_all;
    if (nranks != gar::WORLD || n_in < 2 || ws_bytes < gar::WS_NEED || out_size != gar::DIM * gar::DIM || (unsigned)rank >= (unsigned)gar::WORLD) {
        fprintf(stderr, "unexpected problem shape\n");
        return;
    }
    static bool attr = false;
    if (!attr) {
        hipError_t e = hipFuncSetAttribute((const void*)gar::gemm_allgather_kernel, hipFuncAttributeMaxDynamicSharedMemorySize, gar::LDS_BYTES);
        if (e != hipSuccess) { fprintf(stderr, "hipFuncSetAttribute: %s\n", hipGetErrorString(e)); return; }
        attr = true;
    }
    gar::Params P;
    memset(&P, 0, sizeof P);
    P.x = d_in[0]; P.w = d_in[1]; P.out = d_out; P.rank = rank;
    for (int p = 0; p < gar::WORLD; ++p) P.ws[p] = (char*)d_ws_all[p];
    hipLaunchKernelGGL(gar::gemm_allgather_kernel, dim3(gar::GRID), dim3(512), gar::LDS_BYTES, stream, P);
    hipError_t e = hipGetLastError();
    if (e != hipSuccess) fprintf(stderr, "launch: %s\n", hipGetErrorString(e));
}
```

```cpp
#include <hip/hip_runtime.h>
#include <rccl/rccl.h>
#include <cstdio>
#include <cstdint>
#include <cstring>

namespace gar {

#define LAS __attribute__((address_space(3)))
typedef unsigned u32x4 __attribute__((ext_vector_type(4)));
typedef unsigned short u16x2 __attribute__((ext_vector_type(2)));
typedef short bf16x8 __attribute__((ext_vector_type(8)));
typedef float f32x4 __attribute__((ext_vector_type(4)));
typedef __amdgpu_buffer_rsrc_t rsrc_t;

constexpr int WORLD = 8;
constexpr int DIM = 8192;
constexpr int KSH = 1024;
constexpr int NGEMM = 128;
constexpr int GRID = 256;
#define PULL_WG_PER_LINK 12
#define PULL_DECODERS 7
constexpr int PWL = PULL_WG_PER_LINK;
constexpr int NDEC = PULL_DECODERS;
constexpr int NPULL = 7 * PWL;
constexpr int NPACK = GRID - NGEMM - NPULL;
#define NRAWEND 40
#define WARM_TLB 1
#define OUT_AUX 18
#define IMG_AUX 18
#define STG_AUX AUX_SYS
#define XIN_AUX 0
#define JIT_LEAD 30
#define PAY_AUX AUX_SYS
#define NRAW 48
constexpr int NFIRST = (NPACK + NGEMM) * 4;
#define GEMM_PACK_ROUNDS 1
constexpr int GROUNDS = GEMM_PACK_ROUNDS;
static_assert(NFIRST + GROUNDS * NGEMM * 8 <= 4096 && NFIRST % 8 == 0, "roles");
constexpr int NSEQ = 3584;
constexpr unsigned long long DEADLINE = 40000000ull;

constexpr unsigned OFF_TMO = 128;
constexpr unsigned OFF_LINK = 1024;
constexpr unsigned OFF_CNT = 4096;
constexpr unsigned OFF_XFLAG = OFF_CNT + 256 * 128;
constexpr unsigned OFF_OCNT = OFF_XFLAG + 256 * 128;
constexpr unsigned OFF_GRAN = 131072;
constexpr unsigned OFF_CALL = 640u << 10;
constexpr unsigned OFF_SLOT = 1u << 20;
constexpr unsigned OFF_AIMG = 64u << 20;
constexpr unsigned OFF_BIMG = 128u << 20;
constexpr unsigned OFF_XBUF = 160u << 20;
constexpr unsigned WS_USED = 192u << 20;
constexpr unsigned WS_NEED = WS_USED;
static_assert(OFF_OCNT + 128 * 128 <= OFF_GRAN, "map");
static_assert(OFF_GRAN + WORLD * NSEQ * 8 <= OFF_CALL && OFF_CALL + 2048 * 64 <= OFF_SLOT, "map");
static_assert(OFF_SLOT + 6144u * 8192u <= OFF_AIMG, "map");
constexpr int LDS_BYTES = 131072 + 512;

struct Params {
    const void* x; const void* w; void* out;
    char* ws[WORLD];
    int rank; int pad;
};

constexpr int AUX_SC1 = 16, AUX_SYS = 17;

__device__ __forceinline__ rsrc_t mk_rsrc(const void* p, unsigned bytes) {
    return __builtin_amdgcn_make_buffer_rsrc(const_cast<void*>(p), 0, bytes, 0x00020000);
}
__device__ __forceinline__ unsigned rfl(unsigned v) { return __builtin_amdgcn_readfirstlane(v); }
__device__ __forceinline__ unsigned bfi(unsigned mask, unsigned a, unsigned b) { return (a & mask) | (b & ~mask); }
__device__ __forceinline__ unsigned perm(unsigned hi, unsigned lo, unsigned sel) { return __builtin_amdgcn_perm(hi, lo, sel); }
#define WAIT_VM0() asm volatile("s_waitcnt vmcnt(0)" ::: "memory")

#define STAMP(ws, idx) do { } while (0)

__device__ __forceinline__ unsigned call_draw(char* ws) {
    unsigned t;
    const int zero = 0, one = 1;
    unsigned long long keep;
    asm volatile("s_mov_b64 %1, exec\n\ts_mov_b64 exec, 1\n\tglobal_atomic_add %0, %2, %3, %4 sc0\n\ts_mov_b64 exec, %1\n\ts_waitcnt vmcnt(0)"
                 : "=&v"(t), "=&s"(keep) : "v"(zero), "v"(one), "s"(ws + OFF_CALL + (blockIdx.x * 8 + rfl(threadIdx.x >> 6)) * 64) : "memory");
    return rfl(t) + 1u;
}

__device__ __forceinline__ bool give_up(unsigned long long t0, unsigned* tmo, unsigned call) {
    if (rfl(__hip_atomic_load(tmo, __ATOMIC_RELAXED, __HIP_MEMORY_SCOPE_AGENT)) == call) return true;
    if (__builtin_amdgcn_s_memrealtime() - t0 > DEADLINE) { __hip_atomic_store(tmo, call, __ATOMIC_RELAXED, __HIP_MEMORY_SCOPE_AGENT); return true; }
    return false;
}
__device__ __forceinline__ void wait_word(unsigned* p, unsigned target, unsigned* tmo, unsigned call) {
    if ((int)(rfl(__hip_atomic_load(p, __ATOMIC_RELAXED, __HIP_MEMORY_SCOPE_SYSTEM)) - target) >= 0) return;
    const unsigned long long t0 = __builtin_amdgcn_s_memrealtime();
    while ((int)(rfl(__hip_atomic_load(p, __ATOMIC_RELAXED, __HIP_MEMORY_SCOPE_SYSTEM)) - target) < 0) {
        if (give_up(t0, tmo, call)) break;
        __builtin_amdgcn_s_sleep(4);
    }
}

constexpr unsigned MAX_EXC = 31;
constexpr unsigned RAWB = 3584, AUX_U = 20, AUX_MAX = 2560, U_MAX = 10240, LANE_U_MAX = 224;
constexpr unsigned ENC_HIST = AUX_MAX + 16, ENC_SORT = ENC_HIST + 256, ENC_END = ENC_SORT + 128;

__device__ __forceinline__ unsigned bfi_c(unsigned mask, unsigned a, unsigned b) { unsigned r; asm("v_bfi_b32 %0, %1, %2, %3" : "=v"(r) : "s"(mask), "v"(a), "v"(b)); return r; }
__device__ __forceinline__ unsigned opaque(unsigned v) { asm("" : "+v"(v)); return v; }
__device__ __forceinline__ unsigned bytemask(unsigned x, int sh, unsigned c0c) {
    unsigned t; asm("v_and_or_b32 %0, %1, %2, %3" : "=v"(t) : "v"(x >> sh), "s"(0x01010101u), "v"(c0c));
    return perm(0u, 0u, t);
}
template <int CTRL, int RM = 0xf> __device__ __forceinline__ unsigned dpp0(unsigned v) { return (unsigned)__builtin_amdgcn_update_dpp(0, (int)v, CTRL, RM, 0xf, RM == 0xf); }
__device__ __forceinline__ unsigned wave_scan_incl(unsigned v) {
    v += dpp0<0x111>(v); v += dpp0<0x112>(v); v += dpp0<0x114>(v); v += dpp0<0x118>(v);
    v += dpp0<0x142, 0xa>(v); v += dpp0<0x143, 0xc>(v);
    return v;
}
__device__ __forceinline__ unsigned umin(unsigned a, unsigned b) { return a < b ? a : b; }
__device__ __forceinline__ unsigned umax(unsigned a, unsigned b) { return a > b ? a : b; }
__device__ __forceinline__ unsigned sad4(unsigned x) { return __builtin_amdgcn_sad_u8(x, 0u, 0u); }
__device__ __forceinline__ unsigned lut2x16(const unsigned (&T)[8], unsigned idx, unsigned ms, unsigned c0c) {
    const unsigned sel = idx & 0x07070707u, m8 = bytemask(idx, 3, c0c);
    const unsigned p = bfi(m8, perm(T[3], T[2], sel), perm(T[1], T[0], sel)), n = bfi(m8, perm(T[7], T[6], sel), perm(T[5], T[4], sel));
    return bfi(ms, n, p);
}
#define LDS_FENCE() asm volatile("" ::: "memory")
__device__ __forceinline__ void lds_or2(LAS unsigned char* scr, unsigned p, unsigned w) {
    const unsigned s = p & 31u;
    LAS unsigned* q = (LAS unsigned*)(scr + ((p >> 5) << 2));
    __hip_atomic_fetch_or(q, w << s, __ATOMIC_RELAXED, __HIP_MEMORY_SCOPE_WORKGROUP);
    __hip_atomic_fetch_or(q + 1, (w >> 1) >> (31u - s), __ATOMIC_RELAXED, __HIP_MEMORY_SCOPE_WORKGROUP);
}

__device__ __forceinline__ unsigned encode_block(const u32x4 (&c)[8], unsigned (&P)[14], u32x4 (&a)[3], LAS unsigned char* scr, int lane) {
    asm volatile("" : "+v"(lane));
    const unsigned c0c = opaque(0x0C0C0C0Cu);
    unsigned E[16], R[16];
    u16x2 p1 = {0, 0}, p2 = {0, 0}, n1 = {0, 0}, n2 = {0, 0};
#pragma unroll
    for (int g = 0; g < 16; ++g) {
        const unsigned x = (g & 1) ? c[g >> 1].z : c[g >> 1].x, y = (g & 1) ? c[g >> 1].w : c[g >> 1].y;
        const unsigned H = perm(y, x, 0x07050301u), L = perm(y, x, 0x06040200u);
        E[g] = bfi_c(0xFEFEFEFEu, H << 1, L >> 7);
        R[g] = bfi_c(0x80808080u, L << 1, bfi_c(0x40404040u, H >> 1, L));
        const unsigned en = E[g] & bytemask(H, 7, c0c), ep = E[g] ^ en;
        p1 = __builtin_elementwise_max(p1, __builtin_bit_cast(u16x2, ep)); p2 = __builtin_elementwise_max(p2, __builtin_bit_cast(u16x2, ep << 8));
        n1 = __builtin_elementwise_max(n1, __builtin_bit_cast(u16x2, en)); n2 = __builtin_elementwise_max(n2, __builtin_bit_cast(u16x2, en << 8));
        __builtin_amdgcn_sched_barrier(0);
    }
    const u32x4 z4 = {0u, 0u, 0u, 0u};
    *(LAS u32x4*)(scr + lane * 16) = z4; *(LAS u32x4*)(scr + 1024 + lane * 16) = z4;
    if (lane < 49) *(LAS u32x4*)(scr + 2048 + lane * 16) = z4;
    unsigned baseP, baseN;
    {
        const u16x2 mp = __builtin_elementwise_max(p1, p2), mn = __builtin_elementwise_max(n1, n2);
        u16x2 m = {(unsigned short)((mp.x > mp.y ? mp.x : mp.y) >> 8), (unsigned short)((mn.x > mn.y ? mn.x : mn.y) >> 8)};
        m = __builtin_elementwise_max(m, __builtin_bit_cast(u16x2, dpp0<0x128>(__builtin_bit_cast(unsigned, m)))); m = __builtin_elementwise_max(m, __builtin_bit_cast(u16x2, dpp0<0x124>(__builtin_bit_cast(unsigned, m))));
        m = __builtin_elementwise_max(m, __builtin_bit_cast(u16x2, dpp0<0x122>(__builtin_bit_cast(unsigned, m)))); m = __builtin_elementwise_max(m, __builtin_bit_cast(u16x2, dpp0<0x121>(__builtin_bit_cast(unsigned, m))));
        const unsigned mu = __builtin_bit_cast(unsigned, m);
        const u16x2 r01 = __builtin_elementwise_max(__builtin_bit_cast(u16x2, (unsigned)__builtin_amdgcn_readlane(mu, 0)), __builtin_bit_cast(u16x2, (unsigned)__builtin_amdgcn_readlane(mu, 16)));
        const u16x2 r23 = __builtin_elementwise_max(__builtin_bit_cast(u16x2, (unsigned)__builtin_amdgcn_readlane(mu, 32)), __builtin_bit_cast(u16x2, (unsigned)__builtin_amdgcn_readlane(mu, 48)));
        const unsigned mm = rfl(__builtin_bit_cast(unsigned, __builtin_elementwise_max(r01, r23)));
        baseP = mm & 0xFFu; baseN = mm >> 16;
    }
    const unsigned BP = baseP * 0x01010101u, BN = baseN * 0x01010101u;
    unsigned any = 0, D[16];
#pragma unroll
    for (int g = 0; g < 16; ++g) {
        E[g] = bfi(bytemask(R[g], 6, c0c), BN, BP) - E[g]; any |= E[g];
        D[g] = bfi_c(0x1E1E1E1Eu, E[g] << 1, (R[g] >> 7) & 0x01010101u);
        R[g] &= 0x7F7F7F7Fu;
    }
    LDS_FENCE();
#pragma unroll
    for (int gi = 0; gi < 4; ++gi) {
        const int g = gi * 5;
        const unsigned k4 = ((D[g] & 0x0F0F0F0Fu) | ((R[g] >> 2) & 0x10101010u) | ((D[g] << 1) & 0x20202020u)) << 2;
#pragma unroll
        for (int b = 0; b < 4; ++b)
            __hip_atomic_fetch_add((LAS unsigned*)(scr + ENC_HIST + ((k4 >> (8 * b)) & 0xFFu)), 1u, __ATOMIC_RELAXED, __HIP_MEMORY_SCOPE_WORKGROUP);
    }
    LDS_FENCE();
    const unsigned cnt = *(LAS unsigned*)(scr + ENC_HIST + lane * 4);
    const unsigned key = (cnt << 4) | (15u - (lane & 15));
    unsigned rank = 0;
    rank += dpp0<0x121>(key) > key; rank += dpp0<0x122>(key) > key; rank += dpp0<0x123>(key) > key; rank += dpp0<0x124>(key) > key; rank += dpp0<0x125>(key) > key;
    rank += dpp0<0x126>(key) > key; rank += dpp0<0x127>(key) > key; rank += dpp0<0x128>(key) > key; rank += dpp0<0x129>(key) > key; rank += dpp0<0x12A>(key) > key;
    rank += dpp0<0x12B>(key) > key; rank += dpp0<0x12C>(key) > key; rank += dpp0<0x12D>(key) > key; rank += dpp0<0x12E>(key) > key; rank += dpp0<0x12F>(key) > key;
    unsigned tail = cnt; tail += dpp0<0x128>(tail); tail += dpp0<0x124>(tail); tail += dpp0<0x122>(tail); tail += dpp0<0x121>(tail);
    const int sg = (lane >> 4) & 1;
    const unsigned ct = sg ? __builtin_amdgcn_readlane(tail, 48) : __builtin_amdgcn_readlane(tail, 32);
    if (lane < 32) *(LAS unsigned*)(scr + ENC_SORT + ((lane & 16) + rank) * 4) = cnt;
    LDS_FENCE();
    unsigned best;
    {
        const unsigned cb = lane & 15, s0 = cb / 3u, s1 = cb - 3u * s0, A = 1u << s0, AB = A + (1u << s1);
        unsigned cost = ct * (16u - AB);
#pragma unroll
        for (int q = 0; q < 4; ++q) {
            const u32x4 h = *(LAS u32x4*)(scr + ENC_SORT + sg * 64 + q * 16);
#pragma unroll
            for (int j = 0; j < 4; ++j) {
                const unsigned r = q * 4 + j, len = r < A ? 1u + s0 : r < AB ? 2u + s1 : 4u + ((r - AB) >> 1);
                cost += 2u * len * h[j];
            }
        }
        best = cb < 9 ? (cost << 4) | cb : 0xFFFFFFFFu;
        best = umin(best, dpp0<0x128>(best)); best = umin(best, dpp0<0x124>(best)); best = umin(best, dpp0<0x122>(best)); best = umin(best, dpp0<0x121>(best));
    }
    const unsigned cbP = __builtin_amdgcn_readlane(best, 0) & 15u, cbN = __builtin_amdgcn_readlane(best, 16) & 15u;
    const unsigned s0P = cbP / 3u, s1P = cbP - 3u * s0P, s0N = cbN / 3u, s1N = cbN - 3u * s0N;
    const unsigned ABP = (1u << s0P) + (1u << s1P), ABN = (1u << s0N) + (1u << s1N);
    unsigned F[8];
    {
        const unsigned s0 = sg ? s0N : s0P, s1 = sg ? s1N : s1P, A = 1u << s0, AB = sg ? ABN : ABP;
        const unsigned ent = rank < A ? (rank << 4) | (s0 << 6) : rank < AB ? 1u | ((rank - A) << 4) | (s1 << 6) : (2u + ((rank - AB) >> 1)) | (((rank - AB) & 1u) << 4) | (1u << 6);
        unsigned fv = ent << (8 * (lane & 3));
        fv |= dpp0<0xB1>(fv); fv |= dpp0<0x4E>(fv);
#pragma unroll
        for (int q = 0; q < 8; ++q) F[q] = __builtin_amdgcn_readlane(fv, q * 4);
        if (lane < 32)
            __hip_atomic_fetch_or((LAS unsigned*)(scr + (sg * 2 + (rank >> 3)) * 4), (unsigned)(lane & 15) << (8 * (rank & 3) + 4 * ((rank >> 2) & 1)), __ATOMIC_RELAXED, __HIP_MEMORY_SCOPE_WORKGROUP);
        if (lane == 0) *(LAS unsigned*)(scr + 16) = baseP | (baseN << 8) | (s0P << 16) | (s1P << 18) | (s0N << 20) | (s1N << 22);
    }
    const unsigned ABP4 = ABP * 0x01010101u, ABN4 = ABN * 0x01010101u, S0P4 = s0P * 0x01010101u, S0N4 = s0N * 0x01010101u;
    unsigned KK[16], XW[4], OX[4], TU = 0, TX = 0, bad = 0, xl = 0, xw = 0;
#pragma unroll
    for (int g = 0; g < 16; ++g) {
        const unsigned ms = bytemask(R[g], 6, c0c), m16 = bytemask(D[g], 4, c0c), ent = lut2x16(F, D[g], ms, c0c);
        const unsigned q = ((D[g] | 0x80808080u) - bfi(ms, ABN4, ABP4)) & 0x7F7F7F7Fu;
        unsigned K = bfi(m16, ((q >> 1) & 0x3F3F3F3Fu) + 0x02020202u, ent & 0x0F0F0F0Fu);
        unsigned SUF = bfi(m16, q & 0x01010101u, (ent >> 4) & 0x03030303u);
        unsigned S = bfi(m16, 0x01010101u, (ent >> 6) & 0x03030303u);
        unsigned gl = sad4(K) + 4u;
        if (__ballot(gl > 31u) != 0ull) {
            any |= 0x80u;
            const unsigned km = umax(umax(K & 0xFFu, (K >> 8) & 0xFFu), umax((K >> 16) & 0xFFu, K >> 24));
            const unsigned eq = gl > 31u ? (((K ^ (km * 0x01010101u)) + 0x7F7F7F7Fu) & 0x80808080u) ^ 0x80808080u : 0u, em = bytemask(eq, 7, c0c);
            E[g] |= eq;
            K &= ~em; SUF &= ~em; S = bfi(em, bfi(ms, S0N4, S0P4), S);
            gl = sad4(K) + 4u;
        }
        KK[g] = K; TU += gl; bad = umax(bad, gl);
        xw |= (SUF & 0xFFu) << xl; xl += S & 0xFFu;
        xw |= ((SUF >> 8) & 0xFFu) << xl; xl += (S >> 8) & 0xFFu;
        xw |= ((SUF >> 16) & 0xFFu) << xl; xl += (S >> 16) & 0xFFu;
        xw |= (SUF >> 24) << (xl & 31u); xl += S >> 24;
        if ((g & 3) == 3) { XW[g >> 2] = xw; OX[g >> 2] = TX; TX += xl; xw = 0; xl = 0; }
        __builtin_amdgcn_sched_barrier(0);
    }
    bad = (bad > 31u) | (TU > LANE_U_MAX);
    const unsigned tt = TU | (TX << 16), incl = wave_scan_incl(tt), tot = __builtin_amdgcn_readlane(incl, 63), totU = tot & 0xFFFFu, totX = tot >> 16;
    const unsigned xstart = (AUX_U * 8u + totU + 31u) & ~31u, endbit = xstart + totX;
    unsigned raw = (__ballot(bad != 0) != 0ull) || totU > U_MAX || endbit > (AUX_MAX - 8u) * 8u;
    if (!raw) {
        const unsigned px = xstart + ((incl - tt) >> 16);
        unsigned p = AUX_U * 8u + ((incl - tt) & 0xFFFFu);
#pragma unroll
        for (int g = 0; g < 16; ++g) {
            const unsigned ln = KK[g] + 0x01010101u;
            unsigned pos = (ln & 0xFFu) - 1u, w = 1u << pos;
            pos += (ln >> 8) & 0xFFu; w |= 1u << pos;
            pos += (ln >> 16) & 0xFFu; w |= 1u << pos;
            pos += ln >> 24; w |= 1u << pos;
            lds_or2(scr, p, w); p += pos + 1u;
            __builtin_amdgcn_sched_barrier(0);
        }
#pragma unroll
        for (int q = 0; q < 4; ++q) lds_or2(scr, px + OX[q], XW[q]);
    }
#pragma unroll
    for (int i = 0; i < 7; ++i) { P[i] = R[i] | ((R[14] << (7 - i)) & 0x80808080u); P[7 + i] = R[7 + i] | ((R[15] << (7 - i)) & 0x80808080u); }
    unsigned n = 0;
    const unsigned xo = (endbit + 31u) >> 5;
    if (!raw && __ballot((any & 0xF0F0F0F0u) != 0) != 0ull) {
#pragma unroll
        for (int g = 0; g < 16; ++g) {
            const unsigned f = E[g] & 0xF0F0F0F0u;
            unsigned long long mask = __ballot(f != 0);
            while (mask != 0ull && n <= MAX_EXC) {
                const int L = __builtin_ctzll(mask); mask &= mask - 1;
                const unsigned fl = __builtin_amdgcn_readlane(f, L);
                const unsigned lo = __builtin_amdgcn_readlane((g & 1) ? c[g >> 1].z : c[g >> 1].x, L);
                const unsigned hi = __builtin_amdgcn_readlane((g & 1) ? c[g >> 1].w : c[g >> 1].y, L);
                const bool e = lane < 4 && ((fl >> (8 * (lane & 3))) & 0xFFu) != 0;
                const unsigned long long em = __ballot(e);
                const unsigned slot = xo + n + __builtin_popcountll(em & ((1ull << lane) - 1ull));
                const unsigned val = (((lane & 2) ? hi : lo) >> (16 * (lane & 1))) & 0xFFFFu;
                if (e && slot * 4u < AUX_MAX) *(LAS unsigned*)(scr + slot * 4u) = (unsigned)(L * 64 + g * 4 + (lane & 3)) | (val << 12);
                n += __builtin_popcountll(em);
            }
        }
    }
    raw |= (n > MAX_EXC) | ((xo + n) * 4u > AUX_MAX);
    LDS_FENCE();
    a[0] = *(LAS u32x4*)(scr + lane * 16); a[1] = *(LAS u32x4*)(scr + 1024 + lane * 16); a[2] = *(LAS u32x4*)(scr + 2048 + (lane & 31) * 16);
    return ((xo + n + 3u) >> 2) | (n << 8) | (xo << 16) | (raw << 31);
}

__device__ __forceinline__ void decode_block(LAS unsigned char* src, LAS unsigned char* pos_scr, u32x4 (&c)[8], int lane) {
    LAS unsigned char* aux = src + RAWB;
    const unsigned c0c = opaque(0x0C0C0C0Cu);
    const u32x4 nt = *(LAS const u32x4*)(aux);
    const unsigned prm_v = *(LAS const unsigned*)(aux + 16);
    unsigned u[5];
#pragma unroll
    for (int i = 0; i < 5; ++i) u[i] = *(LAS const unsigned*)(aux + AUX_U + lane * 20 + i * 4);
    u32x4 P4[3];
#pragma unroll
    for (int p = 0; p < 3; ++p) P4[p] = *(LAS const u32x4*)(src + p * 1024 + lane * 16);
    const unsigned long long ph = *(LAS const unsigned long long*)(src + 3072 + lane * 8);
    const unsigned c0 = __builtin_popcount(u[0]), c1 = c0 + __builtin_popcount(u[1]), c2 = c1 + __builtin_popcount(u[2]), c3 = c2 + __builtin_popcount(u[3]), cn = c3 + __builtin_popcount(u[4]);
    const unsigned C = wave_scan_incl(cn) - cn;
    if (lane == 0) *(LAS unsigned*)(pos_scr) = AUX_U * 8u;
    const int rounds = __ballot(cn > 128u) != 0ull ? 3 : 2;
    for (int j = 0; j < rounds; ++j) {
        const unsigned t = (C >> 6) + 1u + j, kth = 64u * t - C;
        const unsigned i = (kth > c0) + (kth > c1) + (kth > c2) + (kth > c3);
        unsigned rem = kth - (i == 0 ? 0u : i == 1 ? c0 : i == 2 ? c1 : i == 3 ? c2 : c3), w = i == 0 ? u[0] : i == 1 ? u[1] : i == 2 ? u[2] : i == 3 ? u[3] : u[4], pos = 0;
#pragma unroll
        for (int st = 16; st >= 1; st >>= 1) {
            const unsigned lo = __builtin_popcount(w & ((1u << st) - 1u));
            const bool up = rem > lo;
            rem -= up ? lo : 0u; w = up ? w >> st : w; pos += up ? st : 0;
        }
        if (kth <= cn && t < 64u) *(LAS unsigned*)(pos_scr + t * 4) = AUX_U * 8u + lane * 160u + i * 32u + pos + 1u;
    }
    LDS_FENCE();
    const unsigned p0 = *(LAS const unsigned*)(pos_scr + lane * 4);
    unsigned S[8];
    {
        LAS const unsigned* q = (LAS const unsigned*)(aux + ((p0 >> 5) << 2));
#pragma unroll
        for (int i = 0; i < 8; ++i) S[i] = q[i];
#pragma unroll
        for (int i = 0; i < 7; ++i) S[i] = __builtin_amdgcn_alignbit(S[i + 1], S[i], p0 & 31u);
    }
    unsigned R[16];
    {
        const unsigned P[14] = {P4[0].x, P4[0].y, P4[0].z, P4[0].w, P4[1].x, P4[1].y, P4[1].z, P4[1].w, P4[2].x, P4[2].y, P4[2].z, P4[2].w, (unsigned)ph, (unsigned)(ph >> 32)};
        unsigned x = 0, y = 0;
#pragma unroll
        for (int i = 0; i < 7; ++i) { x |= (P[i] >> (7 - i)) & (0x01010101u << i); y |= (P[7 + i] >> (7 - i)) & (0x01010101u << i); }
#pragma unroll
        for (int i = 0; i < 14; ++i) R[i] = P[i] & 0x7F7F7F7Fu;
        R[14] = x; R[15] = y;
    }
    const unsigned n0 = rfl(nt.x), n1 = rfl(nt.y), n2 = rfl(nt.z), n3 = rfl(nt.w), prm = rfl(prm_v);
    const unsigned I[8] = {n0 & 0x0F0F0F0Fu, (n0 >> 4) & 0x0F0F0F0Fu, n1 & 0x0F0F0F0Fu, (n1 >> 4) & 0x0F0F0F0Fu, n2 & 0x0F0F0F0Fu, (n2 >> 4) & 0x0F0F0F0Fu, n3 & 0x0F0F0F0Fu, (n3 >> 4) & 0x0F0F0F0Fu};
    const unsigned s0P = (prm >> 16) & 3u, s1P = (prm >> 18) & 3u, s0N = (prm >> 20) & 3u, s1N = (prm >> 22) & 3u;
    const unsigned STP = (s0P << 8) | (s1P << 16) | (1u << 24), STN = (s0N << 8) | (s1N << 16) | (1u << 24);
    const unsigned OTP = (4u << 8) | (((1u << s0P) + 2u) << 16) | (((1u << s0P) + (1u << s1P)) << 24);
    const unsigned OTN = (4u << 8) | (((1u << s0N) + 2u) << 16) | (((1u << s0N) + (1u << s1N)) << 24);
    const unsigned BP = (prm & 0xFFu) * 0x01010101u, BN = ((prm >> 8) & 0xFFu) * 0x01010101u;
    unsigned RB[16], SS[16], TU = 0, TX = 0;
#pragma unroll
    for (int g = 0; g < 16; ++g) {
        const unsigned w0 = S[0], q0 = __builtin_ctz(w0), w1 = w0 & (w0 - 1u), q1 = __builtin_ctz(w1), w2 = w1 & (w1 - 1u), q2 = __builtin_ctz(w2), w3 = w2 & (w2 - 1u), q3 = __builtin_ctz(w3);
        const unsigned Q = q0 | (q1 << 8) | (q2 << 16) | (q3 << 24);
        const unsigned LN = Q - opaque((Q << 8) - 1u), sh = q3 + 1u;
        TU += sh;
        if (g < 15) {
#pragma unroll
            for (int i = 0; i < 7; ++i) if (32 * i < (int)LANE_U_MAX - 4 * (g + 1)) S[i] = (i < 6 && 32 * (i + 1) < (int)LANE_U_MAX - 4 * g) ? __builtin_amdgcn_alignbit(S[i + 1], S[i], sh) : S[i] >> sh;
        }
        const unsigned sel = bfi_c(0x04040404u, R[g] >> 4, bfi(bytemask(LN + 0x7D7D7D7Du, 7, c0c), 0x03030303u, LN));
        SS[g] = perm(STN, STP, sel);
        RB[g] = perm(OTN, OTP, sel) + (LN << 1);
        TX = __builtin_amdgcn_sad_u8(SS[g], 0u, TX);
    }
    const unsigned xstart = (__builtin_amdgcn_readlane(p0 + TU, 63) + 31u) & ~31u, px = xstart + wave_scan_incl(TX) - TX;
    unsigned X[5];
    {
        LAS const unsigned* q = (LAS const unsigned*)(aux + ((px >> 5) << 2));
#pragma unroll
        for (int i = 0; i < 5; ++i) X[i] = q[i];
#pragma unroll
        for (int i = 0; i < 4; ++i) X[i] = __builtin_amdgcn_alignbit(X[i + 1], X[i], px & 31u);
    }
    unsigned xoff = 0;
#pragma unroll
    for (int g = 0; g < 16; ++g) {
        const unsigned s = SS[g], M = perm(0u, 0x00030100u, s);
        const unsigned y = (g & 1) ? X[0] >> xoff : X[0], Y = perm(y, y, 0x01000100u);
        const unsigned t1 = opaque(s << 8), t2 = opaque((t1 << 8) + t1), Cs = (t2 << 16) + t2;
        const u16x2 z02 = __builtin_bit_cast(u16x2, Y) >> __builtin_bit_cast(u16x2, Cs & 0x00FF00FFu);
        const u16x2 z13 = __builtin_bit_cast(u16x2, Y) >> __builtin_bit_cast(u16x2, perm(0u, Cs, 0x0C030C01u));
        const unsigned suf = perm(__builtin_bit_cast(unsigned, z13), __builtin_bit_cast(unsigned, z02), 0x06020400u) & M;
        if (g & 1) {
            if (g < 15) {
                const unsigned t = __builtin_amdgcn_sad_u8(s, 0u, xoff);
#pragma unroll
                for (int i = 0; i < 4; ++i) if (32 * i < 8 * (15 - g)) X[i] = (i < 3 && 32 * (i + 1) < 8 * (17 - g)) ? __builtin_amdgcn_alignbit(X[i + 1], X[i], t) : X[i] >> t;
            }
        } else xoff = sad4(s);
        const unsigned rank = RB[g] + suf - 0x06060606u, ms = bytemask(R[g], 6, c0c);
        const unsigned D = bfi(bytemask(rank, 4, c0c), rank, lut2x16(I, rank, ms, c0c));
        const unsigned E = bfi(ms, BN, BP) - ((D >> 1) & 0x0F0F0F0Fu);
        const unsigned H = bfi_c(0x80808080u, R[g] << 1, E >> 1), L = bfi_c(0x3F3F3F3Fu, R[g], bfi_c(0x40404040u, D << 6, E << 7));
        const unsigned x = perm(H, L, 0x05010400u), yv = perm(H, L, 0x07030602u);
        if (g & 1) { c[g >> 1].z = x; c[g >> 1].w = yv; } else { c[g >> 1].x = x; c[g >> 1].y = yv; }
    }
}

typedef unsigned u32x2 __attribute__((ext_vector_type(2)));
__device__ __forceinline__ void store_granule(char* p, u32x2 g) {
    asm volatile("global_store_dwordx2 %0, %1, off sc0 sc1\n\ts_nop 1" :: "v"(p), "v"(g) : "memory");
}

constexpr unsigned GCTL = 131072;
__device__ __forceinline__ unsigned payload_bytes(unsigned h) { return (h >> 31) ? 8192u : RAWB + ((h & 255u) << 4); }
__device__ __forceinline__ void publish_block(const Params& P, rsrc_t wsr, const u32x4 (&c)[8], unsigned slot, unsigned n, unsigned dmask, unsigned call, LAS unsigned char* lds, LAS unsigned char* scr,
                                              unsigned j, int wid, int lane) {
    unsigned R[14], h = 1u << 31; u32x4 a[3];
    if (n >= NRAW && n < NSEQ - NRAWEND) h = encode_block(c, R, a, scr, lane);
    const unsigned u = j & 3u, gsize = j == 0 ? 4u : 8u;
    volatile LAS unsigned* ctl = (volatile LAS unsigned*)(lds + GCTL);
    while (j >= 4u && (int)(ctl[36 + u] - (j - 3u)) < 0) __builtin_amdgcn_s_sleep(1);
    unsigned so = OFF_SLOT + (slot & ~1u) * 8192u;
    if (wid & 1) {
        unsigned v;
        while (((v = ctl[40 + u * 8 + wid - 1]) >> 16) != ((j + 1u) & 0xFFFFu)) __builtin_amdgcn_s_sleep(1);
        so += (rfl(v) & 0xFFFFu) << 4;
    } else if (lane == 0) ctl[40 + u * 8 + wid] = ((j + 1u) << 16) | (payload_bytes(h) >> 4);
    if (h >> 31) {
#pragma unroll
        for (int q = 0; q < 8; ++q) __builtin_amdgcn_raw_buffer_store_b128(c[q], wsr, lane * 16 + q * 1024, so, PAY_AUX);
    } else {
        const unsigned n16 = h & 255u;
#pragma unroll
        for (int q = 0; q < 3; ++q) __builtin_amdgcn_raw_buffer_store_b128((u32x4){R[4 * q], R[4 * q + 1], R[4 * q + 2], R[4 * q + 3]}, wsr, lane * 16 + q * 1024, so, PAY_AUX);
        __builtin_amdgcn_raw_buffer_store_b64((u32x2){R[12], R[13]}, wsr, lane * 8 + 3072, so, PAY_AUX);
        if ((unsigned)lane < n16) __builtin_amdgcn_raw_buffer_store_b128(a[0], wsr, lane * 16 + RAWB, so, PAY_AUX);
        if ((unsigned)lane + 64u < n16) __builtin_amdgcn_raw_buffer_store_b128(a[1], wsr, lane * 16 + RAWB + 1024, so, PAY_AUX);
        if ((unsigned)lane + 128u < n16) __builtin_amdgcn_raw_buffer_store_b128(a[2], wsr, lane * 16 + RAWB + 2048, so, PAY_AUX);
    }
    WAIT_VM0();
    unsigned arrived = 0;
    if (lane == 0) {
        ctl[u * 8 + wid] = h;
        arrived = __hip_atomic_fetch_add((LAS unsigned*)(lds + GCTL) + 32 + u, 1u, __ATOMIC_RELAXED, __HIP_MEMORY_SCOPE_WORKGROUP);
    }
    if (rfl(arrived) == gsize - 1u) {
        const unsigned d = lane >> 3, w = lane & 7, hw = ctl[u * 8 + w];
        if (w < gsize && ((dmask >> d) & 1u)) store_granule(P.ws[d] + OFF_GRAN + ((unsigned)P.rank * NSEQ + n - wid + w) * 8u, (u32x2){call, hw});
        if (lane == 0) { ctl[32 + u] = 0u; ctl[36 + u] = j + 1u; }
    }
}

__device__ __forceinline__ void signal_cnt(char* ws, unsigned off, int lane) {
    if (off && lane == 0) __hip_atomic_fetch_add((unsigned*)(ws + off), 1u, __ATOMIC_RELAXED, __HIP_MEMORY_SCOPE_SYSTEM);
}
__device__ __forceinline__ unsigned packer_role(const Params& P, LAS unsigned char* lds, int first, int next, int step, int end) {
    unsigned call = 0;
    const int lane = threadIdx.x & 63, wid = rfl(threadIdx.x >> 6);
    char* ws = P.ws[P.rank];
    const rsrc_t wsr = mk_rsrc(ws, WS_USED);
    const rsrc_t xr = mk_rsrc(P.x, DIM * KSH * 2), wr = mk_rsrc(P.w, DIM * KSH * 2);
    const int myi = P.rank >> 2, myj = P.rank & 3;
    LAS unsigned char* my = lds + wid * 8192;
    unsigned j = first < NFIRST ? 0u : 1u;
    const unsigned long long tstart = __builtin_amdgcn_s_memrealtime();
    for (int p = first; p < end; p = p < NFIRST ? next : p + step, ++j) {
        if (p < 4096) {
            const int due = (p < 3072 ? (p / 192) * 580 : 9280 + ((p - 3072) >> 6) * 290) - JIT_LEAD * 100;
            while ((int)(__builtin_amdgcn_s_memrealtime() - tstart) < due) __builtin_amdgcn_s_sleep(16);
        }
        u32x4 c[8];
        unsigned slot, n, dmask, selfcnt = 0;
        if (p >= 4096) {
            const unsigned o = p - 4096, w = o & 7, h = (o >> 3) & 1, tile = (o >> 4) & 63;
            if (call == 0) call = call_draw(ws);
            unsigned* tmo = (unsigned*)(ws + OFF_TMO);
            wait_word((unsigned*)(ws + OFF_OCNT + (o >> 3) * 128), call * 8u, tmo, call);
            const unsigned row0 = (P.rank >> 2) * 4096 + (tile >> 3) * 256 + h * 128 + (w >> 2) * 64, col0 = (P.rank & 3) * 2048 + (tile & 7) * 256 + (w & 3) * 64;
            const rsrc_t outr = mk_rsrc(P.out, (unsigned)DIM * DIM * 2u);
#pragma unroll
            for (int q = 0; q < 8; ++q) c[q] = __builtin_amdgcn_raw_buffer_load_b128(outr, (unsigned)(lane >> 3) * (DIM * 2u) + (unsigned)(lane & 7) * 16u, (row0 * DIM + col0) * 2u + q * (8u * DIM * 2u), AUX_SYS);
            if (rfl(__hip_atomic_load(tmo, __ATOMIC_RELAXED, __HIP_MEMORY_SCOPE_AGENT)) == call) {
#pragma unroll
                for (int q = 0; q < 8; ++q) c[q] = (u32x4){0x7FC07FC0u, 0x7FC07FC0u, 0x7FC07FC0u, 0x7FC07FC0u};
            }
            slot = 4096u + o; n = 1536u + o; dmask = 0xFFu;
        } else {
            bool is_w; int kk, sel, r, bi;
            if (p < 3072) {
                kk = p / 192; const int rem = p - kk * 192;
                bi = rem & 31; r = 0; is_w = rem >= 64; sel = is_w ? ((rem - 64) >> 5) : (rem >> 5);
            } else { const int q = p - 3072; kk = q >> 6; sel = (q >> 5) & 1; bi = q & 31; r = 1; is_w = false; }
            if (!is_w) {
                const int rr = lane >> 3;
                const unsigned vo = (unsigned)rr * 2048u + (unsigned)(((lane & 7) ^ rr) * 16);
                const unsigned so = (unsigned)(sel * 4096 + r * 2048 + bi * 64) * 2048u + (unsigned)kk * 128u;
#pragma unroll
                for (int q = 0; q < 8; ++q) c[q] = __builtin_amdgcn_raw_buffer_load_b128(xr, vo + q * 8 * 2048, so, XIN_AUX);
            } else {
                const int pn = bi >> 2, bj = (bi >> 1) & 1, q = bi & 1;
                const int gk = lane >> 3, cc = lane & 7, wcl = cc >> 2, cq = cc & 3;
                const unsigned vo = (unsigned)(gk * 8) * 16384u + (unsigned)((64 * (2 * q + wcl) + 32 * bj + 8 * cq) * 2);
                const unsigned so = (unsigned)(kk * 64) * 16384u + (unsigned)((sel * 2048 + pn * 256) * 2);
                u32x4 R[8];
#pragma unroll
                for (int jj = 0; jj < 8; ++jj) R[jj] = __builtin_amdgcn_raw_buffer_load_b128(wr, vo + jj * 16384, so, XIN_AUX);
#pragma unroll
                for (int t = 0; t < 8; ++t) {
                    const unsigned s = (t & 1) ? 0x07060302u : 0x05040100u;
                    u32x4 v;
                    v.x = perm(R[1][t >> 1], R[0][t >> 1], s); v.y = perm(R[3][t >> 1], R[2][t >> 1], s);
                    v.z = perm(R[5][t >> 1], R[4][t >> 1], s); v.w = perm(R[7][t >> 1], R[6][t >> 1], s);
                    const int s64 = 32 * wcl + 16 * (t >> 2) + 4 * cq + (t & 3);
                    *(LAS u32x4*)(my + s64 * 128 + ((gk ^ (s64 & 7)) * 16)) = v;
                }
                asm volatile("s_waitcnt lgkmcnt(0)" ::: "memory");
#pragma unroll
                for (int q2 = 0; q2 < 8; ++q2) c[q2] = *(LAS u32x4*)(my + q2 * 1024 + lane * 16);
                asm volatile("s_waitcnt lgkmcnt(0)" ::: "memory");
            }
            if (call == 0) call = call_draw(ws);
            const int g = P.rank * 16 + kk;
            if (is_w ? (sel == myj) : (sel == myi)) {
                const unsigned so = is_w ? OFF_BIMG + (unsigned)(g * 32 + bi) * 8192u : OFF_AIMG + (unsigned)((r * 128 + g) * 32 + bi) * 8192u;
#pragma unroll
                for (int q = 0; q < 8; ++q) __builtin_amdgcn_raw_buffer_store_b128(c[q], wsr, lane * 16 + q * 1024, so, AUX_SC1);
                selfcnt = OFF_CNT + (r * 128 + g) * 128;
            }
            slot = is_w ? 2048u + (unsigned)((sel * 16 + kk) * 32 + bi) : (unsigned)(((sel * 2 + r) * 16 + kk) * 32 + bi);
            n = is_w ? (unsigned)(kk * 64 + 32 + bi) : (r == 0 ? (unsigned)(kk * 64 + bi) : (unsigned)(1024 + kk * 32 + bi));
            dmask = is_w ? (0x11u << sel) : (0x0Fu << (4 * sel));
        }
        publish_block(P, wsr, c, slot, n, dmask & ~(1u << P.rank), call, lds, my, j, wid, lane);
        signal_cnt(ws, selfcnt, lane);
    }
    if (call == 0) call = call_draw(ws);
    return call;
}

struct Blk { unsigned slot, dso, cnt; bool is_out; };
__device__ __forceinline__ Blk locate(unsigned n, int src, int myi, int myj) {
    Blk b; b.is_out = false;
    if (n < 1024) { const int kk = n >> 6, bi = n & 31, g = src * 16 + kk; b.cnt = OFF_CNT + g * 128;
        if ((n & 63) >= 32) { b.slot = 2048u + (myj * 16 + kk) * 32 + bi; b.dso = OFF_BIMG + (unsigned)(g * 32 + bi) * 8192u; }
        else { b.slot = ((myi * 2) * 16 + kk) * 32 + bi; b.dso = OFF_AIMG + (unsigned)(g * 32 + bi) * 8192u; } }
    else if (n < 1536) { const int kk = (n - 1024) >> 5, bi = n & 31, g = src * 16 + kk; b.cnt = OFF_CNT + (128 + g) * 128;
        b.slot = ((myi * 2 + 1) * 16 + kk) * 32 + bi; b.dso = OFF_AIMG + (unsigned)((128 + g) * 32 + bi) * 8192u; }
    else { const int o = n - 1536, w = o & 7, h = (o >> 3) & 1, tile = (o >> 4) & 63, r = o >> 10; b.cnt = 0; b.is_out = true;
        b.slot = 4096u + o;
        const unsigned row0 = (src >> 2) * 4096 + r * 2048 + (tile >> 3) * 256 + h * 128 + (w >> 2) * 64, col0 = (src & 3) * 2048 + (tile & 7) * 256 + (w & 3) * 64;
        b.dso = (row0 * DIM + col0) * 2u; }
    return b;
}
constexpr unsigned NPAIR = NSEQ / 2, NSLOT = 7, CTL = NSLOT * 16384;
constexpr unsigned M_DEAD = 1u << 30, N_STOP = 0xFFFFFFFFu;
__device__ __forceinline__ unsigned lds_ld(LAS unsigned char* p) { return *(volatile LAS unsigned*)p; }
__device__ __forceinline__ void lds_st(LAS unsigned char* p, unsigned v) { *(volatile LAS unsigned*)p = v; }
__device__ __forceinline__ void ring_publish(LAS unsigned char* lds, unsigned i, unsigned n, unsigned ha, unsigned hb, int lane) {
    const unsigned sl = i % NSLOT;
    if (lane == 0) { *(volatile LAS u32x4*)(lds + CTL + sl * 16) = (u32x4){n, ha, hb, 0u}; lds_st(lds + CTL + 128 + sl * 4, i + 1u); }
}
__device__ __forceinline__ bool slot_busy(LAS unsigned char* lds, unsigned i) { return i >= NSLOT && (int)(lds_ld(lds + CTL + 160 + (i % NSLOT) * 4) - 2u * (i / NSLOT)) < 0; }
__device__ __forceinline__ void dma_op(rsrc_t r, LAS unsigned char* dst, unsigned j, unsigned so, int lane) {
    __builtin_amdgcn_raw_ptr_buffer_load_lds(r, (LAS void*)(dst + j * 1024), 16, lane * 16, so + j * 1024, 0, AUX_SYS);
}
__device__ __forceinline__ u32x4 load_pair(rsrc_t wsr, unsigned off) { return __builtin_amdgcn_raw_buffer_load_b128(wsr, 0, off, AUX_SYS); }
#define VM_WAIT_(n) asm volatile("s_waitcnt vmcnt(" #n ")" ::: "memory")
#define VM_WAIT(n) VM_WAIT_(n)
#define G_STR_(x) #x
#define G_STR(x) G_STR_(x)
#define WIN 5

__device__ __forceinline__ void loader_wave(const Params& P, LAS unsigned char* lds, unsigned call, int src, int lane) {
    char* ws = P.ws[P.rank];
    unsigned* tmo = (unsigned*)(ws + OFF_TMO);
    const rsrc_t wsr = mk_rsrc(ws, WS_USED);
    char* const remp = P.ws[src];
    const int myi = P.rank >> 2, myj = P.rank & 3;
    const unsigned gbase = OFF_GRAN + (unsigned)src * NSEQ * 8u;
    char* const gptr = ws + gbase;
    const unsigned tko = OFF_LINK + (unsigned)src * 128u, tbase = (call - 1u) * (NPAIR + 3u * PWL);
    __builtin_amdgcn_s_setprio(3);
    unsigned q, q1, q2;
    {
        unsigned t0 = 0;
        if (lane == 0) t0 = __hip_atomic_fetch_add((unsigned*)(ws + tko), 3u, __ATOMIC_RELAXED, __HIP_MEMORY_SCOPE_AGENT);
        q = rfl(t0) - tbase; q1 = q + 1; q2 = q + 2;
    }
    {
        const rsrc_t rw = mk_rsrc(remp, OFF_AIMG);
        const unsigned v = __builtin_amdgcn_raw_buffer_load_b32(rw, lane < 25 ? OFF_SLOT + (unsigned)lane * (2u << 20) + 8188u - (lane == 24 ? 8192u : 0u) : 0xFFFFFFF0u, 0, AUX_SYS);
        asm volatile("" :: "v"(v));
    }
    unsigned gca, gha, gcb, ghb, hca, hha, hcb, hhb;
    { const u32x4 a = load_pair(wsr, gbase + (q < NPAIR ? q : 0u) * 16u), b = load_pair(wsr, gbase + (q1 < NPAIR ? q1 : 0u) * 16u);
      gca = rfl(a.x); gha = rfl(a.y); gcb = rfl(a.z); ghb = rfl(a.w); hca = rfl(b.x); hha = rfl(b.y); hcb = rfl(b.z); hhb = rfl(b.w); }
    const int zero = 0, one = 1;
    unsigned i = 0;
    bool pend = false; unsigned pn = 0, pa = 0, pb = 0;
    while (q < NPAIR) {
        bool dead = false;
        if ((int)(gca - call) < 0 || (int)(gcb - call) < 0 || slot_busy(lds, i)) {
            if (pend) { VM_WAIT(0); ring_publish(lds, i - 1u, pn, pa, pb, lane); pend = false; }
            const unsigned long long t0 = __builtin_amdgcn_s_memrealtime();
            while ((int)(gca - call) < 0 || (int)(gcb - call) < 0) {
                const u32x4 a = load_pair(wsr, gbase + q * 16u);
                gca = rfl(a.x); gha = rfl(a.y); gcb = rfl(a.z); ghb = rfl(a.w);
                if ((int)(gca - call) >= 0 && (int)(gcb - call) >= 0) break;
                if (give_up(t0, tmo, call)) { dead = true; break; }
                __builtin_amdgcn_s_sleep(2);
                asm volatile("" ::: "memory");
            }
            while (slot_busy(lds, i)) __builtin_amdgcn_s_sleep(1);
        }
        const unsigned n = 2u * q;
        const unsigned so = OFF_SLOT + locate(n, src, myi, myj).slot * 8192u;
        const unsigned bytes = dead ? 0u : payload_bytes(gha) + payload_bytes(ghb), nops = (bytes + 1023u) >> 10;
        const rsrc_t rem = mk_rsrc(remp, so + bytes);
        LAS unsigned char* dst = lds + (i % NSLOT) * 16384;
        u32x4 g2; unsigned tk = 0;
        asm volatile("global_load_dwordx4 %0, %1, %2 sc0 sc1" : "=v"(g2) : "v"(zero), "s"(gptr + (q2 < NPAIR ? q2 : 0u) * 16u) : "memory");
        if (lane == 0) asm volatile("global_atomic_add %0, %1, %2, %3 sc0" : "=v"(tk) : "v"(zero), "v"(one), "s"(ws + tko) : "memory");
#define G_FILL(jj) do { if ((jj) == WIN) asm volatile("s_waitcnt vmcnt(" G_STR(WIN) ")" : "+v"(g2), "+v"(tk) :: "memory"); else VM_WAIT(WIN); \
                        if ((jj) == WIN - 2 && pend) ring_publish(lds, i - 1u, pn, pa, pb, lane); dma_op(rem, dst, jj, so, lane); } while (0)
        G_FILL(0); G_FILL(1); G_FILL(2); G_FILL(3); G_FILL(4); G_FILL(5); G_FILL(6); G_FILL(7);
#undef G_FILL
        for (unsigned j = 8; j < nops; ++j) { VM_WAIT(WIN); dma_op(rem, dst, j, so, lane); }
        pend = true; pn = n; pa = gha | (dead ? M_DEAD : 0u); pb = ghb | (dead ? M_DEAD : 0u);
        q = q1; gca = hca; gha = hha; gcb = hcb; ghb = hhb;
        q1 = q2; hca = rfl(g2.x); hha = rfl(g2.y); hcb = rfl(g2.z); hhb = rfl(g2.w);
        q2 = rfl(tk) - tbase;
        ++i;
    }
    VM_WAIT(0);
    if (pend) ring_publish(lds, i - 1u, pn, pa, pb, lane);
    for (int d = 0; d < (NDEC + 1) / 2 + 1; ++d, ++i) {
        while (slot_busy(lds, i)) __builtin_amdgcn_s_sleep(1);
        ring_publish(lds, i, N_STOP, 0u, 0u, lane);
    }
}

__device__ __forceinline__ void decoder_wave(const Params& P, LAS unsigned char* lds, unsigned call, int src, int d, int lane) {
    char* ws = P.ws[P.rank];
    unsigned* tmo = (unsigned*)(ws + OFF_TMO);
    const rsrc_t wsr = mk_rsrc(ws, WS_USED), outr = mk_rsrc(P.out, (unsigned)DIM * DIM * 2u);
    const int myi = P.rank >> 2, myj = P.rank & 3;
    for (unsigned i = d;; i += NDEC) {
        const unsigned pi = i >> 1, sl = pi % NSLOT;
        while (lds_ld(lds + CTL + 128 + sl * 4) != pi + 1u) __builtin_amdgcn_s_sleep(1);
        asm volatile("" ::: "memory");
        const u32x4 meta = *(volatile LAS u32x4*)(lds + CTL + sl * 16);
        const unsigned n0 = rfl(meta.x), ha = rfl(meta.y), h = (i & 1) ? rfl(meta.z) : ha;
        if (n0 == N_STOP) break;
        const Blk b = locate(n0 + (i & 1), src, myi, myj);
        const bool raw = (h >> 31) != 0;
        bool dead = (h & M_DEAD) != 0;
        if (b.is_out && rfl(__hip_atomic_load(tmo, __ATOMIC_RELAXED, __HIP_MEMORY_SCOPE_AGENT)) == call) dead = true;
        u32x4 c[8];
        unsigned recs = 0;
        const unsigned ne = (dead || raw) ? 0u : (h >> 8) & 63u;
        LAS unsigned char* cur = lds + sl * 16384 + ((i & 1) ? payload_bytes(ha) : 0u);
        if (dead) {
#pragma unroll
            for (int j = 0; j < 8; ++j) c[j] = (u32x4){0x7FC07FC0u, 0x7FC07FC0u, 0x7FC07FC0u, 0x7FC07FC0u};
        } else if (raw) {
#pragma unroll
            for (int j = 0; j < 8; ++j) c[j] = *(LAS u32x4*)(cur + j * 1024 + lane * 16);
        } else {
            recs = *(LAS unsigned*)(cur + RAWB + (((h >> 16) & 0x3FFu) + (lane & 31)) * 4u);
            decode_block(cur, lds + CTL + 256 + d * 256, c, lane);
        }
#pragma unroll
        for (int j = 0; j < 8; ++j) asm volatile("" : "+v"(c[j]));
        asm volatile("" : "+v"(recs));
        if (lane == 0) __hip_atomic_fetch_add((LAS unsigned*)(lds + CTL + 160 + sl * 4), 1u, __ATOMIC_RELAXED, __HIP_MEMORY_SCOPE_WORKGROUP);
        const unsigned sj = b.is_out ? 8u * DIM * 2u : 1024u, sr = b.is_out ? DIM * 2u : 128u;
        const unsigned vo = (unsigned)(lane >> 3) * sr + (unsigned)(lane & 7) * 16u;
        if (b.is_out) {
#pragma unroll
            for (int j = 0; j < 8; ++j) __builtin_amdgcn_raw_buffer_store_b128(c[j], outr, vo, b.dso + j * sj, OUT_AUX);
            for (unsigned e = 0; e < ne; ++e) {
                const unsigned rec = __builtin_amdgcn_readlane(recs, e), pos = rec & 0xFFFu, v = pos & 63u;
                if ((unsigned)lane == (pos >> 6)) __builtin_amdgcn_raw_buffer_store_b16((unsigned short)((rec >> 12) & 0xFFFFu), outr, vo + (v & 7u) * 2u, b.dso + (v >> 3) * sj, OUT_AUX);
            }
        } else {
#pragma unroll
            for (int j = 0; j < 8; ++j) __builtin_amdgcn_raw_buffer_store_b128(c[j], wsr, vo, b.dso + j * sj, IMG_AUX);
            for (unsigned e = 0; e < ne; ++e) {
                const unsigned rec = __builtin_amdgcn_readlane(recs, e), pos = rec & 0xFFFu, v = pos & 63u;
                if ((unsigned)lane == (pos >> 6)) __builtin_amdgcn_raw_buffer_store_b16((unsigned short)((rec >> 12) & 0xFFFFu), wsr, vo + (v & 7u) * 2u, b.dso + (v >> 3) * sj, IMG_AUX);
            }
            WAIT_VM0(); signal_cnt(ws, b.cnt, lane);
        }
    }
}

__device__ __forceinline__ void puller_role(const Params& P, LAS unsigned char* lds, unsigned call, int wg) {
    const int lane = threadIdx.x & 63, wid = rfl(threadIdx.x >> 6);
    const int link = wg % 7, src = rfl((P.rank + 1 + link) & 7);
    if (threadIdx.x < 16) *(volatile LAS unsigned*)(lds + CTL + 128 + threadIdx.x * 4) = 0u;
    __syncthreads();
    if (wid == 0) loader_wave(P, lds, call, src, lane);
    else if (wid <= NDEC) decoder_wave(P, lds, call, src, wid - 1, lane);
}

__device__ __forceinline__ float gelu_f(float y) {
    const float c1 = -2.0f * 0.7978845608028654f * 1.4426950408889634f, c2 = c1 * 0.044715f;
    const float e = __builtin_amdgcn_exp2f(y * (c1 + c2 * y * y));
    return y * __builtin_amdgcn_rcpf(1.0f + e);
}
__device__ __forceinline__ unsigned cvt_pk_bf16(float lo, float hi) { unsigned r; asm("v_cvt_pk_bf16_f32 %0, %1, %2" : "=v"(r) : "v"(lo), "v"(hi)); return r; }

__device__ __forceinline__ void gemm_role(const Params& P, LAS unsigned char* lds, unsigned call, int wg) {
    const int tid = threadIdx.x, wid = rfl(tid >> 6), lane = tid & 63, wr = wid >> 2, wc = wid & 3, fr = lane & 15, fq = lane >> 4;
    char* ws = P.ws[P.rank];
    unsigned* tmo = (unsigned*)(ws + OFF_TMO);
    const rsrc_t wsr = mk_rsrc(ws, WS_USED);
    const int xcd = wg & 7, q = (wg - NPACK) >> 3, h = xcd & 1, blk = xcd >> 1;
    const int tm = (blk >> 1) * 4 + (q >> 2), tn = (blk & 1) * 4 + (q & 3), tile = tm * 8 + tn;
    const unsigned voff = (unsigned)tid * 16u, ldsw = (unsigned)wid * 1024u;
    const int aoff0 = (wr * 64 + fr) * 128 + ((fq ^ (fr & 7)) << 4), aoff1 = aoff0 ^ 64;
    const int boff0 = (wc * 32 + fr) * 128 + ((fq ^ (fr & 7)) << 4), boff1 = boff0 ^ 64;
#define G_SA(b, hh) (((b) * 2 + (hh)) * 16384)
#define G_SB(b, hh) ((4 + (b) * 2 + (hh)) * 16384)
#define G_STAGE(bufoff, soff) do { _Pragma("unroll") for (int _i = 0; _i < 2; ++_i) \
        __builtin_amdgcn_raw_ptr_buffer_load_lds(wsr, (LAS void*)(lds + (bufoff) + ldsw + _i * 8192), 16, voff, (soff) + _i * 8192, 0, STG_AUX); } while (0)
#define G_LDA(dst, b, hh) do { _Pragma("unroll") for (int m = 0; m < 4; ++m) { \
        dst[m][0] = *(const LAS bf16x8*)(lds + G_SA(b, hh) + aoff0 + m * 2048); dst[m][1] = *(const LAS bf16x8*)(lds + G_SA(b, hh) + aoff1 + m * 2048); } } while (0)
#define G_LDB(dst, b, hh) do { _Pragma("unroll") for (int n = 0; n < 2; ++n) { \
        dst[n][0] = *(const LAS bf16x8*)(lds + G_SB(b, hh) + boff0 + n * 2048); dst[n][1] = *(const LAS bf16x8*)(lds + G_SB(b, hh) + boff1 + n * 2048); } } while (0)
#define G_MMA(ai, bj, At, Bt) do { __builtin_amdgcn_s_setprio(1); _Pragma("unroll") for (int m = 0; m < 4; ++m) _Pragma("unroll") for (int n = 0; n < 2; ++n) _Pragma("unroll") for (int k = 0; k < 2; ++k) \
        acc[ai][bj][m][n] = __builtin_amdgcn_mfma_f32_16x16x32_bf16(Bt[n][k], At[m][k], acc[ai][bj][m][n], 0, 0, 0); __builtin_amdgcn_s_setprio(0); } while (0)
#define G_WAIT_V(n) asm volatile("s_waitcnt vmcnt(" #n ")" ::: "memory")
#define G_WAIT_L(n) asm volatile("s_waitcnt lgkmcnt(" #n ")" ::: "memory")
#define G_BAR __builtin_amdgcn_s_barrier()
#define G_SCHED __builtin_amdgcn_sched_barrier(0)
#define G_GIDX(t) ((unsigned)((4 * h + ((t) & 3)) * 16 + ((t) >> 2)))
    const int zero = 0;
    for (int r = 0; r < 2; ++r) {
        const unsigned target = call * (r == 0 ? 64u : 32u);
        const unsigned abase = OFF_AIMG + (unsigned)r * (128u * 262144u) + (unsigned)tm * 32768u, bbase = OFF_BIMG + (unsigned)tn * 32768u;
        char* cbase = ws + OFF_CNT + r * 128 * 128;
        f32x4 acc[2][2][4][2];
#pragma unroll
        for (int a = 0; a < 2; ++a)
#pragma unroll
            for (int b = 0; b < 2; ++b)
#pragma unroll
                for (int m = 0; m < 4; ++m)
#pragma unroll
                    for (int n = 0; n < 2; ++n) acc[a][b][m][n] = (f32x4){0.f, 0.f, 0.f, 0.f};
        bf16x8 At[4][2], B0[2][2], B1[2][2];
        STAMP(ws, r * 8 + 0);
        wait_word((unsigned*)(cbase + G_GIDX(0) * 128), target, tmo, call);
        wait_word((unsigned*)(cbase + G_GIDX(1) * 128), target, tmo, call);
        wait_word((unsigned*)(cbase + G_GIDX(2) * 128), target, tmo, call);
        STAMP(ws, r * 8 + 1);
        {
            const unsigned a0 = abase + G_GIDX(0) * 262144u, b0 = bbase + G_GIDX(0) * 262144u, a1 = abase + G_GIDX(1) * 262144u, b1 = bbase + G_GIDX(1) * 262144u;
            G_STAGE(G_SB(0, 0), b0); G_STAGE(G_SA(0, 0), a0); G_STAGE(G_SB(0, 1), b0 + 16384); G_STAGE(G_SA(0, 1), a0 + 16384);
            if (wr == 1) G_BAR;
            G_WAIT_V(4); G_BAR;
            G_STAGE(G_SB(1, 0), b1); G_STAGE(G_SA(1, 0), a1); G_STAGE(G_SB(1, 1), b1 + 16384);
            G_WAIT_V(6); G_BAR;
        }
        unsigned fA = target, fB;
        for (int t = 0; t < 64; t += 2) {
            const int t2 = t + 2 < 64 ? t + 2 : 63, t3 = t + 3 < 64 ? t + 3 : 63, t4 = t + 4 < 64 ? t + 4 : 63;
            const unsigned g1 = G_GIDX(t + 1), g2 = G_GIDX(t2), g3 = G_GIDX(t3), g4 = G_GIDX(t4);
            const unsigned a1 = abase + g1 * 262144u, a2 = abase + g2 * 262144u, b2 = bbase + g2 * 262144u, a3 = abase + g3 * 262144u, b3 = bbase + g3 * 262144u;
            asm volatile("global_load_dword %0, %1, %2 sc0 sc1" : "=v"(fB) : "v"(zero), "s"(cbase + g3 * 128));
            G_LDB(B0, 0, 0); G_SCHED; G_LDA(At, 0, 0); G_STAGE(G_SA(1, 1), a1 + 16384);
            G_WAIT_L(8); G_BAR; G_WAIT_L(0); G_MMA(0, 0, At, B0); G_BAR; G_SCHED;
            asm volatile("" : "+v"(fA));
            if ((int)(rfl(fA) - target) < 0) wait_word((unsigned*)(cbase + g2 * 128), target, tmo, call);
            G_LDB(B1, 0, 1); G_STAGE(G_SB(0, 0), b2);
            G_BAR; G_WAIT_L(0); G_MMA(0, 1, At, B1); G_BAR;
            G_LDA(At, 0, 1); G_STAGE(G_SA(0, 0), a2);
            G_BAR; G_WAIT_L(0); G_MMA(1, 0, At, B0); G_BAR; G_SCHED;
            G_STAGE(G_SB(0, 1), b2 + 16384);
            G_WAIT_V(6); G_BAR; G_MMA(1, 1, At, B1); G_BAR;
            asm volatile("global_load_dword %0, %1, %2 sc0 sc1" : "=v"(fA) : "v"(zero), "s"(cbase + g4 * 128));
            G_LDB(B0, 1, 0); G_SCHED; G_LDA(At, 1, 0); G_STAGE(G_SA(0, 1), a2 + 16384);
            G_WAIT_L(8); G_BAR; G_WAIT_L(0); G_MMA(0, 0, At, B0); G_BAR; G_SCHED;
            asm volatile("" : "+v"(fB));
            if ((int)(rfl(fB) - target) < 0) wait_word((unsigned*)(cbase + g3 * 128), target, tmo, call);
            G_LDB(B1, 1, 1); G_STAGE(G_SB(1, 0), b3);
            G_BAR; G_WAIT_L(0); G_MMA(0, 1, At, B1); G_BAR;
            G_LDA(At, 1, 1); G_STAGE(G_SA(1, 0), a3);
            G_BAR; G_WAIT_L(0); G_MMA(1, 0, At, B0); G_BAR; G_SCHED;
            G_STAGE(G_SB(1, 1), b3 + 16384);
            G_WAIT_V(6); G_BAR; G_MMA(1, 1, At, B1); G_BAR;
        }
        G_WAIT_V(0);
        asm volatile("" : "+v"(fA));
        if (wr == 0) G_BAR;
        G_BAR;
        STAMP(ws, r * 8 + 2);
        int lane = threadIdx.x & 63;
        asm volatile("" : "+v"(lane));
        const int fr = lane & 15, fq = lane >> 4;
        if (h) {
#pragma unroll
            for (int b = 0; b < 2; ++b)
#pragma unroll
                for (int m = 0; m < 4; ++m)
#pragma unroll
                    for (int n = 0; n < 2; ++n) { const f32x4 tq = acc[0][b][m][n]; acc[0][b][m][n] = acc[1][b][m][n]; acc[1][b][m][n] = tq; }
        }
        const unsigned xl = (unsigned)((r * 64 + tile) * 2);
#pragma unroll
        for (int b = 0; b < 2; ++b)
#pragma unroll
            for (int m = 0; m < 4; ++m)
#pragma unroll
                for (int n = 0; n < 2; ++n)
                    __builtin_amdgcn_raw_buffer_store_b128(__builtin_bit_cast(u32x4, acc[1][b][m][n]), wsr, lane * 16 + ((b * 4 + m) * 2 + n) * 1024, OFF_XBUF + ((xl + h) * 8 + wid) * 16384u, AUX_SC1);
        WAIT_VM0();
        if (lane == 0) __hip_atomic_store((unsigned*)(ws + OFF_XFLAG + (xl + h) * 128 + wid * 16), call, __ATOMIC_RELAXED, __HIP_MEMORY_SCOPE_AGENT);
        wait_word((unsigned*)(ws + OFF_XFLAG + (xl + (h ^ 1)) * 128 + wid * 16), call, tmo, call);
        const bool dead = rfl(__hip_atomic_load(tmo, __ATOMIC_RELAXED, __HIP_MEMORY_SCOPE_AGENT)) == call;
        STAMP(ws, r * 8 + 3);
        LAS unsigned char* my = lds + wid * 8192;
#pragma unroll
        for (int b = 0; b < 2; ++b)
#pragma unroll
            for (int m = 0; m < 4; ++m) {
                const f32x4 p0 = __builtin_bit_cast(f32x4, __builtin_amdgcn_raw_buffer_load_b128(wsr, lane * 16 + ((b * 4 + m) * 2 + 0) * 1024, OFF_XBUF + ((xl + (h ^ 1)) * 8 + wid) * 16384u, AUX_SC1));
                const f32x4 p1 = __builtin_bit_cast(f32x4, __builtin_amdgcn_raw_buffer_load_b128(wsr, lane * 16 + ((b * 4 + m) * 2 + 1) * 1024, OFF_XBUF + ((xl + (h ^ 1)) * 8 + wid) * 16384u, AUX_SC1));
                const f32x4 v0 = acc[0][b][m][0] + p0, v1 = acc[0][b][m][1] + p1;
                u32x4 o;
                o.x = cvt_pk_bf16(gelu_f(v0[0]), gelu_f(v0[1])); o.y = cvt_pk_bf16(gelu_f(v0[2]), gelu_f(v0[3]));
                o.z = cvt_pk_bf16(gelu_f(v1[0]), gelu_f(v1[1])); o.w = cvt_pk_bf16(gelu_f(v1[2]), gelu_f(v1[3]));
                const int row = m * 16 + fr, ch = b * 4 + fq;
                *(LAS u32x4*)(my + row * 128 + ((ch ^ (row & 7)) * 16)) = o;
            }
        asm volatile("s_waitcnt lgkmcnt(0)" ::: "memory");
        u32x4 c[8];
#pragma unroll
        for (int j = 0; j < 8; ++j) { const int row = j * 8 + (lane >> 3); c[j] = *(LAS u32x4*)(my + row * 128 + (((lane & 7) ^ (row & 7)) * 16)); }
        if (dead) {
#pragma unroll
            for (int j = 0; j < 8; ++j) c[j] = (u32x4){0x7FC07FC0u, 0x7FC07FC0u, 0x7FC07FC0u, 0x7FC07FC0u};
        }
        {
            const rsrc_t outr = mk_rsrc(P.out, (unsigned)DIM * DIM * 2u);
            const unsigned row0 = (P.rank >> 2) * 4096 + r * 2048 + tm * 256 + h * 128 + wr * 64, col0 = (P.rank & 3) * 2048 + tn * 256 + wc * 64;
            const unsigned dso = (row0 * DIM + col0) * 2u, vo = (unsigned)(lane >> 3) * (DIM * 2u) + (unsigned)(lane & 7) * 16u;
            if (r == 0) {
#pragma unroll
                for (int j = 0; j < 8; ++j) __builtin_amdgcn_raw_buffer_store_b128(c[j], outr, vo, dso + j * (8u * DIM * 2u), AUX_SC1);
            } else {
#pragma unroll
                for (int j = 0; j < 8; ++j) __builtin_amdgcn_raw_buffer_store_b128(c[j], outr, vo, dso + j * (8u * DIM * 2u), OUT_AUX);
            }
            const unsigned o = (xl + h) * 8 + wid;
            if (r == 0) { WAIT_VM0(); signal_cnt(ws, OFF_OCNT + (o >> 3) * 128, lane); }
            else publish_block(P, wsr, c, 4096u + o, 1536u + o, 0xFFu & ~(1u << P.rank), call, lds, my, 1u + GROUNDS, wid, lane);
        }
        STAMP(ws, r * 8 + 4);
        asm volatile("s_waitcnt lgkmcnt(0)" ::: "memory");
        G_BAR;
    }
#undef G_SA
#undef G_SB
#undef G_STAGE
#undef G_LDA
#undef G_LDB
#undef G_MMA
#undef G_WAIT_V
#undef G_WAIT_L
#undef G_BAR
#undef G_SCHED
#undef G_GIDX
}

__global__ __launch_bounds__(512, 2) void gemm_allgather_kernel(Params P) {
    extern __shared__ __attribute__((aligned(16))) unsigned char shm[];
    LAS unsigned char* lds = (LAS unsigned char*)shm;
    char* ws = P.ws[P.rank];
    const int b = rfl(blockIdx.x);
    STAMP(ws, 60);
    unsigned call;
    if (b >= NPACK + NGEMM) { call = call_draw(ws); puller_role(P, lds, rfl(call), rfl(b - NPACK - NGEMM)); }
    else {
        if (threadIdx.x < 40) *(volatile LAS unsigned*)(lds + GCTL + 128 + threadIdx.x * 4) = 0u;
        __syncthreads();
        const int wid = rfl(threadIdx.x >> 6);
        const bool gemm = b >= NPACK;
        const int next = gemm ? NFIRST + (b - NPACK) * 8 + wid : NFIRST + GROUNDS * NGEMM * 8 + b * 8 + wid;
        call = packer_role(P, lds, wid < 4 ? b * 4 + wid : next, next, gemm ? NGEMM * 8 : NPACK * 8, gemm ? NFIRST + GROUNDS * NGEMM * 8 : 4096 + 1024);
        if (gemm) { __syncthreads(); gemm_role(P, lds, rfl(call), rfl(b)); }
    }
    STAMP(ws, 61);
    STAMP(ws, 58 + (call & 1));
}

}

extern "C" void kernel_launch(int rank, int nranks, ncclComm_t comm,
                              hipStream_t stream,
                              void* const* d_in, const int* in_sizes, int n_in,
                              void* d_out, int out_size,
                              const int* in_dtypes, int out_dtype,
                              void* const* d_out_all,
                              void* const* d_ws_all, size_t ws_bytes) {
    (void)comm; (void)in_sizes; (void)in_dtypes; (void)out_dtype; (void)d_out_all;
    if (nranks != gar::WORLD || n_in < 2 || ws_bytes < gar::WS_NEED || out_size != gar::DIM * gar::DIM || (unsigned)rank >= (unsigned)gar::WORLD) {
        fprintf(stderr, "unexpected problem shape\n");
        return;
    }
    static bool attr = false;
    if (!attr) {
        hipError_t e = hipFuncSetAttribute((const void*)gar::gemm_allgather_kernel, hipFuncAttributeMaxDynamicSharedMemorySize, gar::LDS_BYTES);
        if (e != hipSuccess) { fprintf(stderr, "hipFuncSetAttribute: %s\n", hipGetErrorString(e)); return; }
        attr = true;
    }
    gar::Params P;
    memset(&P, 0, sizeof P);
    P.x = d_in[0]; P.w = d_in[1]; P.out = d_out; P.rank = rank;
    for (int p = 0; p < gar::WORLD; ++p) P.ws[p] = (char*)d_ws_all[p];
    hipLaunchKernelGGL(gar::gemm_allgather_kernel, dim3(gar::GRID), dim3(512), gar::LDS_BYTES, stream, P);
    hipError_t e = hipGetLastError();
    if (e != hipSuccess) fprintf(stderr, "launch: %s\n", hipGetErrorString(e));
}
```

```cpp
#include <hip/hip_runtime.h>
#include <rccl/rccl.h>
#include <cstdio>
#include <cstdint>
#include <cstring>

namespace gar {

#define LAS __attribute__((address_space(3)))
typedef unsigned u32x4 __attribute__((ext_vector_type(4)));
typedef unsigned short u16x2 __attribute__((ext_vector_type(2)));
typedef short bf16x8 __attribute__((ext_vector_type(8)));
typedef float f32x4 __attribute__((ext_vector_type(4)));
typedef __amdgpu_buffer_rsrc_t rsrc_t;

constexpr int WORLD = 8;
constexpr int DIM = 8192;
constexpr int KSH = 1024;
constexpr int NGEMM = 128;
constexpr int GRID = 256;
#define PULL_WG_PER_LINK 12
#define PULL_DECODERS 7
constexpr int PWL = PULL_WG_PER_LINK;
constexpr int NDEC = PULL_DECODERS;
constexpr int NPULL = 7 * PWL;
constexpr int NPACK = GRID - NGEMM - NPULL;
#define NRAWEND 40
#define WARM_TLB 1
#define OUT_AUX 18
#define IMG_AUX 16
#define STG_AUX AUX_SYS
#define XIN_AUX 0
#define JIT_LEAD 30
#define PAY_AUX AUX_SYS
#define NRAW 48
constexpr int NFIRST = (NPACK + NGEMM) * 4;
#define GEMM_PACK_ROUNDS 1
constexpr int GROUNDS = GEMM_PACK_ROUNDS;
constexpr int OUT2 = 4096 + NPACK * 4;
static_assert(NFIRST + GROUNDS * NGEMM * 8 <= 4096 && NFIRST % 8 == 0 && OUT2 % 8 == 0, "roles");
constexpr int NSEQ = 3584;
constexpr unsigned long long DEADLINE = 40000000ull;

constexpr unsigned OFF_TMO = 128;
constexpr unsigned OFF_LINK = 1024;
constexpr unsigned OFF_CNT = 4096;
constexpr unsigned OFF_XFLAG = OFF_CNT + 256 * 128;
constexpr unsigned OFF_OCNT = OFF_XFLAG + 256 * 128;
constexpr unsigned OFF_GRAN = 131072;
constexpr unsigned OFF_CALL = 640u << 10;
constexpr unsigned OFF_SLOT = 1u << 20;
constexpr unsigned OFF_AIMG = 64u << 20;
constexpr unsigned OFF_BIMG = 128u << 20;
constexpr unsigned OFF_XBUF = 160u << 20;
constexpr unsigned WS_USED = 192u << 20;
constexpr unsigned WS_NEED = WS_USED;
static_assert(OFF_OCNT + 128 * 128 <= OFF_GRAN, "map");
static_assert(OFF_GRAN + WORLD * NSEQ * 8 <= OFF_CALL && OFF_CALL + 2048 * 64 <= OFF_SLOT, "map");
static_assert(OFF_SLOT + 6144u * 8192u <= OFF_AIMG, "map");
constexpr int LDS_BYTES = 131072 + 512;

struct Params {
    const void* x; const void* w; void* out;
    char* ws[WORLD];
    int rank; int pad;
};

constexpr int AUX_SC1 = 16, AUX_SYS = 17;

__device__ __forceinline__ rsrc_t mk_rsrc(const void* p, unsigned bytes) {
    return __builtin_amdgcn_make_buffer_rsrc(const_cast<void*>(p), 0, bytes, 0x00020000);
}
__device__ __forceinline__ unsigned rfl(unsigned v) { return __builtin_amdgcn_readfirstlane(v); }
__device__ __forceinline__ unsigned bfi(unsigned mask, unsigned a, unsigned b) { return (a & mask) | (b & ~mask); }
__device__ __forceinline__ unsigned perm(unsigned hi, unsigned lo, unsigned sel) { return __builtin_amdgcn_perm(hi, lo, sel); }
#define WAIT_VM0() asm volatile("s_waitcnt vmcnt(0)" ::: "memory")

#define STAMP(ws, idx) do { } while (0)

__device__ __forceinline__ unsigned call_draw(char* ws) {
    unsigned t;
    const int zero = 0, one = 1;
    unsigned long long keep;
    asm volatile("s_mov_b64 %1, exec\n\ts_mov_b64 exec, 1\n\tglobal_atomic_add %0, %2, %3, %4 sc0\n\ts_mov_b64 exec, %1\n\ts_waitcnt vmcnt(0)"
                 : "=&v"(t), "=&s"(keep) : "v"(zero), "v"(one), "s"(ws + OFF_CALL + (blockIdx.x * 8 + rfl(threadIdx.x >> 6)) * 64) : "memory");
    return rfl(t) + 1u;
}

__device__ __forceinline__ bool give_up(unsigned long long t0, unsigned* tmo, unsigned call) {
    if (rfl(__hip_atomic_load(tmo, __ATOMIC_RELAXED, __HIP_MEMORY_SCOPE_AGENT)) == call) return true;
    if (__builtin_amdgcn_s_memrealtime() - t0 > DEADLINE) { __hip_atomic_store(tmo, call, __ATOMIC_RELAXED, __HIP_MEMORY_SCOPE_AGENT); return true; }
    return false;
}
__device__ __forceinline__ void wait_word(unsigned* p, unsigned target, unsigned* tmo, unsigned call) {
    if ((int)(rfl(__hip_atomic_load(p, __ATOMIC_RELAXED, __HIP_MEMORY_SCOPE_SYSTEM)) - target) >= 0) return;
    const unsigned long long t0 = __builtin_amdgcn_s_memrealtime();
    while ((int)(rfl(__hip_atomic_load(p, __ATOMIC_RELAXED, __HIP_MEMORY_SCOPE_SYSTEM)) - target) < 0) {
        if (give_up(t0, tmo, call)) break;
        __builtin_amdgcn_s_sleep(4);
    }
}

constexpr unsigned MAX_EXC = 31;
constexpr unsigned RAWB = 3584, AUX_U = 20, AUX_MAX = 2560, U_MAX = 10240, LANE_U_MAX = 224;
constexpr unsigned ENC_HIST = AUX_MAX + 16, ENC_SORT = ENC_HIST + 256, ENC_END = ENC_SORT + 128;

__device__ __forceinline__ unsigned bfi_c(unsigned mask, unsigned a, unsigned b) { unsigned r; asm("v_bfi_b32 %0, %1, %2, %3" : "=v"(r) : "s"(mask), "v"(a), "v"(b)); return r; }
__device__ __forceinline__ unsigned opaque(unsigned v) { asm("" : "+v"(v)); return v; }
__device__ __forceinline__ unsigned bytemask(unsigned x, int sh, unsigned c0c) {
    unsigned t; asm("v_and_or_b32 %0, %1, %2, %3" : "=v"(t) : "v"(x >> sh), "s"(0x01010101u), "v"(c0c));
    return perm(0u, 0u, t);
}
template <int CTRL, int RM = 0xf> __device__ __forceinline__ unsigned dpp0(unsigned v) { return (unsigned)__builtin_amdgcn_update_dpp(0, (int)v, CTRL, RM, 0xf, RM == 0xf); }
__device__ __forceinline__ unsigned wave_scan_incl(unsigned v) {
    v += dpp0<0x111>(v); v += dpp0<0x112>(v); v += dpp0<0x114>(v); v += dpp0<0x118>(v);
    v += dpp0<0x142, 0xa>(v); v += dpp0<0x143, 0xc>(v);
    return v;
}
__device__ __forceinline__ unsigned umin(unsigned a, unsigned b) { return a < b ? a : b; }
__device__ __forceinline__ unsigned umax(unsigned a, unsigned b) { return a > b ? a : b; }
__device__ __forceinline__ unsigned sad4(unsigned x) { return __builtin_amdgcn_sad_u8(x, 0u, 0u); }
__device__ __forceinline__ unsigned lut2x16(const unsigned (&T)[8], unsigned idx, unsigned ms, unsigned c0c) {
    const unsigned sel = idx & 0x07070707u, m8 = bytemask(idx, 3, c0c);
    const unsigned p = bfi(m8, perm(T[3], T[2], sel), perm(T[1], T[0], sel)), n = bfi(m8, perm(T[7], T[6], sel), perm(T[5], T[4], sel));
    return bfi(ms, n, p);
}
#define LDS_FENCE() asm volatile("" ::: "memory")
__device__ __forceinline__ void lds_or2(LAS unsigned char* scr, unsigned p, unsigned w) {
    const unsigned s = p & 31u;
    LAS unsigned* q = (LAS unsigned*)(scr + ((p >> 5) << 2));
    __hip_atomic_fetch_or(q, w << s, __ATOMIC_RELAXED, __HIP_MEMORY_SCOPE_WORKGROUP);
    __hip_atomic_fetch_or(q + 1, (w >> 1) >> (31u - s), __ATOMIC_RELAXED, __HIP_MEMORY_SCOPE_WORKGROUP);
}

__device__ __forceinline__ unsigned encode_block(const u32x4 (&c)[8], unsigned (&P)[14], u32x4 (&a)[3], LAS unsigned char* scr, int lane) {
    asm volatile("" : "+v"(lane));
    const unsigned c0c = opaque(0x0C0C0C0Cu);
    unsigned E[16], R[16];
    u16x2 p1 = {0, 0}, p2 = {0, 0}, n1 = {0, 0}, n2 = {0, 0};
#pragma unroll
    for (int g = 0; g < 16; ++g) {
        const unsigned x = (g & 1) ? c[g >> 1].z : c[g >> 1].x, y = (g & 1) ? c[g >> 1].w : c[g >> 1].y;
        const unsigned H = perm(y, x, 0x07050301u), L = perm(y, x, 0x06040200u);
        E[g] = bfi_c(0xFEFEFEFEu, H << 1, L >> 7);
        R[g] = bfi_c(0x80808080u, L << 1, bfi_c(0x40404040u, H >> 1, L));
        const unsigned en = E[g] & bytemask(H, 7, c0c), ep = E[g] ^ en;
        p1 = __builtin_elementwise_max(p1, __builtin_bit_cast(u16x2, ep)); p2 = __builtin_elementwise_max(p2, __builtin_bit_cast(u16x2, ep << 8));
        n1 = __builtin_elementwise_max(n1, __builtin_bit_cast(u16x2, en)); n2 = __builtin_elementwise_max(n2, __builtin_bit_cast(u16x2, en << 8));
        __builtin_amdgcn_sched_barrier(0);
    }
    const u32x4 z4 = {0u, 0u, 0u, 0u};
    *(LAS u32x4*)(scr + lane * 16) = z4; *(LAS u32x4*)(scr + 1024 + lane * 16) = z4;
    if (lane < 49) *(LAS u32x4*)(scr + 2048 + lane * 16) = z4;
    unsigned baseP, baseN;
    {
        const u16x2 mp = __builtin_elementwise_max(p1, p2), mn = __builtin_elementwise_max(n1, n2);
        u16x2 m = {(unsigned short)((mp.x > mp.y ? mp.x : mp.y) >> 8), (unsigned short)((mn.x > mn.y ? mn.x : mn.y) >> 8)};
        m = __builtin_elementwise_max(m, __builtin_bit_cast(u16x2, dpp0<0x128>(__builtin_bit_cast(unsigned, m)))); m = __builtin_elementwise_max(m, __builtin_bit_cast(u16x2, dpp0<0x124>(__builtin_bit_cast(unsigned, m))));
        m = __builtin_elementwise_max(m, __builtin_bit_cast(u16x2, dpp0<0x122>(__builtin_bit_cast(unsigned, m)))); m = __builtin_elementwise_max(m, __builtin_bit_cast(u16x2, dpp0<0x121>(__builtin_bit_cast(unsigned, m))));
        const unsigned mu = __builtin_bit_cast(unsigned, m);
        const u16x2 r01 = __builtin_elementwise_max(__builtin_bit_cast(u16x2, (unsigned)__builtin_amdgcn_readlane(mu, 0)), __builtin_bit_cast(u16x2, (unsigned)__builtin_amdgcn_readlane(mu, 16)));
        const u16x2 r23 = __builtin_elementwise_max(__builtin_bit_cast(u16x2, (unsigned)__builtin_amdgcn_readlane(mu, 32)), __builtin_bit_cast(u16x2, (unsigned)__builtin_amdgcn_readlane(mu, 48)));
        const unsigned mm = rfl(__builtin_bit_cast(unsigned, __builtin_elementwise_max(r01, r23)));
        baseP = mm & 0xFFu; baseN = mm >> 16;
    }
    const unsigned BP = baseP * 0x01010101u, BN = baseN * 0x01010101u;
    unsigned any = 0, D[16];
#pragma unroll
    for (int g = 0; g < 16; ++g) {
        E[g] = bfi(bytemask(R[g], 6, c0c), BN, BP) - E[g]; any |= E[g];
        D[g] = bfi_c(0x1E1E1E1Eu, E[g] << 1, (R[g] >> 7) & 0x01010101u);
        R[g] &= 0x7F7F7F7Fu;
    }
    LDS_FENCE();
#pragma unroll
    for (int gi = 0; gi < 4; ++gi) {
        const int g = gi * 5;
        const unsigned k4 = ((D[g] & 0x0F0F0F0Fu) | ((R[g] >> 2) & 0x10101010u) | ((D[g] << 1) & 0x20202020u)) << 2;
#pragma unroll
        for (int b = 0; b < 4; ++b)
            __hip_atomic_fetch_add((LAS unsigned*)(scr + ENC_HIST + ((k4 >> (8 * b)) & 0xFFu)), 1u, __ATOMIC_RELAXED, __HIP_MEMORY_SCOPE_WORKGROUP);
    }
    LDS_FENCE();
    const unsigned cnt = *(LAS unsigned*)(scr + ENC_HIST + lane * 4);
    const unsigned key = (cnt << 4) | (15u - (lane & 15));
    unsigned rank = 0;
    rank += dpp0<0x121>(key) > key; rank += dpp0<0x122>(key) > key; rank += dpp0<0x123>(key) > key; rank += dpp0<0x124>(key) > key; rank += dpp0<0x125>(key) > key;
    rank += dpp0<0x126>(key) > key; rank += dpp0<0x127>(key) > key; rank += dpp0<0x128>(key) > key; rank += dpp0<0x129>(key) > key; rank += dpp0<0x12A>(key) > key;
    rank += dpp0<0x12B>(key) > key; rank += dpp0<0x12C>(key) > key; rank += dpp0<0x12D>(key) > key; rank += dpp0<0x12E>(key) > key; rank += dpp0<0x12F>(key) > key;
    unsigned tail = cnt; tail += dpp0<0x128>(tail); tail += dpp0<0x124>(tail); tail += dpp0<0x122>(tail); tail += dpp0<0x121>(tail);
    const int sg = (lane >> 4) & 1;
    const unsigned ct = sg ? __builtin_amdgcn_readlane(tail, 48) : __builtin_amdgcn_readlane(tail, 32);
    if (lane < 32) *(LAS unsigned*)(scr + ENC_SORT + ((lane & 16) + rank) * 4) = cnt;
    LDS_FENCE();
    unsigned best;
    {
        const unsigned cb = lane & 15, s0 = cb / 3u, s1 = cb - 3u * s0, A = 1u << s0, AB = A + (1u << s1);
        unsigned cost = ct * (16u - AB);
#pragma unroll
        for (int q = 0; q < 4; ++q) {
            const u32x4 h = *(LAS u32x4*)(scr + ENC_SORT + sg * 64 + q * 16);
#pragma unroll
            for (int j = 0; j < 4; ++j) {
                const unsigned r = q * 4 + j, len = r < A ? 1u + s0 : r < AB ? 2u + s1 : 4u + ((r - AB) >> 1);
                cost += 2u * len * h[j];
            }
        }
        best = cb < 9 ? (cost << 4) | cb : 0xFFFFFFFFu;
        best = umin(best, dpp0<0x128>(best)); best = umin(best, dpp0<0x124>(best)); best = umin(best, dpp0<0x122>(best)); best = umin(best, dpp0<0x121>(best));
    }
    const unsigned cbP = __builtin_amdgcn_readlane(best, 0) & 15u, cbN = __builtin_amdgcn_readlane(best, 16) & 15u;
    const unsigned s0P = cbP / 3u, s1P = cbP - 3u * s0P, s0N = cbN / 3u, s1N = cbN - 3u * s0N;
    const unsigned ABP = (1u << s0P) + (1u << s1P), ABN = (1u << s0N) + (1u << s1N);
    unsigned F[8];
    {
        const unsigned s0 = sg ? s0N : s0P, s1 = sg ? s1N : s1P, A = 1u << s0, AB = sg ? ABN : ABP;
        const unsigned ent = rank < A ? (rank << 4) | (s0 << 6) : rank < AB ? 1u | ((rank - A) << 4) | (s1 << 6) : (2u + ((rank - AB) >> 1)) | (((rank - AB) & 1u) << 4) | (1u << 6);
        unsigned fv = ent << (8 * (lane & 3));
        fv |= dpp0<0xB1>(fv); fv |= dpp0<0x4E>(fv);
#pragma unroll
        for (int q = 0; q < 8; ++q) F[q] = __builtin_amdgcn_readlane(fv, q * 4);
        if (lane < 32)
            __hip_atomic_fetch_or((LAS unsigned*)(scr + (sg * 2 + (rank >> 3)) * 4), (unsigned)(lane & 15) << (8 * (rank & 3) + 4 * ((rank >> 2) & 1)), __ATOMIC_RELAXED, __HIP_MEMORY_SCOPE_WORKGROUP);
        if (lane == 0) *(LAS unsigned*)(scr + 16) = baseP | (baseN << 8) | (s0P << 16) | (s1P << 18) | (s0N << 20) | (s1N << 22);
    }
    const unsigned ABP4 = ABP * 0x01010101u, ABN4 = ABN * 0x01010101u, S0P4 = s0P * 0x01010101u, S0N4 = s0N * 0x01010101u;
    unsigned KK[16], XW[4], OX[4], TU = 0, TX = 0, bad = 0, xl = 0, xw = 0;
#pragma unroll
    for (int g = 0; g < 16; ++g) {
        const unsigned ms = bytemask(R[g], 6, c0c), m16 = bytemask(D[g], 4, c0c), ent = lut2x16(F, D[g], ms, c0c);
        const unsigned q = ((D[g] | 0x80808080u) - bfi(ms, ABN4, ABP4)) & 0x7F7F7F7Fu;
        unsigned K = bfi(m16, ((q >> 1) & 0x3F3F3F3Fu) + 0x02020202u, ent & 0x0F0F0F0Fu);
        unsigned SUF = bfi(m16, q & 0x01010101u, (ent >> 4) & 0x03030303u);
        unsigned S = bfi(m16, 0x01010101u, (ent >> 6) & 0x03030303u);
        unsigned gl = sad4(K) + 4u;
        if (__ballot(gl > 31u) != 0ull) {
            any |= 0x80u;
            const unsigned km = umax(umax(K & 0xFFu, (K >> 8) & 0xFFu), umax((K >> 16) & 0xFFu, K >> 24));
            const unsigned eq = gl > 31u ? (((K ^ (km * 0x01010101u)) + 0x7F7F7F7Fu) & 0x80808080u) ^ 0x80808080u : 0u, em = bytemask(eq, 7, c0c);
            E[g] |= eq;
            K &= ~em; SUF &= ~em; S = bfi(em, bfi(ms, S0N4, S0P4), S);
            gl = sad4(K) + 4u;
        }
        KK[g] = K; TU += gl; bad = umax(bad, gl);
        xw |= (SUF & 0xFFu) << xl; xl += S & 0xFFu;
        xw |= ((SUF >> 8) & 0xFFu) << xl; xl += (S >> 8) & 0xFFu;
        xw |= ((SUF >> 16) & 0xFFu) << xl; xl += (S >> 16) & 0xFFu;
        xw |= (SUF >> 24) << (xl & 31u); xl += S >> 24;
        if ((g & 3) == 3) { XW[g >> 2] = xw; OX[g >> 2] = TX; TX += xl; xw = 0; xl = 0; }
        __builtin_amdgcn_sched_barrier(0);
    }
    bad = (bad > 31u) | (TU > LANE_U_MAX);
    const unsigned tt = TU | (TX << 16), incl = wave_scan_incl(tt), tot = __builtin_amdgcn_readlane(incl, 63), totU = tot & 0xFFFFu, totX = tot >> 16;
    const unsigned xstart = (AUX_U * 8u + totU + 31u) & ~31u, endbit = xstart + totX;
    unsigned raw = (__ballot(bad != 0) != 0ull) || totU > U_MAX || endbit > (AUX_MAX - 8u) * 8u;
    if (!raw) {
        const unsigned px = xstart + ((incl - tt) >> 16);
        unsigned p = AUX_U * 8u + ((incl - tt) & 0xFFFFu);
#pragma unroll
        for (int g = 0; g < 16; ++g) {
            const unsigned ln = KK[g] + 0x01010101u;
            unsigned pos = (ln & 0xFFu) - 1u, w = 1u << pos;
            pos += (ln >> 8) & 0xFFu; w |= 1u << pos;
            pos += (ln >> 16) & 0xFFu; w |= 1u << pos;
            pos += ln >> 24; w |= 1u << pos;
            lds_or2(scr, p, w); p += pos + 1u;
            __builtin_amdgcn_sched_barrier(0);
        }
#pragma unroll
        for (int q = 0; q < 4; ++q) lds_or2(scr, px + OX[q], XW[q]);
    }
#pragma unroll
    for (int i = 0; i < 7; ++i) { P[i] = R[i] | ((R[14] << (7 - i)) & 0x80808080u); P[7 + i] = R[7 + i] | ((R[15] << (7 - i)) & 0x80808080u); }
    unsigned n = 0;
    const unsigned xo = (endbit + 31u) >> 5;
    if (!raw && __ballot((any & 0xF0F0F0F0u) != 0) != 0ull) {
#pragma unroll
        for (int g = 0; g < 16; ++g) {
            const unsigned f = E[g] & 0xF0F0F0F0u;
            unsigned long long mask = __ballot(f != 0);
            while (mask != 0ull && n <= MAX_EXC) {
                const int L = __builtin_ctzll(mask); mask &= mask - 1;
                const unsigned fl = __builtin_amdgcn_readlane(f, L);
                const unsigned lo = __builtin_amdgcn_readlane((g & 1) ? c[g >> 1].z : c[g >> 1].x, L);
                const unsigned hi = __builtin_amdgcn_readlane((g & 1) ? c[g >> 1].w : c[g >> 1].y, L);
                const bool e = lane < 4 && ((fl >> (8 * (lane & 3))) & 0xFFu) != 0;
                const unsigned long long em = __ballot(e);
                const unsigned slot = xo + n + __builtin_popcountll(em & ((1ull << lane) - 1ull));
                const unsigned val = (((lane & 2) ? hi : lo) >> (16 * (lane & 1))) & 0xFFFFu;
                if (e && slot * 4u < AUX_MAX) *(LAS unsigned*)(scr + slot * 4u) = (unsigned)(L * 64 + g * 4 + (lane & 3)) | (val << 12);
                n += __builtin_popcountll(em);
            }
        }
    }
    raw |= (n > MAX_EXC) | ((xo + n) * 4u > AUX_MAX);
    LDS_FENCE();
    a[0] = *(LAS u32x4*)(scr + lane * 16); a[1] = *(LAS u32x4*)(scr + 1024 + lane * 16); a[2] = *(LAS u32x4*)(scr + 2048 + (lane & 31) * 16);
    return ((xo + n + 3u) >> 2) | (n << 8) | (xo << 16) | (raw << 31);
}

__device__ __forceinline__ void decode_block(LAS unsigned char* src, LAS unsigned char* pos_scr, u32x4 (&c)[8], int lane) {
    LAS unsigned char* aux = src + RAWB;
    const unsigned c0c = opaque(0x0C0C0C0Cu);
    const u32x4 nt = *(LAS const u32x4*)(aux);
    const unsigned prm_v = *(LAS const unsigned*)(aux + 16);
    unsigned u[5];
#pragma unroll
    for (int i = 0; i < 5; ++i) u[i] = *(LAS const unsigned*)(aux + AUX_U + lane * 20 + i * 4);
    u32x4 P4[3];
#pragma unroll
    for (int p = 0; p < 3; ++p) P4[p] = *(LAS const u32x4*)(src + p * 1024 + lane * 16);
    const unsigned long long ph = *(LAS const unsigned long long*)(src + 3072 + lane * 8);
    const unsigned c0 = __builtin_popcount(u[0]), c1 = c0 + __builtin_popcount(u[1]), c2 = c1 + __builtin_popcount(u[2]), c3 = c2 + __builtin_popcount(u[3]), cn = c3 + __builtin_popcount(u[4]);
    const unsigned C = wave_scan_incl(cn) - cn;
    if (lane == 0) *(LAS unsigned*)(pos_scr) = AUX_U * 8u;
    const int rounds = __ballot(cn > 128u) != 0ull ? 3 : 2;
    for (int j = 0; j < rounds; ++j) {
        const unsigned t = (C >> 6) + 1u + j, kth = 64u * t - C;
        const unsigned i = (kth > c0) + (kth > c1) + (kth > c2) + (kth > c3);
        unsigned rem = kth - (i == 0 ? 0u : i == 1 ? c0 : i == 2 ? c1 : i == 3 ? c2 : c3), w = i == 0 ? u[0] : i == 1 ? u[1] : i == 2 ? u[2] : i == 3 ? u[3] : u[4], pos = 0;
#pragma unroll
        for (int st = 16; st >= 1; st >>= 1) {
            const unsigned lo = __builtin_popcount(w & ((1u << st) - 1u));
            const bool up = rem > lo;
            rem -= up ? lo : 0u; w = up ? w >> st : w; pos += up ? st : 0;
        }
        if (kth <= cn && t < 64u) *(LAS unsigned*)(pos_scr + t * 4) = AUX_U * 8u + lane * 160u + i * 32u + pos + 1u;
    }
    LDS_FENCE();
    const unsigned p0 = *(LAS const unsigned*)(pos_scr + lane * 4);
    unsigned S[8];
    {
        LAS const unsigned* q = (LAS const unsigned*)(aux + ((p0 >> 5) << 2));
#pragma unroll
        for (int i = 0; i < 8; ++i) S[i] = q[i];
#pragma unroll
        for (int i = 0; i < 7; ++i) S[i] = __builtin_amdgcn_alignbit(S[i + 1], S[i], p0 & 31u);
    }
    unsigned R[16];
    {
        const unsigned P[14] = {P4[0].x, P4[0].y, P4[0].z, P4[0].w, P4[1].x, P4[1].y, P4[1].z, P4[1].w, P4[2].x, P4[2].y, P4[2].z, P4[2].w, (unsigned)ph, (unsigned)(ph >> 32)};
        unsigned x = 0, y = 0;
#pragma unroll
        for (int i = 0; i < 7; ++i) { x |= (P[i] >> (7 - i)) & (0x01010101u << i); y |= (P[7 + i] >> (7 - i)) & (0x01010101u << i); }
#pragma unroll
        for (int i = 0; i < 14; ++i) R[i] = P[i] & 0x7F7F7F7Fu;
        R[14] = x; R[15] = y;
    }
    const unsigned n0 = rfl(nt.x), n1 = rfl(nt.y), n2 = rfl(nt.z), n3 = rfl(nt.w), prm = rfl(prm_v);
    const unsigned I[8] = {n0 & 0x0F0F0F0Fu, (n0 >> 4) & 0x0F0F0F0Fu, n1 & 0x0F0F0F0Fu, (n1 >> 4) & 0x0F0F0F0Fu, n2 & 0x0F0F0F0Fu, (n2 >> 4) & 0x0F0F0F0Fu, n3 & 0x0F0F0F0Fu, (n3 >> 4) & 0x0F0F0F0Fu};
    const unsigned s0P = (prm >> 16) & 3u, s1P = (prm >> 18) & 3u, s0N = (prm >> 20) & 3u, s1N = (prm >> 22) & 3u;
    const unsigned STP = (s0P << 8) | (s1P << 16) | (1u << 24), STN = (s0N << 8) | (s1N << 16) | (1u << 24);
    const unsigned OTP = (4u << 8) | (((1u << s0P) + 2u) << 16) | (((1u << s0P) + (1u << s1P)) << 24);
    const unsigned OTN = (4u << 8) | (((1u << s0N) + 2u) << 16) | (((1u << s0N) + (1u << s1N)) << 24);
    const unsigned BP = (prm & 0xFFu) * 0x01010101u, BN = ((prm >> 8) & 0xFFu) * 0x01010101u;
    unsigned RB[16], SS[16], TU = 0, TX = 0;
#pragma unroll
    for (int g = 0; g < 16; ++g) {
        const unsigned w0 = S[0], q0 = __builtin_ctz(w0), w1 = w0 & (w0 - 1u), q1 = __builtin_ctz(w1), w2 = w1 & (w1 - 1u), q2 = __builtin_ctz(w2), w3 = w2 & (w2 - 1u), q3 = __builtin_ctz(w3);
        const unsigned Q = q0 | (q1 << 8) | (q2 << 16) | (q3 << 24);
        const unsigned LN = Q - opaque((Q << 8) - 1u), sh = q3 + 1u;
        TU += sh;
        if (g < 15) {
#pragma unroll
            for (int i = 0; i < 7; ++i) if (32 * i < (int)LANE_U_MAX - 4 * (g + 1)) S[i] = (i < 6 && 32 * (i + 1) < (int)LANE_U_MAX - 4 * g) ? __builtin_amdgcn_alignbit(S[i + 1], S[i], sh) : S[i] >> sh;
        }
        const unsigned sel = bfi_c(0x04040404u, R[g] >> 4, bfi(bytemask(LN + 0x7D7D7D7Du, 7, c0c), 0x03030303u, LN));
        SS[g] = perm(STN, STP, sel);
        RB[g] = perm(OTN, OTP, sel) + (LN << 1);
        TX = __builtin_amdgcn_sad_u8(SS[g], 0u, TX);
    }
    const unsigned xstart = (__builtin_amdgcn_readlane(p0 + TU, 63) + 31u) & ~31u, px = xstart + wave_scan_incl(TX) - TX;
    unsigned X[5];
    {
        LAS const unsigned* q = (LAS const unsigned*)(aux + ((px >> 5) << 2));
#pragma unroll
        for (int i = 0; i < 5; ++i) X[i] = q[i];
#pragma unroll
        for (int i = 0; i < 4; ++i) X[i] = __builtin_amdgcn_alignbit(X[i + 1], X[i], px & 31u);
    }
    unsigned xoff = 0;
#pragma unroll
    for (int g = 0; g < 16; ++g) {
        const unsigned s = SS[g], M = perm(0u, 0x00030100u, s);
        const unsigned y = (g & 1) ? X[0] >> xoff : X[0], Y = perm(y, y, 0x01000100u);
        const unsigned t1 = opaque(s << 8), t2 = opaque((t1 << 8) + t1), Cs = (t2 << 16) + t2;
        const u16x2 z02 = __builtin_bit_cast(u16x2, Y) >> __builtin_bit_cast(u16x2, Cs & 0x00FF00FFu);
        const u16x2 z13 = __builtin_bit_cast(u16x2, Y) >> __builtin_bit_cast(u16x2, perm(0u, Cs, 0x0C030C01u));
        const unsigned suf = perm(__builtin_bit_cast(unsigned, z13), __builtin_bit_cast(unsigned, z02), 0x06020400u) & M;
        if (g & 1) {
            if (g < 15) {
                const unsigned t = __builtin_amdgcn_sad_u8(s, 0u, xoff);
#pragma unroll
                for (int i = 0; i < 4; ++i) if (32 * i < 8 * (15 - g)) X[i] = (i < 3 && 32 * (i + 1) < 8 * (17 - g)) ? __builtin_amdgcn_alignbit(X[i + 1], X[i], t) : X[i] >> t;
            }
        } else xoff = sad4(s);
        const unsigned rank = RB[g] + suf - 0x06060606u, ms = bytemask(R[g], 6, c0c);
        const unsigned D = bfi(bytemask(rank, 4, c0c), rank, lut2x16(I, rank, ms, c0c));
        const unsigned E = bfi(ms, BN, BP) - ((D >> 1) & 0x0F0F0F0Fu);
        const unsigned H = bfi_c(0x80808080u, R[g] << 1, E >> 1), L = bfi_c(0x3F3F3F3Fu, R[g], bfi_c(0x40404040u, D << 6, E << 7));
        const unsigned x = perm(H, L, 0x05010400u), yv = perm(H, L, 0x07030602u);
        if (g & 1) { c[g >> 1].z = x; c[g >> 1].w = yv; } else { c[g >> 1].x = x; c[g >> 1].y = yv; }
    }
}

typedef unsigned u32x2 __attribute__((ext_vector_type(2)));
__device__ __forceinline__ void store_granule(char* p, u32x2 g) {
    asm volatile("global_store_dwordx2 %0, %1, off sc0 sc1\n\ts_nop 1" :: "v"(p), "v"(g) : "memory");
}

constexpr unsigned GCTL = 131072;
__device__ __forceinline__ unsigned payload_bytes(unsigned h) { return (h >> 31) ? 8192u : RAWB + ((h & 255u) << 4); }
__device__ __forceinline__ void publish_block(const Params& P, rsrc_t wsr, const u32x4 (&c)[8], unsigned slot, unsigned n, unsigned dmask, unsigned call, LAS unsigned char* lds, LAS unsigned char* scr,
                                              unsigned j, unsigned gsize, int wid, int lane) {
    unsigned R[14], h = 1u << 31; u32x4 a[3];
    if (n >= NRAW && n < NSEQ - NRAWEND) h = encode_block(c, R, a, scr, lane);
    const unsigned u = j & 3u;
    volatile LAS unsigned* ctl = (volatile LAS unsigned*)(lds + GCTL);
    while (j >= 4u && (int)(ctl[36 + u] - (j - 3u)) < 0) __builtin_amdgcn_s_sleep(1);
    unsigned so = OFF_SLOT + (slot & ~1u) * 8192u;
    if (wid & 1) {
        unsigned v;
        while (((v = ctl[40 + u * 8 + wid - 1]) >> 16) != ((j + 1u) & 0xFFFFu)) __builtin_amdgcn_s_sleep(1);
        so += (rfl(v) & 0xFFFFu) << 4;
    } else if (lane == 0) ctl[40 + u * 8 + wid] = ((j + 1u) << 16) | (payload_bytes(h) >> 4);
    if (h >> 31) {
#pragma unroll
        for (int q = 0; q < 8; ++q) __builtin_amdgcn_raw_buffer_store_b128(c[q], wsr, lane * 16 + q * 1024, so, PAY_AUX);
    } else {
        const unsigned n16 = h & 255u;
#pragma unroll
        for (int q = 0; q < 3; ++q) __builtin_amdgcn_raw_buffer_store_b128((u32x4){R[4 * q], R[4 * q + 1], R[4 * q + 2], R[4 * q + 3]}, wsr, lane * 16 + q * 1024, so, PAY_AUX);
        __builtin_amdgcn_raw_buffer_store_b64((u32x2){R[12], R[13]}, wsr, lane * 8 + 3072, so, PAY_AUX);
        if ((unsigned)lane < n16) __builtin_amdgcn_raw_buffer_store_b128(a[0], wsr, lane * 16 + RAWB, so, PAY_AUX);
        if ((unsigned)lane + 64u < n16) __builtin_amdgcn_raw_buffer_store_b128(a[1], wsr, lane * 16 + RAWB + 1024, so, PAY_AUX);
        if ((unsigned)lane + 128u < n16) __builtin_amdgcn_raw_buffer_store_b128(a[2], wsr, lane * 16 + RAWB + 2048, so, PAY_AUX);
    }
    WAIT_VM0();
    unsigned arrived = 0;
    if (lane == 0) {
        ctl[u * 8 + wid] = h;
        arrived = __hip_atomic_fetch_add((LAS unsigned*)(lds + GCTL) + 32 + u, 1u, __ATOMIC_RELAXED, __HIP_MEMORY_SCOPE_WORKGROUP);
    }
    if (rfl(arrived) == gsize - 1u) {
        const unsigned d = lane >> 3, w = lane & 7, hw = ctl[u * 8 + w];
        if (w < gsize && ((dmask >> d) & 1u)) store_granule(P.ws[d] + OFF_GRAN + ((unsigned)P.rank * NSEQ + n - wid + w) * 8u, (u32x2){call, hw});
        if (lane == 0) { ctl[32 + u] = 0u; ctl[36 + u] = j + 1u; }
    }
}

__device__ __forceinline__ void signal_cnt(char* ws, unsigned off, int lane) {
    if (off && lane == 0) __hip_atomic_fetch_add((unsigned*)(ws + off), 1u, __ATOMIC_RELAXED, __HIP_MEMORY_SCOPE_SYSTEM);
}
__device__ __forceinline__ unsigned packer_role(const Params& P, LAS unsigned char* lds, int first, int next, int step, int end) {
    unsigned call = 0;
    const int lane = threadIdx.x & 63, wid = rfl(threadIdx.x >> 6);
    char* ws = P.ws[P.rank];
    const rsrc_t wsr = mk_rsrc(ws, WS_USED);
    const rsrc_t xr = mk_rsrc(P.x, DIM * KSH * 2), wr = mk_rsrc(P.w, DIM * KSH * 2);
    const int myi = P.rank >> 2, myj = P.rank & 3;
    LAS unsigned char* my = lds + wid * 8192;
    unsigned j = first < NFIRST ? 0u : 1u;
    const unsigned long long tstart = __builtin_amdgcn_s_memrealtime();
    for (int p = first; p < end; ++j) {
        if (p < 4096) {
            const int due = (p < 3072 ? (p / 192) * 580 : 9280 + ((p - 3072) >> 6) * 290) - JIT_LEAD * 100;
            while ((int)(__builtin_amdgcn_s_memrealtime() - tstart) < due) __builtin_amdgcn_s_sleep(16);
        }
        u32x4 c[8];
        unsigned slot, n, dmask, selfcnt = 0;
        if (p >= 4096) {
            const unsigned o = p - 4096, w = o & 7, h = (o >> 3) & 1, tile = (o >> 4) & 63;
            if (call == 0) call = call_draw(ws);
            unsigned* tmo = (unsigned*)(ws + OFF_TMO);
            wait_word((unsigned*)(ws + OFF_OCNT + (o >> 3) * 128), call * 8u, tmo, call);
            const unsigned row0 = (P.rank >> 2) * 4096 + (tile >> 3) * 256 + h * 128 + (w >> 2) * 64, col0 = (P.rank & 3) * 2048 + (tile & 7) * 256 + (w & 3) * 64;
            const rsrc_t outr = mk_rsrc(P.out, (unsigned)DIM * DIM * 2u);
#pragma unroll
            for (int q = 0; q < 8; ++q) c[q] = __builtin_amdgcn_raw_buffer_load_b128(outr, (unsigned)(lane >> 3) * (DIM * 2u) + (unsigned)(lane & 7) * 16u, (row0 * DIM + col0) * 2u + q * (8u * DIM * 2u), AUX_SYS);
            if (rfl(__hip_atomic_load(tmo, __ATOMIC_RELAXED, __HIP_MEMORY_SCOPE_AGENT)) == call) {
#pragma unroll
                for (int q = 0; q < 8; ++q) c[q] = (u32x4){0x7FC07FC0u, 0x7FC07FC0u, 0x7FC07FC0u, 0x7FC07FC0u};
            }
            slot = 4096u + o; n = 1536u + o; dmask = 0xFFu;
        } else {
            bool is_w; int kk, sel, r, bi;
            if (p < 3072) {
                kk = p / 192; const int rem = p - kk * 192;
                bi = rem & 31; r = 0; is_w = rem >= 64; sel = is_w ? ((rem - 64) >> 5) : (rem >> 5);
            } else { const int q = p - 3072; kk = q >> 6; sel = (q >> 5) & 1; bi = q & 31; r = 1; is_w = false; }
            if (!is_w) {
                const int rr = lane >> 3;
                const unsigned vo = (unsigned)rr * 2048u + (unsigned)(((lane & 7) ^ rr) * 16);
                const unsigned so = (unsigned)(sel * 4096 + r * 2048 + bi * 64) * 2048u + (unsigned)kk * 128u;
#pragma unroll
                for (int q = 0; q < 8; ++q) c[q] = __builtin_amdgcn_raw_buffer_load_b128(xr, vo + q * 8 * 2048, so, XIN_AUX);
            } else {
                const int pn = bi >> 2, bj = (bi >> 1) & 1, q = bi & 1;
                const int gk = lane >> 3, cc = lane & 7, wcl = cc >> 2, cq = cc & 3;
                const unsigned vo = (unsigned)(gk * 8) * 16384u + (unsigned)((64 * (2 * q + wcl) + 32 * bj + 8 * cq) * 2);
                const unsigned so = (unsigned)(kk * 64) * 16384u + (unsigned)((sel * 2048 + pn * 256) * 2);
                u32x4 R[8];
#pragma unroll
                for (int jj = 0; jj < 8; ++jj) R[jj] = __builtin_amdgcn_raw_buffer_load_b128(wr, vo + jj * 16384, so, XIN_AUX);
#pragma unroll
                for (int t = 0; t < 8; ++t) {
                    const unsigned s = (t & 1) ? 0x07060302u : 0x05040100u;
                    u32x4 v;
                    v.x = perm(R[1][t >> 1], R[0][t >> 1], s); v.y = perm(R[3][t >> 1], R[2][t >> 1], s);
                    v.z = perm(R[5][t >> 1], R[4][t >> 1], s); v.w = perm(R[7][t >> 1], R[6][t >> 1], s);
                    const int s64 = 32 * wcl + 16 * (t >> 2) + 4 * cq + (t & 3);
                    *(LAS u32x4*)(my + s64 * 128 + ((gk ^ (s64 & 7)) * 16)) = v;
                }
                asm volatile("s_waitcnt lgkmcnt(0)" ::: "memory");
#pragma unroll
                for (int q2 = 0; q2 < 8; ++q2) c[q2] = *(LAS u32x4*)(my + q2 * 1024 + lane * 16);
                asm volatile("s_waitcnt lgkmcnt(0)" ::: "memory");
            }
            if (call == 0) call = call_draw(ws);
            const int g = P.rank * 16 + kk;
            if (is_w ? (sel == myj) : (sel == myi)) {
                const unsigned so = is_w ? OFF_BIMG + (unsigned)(g * 32 + bi) * 8192u : OFF_AIMG + (unsigned)((r * 128 + g) * 32 + bi) * 8192u;
#pragma unroll
                for (int q = 0; q < 8; ++q) __builtin_amdgcn_raw_buffer_store_b128(c[q], wsr, lane * 16 + q * 1024, so, AUX_SC1);
                selfcnt = OFF_CNT + (r * 128 + g) * 128;
            }
            slot = is_w ? 2048u + (unsigned)((sel * 16 + kk) * 32 + bi) : (unsigned)(((sel * 2 + r) * 16 + kk) * 32 + bi);
            n = is_w ? (unsigned)(kk * 64 + 32 + bi) : (r == 0 ? (unsigned)(kk * 64 + bi) : (unsigned)(1024 + kk * 32 + bi));
            dmask = is_w ? (0x11u << sel) : (0x0Fu << (4 * sel));
        }
        publish_block(P, wsr, c, slot, n, dmask & ~(1u << P.rank), call, lds, my, j, (p < NFIRST || (p >= 4096 && p < OUT2)) ? 4u : 8u, wid, lane);
        signal_cnt(ws, selfcnt, lane);
        if (p < NFIRST) p = next;
        else if (p < 4096) {
            p += step;
            if (p >= 4096 && end > 4096) { if (wid < 4) p = 4096 + (int)blockIdx.x * 4 + wid; else { p = OUT2 + (int)blockIdx.x * 8 + wid; ++j; } }
        } else if (p < OUT2) p = OUT2 + (int)blockIdx.x * 8 + wid;
        else p += step;
    }
    if (call == 0) call = call_draw(ws);
    return call;
}

struct Blk { unsigned slot, dso, cnt; bool is_out; };
__device__ __forceinline__ Blk locate(unsigned n, int src, int myi, int myj) {
    Blk b; b.is_out = false;
    if (n < 1024) { const int kk = n >> 6, bi = n & 31, g = src * 16 + kk; b.cnt = OFF_CNT + g * 128;
        if ((n & 63) >= 32) { b.slot = 2048u + (myj * 16 + kk) * 32 + bi; b.dso = OFF_BIMG + (unsigned)(g * 32 + bi) * 8192u; }
        else { b.slot = ((myi * 2) * 16 + kk) * 32 + bi; b.dso = OFF_AIMG + (unsigned)(g * 32 + bi) * 8192u; } }
    else if (n < 1536) { const int kk = (n - 1024) >> 5, bi = n & 31, g = src * 16 + kk; b.cnt = OFF_CNT + (128 + g) * 128;
        b.slot = ((myi * 2 + 1) * 16 + kk) * 32 + bi; b.dso = OFF_AIMG + (unsigned)((128 + g) * 32 + bi) * 8192u; }
    else { const int o = n - 1536, w = o & 7, h = (o >> 3) & 1, tile = (o >> 4) & 63, r = o >> 10; b.cnt = 0; b.is_out = true;
        b.slot = 4096u + o;
        const unsigned row0 = (src >> 2) * 4096 + r * 2048 + (tile >> 3) * 256 + h * 128 + (w >> 2) * 64, col0 = (src & 3) * 2048 + (tile & 7) * 256 + (w & 3) * 64;
        b.dso = (row0 * DIM + col0) * 2u; }
    return b;
}
constexpr unsigned NPAIR = NSEQ / 2, NSLOT = 7, CTL = NSLOT * 16384;
constexpr unsigned M_DEAD = 1u << 30, N_STOP = 0xFFFFFFFFu;
__device__ __forceinline__ unsigned lds_ld(LAS unsigned char* p) { return *(volatile LAS unsigned*)p; }
__device__ __forceinline__ void lds_st(LAS unsigned char* p, unsigned v) { *(volatile LAS unsigned*)p = v; }
__device__ __forceinline__ void ring_publish(LAS unsigned char* lds, unsigned i, unsigned n, unsigned ha, unsigned hb, int lane) {
    const unsigned sl = i % NSLOT;
    if (lane == 0) { *(volatile LAS u32x4*)(lds + CTL + sl * 16) = (u32x4){n, ha, hb, 0u}; lds_st(lds + CTL + 128 + sl * 4, i + 1u); }
}
__device__ __forceinline__ bool slot_busy(LAS unsigned char* lds, unsigned i) { return i >= NSLOT && (int)(lds_ld(lds + CTL + 160 + (i % NSLOT) * 4) - 2u * (i / NSLOT)) < 0; }
__device__ __forceinline__ void dma_op(rsrc_t r, LAS unsigned char* dst, unsigned j, unsigned so, int lane) {
    __builtin_amdgcn_raw_ptr_buffer_load_lds(r, (LAS void*)(dst + j * 1024), 16, lane * 16, so + j * 1024, 0, AUX_SYS);
}
__device__ __forceinline__ u32x4 load_pair(rsrc_t wsr, unsigned off) { return __builtin_amdgcn_raw_buffer_load_b128(wsr, 0, off, AUX_SYS); }
#define VM_WAIT_(n) asm volatile("s_waitcnt vmcnt(" #n ")" ::: "memory")
#define VM_WAIT(n) VM_WAIT_(n)
#define G_STR_(x) #x
#define G_STR(x) G_STR_(x)
#define WIN 5

__device__ __forceinline__ void loader_wave(const Params& P, LAS unsigned char* lds, unsigned call, int src, int lane) {
    char* ws = P.ws[P.rank];
    unsigned* tmo = (unsigned*)(ws + OFF_TMO);
    const rsrc_t wsr = mk_rsrc(ws, WS_USED);
    char* const remp = P.ws[src];
    const int myi = P.rank >> 2, myj = P.rank & 3;
    const unsigned gbase = OFF_GRAN + (unsigned)src * NSEQ * 8u;
    char* const gptr = ws + gbase;
    const unsigned tko = OFF_LINK + (unsigned)src * 128u, tbase = (call - 1u) * (NPAIR + 3u * PWL);
    __builtin_amdgcn_s_setprio(3);
    unsigned q, q1, q2;
    {
        unsigned t0 = 0;
        if (lane == 0) t0 = __hip_atomic_fetch_add((unsigned*)(ws + tko), 3u, __ATOMIC_RELAXED, __HIP_MEMORY_SCOPE_AGENT);
        q = rfl(t0) - tbase; q1 = q + 1; q2 = q + 2;
    }
    {
        const rsrc_t rw = mk_rsrc(remp, OFF_AIMG);
        const unsigned v = __builtin_amdgcn_raw_buffer_load_b32(rw, lane < 25 ? OFF_SLOT + (unsigned)lane * (2u << 20) + 8188u - (lane == 24 ? 8192u : 0u) : 0xFFFFFFF0u, 0, AUX_SYS);
        asm volatile("" :: "v"(v));
    }
    unsigned gca, gha, gcb, ghb, hca, hha, hcb, hhb;
    { const u32x4 a = load_pair(wsr, gbase + (q < NPAIR ? q : 0u) * 16u), b = load_pair(wsr, gbase + (q1 < NPAIR ? q1 : 0u) * 16u);
      gca = rfl(a.x); gha = rfl(a.y); gcb = rfl(a.z); ghb = rfl(a.w); hca = rfl(b.x); hha = rfl(b.y); hcb = rfl(b.z); hhb = rfl(b.w); }
    const int zero = 0, one = 1;
    unsigned i = 0;
    bool pend = false; unsigned pn = 0, pa = 0, pb = 0;
    while (q < NPAIR) {
        bool dead = false;
        if ((int)(gca - call) < 0 || (int)(gcb - call) < 0 || slot_busy(lds, i)) {
            if (pend) { VM_WAIT(0); ring_publish(lds, i - 1u, pn, pa, pb, lane); pend = false; }
            const unsigned long long t0 = __builtin_amdgcn_s_memrealtime();
            while ((int)(gca - call) < 0 || (int)(gcb - call) < 0) {
                const u32x4 a = load_pair(wsr, gbase + q * 16u);
                gca = rfl(a.x); gha = rfl(a.y); gcb = rfl(a.z); ghb = rfl(a.w);
                if ((int)(gca - call) >= 0 && (int)(gcb - call) >= 0) break;
                if (give_up(t0, tmo, call)) { dead = true; break; }
                __builtin_amdgcn_s_sleep(2);
                asm volatile("" ::: "memory");
            }
            while (slot_busy(lds, i)) __builtin_amdgcn_s_sleep(1);
        }
        const unsigned n = 2u * q;
        const unsigned so = OFF_SLOT + locate(n, src, myi, myj).slot * 8192u;
        const unsigned bytes = dead ? 0u : payload_bytes(gha) + payload_bytes(ghb), nops = (bytes + 1023u) >> 10;
        const rsrc_t rem = mk_rsrc(remp, so + bytes);
        LAS unsigned char* dst = lds + (i % NSLOT) * 16384;
        u32x4 g2; unsigned tk = 0;
        asm volatile("global_load_dwordx4 %0, %1, %2 sc0 sc1" : "=v"(g2) : "v"(zero), "s"(gptr + (q2 < NPAIR ? q2 : 0u) * 16u) : "memory");
        if (lane == 0) asm volatile("global_atomic_add %0, %1, %2, %3 sc0" : "=v"(tk) : "v"(zero), "v"(one), "s"(ws + tko) : "memory");
#define G_FILL(jj) do { if ((jj) == WIN) asm volatile("s_waitcnt vmcnt(" G_STR(WIN) ")" : "+v"(g2), "+v"(tk) :: "memory"); else VM_WAIT(WIN); \
                        if ((jj) == WIN - 2 && pend) ring_publish(lds, i - 1u, pn, pa, pb, lane); dma_op(rem, dst, jj, so, lane); } while (0)
        G_FILL(0); G_FILL(1); G_FILL(2); G_FILL(3); G_FILL(4); G_FILL(5); G_FILL(6); G_FILL(7);
#undef G_FILL
        for (unsigned j = 8; j < nops; ++j) { VM_WAIT(WIN); dma_op(rem, dst, j, so, lane); }
        pend = true; pn = n; pa = gha | (dead ? M_DEAD : 0u); pb = ghb | (dead ? M_DEAD : 0u);
        q = q1; gca = hca; gha = hha; gcb = hcb; ghb = hhb;
        q1 = q2; hca = rfl(g2.x); hha = rfl(g2.y); hcb = rfl(g2.z); hhb = rfl(g2.w);
        q2 = rfl(tk) - tbase;
        ++i;
    }
    VM_WAIT(0);
    if (pend) ring_publish(lds, i - 1u, pn, pa, pb, lane);
    for (int d = 0; d < (NDEC + 1) / 2 + 1; ++d, ++i) {
        while (slot_busy(lds, i)) __builtin_amdgcn_s_sleep(1);
        ring_publish(lds, i, N_STOP, 0u, 0u, lane);
    }
}

__device__ __forceinline__ void decoder_wave(const Params& P, LAS unsigned char* lds, unsigned call, int src, int d, int lane) {
    char* ws = P.ws[P.rank];
    unsigned* tmo = (unsigned*)(ws + OFF_TMO);
    const rsrc_t wsr = mk_rsrc(ws, WS_USED), outr = mk_rsrc(P.out, (unsigned)DIM * DIM * 2u);
    const int myi = P.rank >> 2, myj = P.rank & 3;
    for (unsigned i = d;; i += NDEC) {
        const unsigned pi = i >> 1, sl = pi % NSLOT;
        while (lds_ld(lds + CTL + 128 + sl * 4) != pi + 1u) __builtin_amdgcn_s_sleep(1);
        asm volatile("" ::: "memory");
        const u32x4 meta = *(volatile LAS u32x4*)(lds + CTL + sl * 16);
        const unsigned n0 = rfl(meta.x), ha = rfl(meta.y), h = (i & 1) ? rfl(meta.z) : ha;
        if (n0 == N_STOP) break;
        const Blk b = locate(n0 + (i & 1), src, myi, myj);
        const bool raw = (h >> 31) != 0;
        bool dead = (h & M_DEAD) != 0;
        if (b.is_out && rfl(__hip_atomic_load(tmo, __ATOMIC_RELAXED, __HIP_MEMORY_SCOPE_AGENT)) == call) dead = true;
        u32x4 c[8];
        unsigned recs = 0;
        const unsigned ne = (dead || raw) ? 0u : (h >> 8) & 63u;
        LAS unsigned char* cur = lds + sl * 16384 + ((i & 1) ? payload_bytes(ha) : 0u);
        if (dead) {
#pragma unroll
            for (int j = 0; j < 8; ++j) c[j] = (u32x4){0x7FC07FC0u, 0x7FC07FC0u, 0x7FC07FC0u, 0x7FC07FC0u};
        } else if (raw) {
#pragma unroll
            for (int j = 0; j < 8; ++j) c[j] = *(LAS u32x4*)(cur + j * 1024 + lane * 16);
        } else {
            recs = *(LAS unsigned*)(cur + RAWB + (((h >> 16) & 0x3FFu) + (lane & 31)) * 4u);
            decode_block(cur, lds + CTL + 256 + d * 256, c, lane);
        }
#pragma unroll
        for (int j = 0; j < 8; ++j) asm volatile("" : "+v"(c[j]));
        asm volatile("" : "+v"(recs));
        if (lane == 0) __hip_atomic_fetch_add((LAS unsigned*)(lds + CTL + 160 + sl * 4), 1u, __ATOMIC_RELAXED, __HIP_MEMORY_SCOPE_WORKGROUP);
        const unsigned sj = b.is_out ? 8u * DIM * 2u : 1024u, sr = b.is_out ? DIM * 2u : 128u;
        const unsigned vo = (unsigned)(lane >> 3) * sr + (unsigned)(lane & 7) * 16u;
        if (b.is_out) {
#pragma unroll
            for (int j = 0; j < 8; ++j) __builtin_amdgcn_raw_buffer_store_b128(c[j], outr, vo, b.dso + j * sj, OUT_AUX);
            for (unsigned e = 0; e < ne; ++e) {
                const unsigned rec = __builtin_amdgcn_readlane(recs, e), pos = rec & 0xFFFu, v = pos & 63u;
                if ((unsigned)lane == (pos >> 6)) __builtin_amdgcn_raw_buffer_store_b16((unsigned short)((rec >> 12) & 0xFFFFu), outr, vo + (v & 7u) * 2u, b.dso + (v >> 3) * sj, OUT_AUX);
            }
        } else {
#pragma unroll
            for (int j = 0; j < 8; ++j) __builtin_amdgcn_raw_buffer_store_b128(c[j], wsr, vo, b.dso + j * sj, IMG_AUX);
            for (unsigned e = 0; e < ne; ++e) {
                const unsigned rec = __builtin_amdgcn_readlane(recs, e), pos = rec & 0xFFFu, v = pos & 63u;
                if ((unsigned)lane == (pos >> 6)) __builtin_amdgcn_raw_buffer_store_b16((unsigned short)((rec >> 12) & 0xFFFFu), wsr, vo + (v & 7u) * 2u, b.dso + (v >> 3) * sj, IMG_AUX);
            }
            WAIT_VM0(); signal_cnt(ws, b.cnt, lane);
        }
    }
}

__device__ __forceinline__ void puller_role(const Params& P, LAS unsigned char* lds, unsigned call, int wg) {
    const int lane = threadIdx.x & 63, wid = rfl(threadIdx.x >> 6);
    const int link = wg % 7, src = rfl((P.rank + 1 + link) & 7);
    if (threadIdx.x < 16) *(volatile LAS unsigned*)(lds + CTL + 128 + threadIdx.x * 4) = 0u;
    __syncthreads();
    if (wid == 0) loader_wave(P, lds, call, src, lane);
    else if (wid <= NDEC) decoder_wave(P, lds, call, src, wid - 1, lane);
}

__device__ __forceinline__ float gelu_f(float y) {
    const float c1 = -2.0f * 0.7978845608028654f * 1.4426950408889634f, c2 = c1 * 0.044715f;
    const float e = __builtin_amdgcn_exp2f(y * (c1 + c2 * y * y));
    return y * __builtin_amdgcn_rcpf(1.0f + e);
}
__device__ __forceinline__ unsigned cvt_pk_bf16(float lo, float hi) { unsigned r; asm("v_cvt_pk_bf16_f32 %0, %1, %2" : "=v"(r) : "v"(lo), "v"(hi)); return r; }

__device__ __forceinline__ void gemm_role(const Params& P, LAS unsigned char* lds, unsigned call, int wg) {
    const int tid = threadIdx.x, wid = rfl(tid >> 6), lane = tid & 63, wr = wid >> 2, wc = wid & 3, fr = lane & 15, fq = lane >> 4;
    char* ws = P.ws[P.rank];
    unsigned* tmo = (unsigned*)(ws + OFF_TMO);
    const rsrc_t wsr = mk_rsrc(ws, WS_USED);
    const int xcd = wg & 7, q = (wg - NPACK) >> 3, h = xcd & 1, blk = xcd >> 1;
    const int tm = (blk >> 1) * 4 + (q >> 2), tn = (blk & 1) * 4 + (q & 3), tile = tm * 8 + tn;
    const unsigned voff = (unsigned)tid * 16u, ldsw = (unsigned)wid * 1024u;
    const int aoff0 = (wr * 64 + fr) * 128 + ((fq ^ (fr & 7)) << 4), aoff1 = aoff0 ^ 64;
    const int boff0 = (wc * 32 + fr) * 128 + ((fq ^ (fr & 7)) << 4), boff1 = boff0 ^ 64;
#define G_SA(b, hh) (((b) * 2 + (hh)) * 16384)
#define G_SB(b, hh) ((4 + (b) * 2 + (hh)) * 16384)
#define G_STAGE(bufoff, soff) do { _Pragma("unroll") for (int _i = 0; _i < 2; ++_i) \
        __builtin_amdgcn_raw_ptr_buffer_load_lds(wsr, (LAS void*)(lds + (bufoff) + ldsw + _i * 8192), 16, voff, (soff) + _i * 8192, 0, STG_AUX); } while (0)
#define G_LDA(dst, b, hh) do { _Pragma("unroll") for (int m = 0; m < 4; ++m) { \
        dst[m][0] = *(const LAS bf16x8*)(lds + G_SA(b, hh) + aoff0 + m * 2048); dst[m][1] = *(const LAS bf16x8*)(lds + G_SA(b, hh) + aoff1 + m * 2048); } } while (0)
#define G_LDB(dst, b, hh) do { _Pragma("unroll") for (int n = 0; n < 2; ++n) { \
        dst[n][0] = *(const LAS bf16x8*)(lds + G_SB(b, hh) + boff0 + n * 2048); dst[n][1] = *(const LAS bf16x8*)(lds + G_SB(b, hh) + boff1 + n * 2048); } } while (0)
#define G_MMA(ai, bj, At, Bt) do { __builtin_amdgcn_s_setprio(1); _Pragma("unroll") for (int m = 0; m < 4; ++m) _Pragma("unroll") for (int n = 0; n < 2; ++n) _Pragma("unroll") for (int k = 0; k < 2; ++k) \
        acc[ai][bj][m][n] = __builtin_amdgcn_mfma_f32_16x16x32_bf16(Bt[n][k], At[m][k], acc[ai][bj][m][n], 0, 0, 0); __builtin_amdgcn_s_setprio(0); } while (0)
#define G_WAIT_V(n) asm volatile("s_waitcnt vmcnt(" #n ")" ::: "memory")
#define G_WAIT_L(n) asm volatile("s_waitcnt lgkmcnt(" #n ")" ::: "memory")
#define G_BAR __builtin_amdgcn_s_barrier()
#define G_SCHED __builtin_amdgcn_sched_barrier(0)
#define G_GIDX(t) ((unsigned)((4 * h + ((t) & 3)) * 16 + ((t) >> 2)))
    const int zero = 0;
    for (int r = 0; r < 2; ++r) {
        const unsigned target = call * (r == 0 ? 64u : 32u);
        const unsigned abase = OFF_AIMG + (unsigned)r * (128u * 262144u) + (unsigned)tm * 32768u, bbase = OFF_BIMG + (unsigned)tn * 32768u;
        char* cbase = ws + OFF_CNT + r * 128 * 128;
        f32x4 acc[2][2][4][2];
#pragma unroll
        for (int a = 0; a < 2; ++a)
#pragma unroll
            for (int b = 0; b < 2; ++b)
#pragma unroll
                for (int m = 0; m < 4; ++m)
#pragma unroll
                    for (int n = 0; n < 2; ++n) acc[a][b][m][n] = (f32x4){0.f, 0.f, 0.f, 0.f};
        bf16x8 At[4][2], B0[2][2], B1[2][2];
        STAMP(ws, r * 8 + 0);
        wait_word((unsigned*)(cbase + G_GIDX(0) * 128), target, tmo, call);
        wait_word((unsigned*)(cbase + G_GIDX(1) * 128), target, tmo, call);
        wait_word((unsigned*)(cbase + G_GIDX(2) * 128), target, tmo, call);
        STAMP(ws, r * 8 + 1);
        {
            const unsigned a0 = abase + G_GIDX(0) * 262144u, b0 = bbase + G_GIDX(0) * 262144u, a1 = abase + G_GIDX(1) * 262144u, b1 = bbase + G_GIDX(1) * 262144u;
            G_STAGE(G_SB(0, 0), b0); G_STAGE(G_SA(0, 0), a0); G_STAGE(G_SB(0, 1), b0 + 16384); G_STAGE(G_SA(0, 1), a0 + 16384);
            if (wr == 1) G_BAR;
            G_WAIT_V(4); G_BAR;
            G_STAGE(G_SB(1, 0), b1); G_STAGE(G_SA(1, 0), a1); G_STAGE(G_SB(1, 1), b1 + 16384);
            G_WAIT_V(6); G_BAR;
        }
        unsigned fA = target, fB;
        for (int t = 0; t < 64; t += 2) {
            const int t2 = t + 2 < 64 ? t + 2 : 63, t3 = t + 3 < 64 ? t + 3 : 63, t4 = t + 4 < 64 ? t + 4 : 63;
            const unsigned g1 = G_GIDX(t + 1), g2 = G_GIDX(t2), g3 = G_GIDX(t3), g4 = G_GIDX(t4);
            const unsigned a1 = abase + g1 * 262144u, a2 = abase + g2 * 262144u, b2 = bbase + g2 * 262144u, a3 = abase + g3 * 262144u, b3 = bbase + g3 * 262144u;
            asm volatile("global_load_dword %0, %1, %2 sc0 sc1" : "=v"(fB) : "v"(zero), "s"(cbase + g3 * 128));
            G_LDB(B0, 0, 0); G_SCHED; G_LDA(At, 0, 0); G_STAGE(G_SA(1, 1), a1 + 16384);
            G_WAIT_L(8); G_BAR; G_WAIT_L(0); G_MMA(0, 0, At, B0); G_BAR; G_SCHED;
            asm volatile("" : "+v"(fA));
            if ((int)(rfl(fA) - target) < 0) wait_word((unsigned*)(cbase + g2 * 128), target, tmo, call);
            G_LDB(B1, 0, 1); G_STAGE(G_SB(0, 0), b2);
            G_BAR; G_WAIT_L(0); G_MMA(0, 1, At, B1); G_BAR;
            G_LDA(At, 0, 1); G_STAGE(G_SA(0, 0), a2);
            G_BAR; G_WAIT_L(0); G_MMA(1, 0, At, B0); G_BAR; G_SCHED;
            G_STAGE(G_SB(0, 1), b2 + 16384);
            G_WAIT_V(6); G_BAR; G_MMA(1, 1, At, B1); G_BAR;
            asm volatile("global_load_dword %0, %1, %2 sc0 sc1" : "=v"(fA) : "v"(zero), "s"(cbase + g4 * 128));
            G_LDB(B0, 1, 0); G_SCHED; G_LDA(At, 1, 0); G_STAGE(G_SA(0, 1), a2 + 16384);
            G_WAIT_L(8); G_BAR; G_WAIT_L(0); G_MMA(0, 0, At, B0); G_BAR; G_SCHED;
            asm volatile("" : "+v"(fB));
            if ((int)(rfl(fB) - target) < 0) wait_word((unsigned*)(cbase + g3 * 128), target, tmo, call);
            G_LDB(B1, 1, 1); G_STAGE(G_SB(1, 0), b3);
            G_BAR; G_WAIT_L(0); G_MMA(0, 1, At, B1); G_BAR;
            G_LDA(At, 1, 1); G_STAGE(G_SA(1, 0), a3);
            G_BAR; G_WAIT_L(0); G_MMA(1, 0, At, B0); G_BAR; G_SCHED;
            G_STAGE(G_SB(1, 1), b3 + 16384);
            G_WAIT_V(6); G_BAR; G_MMA(1, 1, At, B1); G_BAR;
        }
        G_WAIT_V(0);
        asm volatile("" : "+v"(fA));
        if (wr == 0) G_BAR;
        G_BAR;
        STAMP(ws, r * 8 + 2);
        int lane = threadIdx.x & 63;
        asm volatile("" : "+v"(lane));
        const int fr = lane & 15, fq = lane >> 4;
        if (h) {
#pragma unroll
            for (int b = 0; b < 2; ++b)
#pragma unroll
                for (int m = 0; m < 4; ++m)
#pragma unroll
                    for (int n = 0; n < 2; ++n) { const f32x4 tq = acc[0][b][m][n]; acc[0][b][m][n] = acc[1][b][m][n]; acc[1][b][m][n] = tq; }
        }
        const unsigned xl = (unsigned)((r * 64 + tile) * 2);
#pragma unroll
        for (int b = 0; b < 2; ++b)
#pragma unroll
            for (int m = 0; m < 4; ++m)
#pragma unroll
                for (int n = 0; n < 2; ++n)
                    __builtin_amdgcn_raw_buffer_store_b128(__builtin_bit_cast(u32x4, acc[1][b][m][n]), wsr, lane * 16 + ((b * 4 + m) * 2 + n) * 1024, OFF_XBUF + ((xl + h) * 8 + wid) * 16384u, AUX_SC1);
        WAIT_VM0();
        if (lane == 0) __hip_atomic_store((unsigned*)(ws + OFF_XFLAG + (xl + h) * 128 + wid * 16), call, __ATOMIC_RELAXED, __HIP_MEMORY_SCOPE_AGENT);
        wait_word((unsigned*)(ws + OFF_XFLAG + (xl + (h ^ 1)) * 128 + wid * 16), call, tmo, call);
        const bool dead = rfl(__hip_atomic_load(tmo, __ATOMIC_RELAXED, __HIP_MEMORY_SCOPE_AGENT)) == call;
        STAMP(ws, r * 8 + 3);
        LAS unsigned char* my = lds + wid * 8192;
#pragma unroll
        for (int b = 0; b < 2; ++b)
#pragma unroll
            for (int m = 0; m < 4; ++m) {
                const f32x4 p0 = __builtin_bit_cast(f32x4, __builtin_amdgcn_raw_buffer_load_b128(wsr, lane * 16 + ((b * 4 + m) * 2 + 0) * 1024, OFF_XBUF + ((xl + (h ^ 1)) * 8 + wid) * 16384u, AUX_SC1));
                const f32x4 p1 = __builtin_bit_cast(f32x4, __builtin_amdgcn_raw_buffer_load_b128(wsr, lane * 16 + ((b * 4 + m) * 2 + 1) * 1024, OFF_XBUF + ((xl + (h ^ 1)) * 8 + wid) * 16384u, AUX_SC1));
                const f32x4 v0 = acc[0][b][m][0] + p0, v1 = acc[0][b][m][1] + p1;
                u32x4 o;
                o.x = cvt_pk_bf16(gelu_f(v0[0]), gelu_f(v0[1])); o.y = cvt_pk_bf16(gelu_f(v0[2]), gelu_f(v0[3]));
                o.z = cvt_pk_bf16(gelu_f(v1[0]), gelu_f(v1[1])); o.w = cvt_pk_bf16(gelu_f(v1[2]), gelu_f(v1[3]));
                const int row = m * 16 + fr, ch = b * 4 + fq;
                *(LAS u32x4*)(my + row * 128 + ((ch ^ (row & 7)) * 16)) = o;
            }
        asm volatile("s_waitcnt lgkmcnt(0)" ::: "memory");
        u32x4 c[8];
#pragma unroll
        for (int j = 0; j < 8; ++j) { const int row = j * 8 + (lane >> 3); c[j] = *(LAS u32x4*)(my + row * 128 + (((lane & 7) ^ (row & 7)) * 16)); }
        if (dead) {
#pragma unroll
            for (int j = 0; j < 8; ++j) c[j] = (u32x4){0x7FC07FC0u, 0x7FC07FC0u, 0x7FC07FC0u, 0x7FC07FC0u};
        }
        {
            const rsrc_t outr = mk_rsrc(P.out, (unsigned)DIM * DIM * 2u);
            const unsigned row0 = (P.rank >> 2) * 4096 + r * 2048 + tm * 256 + h * 128 + wr * 64, col0 = (P.rank & 3) * 2048 + tn * 256 + wc * 64;
            const unsigned dso = (row0 * DIM + col0) * 2u, vo = (unsigned)(lane >> 3) * (DIM * 2u) + (unsigned)(lane & 7) * 16u;
            if (r == 0) {
#pragma unroll
                for (int j = 0; j < 8; ++j) __builtin_amdgcn_raw_buffer_store_b128(c[j], outr, vo, dso + j * (8u * DIM * 2u), AUX_SC1);
            } else {
#pragma unroll
                for (int j = 0; j < 8; ++j) __builtin_amdgcn_raw_buffer_store_b128(c[j], outr, vo, dso + j * (8u * DIM * 2u), OUT_AUX);
            }
            const unsigned o = (xl + h) * 8 + wid;
            if (r == 0) { WAIT_VM0(); signal_cnt(ws, OFF_OCNT + (o >> 3) * 128, lane); }
            else publish_block(P, wsr, c, 4096u + o, 1536u + o, 0xFFu & ~(1u << P.rank), call, lds, my, 1u + GROUNDS, 8u, wid, lane);
        }
        STAMP(ws, r * 8 + 4);
        asm volatile("s_waitcnt lgkmcnt(0)" ::: "memory");
        G_BAR;
    }
#undef G_SA
#undef G_SB
#undef G_STAGE
#undef G_LDA
#undef G_LDB
#undef G_MMA
#undef G_WAIT_V
#undef G_WAIT_L
#undef G_BAR
#undef G_SCHED
#undef G_GIDX
}

__global__ __launch_bounds__(512, 2) void gemm_allgather_kernel(Params P) {
    extern __shared__ __attribute__((aligned(16))) unsigned char shm[];
    LAS unsigned char* lds = (LAS unsigned char*)shm;
    char* ws = P.ws[P.rank];
    const int b = rfl(blockIdx.x);
    STAMP(ws, 60);
    unsigned call;
    if (b >= NPACK + NGEMM) { call = call_draw(ws); puller_role(P, lds, rfl(call), rfl(b - NPACK - NGEMM)); }
    else {
        if (threadIdx.x < 40) *(volatile LAS unsigned*)(lds + GCTL + 128 + threadIdx.x * 4) = 0u;
        __syncthreads();
        const int wid = rfl(threadIdx.x >> 6);
        const bool gemm = b >= NPACK;
        const int next = gemm ? NFIRST + (b - NPACK) * 8 + wid : NFIRST + GROUNDS * NGEMM * 8 + b * 8 + wid;
        call = packer_role(P, lds, wid < 4 ? b * 4 + wid : next, next, gemm ? NGEMM * 8 : NPACK * 8, gemm ? NFIRST + GROUNDS * NGEMM * 8 : 4096 + 1024);
        if (gemm) { __syncthreads(); gemm_role(P, lds, rfl(call), rfl(b)); }
    }
    STAMP(ws, 61);
    STAMP(ws, 58 + (call & 1));
}

}

extern "C" void kernel_launch(int rank, int nranks, ncclComm_t comm,
                              hipStream_t stream,
                              void* const* d_in, const int* in_sizes, int n_in,
                              void* d_out, int out_size,
                              const int* in_dtypes, int out_dtype,
                              void* const* d_out_all,
                              void* const* d_ws_all, size_t ws_bytes) {
    (void)comm; (void)in_sizes; (void)in_dtypes; (void)out_dtype; (void)d_out_all;
    if (nranks != gar::WORLD || n_in < 2 || ws_bytes < gar::WS_NEED || out_size != gar::DIM * gar::DIM || (unsigned)rank >= (unsigned)gar::WORLD) {
        fprintf(stderr, "unexpected problem shape\n");
        return;
    }
    static bool attr = false;
    if (!attr) {
        hipError_t e = hipFuncSetAttribute((const void*)gar::gemm_allgather_kernel, hipFuncAttributeMaxDynamicSharedMemorySize, gar::LDS_BYTES);
        if (e != hipSuccess) { fprintf(stderr, "hipFuncSetAttribute: %s\n", hipGetErrorString(e)); return; }
        attr = true;
    }
    gar::Params P;
    memset(&P, 0, sizeof P);
    P.x = d_in[0]; P.w = d_in[1]; P.out = d_out; P.rank = rank;
    for (int p = 0; p < gar::WORLD; ++p) P.ws[p] = (char*)d_ws_all[p];
    hipLaunchKernelGGL(gar::gemm_allgather_kernel, dim3(gar::GRID), dim3(512), gar::LDS_BYTES, stream, P);
    hipError_t e = hipGetLastError();
    if (e != hipSuccess) fprintf(stderr, "launch: %s\n", hipGetErrorString(e));
}
```

```cpp
#include <hip/hip_runtime.h>
#include <rccl/rccl.h>
#include <cstdio>
#include <cstdint>
#include <cstring>

namespace gar {

#define LAS __attribute__((address_space(3)))
typedef unsigned u32x4 __attribute__((ext_vector_type(4)));
typedef unsigned short u16x2 __attribute__((ext_vector_type(2)));
typedef short bf16x8 __attribute__((ext_vector_type(8)));
typedef float f32x4 __attribute__((ext_vector_type(4)));
typedef __amdgpu_buffer_rsrc_t rsrc_t;

constexpr int WORLD = 8;
constexpr int DIM = 8192;
constexpr int KSH = 1024;
constexpr int NGEMM = 128;
constexpr int GRID = 256;
#define PULL_WG_PER_LINK 12
#define PULL_DECODERS 7
constexpr int PWL = PULL_WG_PER_LINK;
constexpr int NDEC = PULL_DECODERS;
constexpr int NPULL = 7 * PWL;
constexpr int NPACK = GRID - NGEMM - NPULL;
#define NRAWEND 40
#define WARM_TLB 1
#define OUT_AUX 18
#define IMG_AUX 16
#define STG_AUX AUX_SYS
#define XIN_AUX 0
#define JIT_LEAD 0
#define PAY_AUX AUX_SYS
#define NRAW 48
constexpr int NFIRST = (NPACK + NGEMM) * 4;
#define GEMM_PACK_ROUNDS 1
constexpr int GROUNDS = GEMM_PACK_ROUNDS;
constexpr int OUT2 = 4096 + NPACK * 4;
static_assert(NFIRST + GROUNDS * NGEMM * 8 <= 4096 && NFIRST % 8 == 0 && OUT2 % 8 == 0, "roles");
constexpr int NSEQ = 3584;
constexpr unsigned long long DEADLINE = 40000000ull;

constexpr unsigned OFF_TMO = 128;
constexpr unsigned OFF_LINK = 1024;
constexpr unsigned OFF_CNT = 4096;
constexpr unsigned OFF_XFLAG = OFF_CNT + 256 * 128;
constexpr unsigned OFF_OCNT = OFF_XFLAG + 256 * 128;
constexpr unsigned OFF_GRAN = 131072;
constexpr unsigned OFF_CALL = 640u << 10;
constexpr unsigned OFF_SLOT = 1u << 20;
constexpr unsigned OFF_AIMG = 64u << 20;
constexpr unsigned OFF_BIMG = 128u << 20;
constexpr unsigned OFF_XBUF = 160u << 20;
constexpr unsigned WS_USED = 192u << 20;
constexpr unsigned WS_NEED = WS_USED;
static_assert(OFF_OCNT + 128 * 128 <= OFF_GRAN, "map");
static_assert(OFF_GRAN + WORLD * NSEQ * 8 <= OFF_CALL && OFF_CALL + 2048 * 64 <= OFF_SLOT, "map");
static_assert(OFF_SLOT + 6144u * 8192u <= OFF_AIMG, "map");
constexpr int LDS_BYTES = 131072 + 512;

struct Params {
    const void* x; const void* w; void* out;
    char* ws[WORLD];
    int rank; int pad;
};

constexpr int AUX_SC1 = 16, AUX_SYS = 17;

__device__ __forceinline__ rsrc_t mk_rsrc(const void* p, unsigned bytes) {
    return __builtin_amdgcn_make_buffer_rsrc(const_cast<void*>(p), 0, bytes, 0x00020000);
}
__device__ __forceinline__ unsigned rfl(unsigned v) { return __builtin_amdgcn_readfirstlane(v); }
__device__ __forceinline__ unsigned bfi(unsigned mask, unsigned a, unsigned b) { return (a & mask) | (b & ~mask); }
__device__ __forceinline__ unsigned perm(unsigned hi, unsigned lo, unsigned sel) { return __builtin_amdgcn_perm(hi, lo, sel); }
#define WAIT_VM0() asm volatile("s_waitcnt vmcnt(0)" ::: "memory")

#define STAMP(ws, idx) do { } while (0)

__device__ __forceinline__ unsigned call_draw(char* ws) {
    unsigned t;
    const int zero = 0, one = 1;
    unsigned long long keep;
    asm volatile("s_mov_b64 %1, exec\n\ts_mov_b64 exec, 1\n\tglobal_atomic_add %0, %2, %3, %4 sc0\n\ts_mov_b64 exec, %1\n\ts_waitcnt vmcnt(0)"
                 : "=&v"(t), "=&s"(keep) : "v"(zero), "v"(one), "s"(ws + OFF_CALL + (blockIdx.x * 8 + rfl(threadIdx.x >> 6)) * 64) : "memory");
    return rfl(t) + 1u;
}

__device__ __forceinline__ bool give_up(unsigned long long t0, unsigned* tmo, unsigned call) {
    if (rfl(__hip_atomic_load(tmo, __ATOMIC_RELAXED, __HIP_MEMORY_SCOPE_AGENT)) == call) return true;
    if (__builtin_amdgcn_s_memrealtime() - t0 > DEADLINE) { __hip_atomic_store(tmo, call, __ATOMIC_RELAXED, __HIP_MEMORY_SCOPE_AGENT); return true; }
    return false;
}
__device__ __forceinline__ void wait_word(unsigned* p, unsigned target, unsigned* tmo, unsigned call) {
    if ((int)(rfl(__hip_atomic_load(p, __ATOMIC_RELAXED, __HIP_MEMORY_SCOPE_SYSTEM)) - target) >= 0) return;
    const unsigned long long t0 = __builtin_amdgcn_s_memrealtime();
    while ((int)(rfl(__hip_atomic_load(p, __ATOMIC_RELAXED, __HIP_MEMORY_SCOPE_SYSTEM)) - target) < 0) {
        if (give_up(t0, tmo, call)) break;
        __builtin_amdgcn_s_sleep(4);
    }
}

constexpr unsigned MAX_EXC = 31;
constexpr unsigned RAWB = 3584, AUX_U = 20, AUX_MAX = 2560, U_MAX = 10240, LANE_U_MAX = 224;
constexpr unsigned ENC_HIST = AUX_MAX + 16, ENC_SORT = ENC_HIST + 256, ENC_END = ENC_SORT + 128;

__device__ __forceinline__ unsigned bfi_c(unsigned mask, unsigned a, unsigned b) { unsigned r; asm("v_bfi_b32 %0, %1, %2, %3" : "=v"(r) : "s"(mask), "v"(a), "v"(b)); return r; }
__device__ __forceinline__ unsigned opaque(unsigned v) { asm("" : "+v"(v)); return v; }
__device__ __forceinline__ unsigned bytemask(unsigned x, int sh, unsigned c0c) {
    unsigned t; asm("v_and_or_b32 %0, %1, %2, %3" : "=v"(t) : "v"(x >> sh), "s"(0x01010101u), "v"(c0c));
    return perm(0u, 0u, t);
}
template <int CTRL, int RM = 0xf> __device__ __forceinline__ unsigned dpp0(unsigned v) { return (unsigned)__builtin_amdgcn_update_dpp(0, (int)v, CTRL, RM, 0xf, RM == 0xf); }
__device__ __forceinline__ unsigned wave_scan_incl(unsigned v) {
    v += dpp0<0x111>(v); v += dpp0<0x112>(v); v += dpp0<0x114>(v); v += dpp0<0x118>(v);
    v += dpp0<0x142, 0xa>(v); v += dpp0<0x143, 0xc>(v);
    return v;
}
__device__ __forceinline__ unsigned umin(unsigned a, unsigned b) { return a < b ? a : b; }
__device__ __forceinline__ unsigned umax(unsigned a, unsigned b) { return a > b ? a : b; }
__device__ __forceinline__ unsigned sad4(unsigned x) { return __builtin_amdgcn_sad_u8(x, 0u, 0u); }
__device__ __forceinline__ unsigned lut2x16(const unsigned (&T)[8], unsigned idx, unsigned ms, unsigned c0c) {
    const unsigned sel = idx & 0x07070707u, m8 = bytemask(idx, 3, c0c);
    const unsigned p = bfi(m8, perm(T[3], T[2], sel), perm(T[1], T[0], sel)), n = bfi(m8, perm(T[7], T[6], sel), perm(T[5], T[4], sel));
    return bfi(ms, n, p);
}
#define LDS_FENCE() asm volatile("" ::: "memory")
__device__ __forceinline__ void lds_or2(LAS unsigned char* scr, unsigned p, unsigned w) {
    const unsigned s = p & 31u;
    LAS unsigned* q = (LAS unsigned*)(scr + ((p >> 5) << 2));
    __hip_atomic_fetch_or(q, w << s, __ATOMIC_RELAXED, __HIP_MEMORY_SCOPE_WORKGROUP);
    __hip_atomic_fetch_or(q + 1, (w >> 1) >> (31u - s), __ATOMIC_RELAXED, __HIP_MEMORY_SCOPE_WORKGROUP);
}

__device__ __forceinline__ unsigned encode_block(const u32x4 (&c)[8], unsigned (&P)[14], u32x4 (&a)[3], LAS unsigned char* scr, int lane) {
    asm volatile("" : "+v"(lane));
    const unsigned c0c = opaque(0x0C0C0C0Cu);
    unsigned E[16], R[16];
    u16x2 p1 = {0, 0}, p2 = {0, 0}, n1 = {0, 0}, n2 = {0, 0};
#pragma unroll
    for (int g = 0; g < 16; ++g) {
        const unsigned x = (g & 1) ? c[g >> 1].z : c[g >> 1].x, y = (g & 1) ? c[g >> 1].w : c[g >> 1].y;
        const unsigned H = perm(y, x, 0x07050301u), L = perm(y, x, 0x06040200u);
        E[g] = bfi_c(0xFEFEFEFEu, H << 1, L >> 7);
        R[g] = bfi_c(0x80808080u, L << 1, bfi_c(0x40404040u, H >> 1, L));
        const unsigned en = E[g] & bytemask(H, 7, c0c), ep = E[g] ^ en;
        p1 = __builtin_elementwise_max(p1, __builtin_bit_cast(u16x2, ep)); p2 = __builtin_elementwise_max(p2, __builtin_bit_cast(u16x2, ep << 8));
        n1 = __builtin_elementwise_max(n1, __builtin_bit_cast(u16x2, en)); n2 = __builtin_elementwise_max(n2, __builtin_bit_cast(u16x2, en << 8));
        __builtin_amdgcn_sched_barrier(0);
    }
    const u32x4 z4 = {0u, 0u, 0u, 0u};
    *(LAS u32x4*)(scr + lane * 16) = z4; *(LAS u32x4*)(scr + 1024 + lane * 16) = z4;
    if (lane < 49) *(LAS u32x4*)(scr + 2048 + lane * 16) = z4;
    unsigned baseP, baseN;
    {
        const u16x2 mp = __builtin_elementwise_max(p1, p2), mn = __builtin_elementwise_max(n1, n2);
        u16x2 m = {(unsigned short)((mp.x > mp.y ? mp.x : mp.y) >> 8), (unsigned short)((mn.x > mn.y ? mn.x : mn.y) >> 8)};
        m = __builtin_elementwise_max(m, __builtin_bit_cast(u16x2, dpp0<0x128>(__builtin_bit_cast(unsigned, m)))); m = __builtin_elementwise_max(m, __builtin_bit_cast(u16x2, dpp0<0x124>(__builtin_bit_cast(unsigned, m))));
        m = __builtin_elementwise_max(m, __builtin_bit_cast(u16x2, dpp0<0x122>(__builtin_bit_cast(unsigned, m)))); m = __builtin_elementwise_max(m, __builtin_bit_cast(u16x2, dpp0<0x121>(__builtin_bit_cast(unsigned, m))));
        const unsigned mu = __builtin_bit_cast(unsigned, m);
        const u16x2 r01 = __builtin_elementwise_max(__builtin_bit_cast(u16x2, (unsigned)__builtin_amdgcn_readlane(mu, 0)), __builtin_bit_cast(u16x2, (unsigned)__builtin_amdgcn_readlane(mu, 16)));
        const u16x2 r23 = __builtin_elementwise_max(__builtin_bit_cast(u16x2, (unsigned)__builtin_amdgcn_readlane(mu, 32)), __builtin_bit_cast(u16x2, (unsigned)__builtin_amdgcn_readlane(mu, 48)));
        const unsigned mm = rfl(__builtin_bit_cast(unsigned, __builtin_elementwise_max(r01, r23)));
        baseP = mm & 0xFFu; baseN = mm >> 16;
    }
    const unsigned BP = baseP * 0x01010101u, BN = baseN * 0x01010101u;
    unsigned any = 0, D[16];
#pragma unroll
    for (int g = 0; g < 16; ++g) {
        E[g] = bfi(bytemask(R[g], 6, c0c), BN, BP) - E[g]; any |= E[g];
        D[g] = bfi_c(0x1E1E1E1Eu, E[g] << 1, (R[g] >> 7) & 0x01010101u);
        R[g] &= 0x7F7F7F7Fu;
    }
    LDS_FENCE();
#pragma unroll
    for (int gi = 0; gi < 4; ++gi) {
        const int g = gi * 5;
        const unsigned k4 = ((D[g] & 0x0F0F0F0Fu) | ((R[g] >> 2) & 0x10101010u) | ((D[g] << 1) & 0x20202020u)) << 2;
#pragma unroll
        for (int b = 0; b < 4; ++b)
            __hip_atomic_fetch_add((LAS unsigned*)(scr + ENC_HIST + ((k4 >> (8 * b)) & 0xFFu)), 1u, __ATOMIC_RELAXED, __HIP_MEMORY_SCOPE_WORKGROUP);
    }
    LDS_FENCE();
    const unsigned cnt = *(LAS unsigned*)(scr + ENC_HIST + lane * 4);
    const unsigned key = (cnt << 4) | (15u - (lane & 15));
    unsigned rank = 0;
    rank += dpp0<0x121>(key) > key; rank += dpp0<0x122>(key) > key; rank += dpp0<0x123>(key) > key; rank += dpp0<0x124>(key) > key; rank += dpp0<0x125>(key) > key;
    rank += dpp0<0x126>(key) > key; rank += dpp0<0x127>(key) > key; rank += dpp0<0x128>(key) > key; rank += dpp0<0x129>(key) > key; rank += dpp0<0x12A>(key) > key;
    rank += dpp0<0x12B>(key) > key; rank += dpp0<0x12C>(key) > key; rank += dpp0<0x12D>(key) > key; rank += dpp0<0x12E>(key) > key; rank += dpp0<0x12F>(key) > key;
    unsigned tail = cnt; tail += dpp0<0x128>(tail); tail += dpp0<0x124>(tail); tail += dpp0<0x122>(tail); tail += dpp0<0x121>(tail);
    const int sg = (lane >> 4) & 1;
    const unsigned ct = sg ? __builtin_amdgcn_readlane(tail, 48) : __builtin_amdgcn_readlane(tail, 32);
    if (lane < 32) *(LAS unsigned*)(scr + ENC_SORT + ((lane & 16) + rank) * 4) = cnt;
    LDS_FENCE();
    unsigned best;
    {
        const unsigned cb = lane & 15, s0 = cb / 3u, s1 = cb - 3u * s0, A = 1u << s0, AB = A + (1u << s1);
        unsigned cost = ct * (16u - AB);
#pragma unroll
        for (int q = 0; q < 4; ++q) {
            const u32x4 h = *(LAS u32x4*)(scr + ENC_SORT + sg * 64 + q * 16);
#pragma unroll
            for (int j = 0; j < 4; ++j) {
                const unsigned r = q * 4 + j, len = r < A ? 1u + s0 : r < AB ? 2u + s1 : 4u + ((r - AB) >> 1);
                cost += 2u * len * h[j];
            }
        }
        best = cb < 9 ? (cost << 4) | cb : 0xFFFFFFFFu;
        best = umin(best, dpp0<0x128>(best)); best = umin(best, dpp0<0x124>(best)); best = umin(best, dpp0<0x122>(best)); best = umin(best, dpp0<0x121>(best));
    }
    const unsigned cbP = __builtin_amdgcn_readlane(best, 0) & 15u, cbN = __builtin_amdgcn_readlane(best, 16) & 15u;
    const unsigned s0P = cbP / 3u, s1P = cbP - 3u * s0P, s0N = cbN / 3u, s1N = cbN - 3u * s0N;
    const unsigned ABP = (1u << s0P) + (1u << s1P), ABN = (1u << s0N) + (1u << s1N);
    unsigned F[8];
    {
        const unsigned s0 = sg ? s0N : s0P, s1 = sg ? s1N : s1P, A = 1u << s0, AB = sg ? ABN : ABP;
        const unsigned ent = rank < A ? (rank << 4) | (s0 << 6) : rank < AB ? 1u | ((rank - A) << 4) | (s1 << 6) : (2u + ((rank - AB) >> 1)) | (((rank - AB) & 1u) << 4) | (1u << 6);
        unsigned fv = ent << (8 * (lane & 3));
        fv |= dpp0<0xB1>(fv); fv |= dpp0<0x4E>(fv);
#pragma unroll
        for (int q = 0; q < 8; ++q) F[q] = __builtin_amdgcn_readlane(fv, q * 4);
        if (lane < 32)
            __hip_atomic_fetch_or((LAS unsigned*)(scr + (sg * 2 + (rank >> 3)) * 4), (unsigned)(lane & 15) << (8 * (rank & 3) + 4 * ((rank >> 2) & 1)), __ATOMIC_RELAXED, __HIP_MEMORY_SCOPE_WORKGROUP);
        if (lane == 0) *(LAS unsigned*)(scr + 16) = baseP | (baseN << 8) | (s0P << 16) | (s1P << 18) | (s0N << 20) | (s1N << 22);
    }
    const unsigned ABP4 = ABP * 0x01010101u, ABN4 = ABN * 0x01010101u, S0P4 = s0P * 0x01010101u, S0N4 = s0N * 0x01010101u;
    unsigned KK[16], XW[4], OX[4], TU = 0, TX = 0, bad = 0, xl = 0, xw = 0;
#pragma unroll
    for (int g = 0; g < 16; ++g) {
        const unsigned ms = bytemask(R[g], 6, c0c), m16 = bytemask(D[g], 4, c0c), ent = lut2x16(F, D[g], ms, c0c);
        const unsigned q = ((D[g] | 0x80808080u) - bfi(ms, ABN4, ABP4)) & 0x7F7F7F7Fu;
        unsigned K = bfi(m16, ((q >> 1) & 0x3F3F3F3Fu) + 0x02020202u, ent & 0x0F0F0F0Fu);
        unsigned SUF = bfi(m16, q & 0x01010101u, (ent >> 4) & 0x03030303u);
        unsigned S = bfi(m16, 0x01010101u, (ent >> 6) & 0x03030303u);
        unsigned gl = sad4(K) + 4u;
        if (__ballot(gl > 31u) != 0ull) {
            any |= 0x80u;
            const unsigned km = umax(umax(K & 0xFFu, (K >> 8) & 0xFFu), umax((K >> 16) & 0xFFu, K >> 24));
            const unsigned eq = gl > 31u ? (((K ^ (km * 0x01010101u)) + 0x7F7F7F7Fu) & 0x80808080u) ^ 0x80808080u : 0u, em = bytemask(eq, 7, c0c);
            E[g] |= eq;
            K &= ~em; SUF &= ~em; S = bfi(em, bfi(ms, S0N4, S0P4), S);
            gl = sad4(K) + 4u;
        }
        KK[g] = K; TU += gl; bad = umax(bad, gl);
        xw |= (SUF & 0xFFu) << xl; xl += S & 0xFFu;
        xw |= ((SUF >> 8) & 0xFFu) << xl; xl += (S >> 8) & 0xFFu;
        xw |= ((SUF >> 16) & 0xFFu) << xl; xl += (S >> 16) & 0xFFu;
        xw |= (SUF >> 24) << (xl & 31u); xl += S >> 24;
        if ((g & 3) == 3) { XW[g >> 2] = xw; OX[g >> 2] = TX; TX += xl; xw = 0; xl = 0; }
        __builtin_amdgcn_sched_barrier(0);
    }
    bad = (bad > 31u) | (TU > LANE_U_MAX);
    const unsigned tt = TU | (TX << 16), incl = wave_scan_incl(tt), tot = __builtin_amdgcn_readlane(incl, 63), totU = tot & 0xFFFFu, totX = tot >> 16;
    const unsigned xstart = (AUX_U * 8u + totU + 31u) & ~31u, endbit = xstart + totX;
    unsigned raw = (__ballot(bad != 0) != 0ull) || totU > U_MAX || endbit > (AUX_MAX - 8u) * 8u;
    if (!raw) {
        const unsigned px = xstart + ((incl - tt) >> 16);
        unsigned p = AUX_U * 8u + ((incl - tt) & 0xFFFFu);
#pragma unroll
        for (int g = 0; g < 16; ++g) {
            const unsigned ln = KK[g] + 0x01010101u;
            unsigned pos = (ln & 0xFFu) - 1u, w = 1u << pos;
            pos += (ln >> 8) & 0xFFu; w |= 1u << pos;
            pos += (ln >> 16) & 0xFFu; w |= 1u << pos;
            pos += ln >> 24; w |= 1u << pos;
            lds_or2(scr, p, w); p += pos + 1u;
            __builtin_amdgcn_sched_barrier(0);
        }
#pragma unroll
        for (int q = 0; q < 4; ++q) lds_or2(scr, px + OX[q], XW[q]);
    }
#pragma unroll
    for (int i = 0; i < 7; ++i) { P[i] = R[i] | ((R[14] << (7 - i)) & 0x80808080u); P[7 + i] = R[7 + i] | ((R[15] << (7 - i)) & 0x80808080u); }
    unsigned n = 0;
    const unsigned xo = (endbit + 31u) >> 5;
    if (!raw && __ballot((any & 0xF0F0F0F0u) != 0) != 0ull) {
#pragma unroll
        for (int g = 0; g < 16; ++g) {
            const unsigned f = E[g] & 0xF0F0F0F0u;
            unsigned long long mask = __ballot(f != 0);
            while (mask != 0ull && n <= MAX_EXC) {
                const int L = __builtin_ctzll(mask); mask &= mask - 1;
                const unsigned fl = __builtin_amdgcn_readlane(f, L);
                const unsigned lo = __builtin_amdgcn_readlane((g & 1) ? c[g >> 1].z : c[g >> 1].x, L);
                const unsigned hi = __builtin_amdgcn_readlane((g & 1) ? c[g >> 1].w : c[g >> 1].y, L);
                const bool e = lane < 4 && ((fl >> (8 * (lane & 3))) & 0xFFu) != 0;
                const unsigned long long em = __ballot(e);
                const unsigned slot = xo + n + __builtin_popcountll(em & ((1ull << lane) - 1ull));
                const unsigned val = (((lane & 2) ? hi : lo) >> (16 * (lane & 1))) & 0xFFFFu;
                if (e && slot * 4u < AUX_MAX) *(LAS unsigned*)(scr + slot * 4u) = (unsigned)(L * 64 + g * 4 + (lane & 3)) | (val << 12);
                n += __builtin_popcountll(em);
            }
        }
    }
    raw |= (n > MAX_EXC) | ((xo + n) * 4u > AUX_MAX);
    LDS_FENCE();
    a[0] = *(LAS u32x4*)(scr + lane * 16); a[1] = *(LAS u32x4*)(scr + 1024 + lane * 16); a[2] = *(LAS u32x4*)(scr + 2048 + (lane & 31) * 16);
    return ((xo + n + 3u) >> 2) | (n << 8) | (xo << 16) | (raw << 31);
}

__device__ __forceinline__ void decode_block(LAS unsigned char* src, LAS unsigned char* pos_scr, u32x4 (&c)[8], int lane) {
    LAS unsigned char* aux = src + RAWB;
    const unsigned c0c = opaque(0x0C0C0C0Cu);
    const u32x4 nt = *(LAS const u32x4*)(aux);
    const unsigned prm_v = *(LAS const unsigned*)(aux + 16);
    unsigned u[5];
#pragma unroll
    for (int i = 0; i < 5; ++i) u[i] = *(LAS const unsigned*)(aux + AUX_U + lane * 20 + i * 4);
    u32x4 P4[3];
#pragma unroll
    for (int p = 0; p < 3; ++p) P4[p] = *(LAS const u32x4*)(src + p * 1024 + lane * 16);
    const unsigned long long ph = *(LAS const unsigned long long*)(src + 3072 + lane * 8);
    const unsigned c0 = __builtin_popcount(u[0]), c1 = c0 + __builtin_popcount(u[1]), c2 = c1 + __builtin_popcount(u[2]), c3 = c2 + __builtin_popcount(u[3]), cn = c3 + __builtin_popcount(u[4]);
    const unsigned C = wave_scan_incl(cn) - cn;
    if (lane == 0) *(LAS unsigned*)(pos_scr) = AUX_U * 8u;
    const int rounds = __ballot(cn > 128u) != 0ull ? 3 : 2;
    for (int j = 0; j < rounds; ++j) {
        const unsigned t = (C >> 6) + 1u + j, kth = 64u * t - C;
        const unsigned i = (kth > c0) + (kth > c1) + (kth > c2) + (kth > c3);
        unsigned rem = kth - (i == 0 ? 0u : i == 1 ? c0 : i == 2 ? c1 : i == 3 ? c2 : c3), w = i == 0 ? u[0] : i == 1 ? u[1] : i == 2 ? u[2] : i == 3 ? u[3] : u[4], pos = 0;
#pragma unroll
        for (int st = 16; st >= 1; st >>= 1) {
            const unsigned lo = __builtin_popcount(w & ((1u << st) - 1u));
            const bool up = rem > lo;
            rem -= up ? lo : 0u; w = up ? w >> st : w; pos += up ? st : 0;
        }
        if (kth <= cn && t < 64u) *(LAS unsigned*)(pos_scr + t * 4) = AUX_U * 8u + lane * 160u + i * 32u + pos + 1u;
    }
    LDS_FENCE();
    const unsigned p0 = *(LAS const unsigned*)(pos_scr + lane * 4);
    unsigned S[8];
    {
        LAS const unsigned* q = (LAS const unsigned*)(aux + ((p0 >> 5) << 2));
#pragma unroll
        for (int i = 0; i < 8; ++i) S[i] = q[i];
#pragma unroll
        for (int i = 0; i < 7; ++i) S[i] = __builtin_amdgcn_alignbit(S[i + 1], S[i], p0 & 31u);
    }
    unsigned R[16];
    {
        const unsigned P[14] = {P4[0].x, P4[0].y, P4[0].z, P4[0].w, P4[1].x, P4[1].y, P4[1].z, P4[1].w, P4[2].x, P4[2].y, P4[2].z, P4[2].w, (unsigned)ph, (unsigned)(ph >> 32)};
        unsigned x = 0, y = 0;
#pragma unroll
        for (int i = 0; i < 7; ++i) { x |= (P[i] >> (7 - i)) & (0x01010101u << i); y |= (P[7 + i] >> (7 - i)) & (0x01010101u << i); }
#pragma unroll
        for (int i = 0; i < 14; ++i) R[i] = P[i] & 0x7F7F7F7Fu;
        R[14] = x; R[15] = y;
    }
    const unsigned n0 = rfl(nt.x), n1 = rfl(nt.y), n2 = rfl(nt.z), n3 = rfl(nt.w), prm = rfl(prm_v);
    const unsigned I[8] = {n0 & 0x0F0F0F0Fu, (n0 >> 4) & 0x0F0F0F0Fu, n1 & 0x0F0F0F0Fu, (n1 >> 4) & 0x0F0F0F0Fu, n2 & 0x0F0F0F0Fu, (n2 >> 4) & 0x0F0F0F0Fu, n3 & 0x0F0F0F0Fu, (n3 >> 4) & 0x0F0F0F0Fu};
    const unsigned s0P = (prm >> 16) & 3u, s1P = (prm >> 18) & 3u, s0N = (prm >> 20) & 3u, s1N = (prm >> 22) & 3u;
    const unsigned STP = (s0P << 8) | (s1P << 16) | (1u << 24), STN = (s0N << 8) | (s1N << 16) | (1u << 24);
    const unsigned OTP = (4u << 8) | (((1u << s0P) + 2u) << 16) | (((1u << s0P) + (1u << s1P)) << 24);
    const unsigned OTN = (4u << 8) | (((1u << s0N) + 2u) << 16) | (((1u << s0N) + (1u << s1N)) << 24);
    const unsigned BP = (prm & 0xFFu) * 0x01010101u, BN = ((prm >> 8) & 0xFFu) * 0x01010101u;
    unsigned RB[16], SS[16], TU = 0, TX = 0;
#pragma unroll
    for (int g = 0; g < 16; ++g) {
        const unsigned w0 = S[0], q0 = __builtin_ctz(w0), w1 = w0 & (w0 - 1u), q1 = __builtin_ctz(w1), w2 = w1 & (w1 - 1u), q2 = __builtin_ctz(w2), w3 = w2 & (w2 - 1u), q3 = __builtin_ctz(w3);
        const unsigned Q = q0 | (q1 << 8) | (q2 << 16) | (q3 << 24);
        const unsigned LN = Q - opaque((Q << 8) - 1u), sh = q3 + 1u;
        TU += sh;
        if (g < 15) {
#pragma unroll
            for (int i = 0; i < 7; ++i) if (32 * i < (int)LANE_U_MAX - 4 * (g + 1)) S[i] = (i < 6 && 32 * (i + 1) < (int)LANE_U_MAX - 4 * g) ? __builtin_amdgcn_alignbit(S[i + 1], S[i], sh) : S[i] >> sh;
        }
        const unsigned sel = bfi_c(0x04040404u, R[g] >> 4, bfi(bytemask(LN + 0x7D7D7D7Du, 7, c0c), 0x03030303u, LN));
        SS[g] = perm(STN, STP, sel);
        RB[g] = perm(OTN, OTP, sel) + (LN << 1);
        TX = __builtin_amdgcn_sad_u8(SS[g], 0u, TX);
    }
    const unsigned xstart = (__builtin_amdgcn_readlane(p0 + TU, 63) + 31u) & ~31u, px = xstart + wave_scan_incl(TX) - TX;
    unsigned X[5];
    {
        LAS const unsigned* q = (LAS const unsigned*)(aux + ((px >> 5) << 2));
#pragma unroll
        for (int i = 0; i < 5; ++i) X[i] = q[i];
#pragma unroll
        for (int i = 0; i < 4; ++i) X[i] = __builtin_amdgcn_alignbit(X[i + 1], X[i], px & 31u);
    }
    unsigned xoff = 0;
#pragma unroll
    for (int g = 0; g < 16; ++g) {
        const unsigned s = SS[g], M = perm(0u, 0x00030100u, s);
        const unsigned y = (g & 1) ? X[0] >> xoff : X[0], Y = perm(y, y, 0x01000100u);
        const unsigned t1 = opaque(s << 8), t2 = opaque((t1 << 8) + t1), Cs = (t2 << 16) + t2;
        const u16x2 z02 = __builtin_bit_cast(u16x2, Y) >> __builtin_bit_cast(u16x2, Cs & 0x00FF00FFu);
        const u16x2 z13 = __builtin_bit_cast(u16x2, Y) >> __builtin_bit_cast(u16x2, perm(0u, Cs, 0x0C030C01u));
        const unsigned suf = perm(__builtin_bit_cast(unsigned, z13), __builtin_bit_cast(unsigned, z02), 0x06020400u) & M;
        if (g & 1) {
            if (g < 15) {
                const unsigned t = __builtin_amdgcn_sad_u8(s, 0u, xoff);
#pragma unroll
                for (int i = 0; i < 4; ++i) if (32 * i < 8 * (15 - g)) X[i] = (i < 3 && 32 * (i + 1) < 8 * (17 - g)) ? __builtin_amdgcn_alignbit(X[i + 1], X[i], t) : X[i] >> t;
            }
        } else xoff = sad4(s);
        const unsigned rank = RB[g] + suf - 0x06060606u, ms = bytemask(R[g], 6, c0c);
        const unsigned D = bfi(bytemask(rank, 4, c0c), rank, lut2x16(I, rank, ms, c0c));
        const unsigned E = bfi(ms, BN, BP) - ((D >> 1) & 0x0F0F0F0Fu);
        const unsigned H = bfi_c(0x80808080u, R[g] << 1, E >> 1), L = bfi_c(0x3F3F3F3Fu, R[g], bfi_c(0x40404040u, D << 6, E << 7));
        const unsigned x = perm(H, L, 0x05010400u), yv = perm(H, L, 0x07030602u);
        if (g & 1) { c[g >> 1].z = x; c[g >> 1].w = yv; } else { c[g >> 1].x = x; c[g >> 1].y = yv; }
    }
}

typedef unsigned u32x2 __attribute__((ext_vector_type(2)));
__device__ __forceinline__ void store_granule(char* p, u32x2 g) {
    asm volatile("global_store_dwordx2 %0, %1, off sc0 sc1\n\ts_nop 1" :: "v"(p), "v"(g) : "memory");
}

constexpr unsigned GCTL = 131072;
__device__ __forceinline__ unsigned payload_bytes(unsigned h) { return (h >> 31) ? 8192u : RAWB + ((h & 255u) << 4); }
__device__ __forceinline__ void publish_block(const Params& P, rsrc_t wsr, const u32x4 (&c)[8], unsigned slot, unsigned n, unsigned dmask, unsigned call, LAS unsigned char* lds, LAS unsigned char* scr,
                                              unsigned j, unsigned gsize, int wid, int lane) {
    unsigned R[14], h = 1u << 31; u32x4 a[3];
    if (n >= NRAW && n < NSEQ - NRAWEND) h = encode_block(c, R, a, scr, lane);
    const unsigned u = j & 3u;
    volatile LAS unsigned* ctl = (volatile LAS unsigned*)(lds + GCTL);
    while (j >= 4u && (int)(ctl[36 + u] - (j - 3u)) < 0) __builtin_amdgcn_s_sleep(1);
    unsigned so = OFF_SLOT + (slot & ~1u) * 8192u;
    if (wid & 1) {
        unsigned v;
        while (((v = ctl[40 + u * 8 + wid - 1]) >> 16) != ((j + 1u) & 0xFFFFu)) __builtin_amdgcn_s_sleep(1);
        so += (rfl(v) & 0xFFFFu) << 4;
    } else if (lane == 0) ctl[40 + u * 8 + wid] = ((j + 1u) << 16) | (payload_bytes(h) >> 4);
    if (h >> 31) {
#pragma unroll
        for (int q = 0; q < 8; ++q) __builtin_amdgcn_raw_buffer_store_b128(c[q], wsr, lane * 16 + q * 1024, so, PAY_AUX);
    } else {
        const unsigned n16 = h & 255u;
#pragma unroll
        for (int q = 0; q < 3; ++q) __builtin_amdgcn_raw_buffer_store_b128((u32x4){R[4 * q], R[4 * q + 1], R[4 * q + 2], R[4 * q + 3]}, wsr, lane * 16 + q * 1024, so, PAY_AUX);
        __builtin_amdgcn_raw_buffer_store_b64((u32x2){R[12], R[13]}, wsr, lane * 8 + 3072, so, PAY_AUX);
        if ((unsigned)lane < n16) __builtin_amdgcn_raw_buffer_store_b128(a[0], wsr, lane * 16 + RAWB, so, PAY_AUX);
        if ((unsigned)lane + 64u < n16) __builtin_amdgcn_raw_buffer_store_b128(a[1], wsr, lane * 16 + RAWB + 1024, so, PAY_AUX);
        if ((unsigned)lane + 128u < n16) __builtin_amdgcn_raw_buffer_store_b128(a[2], wsr, lane * 16 + RAWB + 2048, so, PAY_AUX);
    }
    WAIT_VM0();
    unsigned arrived = 0;
    if (lane == 0) {
        ctl[u * 8 + wid] = h;
        arrived = __hip_atomic_fetch_add((LAS unsigned*)(lds + GCTL) + 32 + u, 1u, __ATOMIC_RELAXED, __HIP_MEMORY_SCOPE_WORKGROUP);
    }
    if (rfl(arrived) == gsize - 1u) {
        const unsigned d = lane >> 3, w = lane & 7, hw = ctl[u * 8 + w];
        if (w < gsize && ((dmask >> d) & 1u)) store_granule(P.ws[d] + OFF_GRAN + ((unsigned)P.rank * NSEQ + n - wid + w) * 8u, (u32x2){call, hw});
        if (lane == 0) { ctl[32 + u] = 0u; ctl[36 + u] = j + 1u; }
    }
}

__device__ __forceinline__ void signal_cnt(char* ws, unsigned off, int lane) {
    if (off && lane == 0) __hip_atomic_fetch_add((unsigned*)(ws + off), 1u, __ATOMIC_RELAXED, __HIP_MEMORY_SCOPE_SYSTEM);
}
__device__ __forceinline__ unsigned packer_role(const Params& P, LAS unsigned char* lds, int first, int next, int step, int end) {
    unsigned call = 0;
    const int lane = threadIdx.x & 63, wid = rfl(threadIdx.x >> 6);
    char* ws = P.ws[P.rank];
    const rsrc_t wsr = mk_rsrc(ws, WS_USED);
    const rsrc_t xr = mk_rsrc(P.x, DIM * KSH * 2), wr = mk_rsrc(P.w, DIM * KSH * 2);
    const int myi = P.rank >> 2, myj = P.rank & 3;
    LAS unsigned char* my = lds + wid * 8192;
    unsigned j = first < NFIRST ? 0u : 1u;
    for (int p = first; p < end; ++j) {
        u32x4 c[8];
        unsigned slot, n, dmask, selfcnt = 0;
        if (p >= 4096) {
            const unsigned o = p - 4096, w = o & 7, h = (o >> 3) & 1, tile = (o >> 4) & 63;
            if (call == 0) call = call_draw(ws);
            unsigned* tmo = (unsigned*)(ws + OFF_TMO);
            wait_word((unsigned*)(ws + OFF_OCNT + (o >> 3) * 128), call * 8u, tmo, call);
            const unsigned row0 = (P.rank >> 2) * 4096 + (tile >> 3) * 256 + h * 128 + (w >> 2) * 64, col0 = (P.rank & 3) * 2048 + (tile & 7) * 256 + (w & 3) * 64;
            const rsrc_t outr = mk_rsrc(P.out, (unsigned)DIM * DIM * 2u);
#pragma unroll
            for (int q = 0; q < 8; ++q) c[q] = __builtin_amdgcn_raw_buffer_load_b128(outr, (unsigned)(lane >> 3) * (DIM * 2u) + (unsigned)(lane & 7) * 16u, (row0 * DIM + col0) * 2u + q * (8u * DIM * 2u), AUX_SYS);
            if (rfl(__hip_atomic_load(tmo, __ATOMIC_RELAXED, __HIP_MEMORY_SCOPE_AGENT)) == call) {
#pragma unroll
                for (int q = 0; q < 8; ++q) c[q] = (u32x4){0x7FC07FC0u, 0x7FC07FC0u, 0x7FC07FC0u, 0x7FC07FC0u};
            }
            slot = 4096u + o; n = 1536u + o; dmask = 0xFFu;
        } else {
            bool is_w; int kk, sel, r, bi;
            if (p < 3072) {
                kk = p / 192; const int rem = p - kk * 192;
                bi = rem & 31; r = 0; is_w = rem >= 64; sel = is_w ? ((rem - 64) >> 5) : (rem >> 5);
            } else { const int q = p - 3072; kk = q >> 6; sel = (q >> 5) & 1; bi = q & 31; r = 1; is_w = false; }
            if (!is_w) {
                const int rr = lane >> 3;
                const unsigned vo = (unsigned)rr * 2048u + (unsigned)(((lane & 7) ^ rr) * 16);
                const unsigned so = (unsigned)(sel * 4096 + r * 2048 + bi * 64) * 2048u + (unsigned)kk * 128u;
#pragma unroll
                for (int q = 0; q < 8; ++q) c[q] = __builtin_amdgcn_raw_buffer_load_b128(xr, vo + q * 8 * 2048, so, XIN_AUX);
            } else {
                const int pn = bi >> 2, bj = (bi >> 1) & 1, q = bi & 1;
                const int gk = lane >> 3, cc = lane & 7, wcl = cc >> 2, cq = cc & 3;
                const unsigned vo = (unsigned)(gk * 8) * 16384u + (unsigned)((64 * (2 * q + wcl) + 32 * bj + 8 * cq) * 2);
                const unsigned so = (unsigned)(kk * 64) * 16384u + (unsigned)((sel * 2048 + pn * 256) * 2);
                u32x4 R[8];
#pragma unroll
                for (int jj = 0; jj < 8; ++jj) R[jj] = __builtin_amdgcn_raw_buffer_load_b128(wr, vo + jj * 16384, so, XIN_AUX);
#pragma unroll
                for (int t = 0; t < 8; ++t) {
                    const unsigned s = (t & 1) ? 0x07060302u : 0x05040100u;
                    u32x4 v;
                    v.x = perm(R[1][t >> 1], R[0][t >> 1], s); v.y = perm(R[3][t >> 1], R[2][t >> 1], s);
                    v.z = perm(R[5][t >> 1], R[4][t >> 1], s); v.w = perm(R[7][t >> 1], R[6][t >> 1], s);
                    const int s64 = 32 * wcl + 16 * (t >> 2) + 4 * cq + (t & 3);
                    *(LAS u32x4*)(my + s64 * 128 + ((gk ^ (s64 & 7)) * 16)) = v;
                }
                asm volatile("s_waitcnt lgkmcnt(0)" ::: "memory");
#pragma unroll
                for (int q2 = 0; q2 < 8; ++q2) c[q2] = *(LAS u32x4*)(my + q2 * 1024 + lane * 16);
                asm volatile("s_waitcnt lgkmcnt(0)" ::: "memory");
            }
            if (call == 0) call = call_draw(ws);
            const int g = P.rank * 16 + kk;
            if (is_w ? (sel == myj) : (sel == myi)) {
                const unsigned so = is_w ? OFF_BIMG + (unsigned)(g * 32 + bi) * 8192u : OFF_AIMG + (unsigned)((r * 128 + g) * 32 + bi) * 8192u;
#pragma unroll
                for (int q = 0; q < 8; ++q) __builtin_amdgcn_raw_buffer_store_b128(c[q], wsr, lane * 16 + q * 1024, so, AUX_SC1);
                selfcnt = OFF_CNT + (r * 128 + g) * 128;
            }
            slot = is_w ? 2048u + (unsigned)((sel * 16 + kk) * 32 + bi) : (unsigned)(((sel * 2 + r) * 16 + kk) * 32 + bi);
            n = is_w ? (unsigned)(kk * 64 + 32 + bi) : (r == 0 ? (unsigned)(kk * 64 + bi) : (unsigned)(1024 + kk * 32 + bi));
            dmask = is_w ? (0x11u << sel) : (0x0Fu << (4 * sel));
        }
        publish_block(P, wsr, c, slot, n, dmask & ~(1u << P.rank), call, lds, my, j, (p < NFIRST || (p >= 4096 && p < OUT2)) ? 4u : 8u, wid, lane);
        signal_cnt(ws, selfcnt, lane);
        if (p < NFIRST) p = next;
        else if (p < 4096) {
            p += step;
            if (p >= 4096 && end > 4096) { if (wid < 4) p = 4096 + (int)blockIdx.x * 4 + wid; else { p = OUT2 + (int)blockIdx.x * 8 + wid; ++j; } }
        } else if (p < OUT2) p = OUT2 + (int)blockIdx.x * 8 + wid;
        else p += step;
    }
    if (call == 0) call = call_draw(ws);
    return call;
}

struct Blk { unsigned slot, dso, cnt; bool is_out; };
__device__ __forceinline__ Blk locate(unsigned n, int src, int myi, int myj) {
    Blk b; b.is_out = false;
    if (n < 1024) { const int kk = n >> 6, bi = n & 31, g = src * 16 + kk; b.cnt = OFF_CNT + g * 128;
        if ((n & 63) >= 32) { b.slot = 2048u + (myj * 16 + kk) * 32 + bi; b.dso = OFF_BIMG + (unsigned)(g * 32 + bi) * 8192u; }
        else { b.slot = ((myi * 2) * 16 + kk) * 32 + bi; b.dso = OFF_AIMG + (unsigned)(g * 32 + bi) * 8192u; } }
    else if (n < 1536) { const int kk = (n - 1024) >> 5, bi = n & 31, g = src * 16 + kk; b.cnt = OFF_CNT + (128 + g) * 128;
        b.slot = ((myi * 2 + 1) * 16 + kk) * 32 + bi; b.dso = OFF_AIMG + (unsigned)((128 + g) * 32 + bi) * 8192u; }
    else { const int o = n - 1536, w = o & 7, h = (o >> 3) & 1, tile = (o >> 4) & 63, r = o >> 10; b.cnt = 0; b.is_out = true;
        b.slot = 4096u + o;
        const unsigned row0 = (src >> 2) * 4096 + r * 2048 + (tile >> 3) * 256 + h * 128 + (w >> 2) * 64, col0 = (src & 3) * 2048 + (tile & 7) * 256 + (w & 3) * 64;
        b.dso = (row0 * DIM + col0) * 2u; }
    return b;
}
constexpr unsigned NPAIR = NSEQ / 2, NSLOT = 7, CTL = NSLOT * 16384;
constexpr unsigned M_DEAD = 1u << 30, N_STOP = 0xFFFFFFFFu;
__device__ __forceinline__ unsigned lds_ld(LAS unsigned char* p) { return *(volatile LAS unsigned*)p; }
__device__ __forceinline__ void lds_st(LAS unsigned char* p, unsigned v) { *(volatile LAS unsigned*)p = v; }
__device__ __forceinline__ void ring_publish(LAS unsigned char* lds, unsigned i, unsigned n, unsigned ha, unsigned hb, int lane) {
    const unsigned sl = i % NSLOT;
    if (lane == 0) { *(volatile LAS u32x4*)(lds + CTL + sl * 16) = (u32x4){n, ha, hb, 0u}; lds_st(lds + CTL + 128 + sl * 4, i + 1u); }
}
__device__ __forceinline__ bool slot_busy(LAS unsigned char* lds, unsigned i) { return i >= NSLOT && (int)(lds_ld(lds + CTL + 160 + (i % NSLOT) * 4) - 2u * (i / NSLOT)) < 0; }
__device__ __forceinline__ void dma_op(rsrc_t r, LAS unsigned char* dst, unsigned j, unsigned so, int lane) {
    __builtin_amdgcn_raw_ptr_buffer_load_lds(r, (LAS void*)(dst + j * 1024), 16, lane * 16, so + j * 1024, 0, AUX_SYS);
}
__device__ __forceinline__ u32x4 load_pair(rsrc_t wsr, unsigned off) { return __builtin_amdgcn_raw_buffer_load_b128(wsr, 0, off, AUX_SYS); }
#define VM_WAIT_(n) asm volatile("s_waitcnt vmcnt(" #n ")" ::: "memory")
#define VM_WAIT(n) VM_WAIT_(n)
#define G_STR_(x) #x
#define G_STR(x) G_STR_(x)
#define WIN 5

__device__ __forceinline__ void loader_wave(const Params& P, LAS unsigned char* lds, unsigned call, int src, int lane) {
    char* ws = P.ws[P.rank];
    unsigned* tmo = (unsigned*)(ws + OFF_TMO);
    const rsrc_t wsr = mk_rsrc(ws, WS_USED);
    char* const remp = P.ws[src];
    const int myi = P.rank >> 2, myj = P.rank & 3;
    const unsigned gbase = OFF_GRAN + (unsigned)src * NSEQ * 8u;
    char* const gptr = ws + gbase;
    const unsigned tko = OFF_LINK + (unsigned)src * 128u, tbase = (call - 1u) * (NPAIR + 3u * PWL);
    __builtin_amdgcn_s_setprio(3);
    unsigned q, q1, q2;
    {
        unsigned t0 = 0;
        if (lane == 0) t0 = __hip_atomic_fetch_add((unsigned*)(ws + tko), 3u, __ATOMIC_RELAXED, __HIP_MEMORY_SCOPE_AGENT);
        q = rfl(t0) - tbase; q1 = q + 1; q2 = q + 2;
    }
    {
        const rsrc_t rw = mk_rsrc(remp, OFF_AIMG);
        const unsigned v = __builtin_amdgcn_raw_buffer_load_b32(rw, lane < 25 ? OFF_SLOT + (unsigned)lane * (2u << 20) + 8188u - (lane == 24 ? 8192u : 0u) : 0xFFFFFFF0u, 0, AUX_SYS);
        asm volatile("" :: "v"(v));
    }
    unsigned gca, gha, gcb, ghb, hca, hha, hcb, hhb;
    { const u32x4 a = load_pair(wsr, gbase + (q < NPAIR ? q : 0u) * 16u), b = load_pair(wsr, gbase + (q1 < NPAIR ? q1 : 0u) * 16u);
      gca = rfl(a.x); gha = rfl(a.y); gcb = rfl(a.z); ghb = rfl(a.w); hca = rfl(b.x); hha = rfl(b.y); hcb = rfl(b.z); hhb = rfl(b.w); }
    const int zero = 0, one = 1;
    unsigned i = 0;
    bool pend = false; unsigned pn = 0, pa = 0, pb = 0;
    while (q < NPAIR) {
        bool dead = false;
        if ((int)(gca - call) < 0 || (int)(gcb - call) < 0 || slot_busy(lds, i)) {
            if (pend) { VM_WAIT(0); ring_publish(lds, i - 1u, pn, pa, pb, lane); pend = false; }
            const unsigned long long t0 = __builtin_amdgcn_s_memrealtime();
            while ((int)(gca - call) < 0 || (int)(gcb - call) < 0) {
                const u32x4 a = load_pair(wsr, gbase + q * 16u);
                gca = rfl(a.x); gha = rfl(a.y); gcb = rfl(a.z); ghb = rfl(a.w);
                if ((int)(gca - call) >= 0 && (int)(gcb - call) >= 0) break;
                if (give_up(t0, tmo, call)) { dead = true; break; }
                __builtin_amdgcn_s_sleep(2);
                asm volatile("" ::: "memory");
            }
            while (slot_busy(lds, i)) __builtin_amdgcn_s_sleep(1);
        }
        const unsigned n = 2u * q;
        const unsigned so = OFF_SLOT + locate(n, src, myi, myj).slot * 8192u;
        const unsigned bytes = dead ? 0u : payload_bytes(gha) + payload_bytes(ghb), nops = (bytes + 1023u) >> 10;
        const rsrc_t rem = mk_rsrc(remp, so + bytes);
        LAS unsigned char* dst = lds + (i % NSLOT) * 16384;
        u32x4 g2; unsigned tk = 0;
        asm volatile("global_load_dwordx4 %0, %1, %2 sc0 sc1" : "=v"(g2) : "v"(zero), "s"(gptr + (q2 < NPAIR ? q2 : 0u) * 16u) : "memory");
        if (lane == 0) asm volatile("global_atomic_add %0, %1, %2, %3 sc0" : "=v"(tk) : "v"(zero), "v"(one), "s"(ws + tko) : "memory");
#define G_FILL(jj) do { if ((jj) == WIN) asm volatile("s_waitcnt vmcnt(" G_STR(WIN) ")" : "+v"(g2), "+v"(tk) :: "memory"); else VM_WAIT(WIN); \
                        if ((jj) == WIN - 2 && pend) ring_publish(lds, i - 1u, pn, pa, pb, lane); dma_op(rem, dst, jj, so, lane); } while (0)
        G_FILL(0); G_FILL(1); G_FILL(2); G_FILL(3); G_FILL(4); G_FILL(5); G_FILL(6); G_FILL(7);
#undef G_FILL
        for (unsigned j = 8; j < nops; ++j) { VM_WAIT(WIN); dma_op(rem, dst, j, so, lane); }
        pend = true; pn = n; pa = gha | (dead ? M_DEAD : 0u); pb = ghb | (dead ? M_DEAD : 0u);
        q = q1; gca = hca; gha = hha; gcb = hcb; ghb = hhb;
        q1 = q2; hca = rfl(g2.x); hha = rfl(g2.y); hcb = rfl(g2.z); hhb = rfl(g2.w);
        q2 = rfl(tk) - tbase;
        ++i;
    }
    VM_WAIT(0);
    if (pend) ring_publish(lds, i - 1u, pn, pa, pb, lane);
    for (int d = 0; d < (NDEC + 1) / 2 + 1; ++d, ++i) {
        while (slot_busy(lds, i)) __builtin_amdgcn_s_sleep(1);
        ring_publish(lds, i, N_STOP, 0u, 0u, lane);
    }
}

__device__ __forceinline__ void decoder_wave(const Params& P, LAS unsigned char* lds, unsigned call, int src, int d, int lane) {
    char* ws = P.ws[P.rank];
    unsigned* tmo = (unsigned*)(ws + OFF_TMO);
    const rsrc_t wsr = mk_rsrc(ws, WS_USED), outr = mk_rsrc(P.out, (unsigned)DIM * DIM * 2u);
    const int myi = P.rank >> 2, myj = P.rank & 3;
    for (unsigned i = d;; i += NDEC) {
        const unsigned pi = i >> 1, sl = pi % NSLOT;
        while (lds_ld(lds + CTL + 128 + sl * 4) != pi + 1u) __builtin_amdgcn_s_sleep(1);
        asm volatile("" ::: "memory");
        const u32x4 meta = *(volatile LAS u32x4*)(lds + CTL + sl * 16);
        const unsigned n0 = rfl(meta.x), ha = rfl(meta.y), h = (i & 1) ? rfl(meta.z) : ha;
        if (n0 == N_STOP) break;
        const Blk b = locate(n0 + (i & 1), src, myi, myj);
        const bool raw = (h >> 31) != 0;
        bool dead = (h & M_DEAD) != 0;
        if (b.is_out && rfl(__hip_atomic_load(tmo, __ATOMIC_RELAXED, __HIP_MEMORY_SCOPE_AGENT)) == call) dead = true;
        u32x4 c[8];
        unsigned recs = 0;
        const unsigned ne = (dead || raw) ? 0u : (h >> 8) & 63u;
        LAS unsigned char* cur = lds + sl * 16384 + ((i & 1) ? payload_bytes(ha) : 0u);
        if (dead) {
#pragma unroll
            for (int j = 0; j < 8; ++j) c[j] = (u32x4){0x7FC07FC0u, 0x7FC07FC0u, 0x7FC07FC0u, 0x7FC07FC0u};
        } else if (raw) {
#pragma unroll
            for (int j = 0; j < 8; ++j) c[j] = *(LAS u32x4*)(cur + j * 1024 + lane * 16);
        } else {
            recs = *(LAS unsigned*)(cur + RAWB + (((h >> 16) & 0x3FFu) + (lane & 31)) * 4u);
            decode_block(cur, lds + CTL + 256 + d * 256, c, lane);
        }
#pragma unroll
        for (int j = 0; j < 8; ++j) asm volatile("" : "+v"(c[j]));
        asm volatile("" : "+v"(recs));
        if (lane == 0) __hip_atomic_fetch_add((LAS unsigned*)(lds + CTL + 160 + sl * 4), 1u, __ATOMIC_RELAXED, __HIP_MEMORY_SCOPE_WORKGROUP);
        const unsigned sj = b.is_out ? 8u * DIM * 2u : 1024u, sr = b.is_out ? DIM * 2u : 128u;
        const unsigned vo = (unsigned)(lane >> 3) * sr + (unsigned)(lane & 7) * 16u;
        if (b.is_out) {
#pragma unroll
            for (int j = 0; j < 8; ++j) __builtin_amdgcn_raw_buffer_store_b128(c[j], outr, vo, b.dso + j * sj, OUT_AUX);
            for (unsigned e = 0; e < ne; ++e) {
                const unsigned rec = __builtin_amdgcn_readlane(recs, e), pos = rec & 0xFFFu, v = pos & 63u;
                if ((unsigned)lane == (pos >> 6)) __builtin_amdgcn_raw_buffer_store_b16((unsigned short)((rec >> 12) & 0xFFFFu), outr, vo + (v & 7u) * 2u, b.dso + (v >> 3) * sj, OUT_AUX);
            }
        } else {
#pragma unroll
            for (int j = 0; j < 8; ++j) __builtin_amdgcn_raw_buffer_store_b128(c[j], wsr, vo, b.dso + j * sj, IMG_AUX);
            for (unsigned e = 0; e < ne; ++e) {
                const unsigned rec = __builtin_amdgcn_readlane(recs, e), pos = rec & 0xFFFu, v = pos & 63u;
                if ((unsigned)lane == (pos >> 6)) __builtin_amdgcn_raw_buffer_store_b16((unsigned short)((rec >> 12) & 0xFFFFu), wsr, vo + (v & 7u) * 2u, b.dso + (v >> 3) * sj, IMG_AUX);
            }
            WAIT_VM0(); signal_cnt(ws, b.cnt, lane);
        }
    }
}

__device__ __forceinline__ void puller_role(const Params& P, LAS unsigned char* lds, unsigned call, int wg) {
    const int lane = threadIdx.x & 63, wid = rfl(threadIdx.x >> 6);
    const int link = wg % 7, src = rfl((P.rank + 1 + link) & 7);
    if (threadIdx.x < 16) *(volatile LAS unsigned*)(lds + CTL + 128 + threadIdx.x * 4) = 0u;
    __syncthreads();
    if (wid == 0) loader_wave(P, lds, call, src, lane);
    else if (wid <= NDEC) decoder_wave(P, lds, call, src, wid - 1, lane);
}

__device__ __forceinline__ float gelu_f(float y) {
    const float c1 = -2.0f * 0.7978845608028654f * 1.4426950408889634f, c2 = c1 * 0.044715f;
    const float e = __builtin_amdgcn_exp2f(y * (c1 + c2 * y * y));
    return y * __builtin_amdgcn_rcpf(1.0f + e);
}
__device__ __forceinline__ unsigned cvt_pk_bf16(float lo, float hi) { unsigned r; asm("v_cvt_pk_bf16_f32 %0, %1, %2" : "=v"(r) : "v"(lo), "v"(hi)); return r; }

__device__ __forceinline__ void gemm_role(const Params& P, LAS unsigned char* lds, unsigned call, int wg) {
    const int tid = threadIdx.x, wid = rfl(tid >> 6), lane = tid & 63, wr = wid >> 2, wc = wid & 3, fr = lane & 15, fq = lane >> 4;
    char* ws = P.ws[P.rank];
    unsigned* tmo = (unsigned*)(ws + OFF_TMO);
    const rsrc_t wsr = mk_rsrc(ws, WS_USED);
    const int xcd = wg & 7, q = (wg - NPACK) >> 3, h = xcd & 1, blk = xcd >> 1;
    const int tm = (blk >> 1) * 4 + (q >> 2), tn = (blk & 1) * 4 + (q & 3), tile = tm * 8 + tn;
    const unsigned voff = (unsigned)tid * 16u, ldsw = (unsigned)wid * 1024u;
    const int aoff0 = (wr * 64 + fr) * 128 + ((fq ^ (fr & 7)) << 4), aoff1 = aoff0 ^ 64;
    const int boff0 = (wc * 32 + fr) * 128 + ((fq ^ (fr & 7)) << 4), boff1 = boff0 ^ 64;
#define G_SA(b, hh) (((b) * 2 + (hh)) * 16384)
#define G_SB(b, hh) ((4 + (b) * 2 + (hh)) * 16384)
#define G_STAGE(bufoff, soff) do { _Pragma("unroll") for (int _i = 0; _i < 2; ++_i) \
        __builtin_amdgcn_raw_ptr_buffer_load_lds(wsr, (LAS void*)(lds + (bufoff) + ldsw + _i * 8192), 16, voff, (soff) + _i * 8192, 0, STG_AUX); } while (0)
#define G_LDA(dst, b, hh) do { _Pragma("unroll") for (int m = 0; m < 4; ++m) { \
        dst[m][0] = *(const LAS bf16x8*)(lds + G_SA(b, hh) + aoff0 + m * 2048); dst[m][1] = *(const LAS bf16x8*)(lds + G_SA(b, hh) + aoff1 + m * 2048); } } while (0)
#define G_LDB(dst, b, hh) do { _Pragma("unroll") for (int n = 0; n < 2; ++n) { \
        dst[n][0] = *(const LAS bf16x8*)(lds + G_SB(b, hh) + boff0 + n * 2048); dst[n][1] = *(const LAS bf16x8*)(lds + G_SB(b, hh) + boff1 + n * 2048); } } while (0)
#define G_MMA(ai, bj, At, Bt) do { __builtin_amdgcn_s_setprio(1); _Pragma("unroll") for (int m = 0; m < 4; ++m) _Pragma("unroll") for (int n = 0; n < 2; ++n) _Pragma("unroll") for (int k = 0; k < 2; ++k) \
        acc[ai][bj][m][n] = __builtin_amdgcn_mfma_f32_16x16x32_bf16(Bt[n][k], At[m][k], acc[ai][bj][m][n], 0, 0, 0); __builtin_amdgcn_s_setprio(0); } while (0)
#define G_WAIT_V(n) asm volatile("s_waitcnt vmcnt(" #n ")" ::: "memory")
#define G_WAIT_L(n) asm volatile("s_waitcnt lgkmcnt(" #n ")" ::: "memory")
#define G_BAR __builtin_amdgcn_s_barrier()
#define G_SCHED __builtin_amdgcn_sched_barrier(0)
#define G_GIDX(t) ((unsigned)((4 * h + ((t) & 3)) * 16 + ((t) >> 2)))
    const int zero = 0;
    for (int r = 0; r < 2; ++r) {
        const unsigned target = call * (r == 0 ? 64u : 32u);
        const unsigned abase = OFF_AIMG + (unsigned)r * (128u * 262144u) + (unsigned)tm * 32768u, bbase = OFF_BIMG + (unsigned)tn * 32768u;
        char* cbase = ws + OFF_CNT + r * 128 * 128;
        f32x4 acc[2][2][4][2];
#pragma unroll
        for (int a = 0; a < 2; ++a)
#pragma unroll
            for (int b = 0; b < 2; ++b)
#pragma unroll
                for (int m = 0; m < 4; ++m)
#pragma unroll
                    for (int n = 0; n < 2; ++n) acc[a][b][m][n] = (f32x4){0.f, 0.f, 0.f, 0.f};
        bf16x8 At[4][2], B0[2][2], B1[2][2];
        STAMP(ws, r * 8 + 0);
        wait_word((unsigned*)(cbase + G_GIDX(0) * 128), target, tmo, call);
        wait_word((unsigned*)(cbase + G_GIDX(1) * 128), target, tmo, call);
        wait_word((unsigned*)(cbase + G_GIDX(2) * 128), target, tmo, call);
        STAMP(ws, r * 8 + 1);
        {
            const unsigned a0 = abase + G_GIDX(0) * 262144u, b0 = bbase + G_GIDX(0) * 262144u, a1 = abase + G_GIDX(1) * 262144u, b1 = bbase + G_GIDX(1) * 262144u;
            G_STAGE(G_SB(0, 0), b0); G_STAGE(G_SA(0, 0), a0); G_STAGE(G_SB(0, 1), b0 + 16384); G_STAGE(G_SA(0, 1), a0 + 16384);
            if (wr == 1) G_BAR;
            G_WAIT_V(4); G_BAR;
            G_STAGE(G_SB(1, 0), b1); G_STAGE(G_SA(1, 0), a1); G_STAGE(G_SB(1, 1), b1 + 16384);
            G_WAIT_V(6); G_BAR;
        }
        unsigned fA = target, fB;
        for (int t = 0; t < 64; t += 2) {
            const int t2 = t + 2 < 64 ? t + 2 : 63, t3 = t + 3 < 64 ? t + 3 : 63, t4 = t + 4 < 64 ? t + 4 : 63;
            const unsigned g1 = G_GIDX(t + 1), g2 = G_GIDX(t2), g3 = G_GIDX(t3), g4 = G_GIDX(t4);
            const unsigned a1 = abase + g1 * 262144u, a2 = abase + g2 * 262144u, b2 = bbase + g2 * 262144u, a3 = abase + g3 * 262144u, b3 = bbase + g3 * 262144u;
            asm volatile("global_load_dword %0, %1, %2 sc0 sc1" : "=v"(fB) : "v"(zero), "s"(cbase + g3 * 128));
            G_LDB(B0, 0, 0); G_SCHED; G_LDA(At, 0, 0); G_STAGE(G_SA(1, 1), a1 + 16384);
            G_WAIT_L(8); G_BAR; G_WAIT_L(0); G_MMA(0, 0, At, B0); G_BAR; G_SCHED;
            asm volatile("" : "+v"(fA));
            if ((int)(rfl(fA) - target) < 0) wait_word((unsigned*)(cbase + g2 * 128), target, tmo, call);
            G_LDB(B1, 0, 1); G_STAGE(G_SB(0, 0), b2);
            G_BAR; G_WAIT_L(0); G_MMA(0, 1, At, B1); G_BAR;
            G_LDA(At, 0, 1); G_STAGE(G_SA(0, 0), a2);
            G_BAR; G_WAIT_L(0); G_MMA(1, 0, At, B0); G_BAR; G_SCHED;
            G_STAGE(G_SB(0, 1), b2 + 16384);
            G_WAIT_V(6); G_BAR; G_MMA(1, 1, At, B1); G_BAR;
            asm volatile("global_load_dword %0, %1, %2 sc0 sc1" : "=v"(fA) : "v"(zero), "s"(cbase + g4 * 128));
            G_LDB(B0, 1, 0); G_SCHED; G_LDA(At, 1, 0); G_STAGE(G_SA(0, 1), a2 + 16384);
            G_WAIT_L(8); G_BAR; G_WAIT_L(0); G_MMA(0, 0, At, B0); G_BAR; G_SCHED;
            asm volatile("" : "+v"(fB));
            if ((int)(rfl(fB) - target) < 0) wait_word((unsigned*)(cbase + g3 * 128), target, tmo, call);
            G_LDB(B1, 1, 1); G_STAGE(G_SB(1, 0), b3);
            G_BAR; G_WAIT_L(0); G_MMA(0, 1, At, B1); G_BAR;
            G_LDA(At, 1, 1); G_STAGE(G_SA(1, 0), a3);
            G_BAR; G_WAIT_L(0); G_MMA(1, 0, At, B0); G_BAR; G_SCHED;
            G_STAGE(G_SB(1, 1), b3 + 16384);
            G_WAIT_V(6); G_BAR; G_MMA(1, 1, At, B1); G_BAR;
        }
        G_WAIT_V(0);
        asm volatile("" : "+v"(fA));
        if (wr == 0) G_BAR;
        G_BAR;
        STAMP(ws, r * 8 + 2);
        int lane = threadIdx.x & 63;
        asm volatile("" : "+v"(lane));
        const int fr = lane & 15, fq = lane >> 4;
        if (h) {
#pragma unroll
            for (int b = 0; b < 2; ++b)
#pragma unroll
                for (int m = 0; m < 4; ++m)
#pragma unroll
                    for (int n = 0; n < 2; ++n) { const f32x4 tq = acc[0][b][m][n]; acc[0][b][m][n] = acc[1][b][m][n]; acc[1][b][m][n] = tq; }
        }
        const unsigned xl = (unsigned)((r * 64 + tile) * 2);
#pragma unroll
        for (int b = 0; b < 2; ++b)
#pragma unroll
            for (int m = 0; m < 4; ++m)
#pragma unroll
                for (int n = 0; n < 2; ++n)
                    __builtin_amdgcn_raw_buffer_store_b128(__builtin_bit_cast(u32x4, acc[1][b][m][n]), wsr, lane * 16 + ((b * 4 + m) * 2 + n) * 1024, OFF_XBUF + ((xl + h) * 8 + wid) * 16384u, AUX_SC1);
        WAIT_VM0();
        if (lane == 0) __hip_atomic_store((unsigned*)(ws + OFF_XFLAG + (xl + h) * 128 + wid * 16), call, __ATOMIC_RELAXED, __HIP_MEMORY_SCOPE_AGENT);
        wait_word((unsigned*)(ws + OFF_XFLAG + (xl + (h ^ 1)) * 128 + wid * 16), call, tmo, call);
        const bool dead = rfl(__hip_atomic_load(tmo, __ATOMIC_RELAXED, __HIP_MEMORY_SCOPE_AGENT)) == call;
        STAMP(ws, r * 8 + 3);
        LAS unsigned char* my = lds + wid * 8192;
#pragma unroll
        for (int b = 0; b < 2; ++b)
#pragma unroll
            for (int m = 0; m < 4; ++m) {
                const f32x4 p0 = __builtin_bit_cast(f32x4, __builtin_amdgcn_raw_buffer_load_b128(wsr, lane * 16 + ((b * 4 + m) * 2 + 0) * 1024, OFF_XBUF + ((xl + (h ^ 1)) * 8 + wid) * 16384u, AUX_SC1));
                const f32x4 p1 = __builtin_bit_cast(f32x4, __builtin_amdgcn_raw_buffer_load_b128(wsr, lane * 16 + ((b * 4 + m) * 2 + 1) * 1024, OFF_XBUF + ((xl + (h ^ 1)) * 8 + wid) * 16384u, AUX_SC1));
                const f32x4 v0 = acc[0][b][m][0] + p0, v1 = acc[0][b][m][1] + p1;
                u32x4 o;
                o.x = cvt_pk_bf16(gelu_f(v0[0]), gelu_f(v0[1])); o.y = cvt_pk_bf16(gelu_f(v0[2]), gelu_f(v0[3]));
                o.z = cvt_pk_bf16(gelu_f(v1[0]), gelu_f(v1[1])); o.w = cvt_pk_bf16(gelu_f(v1[2]), gelu_f(v1[3]));
                const int row = m * 16 + fr, ch = b * 4 + fq;
                *(LAS u32x4*)(my + row * 128 + ((ch ^ (row & 7)) * 16)) = o;
            }
        asm volatile("s_waitcnt lgkmcnt(0)" ::: "memory");
        u32x4 c[8];
#pragma unroll
        for (int j = 0; j < 8; ++j) { const int row = j * 8 + (lane >> 3); c[j] = *(LAS u32x4*)(my + row * 128 + (((lane & 7) ^ (row & 7)) * 16)); }
        if (dead) {
#pragma unroll
            for (int j = 0; j < 8; ++j) c[j] = (u32x4){0x7FC07FC0u, 0x7FC07FC0u, 0x7FC07FC0u, 0x7FC07FC0u};
        }
        {
            const rsrc_t outr = mk_rsrc(P.out, (unsigned)DIM * DIM * 2u);
            const unsigned row0 = (P.rank >> 2) * 4096 + r * 2048 + tm * 256 + h * 128 + wr * 64, col0 = (P.rank & 3) * 2048 + tn * 256 + wc * 64;
            const unsigned dso = (row0 * DIM + col0) * 2u, vo = (unsigned)(lane >> 3) * (DIM * 2u) + (unsigned)(lane & 7) * 16u;
            if (r == 0) {
#pragma unroll
                for (int j = 0; j < 8; ++j) __builtin_amdgcn_raw_buffer_store_b128(c[j], outr, vo, dso + j * (8u * DIM * 2u), AUX_SC1);
            } else {
#pragma unroll
                for (int j = 0; j < 8; ++j) __builtin_amdgcn_raw_buffer_store_b128(c[j], outr, vo, dso + j * (8u * DIM * 2u), OUT_AUX);
            }
            const unsigned o = (xl + h) * 8 + wid;
            if (r == 0) { WAIT_VM0(); signal_cnt(ws, OFF_OCNT + (o >> 3) * 128, lane); }
            else publish_block(P, wsr, c, 4096u + o, 1536u + o, 0xFFu & ~(1u << P.rank), call, lds, my, 1u + GROUNDS, 8u, wid, lane);
        }
        STAMP(ws, r * 8 + 4);
        asm volatile("s_waitcnt lgkmcnt(0)" ::: "memory");
        G_BAR;
    }
#undef G_SA
#undef G_SB
#undef G_STAGE
#undef G_LDA
#undef G_LDB
#undef G_MMA
#undef G_WAIT_V
#undef G_WAIT_L
#undef G_BAR
#undef G_SCHED
#undef G_GIDX
}

__global__ __launch_bounds__(512, 2) void gemm_allgather_kernel(Params P) {
    extern __shared__ __attribute__((aligned(16))) unsigned char shm[];
    LAS unsigned char* lds = (LAS unsigned char*)shm;
    char* ws = P.ws[P.rank];
    const int b = rfl(blockIdx.x);
    STAMP(ws, 60);
    unsigned call;
    if (b >= NPACK + NGEMM) { call = call_draw(ws); puller_role(P, lds, rfl(call), rfl(b - NPACK - NGEMM)); }
    else {
        if (threadIdx.x < 40) *(volatile LAS unsigned*)(lds + GCTL + 128 + threadIdx.x * 4) = 0u;
        __syncthreads();
        const int wid = rfl(threadIdx.x >> 6);
        const bool gemm = b >= NPACK;
        const int next = gemm ? NFIRST + (b - NPACK) * 8 + wid : NFIRST + GROUNDS * NGEMM * 8 + b * 8 + wid;
        call = packer_role(P, lds, wid < 4 ? b * 4 + wid : next, next, gemm ? NGEMM * 8 : NPACK * 8, gemm ? NFIRST + GROUNDS * NGEMM * 8 : 4096 + 1024);
        if (gemm) { __syncthreads(); gemm_role(P, lds, rfl(call), rfl(b)); }
    }
    STAMP(ws, 61);
    STAMP(ws, 58 + (call & 1));
}

}

extern "C" void kernel_launch(int rank, int nranks, ncclComm_t comm,
                              hipStream_t stream,
                              void* const* d_in, const int* in_sizes, int n_in,
                              void* d_out, int out_size,
                              const int* in_dtypes, int out_dtype,
                              void* const* d_out_all,
                              void* const* d_ws_all, size_t ws_bytes) {
    (void)comm; (void)in_sizes; (void)in_dtypes; (void)out_dtype; (void)d_out_all;
    if (nranks != gar::WORLD || n_in < 2 || ws_bytes < gar::WS_NEED || out_size != gar::DIM * gar::DIM || (unsigned)rank >= (unsigned)gar::WORLD) {
        fprintf(stderr, "unexpected problem shape\n");
        return;
    }
    static bool attr = false;
    if (!attr) {
        hipError_t e = hipFuncSetAttribute((const void*)gar::gemm_allgather_kernel, hipFuncAttributeMaxDynamicSharedMemorySize, gar::LDS_BYTES);
        if (e != hipSuccess) { fprintf(stderr, "hipFuncSetAttribute: %s\n", hipGetErrorString(e)); return; }
        attr = true;
    }
    gar::Params P;
    memset(&P, 0, sizeof P);
    P.x = d_in[0]; P.w = d_in[1]; P.out = d_out; P.rank = rank;
    for (int p = 0; p < gar::WORLD; ++p) P.ws[p] = (char*)d_ws_all[p];
    hipLaunchKernelGGL(gar::gemm_allgather_kernel, dim3(gar::GRID), dim3(512), gar::LDS_BYTES, stream, P);
    hipError_t e = hipGetLastError();
    if (e != hipSuccess) fprintf(stderr, "launch: %s\n", hipGetErrorString(e));
}
```

```cpp
#include <hip/hip_runtime.h>
#include <rccl/rccl.h>
#include <cstdio>
#include <cstdint>
#include <cstring>

namespace gar {

#define LAS __attribute__((address_space(3)))
typedef unsigned u32x4 __attribute__((ext_vector_type(4)));
typedef unsigned short u16x2 __attribute__((ext_vector_type(2)));
typedef short bf16x8 __attribute__((ext_vector_type(8)));
typedef float f32x4 __attribute__((ext_vector_type(4)));
typedef __amdgpu_buffer_rsrc_t rsrc_t;

constexpr int WORLD = 8;
constexpr int DIM = 8192;
constexpr int KSH = 1024;
constexpr int NGEMM = 128;
constexpr int GRID = 256;
#define PULL_WG_PER_LINK 12
#define PULL_DECODERS 7
constexpr int PWL = PULL_WG_PER_LINK;
constexpr int NDEC = PULL_DECODERS;
constexpr int NPULL = 7 * PWL;
constexpr int NPACK = GRID - NGEMM - NPULL;
#define NRAWEND 40
#define WARM_TLB 1
#define OUT_AUX 18
#define IMG_AUX 16
#define STG_AUX AUX_SYS
#define XIN_AUX 0
#define JIT_LEAD 18
#define PAY_AUX AUX_SYS
#define NRAW 48
constexpr int NFIRST = (NPACK + NGEMM) * 4;
#define GEMM_PACK_ROUNDS 1
constexpr int GROUNDS = GEMM_PACK_ROUNDS;
constexpr int OUT2 = 4096 + NPACK * 4;
static_assert(NFIRST + GROUNDS * NGEMM * 8 <= 4096 && NFIRST % 8 == 0 && OUT2 % 8 == 0, "roles");
constexpr int NSEQ = 3584;
constexpr unsigned long long DEADLINE = 40000000ull;

constexpr unsigned OFF_TMO = 128;
constexpr unsigned OFF_LINK = 1024;
constexpr unsigned OFF_CNT = 4096;
constexpr unsigned OFF_XFLAG = OFF_CNT + 256 * 128;
constexpr unsigned OFF_OCNT = OFF_XFLAG + 256 * 128;
constexpr unsigned OFF_GRAN = 131072;
constexpr unsigned OFF_CALL = 640u << 10;
constexpr unsigned OFF_SLOT = 1u << 20;
constexpr unsigned OFF_AIMG = 64u << 20;
constexpr unsigned OFF_BIMG = 128u << 20;
constexpr unsigned OFF_XBUF = 160u << 20;
constexpr unsigned WS_USED = 192u << 20;
constexpr unsigned WS_NEED = WS_USED;
static_assert(OFF_OCNT + 128 * 128 <= OFF_GRAN, "map");
static_assert(OFF_GRAN + WORLD * NSEQ * 8 <= OFF_CALL && OFF_CALL + 2048 * 64 <= OFF_SLOT, "map");
static_assert(OFF_SLOT + 6144u * 8192u <= OFF_AIMG, "map");
constexpr int LDS_BYTES = 131072 + 512;

struct Params {
    const void* x; const void* w; void* out;
    char* ws[WORLD];
    int rank; int pad;
};

constexpr int AUX_SC1 = 16, AUX_SYS = 17;

__device__ __forceinline__ rsrc_t mk_rsrc(const void* p, unsigned bytes) {
    return __builtin_amdgcn_make_buffer_rsrc(const_cast<void*>(p), 0, bytes, 0x00020000);
}
__device__ __forceinline__ unsigned rfl(unsigned v) { return __builtin_amdgcn_readfirstlane(v); }
__device__ __forceinline__ unsigned bfi(unsigned mask, unsigned a, unsigned b) { return (a & mask) | (b & ~mask); }
__device__ __forceinline__ unsigned perm(unsigned hi, unsigned lo, unsigned sel) { return __builtin_amdgcn_perm(hi, lo, sel); }
#define WAIT_VM0() asm volatile("s_waitcnt vmcnt(0)" ::: "memory")

#define STAMP(ws, idx) do { } while (0)

__device__ __forceinline__ unsigned call_draw(char* ws) {
    unsigned t;
    const int zero = 0, one = 1;
    unsigned long long keep;
    asm volatile("s_mov_b64 %1, exec\n\ts_mov_b64 exec, 1\n\tglobal_atomic_add %0, %2, %3, %4 sc0\n\ts_mov_b64 exec, %1\n\ts_waitcnt vmcnt(0)"
                 : "=&v"(t), "=&s"(keep) : "v"(zero), "v"(one), "s"(ws + OFF_CALL + (blockIdx.x * 8 + rfl(threadIdx.x >> 6)) * 64) : "memory");
    return rfl(t) + 1u;
}

__device__ __forceinline__ bool give_up(unsigned long long t0, unsigned* tmo, unsigned call) {
    if (rfl(__hip_atomic_load(tmo, __ATOMIC_RELAXED, __HIP_MEMORY_SCOPE_AGENT)) == call) return true;
    if (__builtin_amdgcn_s_memrealtime() - t0 > DEADLINE) { __hip_atomic_store(tmo, call, __ATOMIC_RELAXED, __HIP_MEMORY_SCOPE_AGENT); return true; }
    return false;
}
__device__ __forceinline__ void wait_word(unsigned* p, unsigned target, unsigned* tmo, unsigned call) {
    if ((int)(rfl(__hip_atomic_load(p, __ATOMIC_RELAXED, __HIP_MEMORY_SCOPE_SYSTEM)) - target) >= 0) return;
    const unsigned long long t0 = __builtin_amdgcn_s_memrealtime();
    while ((int)(rfl(__hip_atomic_load(p, __ATOMIC_RELAXED, __HIP_MEMORY_SCOPE_SYSTEM)) - target) < 0) {
        if (give_up(t0, tmo, call)) break;
        __builtin_amdgcn_s_sleep(4);
    }
}

constexpr unsigned MAX_EXC = 31;
constexpr unsigned RAWB = 3584, AUX_U = 20, AUX_MAX = 2560, U_MAX = 10240, LANE_U_MAX = 224;
constexpr unsigned ENC_HIST = AUX_MAX + 16, ENC_SORT = ENC_HIST + 256, ENC_END = ENC_SORT + 128;

__device__ __forceinline__ unsigned bfi_c(unsigned mask, unsigned a, unsigned b) { unsigned r; asm("v_bfi_b32 %0, %1, %2, %3" : "=v"(r) : "s"(mask), "v"(a), "v"(b)); return r; }
__device__ __forceinline__ unsigned opaque(unsigned v) { asm("" : "+v"(v)); return v; }
__device__ __forceinline__ unsigned bytemask(unsigned x, int sh, unsigned c0c) {
    unsigned t; asm("v_and_or_b32 %0, %1, %2, %3" : "=v"(t) : "v"(x >> sh), "s"(0x01010101u), "v"(c0c));
    return perm(0u, 0u, t);
}
template <int CTRL, int RM = 0xf> __device__ __forceinline__ unsigned dpp0(unsigned v) { return (unsigned)__builtin_amdgcn_update_dpp(0, (int)v, CTRL, RM, 0xf, RM == 0xf); }
__device__ __forceinline__ unsigned wave_scan_incl(unsigned v) {
    v += dpp0<0x111>(v); v += dpp0<0x112>(v); v += dpp0<0x114>(v); v += dpp0<0x118>(v);
    v += dpp0<0x142, 0xa>(v); v += dpp0<0x143, 0xc>(v);
    return v;
}
__device__ __forceinline__ unsigned umin(unsigned a, unsigned b) { return a < b ? a : b; }
__device__ __forceinline__ unsigned umax(unsigned a, unsigned b) { return a > b ? a : b; }
__device__ __forceinline__ unsigned sad4(unsigned x) { return __builtin_amdgcn_sad_u8(x, 0u, 0u); }
__device__ __forceinline__ unsigned lut2x16(const unsigned (&T)[8], unsigned idx, unsigned ms, unsigned c0c) {
    const unsigned sel = idx & 0x07070707u, m8 = bytemask(idx, 3, c0c);
    const unsigned p = bfi(m8, perm(T[3], T[2], sel), perm(T[1], T[0], sel)), n = bfi(m8, perm(T[7], T[6], sel), perm(T[5], T[4], sel));
    return bfi(ms, n, p);
}
#define LDS_FENCE() asm volatile("" ::: "memory")
__device__ __forceinline__ void lds_or2(LAS unsigned char* scr, unsigned p, unsigned w) {
    const unsigned s = p & 31u;
    LAS unsigned* q = (LAS unsigned*)(scr + ((p >> 5) << 2));
    __hip_atomic_fetch_or(q, w << s, __ATOMIC_RELAXED, __HIP_MEMORY_SCOPE_WORKGROUP);
    __hip_atomic_fetch_or(q + 1, (w >> 1) >> (31u - s), __ATOMIC_RELAXED, __HIP_MEMORY_SCOPE_WORKGROUP);
}

__device__ __forceinline__ unsigned encode_block(const u32x4 (&c)[8], unsigned (&P)[14], u32x4 (&a)[3], LAS unsigned char* scr, int lane) {
    asm volatile("" : "+v"(lane));
    const unsigned c0c = opaque(0x0C0C0C0Cu);
    unsigned E[16], R[16];
    u16x2 p1 = {0, 0}, p2 = {0, 0}, n1 = {0, 0}, n2 = {0, 0};
#pragma unroll
    for (int g = 0; g < 16; ++g) {
        const unsigned x = (g & 1) ? c[g >> 1].z : c[g >> 1].x, y = (g & 1) ? c[g >> 1].w : c[g >> 1].y;
        const unsigned H = perm(y, x, 0x07050301u), L = perm(y, x, 0x06040200u);
        E[g] = bfi_c(0xFEFEFEFEu, H << 1, L >> 7);
        R[g] = bfi_c(0x80808080u, L << 1, bfi_c(0x40404040u, H >> 1, L));
        const unsigned en = E[g] & bytemask(H, 7, c0c), ep = E[g] ^ en;
        p1 = __builtin_elementwise_max(p1, __builtin_bit_cast(u16x2, ep)); p2 = __builtin_elementwise_max(p2, __builtin_bit_cast(u16x2, ep << 8));
        n1 = __builtin_elementwise_max(n1, __builtin_bit_cast(u16x2, en)); n2 = __builtin_elementwise_max(n2, __builtin_bit_cast(u16x2, en << 8));
        __builtin_amdgcn_sched_barrier(0);
    }
    const u32x4 z4 = {0u, 0u, 0u, 0u};
    *(LAS u32x4*)(scr + lane * 16) = z4; *(LAS u32x4*)(scr + 1024 + lane * 16) = z4;
    if (lane < 49) *(LAS u32x4*)(scr + 2048 + lane * 16) = z4;
    unsigned baseP, baseN;
    {
        const u16x2 mp = __builtin_elementwise_max(p1, p2), mn = __builtin_elementwise_max(n1, n2);
        u16x2 m = {(unsigned short)((mp.x > mp.y ? mp.x : mp.y) >> 8), (unsigned short)((mn.x > mn.y ? mn.x : mn.y) >> 8)};
        m = __builtin_elementwise_max(m, __builtin_bit_cast(u16x2, dpp0<0x128>(__builtin_bit_cast(unsigned, m)))); m = __builtin_elementwise_max(m, __builtin_bit_cast(u16x2, dpp0<0x124>(__builtin_bit_cast(unsigned, m))));
        m = __builtin_elementwise_max(m, __builtin_bit_cast(u16x2, dpp0<0x122>(__builtin_bit_cast(unsigned, m)))); m = __builtin_elementwise_max(m, __builtin_bit_cast(u16x2, dpp0<0x121>(__builtin_bit_cast(unsigned, m))));
        const unsigned mu = __builtin_bit_cast(unsigned, m);
        const u16x2 r01 = __builtin_elementwise_max(__builtin_bit_cast(u16x2, (unsigned)__builtin_amdgcn_readlane(mu, 0)), __builtin_bit_cast(u16x2, (unsigned)__builtin_amdgcn_readlane(mu, 16)));
        const u16x2 r23 = __builtin_elementwise_max(__builtin_bit_cast(u16x2, (unsigned)__builtin_amdgcn_readlane(mu, 32)), __builtin_bit_cast(u16x2, (unsigned)__builtin_amdgcn_readlane(mu, 48)));
        const unsigned mm = rfl(__builtin_bit_cast(unsigned, __builtin_elementwise_max(r01, r23)));
        baseP = mm & 0xFFu; baseN = mm >> 16;
    }
    const unsigned BP = baseP * 0x01010101u, BN = baseN * 0x01010101u;
    unsigned any = 0, D[16];
#pragma unroll
    for (int g = 0; g < 16; ++g) {
        E[g] = bfi(bytemask(R[g], 6, c0c), BN, BP) - E[g]; any |= E[g];
        D[g] = bfi_c(0x1E1E1E1Eu, E[g] << 1, (R[g] >> 7) & 0x01010101u);
        R[g] &= 0x7F7F7F7Fu;
    }
    LDS_FENCE();
#pragma unroll
    for (int gi = 0; gi < 4; ++gi) {
        const int g = gi * 5;
        const unsigned k4 = ((D[g] & 0x0F0F0F0Fu) | ((R[g] >> 2) & 0x10101010u) | ((D[g] << 1) & 0x20202020u)) << 2;
#pragma unroll
        for (int b = 0; b < 4; ++b)
            __hip_atomic_fetch_add((LAS unsigned*)(scr + ENC_HIST + ((k4 >> (8 * b)) & 0xFFu)), 1u, __ATOMIC_RELAXED, __HIP_MEMORY_SCOPE_WORKGROUP);
    }
    LDS_FENCE();
    const unsigned cnt = *(LAS unsigned*)(scr + ENC_HIST + lane * 4);
    const unsigned key = (cnt << 4) | (15u - (lane & 15));
    unsigned rank = 0;
    rank += dpp0<0x121>(key) > key; rank += dpp0<0x122>(key) > key; rank += dpp0<0x123>(key) > key; rank += dpp0<0x124>(key) > key; rank += dpp0<0x125>(key) > key;
    rank += dpp0<0x126>(key) > key; rank += dpp0<0x127>(key) > key; rank += dpp0<0x128>(key) > key; rank += dpp0<0x129>(key) > key; rank += dpp0<0x12A>(key) > key;
    rank += dpp0<0x12B>(key) > key; rank += dpp0<0x12C>(key) > key; rank += dpp0<0x12D>(key) > key; rank += dpp0<0x12E>(key) > key; rank += dpp0<0x12F>(key) > key;
    unsigned tail = cnt; tail += dpp0<0x128>(tail); tail += dpp0<0x124>(tail); tail += dpp0<0x122>(tail); tail += dpp0<0x121>(tail);
    const int sg = (lane >> 4) & 1;
    const unsigned ct = sg ? __builtin_amdgcn_readlane(tail, 48) : __builtin_amdgcn_readlane(tail, 32);
    if (lane < 32) *(LAS unsigned*)(scr + ENC_SORT + ((lane & 16) + rank) * 4) = cnt;
    LDS_FENCE();
    unsigned best;
    {
        const unsigned cb = lane & 15, s0 = cb / 3u, s1 = cb - 3u * s0, A = 1u << s0, AB = A + (1u << s1);
        unsigned cost = ct * (16u - AB);
#pragma unroll
        for (int q = 0; q < 4; ++q) {
            const u32x4 h = *(LAS u32x4*)(scr + ENC_SORT + sg * 64 + q * 16);
#pragma unroll
            for (int j = 0; j < 4; ++j) {
                const unsigned r = q * 4 + j, len = r < A ? 1u + s0 : r < AB ? 2u + s1 : 4u + ((r - AB) >> 1);
                cost += 2u * len * h[j];
            }
        }
        best = cb < 9 ? (cost << 4) | cb : 0xFFFFFFFFu;
        best = umin(best, dpp0<0x128>(best)); best = umin(best, dpp0<0x124>(best)); best = umin(best, dpp0<0x122>(best)); best = umin(best, dpp0<0x121>(best));
    }
    const unsigned cbP = __builtin_amdgcn_readlane(best, 0) & 15u, cbN = __builtin_amdgcn_readlane(best, 16) & 15u;
    const unsigned s0P = cbP / 3u, s1P = cbP - 3u * s0P, s0N = cbN / 3u, s1N = cbN - 3u * s0N;
    const unsigned ABP = (1u << s0P) + (1u << s1P), ABN = (1u << s0N) + (1u << s1N);
    unsigned F[8];
    {
        const unsigned s0 = sg ? s0N : s0P, s1 = sg ? s1N : s1P, A = 1u << s0, AB = sg ? ABN : ABP;
        const unsigned ent = rank < A ? (rank << 4) | (s0 << 6) : rank < AB ? 1u | ((rank - A) << 4) | (s1 << 6) : (2u + ((rank - AB) >> 1)) | (((rank - AB) & 1u) << 4) | (1u << 6);
        unsigned fv = ent << (8 * (lane & 3));
        fv |= dpp0<0xB1>(fv); fv |= dpp0<0x4E>(fv);
#pragma unroll
        for (int q = 0; q < 8; ++q) F[q] = __builtin_amdgcn_readlane(fv, q * 4);
        if (lane < 32)
            __hip_atomic_fetch_or((LAS unsigned*)(scr + (sg * 2 + (rank >> 3)) * 4), (unsigned)(lane & 15) << (8 * (rank & 3) + 4 * ((rank >> 2) & 1)), __ATOMIC_RELAXED, __HIP_MEMORY_SCOPE_WORKGROUP);
        if (lane == 0) *(LAS unsigned*)(scr + 16) = baseP | (baseN << 8) | (s0P << 16) | (s1P << 18) | (s0N << 20) | (s1N << 22);
    }
    const unsigned ABP4 = ABP * 0x01010101u, ABN4 = ABN * 0x01010101u, S0P4 = s0P * 0x01010101u, S0N4 = s0N * 0x01010101u;
    unsigned KK[16], XW[4], OX[4], TU = 0, TX = 0, bad = 0, xl = 0, xw = 0;
#pragma unroll
    for (int g = 0; g < 16; ++g) {
        const unsigned ms = bytemask(R[g], 6, c0c), m16 = bytemask(D[g], 4, c0c), ent = lut2x16(F, D[g], ms, c0c);
        const unsigned q = ((D[g] | 0x80808080u) - bfi(ms, ABN4, ABP4)) & 0x7F7F7F7Fu;
        unsigned K = bfi(m16, ((q >> 1) & 0x3F3F3F3Fu) + 0x02020202u, ent & 0x0F0F0F0Fu);
        unsigned SUF = bfi(m16, q & 0x01010101u, (ent >> 4) & 0x03030303u);
        unsigned S = bfi(m16, 0x01010101u, (ent >> 6) & 0x03030303u);
        unsigned gl = sad4(K) + 4u;
        if (__ballot(gl > 31u) != 0ull) {
            any |= 0x80u;
            const unsigned km = umax(umax(K & 0xFFu, (K >> 8) & 0xFFu), umax((K >> 16) & 0xFFu, K >> 24));
            const unsigned eq = gl > 31u ? (((K ^ (km * 0x01010101u)) + 0x7F7F7F7Fu) & 0x80808080u) ^ 0x80808080u : 0u, em = bytemask(eq, 7, c0c);
            E[g] |= eq;
            K &= ~em; SUF &= ~em; S = bfi(em, bfi(ms, S0N4, S0P4), S);
            gl = sad4(K) + 4u;
        }
        KK[g] = K; TU += gl; bad = umax(bad, gl);
        xw |= (SUF & 0xFFu) << xl; xl += S & 0xFFu;
        xw |= ((SUF >> 8) & 0xFFu) << xl; xl += (S >> 8) & 0xFFu;
        xw |= ((SUF >> 16) & 0xFFu) << xl; xl += (S >> 16) & 0xFFu;
        xw |= (SUF >> 24) << (xl & 31u); xl += S >> 24;
        if ((g & 3) == 3) { XW[g >> 2] = xw; OX[g >> 2] = TX; TX += xl; xw = 0; xl = 0; }
        __builtin_amdgcn_sched_barrier(0);
    }
    bad = (bad > 31u) | (TU > LANE_U_MAX);
    const unsigned tt = TU | (TX << 16), incl = wave_scan_incl(tt), tot = __builtin_amdgcn_readlane(incl, 63), totU = tot & 0xFFFFu, totX = tot >> 16;
    const unsigned xstart = (AUX_U * 8u + totU + 31u) & ~31u, endbit = xstart + totX;
    unsigned raw = (__ballot(bad != 0) != 0ull) || totU > U_MAX || endbit > (AUX_MAX - 8u) * 8u;
    if (!raw) {
        const unsigned px = xstart + ((incl - tt) >> 16);
        unsigned p = AUX_U * 8u + ((incl - tt) & 0xFFFFu);
#pragma unroll
        for (int g = 0; g < 16; ++g) {
            const unsigned ln = KK[g] + 0x01010101u;
            unsigned pos = (ln & 0xFFu) - 1u, w = 1u << pos;
            pos += (ln >> 8) & 0xFFu; w |= 1u << pos;
            pos += (ln >> 16) & 0xFFu; w |= 1u << pos;
            pos += ln >> 24; w |= 1u << pos;
            lds_or2(scr, p, w); p += pos + 1u;
            __builtin_amdgcn_sched_barrier(0);
        }
#pragma unroll
        for (int q = 0; q < 4; ++q) lds_or2(scr, px + OX[q], XW[q]);
    }
#pragma unroll
    for (int i = 0; i < 7; ++i) { P[i] = R[i] | ((R[14] << (7 - i)) & 0x80808080u); P[7 + i] = R[7 + i] | ((R[15] << (7 - i)) & 0x80808080u); }
    unsigned n = 0;
    const unsigned xo = (endbit + 31u) >> 5;
    if (!raw && __ballot((any & 0xF0F0F0F0u) != 0) != 0ull) {
#pragma unroll
        for (int g = 0; g < 16; ++g) {
            const unsigned f = E[g] & 0xF0F0F0F0u;
            unsigned long long mask = __ballot(f != 0);
            while (mask != 0ull && n <= MAX_EXC) {
                const int L = __builtin_ctzll(mask); mask &= mask - 1;
                const unsigned fl = __builtin_amdgcn_readlane(f, L);
                const unsigned lo = __builtin_amdgcn_readlane((g & 1) ? c[g >> 1].z : c[g >> 1].x, L);
                const unsigned hi = __builtin_amdgcn_readlane((g & 1) ? c[g >> 1].w : c[g >> 1].y, L);
                const bool e = lane < 4 && ((fl >> (8 * (lane & 3))) & 0xFFu) != 0;
                const unsigned long long em = __ballot(e);
                const unsigned slot = xo + n + __builtin_popcountll(em & ((1ull << lane) - 1ull));
                const unsigned val = (((lane & 2) ? hi : lo) >> (16 * (lane & 1))) & 0xFFFFu;
                if (e && slot * 4u < AUX_MAX) *(LAS unsigned*)(scr + slot * 4u) = (unsigned)(L * 64 + g * 4 + (lane & 3)) | (val << 12);
                n += __builtin_popcountll(em);
            }
        }
    }
    raw |= (n > MAX_EXC) | ((xo + n) * 4u > AUX_MAX);
    LDS_FENCE();
    a[0] = *(LAS u32x4*)(scr + lane * 16); a[1] = *(LAS u32x4*)(scr + 1024 + lane * 16); a[2] = *(LAS u32x4*)(scr + 2048 + (lane & 31) * 16);
    return ((xo + n + 3u) >> 2) | (n << 8) | (xo << 16) | (raw << 31);
}

__device__ __forceinline__ void decode_block(LAS unsigned char* src, LAS unsigned char* pos_scr, u32x4 (&c)[8], int lane) {
    LAS unsigned char* aux = src + RAWB;
    const unsigned c0c = opaque(0x0C0C0C0Cu);
    const u32x4 nt = *(LAS const u32x4*)(aux);
    const unsigned prm_v = *(LAS const unsigned*)(aux + 16);
    unsigned u[5];
#pragma unroll
    for (int i = 0; i < 5; ++i) u[i] = *(LAS const unsigned*)(aux + AUX_U + lane * 20 + i * 4);
    u32x4 P4[3];
#pragma unroll
    for (int p = 0; p < 3; ++p) P4[p] = *(LAS const u32x4*)(src + p * 1024 + lane * 16);
    const unsigned long long ph = *(LAS const unsigned long long*)(src + 3072 + lane * 8);
    const unsigned c0 = __builtin_popcount(u[0]), c1 = c0 + __builtin_popcount(u[1]), c2 = c1 + __builtin_popcount(u[2]), c3 = c2 + __builtin_popcount(u[3]), cn = c3 + __builtin_popcount(u[4]);
    const unsigned C = wave_scan_incl(cn) - cn;
    if (lane == 0) *(LAS unsigned*)(pos_scr) = AUX_U * 8u;
    const int rounds = __ballot(cn > 128u) != 0ull ? 3 : 2;
    for (int j = 0; j < rounds; ++j) {
        const unsigned t = (C >> 6) + 1u + j, kth = 64u * t - C;
        const unsigned i = (kth > c0) + (kth > c1) + (kth > c2) + (kth > c3);
        unsigned rem = kth - (i == 0 ? 0u : i == 1 ? c0 : i == 2 ? c1 : i == 3 ? c2 : c3), w = i == 0 ? u[0] : i == 1 ? u[1] : i == 2 ? u[2] : i == 3 ? u[3] : u[4], pos = 0;
#pragma unroll
        for (int st = 16; st >= 1; st >>= 1) {
            const unsigned lo = __builtin_popcount(w & ((1u << st) - 1u));
            const bool up = rem > lo;
            rem -= up ? lo : 0u; w = up ? w >> st : w; pos += up ? st : 0;
        }
        if (kth <= cn && t < 64u) *(LAS unsigned*)(pos_scr + t * 4) = AUX_U * 8u + lane * 160u + i * 32u + pos + 1u;
    }
    LDS_FENCE();
    const unsigned p0 = *(LAS const unsigned*)(pos_scr + lane * 4);
    unsigned S[8];
    {
        LAS const unsigned* q = (LAS const unsigned*)(aux + ((p0 >> 5) << 2));
#pragma unroll
        for (int i = 0; i < 8; ++i) S[i] = q[i];
#pragma unroll
        for (int i = 0; i < 7; ++i) S[i] = __builtin_amdgcn_alignbit(S[i + 1], S[i], p0 & 31u);
    }
    unsigned R[16];
    {
        const unsigned P[14] = {P4[0].x, P4[0].y, P4[0].z, P4[0].w, P4[1].x, P4[1].y, P4[1].z, P4[1].w, P4[2].x, P4[2].y, P4[2].z, P4[2].w, (unsigned)ph, (unsigned)(ph >> 32)};
        unsigned x = 0, y = 0;
#pragma unroll
        for (int i = 0; i < 7; ++i) { x |= (P[i] >> (7 - i)) & (0x01010101u << i); y |= (P[7 + i] >> (7 - i)) & (0x01010101u << i); }
#pragma unroll
        for (int i = 0; i < 14; ++i) R[i] = P[i] & 0x7F7F7F7Fu;
        R[14] = x; R[15] = y;
    }
    const unsigned n0 = rfl(nt.x), n1 = rfl(nt.y), n2 = rfl(nt.z), n3 = rfl(nt.w), prm = rfl(prm_v);
    const unsigned I[8] = {n0 & 0x0F0F0F0Fu, (n0 >> 4) & 0x0F0F0F0Fu, n1 & 0x0F0F0F0Fu, (n1 >> 4) & 0x0F0F0F0Fu, n2 & 0x0F0F0F0Fu, (n2 >> 4) & 0x0F0F0F0Fu, n3 & 0x0F0F0F0Fu, (n3 >> 4) & 0x0F0F0F0Fu};
    const unsigned s0P = (prm >> 16) & 3u, s1P = (prm >> 18) & 3u, s0N = (prm >> 20) & 3u, s1N = (prm >> 22) & 3u;
    const unsigned STP = (s0P << 8) | (s1P << 16) | (1u << 24), STN = (s0N << 8) | (s1N << 16) | (1u << 24);
    const unsigned OTP = (4u << 8) | (((1u << s0P) + 2u) << 16) | (((1u << s0P) + (1u << s1P)) << 24);
    const unsigned OTN = (4u << 8) | (((1u << s0N) + 2u) << 16) | (((1u << s0N) + (1u << s1N)) << 24);
    const unsigned BP = (prm & 0xFFu) * 0x01010101u, BN = ((prm >> 8) & 0xFFu) * 0x01010101u;
    unsigned RB[16], SS[16], TU = 0, TX = 0;
#pragma unroll
    for (int g = 0; g < 16; ++g) {
        const unsigned w0 = S[0], q0 = __builtin_ctz(w0), w1 = w0 & (w0 - 1u), q1 = __builtin_ctz(w1), w2 = w1 & (w1 - 1u), q2 = __builtin_ctz(w2), w3 = w2 & (w2 - 1u), q3 = __builtin_ctz(w3);
        const unsigned Q = q0 | (q1 << 8) | (q2 << 16) | (q3 << 24);
        const unsigned LN = Q - opaque((Q << 8) - 1u), sh = q3 + 1u;
        TU += sh;
        if (g < 15) {
#pragma unroll
            for (int i = 0; i < 7; ++i) if (32 * i < (int)LANE_U_MAX - 4 * (g + 1)) S[i] = (i < 6 && 32 * (i + 1) < (int)LANE_U_MAX - 4 * g) ? __builtin_amdgcn_alignbit(S[i + 1], S[i], sh) : S[i] >> sh;
        }
        const unsigned sel = bfi_c(0x04040404u, R[g] >> 4, bfi(bytemask(LN + 0x7D7D7D7Du, 7, c0c), 0x03030303u, LN));
        SS[g] = perm(STN, STP, sel);
        RB[g] = perm(OTN, OTP, sel) + (LN << 1);
        TX = __builtin_amdgcn_sad_u8(SS[g], 0u, TX);
    }
    const unsigned xstart = (__builtin_amdgcn_readlane(p0 + TU, 63) + 31u) & ~31u, px = xstart + wave_scan_incl(TX) - TX;
    unsigned X[5];
    {
        LAS const unsigned* q = (LAS const unsigned*)(aux + ((px >> 5) << 2));
#pragma unroll
        for (int i = 0; i < 5; ++i) X[i] = q[i];
#pragma unroll
        for (int i = 0; i < 4; ++i) X[i] = __builtin_amdgcn_alignbit(X[i + 1], X[i], px & 31u);
    }
    unsigned xoff = 0;
#pragma unroll
    for (int g = 0; g < 16; ++g) {
        const unsigned s = SS[g], M = perm(0u, 0x00030100u, s);
        const unsigned y = (g & 1) ? X[0] >> xoff : X[0], Y = perm(y, y, 0x01000100u);
        const unsigned t1 = opaque(s << 8), t2 = opaque((t1 << 8) + t1), Cs = (t2 << 16) + t2;
        const u16x2 z02 = __builtin_bit_cast(u16x2, Y) >> __builtin_bit_cast(u16x2, Cs & 0x00FF00FFu);
        const u16x2 z13 = __builtin_bit_cast(u16x2, Y) >> __builtin_bit_cast(u16x2, perm(0u, Cs, 0x0C030C01u));
        const unsigned suf = perm(__builtin_bit_cast(unsigned, z13), __builtin_bit_cast(unsigned, z02), 0x06020400u) & M;
        if (g & 1) {
            if (g < 15) {
                const unsigned t = __builtin_amdgcn_sad_u8(s, 0u, xoff);
#pragma unroll
                for (int i = 0; i < 4; ++i) if (32 * i < 8 * (15 - g)) X[i] = (i < 3 && 32 * (i + 1) < 8 * (17 - g)) ? __builtin_amdgcn_alignbit(X[i + 1], X[i], t) : X[i] >> t;
            }
        } else xoff = sad4(s);
        const unsigned rank = RB[g] + suf - 0x06060606u, ms = bytemask(R[g], 6, c0c);
        const unsigned D = bfi(bytemask(rank, 4, c0c), rank, lut2x16(I, rank, ms, c0c));
        const unsigned E = bfi(ms, BN, BP) - ((D >> 1) & 0x0F0F0F0Fu);
        const unsigned H = bfi_c(0x80808080u, R[g] << 1, E >> 1), L = bfi_c(0x3F3F3F3Fu, R[g], bfi_c(0x40404040u, D << 6, E << 7));
        const unsigned x = perm(H, L, 0x05010400u), yv = perm(H, L, 0x07030602u);
        if (g & 1) { c[g >> 1].z = x; c[g >> 1].w = yv; } else { c[g >> 1].x = x; c[g >> 1].y = yv; }
    }
}

typedef unsigned u32x2 __attribute__((ext_vector_type(2)));
__device__ __forceinline__ void store_granule(char* p, u32x2 g) {
    asm volatile("global_store_dwordx2 %0, %1, off sc0 sc1\n\ts_nop 1" :: "v"(p), "v"(g) : "memory");
}

constexpr unsigned GCTL = 131072;
__device__ __forceinline__ unsigned payload_bytes(unsigned h) { return (h >> 31) ? 8192u : RAWB + ((h & 255u) << 4); }
__device__ __forceinline__ void publish_block(const Params& P, rsrc_t wsr, const u32x4 (&c)[8], unsigned slot, unsigned n, unsigned dmask, unsigned call, LAS unsigned char* lds, LAS unsigned char* scr,
                                              unsigned j, unsigned gsize, int wid, int lane) {
    unsigned R[14], h = 1u << 31; u32x4 a[3];
    if (n >= NRAW && n < NSEQ - NRAWEND) h = encode_block(c, R, a, scr, lane);
    const unsigned u = j & 3u;
    volatile LAS unsigned* ctl = (volatile LAS unsigned*)(lds + GCTL);
    while (j >= 4u && (int)(ctl[36 + u] - (j - 3u)) < 0) __builtin_amdgcn_s_sleep(1);
    unsigned so = OFF_SLOT + (slot & ~1u) * 8192u;
    if (wid & 1) {
        unsigned v;
        while (((v = ctl[40 + u * 8 + wid - 1]) >> 16) != ((j + 1u) & 0xFFFFu)) __builtin_amdgcn_s_sleep(1);
        so += (rfl(v) & 0xFFFFu) << 4;
    } else if (lane == 0) ctl[40 + u * 8 + wid] = ((j + 1u) << 16) | (payload_bytes(h) >> 4);
    if (h >> 31) {
#pragma unroll
        for (int q = 0; q < 8; ++q) __builtin_amdgcn_raw_buffer_store_b128(c[q], wsr, lane * 16 + q * 1024, so, PAY_AUX);
    } else {
        const unsigned n16 = h & 255u;
#pragma unroll
        for (int q = 0; q < 3; ++q) __builtin_amdgcn_raw_buffer_store_b128((u32x4){R[4 * q], R[4 * q + 1], R[4 * q + 2], R[4 * q + 3]}, wsr, lane * 16 + q * 1024, so, PAY_AUX);
        __builtin_amdgcn_raw_buffer_store_b64((u32x2){R[12], R[13]}, wsr, lane * 8 + 3072, so, PAY_AUX);
        if ((unsigned)lane < n16) __builtin_amdgcn_raw_buffer_store_b128(a[0], wsr, lane * 16 + RAWB, so, PAY_AUX);
        if ((unsigned)lane + 64u < n16) __builtin_amdgcn_raw_buffer_store_b128(a[1], wsr, lane * 16 + RAWB + 1024, so, PAY_AUX);
        if ((unsigned)lane + 128u < n16) __builtin_amdgcn_raw_buffer_store_b128(a[2], wsr, lane * 16 + RAWB + 2048, so, PAY_AUX);
    }
    WAIT_VM0();
    unsigned arrived = 0;
    if (lane == 0) {
        ctl[u * 8 + wid] = h;
        arrived = __hip_atomic_fetch_add((LAS unsigned*)(lds + GCTL) + 32 + u, 1u, __ATOMIC_RELAXED, __HIP_MEMORY_SCOPE_WORKGROUP);
    }
    if (rfl(arrived) == gsize - 1u) {
        const unsigned d = lane >> 3, w = lane & 7, hw = ctl[u * 8 + w];
        if (w < gsize && ((dmask >> d) & 1u)) store_granule(P.ws[d] + OFF_GRAN + ((unsigned)P.rank * NSEQ + n - wid + w) * 8u, (u32x2){call, hw});
        if (lane == 0) { ctl[32 + u] = 0u; ctl[36 + u] = j + 1u; }
    }
}

__device__ __forceinline__ void signal_cnt(char* ws, unsigned off, int lane) {
    if (off && lane == 0) __hip_atomic_fetch_add((unsigned*)(ws + off), 1u, __ATOMIC_RELAXED, __HIP_MEMORY_SCOPE_SYSTEM);
}
__device__ __forceinline__ unsigned packer_role(const Params& P, LAS unsigned char* lds, int first, int next, int step, int end) {
    unsigned call = 0;
    const int lane = threadIdx.x & 63, wid = rfl(threadIdx.x >> 6);
    char* ws = P.ws[P.rank];
    const rsrc_t wsr = mk_rsrc(ws, WS_USED);
    const rsrc_t xr = mk_rsrc(P.x, DIM * KSH * 2), wr = mk_rsrc(P.w, DIM * KSH * 2);
    const int myi = P.rank >> 2, myj = P.rank & 3;
    LAS unsigned char* my = lds + wid * 8192;
    unsigned j = first < NFIRST ? 0u : 1u;
    const unsigned long long tstart = __builtin_amdgcn_s_memrealtime();
    for (int p = first; p < end; ++j) {
        if (p < 4096) {
            const int due = (p < 3072 ? (p / 192) * 580 : 9280 + ((p - 3072) >> 6) * 290) - JIT_LEAD * 100;
            while ((int)(__builtin_amdgcn_s_memrealtime() - tstart) < due) __builtin_amdgcn_s_sleep(16);
        }
        u32x4 c[8];
        unsigned slot, n, dmask, selfcnt = 0;
        if (p >= 4096) {
            const unsigned o = p - 4096, w = o & 7, h = (o >> 3) & 1, tile = (o >> 4) & 63;
            if (call == 0) call = call_draw(ws);
            unsigned* tmo = (unsigned*)(ws + OFF_TMO);
            wait_word((unsigned*)(ws + OFF_OCNT + (o >> 3) * 128), call * 8u, tmo, call);
            const unsigned row0 = (P.rank >> 2) * 4096 + (tile >> 3) * 256 + h * 128 + (w >> 2) * 64, col0 = (P.rank & 3) * 2048 + (tile & 7) * 256 + (w & 3) * 64;
            const rsrc_t outr = mk_rsrc(P.out, (unsigned)DIM * DIM * 2u);
#pragma unroll
            for (int q = 0; q < 8; ++q) c[q] = __builtin_amdgcn_raw_buffer_load_b128(outr, (unsigned)(lane >> 3) * (DIM * 2u) + (unsigned)(lane & 7) * 16u, (row0 * DIM + col0) * 2u + q * (8u * DIM * 2u), AUX_SYS);
            if (rfl(__hip_atomic_load(tmo, __ATOMIC_RELAXED, __HIP_MEMORY_SCOPE_AGENT)) == call) {
#pragma unroll
                for (int q = 0; q < 8; ++q) c[q] = (u32x4){0x7FC07FC0u, 0x7FC07FC0u, 0x7FC07FC0u, 0x7FC07FC0u};
            }
            slot = 4096u + o; n = 1536u + o; dmask = 0xFFu;
        } else {
            bool is_w; int kk, sel, r, bi;
            if (p < 3072) {
                kk = p / 192; const int rem = p - kk * 192;
                bi = rem & 31; r = 0; is_w = rem >= 64; sel = is_w ? ((rem - 64) >> 5) : (rem >> 5);
            } else { const int q = p - 3072; kk = q >> 6; sel = (q >> 5) & 1; bi = q & 31; r = 1; is_w = false; }
            if (!is_w) {
                const int rr = lane >> 3;
                const unsigned vo = (unsigned)rr * 2048u + (unsigned)(((lane & 7) ^ rr) * 16);
                const unsigned so = (unsigned)(sel * 4096 + r * 2048 + bi * 64) * 2048u + (unsigned)kk * 128u;
#pragma unroll
                for (int q = 0; q < 8; ++q) c[q] = __builtin_amdgcn_raw_buffer_load_b128(xr, vo + q * 8 * 2048, so, XIN_AUX);
            } else {
                const int pn = bi >> 2, bj = (bi >> 1) & 1, q = bi & 1;
                const int gk = lane >> 3, cc = lane & 7, wcl = cc >> 2, cq = cc & 3;
                const unsigned vo = (unsigned)(gk * 8) * 16384u + (unsigned)((64 * (2 * q + wcl) + 32 * bj + 8 * cq) * 2);
                const unsigned so = (unsigned)(kk * 64) * 16384u + (unsigned)((sel * 2048 + pn * 256) * 2);
                u32x4 R[8];
#pragma unroll
                for (int jj = 0; jj < 8; ++jj) R[jj] = __builtin_amdgcn_raw_buffer_load_b128(wr, vo + jj * 16384, so, XIN_AUX);
#pragma unroll
                for (int t = 0; t < 8; ++t) {
                    const unsigned s = (t & 1) ? 0x07060302u : 0x05040100u;
                    u32x4 v;
                    v.x = perm(R[1][t >> 1], R[0][t >> 1], s); v.y = perm(R[3][t >> 1], R[2][t >> 1], s);
                    v.z = perm(R[5][t >> 1], R[4][t >> 1], s); v.w = perm(R[7][t >> 1], R[6][t >> 1], s);
                    const int s64 = 32 * wcl + 16 * (t >> 2) + 4 * cq + (t & 3);
                    *(LAS u32x4*)(my + s64 * 128 + ((gk ^ (s64 & 7)) * 16)) = v;
                }
                asm volatile("s_waitcnt lgkmcnt(0)" ::: "memory");
#pragma unroll
                for (int q2 = 0; q2 < 8; ++q2) c[q2] = *(LAS u32x4*)(my + q2 * 1024 + lane * 16);
                asm volatile("s_waitcnt lgkmcnt(0)" ::: "memory");
            }
            if (call == 0) call = call_draw(ws);
            const int g = P.rank * 16 + kk;
            if (is_w ? (sel == myj) : (sel == myi)) {
                const unsigned so = is_w ? OFF_BIMG + (unsigned)(g * 32 + bi) * 8192u : OFF_AIMG + (unsigned)((r * 128 + g) * 32 + bi) * 8192u;
#pragma unroll
                for (int q = 0; q < 8; ++q) __builtin_amdgcn_raw_buffer_store_b128(c[q], wsr, lane * 16 + q * 1024, so, AUX_SC1);
                selfcnt = OFF_CNT + (r * 128 + g) * 128;
            }
            slot = is_w ? 2048u + (unsigned)((sel * 16 + kk) * 32 + bi) : (unsigned)(((sel * 2 + r) * 16 + kk) * 32 + bi);
            n = is_w ? (unsigned)(kk * 64 + 32 + bi) : (r == 0 ? (unsigned)(kk * 64 + bi) : (unsigned)(1024 + kk * 32 + bi));
            dmask = is_w ? (0x11u << sel) : (0x0Fu << (4 * sel));
        }
        publish_block(P, wsr, c, slot, n, dmask & ~(1u << P.rank), call, lds, my, j, (p < NFIRST || (p >= 4096 && p < OUT2)) ? 4u : 8u, wid, lane);
        signal_cnt(ws, selfcnt, lane);
        if (p < NFIRST) p = next;
        else if (p < 4096) {
            p += step;
            if (p >= 4096 && end > 4096) { if (wid < 4) p = 4096 + (int)blockIdx.x * 4 + wid; else { p = OUT2 + (int)blockIdx.x * 8 + wid; ++j; } }
        } else if (p < OUT2) p = OUT2 + (int)blockIdx.x * 8 + wid;
        else p += step;
    }
    if (call == 0) call = call_draw(ws);
    return call;
}

struct Blk { unsigned slot, dso, cnt; bool is_out; };
__device__ __forceinline__ Blk locate(unsigned n, int src, int myi, int myj) {
    Blk b; b.is_out = false;
    if (n < 1024) { const int kk = n >> 6, bi = n & 31, g = src * 16 + kk; b.cnt = OFF_CNT + g * 128;
        if ((n & 63) >= 32) { b.slot = 2048u + (myj * 16 + kk) * 32 + bi; b.dso = OFF_BIMG + (unsigned)(g * 32 + bi) * 8192u; }
        else { b.slot = ((myi * 2) * 16 + kk) * 32 + bi; b.dso = OFF_AIMG + (unsigned)(g * 32 + bi) * 8192u; } }
    else if (n < 1536) { const int kk = (n - 1024) >> 5, bi = n & 31, g = src * 16 + kk; b.cnt = OFF_CNT + (128 + g) * 128;
        b.slot = ((myi * 2 + 1) * 16 + kk) * 32 + bi; b.dso = OFF_AIMG + (unsigned)((128 + g) * 32 + bi) * 8192u; }
    else { const int o = n - 1536, w = o & 7, h = (o >> 3) & 1, tile = (o >> 4) & 63, r = o >> 10; b.cnt = 0; b.is_out = true;
        b.slot = 4096u + o;
        const unsigned row0 = (src >> 2) * 4096 + r * 2048 + (tile >> 3) * 256 + h * 128 + (w >> 2) * 64, col0 = (src & 3) * 2048 + (tile & 7) * 256 + (w & 3) * 64;
        b.dso = (row0 * DIM + col0) * 2u; }
    return b;
}
constexpr unsigned NPAIR = NSEQ / 2, NSLOT = 7, CTL = NSLOT * 16384;
constexpr unsigned M_DEAD = 1u << 30, N_STOP = 0xFFFFFFFFu;
__device__ __forceinline__ unsigned lds_ld(LAS unsigned char* p) { return *(volatile LAS unsigned*)p; }
__device__ __forceinline__ void lds_st(LAS unsigned char* p, unsigned v) { *(volatile LAS unsigned*)p = v; }
__device__ __forceinline__ void ring_publish(LAS unsigned char* lds, unsigned i, unsigned n, unsigned ha, unsigned hb, int lane) {
    const unsigned sl = i % NSLOT;
    if (lane == 0) { *(volatile LAS u32x4*)(lds + CTL + sl * 16) = (u32x4){n, ha, hb, 0u}; lds_st(lds + CTL + 128 + sl * 4, i + 1u); }
}
__device__ __forceinline__ bool slot_busy(LAS unsigned char* lds, unsigned i) { return i >= NSLOT && (int)(lds_ld(lds + CTL + 160 + (i % NSLOT) * 4) - 2u * (i / NSLOT)) < 0; }
__device__ __forceinline__ void dma_op(rsrc_t r, LAS unsigned char* dst, unsigned j, unsigned so, int lane) {
    __builtin_amdgcn_raw_ptr_buffer_load_lds(r, (LAS void*)(dst + j * 1024), 16, lane * 16, so + j * 1024, 0, AUX_SYS);
}
__device__ __forceinline__ u32x4 load_pair(rsrc_t wsr, unsigned off) { return __builtin_amdgcn_raw_buffer_load_b128(wsr, 0, off, AUX_SYS); }
#define VM_WAIT_(n) asm volatile("s_waitcnt vmcnt(" #n ")" ::: "memory")
#define VM_WAIT(n) VM_WAIT_(n)
#define G_STR_(x) #x
#define G_STR(x) G_STR_(x)
#define WIN 5

__device__ __forceinline__ void loader_wave(const Params& P, LAS unsigned char* lds, unsigned call, int src, int lane) {
    char* ws = P.ws[P.rank];
    unsigned* tmo = (unsigned*)(ws + OFF_TMO);
    const rsrc_t wsr = mk_rsrc(ws, WS_USED);
    char* const remp = P.ws[src];
    const int myi = P.rank >> 2, myj = P.rank & 3;
    const unsigned gbase = OFF_GRAN + (unsigned)src * NSEQ * 8u;
    char* const gptr = ws + gbase;
    const unsigned tko = OFF_LINK + (unsigned)src * 128u, tbase = (call - 1u) * (NPAIR + 3u * PWL);
    __builtin_amdgcn_s_setprio(3);
    unsigned q, q1, q2;
    {
        unsigned t0 = 0;
        if (lane == 0) t0 = __hip_atomic_fetch_add((unsigned*)(ws + tko), 3u, __ATOMIC_RELAXED, __HIP_MEMORY_SCOPE_AGENT);
        q = rfl(t0) - tbase; q1 = q + 1; q2 = q + 2;
    }
    {
        const rsrc_t rw = mk_rsrc(remp, OFF_AIMG);
        const unsigned v = __builtin_amdgcn_raw_buffer_load_b32(rw, lane < 25 ? OFF_SLOT + (unsigned)lane * (2u << 20) + 8188u - (lane == 24 ? 8192u : 0u) : 0xFFFFFFF0u, 0, AUX_SYS);
        asm volatile("" :: "v"(v));
    }
    unsigned gca, gha, gcb, ghb, hca, hha, hcb, hhb;
    { const u32x4 a = load_pair(wsr, gbase + (q < NPAIR ? q : 0u) * 16u), b = load_pair(wsr, gbase + (q1 < NPAIR ? q1 : 0u) * 16u);
      gca = rfl(a.x); gha = rfl(a.y); gcb = rfl(a.z); ghb = rfl(a.w); hca = rfl(b.x); hha = rfl(b.y); hcb = rfl(b.z); hhb = rfl(b.w); }
    const int zero = 0, one = 1;
    unsigned i = 0;
    bool pend = false; unsigned pn = 0, pa = 0, pb = 0;
    while (q < NPAIR) {
        bool dead = false;
        if ((int)(gca - call) < 0 || (int)(gcb - call) < 0 || slot_busy(lds, i)) {
            if (pend) { VM_WAIT(0); ring_publish(lds, i - 1u, pn, pa, pb, lane); pend = false; }
            const unsigned long long t0 = __builtin_amdgcn_s_memrealtime();
            while ((int)(gca - call) < 0 || (int)(gcb - call) < 0) {
                const u32x4 a = load_pair(wsr, gbase + q * 16u);
                gca = rfl(a.x); gha = rfl(a.y); gcb = rfl(a.z); ghb = rfl(a.w);
                if ((int)(gca - call) >= 0 && (int)(gcb - call) >= 0) break;
                if (give_up(t0, tmo, call)) { dead = true; break; }
                __builtin_amdgcn_s_sleep(2);
                asm volatile("" ::: "memory");
            }
            while (slot_busy(lds, i)) __builtin_amdgcn_s_sleep(1);
        }
        const unsigned n = 2u * q;
        const unsigned so = OFF_SLOT + locate(n, src, myi, myj).slot * 8192u;
        const unsigned bytes = dead ? 0u : payload_bytes(gha) + payload_bytes(ghb), nops = (bytes + 1023u) >> 10;
        const rsrc_t rem = mk_rsrc(remp, so + bytes);
        LAS unsigned char* dst = lds + (i % NSLOT) * 16384;
        u32x4 g2; unsigned tk = 0;
        asm volatile("global_load_dwordx4 %0, %1, %2 sc0 sc1" : "=v"(g2) : "v"(zero), "s"(gptr + (q2 < NPAIR ? q2 : 0u) * 16u) : "memory");
        if (lane == 0) asm volatile("global_atomic_add %0, %1, %2, %3 sc0" : "=v"(tk) : "v"(zero), "v"(one), "s"(ws + tko) : "memory");
#define G_FILL(jj) do { if ((jj) == WIN) asm volatile("s_waitcnt vmcnt(" G_STR(WIN) ")" : "+v"(g2), "+v"(tk) :: "memory"); else VM_WAIT(WIN); \
                        if ((jj) == WIN - 2 && pend) ring_publish(lds, i - 1u, pn, pa, pb, lane); dma_op(rem, dst, jj, so, lane); } while (0)
        G_FILL(0); G_FILL(1); G_FILL(2); G_FILL(3); G_FILL(4); G_FILL(5); G_FILL(6); G_FILL(7);
#undef G_FILL
        for (unsigned j = 8; j < nops; ++j) { VM_WAIT(WIN); dma_op(rem, dst, j, so, lane); }
        pend = true; pn = n; pa = gha | (dead ? M_DEAD : 0u); pb = ghb | (dead ? M_DEAD : 0u);
        q = q1; gca = hca; gha = hha; gcb = hcb; ghb = hhb;
        q1 = q2; hca = rfl(g2.x); hha = rfl(g2.y); hcb = rfl(g2.z); hhb = rfl(g2.w);
        q2 = rfl(tk) - tbase;
        ++i;
    }
    VM_WAIT(0);
    if (pend) ring_publish(lds, i - 1u, pn, pa, pb, lane);
    for (int d = 0; d < (NDEC + 1) / 2 + 1; ++d, ++i) {
        while (slot_busy(lds, i)) __builtin_amdgcn_s_sleep(1);
        ring_publish(lds, i, N_STOP, 0u, 0u, lane);
    }
}

__device__ __forceinline__ void decoder_wave(const Params& P, LAS unsigned char* lds, unsigned call, int src, int d, int lane) {
    char* ws = P.ws[P.rank];
    unsigned* tmo = (unsigned*)(ws + OFF_TMO);
    const rsrc_t wsr = mk_rsrc(ws, WS_USED), outr = mk_rsrc(P.out, (unsigned)DIM * DIM * 2u);
    const int myi = P.rank >> 2, myj = P.rank & 3;
    for (unsigned i = d;; i += NDEC) {
        const unsigned pi = i >> 1, sl = pi % NSLOT;
        while (lds_ld(lds + CTL + 128 + sl * 4) != pi + 1u) __builtin_amdgcn_s_sleep(1);
        asm volatile("" ::: "memory");
        const u32x4 meta = *(volatile LAS u32x4*)(lds + CTL + sl * 16);
        const unsigned n0 = rfl(meta.x), ha = rfl(meta.y), h = (i & 1) ? rfl(meta.z) : ha;
        if (n0 == N_STOP) break;
        const Blk b = locate(n0 + (i & 1), src, myi, myj);
        const bool raw = (h >> 31) != 0;
        bool dead = (h & M_DEAD) != 0;
        if (b.is_out && rfl(__hip_atomic_load(tmo, __ATOMIC_RELAXED, __HIP_MEMORY_SCOPE_AGENT)) == call) dead = true;
        u32x4 c[8];
        unsigned recs = 0;
        const unsigned ne = (dead || raw) ? 0u : (h >> 8) & 63u;
        LAS unsigned char* cur = lds + sl * 16384 + ((i & 1) ? payload_bytes(ha) : 0u);
        if (dead) {
#pragma unroll
            for (int j = 0; j < 8; ++j) c[j] = (u32x4){0x7FC07FC0u, 0x7FC07FC0u, 0x7FC07FC0u, 0x7FC07FC0u};
        } else if (raw) {
#pragma unroll
            for (int j = 0; j < 8; ++j) c[j] = *(LAS u32x4*)(cur + j * 1024 + lane * 16);
        } else {
            recs = *(LAS unsigned*)(cur + RAWB + (((h >> 16) & 0x3FFu) + (lane & 31)) * 4u);
            decode_block(cur, lds + CTL + 256 + d * 256, c, lane);
        }
#pragma unroll
        for (int j = 0; j < 8; ++j) asm volatile("" : "+v"(c[j]));
        asm volatile("" : "+v"(recs));
        if (lane == 0) __hip_atomic_fetch_add((LAS unsigned*)(lds + CTL + 160 + sl * 4), 1u, __ATOMIC_RELAXED, __HIP_MEMORY_SCOPE_WORKGROUP);
        const unsigned sj = b.is_out ? 8u * DIM * 2u : 1024u, sr = b.is_out ? DIM * 2u : 128u;
        const unsigned vo = (unsigned)(lane >> 3) * sr + (unsigned)(lane & 7) * 16u;
        if (b.is_out) {
#pragma unroll
            for (int j = 0; j < 8; ++j) __builtin_amdgcn_raw_buffer_store_b128(c[j], outr, vo, b.dso + j * sj, OUT_AUX);
            for (unsigned e = 0; e < ne; ++e) {
                const unsigned rec = __builtin_amdgcn_readlane(recs, e), pos = rec & 0xFFFu, v = pos & 63u;
                if ((unsigned)lane == (pos >> 6)) __builtin_amdgcn_raw_buffer_store_b16((unsigned short)((rec >> 12) & 0xFFFFu), outr, vo + (v & 7u) * 2u, b.dso + (v >> 3) * sj, OUT_AUX);
            }
        } else {
#pragma unroll
            for (int j = 0; j < 8; ++j) __builtin_amdgcn_raw_buffer_store_b128(c[j], wsr, vo, b.dso + j * sj, IMG_AUX);
            for (unsigned e = 0; e < ne; ++e) {
                const unsigned rec = __builtin_amdgcn_readlane(recs, e), pos = rec & 0xFFFu, v = pos & 63u;
                if ((unsigned)lane == (pos >> 6)) __builtin_amdgcn_raw_buffer_store_b16((unsigned short)((rec >> 12) & 0xFFFFu), wsr, vo + (v & 7u) * 2u, b.dso + (v >> 3) * sj, IMG_AUX);
            }
            WAIT_VM0(); signal_cnt(ws, b.cnt, lane);
        }
    }
}

__device__ __forceinline__ void puller_role(const Params& P, LAS unsigned char* lds, unsigned call, int wg) {
    const int lane = threadIdx.x & 63, wid = rfl(threadIdx.x >> 6);
    const int link = wg % 7, src = rfl((P.rank + 1 + link) & 7);
    if (threadIdx.x < 16) *(volatile LAS unsigned*)(lds + CTL + 128 + threadIdx.x * 4) = 0u;
    __syncthreads();
    if (wid == 0) loader_wave(P, lds, call, src, lane);
    else if (wid <= NDEC) decoder_wave(P, lds, call, src, wid - 1, lane);
}

__device__ __forceinline__ float gelu_f(float y) {
    const float c1 = -2.0f * 0.7978845608028654f * 1.4426950408889634f, c2 = c1 * 0.044715f;
    const float e = __builtin_amdgcn_exp2f(y * (c1 + c2 * y * y));
    return y * __builtin_amdgcn_rcpf(1.0f + e);
}
__device__ __forceinline__ unsigned cvt_pk_bf16(float lo, float hi) { unsigned r; asm("v_cvt_pk_bf16_f32 %0, %1, %2" : "=v"(r) : "v"(lo), "v"(hi)); return r; }

__device__ __forceinline__ void gemm_role(const Params& P, LAS unsigned char* lds, unsigned call, int wg) {
    const int tid = threadIdx.x, wid = rfl(tid >> 6), lane = tid & 63, wr = wid >> 2, wc = wid & 3, fr = lane & 15, fq = lane >> 4;
    char* ws = P.ws[P.rank];
    unsigned* tmo = (unsigned*)(ws + OFF_TMO);
    const rsrc_t wsr = mk_rsrc(ws, WS_USED);
    const int xcd = wg & 7, q = (wg - NPACK) >> 3, h = xcd & 1, blk = xcd >> 1;
    const int tm = (blk >> 1) * 4 + (q >> 2), tn = (blk & 1) * 4 + (q & 3), tile = tm * 8 + tn;
    const unsigned voff = (unsigned)tid * 16u, ldsw = (unsigned)wid * 1024u;
    const int aoff0 = (wr * 64 + fr) * 128 + ((fq ^ (fr & 7)) << 4), aoff1 = aoff0 ^ 64;
    const int boff0 = (wc * 32 + fr) * 128 + ((fq ^ (fr & 7)) << 4), boff1 = boff0 ^ 64;
#define G_SA(b, hh) (((b) * 2 + (hh)) * 16384)
#define G_SB(b, hh) ((4 + (b) * 2 + (hh)) * 16384)
#define G_STAGE(bufoff, soff) do { _Pragma("unroll") for (int _i = 0; _i < 2; ++_i) \
        __builtin_amdgcn_raw_ptr_buffer_load_lds(wsr, (LAS void*)(lds + (bufoff) + ldsw + _i * 8192), 16, voff, (soff) + _i * 8192, 0, STG_AUX); } while (0)
#define G_LDA(dst, b, hh) do { _Pragma("unroll") for (int m = 0; m < 4; ++m) { \
        dst[m][0] = *(const LAS bf16x8*)(lds + G_SA(b, hh) + aoff0 + m * 2048); dst[m][1] = *(const LAS bf16x8*)(lds + G_SA(b, hh) + aoff1 + m * 2048); } } while (0)
#define G_LDB(dst, b, hh) do { _Pragma("unroll") for (int n = 0; n < 2; ++n) { \
        dst[n][0] = *(const LAS bf16x8*)(lds + G_SB(b, hh) + boff0 + n * 2048); dst[n][1] = *(const LAS bf16x8*)(lds + G_SB(b, hh) + boff1 + n * 2048); } } while (0)
#define G_MMA(ai, bj, At, Bt) do { __builtin_amdgcn_s_setprio(1); _Pragma("unroll") for (int m = 0; m < 4; ++m) _Pragma("unroll") for (int n = 0; n < 2; ++n) _Pragma("unroll") for (int k = 0; k < 2; ++k) \
        acc[ai][bj][m][n] = __builtin_amdgcn_mfma_f32_16x16x32_bf16(Bt[n][k], At[m][k], acc[ai][bj][m][n], 0, 0, 0); __builtin_amdgcn_s_setprio(0); } while (0)
#define G_WAIT_V(n) asm volatile("s_waitcnt vmcnt(" #n ")" ::: "memory")
#define G_WAIT_L(n) asm volatile("s_waitcnt lgkmcnt(" #n ")" ::: "memory")
#define G_BAR __builtin_amdgcn_s_barrier()
#define G_SCHED __builtin_amdgcn_sched_barrier(0)
#define G_GIDX(t) ((unsigned)((4 * h + ((t) & 3)) * 16 + ((t) >> 2)))
    const int zero = 0;
    for (int r = 0; r < 2; ++r) {
        const unsigned target = call * (r == 0 ? 64u : 32u);
        const unsigned abase = OFF_AIMG + (unsigned)r * (128u * 262144u) + (unsigned)tm * 32768u, bbase = OFF_BIMG + (unsigned)tn * 32768u;
        char* cbase = ws + OFF_CNT + r * 128 * 128;
        f32x4 acc[2][2][4][2];
#pragma unroll
        for (int a = 0; a < 2; ++a)
#pragma unroll
            for (int b = 0; b < 2; ++b)
#pragma unroll
                for (int m = 0; m < 4; ++m)
#pragma unroll
                    for (int n = 0; n < 2; ++n) acc[a][b][m][n] = (f32x4){0.f, 0.f, 0.f, 0.f};
        bf16x8 At[4][2], B0[2][2], B1[2][2];
        STAMP(ws, r * 8 + 0);
        wait_word((unsigned*)(cbase + G_GIDX(0) * 128), target, tmo, call);
        wait_word((unsigned*)(cbase + G_GIDX(1) * 128), target, tmo, call);
        wait_word((unsigned*)(cbase + G_GIDX(2) * 128), target, tmo, call);
        STAMP(ws, r * 8 + 1);
        {
            const unsigned a0 = abase + G_GIDX(0) * 262144u, b0 = bbase + G_GIDX(0) * 262144u, a1 = abase + G_GIDX(1) * 262144u, b1 = bbase + G_GIDX(1) * 262144u;
            G_STAGE(G_SB(0, 0), b0); G_STAGE(G_SA(0, 0), a0); G_STAGE(G_SB(0, 1), b0 + 16384); G_STAGE(G_SA(0, 1), a0 + 16384);
            if (wr == 1) G_BAR;
            G_WAIT_V(4); G_BAR;
            G_STAGE(G_SB(1, 0), b1); G_STAGE(G_SA(1, 0), a1); G_STAGE(G_SB(1, 1), b1 + 16384);
            G_WAIT_V(6); G_BAR;
        }
        unsigned fA = target, fB;
        for (int t = 0; t < 64; t += 2) {
            const int t2 = t + 2 < 64 ? t + 2 : 63, t3 = t + 3 < 64 ? t + 3 : 63, t4 = t + 4 < 64 ? t + 4 : 63;
            const unsigned g1 = G_GIDX(t + 1), g2 = G_GIDX(t2), g3 = G_GIDX(t3), g4 = G_GIDX(t4);
            const unsigned a1 = abase + g1 * 262144u, a2 = abase + g2 * 262144u, b2 = bbase + g2 * 262144u, a3 = abase + g3 * 262144u, b3 = bbase + g3 * 262144u;
            asm volatile("global_load_dword %0, %1, %2 sc0 sc1" : "=v"(fB) : "v"(zero), "s"(cbase + g3 * 128));
            G_LDB(B0, 0, 0); G_SCHED; G_LDA(At, 0, 0); G_STAGE(G_SA(1, 1), a1 + 16384);
            G_WAIT_L(8); G_BAR; G_WAIT_L(0); G_MMA(0, 0, At, B0); G_BAR; G_SCHED;
            asm volatile("" : "+v"(fA));
            if ((int)(rfl(fA) - target) < 0) wait_word((unsigned*)(cbase + g2 * 128), target, tmo, call);
            G_LDB(B1, 0, 1); G_STAGE(G_SB(0, 0), b2);
            G_BAR; G_WAIT_L(0); G_MMA(0, 1, At, B1); G_BAR;
            G_LDA(At, 0, 1); G_STAGE(G_SA(0, 0), a2);
            G_BAR; G_WAIT_L(0); G_MMA(1, 0, At, B0); G_BAR; G_SCHED;
            G_STAGE(G_SB(0, 1), b2 + 16384);
            G_WAIT_V(6); G_BAR; G_MMA(1, 1, At, B1); G_BAR;
            asm volatile("global_load_dword %0, %1, %2 sc0 sc1" : "=v"(fA) : "v"(zero), "s"(cbase + g4 * 128));
            G_LDB(B0, 1, 0); G_SCHED; G_LDA(At, 1, 0); G_STAGE(G_SA(0, 1), a2 + 16384);
            G_WAIT_L(8); G_BAR; G_WAIT_L(0); G_MMA(0, 0, At, B0); G_BAR; G_SCHED;
            asm volatile("" : "+v"(fB));
            if ((int)(rfl(fB) - target) < 0) wait_word((unsigned*)(cbase + g3 * 128), target, tmo, call);
            G_LDB(B1, 1, 1); G_STAGE(G_SB(1, 0), b3);
            G_BAR; G_WAIT_L(0); G_MMA(0, 1, At, B1); G_BAR;
            G_LDA(At, 1, 1); G_STAGE(G_SA(1, 0), a3);
            G_BAR; G_WAIT_L(0); G_MMA(1, 0, At, B0); G_BAR; G_SCHED;
            G_STAGE(G_SB(1, 1), b3 + 16384);
            G_WAIT_V(6); G_BAR; G_MMA(1, 1, At, B1); G_BAR;
        }
        G_WAIT_V(0);
        asm volatile("" : "+v"(fA));
        if (wr == 0) G_BAR;
        G_BAR;
        STAMP(ws, r * 8 + 2);
        int lane = threadIdx.x & 63;
        asm volatile("" : "+v"(lane));
        const int fr = lane & 15, fq = lane >> 4;
        if (h) {
#pragma unroll
            for (int b = 0; b < 2; ++b)
#pragma unroll
                for (int m = 0; m < 4; ++m)
#pragma unroll
                    for (int n = 0; n < 2; ++n) { const f32x4 tq = acc[0][b][m][n]; acc[0][b][m][n] = acc[1][b][m][n]; acc[1][b][m][n] = tq; }
        }
        const unsigned xl = (unsigned)((r * 64 + tile) * 2);
#pragma unroll
        for (int b = 0; b < 2; ++b)
#pragma unroll
            for (int m = 0; m < 4; ++m)
#pragma unroll
                for (int n = 0; n < 2; ++n)
                    __builtin_amdgcn_raw_buffer_store_b128(__builtin_bit_cast(u32x4, acc[1][b][m][n]), wsr, lane * 16 + ((b * 4 + m) * 2 + n) * 1024, OFF_XBUF + ((xl + h) * 8 + wid) * 16384u, AUX_SC1);
        WAIT_VM0();
        if (lane == 0) __hip_atomic_store((unsigned*)(ws + OFF_XFLAG + (xl + h) * 128 + wid * 16), call, __ATOMIC_RELAXED, __HIP_MEMORY_SCOPE_AGENT);
        wait_word((unsigned*)(ws + OFF_XFLAG + (xl + (h ^ 1)) * 128 + wid * 16), call, tmo, call);
        const bool dead = rfl(__hip_atomic_load(tmo, __ATOMIC_RELAXED, __HIP_MEMORY_SCOPE_AGENT)) == call;
        STAMP(ws, r * 8 + 3);
        LAS unsigned char* my = lds + wid * 8192;
#pragma unroll
        for (int b = 0; b < 2; ++b)
#pragma unroll
            for (int m = 0; m < 4; ++m) {
                const f32x4 p0 = __builtin_bit_cast(f32x4, __builtin_amdgcn_raw_buffer_load_b128(wsr, lane * 16 + ((b * 4 + m) * 2 + 0) * 1024, OFF_XBUF + ((xl + (h ^ 1)) * 8 + wid) * 16384u, AUX_SC1));
                const f32x4 p1 = __builtin_bit_cast(f32x4, __builtin_amdgcn_raw_buffer_load_b128(wsr, lane * 16 + ((b * 4 + m) * 2 + 1) * 1024, OFF_XBUF + ((xl + (h ^ 1)) * 8 + wid) * 16384u, AUX_SC1));
                const f32x4 v0 = acc[0][b][m][0] + p0, v1 = acc[0][b][m][1] + p1;
                u32x4 o;
                o.x = cvt_pk_bf16(gelu_f(v0[0]), gelu_f(v0[1])); o.y = cvt_pk_bf16(gelu_f(v0[2]), gelu_f(v0[3]));
                o.z = cvt_pk_bf16(gelu_f(v1[0]), gelu_f(v1[1])); o.w = cvt_pk_bf16(gelu_f(v1[2]), gelu_f(v1[3]));
                const int row = m * 16 + fr, ch = b * 4 + fq;
                *(LAS u32x4*)(my + row * 128 + ((ch ^ (row & 7)) * 16)) = o;
            }
        asm volatile("s_waitcnt lgkmcnt(0)" ::: "memory");
        u32x4 c[8];
#pragma unroll
        for (int j = 0; j < 8; ++j) { const int row = j * 8 + (lane >> 3); c[j] = *(LAS u32x4*)(my + row * 128 + (((lane & 7) ^ (row & 7)) * 16)); }
        if (dead) {
#pragma unroll
            for (int j = 0; j < 8; ++j) c[j] = (u32x4){0x7FC07FC0u, 0x7FC07FC0u, 0x7FC07FC0u, 0x7FC07FC0u};
        }
        {
            const rsrc_t outr = mk_rsrc(P.out, (unsigned)DIM * DIM * 2u);
            const unsigned row0 = (P.rank >> 2) * 4096 + r * 2048 + tm * 256 + h * 128 + wr * 64, col0 = (P.rank & 3) * 2048 + tn * 256 + wc * 64;
            const unsigned dso = (row0 * DIM + col0) * 2u, vo = (unsigned)(lane >> 3) * (DIM * 2u) + (unsigned)(lane & 7) * 16u;
            if (r == 0) {
#pragma unroll
                for (int j = 0; j < 8; ++j) __builtin_amdgcn_raw_buffer_store_b128(c[j], outr, vo, dso + j * (8u * DIM * 2u), AUX_SC1);
            } else {
#pragma unroll
                for (int j = 0; j < 8; ++j) __builtin_amdgcn_raw_buffer_store_b128(c[j], outr, vo, dso + j * (8u * DIM * 2u), OUT_AUX);
            }
            const unsigned o = (xl + h) * 8 + wid;
            if (r == 0) { WAIT_VM0(); signal_cnt(ws, OFF_OCNT + (o >> 3) * 128, lane); }
            else publish_block(P, wsr, c, 4096u + o, 1536u + o, 0xFFu & ~(1u << P.rank), call, lds, my, 1u + GROUNDS, 8u, wid, lane);
        }
        STAMP(ws, r * 8 + 4);
        asm volatile("s_waitcnt lgkmcnt(0)" ::: "memory");
        G_BAR;
    }
#undef G_SA
#undef G_SB
#undef G_STAGE
#undef G_LDA
#undef G_LDB
#undef G_MMA
#undef G_WAIT_V
#undef G_WAIT_L
#undef G_BAR
#undef G_SCHED
#undef G_GIDX
}

__global__ __launch_bounds__(512, 2) void gemm_allgather_kernel(Params P) {
    extern __shared__ __attribute__((aligned(16))) unsigned char shm[];
    LAS unsigned char* lds = (LAS unsigned char*)shm;
    char* ws = P.ws[P.rank];
    const int b = rfl(blockIdx.x);
    STAMP(ws, 60);
    unsigned call;
    if (b >= NPACK + NGEMM) { call = call_draw(ws); puller_role(P, lds, rfl(call), rfl(b - NPACK - NGEMM)); }
    else {
        if (threadIdx.x < 40) *(volatile LAS unsigned*)(lds + GCTL + 128 + threadIdx.x * 4) = 0u;
        __syncthreads();
        const int wid = rfl(threadIdx.x >> 6);
        const bool gemm = b >= NPACK;
        const int next = gemm ? NFIRST + (b - NPACK) * 8 + wid : NFIRST + GROUNDS * NGEMM * 8 + b * 8 + wid;
        call = packer_role(P, lds, wid < 4 ? b * 4 + wid : next, next, gemm ? NGEMM * 8 : NPACK * 8, gemm ? NFIRST + GROUNDS * NGEMM * 8 : 4096 + 1024);
        if (gemm) { __syncthreads(); gemm_role(P, lds, rfl(call), rfl(b)); }
    }
    STAMP(ws, 61);
    STAMP(ws, 58 + (call & 1));
}

}

extern "C" void kernel_launch(int rank, int nranks, ncclComm_t comm,
                              hipStream_t stream,
                              void* const* d_in, const int* in_sizes, int n_in,
                              void* d_out, int out_size,
                              const int* in_dtypes, int out_dtype,
                              void* const* d_out_all,
                              void* const* d_ws_all, size_t ws_bytes) {
    (void)comm; (void)in_sizes; (void)in_dtypes; (void)out_dtype; (void)d_out_all;
    if (nranks != gar::WORLD || n_in < 2 || ws_bytes < gar::WS_NEED || out_size != gar::DIM * gar::DIM || (unsigned)rank >= (unsigned)gar::WORLD) {
        fprintf(stderr, "unexpected problem shape\n");
        return;
    }
    static bool attr = false;
    if (!attr) {
        hipError_t e = hipFuncSetAttribute((const void*)gar::gemm_allgather_kernel, hipFuncAttributeMaxDynamicSharedMemorySize, gar::LDS_BYTES);
        if (e != hipSuccess) { fprintf(stderr, "hipFuncSetAttribute: %s\n", hipGetErrorString(e)); return; }
        attr = true;
    }
    gar::Params P;
    memset(&P, 0, sizeof P);
    P.x = d_in[0]; P.w = d_in[1]; P.out = d_out; P.rank = rank;
    for (int p = 0; p < gar::WORLD; ++p) P.ws[p] = (char*)d_ws_all[p];
    hipLaunchKernelGGL(gar::gemm_allgather_kernel, dim3(gar::GRID), dim3(512), gar::LDS_BYTES, stream, P);
    hipError_t e = hipGetLastError();
    if (e != hipSuccess) fprintf(stderr, "launch: %s\n", hipGetErrorString(e));
}
```

```cpp
#include <hip/hip_runtime.h>
#include <rccl/rccl.h>
#include <cstdio>
#include <cstdint>
#include <cstring>

namespace gar {

#define LAS __attribute__((address_space(3)))
typedef unsigned u32x4 __attribute__((ext_vector_type(4)));
typedef unsigned short u16x2 __attribute__((ext_vector_type(2)));
typedef short bf16x8 __attribute__((ext_vector_type(8)));
typedef float f32x4 __attribute__((ext_vector_type(4)));
typedef __amdgpu_buffer_rsrc_t rsrc_t;

constexpr int WORLD = 8;
constexpr int DIM = 8192;
constexpr int KSH = 1024;
constexpr int NGEMM = 128;
constexpr int GRID = 256;
#define PULL_WG_PER_LINK 12
#define PULL_DECODERS 7
constexpr int PWL = PULL_WG_PER_LINK;
constexpr int NDEC = PULL_DECODERS;
constexpr int NPULL = 7 * PWL;
constexpr int NPACK = GRID - NGEMM - NPULL;
#define NRAW 48
#define NRAWEND 40
#define WARM_TLB 1
#define JIT_LEAD 30
#define OUT_AUX 18
#define IMG_AUX 16
constexpr int NFIRST = (NPACK + NGEMM) * 4;
#define GEMM_PACK_ROUNDS 1
constexpr int GROUNDS = GEMM_PACK_ROUNDS;
constexpr int OUT2 = 4096 + NPACK * 4;
static_assert(NFIRST + GROUNDS * NGEMM * 8 <= 4096 && NFIRST % 8 == 0 && OUT2 % 8 == 0, "roles");
constexpr int NSEQ = 3584;
constexpr unsigned long long DEADLINE = 40000000ull;

constexpr unsigned OFF_TMO = 128;
constexpr unsigned OFF_LINK = 1024;
constexpr unsigned OFF_CNT = 4096;
constexpr unsigned OFF_XFLAG = OFF_CNT + 256 * 128;
constexpr unsigned OFF_OCNT = OFF_XFLAG + 256 * 128;
constexpr unsigned OFF_GRAN = 131072;
constexpr unsigned OFF_CALL = 640u << 10;
constexpr unsigned OFF_SLOT = 1u << 20;
constexpr unsigned OFF_AIMG = 64u << 20;
constexpr unsigned OFF_BIMG = 128u << 20;
constexpr unsigned OFF_XBUF = 160u << 20;
constexpr unsigned WS_USED = 192u << 20;
constexpr unsigned WS_NEED = WS_USED;
static_assert(OFF_OCNT + 128 * 128 <= OFF_GRAN, "map");
static_assert(OFF_GRAN + WORLD * NSEQ * 8 <= OFF_CALL && OFF_CALL + 2048 * 64 <= OFF_SLOT, "map");
static_assert(OFF_SLOT + 6144u * 8192u <= OFF_AIMG, "map");
constexpr int LDS_BYTES = 131072 + 512;

struct Params {
    const void* x; const void* w; void* out;
    char* ws[WORLD];
    int rank; int pad;
};

constexpr int AUX_SC1 = 16, AUX_SYS = 17;

__device__ __forceinline__ rsrc_t mk_rsrc(const void* p, unsigned bytes) {
    return __builtin_amdgcn_make_buffer_rsrc(const_cast<void*>(p), 0, bytes, 0x00020000);
}
__device__ __forceinline__ unsigned rfl(unsigned v) { return __builtin_amdgcn_readfirstlane(v); }
__device__ __forceinline__ unsigned bfi(unsigned mask, unsigned a, unsigned b) { return (a & mask) | (b & ~mask); }
__device__ __forceinline__ unsigned perm(unsigned hi, unsigned lo, unsigned sel) { return __builtin_amdgcn_perm(hi, lo, sel); }
#define WAIT_VM0() asm volatile("s_waitcnt vmcnt(0)" ::: "memory")

#define STAMP(ws, idx) do { } while (0)

__device__ __forceinline__ unsigned call_draw(char* ws) {
    unsigned t;
    const int zero = 0, one = 1;
    unsigned long long keep;
    asm volatile("s_mov_b64 %1, exec\n\ts_mov_b64 exec, 1\n\tglobal_atomic_add %0, %2, %3, %4 sc0\n\ts_mov_b64 exec, %1\n\ts_waitcnt vmcnt(0)"
                 : "=&v"(t), "=&s"(keep) : "v"(zero), "v"(one), "s"(ws + OFF_CALL + (blockIdx.x * 8 + rfl(threadIdx.x >> 6)) * 64) : "memory");
    return rfl(t) + 1u;
}

__device__ __forceinline__ bool give_up(unsigned long long t0, unsigned* tmo, unsigned call) {
    if (rfl(__hip_atomic_load(tmo, __ATOMIC_RELAXED, __HIP_MEMORY_SCOPE_AGENT)) == call) return true;
    if (__builtin_amdgcn_s_memrealtime() - t0 > DEADLINE) { __hip_atomic_store(tmo, call, __ATOMIC_RELAXED, __HIP_MEMORY_SCOPE_AGENT); return true; }
    return false;
}
__device__ __forceinline__ void wait_word(unsigned* p, unsigned target, unsigned* tmo, unsigned call) {
    if ((int)(rfl(__hip_atomic_load(p, __ATOMIC_RELAXED, __HIP_MEMORY_SCOPE_SYSTEM)) - target) >= 0) return;
    const unsigned long long t0 = __builtin_amdgcn_s_memrealtime();
    while ((int)(rfl(__hip_atomic_load(p, __ATOMIC_RELAXED, __HIP_MEMORY_SCOPE_SYSTEM)) - target) < 0) {
        if (give_up(t0, tmo, call)) break;
        __builtin_amdgcn_s_sleep(4);
    }
}

constexpr unsigned MAX_EXC = 31;
constexpr unsigned RAWB = 3584, AUX_U = 20, AUX_MAX = 2560, U_MAX = 10240, LANE_U_MAX = 224;
constexpr unsigned ENC_HIST = AUX_MAX + 16, ENC_SORT = ENC_HIST + 256, ENC_END = ENC_SORT + 128;

__device__ __forceinline__ unsigned bfi_c(unsigned mask, unsigned a, unsigned b) { unsigned r; asm("v_bfi_b32 %0, %1, %2, %3" : "=v"(r) : "s"(mask), "v"(a), "v"(b)); return r; }
__device__ __forceinline__ unsigned opaque(unsigned v) { asm("" : "+v"(v)); return v; }
__device__ __forceinline__ unsigned bytemask(unsigned x, int sh, unsigned c0c) {
    unsigned t; asm("v_and_or_b32 %0, %1, %2, %3" : "=v"(t) : "v"(x >> sh), "s"(0x01010101u), "v"(c0c));
    return perm(0u, 0u, t);
}
template <int CTRL, int RM = 0xf> __device__ __forceinline__ unsigned dpp0(unsigned v) { return (unsigned)__builtin_amdgcn_update_dpp(0, (int)v, CTRL, RM, 0xf, RM == 0xf); }
__device__ __forceinline__ unsigned wave_scan_incl(unsigned v) {
    v += dpp0<0x111>(v); v += dpp0<0x112>(v); v += dpp0<0x114>(v); v += dpp0<0x118>(v);
    v += dpp0<0x142, 0xa>(v); v += dpp0<0x143, 0xc>(v);
    return v;
}
__device__ __forceinline__ unsigned umin(unsigned a, unsigned b) { return a < b ? a : b; }
__device__ __forceinline__ unsigned umax(unsigned a, unsigned b) { return a > b ? a : b; }
__device__ __forceinline__ unsigned sad4(unsigned x) { return __builtin_amdgcn_sad_u8(x, 0u, 0u); }
__device__ __forceinline__ unsigned lut2x16(const unsigned (&T)[8], unsigned idx, unsigned ms, unsigned c0c) {
    const unsigned sel = idx & 0x07070707u, m8 = bytemask(idx, 3, c0c);
    const unsigned p = bfi(m8, perm(T[3], T[2], sel), perm(T[1], T[0], sel)), n = bfi(m8, perm(T[7], T[6], sel), perm(T[5], T[4], sel));
    return bfi(ms, n, p);
}
#define LDS_FENCE() asm volatile("" ::: "memory")
__device__ __forceinline__ void lds_or2(LAS unsigned char* scr, unsigned p, unsigned w) {
    const unsigned s = p & 31u;
    LAS unsigned* q = (LAS unsigned*)(scr + ((p >> 5) << 2));
    __hip_atomic_fetch_or(q, w << s, __ATOMIC_RELAXED, __HIP_MEMORY_SCOPE_WORKGROUP);
    __hip_atomic_fetch_or(q + 1, (w >> 1) >> (31u - s), __ATOMIC_RELAXED, __HIP_MEMORY_SCOPE_WORKGROUP);
}

__device__ __forceinline__ unsigned encode_block(const u32x4 (&c)[8], unsigned (&P)[14], u32x4 (&a)[3], LAS unsigned char* scr, int lane) {
    asm volatile("" : "+v"(lane));
    const unsigned c0c = opaque(0x0C0C0C0Cu);
    unsigned E[16], R[16];
    u16x2 p1 = {0, 0}, p2 = {0, 0}, n1 = {0, 0}, n2 = {0, 0};
#pragma unroll
    for (int g = 0; g < 16; ++g) {
        const unsigned x = (g & 1) ? c[g >> 1].z : c[g >> 1].x, y = (g & 1) ? c[g >> 1].w : c[g >> 1].y;
        const unsigned H = perm(y, x, 0x07050301u), L = perm(y, x, 0x06040200u);
        E[g] = bfi_c(0xFEFEFEFEu, H << 1, L >> 7);
        R[g] = bfi_c(0x80808080u, L << 1, bfi_c(0x40404040u, H >> 1, L));
        const unsigned en = E[g] & bytemask(H, 7, c0c), ep = E[g] ^ en;
        p1 = __builtin_elementwise_max(p1, __builtin_bit_cast(u16x2, ep)); p2 = __builtin_elementwise_max(p2, __builtin_bit_cast(u16x2, ep << 8));
        n1 = __builtin_elementwise_max(n1, __builtin_bit_cast(u16x2, en)); n2 = __builtin_elementwise_max(n2, __builtin_bit_cast(u16x2, en << 8));
        __builtin_amdgcn_sched_barrier(0);
    }
    const u32x4 z4 = {0u, 0u, 0u, 0u};
    *(LAS u32x4*)(scr + lane * 16) = z4; *(LAS u32x4*)(scr + 1024 + lane * 16) = z4;
    if (lane < 49) *(LAS u32x4*)(scr + 2048 + lane * 16) = z4;
    unsigned baseP, baseN;
    {
        const u16x2 mp = __builtin_elementwise_max(p1, p2), mn = __builtin_elementwise_max(n1, n2);
        u16x2 m = {(unsigned short)((mp.x > mp.y ? mp.x : mp.y) >> 8), (unsigned short)((mn.x > mn.y ? mn.x : mn.y) >> 8)};
        m = __builtin_elementwise_max(m, __builtin_bit_cast(u16x2, dpp0<0x128>(__builtin_bit_cast(unsigned, m)))); m = __builtin_elementwise_max(m, __builtin_bit_cast(u16x2, dpp0<0x124>(__builtin_bit_cast(unsigned, m))));
        m = __builtin_elementwise_max(m, __builtin_bit_cast(u16x2, dpp0<0x122>(__builtin_bit_cast(unsigned, m)))); m = __builtin_elementwise_max(m, __builtin_bit_cast(u16x2, dpp0<0x121>(__builtin_bit_cast(unsigned, m))));
        const unsigned mu = __builtin_bit_cast(unsigned, m);
        const u16x2 r01 = __builtin_elementwise_max(__builtin_bit_cast(u16x2, (unsigned)__builtin_amdgcn_readlane(mu, 0)), __builtin_bit_cast(u16x2, (unsigned)__builtin_amdgcn_readlane(mu, 16)));
        const u16x2 r23 = __builtin_elementwise_max(__builtin_bit_cast(u16x2, (unsigned)__builtin_amdgcn_readlane(mu, 32)), __builtin_bit_cast(u16x2, (unsigned)__builtin_amdgcn_readlane(mu, 48)));
        const unsigned mm = rfl(__builtin_bit_cast(unsigned, __builtin_elementwise_max(r01, r23)));
        baseP = mm & 0xFFu; baseN = mm >> 16;
    }
    const unsigned BP = baseP * 0x01010101u, BN = baseN * 0x01010101u;
    unsigned any = 0, D[16];
#pragma unroll
    for (int g = 0; g < 16; ++g) {
        E[g] = bfi(bytemask(R[g], 6, c0c), BN, BP) - E[g]; any |= E[g];
        D[g] = bfi_c(0x1E1E1E1Eu, E[g] << 1, (R[g] >> 7) & 0x01010101u);
        R[g] &= 0x7F7F7F7Fu;
    }
    LDS_FENCE();
#pragma unroll
    for (int gi = 0; gi < 4; ++gi) {
        const int g = gi * 5;
        const unsigned k4 = ((D[g] & 0x0F0F0F0Fu) | ((R[g] >> 2) & 0x10101010u) | ((D[g] << 1) & 0x20202020u)) << 2;
#pragma unroll
        for (int b = 0; b < 4; ++b)
            __hip_atomic_fetch_add((LAS unsigned*)(scr + ENC_HIST + ((k4 >> (8 * b)) & 0xFFu)), 1u, __ATOMIC_RELAXED, __HIP_MEMORY_SCOPE_WORKGROUP);
    }
    LDS_FENCE();
    const unsigned cnt = *(LAS unsigned*)(scr + ENC_HIST + lane * 4);
    const unsigned key = (cnt << 4) | (15u - (lane & 15));
    unsigned rank = 0;
    rank += dpp0<0x121>(key) > key; rank += dpp0<0x122>(key) > key; rank += dpp0<0x123>(key) > key; rank += dpp0<0x124>(key) > key; rank += dpp0<0x125>(key) > key;
    rank += dpp0<0x126>(key) > key; rank += dpp0<0x127>(key) > key; rank += dpp0<0x128>(key) > key; rank += dpp0<0x129>(key) > key; rank += dpp0<0x12A>(key) > key;
    rank += dpp0<0x12B>(key) > key; rank += dpp0<0x12C>(key) > key; rank += dpp0<0x12D>(key) > key; rank += dpp0<0x12E>(key) > key; rank += dpp0<0x12F>(key) > key;
    unsigned tail = cnt; tail += dpp0<0x128>(tail); tail += dpp0<0x124>(tail); tail += dpp0<0x122>(tail); tail += dpp0<0x121>(tail);
    const int sg = (lane >> 4) & 1;
    const unsigned ct = sg ? __builtin_amdgcn_readlane(tail, 48) : __builtin_amdgcn_readlane(tail, 32);
    if (lane < 32) *(LAS unsigned*)(scr + ENC_SORT + ((lane & 16) + rank) * 4) = cnt;
    LDS_FENCE();
    unsigned best;
    {
        const unsigned cb = lane & 15, s0 = cb / 3u, s1 = cb - 3u * s0, A = 1u << s0, AB = A + (1u << s1);
        unsigned cost = ct * (16u - AB);
#pragma unroll
        for (int q = 0; q < 4; ++q) {
            const u32x4 h = *(LAS u32x4*)(scr + ENC_SORT + sg * 64 + q * 16);
#pragma unroll
            for (int j = 0; j < 4; ++j) {
                const unsigned r = q * 4 + j, len = r < A ? 1u + s0 : r < AB ? 2u + s1 : 4u + ((r - AB) >> 1);
                cost += 2u * len * h[j];
            }
        }
        best = cb < 9 ? (cost << 4) | cb : 0xFFFFFFFFu;
        best = umin(best, dpp0<0x128>(best)); best = umin(best, dpp0<0x124>(best)); best = umin(best, dpp0<0x122>(best)); best = umin(best, dpp0<0x121>(best));
    }
    const unsigned cbP = __builtin_amdgcn_readlane(best, 0) & 15u, cbN = __builtin_amdgcn_readlane(best, 16) & 15u;
    const unsigned s0P = cbP / 3u, s1P = cbP - 3u * s0P, s0N = cbN / 3u, s1N = cbN - 3u * s0N;
    const unsigned ABP = (1u << s0P) + (1u << s1P), ABN = (1u << s0N) + (1u << s1N);
    unsigned F[8];
    {
        const unsigned s0 = sg ? s0N : s0P, s1 = sg ? s1N : s1P, A = 1u << s0, AB = sg ? ABN : ABP;
        const unsigned ent = rank < A ? (rank << 4) | (s0 << 6) : rank < AB ? 1u | ((rank - A) << 4) | (s1 << 6) : (2u + ((rank - AB) >> 1)) | (((rank - AB) & 1u) << 4) | (1u << 6);
        unsigned fv = ent << (8 * (lane & 3));
        fv |= dpp0<0xB1>(fv); fv |= dpp0<0x4E>(fv);
#pragma unroll
        for (int q = 0; q < 8; ++q) F[q] = __builtin_amdgcn_readlane(fv, q * 4);
        if (lane < 32)
            __hip_atomic_fetch_or((LAS unsigned*)(scr + (sg * 2 + (rank >> 3)) * 4), (unsigned)(lane & 15) << (8 * (rank & 3) + 4 * ((rank >> 2) & 1)), __ATOMIC_RELAXED, __HIP_MEMORY_SCOPE_WORKGROUP);
        if (lane == 0) *(LAS unsigned*)(scr + 16) = baseP | (baseN << 8) | (s0P << 16) | (s1P << 18) | (s0N << 20) | (s1N << 22);
    }
    const unsigned ABP4 = ABP * 0x01010101u, ABN4 = ABN * 0x01010101u, S0P4 = s0P * 0x01010101u, S0N4 = s0N * 0x01010101u;
    unsigned KK[16], XW[4], OX[4], TU = 0, TX = 0, bad = 0, xl = 0, xw = 0;
#pragma unroll
    for (int g = 0; g < 16; ++g) {
        const unsigned ms = bytemask(R[g], 6, c0c), m16 = bytemask(D[g], 4, c0c), ent = lut2x16(F, D[g], ms, c0c);
        const unsigned q = ((D[g] | 0x80808080u) - bfi(ms, ABN4, ABP4)) & 0x7F7F7F7Fu;
        unsigned K = bfi(m16, ((q >> 1) & 0x3F3F3F3Fu) + 0x02020202u, ent & 0x0F0F0F0Fu);
        unsigned SUF = bfi(m16, q & 0x01010101u, (ent >> 4) & 0x03030303u);
        unsigned S = bfi(m16, 0x01010101u, (ent >> 6) & 0x03030303u);
        unsigned gl = sad4(K) + 4u;
        if (__ballot(gl > 31u) != 0ull) {
            any |= 0x80u;
            const unsigned km = umax(umax(K & 0xFFu, (K >> 8) & 0xFFu), umax((K >> 16) & 0xFFu, K >> 24));
            const unsigned eq = gl > 31u ? (((K ^ (km * 0x01010101u)) + 0x7F7F7F7Fu) & 0x80808080u) ^ 0x80808080u : 0u, em = bytemask(eq, 7, c0c);
            E[g] |= eq;
            K &= ~em; SUF &= ~em; S = bfi(em, bfi(ms, S0N4, S0P4), S);
            gl = sad4(K) + 4u;
        }
        KK[g] = K; TU += gl; bad = umax(bad, gl);
        xw |= (SUF & 0xFFu) << xl; xl += S & 0xFFu;
        xw |= ((SUF >> 8) & 0xFFu) << xl; xl += (S >> 8) & 0xFFu;
        xw |= ((SUF >> 16) & 0xFFu) << xl; xl += (S >> 16) & 0xFFu;
        xw |= (SUF >> 24) << (xl & 31u); xl += S >> 24;
        if ((g & 3) == 3) { XW[g >> 2] = xw; OX[g >> 2] = TX; TX += xl; xw = 0; xl = 0; }
        __builtin_amdgcn_sched_barrier(0);
    }
    bad = (bad > 31u) | (TU > LANE_U_MAX);
    const unsigned tt = TU | (TX << 16), incl = wave_scan_incl(tt), tot = __builtin_amdgcn_readlane(incl, 63), totU = tot & 0xFFFFu, totX = tot >> 16;
    const unsigned xstart = (AUX_U * 8u + totU + 31u) & ~31u, endbit = xstart + totX;
    unsigned raw = (__ballot(bad != 0) != 0ull) || totU > U_MAX || endbit > (AUX_MAX - 8u) * 8u;
    if (!raw) {
        const unsigned px = xstart + ((incl - tt) >> 16);
        unsigned p = AUX_U * 8u + ((incl - tt) & 0xFFFFu);
#pragma unroll
        for (int g = 0; g < 16; ++g) {
            const unsigned ln = KK[g] + 0x01010101u;
            unsigned pos = (ln & 0xFFu) - 1u, w = 1u << pos;
            pos += (ln >> 8) & 0xFFu; w |= 1u << pos;
            pos += (ln >> 16) & 0xFFu; w |= 1u << pos;
            pos += ln >> 24; w |= 1u << pos;
            lds_or2(scr, p, w); p += pos + 1u;
            __builtin_amdgcn_sched_barrier(0);
        }
#pragma unroll
        for (int q = 0; q < 4; ++q) lds_or2(scr, px + OX[q], XW[q]);
    }
#pragma unroll
    for (int i = 0; i < 7; ++i) { P[i] = R[i] | ((R[14] << (7 - i)) & 0x80808080u); P[7 + i] = R[7 + i] | ((R[15] << (7 - i)) & 0x80808080u); }
    unsigned n = 0;
    const unsigned xo = (endbit + 31u) >> 5;
    if (!raw && __ballot((any & 0xF0F0F0F0u) != 0) != 0ull) {
#pragma unroll
        for (int g = 0; g < 16; ++g) {
            const unsigned f = E[g] & 0xF0F0F0F0u;
            unsigned long long mask = __ballot(f != 0);
            while (mask != 0ull && n <= MAX_EXC) {
                const int L = __builtin_ctzll(mask); mask &= mask - 1;
                const unsigned fl = __builtin_amdgcn_readlane(f, L);
                const unsigned lo = __builtin_amdgcn_readlane((g & 1) ? c[g >> 1].z : c[g >> 1].x, L);
                const unsigned hi = __builtin_amdgcn_readlane((g & 1) ? c[g >> 1].w : c[g >> 1].y, L);
                const bool e = lane < 4 && ((fl >> (8 * (lane & 3))) & 0xFFu) != 0;
                const unsigned long long em = __ballot(e);
                const unsigned slot = xo + n + __builtin_popcountll(em & ((1ull << lane) - 1ull));
                const unsigned val = (((lane & 2) ? hi : lo) >> (16 * (lane & 1))) & 0xFFFFu;
                if (e && slot * 4u < AUX_MAX) *(LAS unsigned*)(scr + slot * 4u) = (unsigned)(L * 64 + g * 4 + (lane & 3)) | (val << 12);
                n += __builtin_popcountll(em);
            }
        }
    }
    raw |= (n > MAX_EXC) | ((xo + n) * 4u > AUX_MAX);
    LDS_FENCE();
    a[0] = *(LAS u32x4*)(scr + lane * 16); a[1] = *(LAS u32x4*)(scr + 1024 + lane * 16); a[2] = *(LAS u32x4*)(scr + 2048 + (lane & 31) * 16);
    return ((xo + n + 3u) >> 2) | (n << 8) | (xo << 16) | (raw << 31);
}

__device__ __forceinline__ void decode_block(LAS unsigned char* src, LAS unsigned char* pos_scr, u32x4 (&c)[8], int lane) {
    LAS unsigned char* aux = src + RAWB;
    const unsigned c0c = opaque(0x0C0C0C0Cu);
    const u32x4 nt = *(LAS const u32x4*)(aux);
    const unsigned prm_v = *(LAS const unsigned*)(aux + 16);
    unsigned u[5];
#pragma unroll
    for (int i = 0; i < 5; ++i) u[i] = *(LAS const unsigned*)(aux + AUX_U + lane * 20 + i * 4);
    u32x4 P4[3];
#pragma unroll
    for (int p = 0; p < 3; ++p) P4[p] = *(LAS const u32x4*)(src + p * 1024 + lane * 16);
    const unsigned long long ph = *(LAS const unsigned long long*)(src + 3072 + lane * 8);
    const unsigned c0 = __builtin_popcount(u[0]), c1 = c0 + __builtin_popcount(u[1]), c2 = c1 + __builtin_popcount(u[2]), c3 = c2 + __builtin_popcount(u[3]), cn = c3 + __builtin_popcount(u[4]);
    const unsigned C = wave_scan_incl(cn) - cn;
    if (lane == 0) *(LAS unsigned*)(pos_scr) = AUX_U * 8u;
    const int rounds = __ballot(cn > 128u) != 0ull ? 3 : 2;
    for (int j = 0; j < rounds; ++j) {
        const unsigned t = (C >> 6) + 1u + j, kth = 64u * t - C;
        const unsigned i = (kth > c0) + (kth > c1) + (kth > c2) + (kth > c3);
        unsigned rem = kth - (i == 0 ? 0u : i == 1 ? c0 : i == 2 ? c1 : i == 3 ? c2 : c3), w = i == 0 ? u[0] : i == 1 ? u[1] : i == 2 ? u[2] : i == 3 ? u[3] : u[4], pos = 0;
#pragma unroll
        for (int st = 16; st >= 1; st >>= 1) {
            const unsigned lo = __builtin_popcount(w & ((1u << st) - 1u));
            const bool up = rem > lo;
            rem -= up ? lo : 0u; w = up ? w >> st : w; pos += up ? st : 0;
        }
        if (kth <= cn && t < 64u) *(LAS unsigned*)(pos_scr + t * 4) = AUX_U * 8u + lane * 160u + i * 32u + pos + 1u;
    }
    LDS_FENCE();
    const unsigned p0 = *(LAS const unsigned*)(pos_scr + lane * 4);
    unsigned S[8];
    {
        LAS const unsigned* q = (LAS const unsigned*)(aux + ((p0 >> 5) << 2));
#pragma unroll
        for (int i = 0; i < 8; ++i) S[i] = q[i];
#pragma unroll
        for (int i = 0; i < 7; ++i) S[i] = __builtin_amdgcn_alignbit(S[i + 1], S[i], p0 & 31u);
    }
    unsigned R[16];
    {
        const unsigned P[14] = {P4[0].x, P4[0].y, P4[0].z, P4[0].w, P4[1].x, P4[1].y, P4[1].z, P4[1].w, P4[2].x, P4[2].y, P4[2].z, P4[2].w, (unsigned)ph, (unsigned)(ph >> 32)};
        unsigned x = 0, y = 0;
#pragma unroll
        for (int i = 0; i < 7; ++i) { x |= (P[i] >> (7 - i)) & (0x01010101u << i); y |= (P[7 + i] >> (7 - i)) & (0x01010101u << i); }
#pragma unroll
        for (int i = 0; i < 14; ++i) R[i] = P[i] & 0x7F7F7F7Fu;
        R[14] = x; R[15] = y;
    }
    const unsigned n0 = rfl(nt.x), n1 = rfl(nt.y), n2 = rfl(nt.z), n3 = rfl(nt.w), prm = rfl(prm_v);
    const unsigned I[8] = {n0 & 0x0F0F0F0Fu, (n0 >> 4) & 0x0F0F0F0Fu, n1 & 0x0F0F0F0Fu, (n1 >> 4) & 0x0F0F0F0Fu, n2 & 0x0F0F0F0Fu, (n2 >> 4) & 0x0F0F0F0Fu, n3 & 0x0F0F0F0Fu, (n3 >> 4) & 0x0F0F0F0Fu};
    const unsigned s0P = (prm >> 16) & 3u, s1P = (prm >> 18) & 3u, s0N = (prm >> 20) & 3u, s1N = (prm >> 22) & 3u;
    const unsigned STP = (s0P << 8) | (s1P << 16) | (1u << 24), STN = (s0N << 8) | (s1N << 16) | (1u << 24);
    const unsigned OTP = (4u << 8) | (((1u << s0P) + 2u) << 16) | (((1u << s0P) + (1u << s1P)) << 24);
    const unsigned OTN = (4u << 8) | (((1u << s0N) + 2u) << 16) | (((1u << s0N) + (1u << s1N)) << 24);
    const unsigned BP = (prm & 0xFFu) * 0x01010101u, BN = ((prm >> 8) & 0xFFu) * 0x01010101u;
    unsigned RB[16], SS[16], TU = 0, TX = 0;
#pragma unroll
    for (int g = 0; g < 16; ++g) {
        const unsigned w0 = S[0], q0 = __builtin_ctz(w0), w1 = w0 & (w0 - 1u), q1 = __builtin_ctz(w1), w2 = w1 & (w1 - 1u), q2 = __builtin_ctz(w2), w3 = w2 & (w2 - 1u), q3 = __builtin_ctz(w3);
        const unsigned Q = q0 | (q1 << 8) | (q2 << 16) | (q3 << 24);
        const unsigned LN = Q - opaque((Q << 8) - 1u), sh = q3 + 1u;
        TU += sh;
        if (g < 15) {
#pragma unroll
            for (int i = 0; i < 7; ++i) if (32 * i < (int)LANE_U_MAX - 4 * (g + 1)) S[i] = (i < 6 && 32 * (i + 1) < (int)LANE_U_MAX - 4 * g) ? __builtin_amdgcn_alignbit(S[i + 1], S[i], sh) : S[i] >> sh;
        }
        const unsigned sel = bfi_c(0x04040404u, R[g] >> 4, bfi(bytemask(LN + 0x7D7D7D7Du, 7, c0c), 0x03030303u, LN));
        SS[g] = perm(STN, STP, sel);
        RB[g] = perm(OTN, OTP, sel) + (LN << 1);
        TX = __builtin_amdgcn_sad_u8(SS[g], 0u, TX);
    }
    const unsigned xstart = (__builtin_amdgcn_readlane(p0 + TU, 63) + 31u) & ~31u, px = xstart + wave_scan_incl(TX) - TX;
    unsigned X[5];
    {
        LAS const unsigned* q = (LAS const unsigned*)(aux + ((px >> 5) << 2));
#pragma unroll
        for (int i = 0; i < 5; ++i) X[i] = q[i];
#pragma unroll
        for (int i = 0; i < 4; ++i) X[i] = __builtin_amdgcn_alignbit(X[i + 1], X[i], px & 31u);
    }
    unsigned xoff = 0;
#pragma unroll
    for (int g = 0; g < 16; ++g) {
        const unsigned s = SS[g], M = perm(0u, 0x00030100u, s);
        const unsigned y = (g & 1) ? X[0] >> xoff : X[0], Y = perm(y, y, 0x01000100u);
        const unsigned t1 = opaque(s << 8), t2 = opaque((t1 << 8) + t1), Cs = (t2 << 16) + t2;
        const u16x2 z02 = __builtin_bit_cast(u16x2, Y) >> __builtin_bit_cast(u16x2, Cs & 0x00FF00FFu);
        const u16x2 z13 = __builtin_bit_cast(u16x2, Y) >> __builtin_bit_cast(u16x2, perm(0u, Cs, 0x0C030C01u));
        const unsigned suf = perm(__builtin_bit_cast(unsigned, z13), __builtin_bit_cast(unsigned, z02), 0x06020400u) & M;
        if (g & 1) {
            if (g < 15) {
                const unsigned t = __builtin_amdgcn_sad_u8(s, 0u, xoff);
#pragma unroll
                for (int i = 0; i < 4; ++i) if (32 * i < 8 * (15 - g)) X[i] = (i < 3 && 32 * (i + 1) < 8 * (17 - g)) ? __builtin_amdgcn_alignbit(X[i + 1], X[i], t) : X[i] >> t;
            }
        } else xoff = sad4(s);
        const unsigned rank = RB[g] + suf - 0x06060606u, ms = bytemask(R[g], 6, c0c);
        const unsigned D = bfi(bytemask(rank, 4, c0c), rank, lut2x16(I, rank, ms, c0c));
        const unsigned E = bfi(ms, BN, BP) - ((D >> 1) & 0x0F0F0F0Fu);
        const unsigned H = bfi_c(0x80808080u, R[g] << 1, E >> 1), L = bfi_c(0x3F3F3F3Fu, R[g], bfi_c(0x40404040u, D << 6, E << 7));
        const unsigned x = perm(H, L, 0x05010400u), yv = perm(H, L, 0x07030602u);
        if (g & 1) { c[g >> 1].z = x; c[g >> 1].w = yv; } else { c[g >> 1].x = x; c[g >> 1].y = yv; }
    }
}

typedef unsigned u32x2 __attribute__((ext_vector_type(2)));
__device__ __forceinline__ void store_granule(char* p, u32x2 g) {
    asm volatile("global_store_dwordx2 %0, %1, off sc0 sc1\n\ts_nop 1" :: "v"(p), "v"(g) : "memory");
}

constexpr unsigned GCTL = 131072;
__device__ __forceinline__ unsigned payload_bytes(unsigned h) { return (h >> 31) ? 8192u : RAWB + ((h & 255u) << 4); }
__device__ __forceinline__ void publish_block(const Params& P, rsrc_t wsr, const u32x4 (&c)[8], unsigned slot, unsigned n, unsigned dmask, unsigned call, LAS unsigned char* lds, LAS unsigned char* scr,
                                              unsigned j, unsigned gsize, int wid, int lane) {
    unsigned R[14], h = 1u << 31; u32x4 a[3];
    if (n >= NRAW && n < NSEQ - NRAWEND) h = encode_block(c, R, a, scr, lane);
    const unsigned u = j & 3u;
    volatile LAS unsigned* ctl = (volatile LAS unsigned*)(lds + GCTL);
    while (j >= 4u && (int)(ctl[36 + u] - (j - 3u)) < 0) __builtin_amdgcn_s_sleep(1);
    unsigned so = OFF_SLOT + (slot & ~1u) * 8192u;
    if (wid & 1) {
        unsigned v;
        while (((v = ctl[40 + u * 8 + wid - 1]) >> 16) != ((j + 1u) & 0xFFFFu)) __builtin_amdgcn_s_sleep(1);
        so += (rfl(v) & 0xFFFFu) << 4;
    } else if (lane == 0) ctl[40 + u * 8 + wid] = ((j + 1u) << 16) | (payload_bytes(h) >> 4);
    if (h >> 31) {
#pragma unroll
        for (int q = 0; q < 8; ++q) __builtin_amdgcn_raw_buffer_store_b128(c[q], wsr, lane * 16 + q * 1024, so, AUX_SYS);
    } else {
        const unsigned n16 = h & 255u;
#pragma unroll
        for (int q = 0; q < 3; ++q) __builtin_amdgcn_raw_buffer_store_b128((u32x4){R[4 * q], R[4 * q + 1], R[4 * q + 2], R[4 * q + 3]}, wsr, lane * 16 + q * 1024, so, AUX_SYS);
        __builtin_amdgcn_raw_buffer_store_b64((u32x2){R[12], R[13]}, wsr, lane * 8 + 3072, so, AUX_SYS);
        if ((unsigned)lane < n16) __builtin_amdgcn_raw_buffer_store_b128(a[0], wsr, lane * 16 + RAWB, so, AUX_SYS);
        if ((unsigned)lane + 64u < n16) __builtin_amdgcn_raw_buffer_store_b128(a[1], wsr, lane * 16 + RAWB + 1024, so, AUX_SYS);
        if ((unsigned)lane + 128u < n16) __builtin_amdgcn_raw_buffer_store_b128(a[2], wsr, lane * 16 + RAWB + 2048, so, AUX_SYS);
    }
    WAIT_VM0();
    unsigned arrived = 0;
    if (lane == 0) {
        ctl[u * 8 + wid] = h;
        arrived = __hip_atomic_fetch_add((LAS unsigned*)(lds + GCTL) + 32 + u, 1u, __ATOMIC_RELAXED, __HIP_MEMORY_SCOPE_WORKGROUP);
    }
    if (rfl(arrived) == gsize - 1u) {
        const unsigned d = lane >> 3, w = lane & 7, hw = ctl[u * 8 + w];
        if (w < gsize && ((dmask >> d) & 1u)) store_granule(P.ws[d] + OFF_GRAN + ((unsigned)P.rank * NSEQ + n - wid + w) * 8u, (u32x2){call, hw});
        if (lane == 0) { ctl[32 + u] = 0u; ctl[36 + u] = j + 1u; }
    }
}

__device__ __forceinline__ void signal_cnt(char* ws, unsigned off, int lane) {
    if (off && lane == 0) __hip_atomic_fetch_add((unsigned*)(ws + off), 1u, __ATOMIC_RELAXED, __HIP_MEMORY_SCOPE_SYSTEM);
}
__device__ __forceinline__ unsigned packer_role(const Params& P, LAS unsigned char* lds, int first, int next, int step, int end) {
    unsigned call = 0;
    const int lane = threadIdx.x & 63, wid = rfl(threadIdx.x >> 6);
    char* ws = P.ws[P.rank];
    const rsrc_t wsr = mk_rsrc(ws, WS_USED);
    const rsrc_t xr = mk_rsrc(P.x, DIM * KSH * 2), wr = mk_rsrc(P.w, DIM * KSH * 2);
    const int myi = P.rank >> 2, myj = P.rank & 3;
    LAS unsigned char* my = lds + wid * 8192;
    unsigned j = first < NFIRST ? 0u : 1u;
    const unsigned long long tstart = __builtin_amdgcn_s_memrealtime();
    for (int p = first; p < end; ++j) {
        if (p < 4096) {
            const int due = (p < 3072 ? (p / 192) * 580 : 9280 + ((p - 3072) >> 6) * 290) - JIT_LEAD * 100;
            while ((int)(__builtin_amdgcn_s_memrealtime() - tstart) < due) __builtin_amdgcn_s_sleep(16);
        }
        u32x4 c[8];
        unsigned slot, n, dmask, selfcnt = 0;
        if (p >= 4096) {
            const unsigned o = p - 4096, w = o & 7, h = (o >> 3) & 1, tile = (o >> 4) & 63;
            if (call == 0) call = call_draw(ws);
            unsigned* tmo = (unsigned*)(ws + OFF_TMO);
            wait_word((unsigned*)(ws + OFF_OCNT + (o >> 3) * 128), call * 8u, tmo, call);
            const unsigned row0 = (P.rank >> 2) * 4096 + (tile >> 3) * 256 + h * 128 + (w >> 2) * 64, col0 = (P.rank & 3) * 2048 + (tile & 7) * 256 + (w & 3) * 64;
            const rsrc_t outr = mk_rsrc(P.out, (unsigned)DIM * DIM * 2u);
#pragma unroll
            for (int q = 0; q < 8; ++q) c[q] = __builtin_amdgcn_raw_buffer_load_b128(outr, (unsigned)(lane >> 3) * (DIM * 2u) + (unsigned)(lane & 7) * 16u, (row0 * DIM + col0) * 2u + q * (8u * DIM * 2u), AUX_SYS);
            if (rfl(__hip_atomic_load(tmo, __ATOMIC_RELAXED, __HIP_MEMORY_SCOPE_AGENT)) == call) {
#pragma unroll
                for (int q = 0; q < 8; ++q) c[q] = (u32x4){0x7FC07FC0u, 0x7FC07FC0u, 0x7FC07FC0u, 0x7FC07FC0u};
            }
            slot = 4096u + o; n = 1536u + o; dmask = 0xFFu;
        } else {
            bool is_w; int kk, sel, r, bi;
            if (p < 3072) {
                kk = p / 192; const int rem = p - kk * 192;
                bi = rem & 31; r = 0; is_w = rem >= 64; sel = is_w ? ((rem - 64) >> 5) : (rem >> 5);
            } else { const int q = p - 3072; kk = q >> 6; sel = (q >> 5) & 1; bi = q & 31; r = 1; is_w = false; }
            if (!is_w) {
                const int rr = lane >> 3;
                const unsigned vo = (unsigned)rr * 2048u + (unsigned)(((lane & 7) ^ rr) * 16);
                const unsigned so = (unsigned)(sel * 4096 + r * 2048 + bi * 64) * 2048u + (unsigned)kk * 128u;
#pragma unroll
                for (int q = 0; q < 8; ++q) c[q] = __builtin_amdgcn_raw_buffer_load_b128(xr, vo + q * 8 * 2048, so, 0);
            } else {
                const int pn = bi >> 2, bj = (bi >> 1) & 1, q = bi & 1;
                const int gk = lane >> 3, cc = lane & 7, wcl = cc >> 2, cq = cc & 3;
                const unsigned vo = (unsigned)(gk * 8) * 16384u + (unsigned)((64 * (2 * q + wcl) + 32 * bj + 8 * cq) * 2);
                const unsigned so = (unsigned)(kk * 64) * 16384u + (unsigned)((sel * 2048 + pn * 256) * 2);
                u32x4 R[8];
#pragma unroll
                for (int jj = 0; jj < 8; ++jj) R[jj] = __builtin_amdgcn_raw_buffer_load_b128(wr, vo + jj * 16384, so, 0);
#pragma unroll
                for (int t = 0; t < 8; ++t) {
                    const unsigned s = (t & 1) ? 0x07060302u : 0x05040100u;
                    u32x4 v;
                    v.x = perm(R[1][t >> 1], R[0][t >> 1], s); v.y = perm(R[3][t >> 1], R[2][t >> 1], s);
                    v.z = perm(R[5][t >> 1], R[4][t >> 1], s); v.w = perm(R[7][t >> 1], R[6][t >> 1], s);
                    const int s64 = 32 * wcl + 16 * (t >> 2) + 4 * cq + (t & 3);
                    *(LAS u32x4*)(my + s64 * 128 + ((gk ^ (s64 & 7)) * 16)) = v;
                }
                asm volatile("s_waitcnt lgkmcnt(0)" ::: "memory");
#pragma unroll
                for (int q2 = 0; q2 < 8; ++q2) c[q2] = *(LAS u32x4*)(my + q2 * 1024 + lane * 16);
                asm volatile("s_waitcnt lgkmcnt(0)" ::: "memory");
            }
            if (call == 0) call = call_draw(ws);
            const int g = P.rank * 16 + kk;
            if (is_w ? (sel == myj) : (sel == myi)) {
                const unsigned so = is_w ? OFF_BIMG + (unsigned)(g * 32 + bi) * 8192u : OFF_AIMG + (unsigned)((r * 128 + g) * 32 + bi) * 8192u;
#pragma unroll
                for (int q = 0; q < 8; ++q) __builtin_amdgcn_raw_buffer_store_b128(c[q], wsr, lane * 16 + q * 1024, so, AUX_SC1);
                selfcnt = OFF_CNT + (r * 128 + g) * 128;
            }
            slot = is_w ? 2048u + (unsigned)((sel * 16 + kk) * 32 + bi) : (unsigned)(((sel * 2 + r) * 16 + kk) * 32 + bi);
            n = is_w ? (unsigned)(kk * 64 + 32 + bi) : (r == 0 ? (unsigned)(kk * 64 + bi) : (unsigned)(1024 + kk * 32 + bi));
            dmask = is_w ? (0x11u << sel) : (0x0Fu << (4 * sel));
        }
        publish_block(P, wsr, c, slot, n, dmask & ~(1u << P.rank), call, lds, my, j, (p < NFIRST || (p >= 4096 && p < OUT2)) ? 4u : 8u, wid, lane);
        signal_cnt(ws, selfcnt, lane);
        if (p < NFIRST) p = next;
        else if (p < 4096) {
            p += step;
            if (p >= 4096 && end > 4096) { if (wid < 4) p = 4096 + (int)blockIdx.x * 4 + wid; else { p = OUT2 + (int)blockIdx.x * 8 + wid; ++j; } }
        } else if (p < OUT2) p = OUT2 + (int)blockIdx.x * 8 + wid;
        else p += step;
    }
    if (call == 0) call = call_draw(ws);
    return call;
}

struct Blk { unsigned slot, dso, cnt; bool is_out; };
__device__ __forceinline__ Blk locate(unsigned n, int src, int myi, int myj) {
    Blk b; b.is_out = false;
    if (n < 1024) { const int kk = n >> 6, bi = n & 31, g = src * 16 + kk; b.cnt = OFF_CNT + g * 128;
        if ((n & 63) >= 32) { b.slot = 2048u + (myj * 16 + kk) * 32 + bi; b.dso = OFF_BIMG + (unsigned)(g * 32 + bi) * 8192u; }
        else { b.slot = ((myi * 2) * 16 + kk) * 32 + bi; b.dso = OFF_AIMG + (unsigned)(g * 32 + bi) * 8192u; } }
    else if (n < 1536) { const int kk = (n - 1024) >> 5, bi = n & 31, g = src * 16 + kk; b.cnt = OFF_CNT + (128 + g) * 128;
        b.slot = ((myi * 2 + 1) * 16 + kk) * 32 + bi; b.dso = OFF_AIMG + (unsigned)((128 + g) * 32 + bi) * 8192u; }
    else { const int o = n - 1536, w = o & 7, h = (o >> 3) & 1, tile = (o >> 4) & 63, r = o >> 10; b.cnt = 0; b.is_out = true;
        b.slot = 4096u + o;
        const unsigned row0 = (src >> 2) * 4096 + r * 2048 + (tile >> 3) * 256 + h * 128 + (w >> 2) * 64, col0 = (src & 3) * 2048 + (tile & 7) * 256 + (w & 3) * 64;
        b.dso = (row0 * DIM + col0) * 2u; }
    return b;
}
constexpr unsigned NPAIR = NSEQ / 2, NSLOT = 7, CTL = NSLOT * 16384;
constexpr unsigned M_DEAD = 1u << 30, N_STOP = 0xFFFFFFFFu;
__device__ __forceinline__ unsigned lds_ld(LAS unsigned char* p) { return *(volatile LAS unsigned*)p; }
__device__ __forceinline__ void lds_st(LAS unsigned char* p, unsigned v) { *(volatile LAS unsigned*)p = v; }
__device__ __forceinline__ void ring_publish(LAS unsigned char* lds, unsigned i, unsigned n, unsigned ha, unsigned hb, int lane) {
    const unsigned sl = i % NSLOT;
    if (lane == 0) { *(volatile LAS u32x4*)(lds + CTL + sl * 16) = (u32x4){n, ha, hb, 0u}; lds_st(lds + CTL + 128 + sl * 4, i + 1u); }
}
__device__ __forceinline__ bool slot_busy(LAS unsigned char* lds, unsigned i) { return i >= NSLOT && (int)(lds_ld(lds + CTL + 160 + (i % NSLOT) * 4) - 2u * (i / NSLOT)) < 0; }
__device__ __forceinline__ void dma_op(rsrc_t r, LAS unsigned char* dst, unsigned j, unsigned so, int lane) {
    __builtin_amdgcn_raw_ptr_buffer_load_lds(r, (LAS void*)(dst + j * 1024), 16, lane * 16, so + j * 1024, 0, AUX_SYS);
}
__device__ __forceinline__ u32x4 load_pair(rsrc_t wsr, unsigned off) { return __builtin_amdgcn_raw_buffer_load_b128(wsr, 0, off, AUX_SYS); }
#define VM_WAIT_(n) asm volatile("s_waitcnt vmcnt(" #n ")" ::: "memory")
#define VM_WAIT(n) VM_WAIT_(n)
#define G_STR_(x) #x
#define G_STR(x) G_STR_(x)
#define WIN 5

__device__ __forceinline__ void loader_wave(const Params& P, LAS unsigned char* lds, unsigned call, int src, int lane) {
    char* ws = P.ws[P.rank];
    unsigned* tmo = (unsigned*)(ws + OFF_TMO);
    const rsrc_t wsr = mk_rsrc(ws, WS_USED);
    char* const remp = P.ws[src];
    const int myi = P.rank >> 2, myj = P.rank & 3;
    const unsigned gbase = OFF_GRAN + (unsigned)src * NSEQ * 8u;
    char* const gptr = ws + gbase;
    const unsigned tko = OFF_LINK + (unsigned)src * 128u, tbase = (call - 1u) * (NPAIR + 3u * PWL);
    __builtin_amdgcn_s_setprio(3);
    unsigned q, q1, q2;
    {
        unsigned t0 = 0;
        if (lane == 0) t0 = __hip_atomic_fetch_add((unsigned*)(ws + tko), 3u, __ATOMIC_RELAXED, __HIP_MEMORY_SCOPE_AGENT);
        q = rfl(t0) - tbase; q1 = q + 1; q2 = q + 2;
    }
    {
        const rsrc_t rw = mk_rsrc(remp, OFF_AIMG);
        const unsigned v = __builtin_amdgcn_raw_buffer_load_b32(rw, lane < 25 ? OFF_SLOT + (unsigned)lane * (2u << 20) + 8188u - (lane == 24 ? 8192u : 0u) : 0xFFFFFFF0u, 0, AUX_SYS);
        asm volatile("" :: "v"(v));
    }
    unsigned gca, gha, gcb, ghb, hca, hha, hcb, hhb;
    { const u32x4 a = load_pair(wsr, gbase + (q < NPAIR ? q : 0u) * 16u), b = load_pair(wsr, gbase + (q1 < NPAIR ? q1 : 0u) * 16u);
      gca = rfl(a.x); gha = rfl(a.y); gcb = rfl(a.z); ghb = rfl(a.w); hca = rfl(b.x); hha = rfl(b.y); hcb = rfl(b.z); hhb = rfl(b.w); }
    const int zero = 0, one = 1;
    unsigned i = 0;
    bool pend = false; unsigned pn = 0, pa = 0, pb = 0;
    while (q < NPAIR) {
        bool dead = false;
        if ((int)(gca - call) < 0 || (int)(gcb - call) < 0 || slot_busy(lds, i)) {
            if (pend) { VM_WAIT(0); ring_publish(lds, i - 1u, pn, pa, pb, lane); pend = false; }
            const unsigned long long t0 = __builtin_amdgcn_s_memrealtime();
            while ((int)(gca - call) < 0 || (int)(gcb - call) < 0) {
                const u32x4 a = load_pair(wsr, gbase + q * 16u);
                gca = rfl(a.x); gha = rfl(a.y); gcb = rfl(a.z); ghb = rfl(a.w);
                if ((int)(gca - call) >= 0 && (int)(gcb - call) >= 0) break;
                if (give_up(t0, tmo, call)) { dead = true; break; }
                __builtin_amdgcn_s_sleep(2);
                asm volatile("" ::: "memory");
            }
            while (slot_busy(lds, i)) __builtin_amdgcn_s_sleep(1);
        }
        const unsigned n = 2u * q;
        const unsigned so = OFF_SLOT + locate(n, src, myi, myj).slot * 8192u;
        const unsigned bytes = dead ? 0u : payload_bytes(gha) + payload_bytes(ghb), nops = (bytes + 1023u) >> 10;
        const rsrc_t rem = mk_rsrc(remp, so + bytes);
        LAS unsigned char* dst = lds + (i % NSLOT) * 16384;
        u32x4 g2; unsigned tk = 0;
        asm volatile("global_load_dwordx4 %0, %1, %2 sc0 sc1" : "=v"(g2) : "v"(zero), "s"(gptr + (q2 < NPAIR ? q2 : 0u) * 16u) : "memory");
        if (lane == 0) asm volatile("global_atomic_add %0, %1, %2, %3 sc0" : "=v"(tk) : "v"(zero), "v"(one), "s"(ws + tko) : "memory");
#define G_FILL(jj) do { if ((jj) == WIN) asm volatile("s_waitcnt vmcnt(" G_STR(WIN) ")" : "+v"(g2), "+v"(tk) :: "memory"); else VM_WAIT(WIN); \
                        if ((jj) == WIN - 2 && pend) ring_publish(lds, i - 1u, pn, pa, pb, lane); dma_op(rem, dst, jj, so, lane); } while (0)
        G_FILL(0); G_FILL(1); G_FILL(2); G_FILL(3); G_FILL(4); G_FILL(5); G_FILL(6); G_FILL(7);
#undef G_FILL
        for (unsigned j = 8; j < nops; ++j) { VM_WAIT(WIN); dma_op(rem, dst, j, so, lane); }
        pend = true; pn = n; pa = gha | (dead ? M_DEAD : 0u); pb = ghb | (dead ? M_DEAD : 0u);
        q = q1; gca = hca; gha = hha; gcb = hcb; ghb = hhb;
        q1 = q2; hca = rfl(g2.x); hha = rfl(g2.y); hcb = rfl(g2.z); hhb = rfl(g2.w);
        q2 = rfl(tk) - tbase;
        ++i;
    }
    VM_WAIT(0);
    if (pend) ring_publish(lds, i - 1u, pn, pa, pb, lane);
    for (int d = 0; d < (NDEC + 1) / 2 + 1; ++d, ++i) {
        while (slot_busy(lds, i)) __builtin_amdgcn_s_sleep(1);
        ring_publish(lds, i, N_STOP, 0u, 0u, lane);
    }
}

__device__ __forceinline__ void decoder_wave(const Params& P, LAS unsigned char* lds, unsigned call, int src, int d, int lane) {
    char* ws = P.ws[P.rank];
    unsigned* tmo = (unsigned*)(ws + OFF_TMO);
    const rsrc_t wsr = mk_rsrc(ws, WS_USED), outr = mk_rsrc(P.out, (unsigned)DIM * DIM * 2u);
    const int myi = P.rank >> 2, myj = P.rank & 3;
    for (unsigned i = d;; i += NDEC) {
        const unsigned pi = i >> 1, sl = pi % NSLOT;
        while (lds_ld(lds + CTL + 128 + sl * 4) != pi + 1u) __builtin_amdgcn_s_sleep(1);
        asm volatile("" ::: "memory");
        const u32x4 meta = *(volatile LAS u32x4*)(lds + CTL + sl * 16);
        const unsigned n0 = rfl(meta.x), ha = rfl(meta.y), h = (i & 1) ? rfl(meta.z) : ha;
        if (n0 == N_STOP) break;
        const Blk b = locate(n0 + (i & 1), src, myi, myj);
        const bool raw = (h >> 31) != 0;
        bool dead = (h & M_DEAD) != 0;
        if (b.is_out && rfl(__hip_atomic_load(tmo, __ATOMIC_RELAXED, __HIP_MEMORY_SCOPE_AGENT)) == call) dead = true;
        u32x4 c[8];
        unsigned recs = 0;
        const unsigned ne = (dead || raw) ? 0u : (h >> 8) & 63u;
        LAS unsigned char* cur = lds + sl * 16384 + ((i & 1) ? payload_bytes(ha) : 0u);
        if (dead) {
#pragma unroll
            for (int j = 0; j < 8; ++j) c[j] = (u32x4){0x7FC07FC0u, 0x7FC07FC0u, 0x7FC07FC0u, 0x7FC07FC0u};
        } else if (raw) {
#pragma unroll
            for (int j = 0; j < 8; ++j) c[j] = *(LAS u32x4*)(cur + j * 1024 + lane * 16);
        } else {
            recs = *(LAS unsigned*)(cur + RAWB + (((h >> 16) & 0x3FFu) + (lane & 31)) * 4u);
            decode_block(cur, lds + CTL + 256 + d * 256, c, lane);
        }
#pragma unroll
        for (int j = 0; j < 8; ++j) asm volatile("" : "+v"(c[j]));
        asm volatile("" : "+v"(recs));
        if (lane == 0) __hip_atomic_fetch_add((LAS unsigned*)(lds + CTL + 160 + sl * 4), 1u, __ATOMIC_RELAXED, __HIP_MEMORY_SCOPE_WORKGROUP);
        const unsigned sj = b.is_out ? 8u * DIM * 2u : 1024u, sr = b.is_out ? DIM * 2u : 128u;
        const unsigned vo = (unsigned)(lane >> 3) * sr + (unsigned)(lane & 7) * 16u;
        if (b.is_out) {
#pragma unroll
            for (int j = 0; j < 8; ++j) __builtin_amdgcn_raw_buffer_store_b128(c[j], outr, vo, b.dso + j * sj, OUT_AUX);
            for (unsigned e = 0; e < ne; ++e) {
                const unsigned rec = __builtin_amdgcn_readlane(recs, e), pos = rec & 0xFFFu, v = pos & 63u;
                if ((unsigned)lane == (pos >> 6)) __builtin_amdgcn_raw_buffer_store_b16((unsigned short)((rec >> 12) & 0xFFFFu), outr, vo + (v & 7u) * 2u, b.dso + (v >> 3) * sj, OUT_AUX);
            }
        } else {
#pragma unroll
            for (int j = 0; j < 8; ++j) __builtin_amdgcn_raw_buffer_store_b128(c[j], wsr, vo, b.dso + j * sj, IMG_AUX);
            for (unsigned e = 0; e < ne; ++e) {
                const unsigned rec = __builtin_amdgcn_readlane(recs, e), pos = rec & 0xFFFu, v = pos & 63u;
                if ((unsigned)lane == (pos >> 6)) __builtin_amdgcn_raw_buffer_store_b16((unsigned short)((rec >> 12) & 0xFFFFu), wsr, vo + (v & 7u) * 2u, b.dso + (v >> 3) * sj, IMG_AUX);
            }
            WAIT_VM0(); signal_cnt(ws, b.cnt, lane);
        }
    }
}

__device__ __forceinline__ void puller_role(const Params& P, LAS unsigned char* lds, unsigned call, int wg) {
    const int lane = threadIdx.x & 63, wid = rfl(threadIdx.x >> 6);
    const int link = wg % 7, src = rfl((P.rank + 1 + link) & 7);
    if (threadIdx.x < 16) *(volatile LAS unsigned*)(lds + CTL + 128 + threadIdx.x * 4) = 0u;
    __syncthreads();
    if (wid == 0) loader_wave(P, lds, call, src, lane);
    else if (wid <= NDEC) decoder_wave(P, lds, call, src, wid - 1, lane);
}

__device__ __forceinline__ float gelu_f(float y) {
    const float c1 = -2.0f * 0.7978845608028654f * 1.4426950408889634f, c2 = c1 * 0.044715f;
    const float e = __builtin_amdgcn_exp2f(y * (c1 + c2 * y * y));
    return y * __builtin_amdgcn_rcpf(1.0f + e);
}
__device__ __forceinline__ unsigned cvt_pk_bf16(float lo, float hi) { unsigned r; asm("v_cvt_pk_bf16_f32 %0, %1, %2" : "=v"(r) : "v"(lo), "v"(hi)); return r; }

__device__ __forceinline__ void gemm_role(const Params& P, LAS unsigned char* lds, unsigned call, int wg) {
    const int tid = threadIdx.x, wid = rfl(tid >> 6), lane = tid & 63, wr = wid >> 2, wc = wid & 3, fr = lane & 15, fq = lane >> 4;
    char* ws = P.ws[P.rank];
    unsigned* tmo = (unsigned*)(ws + OFF_TMO);
    const rsrc_t wsr = mk_rsrc(ws, WS_USED);
    const int xcd = wg & 7, q = (wg - NPACK) >> 3, h = xcd & 1, blk = xcd >> 1;
    const int tm = (blk >> 1) * 4 + (q >> 2), tn = (blk & 1) * 4 + (q & 3), tile = tm * 8 + tn;
    const unsigned voff = (unsigned)tid * 16u, ldsw = (unsigned)wid * 1024u;
    const int aoff0 = (wr * 64 + fr) * 128 + ((fq ^ (fr & 7)) << 4), aoff1 = aoff0 ^ 64;
    const int boff0 = (wc * 32 + fr) * 128 + ((fq ^ (fr & 7)) << 4), boff1 = boff0 ^ 64;
#define G_SA(b, hh) (((b) * 2 + (hh)) * 16384)
#define G_SB(b, hh) ((4 + (b) * 2 + (hh)) * 16384)
#define G_STAGE(bufoff, soff) do { _Pragma("unroll") for (int _i = 0; _i < 2; ++_i) \
        __builtin_amdgcn_raw_ptr_buffer_load_lds(wsr, (LAS void*)(lds + (bufoff) + ldsw + _i * 8192), 16, voff, (soff) + _i * 8192, 0, AUX_SYS); } while (0)
#define G_LDA(dst, b, hh) do { _Pragma("unroll") for (int m = 0; m < 4; ++m) { \
        dst[m][0] = *(const LAS bf16x8*)(lds + G_SA(b, hh) + aoff0 + m * 2048); dst[m][1] = *(const LAS bf16x8*)(lds + G_SA(b, hh) + aoff1 + m * 2048); } } while (0)
#define G_LDB(dst, b, hh) do { _Pragma("unroll") for (int n = 0; n < 2; ++n) { \
        dst[n][0] = *(const LAS bf16x8*)(lds + G_SB(b, hh) + boff0 + n * 2048); dst[n][1] = *(const LAS bf16x8*)(lds + G_SB(b, hh) + boff1 + n * 2048); } } while (0)
#define G_MMA(ai, bj, At, Bt) do { __builtin_amdgcn_s_setprio(1); _Pragma("unroll") for (int m = 0; m < 4; ++m) _Pragma("unroll") for (int n = 0; n < 2; ++n) _Pragma("unroll") for (int k = 0; k < 2; ++k) \
        acc[ai][bj][m][n] = __builtin_amdgcn_mfma_f32_16x16x32_bf16(Bt[n][k], At[m][k], acc[ai][bj][m][n], 0, 0, 0); __builtin_amdgcn_s_setprio(0); } while (0)
#define G_WAIT_V(n) asm volatile("s_waitcnt vmcnt(" #n ")" ::: "memory")
#define G_WAIT_L(n) asm volatile("s_waitcnt lgkmcnt(" #n ")" ::: "memory")
#define G_BAR __builtin_amdgcn_s_barrier()
#define G_SCHED __builtin_amdgcn_sched_barrier(0)
#define G_GIDX(t) ((unsigned)((4 * h + ((t) & 3)) * 16 + ((t) >> 2)))
    const int zero = 0;
    for (int r = 0; r < 2; ++r) {
        const unsigned target = call * (r == 0 ? 64u : 32u);
        const unsigned abase = OFF_AIMG + (unsigned)r * (128u * 262144u) + (unsigned)tm * 32768u, bbase = OFF_BIMG + (unsigned)tn * 32768u;
        char* cbase = ws + OFF_CNT + r * 128 * 128;
        f32x4 acc[2][2][4][2];
#pragma unroll
        for (int a = 0; a < 2; ++a)
#pragma unroll
            for (int b = 0; b < 2; ++b)
#pragma unroll
                for (int m = 0; m < 4; ++m)
#pragma unroll
                    for (int n = 0; n < 2; ++n) acc[a][b][m][n] = (f32x4){0.f, 0.f, 0.f, 0.f};
        bf16x8 At[4][2], B0[2][2], B1[2][2];
        STAMP(ws, r * 8 + 0);
        wait_word((unsigned*)(cbase + G_GIDX(0) * 128), target, tmo, call);
        wait_word((unsigned*)(cbase + G_GIDX(1) * 128), target, tmo, call);
        wait_word((unsigned*)(cbase + G_GIDX(2) * 128), target, tmo, call);
        STAMP(ws, r * 8 + 1);
        {
            const unsigned a0 = abase + G_GIDX(0) * 262144u, b0 = bbase + G_GIDX(0) * 262144u, a1 = abase + G_GIDX(1) * 262144u, b1 = bbase + G_GIDX(1) * 262144u;
            G_STAGE(G_SB(0, 0), b0); G_STAGE(G_SA(0, 0), a0); G_STAGE(G_SB(0, 1), b0 + 16384); G_STAGE(G_SA(0, 1), a0 + 16384);
            if (wr == 1) G_BAR;
            G_WAIT_V(4); G_BAR;
            G_STAGE(G_SB(1, 0), b1); G_STAGE(G_SA(1, 0), a1); G_STAGE(G_SB(1, 1), b1 + 16384);
            G_WAIT_V(6); G_BAR;
        }
        unsigned fA = target, fB;
        for (int t = 0; t < 64; t += 2) {
            const int t2 = t + 2 < 64 ? t + 2 : 63, t3 = t + 3 < 64 ? t + 3 : 63, t4 = t + 4 < 64 ? t + 4 : 63;
            const unsigned g1 = G_GIDX(t + 1), g2 = G_GIDX(t2), g3 = G_GIDX(t3), g4 = G_GIDX(t4);
            const unsigned a1 = abase + g1 * 262144u, a2 = abase + g2 * 262144u, b2 = bbase + g2 * 262144u, a3 = abase + g3 * 262144u, b3 = bbase + g3 * 262144u;
            asm volatile("global_load_dword %0, %1, %2 sc0 sc1" : "=v"(fB) : "v"(zero), "s"(cbase + g3 * 128));
            G_LDB(B0, 0, 0); G_SCHED; G_LDA(At, 0, 0); G_STAGE(G_SA(1, 1), a1 + 16384);
            G_WAIT_L(8); G_BAR; G_WAIT_L(0); G_MMA(0, 0, At, B0); G_BAR; G_SCHED;
            asm volatile("" : "+v"(fA));
            if ((int)(rfl(fA) - target) < 0) wait_word((unsigned*)(cbase + g2 * 128), target, tmo, call);
            G_LDB(B1, 0, 1); G_STAGE(G_SB(0, 0), b2);
            G_BAR; G_WAIT_L(0); G_MMA(0, 1, At, B1); G_BAR;
            G_LDA(At, 0, 1); G_STAGE(G_SA(0, 0), a2);
            G_BAR; G_WAIT_L(0); G_MMA(1, 0, At, B0); G_BAR; G_SCHED;
            G_STAGE(G_SB(0, 1), b2 + 16384);
            G_WAIT_V(6); G_BAR; G_MMA(1, 1, At, B1); G_BAR;
            asm volatile("global_load_dword %0, %1, %2 sc0 sc1" : "=v"(fA) : "v"(zero), "s"(cbase + g4 * 128));
            G_LDB(B0, 1, 0); G_SCHED; G_LDA(At, 1, 0); G_STAGE(G_SA(0, 1), a2 + 16384);
            G_WAIT_L(8); G_BAR; G_WAIT_L(0); G_MMA(0, 0, At, B0); G_BAR; G_SCHED;
            asm volatile("" : "+v"(fB));
            if ((int)(rfl(fB) - target) < 0) wait_word((unsigned*)(cbase + g3 * 128), target, tmo, call);
            G_LDB(B1, 1, 1); G_STAGE(G_SB(1, 0), b3);
            G_BAR; G_WAIT_L(0); G_MMA(0, 1, At, B1); G_BAR;
            G_LDA(At, 1, 1); G_STAGE(G_SA(1, 0), a3);
            G_BAR; G_WAIT_L(0); G_MMA(1, 0, At, B0); G_BAR; G_SCHED;
            G_STAGE(G_SB(1, 1), b3 + 16384);
            G_WAIT_V(6); G_BAR; G_MMA(1, 1, At, B1); G_BAR;
        }
        G_WAIT_V(0);
        asm volatile("" : "+v"(fA));
        if (wr == 0) G_BAR;
        G_BAR;
        STAMP(ws, r * 8 + 2);
        int lane = threadIdx.x & 63;
        asm volatile("" : "+v"(lane));
        const int fr = lane & 15, fq = lane >> 4;
        if (h) {
#pragma unroll
            for (int b = 0; b < 2; ++b)
#pragma unroll
                for (int m = 0; m < 4; ++m)
#pragma unroll
                    for (int n = 0; n < 2; ++n) { const f32x4 tq = acc[0][b][m][n]; acc[0][b][m][n] = acc[1][b][m][n]; acc[1][b][m][n] = tq; }
        }
        const unsigned xl = (unsigned)((r * 64 + tile) * 2);
#pragma unroll
        for (int b = 0; b < 2; ++b)
#pragma unroll
            for (int m = 0; m < 4; ++m)
#pragma unroll
                for (int n = 0; n < 2; ++n)
                    __builtin_amdgcn_raw_buffer_store_b128(__builtin_bit_cast(u32x4, acc[1][b][m][n]), wsr, lane * 16 + ((b * 4 + m) * 2 + n) * 1024, OFF_XBUF + ((xl + h) * 8 + wid) * 16384u, AUX_SC1);
        WAIT_VM0();
        if (lane == 0) __hip_atomic_store((unsigned*)(ws + OFF_XFLAG + (xl + h) * 128 + wid * 16), call, __ATOMIC_RELAXED, __HIP_MEMORY_SCOPE_AGENT);
        wait_word((unsigned*)(ws + OFF_XFLAG + (xl + (h ^ 1)) * 128 + wid * 16), call, tmo, call);
        const bool dead = rfl(__hip_atomic_load(tmo, __ATOMIC_RELAXED, __HIP_MEMORY_SCOPE_AGENT)) == call;
        STAMP(ws, r * 8 + 3);
        LAS unsigned char* my = lds + wid * 8192;
#pragma unroll
        for (int b = 0; b < 2; ++b)
#pragma unroll
            for (int m = 0; m < 4; ++m) {
                const f32x4 p0 = __builtin_bit_cast(f32x4, __builtin_amdgcn_raw_buffer_load_b128(wsr, lane * 16 + ((b * 4 + m) * 2 + 0) * 1024, OFF_XBUF + ((xl + (h ^ 1)) * 8 + wid) * 16384u, AUX_SC1));
                const f32x4 p1 = __builtin_bit_cast(f32x4, __builtin_amdgcn_raw_buffer_load_b128(wsr, lane * 16 + ((b * 4 + m) * 2 + 1) * 1024, OFF_XBUF + ((xl + (h ^ 1)) * 8 + wid) * 16384u, AUX_SC1));
                const f32x4 v0 = acc[0][b][m][0] + p0, v1 = acc[0][b][m][1] + p1;
                u32x4 o;
                o.x = cvt_pk_bf16(gelu_f(v0[0]), gelu_f(v0[1])); o.y = cvt_pk_bf16(gelu_f(v0[2]), gelu_f(v0[3]));
                o.z = cvt_pk_bf16(gelu_f(v1[0]), gelu_f(v1[1])); o.w = cvt_pk_bf16(gelu_f(v1[2]), gelu_f(v1[3]));
                const int row = m * 16 + fr, ch = b * 4 + fq;
                *(LAS u32x4*)(my + row * 128 + ((ch ^ (row & 7)) * 16)) = o;
            }
        asm volatile("s_waitcnt lgkmcnt(0)" ::: "memory");
        u32x4 c[8];
#pragma unroll
        for (int j = 0; j < 8; ++j) { const int row = j * 8 + (lane >> 3); c[j] = *(LAS u32x4*)(my + row * 128 + (((lane & 7) ^ (row & 7)) * 16)); }
        if (dead) {
#pragma unroll
            for (int j = 0; j < 8; ++j) c[j] = (u32x4){0x7FC07FC0u, 0x7FC07FC0u, 0x7FC07FC0u, 0x7FC07FC0u};
        }
        {
            const rsrc_t outr = mk_rsrc(P.out, (unsigned)DIM * DIM * 2u);
            const unsigned row0 = (P.rank >> 2) * 4096 + r * 2048 + tm * 256 + h * 128 + wr * 64, col0 = (P.rank & 3) * 2048 + tn * 256 + wc * 64;
            const unsigned dso = (row0 * DIM + col0) * 2u, vo = (unsigned)(lane >> 3) * (DIM * 2u) + (unsigned)(lane & 7) * 16u;
            if (r == 0) {
#pragma unroll
                for (int j = 0; j < 8; ++j) __builtin_amdgcn_raw_buffer_store_b128(c[j], outr, vo, dso + j * (8u * DIM * 2u), AUX_SC1);
            } else {
#pragma unroll
                for (int j = 0; j < 8; ++j) __builtin_amdgcn_raw_buffer_store_b128(c[j], outr, vo, dso + j * (8u * DIM * 2u), OUT_AUX);
            }
            const unsigned o = (xl + h) * 8 + wid;
            if (r == 0) { WAIT_VM0(); signal_cnt(ws, OFF_OCNT + (o >> 3) * 128, lane); }
            else publish_block(P, wsr, c, 4096u + o, 1536u + o, 0xFFu & ~(1u << P.rank), call, lds, my, 1u + GROUNDS, 8u, wid, lane);
        }
        STAMP(ws, r * 8 + 4);
        asm volatile("s_waitcnt lgkmcnt(0)" ::: "memory");
        G_BAR;
    }
#undef G_SA
#undef G_SB
#undef G_STAGE
#undef G_LDA
#undef G_LDB
#undef G_MMA
#undef G_WAIT_V
#undef G_WAIT_L
#undef G_BAR
#undef G_SCHED
#undef G_GIDX
}

__global__ __launch_bounds__(512, 2) void gemm_allgather_kernel(Params P) {
    extern __shared__ __attribute__((aligned(16))) unsigned char shm[];
    LAS unsigned char* lds = (LAS unsigned char*)shm;
    char* ws = P.ws[P.rank];
    const int b = rfl(blockIdx.x);
    STAMP(ws, 60);
    unsigned call;
    if (b >= NPACK + NGEMM) { call = call_draw(ws); puller_role(P, lds, rfl(call), rfl(b - NPACK - NGEMM)); }
    else {
        if (threadIdx.x < 40) *(volatile LAS unsigned*)(lds + GCTL + 128 + threadIdx.x * 4) = 0u;
        __syncthreads();
        const int wid = rfl(threadIdx.x >> 6);
        const bool gemm = b >= NPACK;
        const int next = gemm ? NFIRST + (b - NPACK) * 8 + wid : NFIRST + GROUNDS * NGEMM * 8 + b * 8 + wid;
        call = packer_role(P, lds, wid < 4 ? b * 4 + wid : next, next, gemm ? NGEMM * 8 : NPACK * 8, gemm ? NFIRST + GROUNDS * NGEMM * 8 : 4096 + 1024);
        if (gemm) { __syncthreads(); gemm_role(P, lds, rfl(call), rfl(b)); }
    }
    STAMP(ws, 61);
    STAMP(ws, 58 + (call & 1));
}

}

extern "C" void kernel_launch(int rank, int nranks, ncclComm_t comm,
                              hipStream_t stream,
                              void* const* d_in, const int* in_sizes, int n_in,
                              void* d_out, int out_size,
                              const int* in_dtypes, int out_dtype,
                              void* const* d_out_all,
                              void* const* d_ws_all, size_t ws_bytes) {
    (void)comm; (void)in_sizes; (void)in_dtypes; (void)out_dtype; (void)d_out_all;
    if (nranks != gar::WORLD || n_in < 2 || ws_bytes < gar::WS_NEED || out_size != gar::DIM * gar::DIM || (unsigned)rank >= (unsigned)gar::WORLD) {
        fprintf(stderr, "unexpected problem shape\n");
        return;
    }
    static bool attr = false;
    if (!attr) {
        hipError_t e = hipFuncSetAttribute((const void*)gar::gemm_allgather_kernel, hipFuncAttributeMaxDynamicSharedMemorySize, gar::LDS_BYTES);
        if (e != hipSuccess) { fprintf(stderr, "hipFuncSetAttribute: %s\n", hipGetErrorString(e)); return; }
        attr = true;
    }
    gar::Params P;
    memset(&P, 0, sizeof P);
    P.x = d_in[0]; P.w = d_in[1]; P.out = d_out; P.rank = rank;
    for (int p = 0; p < gar::WORLD; ++p) P.ws[p] = (char*)d_ws_all[p];
    hipLaunchKernelGGL(gar::gemm_allgather_kernel, dim3(gar::GRID), dim3(512), gar::LDS_BYTES, stream, P);
    hipError_t e = hipGetLastError();
    if (e != hipSuccess) fprintf(stderr, "launch: %s\n", hipGetErrorString(e));
}
```

```cpp
#include <hip/hip_runtime.h>
#include <rccl/rccl.h>
#include <cstdio>
#include <cstdint>
#include <cstring>

namespace gar {

#define LAS __attribute__((address_space(3)))
typedef unsigned u32x4 __attribute__((ext_vector_type(4)));
typedef unsigned short u16x2 __attribute__((ext_vector_type(2)));
typedef short bf16x8 __attribute__((ext_vector_type(8)));
typedef float f32x4 __attribute__((ext_vector_type(4)));
typedef __amdgpu_buffer_rsrc_t rsrc_t;

constexpr int WORLD = 8;
constexpr int DIM = 8192;
constexpr int KSH = 1024;
constexpr int NGEMM = 128;
constexpr int GRID = 256;
#define PULL_WG_PER_LINK 12
#define PULL_DECODERS 7
constexpr int PWL = PULL_WG_PER_LINK;
constexpr int NDEC = PULL_DECODERS;
constexpr int NPULL = 7 * PWL;
constexpr int NPACK = GRID - NGEMM - NPULL;
#define NRAW 48
#define NRAWEND 40
#define WARM_TLB 1
#define JIT_LEAD 30
#define OUT_AUX 18
#define IMG_AUX 16
#define PUSH0 1
constexpr int NSKIP = PUSH0 ? 32 : 0;
constexpr int NFIRST = PUSH0 ? 64 + (NPACK + NGEMM - 64) * 4 : (NPACK + NGEMM) * 4;
#define GEMM_PACK_ROUNDS 1
constexpr int GROUNDS = GEMM_PACK_ROUNDS;
constexpr int OUT2 = 4096 + NPACK * 4;
static_assert(NFIRST + GROUNDS * NGEMM * 8 <= 4096 && NFIRST % 8 == 0 && OUT2 % 8 == 0, "roles");
constexpr int NSEQ = 3584;
constexpr unsigned long long DEADLINE = 40000000ull;

constexpr unsigned OFF_TMO = 128;
constexpr unsigned OFF_LINK = 1024;
constexpr unsigned OFF_CNT = 4096;
constexpr unsigned OFF_XFLAG = OFF_CNT + 256 * 128;
constexpr unsigned OFF_OCNT = OFF_XFLAG + 256 * 128;
constexpr unsigned OFF_GRAN = 131072;
constexpr unsigned OFF_CALL = 640u << 10;
constexpr unsigned OFF_SLOT = 1u << 20;
constexpr unsigned OFF_AIMG = 64u << 20;
constexpr unsigned OFF_BIMG = 128u << 20;
constexpr unsigned OFF_XBUF = 160u << 20;
constexpr unsigned WS_USED = 192u << 20;
constexpr unsigned WS_NEED = WS_USED;
static_assert(OFF_OCNT + 128 * 128 <= OFF_GRAN, "map");
static_assert(OFF_GRAN + WORLD * NSEQ * 8 <= OFF_CALL && OFF_CALL + 2048 * 64 <= OFF_SLOT, "map");
static_assert(OFF_SLOT + 6144u * 8192u <= OFF_AIMG, "map");
constexpr int LDS_BYTES = 131072 + 512;

struct Params {
    const void* x; const void* w; void* out;
    char* ws[WORLD];
    int rank; int pad;
};

constexpr int AUX_SC1 = 16, AUX_SYS = 17;

__device__ __forceinline__ rsrc_t mk_rsrc(const void* p, unsigned bytes) {
    return __builtin_amdgcn_make_buffer_rsrc(const_cast<void*>(p), 0, bytes, 0x00020000);
}
__device__ __forceinline__ unsigned rfl(unsigned v) { return __builtin_amdgcn_readfirstlane(v); }
__device__ __forceinline__ unsigned bfi(unsigned mask, unsigned a, unsigned b) { return (a & mask) | (b & ~mask); }
__device__ __forceinline__ unsigned perm(unsigned hi, unsigned lo, unsigned sel) { return __builtin_amdgcn_perm(hi, lo, sel); }
#define WAIT_VM0() asm volatile("s_waitcnt vmcnt(0)" ::: "memory")

#define STAMP(ws, idx) do { } while (0)

__device__ __forceinline__ unsigned call_draw(char* ws) {
    unsigned t;
    const int zero = 0, one = 1;
    unsigned long long keep;
    asm volatile("s_mov_b64 %1, exec\n\ts_mov_b64 exec, 1\n\tglobal_atomic_add %0, %2, %3, %4 sc0\n\ts_mov_b64 exec, %1\n\ts_waitcnt vmcnt(0)"
                 : "=&v"(t), "=&s"(keep) : "v"(zero), "v"(one), "s"(ws + OFF_CALL + (blockIdx.x * 8 + rfl(threadIdx.x >> 6)) * 64) : "memory");
    return rfl(t) + 1u;
}

__device__ __forceinline__ bool give_up(unsigned long long t0, unsigned* tmo, unsigned call) {
    if (rfl(__hip_atomic_load(tmo, __ATOMIC_RELAXED, __HIP_MEMORY_SCOPE_AGENT)) == call) return true;
    if (__builtin_amdgcn_s_memrealtime() - t0 > DEADLINE) { __hip_atomic_store(tmo, call, __ATOMIC_RELAXED, __HIP_MEMORY_SCOPE_AGENT); return true; }
    return false;
}
__device__ __forceinline__ void wait_word(unsigned* p, unsigned target, unsigned* tmo, unsigned call) {
    if ((int)(rfl(__hip_atomic_load(p, __ATOMIC_RELAXED, __HIP_MEMORY_SCOPE_SYSTEM)) - target) >= 0) return;
    const unsigned long long t0 = __builtin_amdgcn_s_memrealtime();
    while ((int)(rfl(__hip_atomic_load(p, __ATOMIC_RELAXED, __HIP_MEMORY_SCOPE_SYSTEM)) - target) < 0) {
        if (give_up(t0, tmo, call)) break;
        __builtin_amdgcn_s_sleep(4);
    }
}

constexpr unsigned MAX_EXC = 31;
constexpr unsigned RAWB = 3584, AUX_U = 20, AUX_MAX = 2560, U_MAX = 10240, LANE_U_MAX = 224;
constexpr unsigned ENC_HIST = AUX_MAX + 16, ENC_SORT = ENC_HIST + 256, ENC_END = ENC_SORT + 128;

__device__ __forceinline__ unsigned bfi_c(unsigned mask, unsigned a, unsigned b) { unsigned r; asm("v_bfi_b32 %0, %1, %2, %3" : "=v"(r) : "s"(mask), "v"(a), "v"(b)); return r; }
__device__ __forceinline__ unsigned opaque(unsigned v) { asm("" : "+v"(v)); return v; }
__device__ __forceinline__ unsigned bytemask(unsigned x, int sh, unsigned c0c) {
    unsigned t; asm("v_and_or_b32 %0, %1, %2, %3" : "=v"(t) : "v"(x >> sh), "s"(0x01010101u), "v"(c0c));
    return perm(0u, 0u, t);
}
template <int CTRL, int RM = 0xf> __device__ __forceinline__ unsigned dpp0(unsigned v) { return (unsigned)__builtin_amdgcn_update_dpp(0, (int)v, CTRL, RM, 0xf, RM == 0xf); }
__device__ __forceinline__ unsigned wave_scan_incl(unsigned v) {
    v += dpp0<0x111>(v); v += dpp0<0x112>(v); v += dpp0<0x114>(v); v += dpp0<0x118>(v);
    v += dpp0<0x142, 0xa>(v); v += dpp0<0x143, 0xc>(v);
    return v;
}
__device__ __forceinline__ unsigned umin(unsigned a, unsigned b) { return a < b ? a : b; }
__device__ __forceinline__ unsigned umax(unsigned a, unsigned b) { return a > b ? a : b; }
__device__ __forceinline__ unsigned sad4(unsigned x) { return __builtin_amdgcn_sad_u8(x, 0u, 0u); }
__device__ __forceinline__ unsigned lut2x16(const unsigned (&T)[8], unsigned idx, unsigned ms, unsigned c0c) {
    const unsigned sel = idx & 0x07070707u, m8 = bytemask(idx, 3, c0c);
    const unsigned p = bfi(m8, perm(T[3], T[2], sel), perm(T[1], T[0], sel)), n = bfi(m8, perm(T[7], T[6], sel), perm(T[5], T[4], sel));
    return bfi(ms, n, p);
}
#define LDS_FENCE() asm volatile("" ::: "memory")
__device__ __forceinline__ void lds_or2(LAS unsigned char* scr, unsigned p, unsigned w) {
    const unsigned s = p & 31u;
    LAS unsigned* q = (LAS unsigned*)(scr + ((p >> 5) << 2));
    __hip_atomic_fetch_or(q, w << s, __ATOMIC_RELAXED, __HIP_MEMORY_SCOPE_WORKGROUP);
    __hip_atomic_fetch_or(q + 1, (w >> 1) >> (31u - s), __ATOMIC_RELAXED, __HIP_MEMORY_SCOPE_WORKGROUP);
}

__device__ __forceinline__ unsigned encode_block(const u32x4 (&c)[8], unsigned (&P)[14], u32x4 (&a)[3], LAS unsigned char* scr, int lane) {
    asm volatile("" : "+v"(lane));
    const unsigned c0c = opaque(0x0C0C0C0Cu);
    unsigned E[16], R[16];
    u16x2 p1 = {0, 0}, p2 = {0, 0}, n1 = {0, 0}, n2 = {0, 0};
#pragma unroll
    for (int g = 0; g < 16; ++g) {
        const unsigned x = (g & 1) ? c[g >> 1].z : c[g >> 1].x, y = (g & 1) ? c[g >> 1].w : c[g >> 1].y;
        const unsigned H = perm(y, x, 0x07050301u), L = perm(y, x, 0x06040200u);
        E[g] = bfi_c(0xFEFEFEFEu, H << 1, L >> 7);
        R[g] = bfi_c(0x80808080u, L << 1, bfi_c(0x40404040u, H >> 1, L));
        const unsigned en = E[g] & bytemask(H, 7, c0c), ep = E[g] ^ en;
        p1 = __builtin_elementwise_max(p1, __builtin_bit_cast(u16x2, ep)); p2 = __builtin_elementwise_max(p2, __builtin_bit_cast(u16x2, ep << 8));
        n1 = __builtin_elementwise_max(n1, __builtin_bit_cast(u16x2, en)); n2 = __builtin_elementwise_max(n2, __builtin_bit_cast(u16x2, en << 8));
        __builtin_amdgcn_sched_barrier(0);
    }
    const u32x4 z4 = {0u, 0u, 0u, 0u};
    *(LAS u32x4*)(scr + lane * 16) = z4; *(LAS u32x4*)(scr + 1024 + lane * 16) = z4;
    if (lane < 49) *(LAS u32x4*)(scr + 2048 + lane * 16) = z4;
    unsigned baseP, baseN;
    {
        const u16x2 mp = __builtin_elementwise_max(p1, p2), mn = __builtin_elementwise_max(n1, n2);
        u16x2 m = {(unsigned short)((mp.x > mp.y ? mp.x : mp.y) >> 8), (unsigned short)((mn.x > mn.y ? mn.x : mn.y) >> 8)};
        m = __builtin_elementwise_max(m, __builtin_bit_cast(u16x2, dpp0<0x128>(__builtin_bit_cast(unsigned, m)))); m = __builtin_elementwise_max(m, __builtin_bit_cast(u16x2, dpp0<0x124>(__builtin_bit_cast(unsigned, m))));
        m = __builtin_elementwise_max(m, __builtin_bit_cast(u16x2, dpp0<0x122>(__builtin_bit_cast(unsigned, m)))); m = __builtin_elementwise_max(m, __builtin_bit_cast(u16x2, dpp0<0x121>(__builtin_bit_cast(unsigned, m))));
        const unsigned mu = __builtin_bit_cast(unsigned, m);
        const u16x2 r01 = __builtin_elementwise_max(__builtin_bit_cast(u16x2, (unsigned)__builtin_amdgcn_readlane(mu, 0)), __builtin_bit_cast(u16x2, (unsigned)__builtin_amdgcn_readlane(mu, 16)));
        const u16x2 r23 = __builtin_elementwise_max(__builtin_bit_cast(u16x2, (unsigned)__builtin_amdgcn_readlane(mu, 32)), __builtin_bit_cast(u16x2, (unsigned)__builtin_amdgcn_readlane(mu, 48)));
        const unsigned mm = rfl(__builtin_bit_cast(unsigned, __builtin_elementwise_max(r01, r23)));
        baseP = mm & 0xFFu; baseN = mm >> 16;
    }
    const unsigned BP = baseP * 0x01010101u, BN = baseN * 0x01010101u;
    unsigned any = 0, D[16];
#pragma unroll
    for (int g = 0; g < 16; ++g) {
        E[g] = bfi(bytemask(R[g], 6, c0c), BN, BP) - E[g]; any |= E[g];
        D[g] = bfi_c(0x1E1E1E1Eu, E[g] << 1, (R[g] >> 7) & 0x01010101u);
        R[g] &= 0x7F7F7F7Fu;
    }
    LDS_FENCE();
#pragma unroll
    for (int gi = 0; gi < 4; ++gi) {
        const int g = gi * 5;
        const unsigned k4 = ((D[g] & 0x0F0F0F0Fu) | ((R[g] >> 2) & 0x10101010u) | ((D[g] << 1) & 0x20202020u)) << 2;
#pragma unroll
        for (int b = 0; b < 4; ++b)
            __hip_atomic_fetch_add((LAS unsigned*)(scr + ENC_HIST + ((k4 >> (8 * b)) & 0xFFu)), 1u, __ATOMIC_RELAXED, __HIP_MEMORY_SCOPE_WORKGROUP);
    }
    LDS_FENCE();
    const unsigned cnt = *(LAS unsigned*)(scr + ENC_HIST + lane * 4);
    const unsigned key = (cnt << 4) | (15u - (lane & 15));
    unsigned rank = 0;
    rank += dpp0<0x121>(key) > key; rank += dpp0<0x122>(key) > key; rank += dpp0<0x123>(key) > key; rank += dpp0<0x124>(key) > key; rank += dpp0<0x125>(key) > key;
    rank += dpp0<0x126>(key) > key; rank += dpp0<0x127>(key) > key; rank += dpp0<0x128>(key) > key; rank += dpp0<0x129>(key) > key; rank += dpp0<0x12A>(key) > key;
    rank += dpp0<0x12B>(key) > key; rank += dpp0<0x12C>(key) > key; rank += dpp0<0x12D>(key) > key; rank += dpp0<0x12E>(key) > key; rank += dpp0<0x12F>(key) > key;
    unsigned tail = cnt; tail += dpp0<0x128>(tail); tail += dpp0<0x124>(tail); tail += dpp0<0x122>(tail); tail += dpp0<0x121>(tail);
    const int sg = (lane >> 4) & 1;
    const unsigned ct = sg ? __builtin_amdgcn_readlane(tail, 48) : __builtin_amdgcn_readlane(tail, 32);
    if (lane < 32) *(LAS unsigned*)(scr + ENC_SORT + ((lane & 16) + rank) * 4) = cnt;
    LDS_FENCE();
    unsigned best;
    {
        const unsigned cb = lane & 15, s0 = cb / 3u, s1 = cb - 3u * s0, A = 1u << s0, AB = A + (1u << s1);
        unsigned cost = ct * (16u - AB);
#pragma unroll
        for (int q = 0; q < 4; ++q) {
            const u32x4 h = *(LAS u32x4*)(scr + ENC_SORT + sg * 64 + q * 16);
#pragma unroll
            for (int j = 0; j < 4; ++j) {
                const unsigned r = q * 4 + j, len = r < A ? 1u + s0 : r < AB ? 2u + s1 : 4u + ((r - AB) >> 1);
                cost += 2u * len * h[j];
            }
        }
        best = cb < 9 ? (cost << 4) | cb : 0xFFFFFFFFu;
        best = umin(best, dpp0<0x128>(best)); best = umin(best, dpp0<0x124>(best)); best = umin(best, dpp0<0x122>(best)); best = umin(best, dpp0<0x121>(best));
    }
    const unsigned cbP = __builtin_amdgcn_readlane(best, 0) & 15u, cbN = __builtin_amdgcn_readlane(best, 16) & 15u;
    const unsigned s0P = cbP / 3u, s1P = cbP - 3u * s0P, s0N = cbN / 3u, s1N = cbN - 3u * s0N;
    const unsigned ABP = (1u << s0P) + (1u << s1P), ABN = (1u << s0N) + (1u << s1N);
    unsigned F[8];
    {
        const unsigned s0 = sg ? s0N : s0P, s1 = sg ? s1N : s1P, A = 1u << s0, AB = sg ? ABN : ABP;
        const unsigned ent = rank < A ? (rank << 4) | (s0 << 6) : rank < AB ? 1u | ((rank - A) << 4) | (s1 << 6) : (2u + ((rank - AB) >> 1)) | (((rank - AB) & 1u) << 4) | (1u << 6);
        unsigned fv = ent << (8 * (lane & 3));
        fv |= dpp0<0xB1>(fv); fv |= dpp0<0x4E>(fv);
#pragma unroll
        for (int q = 0; q < 8; ++q) F[q] = __builtin_amdgcn_readlane(fv, q * 4);
        if (lane < 32)
            __hip_atomic_fetch_or((LAS unsigned*)(scr + (sg * 2 + (rank >> 3)) * 4), (unsigned)(lane & 15) << (8 * (rank & 3) + 4 * ((rank >> 2) & 1)), __ATOMIC_RELAXED, __HIP_MEMORY_SCOPE_WORKGROUP);
        if (lane == 0) *(LAS unsigned*)(scr + 16) = baseP | (baseN << 8) | (s0P << 16) | (s1P << 18) | (s0N << 20) | (s1N << 22);
    }
    const unsigned ABP4 = ABP * 0x01010101u, ABN4 = ABN * 0x01010101u, S0P4 = s0P * 0x01010101u, S0N4 = s0N * 0x01010101u;
    unsigned KK[16], XW[4], OX[4], TU = 0, TX = 0, bad = 0, xl = 0, xw = 0;
#pragma unroll
    for (int g = 0; g < 16; ++g) {
        const unsigned ms = bytemask(R[g], 6, c0c), m16 = bytemask(D[g], 4, c0c), ent = lut2x16(F, D[g], ms, c0c);
        const unsigned q = ((D[g] | 0x80808080u) - bfi(ms, ABN4, ABP4)) & 0x7F7F7F7Fu;
        unsigned K = bfi(m16, ((q >> 1) & 0x3F3F3F3Fu) + 0x02020202u, ent & 0x0F0F0F0Fu);
        unsigned SUF = bfi(m16, q & 0x01010101u, (ent >> 4) & 0x03030303u);
        unsigned S = bfi(m16, 0x01010101u, (ent >> 6) & 0x03030303u);
        unsigned gl = sad4(K) + 4u;
        if (__ballot(gl > 31u) != 0ull) {
            any |= 0x80u;
            const unsigned km = umax(umax(K & 0xFFu, (K >> 8) & 0xFFu), umax((K >> 16) & 0xFFu, K >> 24));
            const unsigned eq = gl > 31u ? (((K ^ (km * 0x01010101u)) + 0x7F7F7F7Fu) & 0x80808080u) ^ 0x80808080u : 0u, em = bytemask(eq, 7, c0c);
            E[g] |= eq;
            K &= ~em; SUF &= ~em; S = bfi(em, bfi(ms, S0N4, S0P4), S);
            gl = sad4(K) + 4u;
        }
        KK[g] = K; TU += gl; bad = umax(bad, gl);
        xw |= (SUF & 0xFFu) << xl; xl += S & 0xFFu;
        xw |= ((SUF >> 8) & 0xFFu) << xl; xl += (S >> 8) & 0xFFu;
        xw |= ((SUF >> 16) & 0xFFu) << xl; xl += (S >> 16) & 0xFFu;
        xw |= (SUF >> 24) << (xl & 31u); xl += S >> 24;
        if ((g & 3) == 3) { XW[g >> 2] = xw; OX[g >> 2] = TX; TX += xl; xw = 0; xl = 0; }
        __builtin_amdgcn_sched_barrier(0);
    }
    bad = (bad > 31u) | (TU > LANE_U_MAX);
    const unsigned tt = TU | (TX << 16), incl = wave_scan_incl(tt), tot = __builtin_amdgcn_readlane(incl, 63), totU = tot & 0xFFFFu, totX = tot >> 16;
    const unsigned xstart = (AUX_U * 8u + totU + 31u) & ~31u, endbit = xstart + totX;
    unsigned raw = (__ballot(bad != 0) != 0ull) || totU > U_MAX || endbit > (AUX_MAX - 8u) * 8u;
    if (!raw) {
        const unsigned px = xstart + ((incl - tt) >> 16);
        unsigned p = AUX_U * 8u + ((incl - tt) & 0xFFFFu);
#pragma unroll
        for (int g = 0; g < 16; ++g) {
            const unsigned ln = KK[g] + 0x01010101u;
            unsigned pos = (ln & 0xFFu) - 1u, w = 1u << pos;
            pos += (ln >> 8) & 0xFFu; w |= 1u << pos;
            pos += (ln >> 16) & 0xFFu; w |= 1u << pos;
            pos += ln >> 24; w |= 1u << pos;
            lds_or2(scr, p, w); p += pos + 1u;
            __builtin_amdgcn_sched_barrier(0);
        }
#pragma unroll
        for (int q = 0; q < 4; ++q) lds_or2(scr, px + OX[q], XW[q]);
    }
#pragma unroll
    for (int i = 0; i < 7; ++i) { P[i] = R[i] | ((R[14] << (7 - i)) & 0x80808080u); P[7 + i] = R[7 + i] | ((R[15] << (7 - i)) & 0x80808080u); }
    unsigned n = 0;
    const unsigned xo = (endbit + 31u) >> 5;
    if (!raw && __ballot((any & 0xF0F0F0F0u) != 0) != 0ull) {
#pragma unroll
        for (int g = 0; g < 16; ++g) {
            const unsigned f = E[g] & 0xF0F0F0F0u;
            unsigned long long mask = __ballot(f != 0);
            while (mask != 0ull && n <= MAX_EXC) {
                const int L = __builtin_ctzll(mask); mask &= mask - 1;
                const unsigned fl = __builtin_amdgcn_readlane(f, L);
                const unsigned lo = __builtin_amdgcn_readlane((g & 1) ? c[g >> 1].z : c[g >> 1].x, L);
                const unsigned hi = __builtin_amdgcn_readlane((g & 1) ? c[g >> 1].w : c[g >> 1].y, L);
                const bool e = lane < 4 && ((fl >> (8 * (lane & 3))) & 0xFFu) != 0;
                const unsigned long long em = __ballot(e);
                const unsigned slot = xo + n + __builtin_popcountll(em & ((1ull << lane) - 1ull));
                const unsigned val = (((lane & 2) ? hi : lo) >> (16 * (lane & 1))) & 0xFFFFu;
                if (e && slot * 4u < AUX_MAX) *(LAS unsigned*)(scr + slot * 4u) = (unsigned)(L * 64 + g * 4 + (lane & 3)) | (val << 12);
                n += __builtin_popcountll(em);
            }
        }
    }
    raw |= (n > MAX_EXC) | ((xo + n) * 4u > AUX_MAX);
    LDS_FENCE();
    a[0] = *(LAS u32x4*)(scr + lane * 16); a[1] = *(LAS u32x4*)(scr + 1024 + lane * 16); a[2] = *(LAS u32x4*)(scr + 2048 + (lane & 31) * 16);
    return ((xo + n + 3u) >> 2) | (n << 8) | (xo << 16) | (raw << 31);
}

__device__ __forceinline__ void decode_block(LAS unsigned char* src, LAS unsigned char* pos_scr, u32x4 (&c)[8], int lane) {
    LAS unsigned char* aux = src + RAWB;
    const unsigned c0c = opaque(0x0C0C0C0Cu);
    const u32x4 nt = *(LAS const u32x4*)(aux);
    const unsigned prm_v = *(LAS const unsigned*)(aux + 16);
    unsigned u[5];
#pragma unroll
    for (int i = 0; i < 5; ++i) u[i] = *(LAS const unsigned*)(aux + AUX_U + lane * 20 + i * 4);
    u32x4 P4[3];
#pragma unroll
    for (int p = 0; p < 3; ++p) P4[p] = *(LAS const u32x4*)(src + p * 1024 + lane * 16);
    const unsigned long long ph = *(LAS const unsigned long long*)(src + 3072 + lane * 8);
    const unsigned c0 = __builtin_popcount(u[0]), c1 = c0 + __builtin_popcount(u[1]), c2 = c1 + __builtin_popcount(u[2]), c3 = c2 + __builtin_popcount(u[3]), cn = c3 + __builtin_popcount(u[4]);
    const unsigned C = wave_scan_incl(cn) - cn;
    if (lane == 0) *(LAS unsigned*)(pos_scr) = AUX_U * 8u;
    const int rounds = __ballot(cn > 128u) != 0ull ? 3 : 2;
    for (int j = 0; j < rounds; ++j) {
        const unsigned t = (C >> 6) + 1u + j, kth = 64u * t - C;
        const unsigned i = (kth > c0) + (kth > c1) + (kth > c2) + (kth > c3);
        unsigned rem = kth - (i == 0 ? 0u : i == 1 ? c0 : i == 2 ? c1 : i == 3 ? c2 : c3), w = i == 0 ? u[0] : i == 1 ? u[1] : i == 2 ? u[2] : i == 3 ? u[3] : u[4], pos = 0;
#pragma unroll
        for (int st = 16; st >= 1; st >>= 1) {
            const unsigned lo = __builtin_popcount(w & ((1u << st) - 1u));
            const bool up = rem > lo;
            rem -= up ? lo : 0u; w = up ? w >> st : w; pos += up ? st : 0;
        }
        if (kth <= cn && t < 64u) *(LAS unsigned*)(pos_scr + t * 4) = AUX_U * 8u + lane * 160u + i * 32u + pos + 1u;
    }
    LDS_FENCE();
    const unsigned p0 = *(LAS const unsigned*)(pos_scr + lane * 4);
    unsigned S[8];
    {
        LAS const unsigned* q = (LAS const unsigned*)(aux + ((p0 >> 5) << 2));
#pragma unroll
        for (int i = 0; i < 8; ++i) S[i] = q[i];
#pragma unroll
        for (int i = 0; i < 7; ++i) S[i] = __builtin_amdgcn_alignbit(S[i + 1], S[i], p0 & 31u);
    }
    unsigned R[16];
    {
        const unsigned P[14] = {P4[0].x, P4[0].y, P4[0].z, P4[0].w, P4[1].x, P4[1].y, P4[1].z, P4[1].w, P4[2].x, P4[2].y, P4[2].z, P4[2].w, (unsigned)ph, (unsigned)(ph >> 32)};
        unsigned x = 0, y = 0;
#pragma unroll
        for (int i = 0; i < 7; ++i) { x |= (P[i] >> (7 - i)) & (0x01010101u << i); y |= (P[7 + i] >> (7 - i)) & (0x01010101u << i); }
#pragma unroll
        for (int i = 0; i < 14; ++i) R[i] = P[i] & 0x7F7F7F7Fu;
        R[14] = x; R[15] = y;
    }
    const unsigned n0 = rfl(nt.x), n1 = rfl(nt.y), n2 = rfl(nt.z), n3 = rfl(nt.w), prm = rfl(prm_v);
    const unsigned I[8] = {n0 & 0x0F0F0F0Fu, (n0 >> 4) & 0x0F0F0F0Fu, n1 & 0x0F0F0F0Fu, (n1 >> 4) & 0x0F0F0F0Fu, n2 & 0x0F0F0F0Fu, (n2 >> 4) & 0x0F0F0F0Fu, n3 & 0x0F0F0F0Fu, (n3 >> 4) & 0x0F0F0F0Fu};
    const unsigned s0P = (prm >> 16) & 3u, s1P = (prm >> 18) & 3u, s0N = (prm >> 20) & 3u, s1N = (prm >> 22) & 3u;
    const unsigned STP = (s0P << 8) | (s1P << 16) | (1u << 24), STN = (s0N << 8) | (s1N << 16) | (1u << 24);
    const unsigned OTP = (4u << 8) | (((1u << s0P) + 2u) << 16) | (((1u << s0P) + (1u << s1P)) << 24);
    const unsigned OTN = (4u << 8) | (((1u << s0N) + 2u) << 16) | (((1u << s0N) + (1u << s1N)) << 24);
    const unsigned BP = (prm & 0xFFu) * 0x01010101u, BN = ((prm >> 8) & 0xFFu) * 0x01010101u;
    unsigned RB[16], SS[16], TU = 0, TX = 0;
#pragma unroll
    for (int g = 0; g < 16; ++g) {
        const unsigned w0 = S[0], q0 = __builtin_ctz(w0), w1 = w0 & (w0 - 1u), q1 = __builtin_ctz(w1), w2 = w1 & (w1 - 1u), q2 = __builtin_ctz(w2), w3 = w2 & (w2 - 1u), q3 = __builtin_ctz(w3);
        const unsigned Q = q0 | (q1 << 8) | (q2 << 16) | (q3 << 24);
        const unsigned LN = Q - opaque((Q << 8) - 1u), sh = q3 + 1u;
        TU += sh;
        if (g < 15) {
#pragma unroll
            for (int i = 0; i < 7; ++i) if (32 * i < (int)LANE_U_MAX - 4 * (g + 1)) S[i] = (i < 6 && 32 * (i + 1) < (int)LANE_U_MAX - 4 * g) ? __builtin_amdgcn_alignbit(S[i + 1], S[i], sh) : S[i] >> sh;
        }
        const unsigned sel = bfi_c(0x04040404u, R[g] >> 4, bfi(bytemask(LN + 0x7D7D7D7Du, 7, c0c), 0x03030303u, LN));
        SS[g] = perm(STN, STP, sel);
        RB[g] = perm(OTN, OTP, sel) + (LN << 1);
        TX = __builtin_amdgcn_sad_u8(SS[g], 0u, TX);
    }
    const unsigned xstart = (__builtin_amdgcn_readlane(p0 + TU, 63) + 31u) & ~31u, px = xstart + wave_scan_incl(TX) - TX;
    unsigned X[5];
    {
        LAS const unsigned* q = (LAS const unsigned*)(aux + ((px >> 5) << 2));
#pragma unroll
        for (int i = 0; i < 5; ++i) X[i] = q[i];
#pragma unroll
        for (int i = 0; i < 4; ++i) X[i] = __builtin_amdgcn_alignbit(X[i + 1], X[i], px & 31u);
    }
    unsigned xoff = 0;
#pragma unroll
    for (int g = 0; g < 16; ++g) {
        const unsigned s = SS[g], M = perm(0u, 0x00030100u, s);
        const unsigned y = (g & 1) ? X[0] >> xoff : X[0], Y = perm(y, y, 0x01000100u);
        const unsigned t1 = opaque(s << 8), t2 = opaque((t1 << 8) + t1), Cs = (t2 << 16) + t2;
        const u16x2 z02 = __builtin_bit_cast(u16x2, Y) >> __builtin_bit_cast(u16x2, Cs & 0x00FF00FFu);
        const u16x2 z13 = __builtin_bit_cast(u16x2, Y) >> __builtin_bit_cast(u16x2, perm(0u, Cs, 0x0C030C01u));
        const unsigned suf = perm(__builtin_bit_cast(unsigned, z13), __builtin_bit_cast(unsigned, z02), 0x06020400u) & M;
        if (g & 1) {
            if (g < 15) {
                const unsigned t = __builtin_amdgcn_sad_u8(s, 0u, xoff);
#pragma unroll
                for (int i = 0; i < 4; ++i) if (32 * i < 8 * (15 - g)) X[i] = (i < 3 && 32 * (i + 1) < 8 * (17 - g)) ? __builtin_amdgcn_alignbit(X[i + 1], X[i], t) : X[i] >> t;
            }
        } else xoff = sad4(s);
        const unsigned rank = RB[g] + suf - 0x06060606u, ms = bytemask(R[g], 6, c0c);
        const unsigned D = bfi(bytemask(rank, 4, c0c), rank, lut2x16(I, rank, ms, c0c));
        const unsigned E = bfi(ms, BN, BP) - ((D >> 1) & 0x0F0F0F0Fu);
        const unsigned H = bfi_c(0x80808080u, R[g] << 1, E >> 1), L = bfi_c(0x3F3F3F3Fu, R[g], bfi_c(0x40404040u, D << 6, E << 7));
        const unsigned x = perm(H, L, 0x05010400u), yv = perm(H, L, 0x07030602u);
        if (g & 1) { c[g >> 1].z = x; c[g >> 1].w = yv; } else { c[g >> 1].x = x; c[g >> 1].y = yv; }
    }
}

typedef unsigned u32x2 __attribute__((ext_vector_type(2)));
__device__ __forceinline__ void store_granule(char* p, u32x2 g) {
    asm volatile("global_store_dwordx2 %0, %1, off sc0 sc1\n\ts_nop 1" :: "v"(p), "v"(g) : "memory");
}

constexpr unsigned GCTL = 131072;
__device__ __forceinline__ unsigned payload_bytes(unsigned h) { return (h >> 31) ? 8192u : RAWB + ((h & 255u) << 4); }
__device__ __forceinline__ void publish_block(const Params& P, rsrc_t wsr, const u32x4 (&c)[8], unsigned slot, unsigned n, unsigned dmask, unsigned call, LAS unsigned char* lds, LAS unsigned char* scr,
                                              unsigned j, unsigned gsize, int wid, int lane) {
    unsigned R[14], h = 1u << 31; u32x4 a[3];
    if (n >= NRAW && n < NSEQ - NRAWEND) h = encode_block(c, R, a, scr, lane);
    const unsigned u = j & 3u;
    volatile LAS unsigned* ctl = (volatile LAS unsigned*)(lds + GCTL);
    while (j >= 4u && (int)(ctl[36 + u] - (j - 3u)) < 0) __builtin_amdgcn_s_sleep(1);
    unsigned so = OFF_SLOT + (slot & ~1u) * 8192u;
    if (wid & 1) {
        unsigned v;
        while (((v = ctl[40 + u * 8 + wid - 1]) >> 16) != ((j + 1u) & 0xFFFFu)) __builtin_amdgcn_s_sleep(1);
        so += (rfl(v) & 0xFFFFu) << 4;
    } else if (lane == 0) ctl[40 + u * 8 + wid] = ((j + 1u) << 16) | (payload_bytes(h) >> 4);
    if (h >> 31) {
#pragma unroll
        for (int q = 0; q < 8; ++q) __builtin_amdgcn_raw_buffer_store_b128(c[q], wsr, lane * 16 + q * 1024, so, AUX_SYS);
    } else {
        const unsigned n16 = h & 255u;
#pragma unroll
        for (int q = 0; q < 3; ++q) __builtin_amdgcn_raw_buffer_store_b128((u32x4){R[4 * q], R[4 * q + 1], R[4 * q + 2], R[4 * q + 3]}, wsr, lane * 16 + q * 1024, so, AUX_SYS);
        __builtin_amdgcn_raw_buffer_store_b64((u32x2){R[12], R[13]}, wsr, lane * 8 + 3072, so, AUX_SYS);
        if ((unsigned)lane < n16) __builtin_amdgcn_raw_buffer_store_b128(a[0], wsr, lane * 16 + RAWB, so, AUX_SYS);
        if ((unsigned)lane + 64u < n16) __builtin_amdgcn_raw_buffer_store_b128(a[1], wsr, lane * 16 + RAWB + 1024, so, AUX_SYS);
        if ((unsigned)lane + 128u < n16) __builtin_amdgcn_raw_buffer_store_b128(a[2], wsr, lane * 16 + RAWB + 2048, so, AUX_SYS);
    }
    WAIT_VM0();
    unsigned arrived = 0;
    if (lane == 0) {
        ctl[u * 8 + wid] = h;
        arrived = __hip_atomic_fetch_add((LAS unsigned*)(lds + GCTL) + 32 + u, 1u, __ATOMIC_RELAXED, __HIP_MEMORY_SCOPE_WORKGROUP);
    }
    if (rfl(arrived) == gsize - 1u) {
        const unsigned d = lane >> 3, w = lane & 7, hw = ctl[u * 8 + w];
        if (w < gsize && ((dmask >> d) & 1u)) store_granule(P.ws[d] + OFF_GRAN + ((unsigned)P.rank * NSEQ + n - wid + w) * 8u, (u32x2){call, hw});
        if (lane == 0) { ctl[32 + u] = 0u; ctl[36 + u] = j + 1u; }
    }
}

__device__ __forceinline__ void signal_cnt(char* ws, unsigned off, int lane) {
    if (off && lane == 0) __hip_atomic_fetch_add((unsigned*)(ws + off), 1u, __ATOMIC_RELAXED, __HIP_MEMORY_SCOPE_SYSTEM);
}
__device__ __forceinline__ void push_first(const Params& P, int b, int wid) {
    const int lane = threadIdx.x & 63, sel = b >> 5, bi = b & 31, rr = lane >> 3, g = P.rank * 16;
    const rsrc_t xr = mk_rsrc(P.x, DIM * KSH * 2);
    char* dst = P.ws[(sel * 4 + wid) & 7];
    const rsrc_t dr = mk_rsrc(dst, WS_USED);
    u32x4 c[8];
#pragma unroll
    for (int q = 0; q < 8; ++q) c[q] = __builtin_amdgcn_raw_buffer_load_b128(xr, (unsigned)rr * 2048u + (unsigned)(((lane & 7) ^ rr) * 16) + q * 8 * 2048, (unsigned)(sel * 4096 + bi * 64) * 2048u, 0);
#pragma unroll
    for (int q = 0; q < 8; ++q) __builtin_amdgcn_raw_buffer_store_b128(c[q], dr, lane * 16 + q * 1024, OFF_AIMG + (unsigned)(g * 32 + bi) * 8192u, AUX_SYS);
    WAIT_VM0();
    if (lane == 0) __hip_atomic_fetch_add((unsigned*)(dst + OFF_CNT + g * 128), 1u, __ATOMIC_RELAXED, __HIP_MEMORY_SCOPE_SYSTEM);
}
__device__ __forceinline__ unsigned packer_role(const Params& P, LAS unsigned char* lds, int first, int next, int step, int end) {
    unsigned call = 0;
    const int lane = threadIdx.x & 63, wid = rfl(threadIdx.x >> 6);
    char* ws = P.ws[P.rank];
    const rsrc_t wsr = mk_rsrc(ws, WS_USED);
    const rsrc_t xr = mk_rsrc(P.x, DIM * KSH * 2), wr = mk_rsrc(P.w, DIM * KSH * 2);
    const int myi = P.rank >> 2, myj = P.rank & 3;
    LAS unsigned char* my = lds + wid * 8192;
    unsigned j = first < NFIRST ? 0u : 1u;
    const unsigned long long tstart = __builtin_amdgcn_s_memrealtime();
    for (int p = first; p < end; ++j) {
        if (p < 4096) {
            const int due = (p < 3072 ? (p / 192) * 580 : 9280 + ((p - 3072) >> 6) * 290) - JIT_LEAD * 100;
            while ((int)(__builtin_amdgcn_s_memrealtime() - tstart) < due) __builtin_amdgcn_s_sleep(16);
        }
        u32x4 c[8];
        unsigned slot, n, dmask, selfcnt = 0;
        if (p >= 4096) {
            const unsigned o = p - 4096, w = o & 7, h = (o >> 3) & 1, tile = (o >> 4) & 63;
            if (call == 0) call = call_draw(ws);
            unsigned* tmo = (unsigned*)(ws + OFF_TMO);
            wait_word((unsigned*)(ws + OFF_OCNT + (o >> 3) * 128), call * 8u, tmo, call);
            const unsigned row0 = (P.rank >> 2) * 4096 + (tile >> 3) * 256 + h * 128 + (w >> 2) * 64, col0 = (P.rank & 3) * 2048 + (tile & 7) * 256 + (w & 3) * 64;
            const rsrc_t outr = mk_rsrc(P.out, (unsigned)DIM * DIM * 2u);
#pragma unroll
            for (int q = 0; q < 8; ++q) c[q] = __builtin_amdgcn_raw_buffer_load_b128(outr, (unsigned)(lane >> 3) * (DIM * 2u) + (unsigned)(lane & 7) * 16u, (row0 * DIM + col0) * 2u + q * (8u * DIM * 2u), AUX_SYS);
            if (rfl(__hip_atomic_load(tmo, __ATOMIC_RELAXED, __HIP_MEMORY_SCOPE_AGENT)) == call) {
#pragma unroll
                for (int q = 0; q < 8; ++q) c[q] = (u32x4){0x7FC07FC0u, 0x7FC07FC0u, 0x7FC07FC0u, 0x7FC07FC0u};
            }
            slot = 4096u + o; n = 1536u + o; dmask = 0xFFu;
        } else {
            bool is_w; int kk, sel, r, bi;
            if (p < 3072) {
                kk = p / 192; const int rem = p - kk * 192;
                bi = rem & 31; r = 0; is_w = rem >= 64; sel = is_w ? ((rem - 64) >> 5) : (rem >> 5);
            } else { const int q = p - 3072; kk = q >> 6; sel = (q >> 5) & 1; bi = q & 31; r = 1; is_w = false; }
            if (!is_w) {
                const int rr = lane >> 3;
                const unsigned vo = (unsigned)rr * 2048u + (unsigned)(((lane & 7) ^ rr) * 16);
                const unsigned so = (unsigned)(sel * 4096 + r * 2048 + bi * 64) * 2048u + (unsigned)kk * 128u;
#pragma unroll
                for (int q = 0; q < 8; ++q) c[q] = __builtin_amdgcn_raw_buffer_load_b128(xr, vo + q * 8 * 2048, so, 0);
            } else {
                const int pn = bi >> 2, bj = (bi >> 1) & 1, q = bi & 1;
                const int gk = lane >> 3, cc = lane & 7, wcl = cc >> 2, cq = cc & 3;
                const unsigned vo = (unsigned)(gk * 8) * 16384u + (unsigned)((64 * (2 * q + wcl) + 32 * bj + 8 * cq) * 2);
                const unsigned so = (unsigned)(kk * 64) * 16384u + (unsigned)((sel * 2048 + pn * 256) * 2);
                u32x4 R[8];
#pragma unroll
                for (int jj = 0; jj < 8; ++jj) R[jj] = __builtin_amdgcn_raw_buffer_load_b128(wr, vo + jj * 16384, so, 0);
#pragma unroll
                for (int t = 0; t < 8; ++t) {
                    const unsigned s = (t & 1) ? 0x07060302u : 0x05040100u;
                    u32x4 v;
                    v.x = perm(R[1][t >> 1], R[0][t >> 1], s); v.y = perm(R[3][t >> 1], R[2][t >> 1], s);
                    v.z = perm(R[5][t >> 1], R[4][t >> 1], s); v.w = perm(R[7][t >> 1], R[6][t >> 1], s);
                    const int s64 = 32 * wcl + 16 * (t >> 2) + 4 * cq + (t & 3);
                    *(LAS u32x4*)(my + s64 * 128 + ((gk ^ (s64 & 7)) * 16)) = v;
                }
                asm volatile("s_waitcnt lgkmcnt(0)" ::: "memory");
#pragma unroll
                for (int q2 = 0; q2 < 8; ++q2) c[q2] = *(LAS u32x4*)(my + q2 * 1024 + lane * 16);
                asm volatile("s_waitcnt lgkmcnt(0)" ::: "memory");
            }
            if (call == 0) call = call_draw(ws);
            const int g = P.rank * 16 + kk;
            if (is_w ? (sel == myj) : (sel == myi)) {
                const unsigned so = is_w ? OFF_BIMG + (unsigned)(g * 32 + bi) * 8192u : OFF_AIMG + (unsigned)((r * 128 + g) * 32 + bi) * 8192u;
#pragma unroll
                for (int q = 0; q < 8; ++q) __builtin_amdgcn_raw_buffer_store_b128(c[q], wsr, lane * 16 + q * 1024, so, AUX_SC1);
                selfcnt = OFF_CNT + (r * 128 + g) * 128;
            }
            slot = is_w ? 2048u + (unsigned)((sel * 16 + kk) * 32 + bi) : (unsigned)(((sel * 2 + r) * 16 + kk) * 32 + bi);
            n = is_w ? (unsigned)(kk * 64 + 32 + bi) : (r == 0 ? (unsigned)(kk * 64 + bi) : (unsigned)(1024 + kk * 32 + bi));
            dmask = is_w ? (0x11u << sel) : (0x0Fu << (4 * sel));
        }
        publish_block(P, wsr, c, slot, n, dmask & ~(1u << P.rank), call, lds, my, j, (p < NFIRST || (p >= 4096 && p < OUT2)) ? 4u : 8u, wid, lane);
        signal_cnt(ws, selfcnt, lane);
        if (p < NFIRST) p = next;
        else if (p < 4096) {
            p += step;
            if (p >= 4096 && end > 4096) { if (wid < 4) p = 4096 + (int)blockIdx.x * 4 + wid; else { p = OUT2 + (int)blockIdx.x * 8 + wid; ++j; } }
        } else if (p < OUT2) p = OUT2 + (int)blockIdx.x * 8 + wid;
        else p += step;
    }
    if (call == 0) call = call_draw(ws);
    return call;
}

struct Blk { unsigned slot, dso, cnt; bool is_out; };
__device__ __forceinline__ Blk locate(unsigned n, int src, int myi, int myj) {
    Blk b; b.is_out = false;
    if (n < 1024) { const int kk = n >> 6, bi = n & 31, g = src * 16 + kk; b.cnt = OFF_CNT + g * 128;
        if ((n & 63) >= 32) { b.slot = 2048u + (myj * 16 + kk) * 32 + bi; b.dso = OFF_BIMG + (unsigned)(g * 32 + bi) * 8192u; }
        else { b.slot = ((myi * 2) * 16 + kk) * 32 + bi; b.dso = OFF_AIMG + (unsigned)(g * 32 + bi) * 8192u; } }
    else if (n < 1536) { const int kk = (n - 1024) >> 5, bi = n & 31, g = src * 16 + kk; b.cnt = OFF_CNT + (128 + g) * 128;
        b.slot = ((myi * 2 + 1) * 16 + kk) * 32 + bi; b.dso = OFF_AIMG + (unsigned)((128 + g) * 32 + bi) * 8192u; }
    else { const int o = n - 1536, w = o & 7, h = (o >> 3) & 1, tile = (o >> 4) & 63, r = o >> 10; b.cnt = 0; b.is_out = true;
        b.slot = 4096u + o;
        const unsigned row0 = (src >> 2) * 4096 + r * 2048 + (tile >> 3) * 256 + h * 128 + (w >> 2) * 64, col0 = (src & 3) * 2048 + (tile & 7) * 256 + (w & 3) * 64;
        b.dso = (row0 * DIM + col0) * 2u; }
    return b;
}
constexpr unsigned NPAIR = NSEQ / 2, NSLOT = 7, CTL = NSLOT * 16384;
constexpr unsigned M_DEAD = 1u << 30, N_STOP = 0xFFFFFFFFu;
__device__ __forceinline__ unsigned lds_ld(LAS unsigned char* p) { return *(volatile LAS unsigned*)p; }
__device__ __forceinline__ void lds_st(LAS unsigned char* p, unsigned v) { *(volatile LAS unsigned*)p = v; }
__device__ __forceinline__ void ring_publish(LAS unsigned char* lds, unsigned i, unsigned n, unsigned ha, unsigned hb, int lane) {
    const unsigned sl = i % NSLOT;
    if (lane == 0) { *(volatile LAS u32x4*)(lds + CTL + sl * 16) = (u32x4){n, ha, hb, 0u}; lds_st(lds + CTL + 128 + sl * 4, i + 1u); }
}
__device__ __forceinline__ bool slot_busy(LAS unsigned char* lds, unsigned i) { return i >= NSLOT && (int)(lds_ld(lds + CTL + 160 + (i % NSLOT) * 4) - 2u * (i / NSLOT)) < 0; }
__device__ __forceinline__ void dma_op(rsrc_t r, LAS unsigned char* dst, unsigned j, unsigned so, int lane) {
    __builtin_amdgcn_raw_ptr_buffer_load_lds(r, (LAS void*)(dst + j * 1024), 16, lane * 16, so + j * 1024, 0, AUX_SYS);
}
__device__ __forceinline__ u32x4 load_pair(rsrc_t wsr, unsigned off) { return __builtin_amdgcn_raw_buffer_load_b128(wsr, 0, off, AUX_SYS); }
#define VM_WAIT_(n) asm volatile("s_waitcnt vmcnt(" #n ")" ::: "memory")
#define VM_WAIT(n) VM_WAIT_(n)
#define G_STR_(x) #x
#define G_STR(x) G_STR_(x)
#define WIN 5

__device__ __forceinline__ void loader_wave(const Params& P, LAS unsigned char* lds, unsigned call, int src, int lane) {
    char* ws = P.ws[P.rank];
    unsigned* tmo = (unsigned*)(ws + OFF_TMO);
    const rsrc_t wsr = mk_rsrc(ws, WS_USED);
    char* const remp = P.ws[src];
    const int myi = P.rank >> 2, myj = P.rank & 3;
    const unsigned gbase = OFF_GRAN + (unsigned)src * NSEQ * 8u;
    char* const gptr = ws + gbase;
    const unsigned tko = OFF_LINK + (unsigned)src * 128u, tbase = (call - 1u) * (NPAIR - NSKIP / 2 + 3u * PWL) - NSKIP / 2;
    __builtin_amdgcn_s_setprio(3);
    unsigned q, q1, q2;
    {
        unsigned t0 = 0;
        if (lane == 0) t0 = __hip_atomic_fetch_add((unsigned*)(ws + tko), 3u, __ATOMIC_RELAXED, __HIP_MEMORY_SCOPE_AGENT);
        q = rfl(t0) - tbase; q1 = q + 1; q2 = q + 2;
    }
    {
        const rsrc_t rw = mk_rsrc(remp, OFF_AIMG);
        const unsigned v = __builtin_amdgcn_raw_buffer_load_b32(rw, lane < 25 ? OFF_SLOT + (unsigned)lane * (2u << 20) + 8188u - (lane == 24 ? 8192u : 0u) : 0xFFFFFFF0u, 0, AUX_SYS);
        asm volatile("" :: "v"(v));
    }
    unsigned gca, gha, gcb, ghb, hca, hha, hcb, hhb;
    { const u32x4 a = load_pair(wsr, gbase + (q < NPAIR ? q : 0u) * 16u), b = load_pair(wsr, gbase + (q1 < NPAIR ? q1 : 0u) * 16u);
      gca = rfl(a.x); gha = rfl(a.y); gcb = rfl(a.z); ghb = rfl(a.w); hca = rfl(b.x); hha = rfl(b.y); hcb = rfl(b.z); hhb = rfl(b.w); }
    const int zero = 0, one = 1;
    unsigned i = 0;
    bool pend = false; unsigned pn = 0, pa = 0, pb = 0;
    while (q < NPAIR) {
        bool dead = false;
        if ((int)(gca - call) < 0 || (int)(gcb - call) < 0 || slot_busy(lds, i)) {
            if (pend) { VM_WAIT(0); ring_publish(lds, i - 1u, pn, pa, pb, lane); pend = false; }
            const unsigned long long t0 = __builtin_amdgcn_s_memrealtime();
            while ((int)(gca - call) < 0 || (int)(gcb - call) < 0) {
                const u32x4 a = load_pair(wsr, gbase + q * 16u);
                gca = rfl(a.x); gha = rfl(a.y); gcb = rfl(a.z); ghb = rfl(a.w);
                if ((int)(gca - call) >= 0 && (int)(gcb - call) >= 0) break;
                if (give_up(t0, tmo, call)) { dead = true; break; }
                __builtin_amdgcn_s_sleep(2);
                asm volatile("" ::: "memory");
            }
            while (slot_busy(lds, i)) __builtin_amdgcn_s_sleep(1);
        }
        const unsigned n = 2u * q;
        const unsigned so = OFF_SLOT + locate(n, src, myi, myj).slot * 8192u;
        const unsigned bytes = dead ? 0u : payload_bytes(gha) + payload_bytes(ghb), nops = (bytes + 1023u) >> 10;
        const rsrc_t rem = mk_rsrc(remp, so + bytes);
        LAS unsigned char* dst = lds + (i % NSLOT) * 16384;
        u32x4 g2; unsigned tk = 0;
        asm volatile("global_load_dwordx4 %0, %1, %2 sc0 sc1" : "=v"(g2) : "v"(zero), "s"(gptr + (q2 < NPAIR ? q2 : 0u) * 16u) : "memory");
        if (lane == 0) asm volatile("global_atomic_add %0, %1, %2, %3 sc0" : "=v"(tk) : "v"(zero), "v"(one), "s"(ws + tko) : "memory");
#define G_FILL(jj) do { if ((jj) == WIN) asm volatile("s_waitcnt vmcnt(" G_STR(WIN) ")" : "+v"(g2), "+v"(tk) :: "memory"); else VM_WAIT(WIN); \
                        if ((jj) == WIN - 2 && pend) ring_publish(lds, i - 1u, pn, pa, pb, lane); dma_op(rem, dst, jj, so, lane); } while (0)
        G_FILL(0); G_FILL(1); G_FILL(2); G_FILL(3); G_FILL(4); G_FILL(5); G_FILL(6); G_FILL(7);
#undef G_FILL
        for (unsigned j = 8; j < nops; ++j) { VM_WAIT(WIN); dma_op(rem, dst, j, so, lane); }
        pend = true; pn = n; pa = gha | (dead ? M_DEAD : 0u); pb = ghb | (dead ? M_DEAD : 0u);
        q = q1; gca = hca; gha = hha; gcb = hcb; ghb = hhb;
        q1 = q2; hca = rfl(g2.x); hha = rfl(g2.y); hcb = rfl(g2.z); hhb = rfl(g2.w);
        q2 = rfl(tk) - tbase;
        ++i;
    }
    VM_WAIT(0);
    if (pend) ring_publish(lds, i - 1u, pn, pa, pb, lane);
    for (int d = 0; d < (NDEC + 1) / 2 + 1; ++d, ++i) {
        while (slot_busy(lds, i)) __builtin_amdgcn_s_sleep(1);
        ring_publish(lds, i, N_STOP, 0u, 0u, lane);
    }
}

__device__ __forceinline__ void decoder_wave(const Params& P, LAS unsigned char* lds, unsigned call, int src, int d, int lane) {
    char* ws = P.ws[P.rank];
    unsigned* tmo = (unsigned*)(ws + OFF_TMO);
    const rsrc_t wsr = mk_rsrc(ws, WS_USED), outr = mk_rsrc(P.out, (unsigned)DIM * DIM * 2u);
    const int myi = P.rank >> 2, myj = P.rank & 3;
    for (unsigned i = d;; i += NDEC) {
        const unsigned pi = i >> 1, sl = pi % NSLOT;
        while (lds_ld(lds + CTL + 128 + sl * 4) != pi + 1u) __builtin_amdgcn_s_sleep(1);
        asm volatile("" ::: "memory");
        const u32x4 meta = *(volatile LAS u32x4*)(lds + CTL + sl * 16);
        const unsigned n0 = rfl(meta.x), ha = rfl(meta.y), h = (i & 1) ? rfl(meta.z) : ha;
        if (n0 == N_STOP) break;
        const Blk b = locate(n0 + (i & 1), src, myi, myj);
        const bool raw = (h >> 31) != 0;
        bool dead = (h & M_DEAD) != 0;
        if (b.is_out && rfl(__hip_atomic_load(tmo, __ATOMIC_RELAXED, __HIP_MEMORY_SCOPE_AGENT)) == call) dead = true;
        u32x4 c[8];
        unsigned recs = 0;
        const unsigned ne = (dead || raw) ? 0u : (h >> 8) & 63u;
        LAS unsigned char* cur = lds + sl * 16384 + ((i & 1) ? payload_bytes(ha) : 0u);
        if (dead) {
#pragma unroll
            for (int j = 0; j < 8; ++j) c[j] = (u32x4){0x7FC07FC0u, 0x7FC07FC0u, 0x7FC07FC0u, 0x7FC07FC0u};
        } else if (raw) {
#pragma unroll
            for (int j = 0; j < 8; ++j) c[j] = *(LAS u32x4*)(cur + j * 1024 + lane * 16);
        } else {
            recs = *(LAS unsigned*)(cur + RAWB + (((h >> 16) & 0x3FFu) + (lane & 31)) * 4u);
            decode_block(cur, lds + CTL + 256 + d * 256, c, lane);
        }
#pragma unroll
        for (int j = 0; j < 8; ++j) asm volatile("" : "+v"(c[j]));
        asm volatile("" : "+v"(recs));
        if (lane == 0) __hip_atomic_fetch_add((LAS unsigned*)(lds + CTL + 160 + sl * 4), 1u, __ATOMIC_RELAXED, __HIP_MEMORY_SCOPE_WORKGROUP);
        const unsigned sj = b.is_out ? 8u * DIM * 2u : 1024u, sr = b.is_out ? DIM * 2u : 128u;
        const unsigned vo = (unsigned)(lane >> 3) * sr + (unsigned)(lane & 7) * 16u;
        if (b.is_out) {
#pragma unroll
            for (int j = 0; j < 8; ++j) __builtin_amdgcn_raw_buffer_store_b128(c[j], outr, vo, b.dso + j * sj, OUT_AUX);
            for (unsigned e = 0; e < ne; ++e) {
                const unsigned rec = __builtin_amdgcn_readlane(recs, e), pos = rec & 0xFFFu, v = pos & 63u;
                if ((unsigned)lane == (pos >> 6)) __builtin_amdgcn_raw_buffer_store_b16((unsigned short)((rec >> 12) & 0xFFFFu), outr, vo + (v & 7u) * 2u, b.dso + (v >> 3) * sj, OUT_AUX);
            }
        } else {
#pragma unroll
            for (int j = 0; j < 8; ++j) __builtin_amdgcn_raw_buffer_store_b128(c[j], wsr, vo, b.dso + j * sj, IMG_AUX);
            for (unsigned e = 0; e < ne; ++e) {
                const unsigned rec = __builtin_amdgcn_readlane(recs, e), pos = rec & 0xFFFu, v = pos & 63u;
                if ((unsigned)lane == (pos >> 6)) __builtin_amdgcn_raw_buffer_store_b16((unsigned short)((rec >> 12) & 0xFFFFu), wsr, vo + (v & 7u) * 2u, b.dso + (v >> 3) * sj, IMG_AUX);
            }
            WAIT_VM0(); signal_cnt(ws, b.cnt, lane);
        }
    }
}

__device__ __forceinline__ void puller_role(const Params& P, LAS unsigned char* lds, unsigned call, int wg) {
    const int lane = threadIdx.x & 63, wid = rfl(threadIdx.x >> 6);
    const int link = wg % 7, src = rfl((P.rank + 1 + link) & 7);
    if (threadIdx.x < 16) *(volatile LAS unsigned*)(lds + CTL + 128 + threadIdx.x * 4) = 0u;
    __syncthreads();
    if (wid == 0) loader_wave(P, lds, call, src, lane);
    else if (wid <= NDEC) decoder_wave(P, lds, call, src, wid - 1, lane);
}

__device__ __forceinline__ float gelu_f(float y) {
    const float c1 = -2.0f * 0.7978845608028654f * 1.4426950408889634f, c2 = c1 * 0.044715f;
    const float e = __builtin_amdgcn_exp2f(y * (c1 + c2 * y * y));
    return y * __builtin_amdgcn_rcpf(1.0f + e);
}
__device__ __forceinline__ unsigned cvt_pk_bf16(float lo, float hi) { unsigned r; asm("v_cvt_pk_bf16_f32 %0, %1, %2" : "=v"(r) : "v"(lo), "v"(hi)); return r; }

__device__ __forceinline__ void gemm_role(const Params& P, LAS unsigned char* lds, unsigned call, int wg) {
    const int tid = threadIdx.x, wid = rfl(tid >> 6), lane = tid & 63, wr = wid >> 2, wc = wid & 3, fr = lane & 15, fq = lane >> 4;
    char* ws = P.ws[P.rank];
    unsigned* tmo = (unsigned*)(ws + OFF_TMO);
    const rsrc_t wsr = mk_rsrc(ws, WS_USED);
    const int xcd = wg & 7, q = (wg - NPACK) >> 3, h = xcd & 1, blk = xcd >> 1;
    const int tm = (blk >> 1) * 4 + (q >> 2), tn = (blk & 1) * 4 + (q & 3), tile = tm * 8 + tn;
    const unsigned voff = (unsigned)tid * 16u, ldsw = (unsigned)wid * 1024u;
    const int aoff0 = (wr * 64 + fr) * 128 + ((fq ^ (fr & 7)) << 4), aoff1 = aoff0 ^ 64;
    const int boff0 = (wc * 32 + fr) * 128 + ((fq ^ (fr & 7)) << 4), boff1 = boff0 ^ 64;
#define G_SA(b, hh) (((b) * 2 + (hh)) * 16384)
#define G_SB(b, hh) ((4 + (b) * 2 + (hh)) * 16384)
#define G_STAGE(bufoff, soff) do { _Pragma("unroll") for (int _i = 0; _i < 2; ++_i) \
        __builtin_amdgcn_raw_ptr_buffer_load_lds(wsr, (LAS void*)(lds + (bufoff) + ldsw + _i * 8192), 16, voff, (soff) + _i * 8192, 0, AUX_SYS); } while (0)
#define G_LDA(dst, b, hh) do { _Pragma("unroll") for (int m = 0; m < 4; ++m) { \
        dst[m][0] = *(const LAS bf16x8*)(lds + G_SA(b, hh) + aoff0 + m * 2048); dst[m][1] = *(const LAS bf16x8*)(lds + G_SA(b, hh) + aoff1 + m * 2048); } } while (0)
#define G_LDB(dst, b, hh) do { _Pragma("unroll") for (int n = 0; n < 2; ++n) { \
        dst[n][0] = *(const LAS bf16x8*)(lds + G_SB(b, hh) + boff0 + n * 2048); dst[n][1] = *(const LAS bf16x8*)(lds + G_SB(b, hh) + boff1 + n * 2048); } } while (0)
#define G_MMA(ai, bj, At, Bt) do { __builtin_amdgcn_s_setprio(1); _Pragma("unroll") for (int m = 0; m < 4; ++m) _Pragma("unroll") for (int n = 0; n < 2; ++n) _Pragma("unroll") for (int k = 0; k < 2; ++k) \
        acc[ai][bj][m][n] = __builtin_amdgcn_mfma_f32_16x16x32_bf16(Bt[n][k], At[m][k], acc[ai][bj][m][n], 0, 0, 0); __builtin_amdgcn_s_setprio(0); } while (0)
#define G_WAIT_V(n) asm volatile("s_waitcnt vmcnt(" #n ")" ::: "memory")
#define G_WAIT_L(n) asm volatile("s_waitcnt lgkmcnt(" #n ")" ::: "memory")
#define G_BAR __builtin_amdgcn_s_barrier()
#define G_SCHED __builtin_amdgcn_sched_barrier(0)
#define G_GIDX(t) ((unsigned)((4 * h + ((t) & 3)) * 16 + ((t) >> 2)))
    const int zero = 0;
    for (int r = 0; r < 2; ++r) {
        const unsigned target = call * (r == 0 ? 64u : 32u);
        const unsigned abase = OFF_AIMG + (unsigned)r * (128u * 262144u) + (unsigned)tm * 32768u, bbase = OFF_BIMG + (unsigned)tn * 32768u;
        char* cbase = ws + OFF_CNT + r * 128 * 128;
        f32x4 acc[2][2][4][2];
#pragma unroll
        for (int a = 0; a < 2; ++a)
#pragma unroll
            for (int b = 0; b < 2; ++b)
#pragma unroll
                for (int m = 0; m < 4; ++m)
#pragma unroll
                    for (int n = 0; n < 2; ++n) acc[a][b][m][n] = (f32x4){0.f, 0.f, 0.f, 0.f};
        bf16x8 At[4][2], B0[2][2], B1[2][2];
        STAMP(ws, r * 8 + 0);
        wait_word((unsigned*)(cbase + G_GIDX(0) * 128), target, tmo, call);
        wait_word((unsigned*)(cbase + G_GIDX(1) * 128), target, tmo, call);
        wait_word((unsigned*)(cbase + G_GIDX(2) * 128), target, tmo, call);
        STAMP(ws, r * 8 + 1);
        {
            const unsigned a0 = abase + G_GIDX(0) * 262144u, b0 = bbase + G_GIDX(0) * 262144u, a1 = abase + G_GIDX(1) * 262144u, b1 = bbase + G_GIDX(1) * 262144u;
            G_STAGE(G_SB(0, 0), b0); G_STAGE(G_SA(0, 0), a0); G_STAGE(G_SB(0, 1), b0 + 16384); G_STAGE(G_SA(0, 1), a0 + 16384);
            if (wr == 1) G_BAR;
            G_WAIT_V(4); G_BAR;
            G_STAGE(G_SB(1, 0), b1); G_STAGE(G_SA(1, 0), a1); G_STAGE(G_SB(1, 1), b1 + 16384);
            G_WAIT_V(6); G_BAR;
        }
        unsigned fA = target, fB;
        for (int t = 0; t < 64; t += 2) {
            const int t2 = t + 2 < 64 ? t + 2 : 63, t3 = t + 3 < 64 ? t + 3 : 63, t4 = t + 4 < 64 ? t + 4 : 63;
            const unsigned g1 = G_GIDX(t + 1), g2 = G_GIDX(t2), g3 = G_GIDX(t3), g4 = G_GIDX(t4);
            const unsigned a1 = abase + g1 * 262144u, a2 = abase + g2 * 262144u, b2 = bbase + g2 * 262144u, a3 = abase + g3 * 262144u, b3 = bbase + g3 * 262144u;
            asm volatile("global_load_dword %0, %1, %2 sc0 sc1" : "=v"(fB) : "v"(zero), "s"(cbase + g3 * 128));
            G_LDB(B0, 0, 0); G_SCHED; G_LDA(At, 0, 0); G_STAGE(G_SA(1, 1), a1 + 16384);
            G_WAIT_L(8); G_BAR; G_WAIT_L(0); G_MMA(0, 0, At, B0); G_BAR; G_SCHED;
            asm volatile("" : "+v"(fA));
            if ((int)(rfl(fA) - target) < 0) wait_word((unsigned*)(cbase + g2 * 128), target, tmo, call);
            G_LDB(B1, 0, 1); G_STAGE(G_SB(0, 0), b2);
            G_BAR; G_WAIT_L(0); G_MMA(0, 1, At, B1); G_BAR;
            G_LDA(At, 0, 1); G_STAGE(G_SA(0, 0), a2);
            G_BAR; G_WAIT_L(0); G_MMA(1, 0, At, B0); G_BAR; G_SCHED;
            G_STAGE(G_SB(0, 1), b2 + 16384);
            G_WAIT_V(6); G_BAR; G_MMA(1, 1, At, B1); G_BAR;
            asm volatile("global_load_dword %0, %1, %2 sc0 sc1" : "=v"(fA) : "v"(zero), "s"(cbase + g4 * 128));
            G_LDB(B0, 1, 0); G_SCHED; G_LDA(At, 1, 0); G_STAGE(G_SA(0, 1), a2 + 16384);
            G_WAIT_L(8); G_BAR; G_WAIT_L(0); G_MMA(0, 0, At, B0); G_BAR; G_SCHED;
            asm volatile("" : "+v"(fB));
            if ((int)(rfl(fB) - target) < 0) wait_word((unsigned*)(cbase + g3 * 128), target, tmo, call);
            G_LDB(B1, 1, 1); G_STAGE(G_SB(1, 0), b3);
            G_BAR; G_WAIT_L(0); G_MMA(0, 1, At, B1); G_BAR;
            G_LDA(At, 1, 1); G_STAGE(G_SA(1, 0), a3);
            G_BAR; G_WAIT_L(0); G_MMA(1, 0, At, B0); G_BAR; G_SCHED;
            G_STAGE(G_SB(1, 1), b3 + 16384);
            G_WAIT_V(6); G_BAR; G_MMA(1, 1, At, B1); G_BAR;
        }
        G_WAIT_V(0);
        asm volatile("" : "+v"(fA));
        if (wr == 0) G_BAR;
        G_BAR;
        STAMP(ws, r * 8 + 2);
        int lane = threadIdx.x & 63;
        asm volatile("" : "+v"(lane));
        const int fr = lane & 15, fq = lane >> 4;
        if (h) {
#pragma unroll
            for (int b = 0; b < 2; ++b)
#pragma unroll
                for (int m = 0; m < 4; ++m)
#pragma unroll
                    for (int n = 0; n < 2; ++n) { const f32x4 tq = acc[0][b][m][n]; acc[0][b][m][n] = acc[1][b][m][n]; acc[1][b][m][n] = tq; }
        }
        const unsigned xl = (unsigned)((r * 64 + tile) * 2);
#pragma unroll
        for (int b = 0; b < 2; ++b)
#pragma unroll
            for (int m = 0; m < 4; ++m)
#pragma unroll
                for (int n = 0; n < 2; ++n)
                    __builtin_amdgcn_raw_buffer_store_b128(__builtin_bit_cast(u32x4, acc[1][b][m][n]), wsr, lane * 16 + ((b * 4 + m) * 2 + n) * 1024, OFF_XBUF + ((xl + h) * 8 + wid) * 16384u, AUX_SC1);
        WAIT_VM0();
        if (lane == 0) __hip_atomic_store((unsigned*)(ws + OFF_XFLAG + (xl + h) * 128 + wid * 16), call, __ATOMIC_RELAXED, __HIP_MEMORY_SCOPE_AGENT);
        wait_word((unsigned*)(ws + OFF_XFLAG + (xl + (h ^ 1)) * 128 + wid * 16), call, tmo, call);
        const bool dead = rfl(__hip_atomic_load(tmo, __ATOMIC_RELAXED, __HIP_MEMORY_SCOPE_AGENT)) == call;
        STAMP(ws, r * 8 + 3);
        LAS unsigned char* my = lds + wid * 8192;
#pragma unroll
        for (int b = 0; b < 2; ++b)
#pragma unroll
            for (int m = 0; m < 4; ++m) {
                const f32x4 p0 = __builtin_bit_cast(f32x4, __builtin_amdgcn_raw_buffer_load_b128(wsr, lane * 16 + ((b * 4 + m) * 2 + 0) * 1024, OFF_XBUF + ((xl + (h ^ 1)) * 8 + wid) * 16384u, AUX_SC1));
                const f32x4 p1 = __builtin_bit_cast(f32x4, __builtin_amdgcn_raw_buffer_load_b128(wsr, lane * 16 + ((b * 4 + m) * 2 + 1) * 1024, OFF_XBUF + ((xl + (h ^ 1)) * 8 + wid) * 16384u, AUX_SC1));
                const f32x4 v0 = acc[0][b][m][0] + p0, v1 = acc[0][b][m][1] + p1;
                u32x4 o;
                o.x = cvt_pk_bf16(gelu_f(v0[0]), gelu_f(v0[1])); o.y = cvt_pk_bf16(gelu_f(v0[2]), gelu_f(v0[3]));
                o.z = cvt_pk_bf16(gelu_f(v1[0]), gelu_f(v1[1])); o.w = cvt_pk_bf16(gelu_f(v1[2]), gelu_f(v1[3]));
                const int row = m * 16 + fr, ch = b * 4 + fq;
                *(LAS u32x4*)(my + row * 128 + ((ch ^ (row & 7)) * 16)) = o;
            }
        asm volatile("s_waitcnt lgkmcnt(0)" ::: "memory");
        u32x4 c[8];
#pragma unroll
        for (int j = 0; j < 8; ++j) { const int row = j * 8 + (lane >> 3); c[j] = *(LAS u32x4*)(my + row * 128 + (((lane & 7) ^ (row & 7)) * 16)); }
        if (dead) {
#pragma unroll
            for (int j = 0; j < 8; ++j) c[j] = (u32x4){0x7FC07FC0u, 0x7FC07FC0u, 0x7FC07FC0u, 0x7FC07FC0u};
        }
        {
            const rsrc_t outr = mk_rsrc(P.out, (unsigned)DIM * DIM * 2u);
            const unsigned row0 = (P.rank >> 2) * 4096 + r * 2048 + tm * 256 + h * 128 + wr * 64, col0 = (P.rank & 3) * 2048 + tn * 256 + wc * 64;
            const unsigned dso = (row0 * DIM + col0) * 2u, vo = (unsigned)(lane >> 3) * (DIM * 2u) + (unsigned)(lane & 7) * 16u;
            if (r == 0) {
#pragma unroll
                for (int j = 0; j < 8; ++j) __builtin_amdgcn_raw_buffer_store_b128(c[j], outr, vo, dso + j * (8u * DIM * 2u), AUX_SC1);
            } else {
#pragma unroll
                for (int j = 0; j < 8; ++j) __builtin_amdgcn_raw_buffer_store_b128(c[j], outr, vo, dso + j * (8u * DIM * 2u), OUT_AUX);
            }
            const unsigned o = (xl + h) * 8 + wid;
            if (r == 0) { WAIT_VM0(); signal_cnt(ws, OFF_OCNT + (o >> 3) * 128, lane); }
            else publish_block(P, wsr, c, 4096u + o, 1536u + o, 0xFFu & ~(1u << P.rank), call, lds, my, 1u + GROUNDS, 8u, wid, lane);
        }
        STAMP(ws, r * 8 + 4);
        asm volatile("s_waitcnt lgkmcnt(0)" ::: "memory");
        G_BAR;
    }
#undef G_SA
#undef G_SB
#undef G_STAGE
#undef G_LDA
#undef G_LDB
#undef G_MMA
#undef G_WAIT_V
#undef G_WAIT_L
#undef G_BAR
#undef G_SCHED
#undef G_GIDX
}

__global__ __launch_bounds__(512, 2) void gemm_allgather_kernel(Params P) {
    extern __shared__ __attribute__((aligned(16))) unsigned char shm[];
    LAS unsigned char* lds = (LAS unsigned char*)shm;
    char* ws = P.ws[P.rank];
    const int b = rfl(blockIdx.x);
    STAMP(ws, 60);
    unsigned call;
    if (b >= NPACK + NGEMM) { call = call_draw(ws); puller_role(P, lds, rfl(call), rfl(b - NPACK - NGEMM)); }
    else {
        if (threadIdx.x < 40) *(volatile LAS unsigned*)(lds + GCTL + 128 + threadIdx.x * 4) = (PUSH0 && b < 64 && threadIdx.x == 4) ? 1u : 0u;
        __syncthreads();
        const int wid = rfl(threadIdx.x >> 6);
        const bool gemm = b >= NPACK;
        const int next = gemm ? NFIRST + (b - NPACK) * 8 + wid : NFIRST + GROUNDS * NGEMM * 8 + b * 8 + wid;
        int first = next;
        if (wid < 4) {
            if (!PUSH0) first = b * 4 + wid;
            else if (b >= 64) first = 64 + (b - 64) * 4 + wid;
            else push_first(P, b, wid);
        }
        call = packer_role(P, lds, first, next, gemm ? NGEMM * 8 : NPACK * 8, gemm ? NFIRST + GROUNDS * NGEMM * 8 : 4096 + 1024);
        if (gemm) { __syncthreads(); gemm_role(P, lds, rfl(call), rfl(b)); }
    }
    STAMP(ws, 61);
    STAMP(ws, 58 + (call & 1));
}

}

extern "C" void kernel_launch(int rank, int nranks, ncclComm_t comm,
                              hipStream_t stream,
                              void* const* d_in, const int* in_sizes, int n_in,
                              void* d_out, int out_size,
                              const int* in_dtypes, int out_dtype,
                              void* const* d_out_all,
                              void* const* d_ws_all, size_t ws_bytes) {
    (void)comm; (void)in_sizes; (void)in_dtypes; (void)out_dtype; (void)d_out_all;
    if (nranks != gar::WORLD || n_in < 2 || ws_bytes < gar::WS_NEED || out_size != gar::DIM * gar::DIM || (unsigned)rank >= (unsigned)gar::WORLD) {
        fprintf(stderr, "unexpected problem shape\n");
        return;
    }
    static bool attr = false;
    if (!attr) {
        hipError_t e = hipFuncSetAttribute((const void*)gar::gemm_allgather_kernel, hipFuncAttributeMaxDynamicSharedMemorySize, gar::LDS_BYTES);
        if (e != hipSuccess) { fprintf(stderr, "hipFuncSetAttribute: %s\n", hipGetErrorString(e)); return; }
        attr = true;
    }
    gar::Params P;
    memset(&P, 0, sizeof P);
    P.x = d_in[0]; P.w = d_in[1]; P.out = d_out; P.rank = rank;
    for (int p = 0; p < gar::WORLD; ++p) P.ws[p] = (char*)d_ws_all[p];
    hipLaunchKernelGGL(gar::gemm_allgather_kernel, dim3(gar::GRID), dim3(512), gar::LDS_BYTES, stream, P);
    hipError_t e = hipGetLastError();
    if (e != hipSuccess) fprintf(stderr, "launch: %s\n", hipGetErrorString(e));
}
```

```cpp
#include <hip/hip_runtime.h>
#include <rccl/rccl.h>
#include <cstdio>
#include <cstdint>
#include <cstring>

namespace gar {

#define LAS __attribute__((address_space(3)))
typedef unsigned u32x4 __attribute__((ext_vector_type(4)));
typedef unsigned short u16x2 __attribute__((ext_vector_type(2)));
typedef short bf16x8 __attribute__((ext_vector_type(8)));
typedef float f32x4 __attribute__((ext_vector_type(4)));
typedef __amdgpu_buffer_rsrc_t rsrc_t;

constexpr int WORLD = 8;
constexpr int DIM = 8192;
constexpr int KSH = 1024;
constexpr int NGEMM = 128;
constexpr int GRID = 256;
#define PULL_WG_PER_LINK 14
#define PULL_DECODERS 7
constexpr int PWL = PULL_WG_PER_LINK;
constexpr int NDEC = PULL_DECODERS;
constexpr int NPULL = 7 * PWL;
constexpr int NPACK = GRID - NGEMM - NPULL;
#define NRAW 48
#define NRAWEND 40
#define WARM_TLB 1
#define JIT_LEAD 30
#define OUT_AUX 18
#define IMG_AUX 16
constexpr int NFIRST = (NPACK + NGEMM) * 4;
#define GEMM_PACK_ROUNDS 2
constexpr int GROUNDS = GEMM_PACK_ROUNDS;
constexpr int OUT2 = 4096 + NPACK * 4;
static_assert(NFIRST + GROUNDS * NGEMM * 8 <= 4096 && NFIRST % 8 == 0 && OUT2 % 8 == 0, "roles");
constexpr int NSEQ = 3584;
constexpr unsigned long long DEADLINE = 80000000ull;

constexpr unsigned OFF_TMO = 128;
constexpr unsigned OFF_LINK = 1024;
constexpr unsigned OFF_CNT = 4096;
constexpr unsigned OFF_XFLAG = OFF_CNT + 256 * 128;
constexpr unsigned OFF_OCNT = OFF_XFLAG + 256 * 128;
constexpr unsigned OFF_GRAN = 131072;
constexpr unsigned OFF_CALL = 640u << 10;
constexpr unsigned OFF_SLOT = 1u << 20;
constexpr unsigned OFF_AIMG = 64u << 20;
constexpr unsigned OFF_BIMG = 128u << 20;
constexpr unsigned OFF_XBUF = 160u << 20;
constexpr unsigned WS_USED = 192u << 20;
constexpr unsigned WS_NEED = WS_USED;
static_assert(OFF_OCNT + 128 * 128 <= OFF_GRAN, "map");
static_assert(OFF_GRAN + WORLD * NSEQ * 8 <= OFF_CALL && OFF_CALL + 2048 * 64 <= OFF_SLOT, "map");
static_assert(OFF_SLOT + 6144u * 8192u <= OFF_AIMG, "map");
constexpr int LDS_BYTES = 131072 + 512;

struct Params {
    const void* x; const void* w; void* out;
    char* ws[WORLD];
    int rank; int pad;
};

constexpr int AUX_SC1 = 16, AUX_SYS = 17;

__device__ __forceinline__ rsrc_t mk_rsrc(const void* p, unsigned bytes) {
    return __builtin_amdgcn_make_buffer_rsrc(const_cast<void*>(p), 0, bytes, 0x00020000);
}
__device__ __forceinline__ unsigned rfl(unsigned v) { return __builtin_amdgcn_readfirstlane(v); }
__device__ __forceinline__ unsigned bfi(unsigned mask, unsigned a, unsigned b) { return (a & mask) | (b & ~mask); }
__device__ __forceinline__ unsigned perm(unsigned hi, unsigned lo, unsigned sel) { return __builtin_amdgcn_perm(hi, lo, sel); }
#define WAIT_VM0() asm volatile("s_waitcnt vmcnt(0)" ::: "memory")

#define STAMP(ws, idx) do { } while (0)

__device__ __forceinline__ unsigned call_draw(char* ws) {
    unsigned t;
    const int zero = 0, one = 1;
    unsigned long long keep;
    asm volatile("s_mov_b64 %1, exec\n\ts_mov_b64 exec, 1\n\tglobal_atomic_add %0, %2, %3, %4 sc0\n\ts_mov_b64 exec, %1\n\ts_waitcnt vmcnt(0)"
                 : "=&v"(t), "=&s"(keep) : "v"(zero), "v"(one), "s"(ws + OFF_CALL + (blockIdx.x * 8 + rfl(threadIdx.x >> 6)) * 64) : "memory");
    return rfl(t) + 1u;
}

__device__ __forceinline__ bool give_up(unsigned long long t0, unsigned* tmo, unsigned call) {
    if (rfl(__hip_atomic_load(tmo, __ATOMIC_RELAXED, __HIP_MEMORY_SCOPE_AGENT)) == call) return true;
    if (__builtin_amdgcn_s_memrealtime() - t0 > DEADLINE) { __hip_atomic_store(tmo, call, __ATOMIC_RELAXED, __HIP_MEMORY_SCOPE_AGENT); return true; }
    return false;
}
__device__ __forceinline__ void wait_word(unsigned* p, unsigned target, unsigned* tmo, unsigned call) {
    if ((int)(rfl(__hip_atomic_load(p, __ATOMIC_RELAXED, __HIP_MEMORY_SCOPE_SYSTEM)) - target) >= 0) return;
    const unsigned long long t0 = __builtin_amdgcn_s_memrealtime();
    while ((int)(rfl(__hip_atomic_load(p, __ATOMIC_RELAXED, __HIP_MEMORY_SCOPE_SYSTEM)) - target) < 0) {
        if (give_up(t0, tmo, call)) break;
        __builtin_amdgcn_s_sleep(4);
    }
}

constexpr unsigned MAX_EXC = 31;
constexpr unsigned RAWB = 3584, AUX_U = 20, AUX_MAX = 2560, U_MAX = 10240, LANE_U_MAX = 224;
constexpr unsigned ENC_HIST = AUX_MAX + 16, ENC_SORT = ENC_HIST + 256, ENC_END = ENC_SORT + 128;

__device__ __forceinline__ unsigned bfi_c(unsigned mask, unsigned a, unsigned b) { unsigned r; asm("v_bfi_b32 %0, %1, %2, %3" : "=v"(r) : "s"(mask), "v"(a), "v"(b)); return r; }
__device__ __forceinline__ unsigned opaque(unsigned v) { asm("" : "+v"(v)); return v; }
__device__ __forceinline__ unsigned bytemask(unsigned x, int sh, unsigned c0c) {
    unsigned t; asm("v_and_or_b32 %0, %1, %2, %3" : "=v"(t) : "v"(x >> sh), "s"(0x01010101u), "v"(c0c));
    return perm(0u, 0u, t);
}
template <int CTRL, int RM = 0xf> __device__ __forceinline__ unsigned dpp0(unsigned v) { return (unsigned)__builtin_amdgcn_update_dpp(0, (int)v, CTRL, RM, 0xf, RM == 0xf); }
__device__ __forceinline__ unsigned wave_scan_incl(unsigned v) {
    v += dpp0<0x111>(v); v += dpp0<0x112>(v); v += dpp0<0x114>(v); v += dpp0<0x118>(v);
    v += dpp0<0x142, 0xa>(v); v += dpp0<0x143, 0xc>(v);
    return v;
}
__device__ __forceinline__ unsigned umin(unsigned a, unsigned b) { return a < b ? a : b; }
__device__ __forceinline__ unsigned umax(unsigned a, unsigned b) { return a > b ? a : b; }
__device__ __forceinline__ unsigned sad4(unsigned x) { return __builtin_amdgcn_sad_u8(x, 0u, 0u); }
__device__ __forceinline__ unsigned lut2x16(const unsigned (&T)[8], unsigned idx, unsigned ms, unsigned c0c) {
    const unsigned sel = idx & 0x07070707u, m8 = bytemask(idx, 3, c0c);
    const unsigned p = bfi(m8, perm(T[3], T[2], sel), perm(T[1], T[0], sel)), n = bfi(m8, perm(T[7], T[6], sel), perm(T[5], T[4], sel));
    return bfi(ms, n, p);
}
#define LDS_FENCE() asm volatile("" ::: "memory")
__device__ __forceinline__ void lds_or2(LAS unsigned char* scr, unsigned p, unsigned w) {
    const unsigned s = p & 31u;
    LAS unsigned* q = (LAS unsigned*)(scr + ((p >> 5) << 2));
    __hip_atomic_fetch_or(q, w << s, __ATOMIC_RELAXED, __HIP_MEMORY_SCOPE_WORKGROUP);
    __hip_atomic_fetch_or(q + 1, (w >> 1) >> (31u - s), __ATOMIC_RELAXED, __HIP_MEMORY_SCOPE_WORKGROUP);
}

__device__ __forceinline__ unsigned encode_block(const u32x4 (&c)[8], unsigned (&P)[14], u32x4 (&a)[3], LAS unsigned char* scr, int lane) {
    asm volatile("" : "+v"(lane));
    const unsigned c0c = opaque(0x0C0C0C0Cu);
    unsigned E[16], R[16];
    u16x2 p1 = {0, 0}, p2 = {0, 0}, n1 = {0, 0}, n2 = {0, 0};
#pragma unroll
    for (int g = 0; g < 16; ++g) {
        const unsigned x = (g & 1) ? c[g >> 1].z : c[g >> 1].x, y = (g & 1) ? c[g >> 1].w : c[g >> 1].y;
        const unsigned H = perm(y, x, 0x07050301u), L = perm(y, x, 0x06040200u);
        E[g] = bfi_c(0xFEFEFEFEu, H << 1, L >> 7);
        R[g] = bfi_c(0x80808080u, L << 1, bfi_c(0x40404040u, H >> 1, L));
        const unsigned en = E[g] & bytemask(H, 7, c0c), ep = E[g] ^ en;
        p1 = __builtin_elementwise_max(p1, __builtin_bit_cast(u16x2, ep)); p2 = __builtin_elementwise_max(p2, __builtin_bit_cast(u16x2, ep << 8));
        n1 = __builtin_elementwise_max(n1, __builtin_bit_cast(u16x2, en)); n2 = __builtin_elementwise_max(n2, __builtin_bit_cast(u16x2, en << 8));
        __builtin_amdgcn_sched_barrier(0);
    }
    const u32x4 z4 = {0u, 0u, 0u, 0u};
    *(LAS u32x4*)(scr + lane * 16) = z4; *(LAS u32x4*)(scr + 1024 + lane * 16) = z4;
    if (lane < 49) *(LAS u32x4*)(scr + 2048 + lane * 16) = z4;
    unsigned baseP, baseN;
    {
        const u16x2 mp = __builtin_elementwise_max(p1, p2), mn = __builtin_elementwise_max(n1, n2);
        u16x2 m = {(unsigned short)((mp.x > mp.y ? mp.x : mp.y) >> 8), (unsigned short)((mn.x > mn.y ? mn.x : mn.y) >> 8)};
        m = __builtin_elementwise_max(m, __builtin_bit_cast(u16x2, dpp0<0x128>(__builtin_bit_cast(unsigned, m)))); m = __builtin_elementwise_max(m, __builtin_bit_cast(u16x2, dpp0<0x124>(__builtin_bit_cast(unsigned, m))));
        m = __builtin_elementwise_max(m, __builtin_bit_cast(u16x2, dpp0<0x122>(__builtin_bit_cast(unsigned, m)))); m = __builtin_elementwise_max(m, __builtin_bit_cast(u16x2, dpp0<0x121>(__builtin_bit_cast(unsigned, m))));
        const unsigned mu = __builtin_bit_cast(unsigned, m);
        const u16x2 r01 = __builtin_elementwise_max(__builtin_bit_cast(u16x2, (unsigned)__builtin_amdgcn_readlane(mu, 0)), __builtin_bit_cast(u16x2, (unsigned)__builtin_amdgcn_readlane(mu, 16)));
        const u16x2 r23 = __builtin_elementwise_max(__builtin_bit_cast(u16x2, (unsigned)__builtin_amdgcn_readlane(mu, 32)), __builtin_bit_cast(u16x2, (unsigned)__builtin_amdgcn_readlane(mu, 48)));
        const unsigned mm = rfl(__builtin_bit_cast(unsigned, __builtin_elementwise_max(r01, r23)));
        baseP = mm & 0xFFu; baseN = mm >> 16;
    }
    const unsigned BP = baseP * 0x01010101u, BN = baseN * 0x01010101u;
    unsigned any = 0, D[16];
#pragma unroll
    for (int g = 0; g < 16; ++g) {
        E[g] = bfi(bytemask(R[g], 6, c0c), BN, BP) - E[g]; any |= E[g];
        D[g] = bfi_c(0x1E1E1E1Eu, E[g] << 1, (R[g] >> 7) & 0x01010101u);
        R[g] &= 0x7F7F7F7Fu;
    }
    LDS_FENCE();
#pragma unroll
    for (int gi = 0; gi < 4; ++gi) {
        const int g = gi * 5;
        const unsigned k4 = ((D[g] & 0x0F0F0F0Fu) | ((R[g] >> 2) & 0x10101010u) | ((D[g] << 1) & 0x20202020u)) << 2;
#pragma unroll
        for (int b = 0; b < 4; ++b)
            __hip_atomic_fetch_add((LAS unsigned*)(scr + ENC_HIST + ((k4 >> (8 * b)) & 0xFFu)), 1u, __ATOMIC_RELAXED, __HIP_MEMORY_SCOPE_WORKGROUP);
    }
    LDS_FENCE();
    const unsigned cnt = *(LAS unsigned*)(scr + ENC_HIST + lane * 4);
    const unsigned key = (cnt << 4) | (15u - (lane & 15));
    unsigned rank = 0;
    rank += dpp0<0x121>(key) > key; rank += dpp0<0x122>(key) > key; rank += dpp0<0x123>(key) > key; rank += dpp0<0x124>(key) > key; rank += dpp0<0x125>(key) > key;
    rank += dpp0<0x126>(key) > key; rank += dpp0<0x127>(key) > key; rank += dpp0<0x128>(key) > key; rank += dpp0<0x129>(key) > key; rank += dpp0<0x12A>(key) > key;
    rank += dpp0<0x12B>(key) > key; rank += dpp0<0x12C>(key) > key; rank += dpp0<0x12D>(key) > key; rank += dpp0<0x12E>(key) > key; rank += dpp0<0x12F>(key) > key;
    unsigned tail = cnt; tail += dpp0<0x128>(tail); tail += dpp0<0x124>(tail); tail += dpp0<0x122>(tail); tail += dpp0<0x121>(tail);
    const int sg = (lane >> 4) & 1;
    const unsigned ct = sg ? __builtin_amdgcn_readlane(tail, 48) : __builtin_amdgcn_readlane(tail, 32);
    if (lane < 32) *(LAS unsigned*)(scr + ENC_SORT + ((lane & 16) + rank) * 4) = cnt;
    LDS_FENCE();
    unsigned best;
    {
        const unsigned cb = lane & 15, s0 = cb / 3u, s1 = cb - 3u * s0, A = 1u << s0, AB = A + (1u << s1);
        unsigned cost = ct * (16u - AB);
#pragma unroll
        for (int q = 0; q < 4; ++q) {
            const u32x4 h = *(LAS u32x4*)(scr + ENC_SORT + sg * 64 + q * 16);
#pragma unroll
            for (int j = 0; j < 4; ++j) {
                const unsigned r = q * 4 + j, len = r < A ? 1u + s0 : r < AB ? 2u + s1 : 4u + ((r - AB) >> 1);
                cost += 2u * len * h[j];
            }
        }
        best = cb < 9 ? (cost << 4) | cb : 0xFFFFFFFFu;
        best = umin(best, dpp0<0x128>(best)); best = umin(best, dpp0<0x124>(best)); best = umin(best, dpp0<0x122>(best)); best = umin(best, dpp0<0x121>(best));
    }
    const unsigned cbP = __builtin_amdgcn_readlane(best, 0) & 15u, cbN = __builtin_amdgcn_readlane(best, 16) & 15u;
    const unsigned s0P = cbP / 3u, s1P = cbP - 3u * s0P, s0N = cbN / 3u, s1N = cbN - 3u * s0N;
    const unsigned ABP = (1u << s0P) + (1u << s1P), ABN = (1u << s0N) + (1u << s1N);
    unsigned F[8];
    {
        const unsigned s0 = sg ? s0N : s0P, s1 = sg ? s1N : s1P, A = 1u << s0, AB = sg ? ABN : ABP;
        const unsigned ent = rank < A ? (rank << 4) | (s0 << 6) : rank < AB ? 1u | ((rank - A) << 4) | (s1 << 6) : (2u + ((rank - AB) >> 1)) | (((rank - AB) & 1u) << 4) | (1u << 6);
        unsigned fv = ent << (8 * (lane & 3));
        fv |= dpp0<0xB1>(fv); fv |= dpp0<0x4E>(fv);
#pragma unroll
        for (int q = 0; q < 8; ++q) F[q] = __builtin_amdgcn_readlane(fv, q * 4);
        if (lane < 32)
            __hip_atomic_fetch_or((LAS unsigned*)(scr + (sg * 2 + (rank >> 3)) * 4), (unsigned)(lane & 15) << (8 * (rank & 3) + 4 * ((rank >> 2) & 1)), __ATOMIC_RELAXED, __HIP_MEMORY_SCOPE_WORKGROUP);
        if (lane == 0) *(LAS unsigned*)(scr + 16) = baseP | (baseN << 8) | (s0P << 16) | (s1P << 18) | (s0N << 20) | (s1N << 22);
    }
    const unsigned ABP4 = ABP * 0x01010101u, ABN4 = ABN * 0x01010101u, S0P4 = s0P * 0x01010101u, S0N4 = s0N * 0x01010101u;
    unsigned KK[16], XW[4], OX[4], TU = 0, TX = 0, bad = 0, xl = 0, xw = 0;
#pragma unroll
    for (int g = 0; g < 16; ++g) {
        const unsigned ms = bytemask(R[g], 6, c0c), m16 = bytemask(D[g], 4, c0c), ent = lut2x16(F, D[g], ms, c0c);
        const unsigned q = ((D[g] | 0x80808080u) - bfi(ms, ABN4, ABP4)) & 0x7F7F7F7Fu;
        unsigned K = bfi(m16, ((q >> 1) & 0x3F3F3F3Fu) + 0x02020202u, ent & 0x0F0F0F0Fu);
        unsigned SUF = bfi(m16, q & 0x01010101u, (ent >> 4) & 0x03030303u);
        unsigned S = bfi(m16, 0x01010101u, (ent >> 6) & 0x03030303u);
        unsigned gl = sad4(K) + 4u;
        if (__ballot(gl > 31u) != 0ull) {
            any |= 0x80u;
            const unsigned km = umax(umax(K & 0xFFu, (K >> 8) & 0xFFu), umax((K >> 16) & 0xFFu, K >> 24));
            const unsigned eq = gl > 31u ? (((K ^ (km * 0x01010101u)) + 0x7F7F7F7Fu) & 0x80808080u) ^ 0x80808080u : 0u, em = bytemask(eq, 7, c0c);
            E[g] |= eq;
            K &= ~em; SUF &= ~em; S = bfi(em, bfi(ms, S0N4, S0P4), S);
            gl = sad4(K) + 4u;
        }
        KK[g] = K; TU += gl; bad = umax(bad, gl);
        xw |= (SUF & 0xFFu) << xl; xl += S & 0xFFu;
        xw |= ((SUF >> 8) & 0xFFu) << xl; xl += (S >> 8) & 0xFFu;
        xw |= ((SUF >> 16) & 0xFFu) << xl; xl += (S >> 16) & 0xFFu;
        xw |= (SUF >> 24) << (xl & 31u); xl += S >> 24;
        if ((g & 3) == 3) { XW[g >> 2] = xw; OX[g >> 2] = TX; TX += xl; xw = 0; xl = 0; }
        __builtin_amdgcn_sched_barrier(0);
    }
    bad = (bad > 31u) | (TU > LANE_U_MAX);
    const unsigned tt = TU | (TX << 16), incl = wave_scan_incl(tt), tot = __builtin_amdgcn_readlane(incl, 63), totU = tot & 0xFFFFu, totX = tot >> 16;
    const unsigned xstart = (AUX_U * 8u + totU + 31u) & ~31u, endbit = xstart + totX;
    unsigned raw = (__ballot(bad != 0) != 0ull) || totU > U_MAX || endbit > (AUX_MAX - 8u) * 8u;
    if (!raw) {
        const unsigned px = xstart + ((incl - tt) >> 16);
        unsigned p = AUX_U * 8u + ((incl - tt) & 0xFFFFu);
#pragma unroll
        for (int g = 0; g < 16; ++g) {
            const unsigned ln = KK[g] + 0x01010101u;
            unsigned pos = (ln & 0xFFu) - 1u, w = 1u << pos;
            pos += (ln >> 8) & 0xFFu; w |= 1u << pos;
            pos += (ln >> 16) & 0xFFu; w |= 1u << pos;
            pos += ln >> 24; w |= 1u << pos;
            lds_or2(scr, p, w); p += pos + 1u;
            __builtin_amdgcn_sched_barrier(0);
        }
#pragma unroll
        for (int q = 0; q < 4; ++q) lds_or2(scr, px + OX[q], XW[q]);
    }
#pragma unroll
    for (int i = 0; i < 7; ++i) { P[i] = R[i] | ((R[14] << (7 - i)) & 0x80808080u); P[7 + i] = R[7 + i] | ((R[15] << (7 - i)) & 0x80808080u); }
    unsigned n = 0;
    const unsigned xo = (endbit + 31u) >> 5;
    if (!raw && __ballot((any & 0xF0F0F0F0u) != 0) != 0ull) {
#pragma unroll
        for (int g = 0; g < 16; ++g) {
            const unsigned f = E[g] & 0xF0F0F0F0u;
            unsigned long long mask = __ballot(f != 0);
            while (mask != 0ull && n <= MAX_EXC) {
                const int L = __builtin_ctzll(mask); mask &= mask - 1;
                const unsigned fl = __builtin_amdgcn_readlane(f, L);
                const unsigned lo = __builtin_amdgcn_readlane((g & 1) ? c[g >> 1].z : c[g >> 1].x, L);
                const unsigned hi = __builtin_amdgcn_readlane((g & 1) ? c[g >> 1].w : c[g >> 1].y, L);
                const bool e = lane < 4 && ((fl >> (8 * (lane & 3))) & 0xFFu) != 0;
                const unsigned long long em = __ballot(e);
                const unsigned slot = xo + n + __builtin_popcountll(em & ((1ull << lane) - 1ull));
                const unsigned val = (((lane & 2) ? hi : lo) >> (16 * (lane & 1))) & 0xFFFFu;
                if (e && slot * 4u < AUX_MAX) *(LAS unsigned*)(scr + slot * 4u) = (unsigned)(L * 64 + g * 4 + (lane & 3)) | (val << 12);
                n += __builtin_popcountll(em);
            }
        }
    }
    raw |= (n > MAX_EXC) | ((xo + n) * 4u > AUX_MAX);
    LDS_FENCE();
    a[0] = *(LAS u32x4*)(scr + lane * 16); a[1] = *(LAS u32x4*)(scr + 1024 + lane * 16); a[2] = *(LAS u32x4*)(scr + 2048 + (lane & 31) * 16);
    return ((xo + n + 3u) >> 2) | (n << 8) | (xo << 16) | (raw << 31);
}

__device__ __forceinline__ void decode_block(LAS unsigned char* src, LAS unsigned char* pos_scr, u32x4 (&c)[8], int lane) {
    LAS unsigned char* aux = src + RAWB;
    const unsigned c0c = opaque(0x0C0C0C0Cu);
    const u32x4 nt = *(LAS const u32x4*)(aux);
    const unsigned prm_v = *(LAS const unsigned*)(aux + 16);
    unsigned u[5];
#pragma unroll
    for (int i = 0; i < 5; ++i) u[i] = *(LAS const unsigned*)(aux + AUX_U + lane * 20 + i * 4);
    u32x4 P4[3];
#pragma unroll
    for (int p = 0; p < 3; ++p) P4[p] = *(LAS const u32x4*)(src + p * 1024 + lane * 16);
    const unsigned long long ph = *(LAS const unsigned long long*)(src + 3072 + lane * 8);
    const unsigned c0 = __builtin_popcount(u[0]), c1 = c0 + __builtin_popcount(u[1]), c2 = c1 + __builtin_popcount(u[2]), c3 = c2 + __builtin_popcount(u[3]), cn = c3 + __builtin_popcount(u[4]);
    const unsigned C = wave_scan_incl(cn) - cn;
    if (lane == 0) *(LAS unsigned*)(pos_scr) = AUX_U * 8u;
    const int rounds = __ballot(cn > 128u) != 0ull ? 3 : 2;
    for (int j = 0; j < rounds; ++j) {
        const unsigned t = (C >> 6) + 1u + j, kth = 64u * t - C;
        const unsigned i = (kth > c0) + (kth > c1) + (kth > c2) + (kth > c3);
        unsigned rem = kth - (i == 0 ? 0u : i == 1 ? c0 : i == 2 ? c1 : i == 3 ? c2 : c3), w = i == 0 ? u[0] : i == 1 ? u[1] : i == 2 ? u[2] : i == 3 ? u[3] : u[4], pos = 0;
#pragma unroll
        for (int st = 16; st >= 1; st >>= 1) {
            const unsigned lo = __builtin_popcount(w & ((1u << st) - 1u));
            const bool up = rem > lo;
            rem -= up ? lo : 0u; w = up ? w >> st : w; pos += up ? st : 0;
        }
        if (kth <= cn && t < 64u) *(LAS unsigned*)(pos_scr + t * 4) = AUX_U * 8u + lane * 160u + i * 32u + pos + 1u;
    }
    LDS_FENCE();
    const unsigned p0 = *(LAS const unsigned*)(pos_scr + lane * 4);
    unsigned S[8];
    {
        LAS const unsigned* q = (LAS const unsigned*)(aux + ((p0 >> 5) << 2));
#pragma unroll
        for (int i = 0; i < 8; ++i) S[i] = q[i];
#pragma unroll
        for (int i = 0; i < 7; ++i) S[i] = __builtin_amdgcn_alignbit(S[i + 1], S[i], p0 & 31u);
    }
    unsigned R[16];
    {
        const unsigned P[14] = {P4[0].x, P4[0].y, P4[0].z, P4[0].w, P4[1].x, P4[1].y, P4[1].z, P4[1].w, P4[2].x, P4[2].y, P4[2].z, P4[2].w, (unsigned)ph, (unsigned)(ph >> 32)};
        unsigned x = 0, y = 0;
#pragma unroll
        for (int i = 0; i < 7; ++i) { x |= (P[i] >> (7 - i)) & (0x01010101u << i); y |= (P[7 + i] >> (7 - i)) & (0x01010101u << i); }
#pragma unroll
        for (int i = 0; i < 14; ++i) R[i] = P[i] & 0x7F7F7F7Fu;
        R[14] = x; R[15] = y;
    }
    const unsigned n0 = rfl(nt.x), n1 = rfl(nt.y), n2 = rfl(nt.z), n3 = rfl(nt.w), prm = rfl(prm_v);
    const unsigned I[8] = {n0 & 0x0F0F0F0Fu, (n0 >> 4) & 0x0F0F0F0Fu, n1 & 0x0F0F0F0Fu, (n1 >> 4) & 0x0F0F0F0Fu, n2 & 0x0F0F0F0Fu, (n2 >> 4) & 0x0F0F0F0Fu, n3 & 0x0F0F0F0Fu, (n3 >> 4) & 0x0F0F0F0Fu};
    const unsigned s0P = (prm >> 16) & 3u, s1P = (prm >> 18) & 3u, s0N = (prm >> 20) & 3u, s1N = (prm >> 22) & 3u;
    const unsigned STP = (s0P << 8) | (s1P << 16) | (1u << 24), STN = (s0N << 8) | (s1N << 16) | (1u << 24);
    const unsigned OTP = (4u << 8) | (((1u << s0P) + 2u) << 16) | (((1u << s0P) + (1u << s1P)) << 24);
    const unsigned OTN = (4u << 8) | (((1u << s0N) + 2u) << 16) | (((1u << s0N) + (1u << s1N)) << 24);
    const unsigned BP = (prm & 0xFFu) * 0x01010101u, BN = ((prm >> 8) & 0xFFu) * 0x01010101u;
    unsigned RB[16], SS[16], TU = 0, TX = 0;
#pragma unroll
    for (int g = 0; g < 16; ++g) {
        const unsigned w0 = S[0], q0 = __builtin_ctz(w0), w1 = w0 & (w0 - 1u), q1 = __builtin_ctz(w1), w2 = w1 & (w1 - 1u), q2 = __builtin_ctz(w2), w3 = w2 & (w2 - 1u), q3 = __builtin_ctz(w3);
        const unsigned Q = q0 | (q1 << 8) | (q2 << 16) | (q3 << 24);
        const unsigned LN = Q - opaque((Q << 8) - 1u), sh = q3 + 1u;
        TU += sh;
        if (g < 15) {
#pragma unroll
            for (int i = 0; i < 7; ++i) if (32 * i < (int)LANE_U_MAX - 4 * (g + 1)) S[i] = (i < 6 && 32 * (i + 1) < (int)LANE_U_MAX - 4 * g) ? __builtin_amdgcn_alignbit(S[i + 1], S[i], sh) : S[i] >> sh;
        }
        const unsigned sel = bfi_c(0x04040404u, R[g] >> 4, bfi(bytemask(LN + 0x7D7D7D7Du, 7, c0c), 0x03030303u, LN));
        SS[g] = perm(STN, STP, sel);
        RB[g] = perm(OTN, OTP, sel) + (LN << 1);
        TX = __builtin_amdgcn_sad_u8(SS[g], 0u, TX);
    }
    const unsigned xstart = (__builtin_amdgcn_readlane(p0 + TU, 63) + 31u) & ~31u, px = xstart + wave_scan_incl(TX) - TX;
    unsigned X[5];
    {
        LAS const unsigned* q = (LAS const unsigned*)(aux + ((px >> 5) << 2));
#pragma unroll
        for (int i = 0; i < 5; ++i) X[i] = q[i];
#pragma unroll
        for (int i = 0; i < 4; ++i) X[i] = __builtin_amdgcn_alignbit(X[i + 1], X[i], px & 31u);
    }
    unsigned xoff = 0;
#pragma unroll
    for (int g = 0; g < 16; ++g) {
        const unsigned s = SS[g], M = perm(0u, 0x00030100u, s);
        const unsigned y = (g & 1) ? X[0] >> xoff : X[0], Y = perm(y, y, 0x01000100u);
        const unsigned t1 = opaque(s << 8), t2 = opaque((t1 << 8) + t1), Cs = (t2 << 16) + t2;
        const u16x2 z02 = __builtin_bit_cast(u16x2, Y) >> __builtin_bit_cast(u16x2, Cs & 0x00FF00FFu);
        const u16x2 z13 = __builtin_bit_cast(u16x2, Y) >> __builtin_bit_cast(u16x2, perm(0u, Cs, 0x0C030C01u));
        const unsigned suf = perm(__builtin_bit_cast(unsigned, z13), __builtin_bit_cast(unsigned, z02), 0x06020400u) & M;
        if (g & 1) {
            if (g < 15) {
                const unsigned t = __builtin_amdgcn_sad_u8(s, 0u, xoff);
#pragma unroll
                for (int i = 0; i < 4; ++i) if (32 * i < 8 * (15 - g)) X[i] = (i < 3 && 32 * (i + 1) < 8 * (17 - g)) ? __builtin_amdgcn_alignbit(X[i + 1], X[i], t) : X[i] >> t;
            }
        } else xoff = sad4(s);
        const unsigned rank = RB[g] + suf - 0x06060606u, ms = bytemask(R[g], 6, c0c);
        const unsigned D = bfi(bytemask(rank, 4, c0c), rank, lut2x16(I, rank, ms, c0c));
        const unsigned E = bfi(ms, BN, BP) - ((D >> 1) & 0x0F0F0F0Fu);
        const unsigned H = bfi_c(0x80808080u, R[g] << 1, E >> 1), L = bfi_c(0x3F3F3F3Fu, R[g], bfi_c(0x40404040u, D << 6, E << 7));
        const unsigned x = perm(H, L, 0x05010400u), yv = perm(H, L, 0x07030602u);
        if (g & 1) { c[g >> 1].z = x; c[g >> 1].w = yv; } else { c[g >> 1].x = x; c[g >> 1].y = yv; }
    }
}

typedef unsigned u32x2 __attribute__((ext_vector_type(2)));
__device__ __forceinline__ void store_granule(char* p, u32x2 g) {
    asm volatile("global_store_dwordx2 %0, %1, off sc0 sc1\n\ts_nop 1" :: "v"(p), "v"(g) : "memory");
}

constexpr unsigned GCTL = 131072;
__device__ __forceinline__ unsigned payload_bytes(unsigned h) { return (h >> 31) ? 8192u : RAWB + ((h & 255u) << 4); }
__device__ __forceinline__ void publish_block(const Params& P, rsrc_t wsr, const u32x4 (&c)[8], unsigned slot, unsigned n, unsigned dmask, unsigned call, LAS unsigned char* lds, LAS unsigned char* scr,
                                              unsigned j, unsigned gsize, int wid, int lane) {
    unsigned R[14], h = 1u << 31; u32x4 a[3];
    if (n >= NRAW && n < NSEQ - NRAWEND) h = encode_block(c, R, a, scr, lane);
    const unsigned u = j & 3u;
    volatile LAS unsigned* ctl = (volatile LAS unsigned*)(lds + GCTL);
    while (j >= 4u && (int)(ctl[36 + u] - (j - 3u)) < 0) __builtin_amdgcn_s_sleep(1);
    unsigned so = OFF_SLOT + (slot & ~1u) * 8192u;
    if (wid & 1) {
        unsigned v;
        while (((v = ctl[40 + u * 8 + wid - 1]) >> 16) != ((j + 1u) & 0xFFFFu)) __builtin_amdgcn_s_sleep(1);
        so += (rfl(v) & 0xFFFFu) << 4;
    } else if (lane == 0) ctl[40 + u * 8 + wid] = ((j + 1u) << 16) | (payload_bytes(h) >> 4);
    if (h >> 31) {
#pragma unroll
        for (int q = 0; q < 8; ++q) __builtin_amdgcn_raw_buffer_store_b128(c[q], wsr, lane * 16 + q * 1024, so, AUX_SYS);
    } else {
        const unsigned n16 = h & 255u;
#pragma unroll
        for (int q = 0; q < 3; ++q) __builtin_amdgcn_raw_buffer_store_b128((u32x4){R[4 * q], R[4 * q + 1], R[4 * q + 2], R[4 * q + 3]}, wsr, lane * 16 + q * 1024, so, AUX_SYS);
        __builtin_amdgcn_raw_buffer_store_b64((u32x2){R[12], R[13]}, wsr, lane * 8 + 3072, so, AUX_SYS);
        if ((unsigned)lane < n16) __builtin_amdgcn_raw_buffer_store_b128(a[0], wsr, lane * 16 + RAWB, so, AUX_SYS);
        if ((unsigned)lane + 64u < n16) __builtin_amdgcn_raw_buffer_store_b128(a[1], wsr, lane * 16 + RAWB + 1024, so, AUX_SYS);
        if ((unsigned)lane + 128u < n16) __builtin_amdgcn_raw_buffer_store_b128(a[2], wsr, lane * 16 + RAWB + 2048, so, AUX_SYS);
    }
    WAIT_VM0();
    unsigned arrived = 0;
    if (lane == 0) {
        ctl[u * 8 + wid] = h;
        arrived = __hip_atomic_fetch_add((LAS unsigned*)(lds + GCTL) + 32 + u, 1u, __ATOMIC_RELAXED, __HIP_MEMORY_SCOPE_WORKGROUP);
    }
    if (rfl(arrived) == gsize - 1u) {
        const unsigned d = lane >> 3, w = lane & 7, hw = ctl[u * 8 + w];
        if (w < gsize && ((dmask >> d) & 1u)) store_granule(P.ws[d] + OFF_GRAN + ((unsigned)P.rank * NSEQ + n - wid + w) * 8u, (u32x2){call, hw});
        if (lane == 0) { ctl[32 + u] = 0u; ctl[36 + u] = j + 1u; }
    }
}

__device__ __forceinline__ void signal_cnt(char* ws, unsigned off, int lane) {
    if (off && lane == 0) __hip_atomic_fetch_add((unsigned*)(ws + off), 1u, __ATOMIC_RELAXED, __HIP_MEMORY_SCOPE_SYSTEM);
}
__device__ __forceinline__ unsigned packer_role(const Params& P, LAS unsigned char* lds, int first, int next, int step, int end) {
    unsigned call = 0;
    const int lane = threadIdx.x & 63, wid = rfl(threadIdx.x >> 6);
    char* ws = P.ws[P.rank];
    const rsrc_t wsr = mk_rsrc(ws, WS_USED);
    const rsrc_t xr = mk_rsrc(P.x, DIM * KSH * 2), wr = mk_rsrc(P.w, DIM * KSH * 2);
    const int myi = P.rank >> 2, myj = P.rank & 3;
    LAS unsigned char* my = lds + wid * 8192;
    unsigned j = first < NFIRST ? 0u : 1u;
    const unsigned long long tstart = __builtin_amdgcn_s_memrealtime();
    for (int p = first; p < end; ++j) {
        if (p < 4096 && end > 4096) {
            const int due = (p < 3072 ? (p / 192) * 580 : 9280 + ((p - 3072) >> 6) * 290) - JIT_LEAD * 100;
            while ((int)(__builtin_amdgcn_s_memrealtime() - tstart) < due) __builtin_amdgcn_s_sleep(16);
        }
        u32x4 c[8];
        unsigned slot, n, dmask, selfcnt = 0;
        if (p >= 4096) {
            const unsigned o = p - 4096, w = o & 7, h = (o >> 3) & 1, tile = (o >> 4) & 63;
            if (call == 0) call = call_draw(ws);
            unsigned* tmo = (unsigned*)(ws + OFF_TMO);
            wait_word((unsigned*)(ws + OFF_OCNT + (o >> 3) * 128), call * 8u, tmo, call);
            const unsigned row0 = (P.rank >> 2) * 4096 + (tile >> 3) * 256 + h * 128 + (w >> 2) * 64, col0 = (P.rank & 3) * 2048 + (tile & 7) * 256 + (w & 3) * 64;
            const rsrc_t outr = mk_rsrc(P.out, (unsigned)DIM * DIM * 2u);
#pragma unroll
            for (int q = 0; q < 8; ++q) c[q] = __builtin_amdgcn_raw_buffer_load_b128(outr, (unsigned)(lane >> 3) * (DIM * 2u) + (unsigned)(lane & 7) * 16u, (row0 * DIM + col0) * 2u + q * (8u * DIM * 2u), AUX_SYS);
            if (rfl(__hip_atomic_load(tmo, __ATOMIC_RELAXED, __HIP_MEMORY_SCOPE_AGENT)) == call) {
#pragma unroll
                for (int q = 0; q < 8; ++q) c[q] = (u32x4){0x7FC07FC0u, 0x7FC07FC0u, 0x7FC07FC0u, 0x7FC07FC0u};
            }
            slot = 4096u + o; n = 1536u + o; dmask = 0xFFu;
        } else {
            bool is_w; int kk, sel, r, bi;
            if (p < 3072) {
                kk = p / 192; const int rem = p - kk * 192;
                bi = rem & 31; r = 0; is_w = rem >= 64; sel = is_w ? ((rem - 64) >> 5) : (rem >> 5);
            } else { const int q = p - 3072; kk = q >> 6; sel = (q >> 5) & 1; bi = q & 31; r = 1; is_w = false; }
            if (!is_w) {
                const int rr = lane >> 3;
                const unsigned vo = (unsigned)rr * 2048u + (unsigned)(((lane & 7) ^ rr) * 16);
                const unsigned so = (unsigned)(sel * 4096 + r * 2048 + bi * 64) * 2048u + (unsigned)kk * 128u;
#pragma unroll
                for (int q = 0; q < 8; ++q) c[q] = __builtin_amdgcn_raw_buffer_load_b128(xr, vo + q * 8 * 2048, so, 0);
            } else {
                const int pn = bi >> 2, bj = (bi >> 1) & 1, q = bi & 1;
                const int gk = lane >> 3, cc = lane & 7, wcl = cc >> 2, cq = cc & 3;
                const unsigned vo = (unsigned)(gk * 8) * 16384u + (unsigned)((64 * (2 * q + wcl) + 32 * bj + 8 * cq) * 2);
                const unsigned so = (unsigned)(kk * 64) * 16384u + (unsigned)((sel * 2048 + pn * 256) * 2);
                u32x4 R[8];
#pragma unroll
                for (int jj = 0; jj < 8; ++jj) R[jj] = __builtin_amdgcn_raw_buffer_load_b128(wr, vo + jj * 16384, so, 0);
#pragma unroll
                for (int t = 0; t < 8; ++t) {
                    const unsigned s = (t & 1) ? 0x07060302u : 0x05040100u;
                    u32x4 v;
                    v.x = perm(R[1][t >> 1], R[0][t >> 1], s); v.y = perm(R[3][t >> 1], R[2][t >> 1], s);
                    v.z = perm(R[5][t >> 1], R[4][t >> 1], s); v.w = perm(R[7][t >> 1], R[6][t >> 1], s);
                    const int s64 = 32 * wcl + 16 * (t >> 2) + 4 * cq + (t & 3);
                    *(LAS u32x4*)(my + s64 * 128 + ((gk ^ (s64 & 7)) * 16)) = v;
                }
                asm volatile("s_waitcnt lgkmcnt(0)" ::: "memory");
#pragma unroll
                for (int q2 = 0; q2 < 8; ++q2) c[q2] = *(LAS u32x4*)(my + q2 * 1024 + lane * 16);
                asm volatile("s_waitcnt lgkmcnt(0)" ::: "memory");
            }
            if (call == 0) call = call_draw(ws);
            const int g = P.rank * 16 + kk;
            if (is_w ? (sel == myj) : (sel == myi)) {
                const unsigned so = is_w ? OFF_BIMG + (unsigned)(g * 32 + bi) * 8192u : OFF_AIMG + (unsigned)((r * 128 + g) * 32 + bi) * 8192u;
#pragma unroll
                for (int q = 0; q < 8; ++q) __builtin_amdgcn_raw_buffer_store_b128(c[q], wsr, lane * 16 + q * 1024, so, AUX_SC1);
                selfcnt = OFF_CNT + (r * 128 + g) * 128;
            }
            slot = is_w ? 2048u + (unsigned)((sel * 16 + kk) * 32 + bi) : (unsigned)(((sel * 2 + r) * 16 + kk) * 32 + bi);
            n = is_w ? (unsigned)(kk * 64 + 32 + bi) : (r == 0 ? (unsigned)(kk * 64 + bi) : (unsigned)(1024 + kk * 32 + bi));
            dmask = is_w ? (0x11u << sel) : (0x0Fu << (4 * sel));
        }
        publish_block(P, wsr, c, slot, n, dmask & ~(1u << P.rank), call, lds, my, j, (p < NFIRST || (p >= 4096 && p < OUT2)) ? 4u : 8u, wid, lane);
        signal_cnt(ws, selfcnt, lane);
        if (p < NFIRST) p = next;
        else if (p < 4096) {
            p += step;
            if (p >= 4096 && end > 4096) { if (wid < 4) p = 4096 + (int)blockIdx.x * 4 + wid; else { p = OUT2 + (int)blockIdx.x * 8 + wid; ++j; } }
        } else if (p < OUT2) p = OUT2 + (int)blockIdx.x * 8 + wid;
        else p += step;
    }
    if (call == 0) call = call_draw(ws);
    return call;
}

struct Blk { unsigned slot, dso, cnt; bool is_out; };
__device__ __forceinline__ Blk locate(unsigned n, int src, int myi, int myj) {
    Blk b; b.is_out = false;
    if (n < 1024) { const int kk = n >> 6, bi = n & 31, g = src * 16 + kk; b.cnt = OFF_CNT + g * 128;
        if ((n & 63) >= 32) { b.slot = 2048u + (myj * 16 + kk) * 32 + bi; b.dso = OFF_BIMG + (unsigned)(g * 32 + bi) * 8192u; }
        else { b.slot = ((myi * 2) * 16 + kk) * 32 + bi; b.dso = OFF_AIMG + (unsigned)(g * 32 + bi) * 8192u; } }
    else if (n < 1536) { const int kk = (n - 1024) >> 5, bi = n & 31, g = src * 16 + kk; b.cnt = OFF_CNT + (128 + g) * 128;
        b.slot = ((myi * 2 + 1) * 16 + kk) * 32 + bi; b.dso = OFF_AIMG + (unsigned)((128 + g) * 32 + bi) * 8192u; }
    else { const int o = n - 1536, w = o & 7, h = (o >> 3) & 1, tile = (o >> 4) & 63, r = o >> 10; b.cnt = 0; b.is_out = true;
        b.slot = 4096u + o;
        const unsigned row0 = (src >> 2) * 4096 + r * 2048 + (tile >> 3) * 256 + h * 128 + (w >> 2) * 64, col0 = (src & 3) * 2048 + (tile & 7) * 256 + (w & 3) * 64;
        b.dso = (row0 * DIM + col0) * 2u; }
    return b;
}
constexpr unsigned NPAIR = NSEQ / 2, NSLOT = 7, CTL = NSLOT * 16384;
constexpr unsigned M_DEAD = 1u << 30, N_STOP = 0xFFFFFFFFu;
__device__ __forceinline__ unsigned lds_ld(LAS unsigned char* p) { return *(volatile LAS unsigned*)p; }
__device__ __forceinline__ void lds_st(LAS unsigned char* p, unsigned v) { *(volatile LAS unsigned*)p = v; }
__device__ __forceinline__ void ring_publish(LAS unsigned char* lds, unsigned i, unsigned n, unsigned ha, unsigned hb, int lane) {
    const unsigned sl = i % NSLOT;
    if (lane == 0) { *(volatile LAS u32x4*)(lds + CTL + sl * 16) = (u32x4){n, ha, hb, 0u}; lds_st(lds + CTL + 128 + sl * 4, i + 1u); }
}
__device__ __forceinline__ bool slot_busy(LAS unsigned char* lds, unsigned i) { return i >= NSLOT && (int)(lds_ld(lds + CTL + 160 + (i % NSLOT) * 4) - 2u * (i / NSLOT)) < 0; }
__device__ __forceinline__ void dma_op(rsrc_t r, LAS unsigned char* dst, unsigned j, unsigned so, int lane) {
    __builtin_amdgcn_raw_ptr_buffer_load_lds(r, (LAS void*)(dst + j * 1024), 16, lane * 16, so + j * 1024, 0, AUX_SYS);
}
__device__ __forceinline__ u32x4 load_pair(rsrc_t wsr, unsigned off) { return __builtin_amdgcn_raw_buffer_load_b128(wsr, 0, off, AUX_SYS); }
#define VM_WAIT_(n) asm volatile("s_waitcnt vmcnt(" #n ")" ::: "memory")
#define VM_WAIT(n) VM_WAIT_(n)
#define G_STR_(x) #x
#define G_STR(x) G_STR_(x)
#define WIN 4

__device__ __forceinline__ void loader_wave(const Params& P, LAS unsigned char* lds, unsigned call, int src, int lane) {
    char* ws = P.ws[P.rank];
    unsigned* tmo = (unsigned*)(ws + OFF_TMO);
    const rsrc_t wsr = mk_rsrc(ws, WS_USED);
    char* const remp = P.ws[src];
    const int myi = P.rank >> 2, myj = P.rank & 3;
    const unsigned gbase = OFF_GRAN + (unsigned)src * NSEQ * 8u;
    char* const gptr = ws + gbase;
    const unsigned tko = OFF_LINK + (unsigned)src * 128u, tbase = (call - 1u) * (NPAIR + 3u * PWL);
    __builtin_amdgcn_s_setprio(3);
    unsigned q, q1, q2;
    {
        unsigned t0 = 0;
        if (lane == 0) t0 = __hip_atomic_fetch_add((unsigned*)(ws + tko), 3u, __ATOMIC_RELAXED, __HIP_MEMORY_SCOPE_AGENT);
        q = rfl(t0) - tbase; q1 = q + 1; q2 = q + 2;
    }
    {
        const rsrc_t rw = mk_rsrc(remp, OFF_AIMG);
        const unsigned v = __builtin_amdgcn_raw_buffer_load_b32(rw, lane < 25 ? OFF_SLOT + (unsigned)lane * (2u << 20) + 8188u - (lane == 24 ? 8192u : 0u) : 0xFFFFFFF0u, 0, AUX_SYS);
        asm volatile("" :: "v"(v));
    }
    unsigned gca, gha, gcb, ghb, hca, hha, hcb, hhb;
    { const u32x4 a = load_pair(wsr, gbase + (q < NPAIR ? q : 0u) * 16u), b = load_pair(wsr, gbase + (q1 < NPAIR ? q1 : 0u) * 16u);
      gca = rfl(a.x); gha = rfl(a.y); gcb = rfl(a.z); ghb = rfl(a.w); hca = rfl(b.x); hha = rfl(b.y); hcb = rfl(b.z); hhb = rfl(b.w); }
    const int zero = 0, one = 1;
    unsigned i = 0;
    bool pend = false; unsigned pn = 0, pa = 0, pb = 0;
    while (q < NPAIR) {
        bool dead = false;
        if ((int)(gca - call) < 0 || (int)(gcb - call) < 0 || slot_busy(lds, i)) {
            if (pend) { VM_WAIT(0); ring_publish(lds, i - 1u, pn, pa, pb, lane); pend = false; }
            const unsigned long long t0 = __builtin_amdgcn_s_memrealtime();
            while ((int)(gca - call) < 0 || (int)(gcb - call) < 0) {
                const u32x4 a = load_pair(wsr, gbase + q * 16u);
                gca = rfl(a.x); gha = rfl(a.y); gcb = rfl(a.z); ghb = rfl(a.w);
                if ((int)(gca - call) >= 0 && (int)(gcb - call) >= 0) break;
                if (give_up(t0, tmo, call)) { dead = true; break; }
                __builtin_amdgcn_s_sleep(2);
                asm volatile("" ::: "memory");
            }
            while (slot_busy(lds, i)) __builtin_amdgcn_s_sleep(1);
        }
        const unsigned n = 2u * q;
        const unsigned so = OFF_SLOT + locate(n, src, myi, myj).slot * 8192u;
        const unsigned bytes = dead ? 0u : payload_bytes(gha) + payload_bytes(ghb), nops = (bytes + 1023u) >> 10;
        const rsrc_t rem = mk_rsrc(remp, so + bytes);
        LAS unsigned char* dst = lds + (i % NSLOT) * 16384;
        u32x4 g2; unsigned tk = 0;
        asm volatile("global_load_dwordx4 %0, %1, %2 sc0 sc1" : "=v"(g2) : "v"(zero), "s"(gptr + (q2 < NPAIR ? q2 : 0u) * 16u) : "memory");
        if (lane == 0) asm volatile("global_atomic_add %0, %1, %2, %3 sc0" : "=v"(tk) : "v"(zero), "v"(one), "s"(ws + tko) : "memory");
#define G_FILL(jj) do { if ((jj) == WIN) asm volatile("s_waitcnt vmcnt(" G_STR(WIN) ")" : "+v"(g2), "+v"(tk) :: "memory"); else VM_WAIT(WIN); \
                        if ((jj) == WIN - 2 && pend) ring_publish(lds, i - 1u, pn, pa, pb, lane); dma_op(rem, dst, jj, so, lane); } while (0)
        G_FILL(0); G_FILL(1); G_FILL(2); G_FILL(3); G_FILL(4); G_FILL(5); G_FILL(6); G_FILL(7);
#undef G_FILL
        for (unsigned j = 8; j < nops; ++j) { VM_WAIT(WIN); dma_op(rem, dst, j, so, lane); }
        pend = true; pn = n; pa = gha | (dead ? M_DEAD : 0u); pb = ghb | (dead ? M_DEAD : 0u);
        q = q1; gca = hca; gha = hha; gcb = hcb; ghb = hhb;
        q1 = q2; hca = rfl(g2.x); hha = rfl(g2.y); hcb = rfl(g2.z); hhb = rfl(g2.w);
        q2 = rfl(tk) - tbase;
        ++i;
    }
    VM_WAIT(0);
    if (pend) ring_publish(lds, i - 1u, pn, pa, pb, lane);
    for (int d = 0; d < (NDEC + 1) / 2 + 1; ++d, ++i) {
        while (slot_busy(lds, i)) __builtin_amdgcn_s_sleep(1);
        ring_publish(lds, i, N_STOP, 0u, 0u, lane);
    }
}

__device__ __forceinline__ void decoder_wave(const Params& P, LAS unsigned char* lds, unsigned call, int src, int d, int lane) {
    char* ws = P.ws[P.rank];
    unsigned* tmo = (unsigned*)(ws + OFF_TMO);
    const rsrc_t wsr = mk_rsrc(ws, WS_USED), outr = mk_rsrc(P.out, (unsigned)DIM * DIM * 2u);
    const int myi = P.rank >> 2, myj = P.rank & 3;
    for (unsigned i = d;; i += NDEC) {
        const unsigned pi = i >> 1, sl = pi % NSLOT;
        while (lds_ld(lds + CTL + 128 + sl * 4) != pi + 1u) __builtin_amdgcn_s_sleep(1);
        asm volatile("" ::: "memory");
        const u32x4 meta = *(volatile LAS u32x4*)(lds + CTL + sl * 16);
        const unsigned n0 = rfl(meta.x), ha = rfl(meta.y), h = (i & 1) ? rfl(meta.z) : ha;
        if (n0 == N_STOP) break;
        const Blk b = locate(n0 + (i & 1), src, myi, myj);
        const bool raw = (h >> 31) != 0;
        bool dead = (h & M_DEAD) != 0;
        if (b.is_out && rfl(__hip_atomic_load(tmo, __ATOMIC_RELAXED, __HIP_MEMORY_SCOPE_AGENT)) == call) dead = true;
        u32x4 c[8];
        unsigned recs = 0;
        const unsigned ne = (dead || raw) ? 0u : (h >> 8) & 63u;
        LAS unsigned char* cur = lds + sl * 16384 + ((i & 1) ? payload_bytes(ha) : 0u);
        if (dead) {
#pragma unroll
            for (int j = 0; j < 8; ++j) c[j] = (u32x4){0x7FC07FC0u, 0x7FC07FC0u, 0x7FC07FC0u, 0x7FC07FC0u};
        } else if (raw) {
#pragma unroll
            for (int j = 0; j < 8; ++j) c[j] = *(LAS u32x4*)(cur + j * 1024 + lane * 16);
        } else {
            recs = *(LAS unsigned*)(cur + RAWB + (((h >> 16) & 0x3FFu) + (lane & 31)) * 4u);
            decode_block(cur, lds + CTL + 256 + d * 256, c, lane);
        }
#pragma unroll
        for (int j = 0; j < 8; ++j) asm volatile("" : "+v"(c[j]));
        asm volatile("" : "+v"(recs));
        if (lane == 0) __hip_atomic_fetch_add((LAS unsigned*)(lds + CTL + 160 + sl * 4), 1u, __ATOMIC_RELAXED, __HIP_MEMORY_SCOPE_WORKGROUP);
        const unsigned sj = b.is_out ? 8u * DIM * 2u : 1024u, sr = b.is_out ? DIM * 2u : 128u;
        const unsigned vo = (unsigned)(lane >> 3) * sr + (unsigned)(lane & 7) * 16u;
        if (b.is_out) {
#pragma unroll
            for (int j = 0; j < 8; ++j) __builtin_amdgcn_raw_buffer_store_b128(c[j], outr, vo, b.dso + j * sj, OUT_AUX);
            for (unsigned e = 0; e < ne; ++e) {
                const unsigned rec = __builtin_amdgcn_readlane(recs, e), pos = rec & 0xFFFu, v = pos & 63u;
                if ((unsigned)lane == (pos >> 6)) __builtin_amdgcn_raw_buffer_store_b16((unsigned short)((rec >> 12) & 0xFFFFu), outr, vo + (v & 7u) * 2u, b.dso + (v >> 3) * sj, OUT_AUX);
            }
        } else {
#pragma unroll
            for (int j = 0; j < 8; ++j) __builtin_amdgcn_raw_buffer_store_b128(c[j], wsr, vo, b.dso + j * sj, IMG_AUX);
            for (unsigned e = 0; e < ne; ++e) {
                const unsigned rec = __builtin_amdgcn_readlane(recs, e), pos = rec & 0xFFFu, v = pos & 63u;
                if ((unsigned)lane == (pos >> 6)) __builtin_amdgcn_raw_buffer_store_b16((unsigned short)((rec >> 12) & 0xFFFFu), wsr, vo + (v & 7u) * 2u, b.dso + (v >> 3) * sj, IMG_AUX);
            }
            WAIT_VM0(); signal_cnt(ws, b.cnt, lane);
        }
    }
}

__device__ __forceinline__ void puller_role(const Params& P, LAS unsigned char* lds, unsigned call, int wg) {
    const int lane = threadIdx.x & 63, wid = rfl(threadIdx.x >> 6);
    const int link = wg % 7, src = rfl((P.rank + 1 + link) & 7);
    if (threadIdx.x < 16) *(volatile LAS unsigned*)(lds + CTL + 128 + threadIdx.x * 4) = 0u;
    __syncthreads();
    if (wid == 0) loader_wave(P, lds, call, src, lane);
    else if (wid <= NDEC) decoder_wave(P, lds, call, src, wid - 1, lane);
}

__device__ __forceinline__ float gelu_f(float y) {
    const float c1 = -2.0f * 0.7978845608028654f * 1.4426950408889634f, c2 = c1 * 0.044715f;
    const float e = __builtin_amdgcn_exp2f(y * (c1 + c2 * y * y));
    return y * __builtin_amdgcn_rcpf(1.0f + e);
}
__device__ __forceinline__ unsigned cvt_pk_bf16(float lo, float hi) { unsigned r; asm("v_cvt_pk_bf16_f32 %0, %1, %2" : "=v"(r) : "v"(lo), "v"(hi)); return r; }

__device__ __forceinline__ void gemm_role(const Params& P, LAS unsigned char* lds, unsigned call, int wg) {
    const int tid = threadIdx.x, wid = rfl(tid >> 6), lane = tid & 63, wr = wid >> 2, wc = wid & 3, fr = lane & 15, fq = lane >> 4;
    char* ws = P.ws[P.rank];
    unsigned* tmo = (unsigned*)(ws + OFF_TMO);
    const rsrc_t wsr = mk_rsrc(ws, WS_USED);
    const int xcd = wg & 7, q = (wg - NPACK) >> 3, h = xcd & 1, blk = xcd >> 1;
    const int tm = (blk >> 1) * 4 + (q >> 2), tn = (blk & 1) * 4 + (q & 3), tile = tm * 8 + tn;
    const unsigned voff = (unsigned)tid * 16u, ldsw = (unsigned)wid * 1024u;
    const int aoff0 = (wr * 64 + fr) * 128 + ((fq ^ (fr & 7)) << 4), aoff1 = aoff0 ^ 64;
    const int boff0 = (wc * 32 + fr) * 128 + ((fq ^ (fr & 7)) << 4), boff1 = boff0 ^ 64;
#define G_SA(b, hh) (((b) * 2 + (hh)) * 16384)
#define G_SB(b, hh) ((4 + (b) * 2 + (hh)) * 16384)
#define G_STAGE(bufoff, soff) do { _Pragma("unroll") for (int _i = 0; _i < 2; ++_i) \
        __builtin_amdgcn_raw_ptr_buffer_load_lds(wsr, (LAS void*)(lds + (bufoff) + ldsw + _i * 8192), 16, voff, (soff) + _i * 8192, 0, AUX_SYS); } while (0)
#define G_LDA(dst, b, hh) do { _Pragma("unroll") for (int m = 0; m < 4; ++m) { \
        dst[m][0] = *(const LAS bf16x8*)(lds + G_SA(b, hh) + aoff0 + m * 2048); dst[m][1] = *(const LAS bf16x8*)(lds + G_SA(b, hh) + aoff1 + m * 2048); } } while (0)
#define G_LDB(dst, b, hh) do { _Pragma("unroll") for (int n = 0; n < 2; ++n) { \
        dst[n][0] = *(const LAS bf16x8*)(lds + G_SB(b, hh) + boff0 + n * 2048); dst[n][1] = *(const LAS bf16x8*)(lds + G_SB(b, hh) + boff1 + n * 2048); } } while (0)
#define G_MMA(ai, bj, At, Bt) do { __builtin_amdgcn_s_setprio(1); _Pragma("unroll") for (int m = 0; m < 4; ++m) _Pragma("unroll") for (int n = 0; n < 2; ++n) _Pragma("unroll") for (int k = 0; k < 2; ++k) \
        acc[ai][bj][m][n] = __builtin_amdgcn_mfma_f32_16x16x32_bf16(Bt[n][k], At[m][k], acc[ai][bj][m][n], 0, 0, 0); __builtin_amdgcn_s_setprio(0); } while (0)
#define G_WAIT_V(n) asm volatile("s_waitcnt vmcnt(" #n ")" ::: "memory")
#define G_WAIT_L(n) asm volatile("s_waitcnt lgkmcnt(" #n ")" ::: "memory")
#define G_BAR __builtin_amdgcn_s_barrier()
#define G_SCHED __builtin_amdgcn_sched_barrier(0)
#define G_GIDX(t) ((unsigned)((4 * h + ((t) & 3)) * 16 + ((t) >> 2)))
    const int zero = 0;
    for (int r = 0; r < 2; ++r) {
        const unsigned target = call * (r == 0 ? 64u : 32u);
        const unsigned abase = OFF_AIMG + (unsigned)r * (128u * 262144u) + (unsigned)tm * 32768u, bbase = OFF_BIMG + (unsigned)tn * 32768u;
        char* cbase = ws + OFF_CNT + r * 128 * 128;
        f32x4 acc[2][2][4][2];
#pragma unroll
        for (int a = 0; a < 2; ++a)
#pragma unroll
            for (int b = 0; b < 2; ++b)
#pragma unroll
                for (int m = 0; m < 4; ++m)
#pragma unroll
                    for (int n = 0; n < 2; ++n) acc[a][b][m][n] = (f32x4){0.f, 0.f, 0.f, 0.f};
        bf16x8 At[4][2], B0[2][2], B1[2][2];
        STAMP(ws, r * 8 + 0);
        wait_word((unsigned*)(cbase + G_GIDX(0) * 128), target, tmo, call);
        wait_word((unsigned*)(cbase + G_GIDX(1) * 128), target, tmo, call);
        wait_word((unsigned*)(cbase + G_GIDX(2) * 128), target, tmo, call);
        STAMP(ws, r * 8 + 1);
        {
            const unsigned a0 = abase + G_GIDX(0) * 262144u, b0 = bbase + G_GIDX(0) * 262144u, a1 = abase + G_GIDX(1) * 262144u, b1 = bbase + G_GIDX(1) * 262144u;
            G_STAGE(G_SB(0, 0), b0); G_STAGE(G_SA(0, 0), a0); G_STAGE(G_SB(0, 1), b0 + 16384); G_STAGE(G_SA(0, 1), a0 + 16384);
            if (wr == 1) G_BAR;
            G_WAIT_V(4); G_BAR;
            G_STAGE(G_SB(1, 0), b1); G_STAGE(G_SA(1, 0), a1); G_STAGE(G_SB(1, 1), b1 + 16384);
            G_WAIT_V(6); G_BAR;
        }
        unsigned fA = target, fB;
        for (int t = 0; t < 64; t += 2) {
            const int t2 = t + 2 < 64 ? t + 2 : 63, t3 = t + 3 < 64 ? t + 3 : 63, t4 = t + 4 < 64 ? t + 4 : 63;
            const unsigned g1 = G_GIDX(t + 1), g2 = G_GIDX(t2), g3 = G_GIDX(t3), g4 = G_GIDX(t4);
            const unsigned a1 = abase + g1 * 262144u, a2 = abase + g2 * 262144u, b2 = bbase + g2 * 262144u, a3 = abase + g3 * 262144u, b3 = bbase + g3 * 262144u;
            asm volatile("global_load_dword %0, %1, %2 sc0 sc1" : "=v"(fB) : "v"(zero), "s"(cbase + g3 * 128));
            G_LDB(B0, 0, 0); G_SCHED; G_LDA(At, 0, 0); G_STAGE(G_SA(1, 1), a1 + 16384);
            G_WAIT_L(8); G_BAR; G_WAIT_L(0); G_MMA(0, 0, At, B0); G_BAR; G_SCHED;
            asm volatile("" : "+v"(fA));
            if ((int)(rfl(fA) - target) < 0) wait_word((unsigned*)(cbase + g2 * 128), target, tmo, call);
            G_LDB(B1, 0, 1); G_STAGE(G_SB(0, 0), b2);
            G_BAR; G_WAIT_L(0); G_MMA(0, 1, At, B1); G_BAR;
            G_LDA(At, 0, 1); G_STAGE(G_SA(0, 0), a2);
            G_BAR; G_WAIT_L(0); G_MMA(1, 0, At, B0); G_BAR; G_SCHED;
            G_STAGE(G_SB(0, 1), b2 + 16384);
            G_WAIT_V(6); G_BAR; G_MMA(1, 1, At, B1); G_BAR;
            asm volatile("global_load_dword %0, %1, %2 sc0 sc1" : "=v"(fA) : "v"(zero), "s"(cbase + g4 * 128));
            G_LDB(B0, 1, 0); G_SCHED; G_LDA(At, 1, 0); G_STAGE(G_SA(0, 1), a2 + 16384);
            G_WAIT_L(8); G_BAR; G_WAIT_L(0); G_MMA(0, 0, At, B0); G_BAR; G_SCHED;
            asm volatile("" : "+v"(fB));
            if ((int)(rfl(fB) - target) < 0) wait_word((unsigned*)(cbase + g3 * 128), target, tmo, call);
            G_LDB(B1, 1, 1); G_STAGE(G_SB(1, 0), b3);
            G_BAR; G_WAIT_L(0); G_MMA(0, 1, At, B1); G_BAR;
            G_LDA(At, 1, 1); G_STAGE(G_SA(1, 0), a3);
            G_BAR; G_WAIT_L(0); G_MMA(1, 0, At, B0); G_BAR; G_SCHED;
            G_STAGE(G_SB(1, 1), b3 + 16384);
            G_WAIT_V(6); G_BAR; G_MMA(1, 1, At, B1); G_BAR;
        }
        G_WAIT_V(0);
        asm volatile("" : "+v"(fA));
        if (wr == 0) G_BAR;
        G_BAR;
        STAMP(ws, r * 8 + 2);
        int lane = threadIdx.x & 63;
        asm volatile("" : "+v"(lane));
        const int fr = lane & 15, fq = lane >> 4;
        if (h) {
#pragma unroll
            for (int b = 0; b < 2; ++b)
#pragma unroll
                for (int m = 0; m < 4; ++m)
#pragma unroll
                    for (int n = 0; n < 2; ++n) { const f32x4 tq = acc[0][b][m][n]; acc[0][b][m][n] = acc[1][b][m][n]; acc[1][b][m][n] = tq; }
        }
        const unsigned xl = (unsigned)((r * 64 + tile) * 2);
#pragma unroll
        for (int b = 0; b < 2; ++b)
#pragma unroll
            for (int m = 0; m < 4; ++m)
#pragma unroll
                for (int n = 0; n < 2; ++n)
                    __builtin_amdgcn_raw_buffer_store_b128(__builtin_bit_cast(u32x4, acc[1][b][m][n]), wsr, lane * 16 + ((b * 4 + m) * 2 + n) * 1024, OFF_XBUF + ((xl + h) * 8 + wid) * 16384u, AUX_SC1);
        WAIT_VM0();
        if (lane == 0) __hip_atomic_store((unsigned*)(ws + OFF_XFLAG + (xl + h) * 128 + wid * 16), call, __ATOMIC_RELAXED, __HIP_MEMORY_SCOPE_AGENT);
        wait_word((unsigned*)(ws + OFF_XFLAG + (xl + (h ^ 1)) * 128 + wid * 16), call, tmo, call);
        const bool dead = rfl(__hip_atomic_load(tmo, __ATOMIC_RELAXED, __HIP_MEMORY_SCOPE_AGENT)) == call;
        STAMP(ws, r * 8 + 3);
        LAS unsigned char* my = lds + wid * 8192;
#pragma unroll
        for (int b = 0; b < 2; ++b)
#pragma unroll
            for (int m = 0; m < 4; ++m) {
                const f32x4 p0 = __builtin_bit_cast(f32x4, __builtin_amdgcn_raw_buffer_load_b128(wsr, lane * 16 + ((b * 4 + m) * 2 + 0) * 1024, OFF_XBUF + ((xl + (h ^ 1)) * 8 + wid) * 16384u, AUX_SC1));
                const f32x4 p1 = __builtin_bit_cast(f32x4, __builtin_amdgcn_raw_buffer_load_b128(wsr, lane * 16 + ((b * 4 + m) * 2 + 1) * 1024, OFF_XBUF + ((xl + (h ^ 1)) * 8 + wid) * 16384u, AUX_SC1));
                const f32x4 v0 = acc[0][b][m][0] + p0, v1 = acc[0][b][m][1] + p1;
                u32x4 o;
                o.x = cvt_pk_bf16(gelu_f(v0[0]), gelu_f(v0[1])); o.y = cvt_pk_bf16(gelu_f(v0[2]), gelu_f(v0[3]));
                o.z = cvt_pk_bf16(gelu_f(v1[0]), gelu_f(v1[1])); o.w = cvt_pk_bf16(gelu_f(v1[2]), gelu_f(v1[3]));
                const int row = m * 16 + fr, ch = b * 4 + fq;
                *(LAS u32x4*)(my + row * 128 + ((ch ^ (row & 7)) * 16)) = o;
            }
        asm volatile("s_waitcnt lgkmcnt(0)" ::: "memory");
        u32x4 c[8];
#pragma unroll
        for (int j = 0; j < 8; ++j) { const int row = j * 8 + (lane >> 3); c[j] = *(LAS u32x4*)(my + row * 128 + (((lane & 7) ^ (row & 7)) * 16)); }
        if (dead) {
#pragma unroll
            for (int j = 0; j < 8; ++j) c[j] = (u32x4){0x7FC07FC0u, 0x7FC07FC0u, 0x7FC07FC0u, 0x7FC07FC0u};
        }
        {
            const rsrc_t outr = mk_rsrc(P.out, (unsigned)DIM * DIM * 2u);
            const unsigned row0 = (P.rank >> 2) * 4096 + r * 2048 + tm * 256 + h * 128 + wr * 64, col0 = (P.rank & 3) * 2048 + tn * 256 + wc * 64;
            const unsigned dso = (row0 * DIM + col0) * 2u, vo = (unsigned)(lane >> 3) * (DIM * 2u) + (unsigned)(lane & 7) * 16u;
            if (r == 0) {
#pragma unroll
                for (int j = 0; j < 8; ++j) __builtin_amdgcn_raw_buffer_store_b128(c[j], outr, vo, dso + j * (8u * DIM * 2u), AUX_SC1);
            } else {
#pragma unroll
                for (int j = 0; j < 8; ++j) __builtin_amdgcn_raw_buffer_store_b128(c[j], outr, vo, dso + j * (8u * DIM * 2u), OUT_AUX);
            }
            const unsigned o = (xl + h) * 8 + wid;
            if (r == 0) { WAIT_VM0(); signal_cnt(ws, OFF_OCNT + (o >> 3) * 128, lane); }
            else publish_block(P, wsr, c, 4096u + o, 1536u + o, 0xFFu & ~(1u << P.rank), call, lds, my, 1u + GROUNDS, 8u, wid, lane);
        }
        STAMP(ws, r * 8 + 4);
        asm volatile("s_waitcnt lgkmcnt(0)" ::: "memory");
        G_BAR;
    }
#undef G_SA
#undef G_SB
#undef G_STAGE
#undef G_LDA
#undef G_LDB
#undef G_MMA
#undef G_WAIT_V
#undef G_WAIT_L
#undef G_BAR
#undef G_SCHED
#undef G_GIDX
}

__global__ __launch_bounds__(512, 2) void gemm_allgather_kernel(Params P) {
    extern __shared__ __attribute__((aligned(16))) unsigned char shm[];
    LAS unsigned char* lds = (LAS unsigned char*)shm;
    char* ws = P.ws[P.rank];
    const int b = rfl(blockIdx.x);
    STAMP(ws, 60);
    unsigned call;
    if (b >= NPACK + NGEMM) { call = call_draw(ws); puller_role(P, lds, rfl(call), rfl(b - NPACK - NGEMM)); }
    else {
        if (threadIdx.x < 40) *(volatile LAS unsigned*)(lds + GCTL + 128 + threadIdx.x * 4) = 0u;
        __syncthreads();
        const int wid = rfl(threadIdx.x >> 6);
        const bool gemm = b >= NPACK;
        const int next = gemm ? NFIRST + (b - NPACK) * 8 + wid : NFIRST + GROUNDS * NGEMM * 8 + b * 8 + wid;
        call = packer_role(P, lds, wid < 4 ? b * 4 + wid : next, next, gemm ? NGEMM * 8 : NPACK * 8, gemm ? NFIRST + GROUNDS * NGEMM * 8 : 4096 + 1024);
        if (gemm) { __syncthreads(); gemm_role(P, lds, rfl(call), rfl(b)); }
    }
    STAMP(ws, 61);
    STAMP(ws, 58 + (call & 1));
}

}

extern "C" void kernel_launch(int rank, int nranks, ncclComm_t comm,
                              hipStream_t stream,
                              void* const* d_in, const int* in_sizes, int n_in,
                              void* d_out, int out_size,
                              const int* in_dtypes, int out_dtype,
                              void* const* d_out_all,
                              void* const* d_ws_all, size_t ws_bytes) {
    (void)comm; (void)in_sizes; (void)in_dtypes; (void)out_dtype; (void)d_out_all;
    if (nranks != gar::WORLD || n_in < 2 || ws_bytes < gar::WS_NEED || out_size != gar::DIM * gar::DIM || (unsigned)rank >= (unsigned)gar::WORLD) {
        fprintf(stderr, "unexpected problem shape\n");
        return;
    }
    static bool attr = false;
    if (!attr) {
        hipError_t e = hipFuncSetAttribute((const void*)gar::gemm_allgather_kernel, hipFuncAttributeMaxDynamicSharedMemorySize, gar::LDS_BYTES);
        if (e != hipSuccess) { fprintf(stderr, "hipFuncSetAttribute: %s\n", hipGetErrorString(e)); return; }
        attr = true;
    }
    gar::Params P;
    memset(&P, 0, sizeof P);
    P.x = d_in[0]; P.w = d_in[1]; P.out = d_out; P.rank = rank;
    for (int p = 0; p < gar::WORLD; ++p) P.ws[p] = (char*)d_ws_all[p];
    hipLaunchKernelGGL(gar::gemm_allgather_kernel, dim3(gar::GRID), dim3(512), gar::LDS_BYTES, stream, P);
    hipError_t e = hipGetLastError();
    if (e != hipSuccess) fprintf(stderr, "launch: %s\n", hipGetErrorString(e));
}
```
